```python
import numpy as np

kernel_name = "gpu_a2a_gemm_m4096_k4096_n16384_f32_gelu"


ABSMAX_RTOL = 0.001

COMPUTE_DTYPE = "float32"
EPILOGUE = "gelu"
IN_QUANT = None
OUT_QUANT = None
IN_DTYPES = {"x": "float32", "w_mat": "float32"}
WORLD_SIZE = 8

def setup_inputs():

    x = np.random.randn(4096, 4096).astype(np.float32)
    w_mat = (np.random.randn(4096, 16384) / np.sqrt(4096)).astype(np.float32)
    return {"x": x, "w_mat": w_mat}

def _epilogue(y):
    c = 0.7978845608028654
    return (0.5 * y * (1.0 + np.tanh(c * (y + 0.044715 * y ** 3)))).astype(np.float32)

def reference(x, w_mat):
    y = (x @ w_mat).astype(np.float32)
    return _epilogue(y)
```

```cpp
#include <hip/hip_runtime.h>
#include <rccl/rccl.h>
#include <cstdio>

#define GEMM_ASM1 \
  "v_mov_b32 v249, 0\n" \
  "global_load_dwordx2 v[250:251], v249, %[callp] sc1\n" \
  "s_cmp_lt_i32 %[slot], 0\n" \
  "s_cbranch_scc0 L_a1_%=\n" \
  "s_mov_b64 s[48:49], %[xptr]\n" \
  "s_add_u32 s48, s48, 0x4000\n" \
  "s_addc_u32 s49, s49, 0\n" \
  "s_add_u32 m0, %[xlds], 0\n" \
  "s_nop 0\n" \
  "global_load_lds_dwordx4 %[xoff0], %[xptr] offset:0 nt\n" \
  "s_add_u32 m0, %[xlds], 1024\n" \
  "s_nop 0\n" \
  "global_load_lds_dwordx4 %[xoff0], s[48:49] offset:0 nt\n" \
  "s_add_u32 m0, %[xlds], 1792\n" \
  "s_nop 0\n" \
  "global_load_lds_dwordx4 %[xoff0], %[xptr] offset:256 nt\n" \
  "s_add_u32 m0, %[xlds], 2816\n" \
  "s_nop 0\n" \
  "global_load_lds_dwordx4 %[xoff0], s[48:49] offset:256 nt\n" \
  "s_add_u32 m0, %[xlds], 4096\n" \
  "s_nop 0\n" \
  "global_load_lds_dwordx4 %[xoff1], %[xptr] offset:0 nt\n" \
  "s_add_u32 m0, %[xlds], 5120\n" \
  "s_nop 0\n" \
  "global_load_lds_dwordx4 %[xoff1], s[48:49] offset:0 nt\n" \
  "s_add_u32 m0, %[xlds], 5888\n" \
  "s_nop 0\n" \
  "global_load_lds_dwordx4 %[xoff1], %[xptr] offset:256 nt\n" \
  "s_add_u32 m0, %[xlds], 6912\n" \
  "s_nop 0\n" \
  "global_load_lds_dwordx4 %[xoff1], s[48:49] offset:256 nt\n" \
  "L_a1_%=:\n" \
  ""
#define GEMM_CLOB1 "v249","v250","v251","s48","s49","memory","scc"
#define GEMM_ASM \
  "s_load_dwordx2 s[52:53], %[tabp], 0x0\n" \
  "s_memrealtime s[58:59]\n" \
  "s_mov_b32 s77, %[slot]\n" \
  "s_mov_b32 s80, 0\n" \
  "s_mov_b32 s72, 8\n" \
  "s_mov_b64 s[36:37], %[wptr]\n" \
  "s_mov_b32 s38, 0x10000000\n" \
  "s_mov_b32 s39, 0x20000\n" \
  "s_cmp_lt_i32 s77, 0\n" \
  "s_cbranch_scc0 L_normal_%=\n" \
  "v_and_b32 v230, 0xf0, %[bxr]\n" \
  "v_lshrrev_b32 v231, 8, %[bxr]\n" \
  "v_lshlrev_b32 v230, 3, v230\n" \
  "v_lshl_add_u32 v230, v231, 4, v230\n" \
  "v_add_u32 v230, %[xlds], v230\n" \
  "s_waitcnt vmcnt(0) lgkmcnt(0)\n" \
  "ds_read_b128 v[14:17], v230 offset:0\n" \
  "ds_read_b128 v[18:21], v230 offset:64\n" \
  "ds_read_b128 v[22:25], v230 offset:2048\n" \
  "ds_read_b128 v[26:29], v230 offset:2112\n" \
  "ds_read_b128 v[30:33], v230 offset:4096\n" \
  "ds_read_b128 v[34:37], v230 offset:4160\n" \
  "ds_read_b128 v[38:41], v230 offset:6144\n" \
  "ds_read_b128 v[42:45], v230 offset:6208\n" \
  "v_readfirstlane_b32 s50, v251\n" \
  "s_nop 0\n" \
  "s_sub_u32 s50, s58, s50\n" \
  "s_mov_b32 s88, 157\n" \
  "s_mov_b32 s89, 176\n" \
  "s_mov_b32 s86, 205\n" \
  "s_mov_b32 s87, 234\n" \
  "s_mov_b32 s82, 460\n" \
  "s_cmp_lt_u32 s50, 3000\n" \
  "s_cselect_b32 s88, 157, s88\n" \
  "s_cselect_b32 s89, 163, s89\n" \
  "s_cselect_b32 s86, 177, s86\n" \
  "s_cselect_b32 s87, 195, s87\n" \
  "s_cselect_b32 s81, 365, s82\n" \
  "s_cselect_b32 s93, 0, -1\n" \
  "s_cselect_b32 s94, 1, 4\n" \
  "v_readfirstlane_b32 s76, v250\n" \
  "s_nop 0\n" \
  "s_add_u32 s76, s76, 1\n" \
  "s_and_b32 s85, s76, 1\n" \
  "s_lshl_b32 s85, s85, 22\n" \
  "s_add_u32 s56, s52, %[foff]\n" \
  "s_addc_u32 s57, s53, 0\n" \
  "s_add_u32 s48, s85, %[doff]\n" \
  "s_add_u32 s52, s52, s48\n" \
  "s_addc_u32 s53, s53, 0\n" \
  "s_waitcnt lgkmcnt(0)\n" \
  "v_cvt_pk_f16_f32 v14, v14, v15\n" \
  "v_cvt_pk_f16_f32 v15, v16, v17\n" \
  "v_cvt_pk_f16_f32 v16, v18, v19\n" \
  "v_cvt_pk_f16_f32 v17, v20, v21\n" \
  "global_store_dwordx4 %[bxr], v[14:17], s[52:53] offset:0 sc0 sc1\n" \
  "v_cvt_pk_f16_f32 v22, v22, v23\n" \
  "v_cvt_pk_f16_f32 v23, v24, v25\n" \
  "v_cvt_pk_f16_f32 v24, v26, v27\n" \
  "v_cvt_pk_f16_f32 v25, v28, v29\n" \
  "global_store_dwordx4 %[bxr], v[22:25], s[52:53] offset:1024 sc0 sc1\n" \
  "v_cvt_pk_f16_f32 v30, v30, v31\n" \
  "v_cvt_pk_f16_f32 v31, v32, v33\n" \
  "v_cvt_pk_f16_f32 v32, v34, v35\n" \
  "v_cvt_pk_f16_f32 v33, v36, v37\n" \
  "global_store_dwordx4 %[bxr], v[30:33], s[52:53] offset:2048 sc0 sc1\n" \
  "v_cvt_pk_f16_f32 v38, v38, v39\n" \
  "v_cvt_pk_f16_f32 v39, v40, v41\n" \
  "v_cvt_pk_f16_f32 v40, v42, v43\n" \
  "v_cvt_pk_f16_f32 v41, v44, v45\n" \
  "global_store_dwordx4 %[bxr], v[38:41], s[52:53] offset:3072 sc0 sc1\n" \
  "s_cmp_eq_u32 s77, -2\n" \
  "s_cbranch_scc0 L_rem_%=\n" \
  "s_waitcnt vmcnt(0)\n" \
  "v_mov_b32 v250, s76\n" \
  "s_nop 0\n" \
  "global_store_dword v249, v250, s[56:57] sc0 sc1\n" \
  "v_readlane_b32 s75, %[vtabW], 0\n" \
  "s_nop 0\n" \
  "s_add_u32 s60, s75, 0x0\n" \
  "s_add_u32 s61, s75, 0x10000\n" \
  "s_add_u32 s62, s75, 0x20000\n" \
  "s_add_u32 s63, s75, 0x30000\n" \
  "s_add_u32 s64, s75, 0x100000\n" \
  "s_add_u32 s65, s75, 0x110000\n" \
  "s_add_u32 s66, s75, 0x120000\n" \
  "s_add_u32 s67, s75, 0x130000\n" \
  "buffer_load_dwordx4 v[102:105], %[voffW], s[36:39], s60 offen nt\n" \
  "buffer_load_dwordx4 v[106:109], %[voffW], s[36:39], s61 offen nt\n" \
  "buffer_load_dwordx4 v[110:113], %[voffW], s[36:39], s62 offen nt\n" \
  "buffer_load_dwordx4 v[114:117], %[voffW], s[36:39], s63 offen nt\n" \
  "buffer_load_dwordx4 v[118:121], %[voffW], s[36:39], s64 offen nt\n" \
  "buffer_load_dwordx4 v[122:125], %[voffW], s[36:39], s65 offen nt\n" \
  "buffer_load_dwordx4 v[126:129], %[voffW], s[36:39], s66 offen nt\n" \
  "buffer_load_dwordx4 v[130:133], %[voffW], s[36:39], s67 offen nt\n" \
  "s_branch L_w1_%=\n" \
  "L_rem_%=:\n" \
  "v_readlane_b32 s75, %[vtabW], 0\n" \
  "s_nop 0\n" \
  "s_add_u32 s60, s75, 0x0\n" \
  "s_add_u32 s61, s75, 0x10000\n" \
  "s_add_u32 s62, s75, 0x20000\n" \
  "s_add_u32 s63, s75, 0x30000\n" \
  "s_add_u32 s64, s75, 0x100000\n" \
  "s_add_u32 s65, s75, 0x110000\n" \
  "s_add_u32 s66, s75, 0x120000\n" \
  "s_add_u32 s67, s75, 0x130000\n" \
  "buffer_load_dwordx4 v[102:105], %[voffW], s[36:39], s60 offen nt\n" \
  "buffer_load_dwordx4 v[106:109], %[voffW], s[36:39], s61 offen nt\n" \
  "buffer_load_dwordx4 v[110:113], %[voffW], s[36:39], s62 offen nt\n" \
  "buffer_load_dwordx4 v[114:117], %[voffW], s[36:39], s63 offen nt\n" \
  "buffer_load_dwordx4 v[118:121], %[voffW], s[36:39], s64 offen nt\n" \
  "buffer_load_dwordx4 v[122:125], %[voffW], s[36:39], s65 offen nt\n" \
  "buffer_load_dwordx4 v[126:129], %[voffW], s[36:39], s66 offen nt\n" \
  "buffer_load_dwordx4 v[130:133], %[voffW], s[36:39], s67 offen nt\n" \
  "s_waitcnt vmcnt(8)\n" \
  "v_mov_b32 v250, s76\n" \
  "s_nop 0\n" \
  "global_store_dword v249, v250, s[56:57] sc0 sc1\n" \
  "s_branch L_w1_%=\n" \
    \
  "L_normal_%=:\n" \
  "v_readlane_b32 s75, %[vtabW], 0\n" \
  "s_nop 0\n" \
  "s_add_u32 s60, s75, 0x0\n" \
  "s_add_u32 s61, s75, 0x10000\n" \
  "s_add_u32 s62, s75, 0x20000\n" \
  "s_add_u32 s63, s75, 0x30000\n" \
  "s_add_u32 s64, s75, 0x100000\n" \
  "s_add_u32 s65, s75, 0x110000\n" \
  "s_add_u32 s66, s75, 0x120000\n" \
  "s_add_u32 s67, s75, 0x130000\n" \
  "buffer_load_dwordx4 v[102:105], %[voffW], s[36:39], s60 offen nt\n" \
  "buffer_load_dwordx4 v[106:109], %[voffW], s[36:39], s61 offen nt\n" \
  "buffer_load_dwordx4 v[110:113], %[voffW], s[36:39], s62 offen nt\n" \
  "buffer_load_dwordx4 v[114:117], %[voffW], s[36:39], s63 offen nt\n" \
  "buffer_load_dwordx4 v[118:121], %[voffW], s[36:39], s64 offen nt\n" \
  "buffer_load_dwordx4 v[122:125], %[voffW], s[36:39], s65 offen nt\n" \
  "buffer_load_dwordx4 v[126:129], %[voffW], s[36:39], s66 offen nt\n" \
  "buffer_load_dwordx4 v[130:133], %[voffW], s[36:39], s67 offen nt\n" \
  "s_waitcnt lgkmcnt(0)\n" \
  "s_cmp_eq_u32 %[prb], -1\n" \
  "s_cbranch_scc1 L_nr_%=\n" \
  "s_mov_b64 s[54:55], %[pbase]\n" \
  "s_mul_i32 s48, s58, 0x9e3779b1\n" \
  "s_lshr_b32 s48, s48, 9\n" \
  "s_and_b32 s48, s48, 0x1c0f80\n" \
  "s_add_u32 s54, s54, s48\n" \
  "s_addc_u32 s55, s55, 0\n" \
  "s_add_u32 s48, s54, %[prb]\n" \
  "s_addc_u32 s49, s55, 0\n" \
  "s_xor_b32 s50, %[prb], 0x2000\n" \
  "s_add_u32 s50, s54, s50\n" \
  "s_addc_u32 s51, s55, 0\n" \
  "s_mov_b32 s72, 8\n" \
  "s_waitcnt lgkmcnt(0)\n" \
  "s_mov_b32 s83, 0x7fd3a5c1\n" \
  "s_mov_b32 s84, 0x7fd3a5c1\n" \
  "s_load_dword s83, s[48:49], 0x0\n" \
  "s_load_dword s84, s[50:51], 0x0\n" \
  "s_waitcnt lgkmcnt(1)\n" \
  "s_cmp_eq_u32 s83, 0x7fd3a5c1\n" \
  "s_addc_u32 s72, s72, 0\n" \
  "s_cmp_eq_u32 s84, 0x7fd3a5c1\n" \
  "s_subb_u32 s72, s72, 0\n" \
  "s_waitcnt lgkmcnt(0)\n" \
  "s_mov_b32 s83, 0x7fd3a5c1\n" \
  "s_mov_b32 s84, 0x7fd3a5c1\n" \
  "s_load_dword s84, s[50:51], 0x14000\n" \
  "s_load_dword s83, s[48:49], 0x14000\n" \
  "s_waitcnt lgkmcnt(1)\n" \
  "s_cmp_eq_u32 s83, 0x7fd3a5c1\n" \
  "s_addc_u32 s72, s72, 0\n" \
  "s_cmp_eq_u32 s84, 0x7fd3a5c1\n" \
  "s_subb_u32 s72, s72, 0\n" \
  "s_waitcnt lgkmcnt(0)\n" \
  "L_nr_%=:\n" \
  "s_waitcnt vmcnt(8)\n" \
  "v_readfirstlane_b32 s50, v251\n" \
  "s_nop 0\n" \
  "s_sub_u32 s50, s58, s50\n" \
  "s_mov_b32 s88, 157\n" \
  "s_mov_b32 s89, 176\n" \
  "s_mov_b32 s86, 205\n" \
  "s_mov_b32 s87, 234\n" \
  "s_mov_b32 s82, 460\n" \
  "s_cmp_lt_u32 s50, 3000\n" \
  "s_cselect_b32 s88, 157, s88\n" \
  "s_cselect_b32 s89, 163, s89\n" \
  "s_cselect_b32 s86, 177, s86\n" \
  "s_cselect_b32 s87, 195, s87\n" \
  "s_cselect_b32 s81, 365, s82\n" \
  "s_cselect_b32 s93, 0, -1\n" \
  "s_cselect_b32 s94, 1, 4\n" \
  "v_readfirstlane_b32 s76, v250\n" \
  "s_nop 0\n" \
  "s_add_u32 s76, s76, 1\n" \
  "s_and_b32 s85, s76, 1\n" \
  "s_lshl_b32 s85, s85, 22\n" \
  "s_add_u32 s56, s52, %[foff]\n" \
  "s_addc_u32 s57, s53, 0\n" \
  "s_add_u32 s48, s85, %[doff]\n" \
  "s_add_u32 s52, s52, s48\n" \
  "s_addc_u32 s53, s53, 0\n" \
    \
  "L_w1_%=:\n" \
  "s_getpc_b64 s[48:49]\n" \
  "L_cw_%=:\n" \
  "v_min_u32 v248, L_done_%=-L_cw_%=, %[voffC]\n" \
  "global_load_dword v252, v248, s[48:49]\n" \
  "s_cmp_eq_u32 %[prb], -1\n" \
  "s_cbranch_scc1 L_join_%=\n" \
  "s_getreg_b32 s83, hwreg(HW_REG_XCC_ID)\n" \
  "s_and_b32 s83, s83, 7\n" \
  "s_lshr_b32 s84, %[prb], 11\n" \
  "s_xor_b32 s84, s84, s83\n" \
  "s_sub_u32 s48, 16, s72\n" \
  "s_bitcmp1_b32 s84, 2\n" \
  "s_cselect_b32 s72, s48, s72\n" \
  "s_lshl_b32 s83, s83, 12\n" \
  "s_or_b32 s72, s72, s83\n" \
  "s_lshl_b32 s83, s76, 16\n" \
  "s_or_b32 s72, s72, s83\n" \
  "v_mov_b32 v251, s72\n" \
  "s_nop 0\n" \
  "global_store_dword v249, v251, %[votep] sc1\n" \
    \
  "L_join_%=:\n" \
  "s_add_u32 s81, s81, s58\n" \
  "s_add_u32 s81, s81, %[phi]\n" \
  "s_mov_b64 s[40:41], %[wsp]\n" \
  "s_add_u32 s85, s85, 0x100000\n" \
  "s_add_u32 s40, s40, s85\n" \
  "s_addc_u32 s41, s41, 0\n" \
  "s_mov_b32 s42, 0x400000\n" \
  "s_mov_b32 s43, 0x20000\n" \
  "s_mov_b64 s[44:45], %[optr]\n" \
  "s_mov_b32 s46, 0x2000000\n" \
  "s_mov_b32 s47, 0x20000\n" \
  "s_mov_b32 s78, 0\n" \
  "s_mov_b64 s[34:35], 0\n" \
  "v_add_u32 v242, %[w4k], %[bxr]\n" \
  "v_add_u32 v243, 0x10000, v242\n" \
  "v_mov_b32 v245, %[bxr]\n" \
  "v_add_u32 v246, 0x10000, %[bxr]\n" \
  "v_add_u32 v244, 0x20000, v242\n" \
  "v_add_u32 v247, 0x20000, %[bxr]\n" \
  "s_lshl_b32 s48, %[w4k], 2\n" \
  "s_add_u32 s68, s48, 4096\n" \
  "s_add_u32 s69, s48, 12288\n" \
  "s_add_u32 s70, s48, 0\n" \
  "s_add_u32 s71, s48, 8192\n" \
  "s_waitcnt vmcnt(0)\n" \
  "v_cvt_pk_f16_f32 v214, v102, v106\n" \
  "v_cvt_pk_f16_f32 v218, v103, v107\n" \
  "v_cvt_pk_f16_f32 v222, v104, v108\n" \
  "v_cvt_pk_f16_f32 v226, v105, v109\n" \
  "v_cvt_pk_f16_f32 v215, v110, v114\n" \
  "v_cvt_pk_f16_f32 v219, v111, v115\n" \
  "v_cvt_pk_f16_f32 v223, v112, v116\n" \
  "v_cvt_pk_f16_f32 v227, v113, v117\n" \
  "v_cvt_pk_f16_f32 v216, v118, v122\n" \
  "v_cvt_pk_f16_f32 v220, v119, v123\n" \
  "v_cvt_pk_f16_f32 v224, v120, v124\n" \
  "v_cvt_pk_f16_f32 v228, v121, v125\n" \
  "v_cvt_pk_f16_f32 v217, v126, v130\n" \
  "v_cvt_pk_f16_f32 v221, v127, v131\n" \
  "v_cvt_pk_f16_f32 v225, v128, v132\n" \
  "v_cvt_pk_f16_f32 v229, v129, v133\n" \
  "ds_write_b128 v242, v[214:217] offset:0\n" \
  "ds_write_b128 v242, v[218:221] offset:1024\n" \
  "ds_write_b128 v242, v[222:225] offset:2048\n" \
  "ds_write_b128 v242, v[226:229] offset:3072\n" \
  "v_readlane_b32 s74, %[vtabA], 0\n" \
  "v_readlane_b32 s48, %[voffF], 0\n" \
  "v_and_b32 v251, 0x0, %[bxr]\n" \
  "s_nop 2\n" \
  "v_add_u32 v251, s48, v251\n" \
  "s_mov_b32 s50, 256\n" \
  "L_fs_%=:\n" \
  "global_load_dwordx4 v[238:241], v251, %[wsp] sc0 sc1\n" \
  "s_waitcnt vmcnt(0)\n" \
  "v_min3_u32 v248, v238, v239, v240\n" \
  "v_min_u32 v248, v248, v241\n" \
  "v_cmp_le_u32 s[34:35], s76, v248\n" \
  "s_sub_u32 s50, s50, 1\n" \
  "s_cbranch_scc1 L_fse_%=\n" \
  "s_nop 1\n" \
  "s_bitcmp1_b32 s34, 0\n" \
  "s_cbranch_scc1 L_fse_%=\n" \
  "s_sleep 1\n" \
  "s_branch L_fs_%=\n" \
  "L_fse_%=:\n" \
  "s_nop 2\n" \
  "s_and_b32 s34, s34, 1\n" \
  "s_bitcmp1_b32 s34, 0\n" \
  "s_cbranch_scc0 L_s95_%=\n" \
  "L_b95_%=:\n" \
  "global_load_dwordx4 v[238:241], %[voffF], %[wsp] sc0 sc1\n" \
  "s_add_u32 s72, s74, s68\n" \
  "buffer_load_dwordx4 v[14:17], %[bxr], s[40:43], s72 offen offset:0 sc0 sc1\n" \
  "buffer_load_dwordx4 v[18:21], %[bxr], s[40:43], s72 offen offset:1024 sc0 sc1\n" \
  "buffer_load_dwordx4 v[22:25], %[bxr], s[40:43], s72 offen offset:2048 sc0 sc1\n" \
  "buffer_load_dwordx4 v[26:29], %[bxr], s[40:43], s72 offen offset:3072 sc0 sc1\n" \
  "s_add_u32 s73, s74, s69\n" \
  "buffer_load_dwordx4 v[30:33], %[bxr], s[40:43], s73 offen offset:0 sc0 sc1\n" \
  "buffer_load_dwordx4 v[34:37], %[bxr], s[40:43], s73 offen offset:1024 sc0 sc1\n" \
  "buffer_load_dwordx4 v[38:41], %[bxr], s[40:43], s73 offen offset:2048 sc0 sc1\n" \
  "buffer_load_dwordx4 v[42:45], %[bxr], s[40:43], s73 offen offset:3072 sc0 sc1\n" \
  "v_readlane_b32 s75, %[vtabW], 1\n" \
  "s_nop 0\n" \
  "s_add_u32 s60, s75, 0x0\n" \
  "buffer_load_dwordx4 v[198:201], %[voffW], s[36:39], s60 offen nt\n" \
  "s_add_u32 s61, s75, 0x10000\n" \
  "buffer_load_dwordx4 v[202:205], %[voffW], s[36:39], s61 offen nt\n" \
  "s_add_u32 s62, s75, 0x20000\n" \
  "buffer_load_dwordx4 v[206:209], %[voffW], s[36:39], s62 offen nt\n" \
  "s_add_u32 s63, s75, 0x30000\n" \
  "buffer_load_dwordx4 v[210:213], %[voffW], s[36:39], s63 offen nt\n" \
  "s_add_u32 s64, s75, 0x100000\n" \
  "buffer_load_dwordx4 v[166:169], %[voffW], s[36:39], s64 offen nt\n" \
  "s_add_u32 s65, s75, 0x110000\n" \
  "buffer_load_dwordx4 v[170:173], %[voffW], s[36:39], s65 offen nt\n" \
  "s_add_u32 s66, s75, 0x120000\n" \
  "buffer_load_dwordx4 v[174:177], %[voffW], s[36:39], s66 offen nt\n" \
  "s_add_u32 s67, s75, 0x130000\n" \
  "buffer_load_dwordx4 v[178:181], %[voffW], s[36:39], s67 offen nt\n" \
  "v_readlane_b32 s75, %[vtabW], 2\n" \
  "s_nop 0\n" \
  "s_add_u32 s60, s75, 0x0\n" \
  "buffer_load_dwordx4 v[182:185], %[voffW], s[36:39], s60 offen nt\n" \
  "s_add_u32 s61, s75, 0x10000\n" \
  "buffer_load_dwordx4 v[186:189], %[voffW], s[36:39], s61 offen nt\n" \
  "s_add_u32 s62, s75, 0x20000\n" \
  "buffer_load_dwordx4 v[190:193], %[voffW], s[36:39], s62 offen nt\n" \
  "s_add_u32 s63, s75, 0x30000\n" \
  "buffer_load_dwordx4 v[194:197], %[voffW], s[36:39], s63 offen nt\n" \
  "buffer_load_dword v252, %[voffT], s[36:39], 0 offen nt\n" \
  "s_add_u32 s72, s74, s70\n" \
  "v_mov_b32 v248, s76\n" \
  "s_nop 0\n" \
  "global_store_dword v249, v248, %[callp] sc1\n" \
  "s_waitcnt lgkmcnt(0)\n" \
  "s_barrier\n" \
  "ds_read_b128 v[134:137], v245 offset:8192\n" \
  "ds_read_b128 v[102:105], v245 offset:0\n" \
  "ds_read_b128 v[138:141], v245 offset:9216\n" \
  "ds_read_b128 v[106:109], v245 offset:1024\n" \
  "ds_read_b128 v[142:145], v245 offset:10240\n" \
  "ds_read_b128 v[110:113], v245 offset:2048\n" \
  "ds_read_b128 v[146:149], v245 offset:11264\n" \
  "ds_read_b128 v[114:117], v245 offset:3072\n" \
  "ds_read_b128 v[150:153], v245 offset:12288\n" \
  "ds_read_b128 v[118:121], v245 offset:4096\n" \
  "ds_read_b128 v[154:157], v245 offset:13312\n" \
  "ds_read_b128 v[122:125], v245 offset:5120\n" \
  "ds_read_b128 v[158:161], v245 offset:14336\n" \
  "ds_read_b128 v[126:129], v245 offset:6144\n" \
  "ds_read_b128 v[162:165], v245 offset:15360\n" \
  "ds_read_b128 v[130:133], v245 offset:7168\n" \
  "s_memrealtime s[58:59]\n" \
  "s_waitcnt vmcnt(18)\n" \
  "s_waitcnt lgkmcnt(0)\n" \
    \
  "v_mfma_f32_16x16x32_f16 a[64:67], v[14:17], v[102:105], 0\n" \
  "buffer_load_dwordx4 v[46:49], %[bxr], s[40:43], s72 offen offset:0 sc0 sc1\n" \
  "v_mfma_f32_16x16x32_f16 a[64:67], v[18:21], v[134:137], a[64:67]\n" \
  "s_add_u32 s73, s74, s71\n" \
  "v_mfma_f32_16x16x32_f16 a[96:99], v[26:29], v[134:137], 0\n" \
  "buffer_load_dwordx4 v[50:53], %[bxr], s[40:43], s72 offen offset:1024 sc0 sc1\n" \
  "v_mfma_f32_16x16x32_f16 a[96:99], v[22:25], v[102:105], a[96:99]\n" \
  "v_min3_u32 v248, v238, v239, v240\n" \
  "v_min_u32 v248, v248, v241\n" \
  "v_mfma_f32_16x16x32_f16 a[100:103], v[22:25], v[106:109], 0\n" \
  "buffer_load_dwordx4 v[54:57], %[bxr], s[40:43], s72 offen offset:2048 sc0 sc1\n" \
  "v_mfma_f32_16x16x32_f16 a[100:103], v[26:29], v[138:141], a[100:103]\n" \
  "v_cmp_le_u32 s[34:35], s76, v248\n" \
  "v_mfma_f32_16x16x32_f16 a[68:71], v[18:21], v[138:141], 0\n" \
  "buffer_load_dwordx4 v[58:61], %[bxr], s[40:43], s72 offen offset:3072 sc0 sc1\n" \
  "v_mfma_f32_16x16x32_f16 a[68:71], v[14:17], v[106:109], a[68:71]\n" \
  "v_mfma_f32_16x16x32_f16 a[72:75], v[14:17], v[110:113], 0\n" \
  "buffer_load_dwordx4 v[62:65], %[bxr], s[40:43], s73 offen offset:0 sc0 sc1\n" \
  "v_mfma_f32_16x16x32_f16 a[72:75], v[18:21], v[142:145], a[72:75]\n" \
  "v_readlane_b32 s74, %[vtabA], 1\n" \
  "v_mfma_f32_16x16x32_f16 a[104:107], v[26:29], v[142:145], 0\n" \
  "buffer_load_dwordx4 v[66:69], %[bxr], s[40:43], s73 offen offset:1024 sc0 sc1\n" \
  "v_mfma_f32_16x16x32_f16 a[104:107], v[22:25], v[110:113], a[104:107]\n" \
  "s_bitcmp1_b32 s34, 1\n" \
  "s_cbranch_scc0 L_s46_%=\n" \
  "L_b46_%=:\n" \
  "v_mfma_f32_16x16x32_f16 a[108:111], v[22:25], v[114:117], 0\n" \
  "buffer_load_dwordx4 v[70:73], %[bxr], s[40:43], s73 offen offset:2048 sc0 sc1\n" \
  "v_mfma_f32_16x16x32_f16 a[108:111], v[26:29], v[146:149], a[108:111]\n" \
  "s_add_u32 s72, s74, s68\n" \
  "v_mfma_f32_16x16x32_f16 a[76:79], v[18:21], v[146:149], 0\n" \
  "buffer_load_dwordx4 v[74:77], %[bxr], s[40:43], s73 offen offset:3072 sc0 sc1\n" \
  "v_mfma_f32_16x16x32_f16 a[76:79], v[14:17], v[114:117], a[76:79]\n" \
  "v_mfma_f32_16x16x32_f16 a[80:83], v[14:17], v[118:121], 0\n" \
  "buffer_load_dwordx4 v[78:81], %[bxr], s[40:43], s72 offen offset:0 sc0 sc1\n" \
  "v_mfma_f32_16x16x32_f16 a[80:83], v[18:21], v[150:153], a[80:83]\n" \
  "s_add_u32 s73, s74, s69\n" \
  "v_mfma_f32_16x16x32_f16 a[112:115], v[26:29], v[150:153], 0\n" \
  "buffer_load_dwordx4 v[82:85], %[bxr], s[40:43], s72 offen offset:1024 sc0 sc1\n" \
  "v_mfma_f32_16x16x32_f16 a[112:115], v[22:25], v[118:121], a[112:115]\n" \
  "v_mfma_f32_16x16x32_f16 a[116:119], v[22:25], v[122:125], 0\n" \
  "buffer_load_dwordx4 v[86:89], %[bxr], s[40:43], s72 offen offset:2048 sc0 sc1\n" \
  "v_mfma_f32_16x16x32_f16 a[116:119], v[26:29], v[154:157], a[116:119]\n" \
  "v_mfma_f32_16x16x32_f16 a[84:87], v[18:21], v[154:157], 0\n" \
  "buffer_load_dwordx4 v[90:93], %[bxr], s[40:43], s72 offen offset:3072 sc0 sc1\n" \
  "v_mfma_f32_16x16x32_f16 a[84:87], v[14:17], v[122:125], a[84:87]\n" \
  "s_cmp_lt_i32 s77, 0\n" \
  "s_cbranch_scc1 L_nox_%=\n" \
  "s_mov_b64 s[48:49], %[xptr]\n" \
  "s_add_u32 s48, s48, 0x4000\n" \
  "s_addc_u32 s49, s49, 0\n" \
  "s_add_u32 m0, %[xlds], 0\n" \
  "s_nop 0\n" \
  "global_load_lds_dwordx4 %[xoff0], %[xptr] offset:0 nt\n" \
  "s_add_u32 m0, %[xlds], 1024\n" \
  "s_nop 0\n" \
  "global_load_lds_dwordx4 %[xoff0], s[48:49] offset:0 nt\n" \
  "s_add_u32 m0, %[xlds], 1792\n" \
  "s_nop 0\n" \
  "global_load_lds_dwordx4 %[xoff0], %[xptr] offset:256 nt\n" \
  "s_add_u32 m0, %[xlds], 2816\n" \
  "s_nop 0\n" \
  "global_load_lds_dwordx4 %[xoff0], s[48:49] offset:256 nt\n" \
  "s_add_u32 m0, %[xlds], 4096\n" \
  "s_nop 0\n" \
  "global_load_lds_dwordx4 %[xoff1], %[xptr] offset:0 nt\n" \
  "s_add_u32 m0, %[xlds], 5120\n" \
  "s_nop 0\n" \
  "global_load_lds_dwordx4 %[xoff1], s[48:49] offset:0 nt\n" \
  "s_add_u32 m0, %[xlds], 5888\n" \
  "s_nop 0\n" \
  "global_load_lds_dwordx4 %[xoff1], %[xptr] offset:256 nt\n" \
  "s_add_u32 m0, %[xlds], 6912\n" \
  "s_nop 0\n" \
  "global_load_lds_dwordx4 %[xoff1], s[48:49] offset:256 nt\n" \
  "L_nox_%=:\n" \
  "v_mfma_f32_16x16x32_f16 a[88:91], v[14:17], v[126:129], 0\n" \
  "buffer_load_dwordx4 v[94:97], %[bxr], s[40:43], s73 offen offset:0 sc0 sc1\n" \
  "v_mfma_f32_16x16x32_f16 a[88:91], v[18:21], v[158:161], a[88:91]\n" \
  "v_mfma_f32_16x16x32_f16 a[120:123], v[26:29], v[158:161], 0\n" \
  "buffer_load_dwordx4 v[98:101], %[bxr], s[40:43], s73 offen offset:1024 sc0 sc1\n" \
  "v_mfma_f32_16x16x32_f16 a[120:123], v[22:25], v[126:129], a[120:123]\n" \
  "v_mfma_f32_16x16x32_f16 a[124:127], v[22:25], v[130:133], 0\n" \
  "v_mfma_f32_16x16x32_f16 a[124:127], v[26:29], v[162:165], a[124:127]\n" \
  "v_mfma_f32_16x16x32_f16 a[92:95], v[18:21], v[162:165], 0\n" \
  "buffer_load_dwordx4 v[22:25], %[bxr], s[40:43], s73 offen offset:2048 sc0 sc1\n" \
  "v_mfma_f32_16x16x32_f16 a[92:95], v[14:17], v[130:133], a[92:95]\n" \
  "s_waitcnt vmcnt(28)\n" \
  "s_add_u32 s72, s74, s70\n" \
  "v_mfma_f32_16x16x32_f16 a[192:195], v[30:33], v[102:105], 0\n" \
  "v_lshrrev_b32 v238, 1, %[bxr]\n" \
  "global_load_dwordx2 v[238:239], v238, %[wsp] offset:1024 sc1\n" \
  "v_mfma_f32_16x16x32_f16 a[192:195], v[34:37], v[134:137], a[192:195]\n" \
  "s_memrealtime s[58:59]\n" \
  "v_mfma_f32_16x16x32_f16 a[224:227], v[42:45], v[134:137], 0\n" \
  "buffer_load_dwordx4 v[26:29], %[bxr], s[40:43], s73 offen offset:3072 sc0 sc1\n" \
  "v_mfma_f32_16x16x32_f16 a[224:227], v[38:41], v[102:105], a[224:227]\n" \
  "v_mfma_f32_16x16x32_f16 a[228:231], v[38:41], v[106:109], 0\n" \
  "v_mfma_f32_16x16x32_f16 a[228:231], v[42:45], v[138:141], a[228:231]\n" \
  "v_mfma_f32_16x16x32_f16 a[196:199], v[34:37], v[138:141], 0\n" \
  "buffer_load_dwordx4 v[18:21], %[bxr], s[40:43], s72 offen offset:0 sc0 sc1\n" \
  "v_mfma_f32_16x16x32_f16 a[196:199], v[30:33], v[106:109], a[196:199]\n" \
  "v_mfma_f32_16x16x32_f16 a[200:203], v[30:33], v[110:113], 0\n" \
  "v_mfma_f32_16x16x32_f16 a[200:203], v[34:37], v[142:145], a[200:203]\n" \
  "s_waitcnt vmcnt(28)\n" \
  "s_waitcnt lgkmcnt(0)\n" \
  "v_mfma_f32_16x16x32_f16 a[232:235], v[42:45], v[142:145], 0\n" \
  "s_sub_u32 s82, s81, s58\n" \
  "v_cvt_pk_f16_f32 v214, v198, v202\n" \
  "v_cvt_pk_f16_f32 v218, v199, v203\n" \
  "v_mfma_f32_16x16x32_f16 a[232:235], v[38:41], v[110:113], a[232:235]\n" \
  "s_waitcnt vmcnt(26)\n" \
  "v_cvt_pk_f16_f32 v222, v200, v204\n" \
  "v_cvt_pk_f16_f32 v226, v201, v205\n" \
  "v_mfma_f32_16x16x32_f16 a[236:239], v[38:41], v[114:117], 0\n" \
  "v_cvt_pk_f16_f32 v215, v206, v210\n" \
  "v_cvt_pk_f16_f32 v219, v207, v211\n" \
  "v_mfma_f32_16x16x32_f16 a[236:239], v[42:45], v[146:149], a[236:239]\n" \
  "s_waitcnt vmcnt(24)\n" \
  "v_cvt_pk_f16_f32 v223, v208, v212\n" \
  "v_cvt_pk_f16_f32 v227, v209, v213\n" \
  "v_mfma_f32_16x16x32_f16 a[204:207], v[34:37], v[146:149], 0\n" \
  "v_cvt_pk_f16_f32 v216, v166, v170\n" \
  "v_cvt_pk_f16_f32 v220, v167, v171\n" \
  "v_mfma_f32_16x16x32_f16 a[204:207], v[30:33], v[114:117], a[204:207]\n" \
  "s_waitcnt vmcnt(22)\n" \
  "v_cvt_pk_f16_f32 v224, v168, v172\n" \
  "v_cvt_pk_f16_f32 v228, v169, v173\n" \
  "v_mfma_f32_16x16x32_f16 a[208:211], v[30:33], v[118:121], 0\n" \
  "v_cvt_pk_f16_f32 v217, v174, v178\n" \
  "v_cvt_pk_f16_f32 v221, v175, v179\n" \
  "v_mfma_f32_16x16x32_f16 a[208:211], v[34:37], v[150:153], a[208:211]\n" \
  "v_cvt_pk_f16_f32 v225, v176, v180\n" \
  "v_cvt_pk_f16_f32 v229, v177, v181\n" \
  "v_mfma_f32_16x16x32_f16 a[240:243], v[42:45], v[150:153], 0\n" \
  "ds_write_b128 v242, v[214:217] offset:16384\n" \
  "v_mfma_f32_16x16x32_f16 a[240:243], v[38:41], v[118:121], a[240:243]\n" \
  "buffer_load_dwordx4 v[14:17], %[bxr], s[40:43], s72 offen offset:1024 sc0 sc1\n" \
  "v_mfma_f32_16x16x32_f16 a[244:247], v[38:41], v[122:125], 0\n" \
  "s_cmp_gt_i32 s82, 0\n" \
  "s_cbranch_scc1 L_s22_%=\n" \
  "L_b22_%=:\n" \
  "s_max_i32 s82, s82, -150\n" \
  "v_mfma_f32_16x16x32_f16 a[244:247], v[42:45], v[154:157], a[244:247]\n" \
  "s_add_u32 s81, s58, s82\n" \
  "s_add_u32 s81, s81, s87\n" \
  "v_mfma_f32_16x16x32_f16 a[212:215], v[34:37], v[154:157], 0\n" \
  "ds_write_b128 v242, v[218:221] offset:17408\n" \
  "v_mfma_f32_16x16x32_f16 a[212:215], v[30:33], v[122:125], a[212:215]\n" \
  "v_mfma_f32_16x16x32_f16 a[216:219], v[30:33], v[126:129], 0\n" \
  "v_mfma_f32_16x16x32_f16 a[216:219], v[34:37], v[158:161], a[216:219]\n" \
  "v_mfma_f32_16x16x32_f16 a[248:251], v[42:45], v[158:161], 0\n" \
  "ds_write_b128 v242, v[222:225] offset:18432\n" \
  "v_mfma_f32_16x16x32_f16 a[248:251], v[38:41], v[126:129], a[248:251]\n" \
  "v_mfma_f32_16x16x32_f16 a[252:255], v[38:41], v[130:133], 0\n" \
  "buffer_load_dwordx4 v[38:41], %[bxr], s[40:43], s72 offen offset:2048 sc0 sc1\n" \
  "v_mfma_f32_16x16x32_f16 a[252:255], v[42:45], v[162:165], a[252:255]\n" \
  "v_mfma_f32_16x16x32_f16 a[220:223], v[34:37], v[162:165], 0\n" \
  "ds_write_b128 v242, v[226:229] offset:19456\n" \
  "s_waitcnt vmcnt(4)\n" \
  "s_lshl_b32 s85, s76, 16\n" \
  "s_nop 0\n" \
  "v_xor_b32 v240, s85, v238\n" \
  "v_xor_b32 v241, s85, v239\n" \
  "v_or_b32 v248, v240, v241\n" \
  "v_cmp_lt_u32 vcc, 0xffff, v248\n" \
  "s_nop 4\n" \
  "s_cmp_eq_u64 vcc, 0\n" \
  "s_cbranch_scc0 L_s45_%=\n" \
  "L_b45_%=:\n" \
  "v_readlane_b32 s50, v238, %[cls]\n" \
  "s_nop 3\n" \
  "s_lshr_b32 s50, s50, 14\n" \
  "s_xor_b32 s50, s50, %[khx]\n" \
  "s_and_b32 s50, s50, 1\n" \
  "v_and_b32 v248, 0xfff, v239\n" \
  "v_cmp_lt_u32 vcc, 8, v248\n" \
  "v_cmp_gt_u32 s[48:49], 8, v248\n" \
  "s_nop 3\n" \
  "s_bcnt1_i32_b64 s79, vcc\n" \
  "s_bcnt1_i32_b64 s48, s[48:49]\n" \
  "s_cmp_gt_u32 s79, s48\n" \
  "s_cselect_b32 s92, 1, 0\n" \
  "s_xor_b32 s92, s92, s50\n" \
  "s_lshl_b32 s92, s92, 12\n" \
  "s_xor_b32 s91, s70, s92\n" \
  "s_lshl_b32 s91, s91, 9\n" \
  "s_xor_b32 s90, s68, s92\n" \
  "s_lshl_b32 s90, s90, 9\n" \
  "v_and_b32 v248, 0xfff, v238\n" \
  "v_cmp_lt_u32 vcc, 8, v248\n" \
  "v_cmp_gt_u32 s[48:49], 8, v248\n" \
  "s_nop 3\n" \
  "s_bcnt1_i32_b64 s79, vcc\n" \
  "s_bcnt1_i32_b64 s48, s[48:49]\n" \
  "s_cmp_gt_u32 s79, s48\n" \
  "s_cselect_b32 s48, 1, 0\n" \
  "s_xor_b32 s48, s48, s50\n" \
  "s_mov_b32 vcc_lo, 0xfffffff8\n" \
  "s_mov_b32 vcc_hi, vcc_lo\n" \
  "v_mov_b32 v248, s48\n" \
  "s_nop 1\n" \
  "v_cndmask_b32 v248, 0, v248, vcc\n" \
  "v_lshl_add_u32 %[vtabW], v248, 21, %[vtabW]\n" \
  "v_lshl_add_u32 %[vtabA], v248, 16, %[vtabA]\n" \
  "v_lshl_add_u32 %[voffF], v248, 9, %[voffF]\n" \
  "s_and_b32 s34, s34, 7\n" \
  "v_mov_b32 v238, 0\n" \
  "v_mfma_f32_16x16x32_f16 a[220:223], v[30:33], v[130:133], a[220:223]\n" \
  "s_add_u32 s64, s75, 0x100000\n" \
  "s_add_u32 s65, s75, 0x110000\n" \
  "v_mfma_f32_16x16x32_f16 a[0:3], v[46:49], v[102:105], 0\n" \
  "buffer_load_dwordx4 v[42:45], %[bxr], s[40:43], s72 offen offset:3072 sc0 sc1\n" \
  "v_mfma_f32_16x16x32_f16 a[0:3], v[50:53], v[134:137], a[0:3]\n" \
  "s_add_u32 s66, s75, 0x120000\n" \
  "s_add_u32 s67, s75, 0x130000\n" \
  "s_add_u32 s73, s74, s71\n" \
  "v_mfma_f32_16x16x32_f16 a[32:35], v[58:61], v[134:137], 0\n" \
  "v_readlane_b32 s75, %[vtabW], 3\n" \
  "v_mfma_f32_16x16x32_f16 a[32:35], v[54:57], v[102:105], a[32:35]\n" \
  "v_cvt_pk_f16_f32 v214, v182, v186\n" \
  "v_cvt_pk_f16_f32 v218, v183, v187\n" \
  "v_mfma_f32_16x16x32_f16 a[36:39], v[54:57], v[106:109], 0\n" \
  "v_cvt_pk_f16_f32 v222, v184, v188\n" \
  "v_cvt_pk_f16_f32 v226, v185, v189\n" \
  "v_mfma_f32_16x16x32_f16 a[36:39], v[58:61], v[138:141], a[36:39]\n" \
  "s_add_u32 s60, s75, 0x0\n" \
  "v_cvt_pk_f16_f32 v215, v190, v194\n" \
  "v_cvt_pk_f16_f32 v219, v191, v195\n" \
  "v_mfma_f32_16x16x32_f16 a[4:7], v[50:53], v[138:141], 0\n" \
  "s_add_u32 s61, s75, 0x10000\n" \
  "v_cvt_pk_f16_f32 v223, v192, v196\n" \
  "v_cvt_pk_f16_f32 v227, v193, v197\n" \
  "v_mfma_f32_16x16x32_f16 a[4:7], v[46:49], v[106:109], a[4:7]\n" \
  "buffer_load_dwordx4 v[34:37], %[bxr], s[40:43], s73 offen offset:0 sc0 sc1\n" \
  "v_mfma_f32_16x16x32_f16 a[8:11], v[46:49], v[110:113], 0\n" \
  "v_mfma_f32_16x16x32_f16 a[8:11], v[50:53], v[142:145], a[8:11]\n" \
  "v_mfma_f32_16x16x32_f16 a[40:43], v[58:61], v[142:145], 0\n" \
  "v_mfma_f32_16x16x32_f16 a[40:43], v[54:57], v[110:113], a[40:43]\n" \
  "buffer_load_dwordx4 v[30:33], %[bxr], s[40:43], s73 offen offset:1024 sc0 sc1\n" \
  "v_mfma_f32_16x16x32_f16 a[44:47], v[54:57], v[114:117], 0\n" \
  "v_mfma_f32_16x16x32_f16 a[44:47], v[58:61], v[146:149], a[44:47]\n" \
  "v_mfma_f32_16x16x32_f16 a[12:15], v[50:53], v[146:149], 0\n" \
  "v_mfma_f32_16x16x32_f16 a[12:15], v[46:49], v[114:117], a[12:15]\n" \
  "v_mfma_f32_16x16x32_f16 a[16:19], v[46:49], v[118:121], 0\n" \
  "v_mfma_f32_16x16x32_f16 a[16:19], v[50:53], v[150:153], a[16:19]\n" \
  "v_mfma_f32_16x16x32_f16 a[48:51], v[58:61], v[150:153], 0\n" \
  "v_mfma_f32_16x16x32_f16 a[48:51], v[54:57], v[118:121], a[48:51]\n" \
  "v_mfma_f32_16x16x32_f16 a[52:55], v[54:57], v[122:125], 0\n" \
  "v_mfma_f32_16x16x32_f16 a[52:55], v[58:61], v[154:157], a[52:55]\n" \
  "v_mfma_f32_16x16x32_f16 a[20:23], v[50:53], v[154:157], 0\n" \
  "v_mfma_f32_16x16x32_f16 a[20:23], v[46:49], v[122:125], a[20:23]\n" \
  "s_andn2_b32 s79, 0x1f, s34\n" \
  "s_cbranch_scc1 L_pl1_%=\n" \
  "L_pb1_%=:\n" \
  "v_mfma_f32_16x16x32_f16 a[24:27], v[46:49], v[126:129], 0\n" \
  "v_mfma_f32_16x16x32_f16 a[24:27], v[50:53], v[158:161], a[24:27]\n" \
  "s_barrier\n" \
  "v_mfma_f32_16x16x32_f16 a[56:59], v[58:61], v[158:161], 0\n" \
  "buffer_load_dwordx4 v[198:201], %[voffW], s[36:39], s64 offen nt\n" \
  "v_mfma_f32_16x16x32_f16 a[56:59], v[54:57], v[126:129], a[56:59]\n" \
  "v_mfma_f32_16x16x32_f16 a[60:63], v[54:57], v[130:133], 0\n" \
  "buffer_load_dwordx4 v[202:205], %[voffW], s[36:39], s65 offen nt\n" \
  "v_mfma_f32_16x16x32_f16 a[60:63], v[58:61], v[162:165], a[60:63]\n" \
  "v_mfma_f32_16x16x32_f16 a[28:31], v[50:53], v[162:165], 0\n" \
  "buffer_load_dwordx4 v[206:209], %[voffW], s[36:39], s66 offen nt\n" \
  "v_mfma_f32_16x16x32_f16 a[28:31], v[46:49], v[130:133], a[28:31]\n" \
  "v_mfma_f32_16x16x32_f16 a[128:131], v[62:65], v[102:105], 0\n" \
  "buffer_load_dwordx4 v[210:213], %[voffW], s[36:39], s67 offen nt\n" \
  "v_mfma_f32_16x16x32_f16 a[128:131], v[66:69], v[134:137], a[128:131]\n" \
  "s_waitcnt lgkmcnt(0)\n" \
  "s_barrier\n" \
  "v_mfma_f32_16x16x32_f16 a[160:163], v[74:77], v[134:137], 0\n" \
  "ds_read_b128 v[134:137], v245 offset:24576\n" \
  "v_mfma_f32_16x16x32_f16 a[160:163], v[70:73], v[102:105], a[160:163]\n" \
  "ds_read_b128 v[102:105], v245 offset:16384\n" \
  "v_mfma_f32_16x16x32_f16 a[164:167], v[70:73], v[106:109], 0\n" \
  "buffer_load_dwordx4 v[166:169], %[voffW], s[36:39], s60 offen nt\n" \
  "v_mfma_f32_16x16x32_f16 a[164:167], v[74:77], v[138:141], a[164:167]\n" \
  "v_mfma_f32_16x16x32_f16 a[132:135], v[66:69], v[138:141], 0\n" \
  "ds_read_b128 v[138:141], v245 offset:25600\n" \
  "v_mfma_f32_16x16x32_f16 a[132:135], v[62:65], v[106:109], a[132:135]\n" \
  "ds_read_b128 v[106:109], v245 offset:17408\n" \
  "v_mfma_f32_16x16x32_f16 a[136:139], v[62:65], v[110:113], 0\n" \
  "buffer_load_dwordx4 v[170:173], %[voffW], s[36:39], s61 offen nt\n" \
  "v_mfma_f32_16x16x32_f16 a[136:139], v[66:69], v[142:145], a[136:139]\n" \
  "s_cmp_eq_u32 s77, 0\n" \
  "s_cbranch_scc1 L_s77_%=\n" \
  "L_b77_%=:\n" \
  "v_mfma_f32_16x16x32_f16 a[168:171], v[74:77], v[142:145], 0\n" \
  "ds_read_b128 v[142:145], v245 offset:26624\n" \
  "v_mfma_f32_16x16x32_f16 a[168:171], v[70:73], v[110:113], a[168:171]\n" \
  "ds_read_b128 v[110:113], v245 offset:18432\n" \
  "v_mfma_f32_16x16x32_f16 a[172:175], v[70:73], v[114:117], 0\n" \
  "v_mfma_f32_16x16x32_f16 a[172:175], v[74:77], v[146:149], a[172:175]\n" \
  "v_mfma_f32_16x16x32_f16 a[140:143], v[66:69], v[146:149], 0\n" \
  "ds_read_b128 v[146:149], v245 offset:27648\n" \
  "v_mfma_f32_16x16x32_f16 a[140:143], v[62:65], v[114:117], a[140:143]\n" \
  "ds_read_b128 v[114:117], v245 offset:19456\n" \
  "v_mfma_f32_16x16x32_f16 a[144:147], v[62:65], v[118:121], 0\n" \
  "v_mfma_f32_16x16x32_f16 a[144:147], v[66:69], v[150:153], a[144:147]\n" \
  "v_mfma_f32_16x16x32_f16 a[176:179], v[74:77], v[150:153], 0\n" \
  "ds_read_b128 v[150:153], v245 offset:28672\n" \
  "v_mfma_f32_16x16x32_f16 a[176:179], v[70:73], v[118:121], a[176:179]\n" \
  "ds_read_b128 v[118:121], v245 offset:20480\n" \
  "v_mfma_f32_16x16x32_f16 a[180:183], v[70:73], v[122:125], 0\n" \
  "v_mfma_f32_16x16x32_f16 a[180:183], v[74:77], v[154:157], a[180:183]\n" \
  "v_mfma_f32_16x16x32_f16 a[148:151], v[66:69], v[154:157], 0\n" \
  "ds_read_b128 v[154:157], v245 offset:29696\n" \
  "v_mfma_f32_16x16x32_f16 a[148:151], v[62:65], v[122:125], a[148:151]\n" \
  "ds_read_b128 v[122:125], v245 offset:21504\n" \
  "v_mfma_f32_16x16x32_f16 a[152:155], v[62:65], v[126:129], 0\n" \
  "buffer_load_dwordx4 v[54:57], %[bxr], s[40:43], s73 offen offset:2048 sc0 sc1\n" \
  "v_mfma_f32_16x16x32_f16 a[152:155], v[66:69], v[158:161], a[152:155]\n" \
  "v_readlane_b32 s74, %[vtabA], 2\n" \
  "v_mfma_f32_16x16x32_f16 a[184:187], v[74:77], v[158:161], 0\n" \
  "s_bitcmp1_b32 s34, 2\n" \
  "s_cbranch_scc0 L_s47_%=\n" \
  "L_b47_%=:\n" \
  "ds_read_b128 v[158:161], v245 offset:30720\n" \
  "v_mfma_f32_16x16x32_f16 a[184:187], v[70:73], v[126:129], a[184:187]\n" \
  "s_add_u32 s72, s74, s68\n" \
  "ds_read_b128 v[126:129], v245 offset:22528\n" \
  "v_mfma_f32_16x16x32_f16 a[188:191], v[70:73], v[130:133], 0\n" \
  "buffer_load_dwordx4 v[58:61], %[bxr], s[40:43], s73 offen offset:3072 sc0 sc1\n" \
  "v_mfma_f32_16x16x32_f16 a[188:191], v[74:77], v[162:165], a[188:191]\n" \
  "s_add_u32 s62, s75, 0x20000\n" \
  "s_add_u32 s63, s75, 0x30000\n" \
  "s_add_u32 s64, s75, 0x100000\n" \
  "v_mfma_f32_16x16x32_f16 a[156:159], v[66:69], v[162:165], 0\n" \
  "s_add_u32 s65, s75, 0x110000\n" \
  "ds_read_b128 v[162:165], v245 offset:31744\n" \
  "s_add_u32 s66, s75, 0x120000\n" \
  "v_mfma_f32_16x16x32_f16 a[156:159], v[62:65], v[130:133], a[156:159]\n" \
  "ds_read_b128 v[130:133], v245 offset:23552\n" \
  "s_waitcnt lgkmcnt(14)\n" \
    \
  "v_mfma_f32_16x16x32_f16 a[64:67], v[78:81], v[102:105], a[64:67]\n" \
  "buffer_load_dwordx4 v[50:53], %[bxr], s[40:43], s72 offen offset:0 sc0 sc1\n" \
  "v_mfma_f32_16x16x32_f16 a[64:67], v[82:85], v[134:137], a[64:67]\n" \
  "s_add_u32 s67, s75, 0x130000\n" \
  "v_mfma_f32_16x16x32_f16 a[96:99], v[90:93], v[134:137], a[96:99]\n" \
  "v_mfma_f32_16x16x32_f16 a[96:99], v[86:89], v[102:105], a[96:99]\n" \
  "s_waitcnt lgkmcnt(12)\n" \
  "v_mfma_f32_16x16x32_f16 a[100:103], v[86:89], v[106:109], a[100:103]\n" \
  "buffer_load_dwordx4 v[46:49], %[bxr], s[40:43], s72 offen offset:1024 sc0 sc1\n" \
  "v_mfma_f32_16x16x32_f16 a[100:103], v[90:93], v[138:141], a[100:103]\n" \
  "v_mfma_f32_16x16x32_f16 a[68:71], v[82:85], v[138:141], a[68:71]\n" \
  "v_mfma_f32_16x16x32_f16 a[68:71], v[78:81], v[106:109], a[68:71]\n" \
  "s_waitcnt lgkmcnt(10)\n" \
  "v_mfma_f32_16x16x32_f16 a[72:75], v[78:81], v[110:113], a[72:75]\n" \
  "buffer_load_dwordx4 v[70:73], %[bxr], s[40:43], s72 offen offset:2048 sc0 sc1\n" \
  "v_mfma_f32_16x16x32_f16 a[72:75], v[82:85], v[142:145], a[72:75]\n" \
  "s_add_u32 s73, s74, s69\n" \
  "v_mfma_f32_16x16x32_f16 a[104:107], v[90:93], v[142:145], a[104:107]\n" \
  "v_mfma_f32_16x16x32_f16 a[104:107], v[86:89], v[110:113], a[104:107]\n" \
  "s_waitcnt lgkmcnt(8)\n" \
  "v_mfma_f32_16x16x32_f16 a[108:111], v[86:89], v[114:117], a[108:111]\n" \
  "buffer_load_dwordx4 v[74:77], %[bxr], s[40:43], s72 offen offset:3072 sc0 sc1\n" \
  "v_mfma_f32_16x16x32_f16 a[108:111], v[90:93], v[146:149], a[108:111]\n" \
  "v_mfma_f32_16x16x32_f16 a[76:79], v[82:85], v[146:149], a[76:79]\n" \
  "s_waitcnt vmcnt(12)\n" \
  "v_mfma_f32_16x16x32_f16 a[76:79], v[78:81], v[114:117], a[76:79]\n" \
  "s_waitcnt lgkmcnt(6)\n" \
  "v_min3_u32 v248, v238, v239, v240\n" \
  "v_min_u32 v248, v248, v241\n" \
  "v_mfma_f32_16x16x32_f16 a[80:83], v[78:81], v[118:121], a[80:83]\n" \
  "buffer_load_dwordx4 v[66:69], %[bxr], s[40:43], s73 offen offset:0 sc0 sc1\n" \
  "v_mfma_f32_16x16x32_f16 a[80:83], v[82:85], v[150:153], a[80:83]\n" \
  "v_cmp_le_u32 s[34:35], s76, v248\n" \
  "v_mfma_f32_16x16x32_f16 a[112:115], v[90:93], v[150:153], a[112:115]\n" \
  "v_mfma_f32_16x16x32_f16 a[112:115], v[86:89], v[118:121], a[112:115]\n" \
  "s_waitcnt lgkmcnt(4)\n" \
  "v_mfma_f32_16x16x32_f16 a[116:119], v[86:89], v[122:125], a[116:119]\n" \
  "buffer_load_dwordx4 v[62:65], %[bxr], s[40:43], s73 offen offset:1024 sc0 sc1\n" \
  "v_mfma_f32_16x16x32_f16 a[116:119], v[90:93], v[154:157], a[116:119]\n" \
  "v_mfma_f32_16x16x32_f16 a[84:87], v[82:85], v[154:157], a[84:87]\n" \
  "v_mfma_f32_16x16x32_f16 a[84:87], v[78:81], v[122:125], a[84:87]\n" \
  "s_waitcnt lgkmcnt(2)\n" \
  "v_mfma_f32_16x16x32_f16 a[88:91], v[78:81], v[126:129], a[88:91]\n" \
  "v_mfma_f32_16x16x32_f16 a[88:91], v[82:85], v[158:161], a[88:91]\n" \
  "s_barrier\n" \
  "v_mfma_f32_16x16x32_f16 a[120:123], v[90:93], v[158:161], a[120:123]\n" \
  "buffer_load_dwordx4 v[174:177], %[voffW], s[36:39], s62 offen nt\n" \
  "v_mfma_f32_16x16x32_f16 a[120:123], v[86:89], v[126:129], a[120:123]\n" \
  "s_waitcnt lgkmcnt(0)\n" \
  "v_mfma_f32_16x16x32_f16 a[124:127], v[86:89], v[130:133], a[124:127]\n" \
  "buffer_load_dwordx4 v[178:181], %[voffW], s[36:39], s63 offen nt\n" \
  "v_mfma_f32_16x16x32_f16 a[124:127], v[90:93], v[162:165], a[124:127]\n" \
  "v_mfma_f32_16x16x32_f16 a[92:95], v[82:85], v[162:165], a[92:95]\n" \
  "buffer_load_dwordx4 v[182:185], %[voffW], s[36:39], s64 offen nt\n" \
  "v_mfma_f32_16x16x32_f16 a[92:95], v[78:81], v[130:133], a[92:95]\n" \
  "v_mfma_f32_16x16x32_f16 a[192:195], v[94:97], v[102:105], a[192:195]\n" \
  "buffer_load_dwordx4 v[186:189], %[voffW], s[36:39], s65 offen nt\n" \
  "v_mfma_f32_16x16x32_f16 a[192:195], v[98:101], v[134:137], a[192:195]\n" \
  "s_memrealtime s[58:59]\n" \
  "v_mfma_f32_16x16x32_f16 a[224:227], v[26:29], v[134:137], a[224:227]\n" \
  "buffer_load_dwordx4 v[190:193], %[voffW], s[36:39], s66 offen nt\n" \
  "v_mfma_f32_16x16x32_f16 a[224:227], v[22:25], v[102:105], a[224:227]\n" \
  "v_mfma_f32_16x16x32_f16 a[228:231], v[22:25], v[106:109], a[228:231]\n" \
  "buffer_load_dwordx4 v[194:197], %[voffW], s[36:39], s67 offen nt\n" \
  "v_mfma_f32_16x16x32_f16 a[228:231], v[26:29], v[138:141], a[228:231]\n" \
  "s_cmp_eq_u32 s77, 1\n" \
  "s_cbranch_scc1 L_s78_%=\n" \
  "L_b78_%=:\n" \
  "v_mfma_f32_16x16x32_f16 a[196:199], v[98:101], v[138:141], a[196:199]\n" \
  "v_mfma_f32_16x16x32_f16 a[196:199], v[94:97], v[106:109], a[196:199]\n" \
  "v_mfma_f32_16x16x32_f16 a[200:203], v[94:97], v[110:113], a[200:203]\n" \
  "v_mfma_f32_16x16x32_f16 a[200:203], v[98:101], v[142:145], a[200:203]\n" \
  "s_waitcnt lgkmcnt(0)\n" \
  "v_mfma_f32_16x16x32_f16 a[232:235], v[26:29], v[142:145], a[232:235]\n" \
  "s_sub_u32 s82, s81, s58\n" \
  "s_cmp_gt_i32 s82, 0\n" \
  "s_cbranch_scc1 L_s23_%=\n" \
  "L_b23_%=:\n" \
  "v_mfma_f32_16x16x32_f16 a[232:235], v[22:25], v[110:113], a[232:235]\n" \
  "s_max_i32 s82, s82, -150\n" \
  "s_add_u32 s81, s58, s82\n" \
  "s_add_u32 s81, s81, s87\n" \
  "v_mfma_f32_16x16x32_f16 a[236:239], v[22:25], v[114:117], a[236:239]\n" \
  "v_mfma_f32_16x16x32_f16 a[236:239], v[26:29], v[146:149], a[236:239]\n" \
  "v_mfma_f32_16x16x32_f16 a[204:207], v[98:101], v[146:149], a[204:207]\n" \
  "v_mfma_f32_16x16x32_f16 a[204:207], v[94:97], v[114:117], a[204:207]\n" \
  "v_mfma_f32_16x16x32_f16 a[208:211], v[94:97], v[118:121], a[208:211]\n" \
  "v_mfma_f32_16x16x32_f16 a[208:211], v[98:101], v[150:153], a[208:211]\n" \
  "v_mfma_f32_16x16x32_f16 a[240:243], v[26:29], v[150:153], a[240:243]\n" \
  "v_mfma_f32_16x16x32_f16 a[240:243], v[22:25], v[118:121], a[240:243]\n" \
  "v_mfma_f32_16x16x32_f16 a[244:247], v[22:25], v[122:125], a[244:247]\n" \
  "v_mfma_f32_16x16x32_f16 a[244:247], v[26:29], v[154:157], a[244:247]\n" \
  "s_waitcnt vmcnt(18)\n" \
  "v_mfma_f32_16x16x32_f16 a[212:215], v[98:101], v[154:157], a[212:215]\n" \
  "v_cvt_pk_f16_f32 v216, v198, v202\n" \
  "v_cvt_pk_f16_f32 v220, v199, v203\n" \
  "v_mfma_f32_16x16x32_f16 a[212:215], v[94:97], v[122:125], a[212:215]\n" \
  "s_waitcnt vmcnt(16)\n" \
  "v_cvt_pk_f16_f32 v224, v200, v204\n" \
  "v_cvt_pk_f16_f32 v228, v201, v205\n" \
  "v_mfma_f32_16x16x32_f16 a[216:219], v[94:97], v[126:129], a[216:219]\n" \
  "v_cvt_pk_f16_f32 v217, v206, v210\n" \
  "v_cvt_pk_f16_f32 v221, v207, v211\n" \
  "v_mfma_f32_16x16x32_f16 a[216:219], v[98:101], v[158:161], a[216:219]\n" \
  "v_cvt_pk_f16_f32 v225, v208, v212\n" \
  "v_cvt_pk_f16_f32 v229, v209, v213\n" \
  "v_mfma_f32_16x16x32_f16 a[248:251], v[26:29], v[158:161], a[248:251]\n" \
  "ds_write_b128 v242, v[214:217] offset:32768\n" \
  "v_mfma_f32_16x16x32_f16 a[248:251], v[22:25], v[126:129], a[248:251]\n" \
  "buffer_load_dwordx4 v[86:89], %[bxr], s[40:43], s73 offen offset:2048 sc0 sc1\n" \
  "v_mfma_f32_16x16x32_f16 a[252:255], v[22:25], v[130:133], a[252:255]\n" \
  "s_add_u32 s72, s74, s70\n" \
  "v_readlane_b32 s75, %[vtabW], 4\n" \
  "v_mfma_f32_16x16x32_f16 a[252:255], v[26:29], v[162:165], a[252:255]\n" \
  "v_mfma_f32_16x16x32_f16 a[220:223], v[98:101], v[162:165], a[220:223]\n" \
  "ds_write_b128 v242, v[218:221] offset:33792\n" \
  "v_mfma_f32_16x16x32_f16 a[220:223], v[94:97], v[130:133], a[220:223]\n" \
  "buffer_load_dwordx4 v[90:93], %[bxr], s[40:43], s73 offen offset:3072 sc0 sc1\n" \
  "v_mfma_f32_16x16x32_f16 a[0:3], v[18:21], v[102:105], a[0:3]\n" \
  "s_add_u32 s60, s75, 0x0\n" \
  "s_add_u32 s61, s75, 0x10000\n" \
  "s_add_u32 s62, s75, 0x20000\n" \
  "v_mfma_f32_16x16x32_f16 a[0:3], v[14:17], v[134:137], a[0:3]\n" \
  "s_add_u32 s63, s75, 0x30000\n" \
  "s_add_u32 s64, s75, 0x100000\n" \
  "s_add_u32 s65, s75, 0x110000\n" \
  "v_mfma_f32_16x16x32_f16 a[32:35], v[42:45], v[134:137], a[32:35]\n" \
  "ds_write_b128 v242, v[222:225] offset:34816\n" \
  "v_mfma_f32_16x16x32_f16 a[32:35], v[38:41], v[102:105], a[32:35]\n" \
  "buffer_load_dwordx4 v[82:85], %[bxr], s[40:43], s72 offen offset:0 sc0 sc1\n" \
  "v_mfma_f32_16x16x32_f16 a[36:39], v[38:41], v[106:109], a[36:39]\n" \
  "v_mfma_f32_16x16x32_f16 a[36:39], v[42:45], v[138:141], a[36:39]\n" \
  "v_mfma_f32_16x16x32_f16 a[4:7], v[14:17], v[138:141], a[4:7]\n" \
  "ds_write_b128 v242, v[226:229] offset:35840\n" \
  "v_mfma_f32_16x16x32_f16 a[4:7], v[18:21], v[106:109], a[4:7]\n" \
  "buffer_load_dwordx4 v[78:81], %[bxr], s[40:43], s72 offen offset:1024 sc0 sc1\n" \
  "v_mfma_f32_16x16x32_f16 a[8:11], v[18:21], v[110:113], a[8:11]\n" \
  "v_mfma_f32_16x16x32_f16 a[8:11], v[14:17], v[142:145], a[8:11]\n" \
  "v_mfma_f32_16x16x32_f16 a[40:43], v[42:45], v[142:145], a[40:43]\n" \
  "s_waitcnt vmcnt(18)\n" \
  "v_mfma_f32_16x16x32_f16 a[40:43], v[38:41], v[110:113], a[40:43]\n" \
  "v_cvt_pk_f16_f32 v214, v166, v170\n" \
  "v_cvt_pk_f16_f32 v218, v167, v171\n" \
  "v_mfma_f32_16x16x32_f16 a[44:47], v[38:41], v[114:117], a[44:47]\n" \
  "v_cvt_pk_f16_f32 v222, v168, v172\n" \
  "v_cvt_pk_f16_f32 v226, v169, v173\n" \
  "v_mfma_f32_16x16x32_f16 a[44:47], v[42:45], v[146:149], a[44:47]\n" \
  "buffer_load_dwordx4 v[22:25], %[bxr], s[40:43], s72 offen offset:2048 sc0 sc1\n" \
  "v_mfma_f32_16x16x32_f16 a[12:15], v[14:17], v[146:149], a[12:15]\n" \
  "s_add_u32 s73, s74, s71\n" \
  "v_mfma_f32_16x16x32_f16 a[12:15], v[18:21], v[114:117], a[12:15]\n" \
  "v_mfma_f32_16x16x32_f16 a[16:19], v[18:21], v[118:121], a[16:19]\n" \
  "v_mfma_f32_16x16x32_f16 a[16:19], v[14:17], v[150:153], a[16:19]\n" \
  "buffer_load_dwordx4 v[26:29], %[bxr], s[40:43], s72 offen offset:3072 sc0 sc1\n" \
  "v_mfma_f32_16x16x32_f16 a[48:51], v[42:45], v[150:153], a[48:51]\n" \
  "v_mfma_f32_16x16x32_f16 a[48:51], v[38:41], v[118:121], a[48:51]\n" \
  "v_mfma_f32_16x16x32_f16 a[52:55], v[38:41], v[122:125], a[52:55]\n" \
  "v_mfma_f32_16x16x32_f16 a[52:55], v[42:45], v[154:157], a[52:55]\n" \
  "buffer_load_dwordx4 v[98:101], %[bxr], s[40:43], s73 offen offset:0 sc0 sc1\n" \
  "v_mfma_f32_16x16x32_f16 a[20:23], v[14:17], v[154:157], a[20:23]\n" \
  "v_mfma_f32_16x16x32_f16 a[20:23], v[18:21], v[122:125], a[20:23]\n" \
  "s_andn2_b32 s79, 0x3f, s34\n" \
  "s_cbranch_scc1 L_pl2_%=\n" \
  "L_pb2_%=:\n" \
  "v_mfma_f32_16x16x32_f16 a[24:27], v[18:21], v[126:129], a[24:27]\n" \
  "v_mfma_f32_16x16x32_f16 a[24:27], v[14:17], v[158:161], a[24:27]\n" \
  "s_barrier\n" \
  "v_mfma_f32_16x16x32_f16 a[56:59], v[42:45], v[158:161], a[56:59]\n" \
  "buffer_load_dwordx4 v[198:201], %[voffW], s[36:39], s60 offen nt\n" \
  "v_mfma_f32_16x16x32_f16 a[56:59], v[38:41], v[126:129], a[56:59]\n" \
  "v_mfma_f32_16x16x32_f16 a[60:63], v[38:41], v[130:133], a[60:63]\n" \
  "buffer_load_dwordx4 v[202:205], %[voffW], s[36:39], s61 offen nt\n" \
  "v_mfma_f32_16x16x32_f16 a[60:63], v[42:45], v[162:165], a[60:63]\n" \
  "v_mfma_f32_16x16x32_f16 a[28:31], v[14:17], v[162:165], a[28:31]\n" \
  "buffer_load_dwordx4 v[206:209], %[voffW], s[36:39], s62 offen nt\n" \
  "v_mfma_f32_16x16x32_f16 a[28:31], v[18:21], v[130:133], a[28:31]\n" \
  "s_waitcnt vmcnt(22)\n" \
  "v_mfma_f32_16x16x32_f16 a[128:131], v[34:37], v[102:105], a[128:131]\n" \
  "buffer_load_dwordx4 v[210:213], %[voffW], s[36:39], s63 offen nt\n" \
  "v_mfma_f32_16x16x32_f16 a[128:131], v[30:33], v[134:137], a[128:131]\n" \
  "s_waitcnt lgkmcnt(0)\n" \
  "s_barrier\n" \
  "v_mfma_f32_16x16x32_f16 a[160:163], v[58:61], v[134:137], a[160:163]\n" \
  "ds_read_b128 v[134:137], v245 offset:40960\n" \
  "v_mfma_f32_16x16x32_f16 a[160:163], v[54:57], v[102:105], a[160:163]\n" \
  "ds_read_b128 v[102:105], v245 offset:32768\n" \
  "v_mfma_f32_16x16x32_f16 a[164:167], v[54:57], v[106:109], a[164:167]\n" \
  "buffer_load_dwordx4 v[166:169], %[voffW], s[36:39], s64 offen nt\n" \
  "v_mfma_f32_16x16x32_f16 a[164:167], v[58:61], v[138:141], a[164:167]\n" \
  "v_mfma_f32_16x16x32_f16 a[132:135], v[30:33], v[138:141], a[132:135]\n" \
  "ds_read_b128 v[138:141], v245 offset:41984\n" \
  "v_mfma_f32_16x16x32_f16 a[132:135], v[34:37], v[106:109], a[132:135]\n" \
  "ds_read_b128 v[106:109], v245 offset:33792\n" \
  "v_mfma_f32_16x16x32_f16 a[136:139], v[34:37], v[110:113], a[136:139]\n" \
  "buffer_load_dwordx4 v[170:173], %[voffW], s[36:39], s65 offen nt\n" \
  "v_mfma_f32_16x16x32_f16 a[136:139], v[30:33], v[142:145], a[136:139]\n" \
  "s_cmp_eq_u32 s77, 2\n" \
  "s_cbranch_scc1 L_s79_%=\n" \
  "L_b79_%=:\n" \
  "v_mfma_f32_16x16x32_f16 a[168:171], v[58:61], v[142:145], a[168:171]\n" \
  "ds_read_b128 v[142:145], v245 offset:43008\n" \
  "v_mfma_f32_16x16x32_f16 a[168:171], v[54:57], v[110:113], a[168:171]\n" \
  "ds_read_b128 v[110:113], v245 offset:34816\n" \
  "v_mfma_f32_16x16x32_f16 a[172:175], v[54:57], v[114:117], a[172:175]\n" \
  "v_mfma_f32_16x16x32_f16 a[172:175], v[58:61], v[146:149], a[172:175]\n" \
  "v_mfma_f32_16x16x32_f16 a[140:143], v[30:33], v[146:149], a[140:143]\n" \
  "ds_read_b128 v[146:149], v245 offset:44032\n" \
  "v_mfma_f32_16x16x32_f16 a[140:143], v[34:37], v[114:117], a[140:143]\n" \
  "ds_read_b128 v[114:117], v245 offset:35840\n" \
  "v_mfma_f32_16x16x32_f16 a[144:147], v[34:37], v[118:121], a[144:147]\n" \
  "v_mfma_f32_16x16x32_f16 a[144:147], v[30:33], v[150:153], a[144:147]\n" \
  "v_mfma_f32_16x16x32_f16 a[176:179], v[58:61], v[150:153], a[176:179]\n" \
  "ds_read_b128 v[150:153], v245 offset:45056\n" \
  "v_mfma_f32_16x16x32_f16 a[176:179], v[54:57], v[118:121], a[176:179]\n" \
  "ds_read_b128 v[118:121], v245 offset:36864\n" \
  "v_mfma_f32_16x16x32_f16 a[180:183], v[54:57], v[122:125], a[180:183]\n" \
  "v_mfma_f32_16x16x32_f16 a[180:183], v[58:61], v[154:157], a[180:183]\n" \
  "v_mfma_f32_16x16x32_f16 a[148:151], v[30:33], v[154:157], a[148:151]\n" \
  "ds_read_b128 v[154:157], v245 offset:46080\n" \
  "v_mfma_f32_16x16x32_f16 a[148:151], v[34:37], v[122:125], a[148:151]\n" \
  "ds_read_b128 v[122:125], v245 offset:37888\n" \
  "v_mfma_f32_16x16x32_f16 a[152:155], v[34:37], v[126:129], a[152:155]\n" \
  "buffer_load_dwordx4 v[94:97], %[bxr], s[40:43], s73 offen offset:1024 sc0 sc1\n" \
  "v_mfma_f32_16x16x32_f16 a[152:155], v[30:33], v[158:161], a[152:155]\n" \
  "v_mfma_f32_16x16x32_f16 a[184:187], v[58:61], v[158:161], a[184:187]\n" \
  "ds_read_b128 v[158:161], v245 offset:47104\n" \
  "v_mfma_f32_16x16x32_f16 a[184:187], v[54:57], v[126:129], a[184:187]\n" \
  "ds_read_b128 v[126:129], v245 offset:38912\n" \
  "v_mfma_f32_16x16x32_f16 a[188:191], v[54:57], v[130:133], a[188:191]\n" \
  "buffer_load_dwordx4 v[38:41], %[bxr], s[40:43], s73 offen offset:2048 sc0 sc1\n" \
  "v_mfma_f32_16x16x32_f16 a[188:191], v[58:61], v[162:165], a[188:191]\n" \
  "s_add_u32 s66, s75, 0x120000\n" \
  "v_readlane_b32 s74, %[vtabA], 3\n" \
  "s_add_u32 s67, s75, 0x130000\n" \
  "v_mfma_f32_16x16x32_f16 a[156:159], v[30:33], v[162:165], a[156:159]\n" \
  "ds_read_b128 v[162:165], v245 offset:48128\n" \
  "v_readlane_b32 s75, %[vtabW], 5\n" \
  "v_mfma_f32_16x16x32_f16 a[156:159], v[34:37], v[130:133], a[156:159]\n" \
  "s_waitcnt vmcnt(23)\n" \
  "ds_read_b128 v[130:133], v245 offset:39936\n" \
  "s_waitcnt lgkmcnt(14)\n" \
    \
  "v_mfma_f32_16x16x32_f16 a[64:67], v[50:53], v[102:105], a[64:67]\n" \
  "buffer_load_dwordx4 v[42:45], %[bxr], s[40:43], s73 offen offset:3072 sc0 sc1\n" \
  "v_mfma_f32_16x16x32_f16 a[64:67], v[46:49], v[134:137], a[64:67]\n" \
  "s_add_u32 s60, s75, 0x0\n" \
  "s_add_u32 s61, s75, 0x10000\n" \
  "s_add_u32 s62, s75, 0x20000\n" \
  "v_mfma_f32_16x16x32_f16 a[96:99], v[74:77], v[134:137], a[96:99]\n" \
  "s_add_u32 s63, s75, 0x30000\n" \
  "s_bitcmp1_b32 s34, 3\n" \
  "s_cbranch_scc0 L_s48_%=\n" \
  "L_b48_%=:\n" \
  "v_mfma_f32_16x16x32_f16 a[96:99], v[70:73], v[102:105], a[96:99]\n" \
  "s_waitcnt lgkmcnt(12)\n" \
  "s_add_u32 s72, s74, s68\n" \
  "v_mfma_f32_16x16x32_f16 a[100:103], v[70:73], v[106:109], a[100:103]\n" \
  "buffer_load_dwordx4 v[14:17], %[bxr], s[40:43], s72 offen offset:0 sc0 sc1\n" \
  "v_mfma_f32_16x16x32_f16 a[100:103], v[74:77], v[138:141], a[100:103]\n" \
  "v_mfma_f32_16x16x32_f16 a[68:71], v[46:49], v[138:141], a[68:71]\n" \
  "v_mfma_f32_16x16x32_f16 a[68:71], v[50:53], v[106:109], a[68:71]\n" \
  "s_waitcnt lgkmcnt(10)\n" \
  "v_mfma_f32_16x16x32_f16 a[72:75], v[50:53], v[110:113], a[72:75]\n" \
  "buffer_load_dwordx4 v[18:21], %[bxr], s[40:43], s72 offen offset:1024 sc0 sc1\n" \
  "v_mfma_f32_16x16x32_f16 a[72:75], v[46:49], v[142:145], a[72:75]\n" \
  "v_mfma_f32_16x16x32_f16 a[104:107], v[74:77], v[142:145], a[104:107]\n" \
  "s_waitcnt vmcnt(22)\n" \
  "v_mfma_f32_16x16x32_f16 a[104:107], v[70:73], v[110:113], a[104:107]\n" \
  "s_waitcnt lgkmcnt(8)\n" \
  "v_cvt_pk_f16_f32 v215, v174, v178\n" \
  "v_cvt_pk_f16_f32 v219, v175, v179\n" \
  "v_mfma_f32_16x16x32_f16 a[108:111], v[70:73], v[114:117], a[108:111]\n" \
  "s_waitcnt vmcnt(20)\n" \
  "v_cvt_pk_f16_f32 v223, v176, v180\n" \
  "v_cvt_pk_f16_f32 v227, v177, v181\n" \
  "v_mfma_f32_16x16x32_f16 a[108:111], v[74:77], v[146:149], a[108:111]\n" \
  "s_waitcnt vmcnt(11)\n" \
  "v_cvt_pk_f16_f32 v216, v182, v186\n" \
  "v_cvt_pk_f16_f32 v220, v183, v187\n" \
  "v_mfma_f32_16x16x32_f16 a[76:79], v[46:49], v[146:149], a[76:79]\n" \
  "v_cvt_pk_f16_f32 v224, v184, v188\n" \
  "v_cvt_pk_f16_f32 v228, v185, v189\n" \
  "v_mfma_f32_16x16x32_f16 a[76:79], v[50:53], v[114:117], a[76:79]\n" \
  "s_waitcnt lgkmcnt(6)\n" \
  "v_cvt_pk_f16_f32 v217, v190, v194\n" \
  "v_cvt_pk_f16_f32 v221, v191, v195\n" \
  "v_mfma_f32_16x16x32_f16 a[80:83], v[50:53], v[118:121], a[80:83]\n" \
  "v_cvt_pk_f16_f32 v225, v192, v196\n" \
  "v_cvt_pk_f16_f32 v229, v193, v197\n" \
  "v_mfma_f32_16x16x32_f16 a[80:83], v[46:49], v[150:153], a[80:83]\n" \
  "ds_write_b128 v242, v[214:217] offset:49152\n" \
  "v_mfma_f32_16x16x32_f16 a[112:115], v[74:77], v[150:153], a[112:115]\n" \
  "buffer_load_dwordx4 v[54:57], %[bxr], s[40:43], s72 offen offset:2048 sc0 sc1\n" \
  "v_mfma_f32_16x16x32_f16 a[112:115], v[70:73], v[118:121], a[112:115]\n" \
  "s_waitcnt lgkmcnt(5)\n" \
  "v_min3_u32 v248, v238, v239, v240\n" \
  "v_min_u32 v248, v248, v241\n" \
  "v_mfma_f32_16x16x32_f16 a[116:119], v[70:73], v[122:125], a[116:119]\n" \
  "v_cmp_le_u32 s[34:35], s76, v248\n" \
  "v_mfma_f32_16x16x32_f16 a[116:119], v[74:77], v[154:157], a[116:119]\n" \
  "ds_write_b128 v242, v[218:221] offset:50176\n" \
  "v_mfma_f32_16x16x32_f16 a[84:87], v[46:49], v[154:157], a[84:87]\n" \
  "buffer_load_dwordx4 v[58:61], %[bxr], s[40:43], s72 offen offset:3072 sc0 sc1\n" \
  "v_mfma_f32_16x16x32_f16 a[84:87], v[50:53], v[122:125], a[84:87]\n" \
  "s_waitcnt lgkmcnt(4)\n" \
  "v_mfma_f32_16x16x32_f16 a[88:91], v[50:53], v[126:129], a[88:91]\n" \
  "s_barrier\n" \
  "v_mfma_f32_16x16x32_f16 a[88:91], v[46:49], v[158:161], a[88:91]\n" \
  "ds_write_b128 v242, v[222:225] offset:51200\n" \
  "v_mfma_f32_16x16x32_f16 a[120:123], v[74:77], v[158:161], a[120:123]\n" \
  "buffer_load_dwordx4 v[174:177], %[voffW], s[36:39], s66 offen nt\n" \
  "v_mfma_f32_16x16x32_f16 a[120:123], v[70:73], v[126:129], a[120:123]\n" \
  "s_waitcnt lgkmcnt(3)\n" \
  "v_mfma_f32_16x16x32_f16 a[124:127], v[70:73], v[130:133], a[124:127]\n" \
  "buffer_load_dwordx4 v[178:181], %[voffW], s[36:39], s67 offen nt\n" \
  "v_mfma_f32_16x16x32_f16 a[124:127], v[74:77], v[162:165], a[124:127]\n" \
  "ds_write_b128 v242, v[226:229] offset:52224\n" \
  "v_mfma_f32_16x16x32_f16 a[92:95], v[46:49], v[162:165], a[92:95]\n" \
  "buffer_load_dwordx4 v[182:185], %[voffW], s[36:39], s60 offen nt\n" \
  "v_mfma_f32_16x16x32_f16 a[92:95], v[50:53], v[130:133], a[92:95]\n" \
  "v_mfma_f32_16x16x32_f16 a[192:195], v[66:69], v[102:105], a[192:195]\n" \
  "buffer_load_dwordx4 v[186:189], %[voffW], s[36:39], s61 offen nt\n" \
  "v_mfma_f32_16x16x32_f16 a[192:195], v[62:65], v[134:137], a[192:195]\n" \
  "s_memrealtime s[58:59]\n" \
  "v_mfma_f32_16x16x32_f16 a[224:227], v[90:93], v[134:137], a[224:227]\n" \
  "buffer_load_dwordx4 v[190:193], %[voffW], s[36:39], s62 offen nt\n" \
  "v_mfma_f32_16x16x32_f16 a[224:227], v[86:89], v[102:105], a[224:227]\n" \
  "v_mfma_f32_16x16x32_f16 a[228:231], v[86:89], v[106:109], a[228:231]\n" \
  "buffer_load_dwordx4 v[194:197], %[voffW], s[36:39], s63 offen nt\n" \
  "v_mfma_f32_16x16x32_f16 a[228:231], v[90:93], v[138:141], a[228:231]\n" \
  "s_cmp_eq_u32 s77, 3\n" \
  "s_cbranch_scc1 L_s80_%=\n" \
  "L_b80_%=:\n" \
  "v_mfma_f32_16x16x32_f16 a[196:199], v[62:65], v[138:141], a[196:199]\n" \
  "v_mfma_f32_16x16x32_f16 a[196:199], v[66:69], v[106:109], a[196:199]\n" \
  "v_mfma_f32_16x16x32_f16 a[200:203], v[66:69], v[110:113], a[200:203]\n" \
  "v_mfma_f32_16x16x32_f16 a[200:203], v[62:65], v[142:145], a[200:203]\n" \
  "s_waitcnt lgkmcnt(0)\n" \
  "v_mfma_f32_16x16x32_f16 a[232:235], v[90:93], v[142:145], a[232:235]\n" \
  "s_sub_u32 s82, s81, s58\n" \
  "s_cmp_gt_i32 s82, 0\n" \
  "s_cbranch_scc1 L_s24_%=\n" \
  "L_b24_%=:\n" \
  "v_mfma_f32_16x16x32_f16 a[232:235], v[86:89], v[110:113], a[232:235]\n" \
  "s_max_i32 s82, s82, -150\n" \
  "s_add_u32 s81, s58, s82\n" \
  "s_add_u32 s81, s81, s87\n" \
  "v_mfma_f32_16x16x32_f16 a[236:239], v[86:89], v[114:117], a[236:239]\n" \
  "v_mfma_f32_16x16x32_f16 a[236:239], v[90:93], v[146:149], a[236:239]\n" \
  "v_mfma_f32_16x16x32_f16 a[204:207], v[62:65], v[146:149], a[204:207]\n" \
  "s_add_u32 s73, s74, s69\n" \
  "v_mfma_f32_16x16x32_f16 a[204:207], v[66:69], v[114:117], a[204:207]\n" \
  "v_mfma_f32_16x16x32_f16 a[208:211], v[66:69], v[118:121], a[208:211]\n" \
  "v_mfma_f32_16x16x32_f16 a[208:211], v[62:65], v[150:153], a[208:211]\n" \
  "v_mfma_f32_16x16x32_f16 a[240:243], v[90:93], v[150:153], a[240:243]\n" \
  "v_mfma_f32_16x16x32_f16 a[240:243], v[86:89], v[118:121], a[240:243]\n" \
  "v_mfma_f32_16x16x32_f16 a[244:247], v[86:89], v[122:125], a[244:247]\n" \
  "v_mfma_f32_16x16x32_f16 a[244:247], v[90:93], v[154:157], a[244:247]\n" \
  "v_mfma_f32_16x16x32_f16 a[212:215], v[62:65], v[154:157], a[212:215]\n" \
  "buffer_load_dwordx4 v[30:33], %[bxr], s[40:43], s73 offen offset:0 sc0 sc1\n" \
  "v_mfma_f32_16x16x32_f16 a[212:215], v[66:69], v[122:125], a[212:215]\n" \
  "v_mfma_f32_16x16x32_f16 a[216:219], v[66:69], v[126:129], a[216:219]\n" \
  "v_mfma_f32_16x16x32_f16 a[216:219], v[62:65], v[158:161], a[216:219]\n" \
  "v_mfma_f32_16x16x32_f16 a[248:251], v[90:93], v[158:161], a[248:251]\n" \
  "buffer_load_dwordx4 v[34:37], %[bxr], s[40:43], s73 offen offset:1024 sc0 sc1\n" \
  "v_mfma_f32_16x16x32_f16 a[248:251], v[86:89], v[126:129], a[248:251]\n" \
  "v_mfma_f32_16x16x32_f16 a[252:255], v[86:89], v[130:133], a[252:255]\n" \
  "s_add_u32 s64, s75, 0x100000\n" \
  "s_add_u32 s65, s75, 0x110000\n" \
  "s_add_u32 s66, s75, 0x120000\n" \
  "v_mfma_f32_16x16x32_f16 a[252:255], v[90:93], v[162:165], a[252:255]\n" \
  "s_add_u32 s67, s75, 0x130000\n" \
  "v_mfma_f32_16x16x32_f16 a[220:223], v[62:65], v[162:165], a[220:223]\n" \
  "buffer_load_dwordx4 v[70:73], %[bxr], s[40:43], s73 offen offset:2048 sc0 sc1\n" \
  "v_mfma_f32_16x16x32_f16 a[220:223], v[66:69], v[130:133], a[220:223]\n" \
  "v_readlane_b32 s75, %[vtabW], 6\n" \
  "s_add_u32 s72, s74, s70\n" \
  "v_mfma_f32_16x16x32_f16 a[0:3], v[82:85], v[102:105], a[0:3]\n" \
  "v_mfma_f32_16x16x32_f16 a[0:3], v[78:81], v[134:137], a[0:3]\n" \
  "s_add_u32 s60, s75, 0x0\n" \
  "s_add_u32 s61, s75, 0x10000\n" \
  "v_mfma_f32_16x16x32_f16 a[32:35], v[26:29], v[134:137], a[32:35]\n" \
  "buffer_load_dwordx4 v[74:77], %[bxr], s[40:43], s73 offen offset:3072 sc0 sc1\n" \
  "v_mfma_f32_16x16x32_f16 a[32:35], v[22:25], v[102:105], a[32:35]\n" \
  "v_mfma_f32_16x16x32_f16 a[36:39], v[22:25], v[106:109], a[36:39]\n" \
  "v_mfma_f32_16x16x32_f16 a[36:39], v[26:29], v[138:141], a[36:39]\n" \
  "v_mfma_f32_16x16x32_f16 a[4:7], v[78:81], v[138:141], a[4:7]\n" \
  "buffer_load_dwordx4 v[46:49], %[bxr], s[40:43], s72 offen offset:0 sc0 sc1\n" \
  "v_mfma_f32_16x16x32_f16 a[4:7], v[82:85], v[106:109], a[4:7]\n" \
  "v_mfma_f32_16x16x32_f16 a[8:11], v[82:85], v[110:113], a[8:11]\n" \
  "v_mfma_f32_16x16x32_f16 a[8:11], v[78:81], v[142:145], a[8:11]\n" \
  "v_mfma_f32_16x16x32_f16 a[40:43], v[26:29], v[142:145], a[40:43]\n" \
  "s_waitcnt vmcnt(22)\n" \
  "v_mfma_f32_16x16x32_f16 a[40:43], v[22:25], v[110:113], a[40:43]\n" \
  "v_cvt_pk_f16_f32 v214, v198, v202\n" \
  "v_cvt_pk_f16_f32 v218, v199, v203\n" \
  "v_mfma_f32_16x16x32_f16 a[44:47], v[22:25], v[114:117], a[44:47]\n" \
  "s_waitcnt vmcnt(20)\n" \
  "v_cvt_pk_f16_f32 v222, v200, v204\n" \
  "v_cvt_pk_f16_f32 v226, v201, v205\n" \
  "v_mfma_f32_16x16x32_f16 a[44:47], v[26:29], v[146:149], a[44:47]\n" \
  "v_cvt_pk_f16_f32 v215, v206, v210\n" \
  "v_cvt_pk_f16_f32 v219, v207, v211\n" \
  "v_mfma_f32_16x16x32_f16 a[12:15], v[78:81], v[146:149], a[12:15]\n" \
  "s_waitcnt vmcnt(18)\n" \
  "v_cvt_pk_f16_f32 v223, v208, v212\n" \
  "v_cvt_pk_f16_f32 v227, v209, v213\n" \
  "v_mfma_f32_16x16x32_f16 a[12:15], v[82:85], v[114:117], a[12:15]\n" \
  "v_cvt_pk_f16_f32 v216, v166, v170\n" \
  "v_cvt_pk_f16_f32 v220, v167, v171\n" \
  "v_mfma_f32_16x16x32_f16 a[16:19], v[82:85], v[118:121], a[16:19]\n" \
  "v_cvt_pk_f16_f32 v224, v168, v172\n" \
  "v_cvt_pk_f16_f32 v228, v169, v173\n" \
  "v_mfma_f32_16x16x32_f16 a[16:19], v[78:81], v[150:153], a[16:19]\n" \
  "buffer_load_dwordx4 v[50:53], %[bxr], s[40:43], s72 offen offset:1024 sc0 sc1\n" \
  "v_mfma_f32_16x16x32_f16 a[48:51], v[26:29], v[150:153], a[48:51]\n" \
  "v_mfma_f32_16x16x32_f16 a[48:51], v[22:25], v[118:121], a[48:51]\n" \
  "v_mfma_f32_16x16x32_f16 a[52:55], v[22:25], v[122:125], a[52:55]\n" \
  "v_mfma_f32_16x16x32_f16 a[52:55], v[26:29], v[154:157], a[52:55]\n" \
  "buffer_load_dwordx4 v[86:89], %[bxr], s[40:43], s72 offen offset:2048 sc0 sc1\n" \
  "v_mfma_f32_16x16x32_f16 a[20:23], v[78:81], v[154:157], a[20:23]\n" \
  "v_mfma_f32_16x16x32_f16 a[20:23], v[82:85], v[122:125], a[20:23]\n" \
  "s_andn2_b32 s79, 0x7f, s34\n" \
  "s_cbranch_scc1 L_pl3_%=\n" \
  "L_pb3_%=:\n" \
  "v_mfma_f32_16x16x32_f16 a[24:27], v[82:85], v[126:129], a[24:27]\n" \
  "v_mfma_f32_16x16x32_f16 a[24:27], v[78:81], v[158:161], a[24:27]\n" \
  "s_barrier\n" \
  "v_mfma_f32_16x16x32_f16 a[56:59], v[26:29], v[158:161], a[56:59]\n" \
  "buffer_load_dwordx4 v[198:201], %[voffW], s[36:39], s64 offen nt\n" \
  "v_mfma_f32_16x16x32_f16 a[56:59], v[22:25], v[126:129], a[56:59]\n" \
  "v_mfma_f32_16x16x32_f16 a[60:63], v[22:25], v[130:133], a[60:63]\n" \
  "buffer_load_dwordx4 v[202:205], %[voffW], s[36:39], s65 offen nt\n" \
  "v_mfma_f32_16x16x32_f16 a[60:63], v[26:29], v[162:165], a[60:63]\n" \
  "v_mfma_f32_16x16x32_f16 a[28:31], v[78:81], v[162:165], a[28:31]\n" \
  "buffer_load_dwordx4 v[206:209], %[voffW], s[36:39], s66 offen nt\n" \
  "v_mfma_f32_16x16x32_f16 a[28:31], v[82:85], v[130:133], a[28:31]\n" \
  "s_waitcnt vmcnt(20)\n" \
  "v_mfma_f32_16x16x32_f16 a[128:131], v[98:101], v[102:105], a[128:131]\n" \
  "buffer_load_dwordx4 v[210:213], %[voffW], s[36:39], s67 offen nt\n" \
  "v_mfma_f32_16x16x32_f16 a[128:131], v[94:97], v[134:137], a[128:131]\n" \
  "v_mfma_f32_16x16x32_f16 a[160:163], v[42:45], v[134:137], a[160:163]\n" \
  "ds_read_b128 v[134:137], v245 offset:57344\n" \
  "v_mfma_f32_16x16x32_f16 a[160:163], v[38:41], v[102:105], a[160:163]\n" \
  "ds_read_b128 v[102:105], v245 offset:49152\n" \
  "v_mfma_f32_16x16x32_f16 a[164:167], v[38:41], v[106:109], a[164:167]\n" \
  "buffer_load_dwordx4 v[166:169], %[voffW], s[36:39], s60 offen nt\n" \
  "v_mfma_f32_16x16x32_f16 a[164:167], v[42:45], v[138:141], a[164:167]\n" \
  "v_mfma_f32_16x16x32_f16 a[132:135], v[94:97], v[138:141], a[132:135]\n" \
  "ds_read_b128 v[138:141], v245 offset:58368\n" \
  "v_mfma_f32_16x16x32_f16 a[132:135], v[98:101], v[106:109], a[132:135]\n" \
  "ds_read_b128 v[106:109], v245 offset:50176\n" \
  "v_mfma_f32_16x16x32_f16 a[136:139], v[98:101], v[110:113], a[136:139]\n" \
  "buffer_load_dwordx4 v[170:173], %[voffW], s[36:39], s61 offen nt\n" \
  "v_mfma_f32_16x16x32_f16 a[136:139], v[94:97], v[142:145], a[136:139]\n" \
  "s_cmp_eq_u32 s77, 4\n" \
  "s_cbranch_scc1 L_s81_%=\n" \
  "L_b81_%=:\n" \
  "v_mfma_f32_16x16x32_f16 a[168:171], v[42:45], v[142:145], a[168:171]\n" \
  "ds_read_b128 v[142:145], v245 offset:59392\n" \
  "v_mfma_f32_16x16x32_f16 a[168:171], v[38:41], v[110:113], a[168:171]\n" \
  "ds_read_b128 v[110:113], v245 offset:51200\n" \
  "v_mfma_f32_16x16x32_f16 a[172:175], v[38:41], v[114:117], a[172:175]\n" \
  "v_mfma_f32_16x16x32_f16 a[172:175], v[42:45], v[146:149], a[172:175]\n" \
  "v_mfma_f32_16x16x32_f16 a[140:143], v[94:97], v[146:149], a[140:143]\n" \
  "ds_read_b128 v[146:149], v245 offset:60416\n" \
  "v_mfma_f32_16x16x32_f16 a[140:143], v[98:101], v[114:117], a[140:143]\n" \
  "ds_read_b128 v[114:117], v245 offset:52224\n" \
  "v_mfma_f32_16x16x32_f16 a[144:147], v[98:101], v[118:121], a[144:147]\n" \
  "v_mfma_f32_16x16x32_f16 a[144:147], v[94:97], v[150:153], a[144:147]\n" \
  "v_mfma_f32_16x16x32_f16 a[176:179], v[42:45], v[150:153], a[176:179]\n" \
  "ds_read_b128 v[150:153], v245 offset:61440\n" \
  "v_mfma_f32_16x16x32_f16 a[176:179], v[38:41], v[118:121], a[176:179]\n" \
  "ds_read_b128 v[118:121], v245 offset:53248\n" \
  "v_mfma_f32_16x16x32_f16 a[180:183], v[38:41], v[122:125], a[180:183]\n" \
  "v_mfma_f32_16x16x32_f16 a[180:183], v[42:45], v[154:157], a[180:183]\n" \
  "s_waitcnt vmcnt(17)\n" \
  "v_mfma_f32_16x16x32_f16 a[148:151], v[94:97], v[154:157], a[148:151]\n" \
  "ds_read_b128 v[154:157], v245 offset:62464\n" \
  "v_cvt_pk_f16_f32 v217, v174, v178\n" \
  "v_mfma_f32_16x16x32_f16 a[148:151], v[98:101], v[122:125], a[148:151]\n" \
  "ds_read_b128 v[122:125], v245 offset:54272\n" \
  "v_cvt_pk_f16_f32 v221, v175, v179\n" \
  "v_mfma_f32_16x16x32_f16 a[152:155], v[98:101], v[126:129], a[152:155]\n" \
  "s_add_u32 s73, s74, s71\n" \
  "v_cvt_pk_f16_f32 v225, v176, v180\n" \
  "v_cvt_pk_f16_f32 v229, v177, v181\n" \
  "v_mfma_f32_16x16x32_f16 a[152:155], v[94:97], v[158:161], a[152:155]\n" \
  "ds_write_b128 v243, v[214:217] offset:0\n" \
  "v_mfma_f32_16x16x32_f16 a[184:187], v[42:45], v[158:161], a[184:187]\n" \
  "ds_read_b128 v[158:161], v245 offset:63488\n" \
  "v_mfma_f32_16x16x32_f16 a[184:187], v[38:41], v[126:129], a[184:187]\n" \
  "ds_read_b128 v[126:129], v245 offset:55296\n" \
  "v_mfma_f32_16x16x32_f16 a[188:191], v[38:41], v[130:133], a[188:191]\n" \
  "buffer_load_dwordx4 v[90:93], %[bxr], s[40:43], s72 offen offset:3072 sc0 sc1\n" \
  "v_mfma_f32_16x16x32_f16 a[188:191], v[42:45], v[162:165], a[188:191]\n" \
  "ds_write_b128 v243, v[218:221] offset:1024\n" \
  "v_mfma_f32_16x16x32_f16 a[156:159], v[94:97], v[162:165], a[156:159]\n" \
  "s_add_u32 s62, s75, 0x20000\n" \
  "ds_read_b128 v[162:165], v245 offset:64512\n" \
  "s_add_u32 s63, s75, 0x30000\n" \
  "v_mfma_f32_16x16x32_f16 a[156:159], v[98:101], v[130:133], a[156:159]\n" \
  "ds_read_b128 v[130:133], v245 offset:56320\n" \
  "s_waitcnt lgkmcnt(14)\n" \
    \
  "v_mfma_f32_16x16x32_f16 a[64:67], v[14:17], v[102:105], a[64:67]\n" \
  "buffer_load_dwordx4 v[62:65], %[bxr], s[40:43], s73 offen offset:0 sc0 sc1\n" \
  "v_mfma_f32_16x16x32_f16 a[64:67], v[18:21], v[134:137], a[64:67]\n" \
  "ds_write_b128 v243, v[222:225] offset:2048\n" \
  "v_mfma_f32_16x16x32_f16 a[96:99], v[58:61], v[134:137], a[96:99]\n" \
  "s_add_u32 s64, s75, 0x100000\n" \
  "s_add_u32 s65, s75, 0x110000\n" \
  "s_add_u32 s66, s75, 0x120000\n" \
  "v_mfma_f32_16x16x32_f16 a[96:99], v[54:57], v[102:105], a[96:99]\n" \
  "s_waitcnt lgkmcnt(14)\n" \
  "s_add_u32 s67, s75, 0x130000\n" \
  "v_mfma_f32_16x16x32_f16 a[100:103], v[54:57], v[106:109], a[100:103]\n" \
  "buffer_load_dwordx4 v[66:69], %[bxr], s[40:43], s73 offen offset:1024 sc0 sc1\n" \
  "v_mfma_f32_16x16x32_f16 a[100:103], v[58:61], v[138:141], a[100:103]\n" \
  "ds_write_b128 v243, v[226:229] offset:3072\n" \
  "v_mfma_f32_16x16x32_f16 a[68:71], v[18:21], v[138:141], a[68:71]\n" \
  "v_mfma_f32_16x16x32_f16 a[68:71], v[14:17], v[106:109], a[68:71]\n" \
  "s_waitcnt lgkmcnt(14)\n" \
  "v_mfma_f32_16x16x32_f16 a[72:75], v[14:17], v[110:113], a[72:75]\n" \
  "buffer_load_dwordx4 v[22:25], %[bxr], s[40:43], s73 offen offset:2048 sc0 sc1\n" \
  "v_mfma_f32_16x16x32_f16 a[72:75], v[18:21], v[142:145], a[72:75]\n" \
  "s_waitcnt vmcnt(19)\n" \
  "v_mfma_f32_16x16x32_f16 a[104:107], v[58:61], v[142:145], a[104:107]\n" \
  "s_waitcnt lgkmcnt(12)\n" \
  "v_cvt_pk_f16_f32 v214, v182, v186\n" \
  "v_cvt_pk_f16_f32 v218, v183, v187\n" \
  "v_mfma_f32_16x16x32_f16 a[104:107], v[54:57], v[110:113], a[104:107]\n" \
  "s_waitcnt vmcnt(17)\n" \
  "v_cvt_pk_f16_f32 v222, v184, v188\n" \
  "v_cvt_pk_f16_f32 v226, v185, v189\n" \
  "v_mfma_f32_16x16x32_f16 a[108:111], v[54:57], v[114:117], a[108:111]\n" \
  "v_cvt_pk_f16_f32 v215, v190, v194\n" \
  "v_cvt_pk_f16_f32 v219, v191, v195\n" \
  "v_mfma_f32_16x16x32_f16 a[108:111], v[58:61], v[146:149], a[108:111]\n" \
  "s_waitcnt vmcnt(10)\n" \
  "v_cvt_pk_f16_f32 v223, v192, v196\n" \
  "v_cvt_pk_f16_f32 v227, v193, v197\n" \
  "v_mfma_f32_16x16x32_f16 a[76:79], v[18:21], v[146:149], a[76:79]\n" \
  "buffer_load_dwordx4 v[26:29], %[bxr], s[40:43], s73 offen offset:3072 sc0 sc1\n" \
  "v_mfma_f32_16x16x32_f16 a[76:79], v[14:17], v[114:117], a[76:79]\n" \
  "s_waitcnt lgkmcnt(10)\n" \
  "v_readlane_b32 s74, %[vtabA], 4\n" \
  "v_min3_u32 v248, v238, v239, v240\n" \
  "v_mfma_f32_16x16x32_f16 a[80:83], v[14:17], v[118:121], a[80:83]\n" \
  "s_bitcmp1_b32 s34, 4\n" \
  "s_cbranch_scc0 L_s49_%=\n" \
  "L_b49_%=:\n" \
  "v_min_u32 v248, v248, v241\n" \
  "v_mfma_f32_16x16x32_f16 a[80:83], v[18:21], v[150:153], a[80:83]\n" \
  "s_add_u32 s72, s74, s68\n" \
  "v_cmp_le_u32 s[34:35], s76, v248\n" \
  "v_mfma_f32_16x16x32_f16 a[112:115], v[58:61], v[150:153], a[112:115]\n" \
  "buffer_load_dwordx4 v[78:81], %[bxr], s[40:43], s72 offen offset:0 sc0 sc1\n" \
  "v_mfma_f32_16x16x32_f16 a[112:115], v[54:57], v[118:121], a[112:115]\n" \
  "s_waitcnt lgkmcnt(8)\n" \
  "v_mfma_f32_16x16x32_f16 a[116:119], v[54:57], v[122:125], a[116:119]\n" \
  "v_mfma_f32_16x16x32_f16 a[116:119], v[58:61], v[154:157], a[116:119]\n" \
  "v_mfma_f32_16x16x32_f16 a[84:87], v[18:21], v[154:157], a[84:87]\n" \
  "buffer_load_dwordx4 v[82:85], %[bxr], s[40:43], s72 offen offset:1024 sc0 sc1\n" \
  "v_mfma_f32_16x16x32_f16 a[84:87], v[14:17], v[122:125], a[84:87]\n" \
  "s_waitcnt lgkmcnt(5)\n" \
  "v_mfma_f32_16x16x32_f16 a[88:91], v[14:17], v[126:129], a[88:91]\n" \
  "v_mfma_f32_16x16x32_f16 a[88:91], v[18:21], v[158:161], a[88:91]\n" \
  "s_barrier\n" \
  "v_mfma_f32_16x16x32_f16 a[120:123], v[58:61], v[158:161], a[120:123]\n" \
  "buffer_load_dwordx4 v[174:177], %[voffW], s[36:39], s62 offen nt\n" \
  "v_mfma_f32_16x16x32_f16 a[120:123], v[54:57], v[126:129], a[120:123]\n" \
  "s_waitcnt lgkmcnt(2)\n" \
  "v_mfma_f32_16x16x32_f16 a[124:127], v[54:57], v[130:133], a[124:127]\n" \
  "buffer_load_dwordx4 v[178:181], %[voffW], s[36:39], s63 offen nt\n" \
  "v_mfma_f32_16x16x32_f16 a[124:127], v[58:61], v[162:165], a[124:127]\n" \
  "v_mfma_f32_16x16x32_f16 a[92:95], v[18:21], v[162:165], a[92:95]\n" \
  "buffer_load_dwordx4 v[182:185], %[voffW], s[36:39], s64 offen nt\n" \
  "v_mfma_f32_16x16x32_f16 a[92:95], v[14:17], v[130:133], a[92:95]\n" \
  "v_mfma_f32_16x16x32_f16 a[192:195], v[30:33], v[102:105], a[192:195]\n" \
  "buffer_load_dwordx4 v[186:189], %[voffW], s[36:39], s65 offen nt\n" \
  "v_mfma_f32_16x16x32_f16 a[192:195], v[34:37], v[134:137], a[192:195]\n" \
  "s_memrealtime s[58:59]\n" \
  "v_mfma_f32_16x16x32_f16 a[224:227], v[74:77], v[134:137], a[224:227]\n" \
  "buffer_load_dwordx4 v[190:193], %[voffW], s[36:39], s66 offen nt\n" \
  "v_mfma_f32_16x16x32_f16 a[224:227], v[70:73], v[102:105], a[224:227]\n" \
  "v_mfma_f32_16x16x32_f16 a[228:231], v[70:73], v[106:109], a[228:231]\n" \
  "buffer_load_dwordx4 v[194:197], %[voffW], s[36:39], s67 offen nt\n" \
  "v_mfma_f32_16x16x32_f16 a[228:231], v[74:77], v[138:141], a[228:231]\n" \
  "s_cmp_eq_u32 s77, 5\n" \
  "s_cbranch_scc1 L_s82_%=\n" \
  "L_b82_%=:\n" \
  "v_mfma_f32_16x16x32_f16 a[196:199], v[34:37], v[138:141], a[196:199]\n" \
  "v_mfma_f32_16x16x32_f16 a[196:199], v[30:33], v[106:109], a[196:199]\n" \
  "v_mfma_f32_16x16x32_f16 a[200:203], v[30:33], v[110:113], a[200:203]\n" \
  "v_mfma_f32_16x16x32_f16 a[200:203], v[34:37], v[142:145], a[200:203]\n" \
  "s_waitcnt lgkmcnt(0)\n" \
  "v_mfma_f32_16x16x32_f16 a[232:235], v[74:77], v[142:145], a[232:235]\n" \
  "s_sub_u32 s82, s81, s58\n" \
  "s_cmp_gt_i32 s82, 0\n" \
  "s_cbranch_scc1 L_s25_%=\n" \
  "L_b25_%=:\n" \
  "v_mfma_f32_16x16x32_f16 a[232:235], v[70:73], v[110:113], a[232:235]\n" \
  "s_max_i32 s82, s82, -150\n" \
  "s_add_u32 s81, s58, s82\n" \
  "s_add_u32 s81, s81, s87\n" \
  "v_mfma_f32_16x16x32_f16 a[236:239], v[70:73], v[114:117], a[236:239]\n" \
  "v_mfma_f32_16x16x32_f16 a[236:239], v[74:77], v[146:149], a[236:239]\n" \
  "v_mfma_f32_16x16x32_f16 a[204:207], v[34:37], v[146:149], a[204:207]\n" \
  "v_mfma_f32_16x16x32_f16 a[204:207], v[30:33], v[114:117], a[204:207]\n" \
  "v_mfma_f32_16x16x32_f16 a[208:211], v[30:33], v[118:121], a[208:211]\n" \
  "v_mfma_f32_16x16x32_f16 a[208:211], v[34:37], v[150:153], a[208:211]\n" \
  "v_mfma_f32_16x16x32_f16 a[240:243], v[74:77], v[150:153], a[240:243]\n" \
  "v_mfma_f32_16x16x32_f16 a[240:243], v[70:73], v[118:121], a[240:243]\n" \
  "v_mfma_f32_16x16x32_f16 a[244:247], v[70:73], v[122:125], a[244:247]\n" \
  "v_mfma_f32_16x16x32_f16 a[244:247], v[74:77], v[154:157], a[244:247]\n" \
  "s_waitcnt vmcnt(17)\n" \
  "v_mfma_f32_16x16x32_f16 a[212:215], v[34:37], v[154:157], a[212:215]\n" \
  "v_cvt_pk_f16_f32 v216, v198, v202\n" \
  "v_cvt_pk_f16_f32 v220, v199, v203\n" \
  "v_mfma_f32_16x16x32_f16 a[212:215], v[30:33], v[122:125], a[212:215]\n" \
  "s_waitcnt vmcnt(15)\n" \
  "v_cvt_pk_f16_f32 v224, v200, v204\n" \
  "v_cvt_pk_f16_f32 v228, v201, v205\n" \
  "v_mfma_f32_16x16x32_f16 a[216:219], v[30:33], v[126:129], a[216:219]\n" \
  "v_cvt_pk_f16_f32 v217, v206, v210\n" \
  "v_cvt_pk_f16_f32 v221, v207, v211\n" \
  "v_mfma_f32_16x16x32_f16 a[216:219], v[34:37], v[158:161], a[216:219]\n" \
  "v_cvt_pk_f16_f32 v225, v208, v212\n" \
  "v_cvt_pk_f16_f32 v229, v209, v213\n" \
  "v_mfma_f32_16x16x32_f16 a[248:251], v[74:77], v[158:161], a[248:251]\n" \
  "ds_write_b128 v243, v[214:217] offset:16384\n" \
  "v_mfma_f32_16x16x32_f16 a[248:251], v[70:73], v[126:129], a[248:251]\n" \
  "buffer_load_dwordx4 v[38:41], %[bxr], s[40:43], s72 offen offset:2048 sc0 sc1\n" \
  "v_mfma_f32_16x16x32_f16 a[252:255], v[70:73], v[130:133], a[252:255]\n" \
  "s_add_u32 s73, s74, s69\n" \
  "v_readlane_b32 s75, %[vtabW], 7\n" \
  "v_mfma_f32_16x16x32_f16 a[252:255], v[74:77], v[162:165], a[252:255]\n" \
  "s_waitcnt vmcnt(13)\n" \
  "v_mfma_f32_16x16x32_f16 a[220:223], v[34:37], v[162:165], a[220:223]\n" \
  "ds_write_b128 v243, v[218:221] offset:17408\n" \
  "v_mfma_f32_16x16x32_f16 a[220:223], v[30:33], v[130:133], a[220:223]\n" \
  "buffer_load_dwordx4 v[42:45], %[bxr], s[40:43], s72 offen offset:3072 sc0 sc1\n" \
  "v_mfma_f32_16x16x32_f16 a[0:3], v[46:49], v[102:105], a[0:3]\n" \
  "s_add_u32 s60, s75, 0x0\n" \
  "s_add_u32 s61, s75, 0x10000\n" \
  "s_add_u32 s62, s75, 0x20000\n" \
  "v_mfma_f32_16x16x32_f16 a[0:3], v[50:53], v[134:137], a[0:3]\n" \
  "s_add_u32 s63, s75, 0x30000\n" \
  "s_add_u32 s64, s75, 0x100000\n" \
  "s_add_u32 s65, s75, 0x110000\n" \
  "v_mfma_f32_16x16x32_f16 a[32:35], v[90:93], v[134:137], a[32:35]\n" \
  "ds_write_b128 v243, v[222:225] offset:18432\n" \
  "v_mfma_f32_16x16x32_f16 a[32:35], v[86:89], v[102:105], a[32:35]\n" \
  "buffer_load_dwordx4 v[94:97], %[bxr], s[40:43], s73 offen offset:0 sc0 sc1\n" \
  "v_mfma_f32_16x16x32_f16 a[36:39], v[86:89], v[106:109], a[36:39]\n" \
  "v_mfma_f32_16x16x32_f16 a[36:39], v[90:93], v[138:141], a[36:39]\n" \
  "v_mfma_f32_16x16x32_f16 a[4:7], v[50:53], v[138:141], a[4:7]\n" \
  "ds_write_b128 v243, v[226:229] offset:19456\n" \
  "v_mfma_f32_16x16x32_f16 a[4:7], v[46:49], v[106:109], a[4:7]\n" \
  "buffer_load_dwordx4 v[98:101], %[bxr], s[40:43], s73 offen offset:1024 sc0 sc1\n" \
  "v_mfma_f32_16x16x32_f16 a[8:11], v[46:49], v[110:113], a[8:11]\n" \
  "v_mfma_f32_16x16x32_f16 a[8:11], v[50:53], v[142:145], a[8:11]\n" \
  "v_mfma_f32_16x16x32_f16 a[40:43], v[90:93], v[142:145], a[40:43]\n" \
  "v_mfma_f32_16x16x32_f16 a[40:43], v[86:89], v[110:113], a[40:43]\n" \
  "v_cvt_pk_f16_f32 v214, v166, v170\n" \
  "v_cvt_pk_f16_f32 v218, v167, v171\n" \
  "v_mfma_f32_16x16x32_f16 a[44:47], v[86:89], v[114:117], a[44:47]\n" \
  "v_cvt_pk_f16_f32 v222, v168, v172\n" \
  "v_cvt_pk_f16_f32 v226, v169, v173\n" \
  "v_mfma_f32_16x16x32_f16 a[44:47], v[90:93], v[146:149], a[44:47]\n" \
  "buffer_load_dwordx4 v[54:57], %[bxr], s[40:43], s73 offen offset:2048 sc0 sc1\n" \
  "v_mfma_f32_16x16x32_f16 a[12:15], v[50:53], v[146:149], a[12:15]\n" \
  "s_add_u32 s72, s74, s70\n" \
  "v_mfma_f32_16x16x32_f16 a[12:15], v[46:49], v[114:117], a[12:15]\n" \
  "v_mfma_f32_16x16x32_f16 a[16:19], v[46:49], v[118:121], a[16:19]\n" \
  "v_mfma_f32_16x16x32_f16 a[16:19], v[50:53], v[150:153], a[16:19]\n" \
  "buffer_load_dwordx4 v[58:61], %[bxr], s[40:43], s73 offen offset:3072 sc0 sc1\n" \
  "v_mfma_f32_16x16x32_f16 a[48:51], v[90:93], v[150:153], a[48:51]\n" \
  "v_mfma_f32_16x16x32_f16 a[48:51], v[86:89], v[118:121], a[48:51]\n" \
  "v_mfma_f32_16x16x32_f16 a[52:55], v[86:89], v[122:125], a[52:55]\n" \
  "v_mfma_f32_16x16x32_f16 a[52:55], v[90:93], v[154:157], a[52:55]\n" \
  "buffer_load_dwordx4 v[18:21], %[bxr], s[40:43], s72 offen offset:0 sc0 sc1\n" \
  "v_mfma_f32_16x16x32_f16 a[20:23], v[50:53], v[154:157], a[20:23]\n" \
  "v_mfma_f32_16x16x32_f16 a[20:23], v[46:49], v[122:125], a[20:23]\n" \
  "s_andn2_b32 s79, 0xff, s34\n" \
  "s_cbranch_scc1 L_pl4_%=\n" \
  "L_pb4_%=:\n" \
  "v_mfma_f32_16x16x32_f16 a[24:27], v[46:49], v[126:129], a[24:27]\n" \
  "v_mfma_f32_16x16x32_f16 a[24:27], v[50:53], v[158:161], a[24:27]\n" \
  "s_barrier\n" \
  "v_mfma_f32_16x16x32_f16 a[56:59], v[90:93], v[158:161], a[56:59]\n" \
  "buffer_load_dwordx4 v[198:201], %[voffW], s[36:39], s60 offen nt\n" \
  "v_mfma_f32_16x16x32_f16 a[56:59], v[86:89], v[126:129], a[56:59]\n" \
  "v_mfma_f32_16x16x32_f16 a[60:63], v[86:89], v[130:133], a[60:63]\n" \
  "buffer_load_dwordx4 v[202:205], %[voffW], s[36:39], s61 offen nt\n" \
  "v_mfma_f32_16x16x32_f16 a[60:63], v[90:93], v[162:165], a[60:63]\n" \
  "v_mfma_f32_16x16x32_f16 a[28:31], v[50:53], v[162:165], a[28:31]\n" \
  "buffer_load_dwordx4 v[206:209], %[voffW], s[36:39], s62 offen nt\n" \
  "v_mfma_f32_16x16x32_f16 a[28:31], v[46:49], v[130:133], a[28:31]\n" \
  "s_waitcnt vmcnt(18)\n" \
  "v_mfma_f32_16x16x32_f16 a[128:131], v[62:65], v[102:105], a[128:131]\n" \
  "buffer_load_dwordx4 v[210:213], %[voffW], s[36:39], s63 offen nt\n" \
  "v_mfma_f32_16x16x32_f16 a[128:131], v[66:69], v[134:137], a[128:131]\n" \
  "v_mfma_f32_16x16x32_f16 a[160:163], v[26:29], v[134:137], a[160:163]\n" \
  "ds_read_b128 v[134:137], v246 offset:8192\n" \
  "v_mfma_f32_16x16x32_f16 a[160:163], v[22:25], v[102:105], a[160:163]\n" \
  "ds_read_b128 v[102:105], v246 offset:0\n" \
  "v_mfma_f32_16x16x32_f16 a[164:167], v[22:25], v[106:109], a[164:167]\n" \
  "buffer_load_dwordx4 v[166:169], %[voffW], s[36:39], s64 offen nt\n" \
  "v_mfma_f32_16x16x32_f16 a[164:167], v[26:29], v[138:141], a[164:167]\n" \
  "v_mfma_f32_16x16x32_f16 a[132:135], v[66:69], v[138:141], a[132:135]\n" \
  "ds_read_b128 v[138:141], v246 offset:9216\n" \
  "v_mfma_f32_16x16x32_f16 a[132:135], v[62:65], v[106:109], a[132:135]\n" \
  "ds_read_b128 v[106:109], v246 offset:1024\n" \
  "v_mfma_f32_16x16x32_f16 a[136:139], v[62:65], v[110:113], a[136:139]\n" \
  "buffer_load_dwordx4 v[170:173], %[voffW], s[36:39], s65 offen nt\n" \
  "v_mfma_f32_16x16x32_f16 a[136:139], v[66:69], v[142:145], a[136:139]\n" \
  "s_cmp_eq_u32 s77, 6\n" \
  "s_cbranch_scc1 L_s83_%=\n" \
  "L_b83_%=:\n" \
  "v_mfma_f32_16x16x32_f16 a[168:171], v[26:29], v[142:145], a[168:171]\n" \
  "ds_read_b128 v[142:145], v246 offset:10240\n" \
  "v_mfma_f32_16x16x32_f16 a[168:171], v[22:25], v[110:113], a[168:171]\n" \
  "ds_read_b128 v[110:113], v246 offset:2048\n" \
  "v_mfma_f32_16x16x32_f16 a[172:175], v[22:25], v[114:117], a[172:175]\n" \
  "v_mfma_f32_16x16x32_f16 a[172:175], v[26:29], v[146:149], a[172:175]\n" \
  "v_mfma_f32_16x16x32_f16 a[140:143], v[66:69], v[146:149], a[140:143]\n" \
  "ds_read_b128 v[146:149], v246 offset:11264\n" \
  "v_mfma_f32_16x16x32_f16 a[140:143], v[62:65], v[114:117], a[140:143]\n" \
  "ds_read_b128 v[114:117], v246 offset:3072\n" \
  "v_mfma_f32_16x16x32_f16 a[144:147], v[62:65], v[118:121], a[144:147]\n" \
  "v_mfma_f32_16x16x32_f16 a[144:147], v[66:69], v[150:153], a[144:147]\n" \
  "v_mfma_f32_16x16x32_f16 a[176:179], v[26:29], v[150:153], a[176:179]\n" \
  "ds_read_b128 v[150:153], v246 offset:12288\n" \
  "v_mfma_f32_16x16x32_f16 a[176:179], v[22:25], v[118:121], a[176:179]\n" \
  "ds_read_b128 v[118:121], v246 offset:4096\n" \
  "v_mfma_f32_16x16x32_f16 a[180:183], v[22:25], v[122:125], a[180:183]\n" \
  "v_mfma_f32_16x16x32_f16 a[180:183], v[26:29], v[154:157], a[180:183]\n" \
  "v_mfma_f32_16x16x32_f16 a[148:151], v[66:69], v[154:157], a[148:151]\n" \
  "ds_read_b128 v[154:157], v246 offset:13312\n" \
  "v_mfma_f32_16x16x32_f16 a[148:151], v[62:65], v[122:125], a[148:151]\n" \
  "ds_read_b128 v[122:125], v246 offset:5120\n" \
  "v_mfma_f32_16x16x32_f16 a[152:155], v[62:65], v[126:129], a[152:155]\n" \
  "buffer_load_dwordx4 v[14:17], %[bxr], s[40:43], s72 offen offset:1024 sc0 sc1\n" \
  "v_mfma_f32_16x16x32_f16 a[152:155], v[66:69], v[158:161], a[152:155]\n" \
  "v_mfma_f32_16x16x32_f16 a[184:187], v[26:29], v[158:161], a[184:187]\n" \
  "ds_read_b128 v[158:161], v246 offset:14336\n" \
  "v_mfma_f32_16x16x32_f16 a[184:187], v[22:25], v[126:129], a[184:187]\n" \
  "ds_read_b128 v[126:129], v246 offset:6144\n" \
  "v_mfma_f32_16x16x32_f16 a[188:191], v[22:25], v[130:133], a[188:191]\n" \
  "buffer_load_dwordx4 v[70:73], %[bxr], s[40:43], s72 offen offset:2048 sc0 sc1\n" \
  "v_mfma_f32_16x16x32_f16 a[188:191], v[26:29], v[162:165], a[188:191]\n" \
  "s_add_u32 s66, s75, 0x120000\n" \
  "s_add_u32 s67, s75, 0x130000\n" \
  "s_add_u32 s73, s74, s71\n" \
  "v_mfma_f32_16x16x32_f16 a[156:159], v[66:69], v[162:165], a[156:159]\n" \
  "ds_read_b128 v[162:165], v246 offset:15360\n" \
  "v_readlane_b32 s75, %[vtabW], 8\n" \
  "v_mfma_f32_16x16x32_f16 a[156:159], v[62:65], v[130:133], a[156:159]\n" \
  "s_waitcnt vmcnt(13)\n" \
  "ds_read_b128 v[130:133], v246 offset:7168\n" \
  "s_waitcnt lgkmcnt(14)\n" \
    \
  "v_mfma_f32_16x16x32_f16 a[64:67], v[78:81], v[102:105], a[64:67]\n" \
  "buffer_load_dwordx4 v[74:77], %[bxr], s[40:43], s72 offen offset:3072 sc0 sc1\n" \
  "v_mfma_f32_16x16x32_f16 a[64:67], v[82:85], v[134:137], a[64:67]\n" \
  "s_add_u32 s60, s75, 0x0\n" \
  "s_add_u32 s61, s75, 0x10000\n" \
  "s_add_u32 s62, s75, 0x20000\n" \
  "v_mfma_f32_16x16x32_f16 a[96:99], v[42:45], v[134:137], a[96:99]\n" \
  "s_add_u32 s63, s75, 0x30000\n" \
  "v_mfma_f32_16x16x32_f16 a[96:99], v[38:41], v[102:105], a[96:99]\n" \
  "s_waitcnt lgkmcnt(12)\n" \
  "v_mfma_f32_16x16x32_f16 a[100:103], v[38:41], v[106:109], a[100:103]\n" \
  "buffer_load_dwordx4 v[34:37], %[bxr], s[40:43], s73 offen offset:0 sc0 sc1\n" \
  "v_mfma_f32_16x16x32_f16 a[100:103], v[42:45], v[138:141], a[100:103]\n" \
  "v_mfma_f32_16x16x32_f16 a[68:71], v[82:85], v[138:141], a[68:71]\n" \
  "v_mfma_f32_16x16x32_f16 a[68:71], v[78:81], v[106:109], a[68:71]\n" \
  "s_waitcnt lgkmcnt(10)\n" \
  "v_mfma_f32_16x16x32_f16 a[72:75], v[78:81], v[110:113], a[72:75]\n" \
  "buffer_load_dwordx4 v[30:33], %[bxr], s[40:43], s73 offen offset:1024 sc0 sc1\n" \
  "v_mfma_f32_16x16x32_f16 a[72:75], v[82:85], v[142:145], a[72:75]\n" \
  "v_mfma_f32_16x16x32_f16 a[104:107], v[42:45], v[142:145], a[104:107]\n" \
  "v_mfma_f32_16x16x32_f16 a[104:107], v[38:41], v[110:113], a[104:107]\n" \
  "s_waitcnt lgkmcnt(8)\n" \
  "v_cvt_pk_f16_f32 v215, v174, v178\n" \
  "v_cvt_pk_f16_f32 v219, v175, v179\n" \
  "v_mfma_f32_16x16x32_f16 a[108:111], v[38:41], v[114:117], a[108:111]\n" \
  "v_cvt_pk_f16_f32 v223, v176, v180\n" \
  "v_cvt_pk_f16_f32 v227, v177, v181\n" \
  "v_mfma_f32_16x16x32_f16 a[108:111], v[42:45], v[146:149], a[108:111]\n" \
  "s_waitcnt vmcnt(11)\n" \
  "v_cvt_pk_f16_f32 v216, v182, v186\n" \
  "v_cvt_pk_f16_f32 v220, v183, v187\n" \
  "v_mfma_f32_16x16x32_f16 a[76:79], v[82:85], v[146:149], a[76:79]\n" \
  "v_cvt_pk_f16_f32 v224, v184, v188\n" \
  "v_cvt_pk_f16_f32 v228, v185, v189\n" \
  "v_mfma_f32_16x16x32_f16 a[76:79], v[78:81], v[114:117], a[76:79]\n" \
  "s_waitcnt lgkmcnt(6)\n" \
  "v_cvt_pk_f16_f32 v217, v190, v194\n" \
  "v_cvt_pk_f16_f32 v221, v191, v195\n" \
  "v_mfma_f32_16x16x32_f16 a[80:83], v[78:81], v[118:121], a[80:83]\n" \
  "v_cvt_pk_f16_f32 v225, v192, v196\n" \
  "v_cvt_pk_f16_f32 v229, v193, v197\n" \
  "v_mfma_f32_16x16x32_f16 a[80:83], v[82:85], v[150:153], a[80:83]\n" \
  "ds_write_b128 v243, v[214:217] offset:32768\n" \
  "v_mfma_f32_16x16x32_f16 a[112:115], v[42:45], v[150:153], a[112:115]\n" \
  "buffer_load_dwordx4 v[86:89], %[bxr], s[40:43], s73 offen offset:2048 sc0 sc1\n" \
  "v_mfma_f32_16x16x32_f16 a[112:115], v[38:41], v[118:121], a[112:115]\n" \
  "s_waitcnt lgkmcnt(5)\n" \
  "v_min3_u32 v248, v238, v239, v240\n" \
  "v_min_u32 v248, v248, v241\n" \
  "v_mfma_f32_16x16x32_f16 a[116:119], v[38:41], v[122:125], a[116:119]\n" \
  "v_cmp_le_u32 s[34:35], s76, v248\n" \
  "v_mfma_f32_16x16x32_f16 a[116:119], v[42:45], v[154:157], a[116:119]\n" \
  "ds_write_b128 v243, v[218:221] offset:33792\n" \
  "v_mfma_f32_16x16x32_f16 a[84:87], v[82:85], v[154:157], a[84:87]\n" \
  "buffer_load_dwordx4 v[90:93], %[bxr], s[40:43], s73 offen offset:3072 sc0 sc1\n" \
  "v_mfma_f32_16x16x32_f16 a[84:87], v[78:81], v[122:125], a[84:87]\n" \
  "s_waitcnt lgkmcnt(4)\n" \
  "v_mfma_f32_16x16x32_f16 a[88:91], v[78:81], v[126:129], a[88:91]\n" \
  "s_barrier\n" \
  "v_mfma_f32_16x16x32_f16 a[88:91], v[82:85], v[158:161], a[88:91]\n" \
  "ds_write_b128 v243, v[222:225] offset:34816\n" \
  "v_mfma_f32_16x16x32_f16 a[120:123], v[42:45], v[158:161], a[120:123]\n" \
  "buffer_load_dwordx4 v[174:177], %[voffW], s[36:39], s66 offen nt\n" \
  "v_mfma_f32_16x16x32_f16 a[120:123], v[38:41], v[126:129], a[120:123]\n" \
  "s_waitcnt lgkmcnt(3)\n" \
  "v_mfma_f32_16x16x32_f16 a[124:127], v[38:41], v[130:133], a[124:127]\n" \
  "buffer_load_dwordx4 v[178:181], %[voffW], s[36:39], s67 offen nt\n" \
  "v_mfma_f32_16x16x32_f16 a[124:127], v[42:45], v[162:165], a[124:127]\n" \
  "ds_write_b128 v243, v[226:229] offset:35840\n" \
  "v_mfma_f32_16x16x32_f16 a[92:95], v[82:85], v[162:165], a[92:95]\n" \
  "buffer_load_dwordx4 v[182:185], %[voffW], s[36:39], s60 offen nt\n" \
  "v_mfma_f32_16x16x32_f16 a[92:95], v[78:81], v[130:133], a[92:95]\n" \
  "v_mfma_f32_16x16x32_f16 a[192:195], v[94:97], v[102:105], a[192:195]\n" \
  "buffer_load_dwordx4 v[186:189], %[voffW], s[36:39], s61 offen nt\n" \
  "v_mfma_f32_16x16x32_f16 a[192:195], v[98:101], v[134:137], a[192:195]\n" \
  "s_memrealtime s[58:59]\n" \
  "v_mfma_f32_16x16x32_f16 a[224:227], v[58:61], v[134:137], a[224:227]\n" \
  "buffer_load_dwordx4 v[190:193], %[voffW], s[36:39], s62 offen nt\n" \
  "v_mfma_f32_16x16x32_f16 a[224:227], v[54:57], v[102:105], a[224:227]\n" \
  "v_mfma_f32_16x16x32_f16 a[228:231], v[54:57], v[106:109], a[228:231]\n" \
  "buffer_load_dwordx4 v[194:197], %[voffW], s[36:39], s63 offen nt\n" \
  "v_mfma_f32_16x16x32_f16 a[228:231], v[58:61], v[138:141], a[228:231]\n" \
  "s_cmp_eq_u32 s77, 7\n" \
  "s_cbranch_scc1 L_s84_%=\n" \
  "L_b84_%=:\n" \
  "v_mfma_f32_16x16x32_f16 a[196:199], v[98:101], v[138:141], a[196:199]\n" \
  "v_mfma_f32_16x16x32_f16 a[196:199], v[94:97], v[106:109], a[196:199]\n" \
  "v_mfma_f32_16x16x32_f16 a[200:203], v[94:97], v[110:113], a[200:203]\n" \
  "v_mfma_f32_16x16x32_f16 a[200:203], v[98:101], v[142:145], a[200:203]\n" \
  "s_waitcnt lgkmcnt(0)\n" \
  "v_mfma_f32_16x16x32_f16 a[232:235], v[58:61], v[142:145], a[232:235]\n" \
  "s_sub_u32 s82, s81, s58\n" \
  "s_branch L_s26_%=\n" \
  "L_b26_%=:\n" \
  "v_mfma_f32_16x16x32_f16 a[232:235], v[54:57], v[110:113], a[232:235]\n" \
  "s_max_i32 s82, s82, -150\n" \
  "s_add_u32 s81, s58, s82\n" \
  "s_add_u32 s81, s81, s87\n" \
  "v_mfma_f32_16x16x32_f16 a[236:239], v[54:57], v[114:117], a[236:239]\n" \
  "v_mfma_f32_16x16x32_f16 a[236:239], v[58:61], v[146:149], a[236:239]\n" \
  "v_mfma_f32_16x16x32_f16 a[204:207], v[98:101], v[146:149], a[204:207]\n" \
  "v_readlane_b32 s74, %[vtabA], 5\n" \
  "v_mfma_f32_16x16x32_f16 a[204:207], v[94:97], v[114:117], a[204:207]\n" \
  "s_bitcmp1_b32 s34, 5\n" \
  "s_cbranch_scc0 L_s50_%=\n" \
  "L_b50_%=:\n" \
  "v_mfma_f32_16x16x32_f16 a[208:211], v[94:97], v[118:121], a[208:211]\n" \
  "s_add_u32 s72, s74, s68\n" \
  "v_mfma_f32_16x16x32_f16 a[208:211], v[98:101], v[150:153], a[208:211]\n" \
  "v_mfma_f32_16x16x32_f16 a[240:243], v[58:61], v[150:153], a[240:243]\n" \
  "v_mfma_f32_16x16x32_f16 a[240:243], v[54:57], v[118:121], a[240:243]\n" \
  "v_mfma_f32_16x16x32_f16 a[244:247], v[54:57], v[122:125], a[244:247]\n" \
  "v_mfma_f32_16x16x32_f16 a[244:247], v[58:61], v[154:157], a[244:247]\n" \
  "v_mfma_f32_16x16x32_f16 a[212:215], v[98:101], v[154:157], a[212:215]\n" \
  "buffer_load_dwordx4 v[50:53], %[bxr], s[40:43], s72 offen offset:0 sc0 sc1\n" \
  "v_mfma_f32_16x16x32_f16 a[212:215], v[94:97], v[122:125], a[212:215]\n" \
  "v_mfma_f32_16x16x32_f16 a[216:219], v[94:97], v[126:129], a[216:219]\n" \
  "v_mfma_f32_16x16x32_f16 a[216:219], v[98:101], v[158:161], a[216:219]\n" \
  "v_mfma_f32_16x16x32_f16 a[248:251], v[58:61], v[158:161], a[248:251]\n" \
  "buffer_load_dwordx4 v[46:49], %[bxr], s[40:43], s72 offen offset:1024 sc0 sc1\n" \
  "v_mfma_f32_16x16x32_f16 a[248:251], v[54:57], v[126:129], a[248:251]\n" \
  "v_mfma_f32_16x16x32_f16 a[252:255], v[54:57], v[130:133], a[252:255]\n" \
  "s_add_u32 s64, s75, 0x100000\n" \
  "s_add_u32 s65, s75, 0x110000\n" \
  "s_add_u32 s66, s75, 0x120000\n" \
  "v_mfma_f32_16x16x32_f16 a[252:255], v[58:61], v[162:165], a[252:255]\n" \
  "s_add_u32 s67, s75, 0x130000\n" \
  "v_mfma_f32_16x16x32_f16 a[220:223], v[98:101], v[162:165], a[220:223]\n" \
  "buffer_load_dwordx4 v[22:25], %[bxr], s[40:43], s72 offen offset:2048 sc0 sc1\n" \
  "v_mfma_f32_16x16x32_f16 a[220:223], v[94:97], v[130:133], a[220:223]\n" \
  "s_waitcnt vmcnt(13)\n" \
  "v_readlane_b32 s75, %[vtabW], 9\n" \
  "s_add_u32 s73, s74, s69\n" \
  "v_mfma_f32_16x16x32_f16 a[0:3], v[18:21], v[102:105], a[0:3]\n" \
  "v_mfma_f32_16x16x32_f16 a[0:3], v[14:17], v[134:137], a[0:3]\n" \
  "s_add_u32 s60, s75, 0x0\n" \
  "s_add_u32 s61, s75, 0x10000\n" \
  "v_mfma_f32_16x16x32_f16 a[32:35], v[74:77], v[134:137], a[32:35]\n" \
  "buffer_load_dwordx4 v[26:29], %[bxr], s[40:43], s72 offen offset:3072 sc0 sc1\n" \
  "v_mfma_f32_16x16x32_f16 a[32:35], v[70:73], v[102:105], a[32:35]\n" \
  "v_mfma_f32_16x16x32_f16 a[36:39], v[70:73], v[106:109], a[36:39]\n" \
  "v_mfma_f32_16x16x32_f16 a[36:39], v[74:77], v[138:141], a[36:39]\n" \
  "v_mfma_f32_16x16x32_f16 a[4:7], v[14:17], v[138:141], a[4:7]\n" \
  "buffer_load_dwordx4 v[66:69], %[bxr], s[40:43], s73 offen offset:0 sc0 sc1\n" \
  "v_mfma_f32_16x16x32_f16 a[4:7], v[18:21], v[106:109], a[4:7]\n" \
  "v_mfma_f32_16x16x32_f16 a[8:11], v[18:21], v[110:113], a[8:11]\n" \
  "v_mfma_f32_16x16x32_f16 a[8:11], v[14:17], v[142:145], a[8:11]\n" \
  "v_mfma_f32_16x16x32_f16 a[40:43], v[74:77], v[142:145], a[40:43]\n" \
  "v_mfma_f32_16x16x32_f16 a[40:43], v[70:73], v[110:113], a[40:43]\n" \
  "v_cvt_pk_f16_f32 v214, v198, v202\n" \
  "v_cvt_pk_f16_f32 v218, v199, v203\n" \
  "v_mfma_f32_16x16x32_f16 a[44:47], v[70:73], v[114:117], a[44:47]\n" \
  "v_cvt_pk_f16_f32 v222, v200, v204\n" \
  "v_cvt_pk_f16_f32 v226, v201, v205\n" \
  "v_mfma_f32_16x16x32_f16 a[44:47], v[74:77], v[146:149], a[44:47]\n" \
  "v_cvt_pk_f16_f32 v215, v206, v210\n" \
  "v_cvt_pk_f16_f32 v219, v207, v211\n" \
  "v_mfma_f32_16x16x32_f16 a[12:15], v[14:17], v[146:149], a[12:15]\n" \
  "v_cvt_pk_f16_f32 v223, v208, v212\n" \
  "v_cvt_pk_f16_f32 v227, v209, v213\n" \
  "v_mfma_f32_16x16x32_f16 a[12:15], v[18:21], v[114:117], a[12:15]\n" \
  "v_cvt_pk_f16_f32 v216, v166, v170\n" \
  "v_cvt_pk_f16_f32 v220, v167, v171\n" \
  "v_mfma_f32_16x16x32_f16 a[16:19], v[18:21], v[118:121], a[16:19]\n" \
  "v_cvt_pk_f16_f32 v224, v168, v172\n" \
  "v_cvt_pk_f16_f32 v228, v169, v173\n" \
  "v_mfma_f32_16x16x32_f16 a[16:19], v[14:17], v[150:153], a[16:19]\n" \
  "buffer_load_dwordx4 v[62:65], %[bxr], s[40:43], s73 offen offset:1024 sc0 sc1\n" \
  "v_mfma_f32_16x16x32_f16 a[48:51], v[74:77], v[150:153], a[48:51]\n" \
  "v_mfma_f32_16x16x32_f16 a[48:51], v[70:73], v[118:121], a[48:51]\n" \
  "v_mfma_f32_16x16x32_f16 a[52:55], v[70:73], v[122:125], a[52:55]\n" \
  "v_mfma_f32_16x16x32_f16 a[52:55], v[74:77], v[154:157], a[52:55]\n" \
  "buffer_load_dwordx4 v[38:41], %[bxr], s[40:43], s73 offen offset:2048 sc0 sc1\n" \
  "v_mfma_f32_16x16x32_f16 a[20:23], v[14:17], v[154:157], a[20:23]\n" \
  "v_mfma_f32_16x16x32_f16 a[20:23], v[18:21], v[122:125], a[20:23]\n" \
  "s_andn2_b32 s79, 0x1ff, s34\n" \
  "s_cbranch_scc1 L_pl5_%=\n" \
  "L_pb5_%=:\n" \
  "v_mfma_f32_16x16x32_f16 a[24:27], v[18:21], v[126:129], a[24:27]\n" \
  "s_barrier\n" \
  "v_mfma_f32_16x16x32_f16 a[24:27], v[14:17], v[158:161], a[24:27]\n" \
  "buffer_load_dwordx4 v[42:45], %[bxr], s[40:43], s73 offen offset:3072 sc0 sc1\n" \
  "v_mfma_f32_16x16x32_f16 a[56:59], v[74:77], v[158:161], a[56:59]\n" \
  "buffer_load_dwordx4 v[198:201], %[voffW], s[36:39], s64 offen nt\n" \
  "v_mfma_f32_16x16x32_f16 a[56:59], v[70:73], v[126:129], a[56:59]\n" \
  "v_mfma_f32_16x16x32_f16 a[60:63], v[70:73], v[130:133], a[60:63]\n" \
  "buffer_load_dwordx4 v[202:205], %[voffW], s[36:39], s65 offen nt\n" \
  "v_mfma_f32_16x16x32_f16 a[60:63], v[74:77], v[162:165], a[60:63]\n" \
  "v_mfma_f32_16x16x32_f16 a[28:31], v[14:17], v[162:165], a[28:31]\n" \
  "buffer_load_dwordx4 v[206:209], %[voffW], s[36:39], s66 offen nt\n" \
  "v_mfma_f32_16x16x32_f16 a[28:31], v[18:21], v[130:133], a[28:31]\n" \
  "s_waitcnt vmcnt(17)\n" \
  "v_mfma_f32_16x16x32_f16 a[128:131], v[34:37], v[102:105], a[128:131]\n" \
  "buffer_load_dwordx4 v[210:213], %[voffW], s[36:39], s67 offen nt\n" \
  "v_mfma_f32_16x16x32_f16 a[128:131], v[30:33], v[134:137], a[128:131]\n" \
  "v_mfma_f32_16x16x32_f16 a[160:163], v[90:93], v[134:137], a[160:163]\n" \
  "ds_read_b128 v[134:137], v246 offset:24576\n" \
  "v_mfma_f32_16x16x32_f16 a[160:163], v[86:89], v[102:105], a[160:163]\n" \
  "ds_read_b128 v[102:105], v246 offset:16384\n" \
  "v_mfma_f32_16x16x32_f16 a[164:167], v[86:89], v[106:109], a[164:167]\n" \
  "buffer_load_dwordx4 v[166:169], %[voffW], s[36:39], s60 offen nt\n" \
  "v_mfma_f32_16x16x32_f16 a[164:167], v[90:93], v[138:141], a[164:167]\n" \
  "v_mfma_f32_16x16x32_f16 a[132:135], v[30:33], v[138:141], a[132:135]\n" \
  "ds_read_b128 v[138:141], v246 offset:25600\n" \
  "v_mfma_f32_16x16x32_f16 a[132:135], v[34:37], v[106:109], a[132:135]\n" \
  "ds_read_b128 v[106:109], v246 offset:17408\n" \
  "v_mfma_f32_16x16x32_f16 a[136:139], v[34:37], v[110:113], a[136:139]\n" \
  "buffer_load_dwordx4 v[170:173], %[voffW], s[36:39], s61 offen nt\n" \
  "v_mfma_f32_16x16x32_f16 a[136:139], v[30:33], v[142:145], a[136:139]\n" \
  "s_cmp_eq_u32 s77, 8\n" \
  "s_cbranch_scc1 L_s85_%=\n" \
  "L_b85_%=:\n" \
  "v_mfma_f32_16x16x32_f16 a[168:171], v[90:93], v[142:145], a[168:171]\n" \
  "ds_read_b128 v[142:145], v246 offset:26624\n" \
  "v_mfma_f32_16x16x32_f16 a[168:171], v[86:89], v[110:113], a[168:171]\n" \
  "ds_read_b128 v[110:113], v246 offset:18432\n" \
  "v_mfma_f32_16x16x32_f16 a[172:175], v[86:89], v[114:117], a[172:175]\n" \
  "v_mfma_f32_16x16x32_f16 a[172:175], v[90:93], v[146:149], a[172:175]\n" \
  "v_mfma_f32_16x16x32_f16 a[140:143], v[30:33], v[146:149], a[140:143]\n" \
  "ds_read_b128 v[146:149], v246 offset:27648\n" \
  "v_mfma_f32_16x16x32_f16 a[140:143], v[34:37], v[114:117], a[140:143]\n" \
  "ds_read_b128 v[114:117], v246 offset:19456\n" \
  "v_mfma_f32_16x16x32_f16 a[144:147], v[34:37], v[118:121], a[144:147]\n" \
  "v_mfma_f32_16x16x32_f16 a[144:147], v[30:33], v[150:153], a[144:147]\n" \
  "v_mfma_f32_16x16x32_f16 a[176:179], v[90:93], v[150:153], a[176:179]\n" \
  "ds_read_b128 v[150:153], v246 offset:28672\n" \
  "v_mfma_f32_16x16x32_f16 a[176:179], v[86:89], v[118:121], a[176:179]\n" \
  "ds_read_b128 v[118:121], v246 offset:20480\n" \
  "v_mfma_f32_16x16x32_f16 a[180:183], v[86:89], v[122:125], a[180:183]\n" \
  "s_add_u32 s72, s74, s70\n" \
  "v_mfma_f32_16x16x32_f16 a[180:183], v[90:93], v[154:157], a[180:183]\n" \
  "s_waitcnt vmcnt(18)\n" \
  "v_mfma_f32_16x16x32_f16 a[148:151], v[30:33], v[154:157], a[148:151]\n" \
  "ds_read_b128 v[154:157], v246 offset:29696\n" \
  "v_cvt_pk_f16_f32 v217, v174, v178\n" \
  "v_mfma_f32_16x16x32_f16 a[148:151], v[34:37], v[122:125], a[148:151]\n" \
  "ds_read_b128 v[122:125], v246 offset:21504\n" \
  "v_cvt_pk_f16_f32 v221, v175, v179\n" \
  "v_mfma_f32_16x16x32_f16 a[152:155], v[34:37], v[126:129], a[152:155]\n" \
  "v_cvt_pk_f16_f32 v225, v176, v180\n" \
  "v_cvt_pk_f16_f32 v229, v177, v181\n" \
  "v_mfma_f32_16x16x32_f16 a[152:155], v[30:33], v[158:161], a[152:155]\n" \
  "ds_write_b128 v243, v[214:217] offset:49152\n" \
  "v_mfma_f32_16x16x32_f16 a[184:187], v[90:93], v[158:161], a[184:187]\n" \
  "ds_read_b128 v[158:161], v246 offset:30720\n" \
  "v_mfma_f32_16x16x32_f16 a[184:187], v[86:89], v[126:129], a[184:187]\n" \
  "ds_read_b128 v[126:129], v246 offset:22528\n" \
  "v_mfma_f32_16x16x32_f16 a[188:191], v[86:89], v[130:133], a[188:191]\n" \
  "buffer_load_dwordx4 v[82:85], %[bxr], s[40:43], s72 offen offset:0 sc0 sc1\n" \
  "v_mfma_f32_16x16x32_f16 a[188:191], v[90:93], v[162:165], a[188:191]\n" \
  "ds_write_b128 v243, v[218:221] offset:50176\n" \
  "v_mfma_f32_16x16x32_f16 a[156:159], v[30:33], v[162:165], a[156:159]\n" \
  "s_add_u32 s62, s75, 0x20000\n" \
  "ds_read_b128 v[162:165], v246 offset:31744\n" \
  "s_add_u32 s63, s75, 0x30000\n" \
  "v_mfma_f32_16x16x32_f16 a[156:159], v[34:37], v[130:133], a[156:159]\n" \
  "s_waitcnt vmcnt(11)\n" \
  "ds_read_b128 v[130:133], v246 offset:23552\n" \
  "s_waitcnt lgkmcnt(14)\n" \
    \
  "v_mfma_f32_16x16x32_f16 a[64:67], v[50:53], v[102:105], a[64:67]\n" \
  "buffer_load_dwordx4 v[78:81], %[bxr], s[40:43], s72 offen offset:1024 sc0 sc1\n" \
  "v_mfma_f32_16x16x32_f16 a[64:67], v[46:49], v[134:137], a[64:67]\n" \
  "ds_write_b128 v243, v[222:225] offset:51200\n" \
  "v_mfma_f32_16x16x32_f16 a[96:99], v[26:29], v[134:137], a[96:99]\n" \
  "s_add_u32 s64, s75, 0x100000\n" \
  "s_add_u32 s65, s75, 0x110000\n" \
  "s_add_u32 s66, s75, 0x120000\n" \
  "v_mfma_f32_16x16x32_f16 a[96:99], v[22:25], v[102:105], a[96:99]\n" \
  "s_waitcnt lgkmcnt(14)\n" \
  "s_add_u32 s67, s75, 0x130000\n" \
  "v_mfma_f32_16x16x32_f16 a[100:103], v[22:25], v[106:109], a[100:103]\n" \
  "buffer_load_dwordx4 v[54:57], %[bxr], s[40:43], s72 offen offset:2048 sc0 sc1\n" \
  "v_mfma_f32_16x16x32_f16 a[100:103], v[26:29], v[138:141], a[100:103]\n" \
  "ds_write_b128 v243, v[226:229] offset:52224\n" \
  "v_mfma_f32_16x16x32_f16 a[68:71], v[46:49], v[138:141], a[68:71]\n" \
  "s_add_u32 s73, s74, s71\n" \
  "v_mfma_f32_16x16x32_f16 a[68:71], v[50:53], v[106:109], a[68:71]\n" \
  "s_waitcnt lgkmcnt(14)\n" \
  "v_mfma_f32_16x16x32_f16 a[72:75], v[50:53], v[110:113], a[72:75]\n" \
  "buffer_load_dwordx4 v[58:61], %[bxr], s[40:43], s72 offen offset:3072 sc0 sc1\n" \
  "v_mfma_f32_16x16x32_f16 a[72:75], v[46:49], v[142:145], a[72:75]\n" \
  "v_mfma_f32_16x16x32_f16 a[104:107], v[26:29], v[142:145], a[104:107]\n" \
  "s_waitcnt lgkmcnt(12)\n" \
  "v_cvt_pk_f16_f32 v214, v182, v186\n" \
  "v_cvt_pk_f16_f32 v218, v183, v187\n" \
  "v_mfma_f32_16x16x32_f16 a[104:107], v[22:25], v[110:113], a[104:107]\n" \
  "v_cvt_pk_f16_f32 v222, v184, v188\n" \
  "v_cvt_pk_f16_f32 v226, v185, v189\n" \
  "v_mfma_f32_16x16x32_f16 a[108:111], v[22:25], v[114:117], a[108:111]\n" \
  "v_cvt_pk_f16_f32 v215, v190, v194\n" \
  "v_cvt_pk_f16_f32 v219, v191, v195\n" \
  "v_mfma_f32_16x16x32_f16 a[108:111], v[26:29], v[146:149], a[108:111]\n" \
  "s_waitcnt vmcnt(11)\n" \
  "v_cvt_pk_f16_f32 v223, v192, v196\n" \
  "v_cvt_pk_f16_f32 v227, v193, v197\n" \
  "v_mfma_f32_16x16x32_f16 a[76:79], v[46:49], v[146:149], a[76:79]\n" \
  "buffer_load_dwordx4 v[98:101], %[bxr], s[40:43], s73 offen offset:0 sc0 sc1\n" \
  "v_mfma_f32_16x16x32_f16 a[76:79], v[50:53], v[114:117], a[76:79]\n" \
  "s_waitcnt lgkmcnt(10)\n" \
  "v_min3_u32 v248, v238, v239, v240\n" \
  "v_min_u32 v248, v248, v241\n" \
  "v_mfma_f32_16x16x32_f16 a[80:83], v[50:53], v[118:121], a[80:83]\n" \
  "v_cmp_le_u32 s[34:35], s76, v248\n" \
  "v_mfma_f32_16x16x32_f16 a[80:83], v[46:49], v[150:153], a[80:83]\n" \
  "v_mfma_f32_16x16x32_f16 a[112:115], v[26:29], v[150:153], a[112:115]\n" \
  "buffer_load_dwordx4 v[94:97], %[bxr], s[40:43], s73 offen offset:1024 sc0 sc1\n" \
  "v_mfma_f32_16x16x32_f16 a[112:115], v[22:25], v[118:121], a[112:115]\n" \
  "s_waitcnt lgkmcnt(8)\n" \
  "v_mfma_f32_16x16x32_f16 a[116:119], v[22:25], v[122:125], a[116:119]\n" \
  "v_mfma_f32_16x16x32_f16 a[116:119], v[26:29], v[154:157], a[116:119]\n" \
  "v_mfma_f32_16x16x32_f16 a[84:87], v[46:49], v[154:157], a[84:87]\n" \
  "buffer_load_dwordx4 v[70:73], %[bxr], s[40:43], s73 offen offset:2048 sc0 sc1\n" \
  "v_mfma_f32_16x16x32_f16 a[84:87], v[50:53], v[122:125], a[84:87]\n" \
  "s_waitcnt lgkmcnt(5)\n" \
  "v_mfma_f32_16x16x32_f16 a[88:91], v[50:53], v[126:129], a[88:91]\n" \
  "v_mfma_f32_16x16x32_f16 a[88:91], v[46:49], v[158:161], a[88:91]\n" \
  "s_waitcnt lgkmcnt(2)\n" \
  "s_barrier\n" \
  "v_mfma_f32_16x16x32_f16 a[120:123], v[26:29], v[158:161], a[120:123]\n" \
  "buffer_load_dwordx4 v[174:177], %[voffW], s[36:39], s62 offen nt\n" \
  "v_mfma_f32_16x16x32_f16 a[120:123], v[22:25], v[126:129], a[120:123]\n" \
  "buffer_load_dwordx4 v[74:77], %[bxr], s[40:43], s73 offen offset:3072 sc0 sc1\n" \
  "v_mfma_f32_16x16x32_f16 a[124:127], v[22:25], v[130:133], a[124:127]\n" \
  "buffer_load_dwordx4 v[178:181], %[voffW], s[36:39], s63 offen nt\n" \
  "v_mfma_f32_16x16x32_f16 a[124:127], v[26:29], v[162:165], a[124:127]\n" \
  "v_mfma_f32_16x16x32_f16 a[92:95], v[46:49], v[162:165], a[92:95]\n" \
  "buffer_load_dwordx4 v[182:185], %[voffW], s[36:39], s64 offen nt\n" \
  "v_mfma_f32_16x16x32_f16 a[92:95], v[50:53], v[130:133], a[92:95]\n" \
  "s_waitcnt vmcnt(17)\n" \
  "v_mfma_f32_16x16x32_f16 a[192:195], v[66:69], v[102:105], a[192:195]\n" \
  "buffer_load_dwordx4 v[186:189], %[voffW], s[36:39], s65 offen nt\n" \
  "v_mfma_f32_16x16x32_f16 a[192:195], v[62:65], v[134:137], a[192:195]\n" \
  "s_memrealtime s[58:59]\n" \
  "v_mfma_f32_16x16x32_f16 a[224:227], v[42:45], v[134:137], a[224:227]\n" \
  "buffer_load_dwordx4 v[190:193], %[voffW], s[36:39], s66 offen nt\n" \
  "v_mfma_f32_16x16x32_f16 a[224:227], v[38:41], v[102:105], a[224:227]\n" \
  "v_mfma_f32_16x16x32_f16 a[228:231], v[38:41], v[106:109], a[228:231]\n" \
  "buffer_load_dwordx4 v[194:197], %[voffW], s[36:39], s67 offen nt\n" \
  "v_mfma_f32_16x16x32_f16 a[228:231], v[42:45], v[138:141], a[228:231]\n" \
  "s_cmp_eq_u32 s77, 9\n" \
  "s_cbranch_scc1 L_s86_%=\n" \
  "L_b86_%=:\n" \
  "v_mfma_f32_16x16x32_f16 a[196:199], v[62:65], v[138:141], a[196:199]\n" \
  "v_mfma_f32_16x16x32_f16 a[196:199], v[66:69], v[106:109], a[196:199]\n" \
  "v_mfma_f32_16x16x32_f16 a[200:203], v[66:69], v[110:113], a[200:203]\n" \
  "v_mfma_f32_16x16x32_f16 a[200:203], v[62:65], v[142:145], a[200:203]\n" \
  "s_waitcnt lgkmcnt(0)\n" \
  "v_mfma_f32_16x16x32_f16 a[232:235], v[42:45], v[142:145], a[232:235]\n" \
  "s_sub_u32 s82, s81, s58\n" \
  "s_branch L_s27_%=\n" \
  "L_b27_%=:\n" \
  "v_mfma_f32_16x16x32_f16 a[232:235], v[38:41], v[110:113], a[232:235]\n" \
  "s_max_i32 s82, s82, -150\n" \
  "s_add_u32 s81, s58, s82\n" \
  "s_add_u32 s81, s81, s87\n" \
  "v_mfma_f32_16x16x32_f16 a[236:239], v[38:41], v[114:117], a[236:239]\n" \
  "v_mfma_f32_16x16x32_f16 a[236:239], v[42:45], v[146:149], a[236:239]\n" \
  "v_mfma_f32_16x16x32_f16 a[204:207], v[62:65], v[146:149], a[204:207]\n" \
  "v_mfma_f32_16x16x32_f16 a[204:207], v[66:69], v[114:117], a[204:207]\n" \
  "v_mfma_f32_16x16x32_f16 a[208:211], v[66:69], v[118:121], a[208:211]\n" \
  "v_mfma_f32_16x16x32_f16 a[208:211], v[62:65], v[150:153], a[208:211]\n" \
  "v_mfma_f32_16x16x32_f16 a[240:243], v[42:45], v[150:153], a[240:243]\n" \
  "v_mfma_f32_16x16x32_f16 a[240:243], v[38:41], v[118:121], a[240:243]\n" \
  "v_readlane_b32 s74, %[vtabA], 6\n" \
  "v_mfma_f32_16x16x32_f16 a[244:247], v[38:41], v[122:125], a[244:247]\n" \
  "s_bitcmp1_b32 s34, 6\n" \
  "s_cbranch_scc0 L_s51_%=\n" \
  "L_b51_%=:\n" \
  "v_mfma_f32_16x16x32_f16 a[244:247], v[42:45], v[154:157], a[244:247]\n" \
  "s_waitcnt vmcnt(18)\n" \
  "s_add_u32 s72, s74, s68\n" \
  "v_mfma_f32_16x16x32_f16 a[212:215], v[62:65], v[154:157], a[212:215]\n" \
  "v_cvt_pk_f16_f32 v216, v198, v202\n" \
  "v_cvt_pk_f16_f32 v220, v199, v203\n" \
  "v_mfma_f32_16x16x32_f16 a[212:215], v[66:69], v[122:125], a[212:215]\n" \
  "s_waitcnt vmcnt(16)\n" \
  "v_cvt_pk_f16_f32 v224, v200, v204\n" \
  "v_cvt_pk_f16_f32 v228, v201, v205\n" \
  "v_mfma_f32_16x16x32_f16 a[216:219], v[66:69], v[126:129], a[216:219]\n" \
  "v_cvt_pk_f16_f32 v217, v206, v210\n" \
  "v_cvt_pk_f16_f32 v221, v207, v211\n" \
  "v_mfma_f32_16x16x32_f16 a[216:219], v[62:65], v[158:161], a[216:219]\n" \
  "v_cvt_pk_f16_f32 v225, v208, v212\n" \
  "v_cvt_pk_f16_f32 v229, v209, v213\n" \
  "v_mfma_f32_16x16x32_f16 a[248:251], v[42:45], v[158:161], a[248:251]\n" \
  "ds_write_b128 v242, v[214:217] offset:0\n" \
  "v_mfma_f32_16x16x32_f16 a[248:251], v[38:41], v[126:129], a[248:251]\n" \
  "buffer_load_dwordx4 v[14:17], %[bxr], s[40:43], s72 offen offset:0 sc0 sc1\n" \
  "v_mfma_f32_16x16x32_f16 a[252:255], v[38:41], v[130:133], a[252:255]\n" \
  "v_readlane_b32 s75, %[vtabW], 10\n" \
  "v_mfma_f32_16x16x32_f16 a[252:255], v[42:45], v[162:165], a[252:255]\n" \
  "s_waitcnt vmcnt(11)\n" \
  "v_mfma_f32_16x16x32_f16 a[220:223], v[62:65], v[162:165], a[220:223]\n" \
  "ds_write_b128 v242, v[218:221] offset:1024\n" \
  "v_mfma_f32_16x16x32_f16 a[220:223], v[66:69], v[130:133], a[220:223]\n" \
  "buffer_load_dwordx4 v[18:21], %[bxr], s[40:43], s72 offen offset:1024 sc0 sc1\n" \
  "v_mfma_f32_16x16x32_f16 a[0:3], v[82:85], v[102:105], a[0:3]\n" \
  "s_add_u32 s60, s75, 0x0\n" \
  "s_add_u32 s61, s75, 0x10000\n" \
  "s_add_u32 s62, s75, 0x20000\n" \
  "v_mfma_f32_16x16x32_f16 a[0:3], v[78:81], v[134:137], a[0:3]\n" \
  "s_add_u32 s63, s75, 0x30000\n" \
  "s_add_u32 s64, s75, 0x100000\n" \
  "s_add_u32 s65, s75, 0x110000\n" \
  "v_mfma_f32_16x16x32_f16 a[32:35], v[58:61], v[134:137], a[32:35]\n" \
  "ds_write_b128 v242, v[222:225] offset:2048\n" \
  "v_mfma_f32_16x16x32_f16 a[32:35], v[54:57], v[102:105], a[32:35]\n" \
  "buffer_load_dwordx4 v[86:89], %[bxr], s[40:43], s72 offen offset:2048 sc0 sc1\n" \
  "v_mfma_f32_16x16x32_f16 a[36:39], v[54:57], v[106:109], a[36:39]\n" \
  "v_mfma_f32_16x16x32_f16 a[36:39], v[58:61], v[138:141], a[36:39]\n" \
  "s_add_u32 s73, s74, s69\n" \
  "v_mfma_f32_16x16x32_f16 a[4:7], v[78:81], v[138:141], a[4:7]\n" \
  "ds_write_b128 v242, v[226:229] offset:3072\n" \
  "v_mfma_f32_16x16x32_f16 a[4:7], v[82:85], v[106:109], a[4:7]\n" \
  "buffer_load_dwordx4 v[90:93], %[bxr], s[40:43], s72 offen offset:3072 sc0 sc1\n" \
  "v_mfma_f32_16x16x32_f16 a[8:11], v[82:85], v[110:113], a[8:11]\n" \
  "v_mfma_f32_16x16x32_f16 a[8:11], v[78:81], v[142:145], a[8:11]\n" \
  "v_mfma_f32_16x16x32_f16 a[40:43], v[58:61], v[142:145], a[40:43]\n" \
  "v_mfma_f32_16x16x32_f16 a[40:43], v[54:57], v[110:113], a[40:43]\n" \
  "v_cvt_pk_f16_f32 v214, v166, v170\n" \
  "v_cvt_pk_f16_f32 v218, v167, v171\n" \
  "v_mfma_f32_16x16x32_f16 a[44:47], v[54:57], v[114:117], a[44:47]\n" \
  "v_cvt_pk_f16_f32 v222, v168, v172\n" \
  "v_cvt_pk_f16_f32 v226, v169, v173\n" \
  "v_mfma_f32_16x16x32_f16 a[44:47], v[58:61], v[146:149], a[44:47]\n" \
  "buffer_load_dwordx4 v[30:33], %[bxr], s[40:43], s73 offen offset:0 sc0 sc1\n" \
  "v_mfma_f32_16x16x32_f16 a[12:15], v[78:81], v[146:149], a[12:15]\n" \
  "v_mfma_f32_16x16x32_f16 a[12:15], v[82:85], v[114:117], a[12:15]\n" \
  "v_mfma_f32_16x16x32_f16 a[16:19], v[82:85], v[118:121], a[16:19]\n" \
  "v_mfma_f32_16x16x32_f16 a[16:19], v[78:81], v[150:153], a[16:19]\n" \
  "buffer_load_dwordx4 v[34:37], %[bxr], s[40:43], s73 offen offset:1024 sc0 sc1\n" \
  "v_mfma_f32_16x16x32_f16 a[48:51], v[58:61], v[150:153], a[48:51]\n" \
  "v_mfma_f32_16x16x32_f16 a[48:51], v[54:57], v[118:121], a[48:51]\n" \
  "v_mfma_f32_16x16x32_f16 a[52:55], v[54:57], v[122:125], a[52:55]\n" \
  "v_mfma_f32_16x16x32_f16 a[52:55], v[58:61], v[154:157], a[52:55]\n" \
  "buffer_load_dwordx4 v[22:25], %[bxr], s[40:43], s73 offen offset:2048 sc0 sc1\n" \
  "v_mfma_f32_16x16x32_f16 a[20:23], v[78:81], v[154:157], a[20:23]\n" \
  "v_mfma_f32_16x16x32_f16 a[20:23], v[82:85], v[122:125], a[20:23]\n" \
  "s_andn2_b32 s79, 0x3ff, s34\n" \
  "s_cbranch_scc1 L_pl6_%=\n" \
  "L_pb6_%=:\n" \
  "v_mfma_f32_16x16x32_f16 a[24:27], v[82:85], v[126:129], a[24:27]\n" \
  "s_barrier\n" \
  "v_mfma_f32_16x16x32_f16 a[24:27], v[78:81], v[158:161], a[24:27]\n" \
  "buffer_load_dwordx4 v[26:29], %[bxr], s[40:43], s73 offen offset:3072 sc0 sc1\n" \
  "v_mfma_f32_16x16x32_f16 a[56:59], v[58:61], v[158:161], a[56:59]\n" \
  "buffer_load_dwordx4 v[198:201], %[voffW], s[36:39], s60 offen nt\n" \
  "v_mfma_f32_16x16x32_f16 a[56:59], v[54:57], v[126:129], a[56:59]\n" \
  "v_mfma_f32_16x16x32_f16 a[60:63], v[54:57], v[130:133], a[60:63]\n" \
  "buffer_load_dwordx4 v[202:205], %[voffW], s[36:39], s61 offen nt\n" \
  "v_mfma_f32_16x16x32_f16 a[60:63], v[58:61], v[162:165], a[60:63]\n" \
  "v_mfma_f32_16x16x32_f16 a[28:31], v[78:81], v[162:165], a[28:31]\n" \
  "buffer_load_dwordx4 v[206:209], %[voffW], s[36:39], s62 offen nt\n" \
  "v_mfma_f32_16x16x32_f16 a[28:31], v[82:85], v[130:133], a[28:31]\n" \
  "s_waitcnt vmcnt(16)\n" \
  "v_mfma_f32_16x16x32_f16 a[128:131], v[98:101], v[102:105], a[128:131]\n" \
  "buffer_load_dwordx4 v[210:213], %[voffW], s[36:39], s63 offen nt\n" \
  "v_mfma_f32_16x16x32_f16 a[128:131], v[94:97], v[134:137], a[128:131]\n" \
  "v_mfma_f32_16x16x32_f16 a[160:163], v[74:77], v[134:137], a[160:163]\n" \
  "ds_read_b128 v[134:137], v246 offset:40960\n" \
  "v_mfma_f32_16x16x32_f16 a[160:163], v[70:73], v[102:105], a[160:163]\n" \
  "ds_read_b128 v[102:105], v246 offset:32768\n" \
  "v_mfma_f32_16x16x32_f16 a[164:167], v[70:73], v[106:109], a[164:167]\n" \
  "buffer_load_dwordx4 v[166:169], %[voffW], s[36:39], s64 offen nt\n" \
  "v_mfma_f32_16x16x32_f16 a[164:167], v[74:77], v[138:141], a[164:167]\n" \
  "v_mfma_f32_16x16x32_f16 a[132:135], v[94:97], v[138:141], a[132:135]\n" \
  "ds_read_b128 v[138:141], v246 offset:41984\n" \
  "v_mfma_f32_16x16x32_f16 a[132:135], v[98:101], v[106:109], a[132:135]\n" \
  "ds_read_b128 v[106:109], v246 offset:33792\n" \
  "v_mfma_f32_16x16x32_f16 a[136:139], v[98:101], v[110:113], a[136:139]\n" \
  "buffer_load_dwordx4 v[170:173], %[voffW], s[36:39], s65 offen nt\n" \
  "v_mfma_f32_16x16x32_f16 a[136:139], v[94:97], v[142:145], a[136:139]\n" \
  "s_cmp_eq_u32 s77, 10\n" \
  "s_cbranch_scc1 L_s87_%=\n" \
  "L_b87_%=:\n" \
  "v_mfma_f32_16x16x32_f16 a[168:171], v[74:77], v[142:145], a[168:171]\n" \
  "ds_read_b128 v[142:145], v246 offset:43008\n" \
  "v_mfma_f32_16x16x32_f16 a[168:171], v[70:73], v[110:113], a[168:171]\n" \
  "ds_read_b128 v[110:113], v246 offset:34816\n" \
  "v_mfma_f32_16x16x32_f16 a[172:175], v[70:73], v[114:117], a[172:175]\n" \
  "v_mfma_f32_16x16x32_f16 a[172:175], v[74:77], v[146:149], a[172:175]\n" \
  "v_mfma_f32_16x16x32_f16 a[140:143], v[94:97], v[146:149], a[140:143]\n" \
  "ds_read_b128 v[146:149], v246 offset:44032\n" \
  "v_mfma_f32_16x16x32_f16 a[140:143], v[98:101], v[114:117], a[140:143]\n" \
  "ds_read_b128 v[114:117], v246 offset:35840\n" \
  "v_mfma_f32_16x16x32_f16 a[144:147], v[98:101], v[118:121], a[144:147]\n" \
  "s_add_u32 s72, s74, s70\n" \
  "v_mfma_f32_16x16x32_f16 a[144:147], v[94:97], v[150:153], a[144:147]\n" \
  "v_mfma_f32_16x16x32_f16 a[176:179], v[74:77], v[150:153], a[176:179]\n" \
  "ds_read_b128 v[150:153], v246 offset:45056\n" \
  "v_mfma_f32_16x16x32_f16 a[176:179], v[70:73], v[118:121], a[176:179]\n" \
  "ds_read_b128 v[118:121], v246 offset:36864\n" \
  "v_mfma_f32_16x16x32_f16 a[180:183], v[70:73], v[122:125], a[180:183]\n" \
  "v_mfma_f32_16x16x32_f16 a[180:183], v[74:77], v[154:157], a[180:183]\n" \
  "v_mfma_f32_16x16x32_f16 a[148:151], v[94:97], v[154:157], a[148:151]\n" \
  "ds_read_b128 v[154:157], v246 offset:46080\n" \
  "v_mfma_f32_16x16x32_f16 a[148:151], v[98:101], v[122:125], a[148:151]\n" \
  "ds_read_b128 v[122:125], v246 offset:37888\n" \
  "v_mfma_f32_16x16x32_f16 a[152:155], v[98:101], v[126:129], a[152:155]\n" \
  "buffer_load_dwordx4 v[46:49], %[bxr], s[40:43], s72 offen offset:0 sc0 sc1\n" \
  "v_mfma_f32_16x16x32_f16 a[152:155], v[94:97], v[158:161], a[152:155]\n" \
  "v_mfma_f32_16x16x32_f16 a[184:187], v[74:77], v[158:161], a[184:187]\n" \
  "ds_read_b128 v[158:161], v246 offset:47104\n" \
  "v_mfma_f32_16x16x32_f16 a[184:187], v[70:73], v[126:129], a[184:187]\n" \
  "ds_read_b128 v[126:129], v246 offset:38912\n" \
  "v_mfma_f32_16x16x32_f16 a[188:191], v[70:73], v[130:133], a[188:191]\n" \
  "buffer_load_dwordx4 v[50:53], %[bxr], s[40:43], s72 offen offset:1024 sc0 sc1\n" \
  "v_mfma_f32_16x16x32_f16 a[188:191], v[74:77], v[162:165], a[188:191]\n" \
  "s_add_u32 s66, s75, 0x120000\n" \
  "s_add_u32 s67, s75, 0x130000\n" \
  "v_mfma_f32_16x16x32_f16 a[156:159], v[94:97], v[162:165], a[156:159]\n" \
  "ds_read_b128 v[162:165], v246 offset:48128\n" \
  "v_readlane_b32 s75, %[vtabW], 11\n" \
  "v_mfma_f32_16x16x32_f16 a[156:159], v[98:101], v[130:133], a[156:159]\n" \
  "s_waitcnt vmcnt(12)\n" \
  "ds_read_b128 v[130:133], v246 offset:39936\n" \
  "s_waitcnt lgkmcnt(14)\n" \
    \
  "v_mfma_f32_16x16x32_f16 a[64:67], v[14:17], v[102:105], a[64:67]\n" \
  "buffer_load_dwordx4 v[38:41], %[bxr], s[40:43], s72 offen offset:2048 sc0 sc1\n" \
  "v_mfma_f32_16x16x32_f16 a[64:67], v[18:21], v[134:137], a[64:67]\n" \
  "s_add_u32 s60, s75, 0x0\n" \
  "s_add_u32 s61, s75, 0x10000\n" \
  "s_add_u32 s62, s75, 0x20000\n" \
  "v_mfma_f32_16x16x32_f16 a[96:99], v[90:93], v[134:137], a[96:99]\n" \
  "s_add_u32 s63, s75, 0x30000\n" \
  "s_add_u32 s73, s74, s71\n" \
  "v_mfma_f32_16x16x32_f16 a[96:99], v[86:89], v[102:105], a[96:99]\n" \
  "s_waitcnt lgkmcnt(12)\n" \
  "v_mfma_f32_16x16x32_f16 a[100:103], v[86:89], v[106:109], a[100:103]\n" \
  "buffer_load_dwordx4 v[42:45], %[bxr], s[40:43], s72 offen offset:3072 sc0 sc1\n" \
  "v_mfma_f32_16x16x32_f16 a[100:103], v[90:93], v[138:141], a[100:103]\n" \
  "v_mfma_f32_16x16x32_f16 a[68:71], v[18:21], v[138:141], a[68:71]\n" \
  "v_mfma_f32_16x16x32_f16 a[68:71], v[14:17], v[106:109], a[68:71]\n" \
  "s_waitcnt lgkmcnt(10)\n" \
  "v_mfma_f32_16x16x32_f16 a[72:75], v[14:17], v[110:113], a[72:75]\n" \
  "buffer_load_dwordx4 v[62:65], %[bxr], s[40:43], s73 offen offset:0 sc0 sc1\n" \
  "v_mfma_f32_16x16x32_f16 a[72:75], v[18:21], v[142:145], a[72:75]\n" \
  "v_mfma_f32_16x16x32_f16 a[104:107], v[90:93], v[142:145], a[104:107]\n" \
  "v_mfma_f32_16x16x32_f16 a[104:107], v[86:89], v[110:113], a[104:107]\n" \
  "s_waitcnt lgkmcnt(8)\n" \
  "v_cvt_pk_f16_f32 v215, v174, v178\n" \
  "v_cvt_pk_f16_f32 v219, v175, v179\n" \
  "v_mfma_f32_16x16x32_f16 a[108:111], v[86:89], v[114:117], a[108:111]\n" \
  "v_cvt_pk_f16_f32 v223, v176, v180\n" \
  "v_cvt_pk_f16_f32 v227, v177, v181\n" \
  "v_mfma_f32_16x16x32_f16 a[108:111], v[90:93], v[146:149], a[108:111]\n" \
  "s_waitcnt vmcnt(12)\n" \
  "v_cvt_pk_f16_f32 v216, v182, v186\n" \
  "v_cvt_pk_f16_f32 v220, v183, v187\n" \
  "v_mfma_f32_16x16x32_f16 a[76:79], v[18:21], v[146:149], a[76:79]\n" \
  "v_cvt_pk_f16_f32 v224, v184, v188\n" \
  "v_cvt_pk_f16_f32 v228, v185, v189\n" \
  "v_mfma_f32_16x16x32_f16 a[76:79], v[14:17], v[114:117], a[76:79]\n" \
  "s_waitcnt lgkmcnt(6)\n" \
  "v_cvt_pk_f16_f32 v217, v190, v194\n" \
  "v_cvt_pk_f16_f32 v221, v191, v195\n" \
  "v_mfma_f32_16x16x32_f16 a[80:83], v[14:17], v[118:121], a[80:83]\n" \
  "v_cvt_pk_f16_f32 v225, v192, v196\n" \
  "v_cvt_pk_f16_f32 v229, v193, v197\n" \
  "v_mfma_f32_16x16x32_f16 a[80:83], v[18:21], v[150:153], a[80:83]\n" \
  "ds_write_b128 v242, v[214:217] offset:16384\n" \
  "v_mfma_f32_16x16x32_f16 a[112:115], v[90:93], v[150:153], a[112:115]\n" \
  "buffer_load_dwordx4 v[66:69], %[bxr], s[40:43], s73 offen offset:1024 sc0 sc1\n" \
  "v_mfma_f32_16x16x32_f16 a[112:115], v[86:89], v[118:121], a[112:115]\n" \
  "s_waitcnt lgkmcnt(5)\n" \
  "v_min3_u32 v248, v238, v239, v240\n" \
  "v_min_u32 v248, v248, v241\n" \
  "v_mfma_f32_16x16x32_f16 a[116:119], v[86:89], v[122:125], a[116:119]\n" \
  "v_cmp_le_u32 s[34:35], s76, v248\n" \
  "v_mfma_f32_16x16x32_f16 a[116:119], v[90:93], v[154:157], a[116:119]\n" \
  "ds_write_b128 v242, v[218:221] offset:17408\n" \
  "v_mfma_f32_16x16x32_f16 a[84:87], v[18:21], v[154:157], a[84:87]\n" \
  "buffer_load_dwordx4 v[54:57], %[bxr], s[40:43], s73 offen offset:2048 sc0 sc1\n" \
  "v_mfma_f32_16x16x32_f16 a[84:87], v[14:17], v[122:125], a[84:87]\n" \
  "s_waitcnt lgkmcnt(4)\n" \
  "v_mfma_f32_16x16x32_f16 a[88:91], v[14:17], v[126:129], a[88:91]\n" \
  "s_waitcnt lgkmcnt(2)\n" \
  "s_barrier\n" \
  "v_mfma_f32_16x16x32_f16 a[88:91], v[18:21], v[158:161], a[88:91]\n" \
  "ds_write_b128 v242, v[222:225] offset:18432\n" \
  "v_mfma_f32_16x16x32_f16 a[120:123], v[90:93], v[158:161], a[120:123]\n" \
  "buffer_load_dwordx4 v[174:177], %[voffW], s[36:39], s66 offen nt\n" \
  "v_mfma_f32_16x16x32_f16 a[120:123], v[86:89], v[126:129], a[120:123]\n" \
  "buffer_load_dwordx4 v[58:61], %[bxr], s[40:43], s73 offen offset:3072 sc0 sc1\n" \
  "v_mfma_f32_16x16x32_f16 a[124:127], v[86:89], v[130:133], a[124:127]\n" \
  "buffer_load_dwordx4 v[178:181], %[voffW], s[36:39], s67 offen nt\n" \
  "v_mfma_f32_16x16x32_f16 a[124:127], v[90:93], v[162:165], a[124:127]\n" \
  "ds_write_b128 v242, v[226:229] offset:19456\n" \
  "v_mfma_f32_16x16x32_f16 a[92:95], v[18:21], v[162:165], a[92:95]\n" \
  "buffer_load_dwordx4 v[182:185], %[voffW], s[36:39], s60 offen nt\n" \
  "v_mfma_f32_16x16x32_f16 a[92:95], v[14:17], v[130:133], a[92:95]\n" \
  "s_waitcnt vmcnt(17)\n" \
  "v_mfma_f32_16x16x32_f16 a[192:195], v[30:33], v[102:105], a[192:195]\n" \
  "buffer_load_dwordx4 v[186:189], %[voffW], s[36:39], s61 offen nt\n" \
  "v_mfma_f32_16x16x32_f16 a[192:195], v[34:37], v[134:137], a[192:195]\n" \
  "s_memrealtime s[58:59]\n" \
  "v_mfma_f32_16x16x32_f16 a[224:227], v[26:29], v[134:137], a[224:227]\n" \
  "buffer_load_dwordx4 v[190:193], %[voffW], s[36:39], s62 offen nt\n" \
  "v_mfma_f32_16x16x32_f16 a[224:227], v[22:25], v[102:105], a[224:227]\n" \
  "v_mfma_f32_16x16x32_f16 a[228:231], v[22:25], v[106:109], a[228:231]\n" \
  "buffer_load_dwordx4 v[194:197], %[voffW], s[36:39], s63 offen nt\n" \
  "v_mfma_f32_16x16x32_f16 a[228:231], v[26:29], v[138:141], a[228:231]\n" \
  "s_cmp_eq_u32 s77, 11\n" \
  "s_cbranch_scc1 L_s88_%=\n" \
  "L_b88_%=:\n" \
  "v_mfma_f32_16x16x32_f16 a[196:199], v[34:37], v[138:141], a[196:199]\n" \
  "v_mfma_f32_16x16x32_f16 a[196:199], v[30:33], v[106:109], a[196:199]\n" \
  "v_mfma_f32_16x16x32_f16 a[200:203], v[30:33], v[110:113], a[200:203]\n" \
  "v_mfma_f32_16x16x32_f16 a[200:203], v[34:37], v[142:145], a[200:203]\n" \
  "s_waitcnt lgkmcnt(0)\n" \
  "v_mfma_f32_16x16x32_f16 a[232:235], v[26:29], v[142:145], a[232:235]\n" \
  "s_sub_u32 s82, s81, s58\n" \
  "s_branch L_s28_%=\n" \
  "L_b28_%=:\n" \
  "v_mfma_f32_16x16x32_f16 a[232:235], v[22:25], v[110:113], a[232:235]\n" \
  "s_max_i32 s82, s82, -150\n" \
  "s_add_u32 s81, s58, s82\n" \
  "s_add_u32 s81, s81, s87\n" \
  "v_mfma_f32_16x16x32_f16 a[236:239], v[22:25], v[114:117], a[236:239]\n" \
  "v_mfma_f32_16x16x32_f16 a[236:239], v[26:29], v[146:149], a[236:239]\n" \
  "v_mfma_f32_16x16x32_f16 a[204:207], v[34:37], v[146:149], a[204:207]\n" \
  "v_readlane_b32 s74, %[vtabA], 7\n" \
  "v_mfma_f32_16x16x32_f16 a[204:207], v[30:33], v[114:117], a[204:207]\n" \
  "s_bitcmp1_b32 s34, 7\n" \
  "s_cbranch_scc0 L_s52_%=\n" \
  "L_b52_%=:\n" \
  "v_mfma_f32_16x16x32_f16 a[208:211], v[30:33], v[118:121], a[208:211]\n" \
  "s_add_u32 s72, s74, s68\n" \
  "v_mfma_f32_16x16x32_f16 a[208:211], v[34:37], v[150:153], a[208:211]\n" \
  "v_mfma_f32_16x16x32_f16 a[240:243], v[26:29], v[150:153], a[240:243]\n" \
  "v_mfma_f32_16x16x32_f16 a[240:243], v[22:25], v[118:121], a[240:243]\n" \
  "v_mfma_f32_16x16x32_f16 a[244:247], v[22:25], v[122:125], a[244:247]\n" \
  "v_mfma_f32_16x16x32_f16 a[244:247], v[26:29], v[154:157], a[244:247]\n" \
  "v_mfma_f32_16x16x32_f16 a[212:215], v[34:37], v[154:157], a[212:215]\n" \
  "buffer_load_dwordx4 v[78:81], %[bxr], s[40:43], s72 offen offset:0 sc0 sc1\n" \
  "v_mfma_f32_16x16x32_f16 a[212:215], v[30:33], v[122:125], a[212:215]\n" \
  "v_mfma_f32_16x16x32_f16 a[216:219], v[30:33], v[126:129], a[216:219]\n" \
  "v_mfma_f32_16x16x32_f16 a[216:219], v[34:37], v[158:161], a[216:219]\n" \
  "v_mfma_f32_16x16x32_f16 a[248:251], v[26:29], v[158:161], a[248:251]\n" \
  "buffer_load_dwordx4 v[82:85], %[bxr], s[40:43], s72 offen offset:1024 sc0 sc1\n" \
  "v_mfma_f32_16x16x32_f16 a[248:251], v[22:25], v[126:129], a[248:251]\n" \
  "v_mfma_f32_16x16x32_f16 a[252:255], v[22:25], v[130:133], a[252:255]\n" \
  "s_add_u32 s64, s75, 0x100000\n" \
  "s_add_u32 s65, s75, 0x110000\n" \
  "s_add_u32 s66, s75, 0x120000\n" \
  "v_mfma_f32_16x16x32_f16 a[252:255], v[26:29], v[162:165], a[252:255]\n" \
  "s_add_u32 s67, s75, 0x130000\n" \
  "v_mfma_f32_16x16x32_f16 a[220:223], v[34:37], v[162:165], a[220:223]\n" \
  "buffer_load_dwordx4 v[70:73], %[bxr], s[40:43], s72 offen offset:2048 sc0 sc1\n" \
  "v_mfma_f32_16x16x32_f16 a[220:223], v[30:33], v[130:133], a[220:223]\n" \
  "s_waitcnt vmcnt(13)\n" \
  "v_readlane_b32 s75, %[vtabW], 12\n" \
  "s_add_u32 s73, s74, s69\n" \
  "v_mfma_f32_16x16x32_f16 a[0:3], v[46:49], v[102:105], a[0:3]\n" \
  "v_mfma_f32_16x16x32_f16 a[0:3], v[50:53], v[134:137], a[0:3]\n" \
  "s_add_u32 s60, s75, 0x0\n" \
  "s_add_u32 s61, s75, 0x10000\n" \
  "v_mfma_f32_16x16x32_f16 a[32:35], v[42:45], v[134:137], a[32:35]\n" \
  "buffer_load_dwordx4 v[74:77], %[bxr], s[40:43], s72 offen offset:3072 sc0 sc1\n" \
  "v_mfma_f32_16x16x32_f16 a[32:35], v[38:41], v[102:105], a[32:35]\n" \
  "v_mfma_f32_16x16x32_f16 a[36:39], v[38:41], v[106:109], a[36:39]\n" \
  "v_mfma_f32_16x16x32_f16 a[36:39], v[42:45], v[138:141], a[36:39]\n" \
  "v_mfma_f32_16x16x32_f16 a[4:7], v[50:53], v[138:141], a[4:7]\n" \
  "buffer_load_dwordx4 v[94:97], %[bxr], s[40:43], s73 offen offset:0 sc0 sc1\n" \
  "v_mfma_f32_16x16x32_f16 a[4:7], v[46:49], v[106:109], a[4:7]\n" \
  "v_mfma_f32_16x16x32_f16 a[8:11], v[46:49], v[110:113], a[8:11]\n" \
  "v_mfma_f32_16x16x32_f16 a[8:11], v[50:53], v[142:145], a[8:11]\n" \
  "v_mfma_f32_16x16x32_f16 a[40:43], v[42:45], v[142:145], a[40:43]\n" \
  "v_mfma_f32_16x16x32_f16 a[40:43], v[38:41], v[110:113], a[40:43]\n" \
  "v_cvt_pk_f16_f32 v214, v198, v202\n" \
  "v_cvt_pk_f16_f32 v218, v199, v203\n" \
  "v_mfma_f32_16x16x32_f16 a[44:47], v[38:41], v[114:117], a[44:47]\n" \
  "v_cvt_pk_f16_f32 v222, v200, v204\n" \
  "v_cvt_pk_f16_f32 v226, v201, v205\n" \
  "v_mfma_f32_16x16x32_f16 a[44:47], v[42:45], v[146:149], a[44:47]\n" \
  "v_cvt_pk_f16_f32 v215, v206, v210\n" \
  "v_cvt_pk_f16_f32 v219, v207, v211\n" \
  "v_mfma_f32_16x16x32_f16 a[12:15], v[50:53], v[146:149], a[12:15]\n" \
  "v_cvt_pk_f16_f32 v223, v208, v212\n" \
  "v_cvt_pk_f16_f32 v227, v209, v213\n" \
  "v_mfma_f32_16x16x32_f16 a[12:15], v[46:49], v[114:117], a[12:15]\n" \
  "v_cvt_pk_f16_f32 v216, v166, v170\n" \
  "v_cvt_pk_f16_f32 v220, v167, v171\n" \
  "v_mfma_f32_16x16x32_f16 a[16:19], v[46:49], v[118:121], a[16:19]\n" \
  "v_cvt_pk_f16_f32 v224, v168, v172\n" \
  "v_cvt_pk_f16_f32 v228, v169, v173\n" \
  "v_mfma_f32_16x16x32_f16 a[16:19], v[50:53], v[150:153], a[16:19]\n" \
  "buffer_load_dwordx4 v[98:101], %[bxr], s[40:43], s73 offen offset:1024 sc0 sc1\n" \
  "v_mfma_f32_16x16x32_f16 a[48:51], v[42:45], v[150:153], a[48:51]\n" \
  "v_mfma_f32_16x16x32_f16 a[48:51], v[38:41], v[118:121], a[48:51]\n" \
  "v_mfma_f32_16x16x32_f16 a[52:55], v[38:41], v[122:125], a[52:55]\n" \
  "v_mfma_f32_16x16x32_f16 a[52:55], v[42:45], v[154:157], a[52:55]\n" \
  "buffer_load_dwordx4 v[86:89], %[bxr], s[40:43], s73 offen offset:2048 sc0 sc1\n" \
  "v_mfma_f32_16x16x32_f16 a[20:23], v[50:53], v[154:157], a[20:23]\n" \
  "v_mfma_f32_16x16x32_f16 a[20:23], v[46:49], v[122:125], a[20:23]\n" \
  "s_andn2_b32 s79, 0x7ff, s34\n" \
  "s_cbranch_scc1 L_pl7_%=\n" \
  "L_pb7_%=:\n" \
  "v_mfma_f32_16x16x32_f16 a[24:27], v[46:49], v[126:129], a[24:27]\n" \
  "s_barrier\n" \
  "v_mfma_f32_16x16x32_f16 a[24:27], v[50:53], v[158:161], a[24:27]\n" \
  "buffer_load_dwordx4 v[90:93], %[bxr], s[40:43], s73 offen offset:3072 sc0 sc1\n" \
  "v_mfma_f32_16x16x32_f16 a[56:59], v[42:45], v[158:161], a[56:59]\n" \
  "buffer_load_dwordx4 v[198:201], %[voffW], s[36:39], s64 offen nt\n" \
  "v_mfma_f32_16x16x32_f16 a[56:59], v[38:41], v[126:129], a[56:59]\n" \
  "v_mfma_f32_16x16x32_f16 a[60:63], v[38:41], v[130:133], a[60:63]\n" \
  "buffer_load_dwordx4 v[202:205], %[voffW], s[36:39], s65 offen nt\n" \
  "v_mfma_f32_16x16x32_f16 a[60:63], v[42:45], v[162:165], a[60:63]\n" \
  "v_mfma_f32_16x16x32_f16 a[28:31], v[50:53], v[162:165], a[28:31]\n" \
  "buffer_load_dwordx4 v[206:209], %[voffW], s[36:39], s66 offen nt\n" \
  "v_mfma_f32_16x16x32_f16 a[28:31], v[46:49], v[130:133], a[28:31]\n" \
  "s_waitcnt vmcnt(16)\n" \
  "v_mfma_f32_16x16x32_f16 a[128:131], v[62:65], v[102:105], a[128:131]\n" \
  "buffer_load_dwordx4 v[210:213], %[voffW], s[36:39], s67 offen nt\n" \
  "v_mfma_f32_16x16x32_f16 a[128:131], v[66:69], v[134:137], a[128:131]\n" \
  "v_mfma_f32_16x16x32_f16 a[160:163], v[58:61], v[134:137], a[160:163]\n" \
  "ds_read_b128 v[134:137], v246 offset:57344\n" \
  "v_mfma_f32_16x16x32_f16 a[160:163], v[54:57], v[102:105], a[160:163]\n" \
  "ds_read_b128 v[102:105], v246 offset:49152\n" \
  "v_mfma_f32_16x16x32_f16 a[164:167], v[54:57], v[106:109], a[164:167]\n" \
  "buffer_load_dwordx4 v[166:169], %[voffW], s[36:39], s60 offen nt\n" \
  "v_mfma_f32_16x16x32_f16 a[164:167], v[58:61], v[138:141], a[164:167]\n" \
  "v_mfma_f32_16x16x32_f16 a[132:135], v[66:69], v[138:141], a[132:135]\n" \
  "ds_read_b128 v[138:141], v246 offset:58368\n" \
  "v_mfma_f32_16x16x32_f16 a[132:135], v[62:65], v[106:109], a[132:135]\n" \
  "ds_read_b128 v[106:109], v246 offset:50176\n" \
  "v_mfma_f32_16x16x32_f16 a[136:139], v[62:65], v[110:113], a[136:139]\n" \
  "buffer_load_dwordx4 v[170:173], %[voffW], s[36:39], s61 offen nt\n" \
  "v_mfma_f32_16x16x32_f16 a[136:139], v[66:69], v[142:145], a[136:139]\n" \
  "s_cmp_eq_u32 s77, 12\n" \
  "s_cbranch_scc1 L_s89_%=\n" \
  "L_b89_%=:\n" \
  "v_mfma_f32_16x16x32_f16 a[168:171], v[58:61], v[142:145], a[168:171]\n" \
  "ds_read_b128 v[142:145], v246 offset:59392\n" \
  "v_mfma_f32_16x16x32_f16 a[168:171], v[54:57], v[110:113], a[168:171]\n" \
  "ds_read_b128 v[110:113], v246 offset:51200\n" \
  "v_mfma_f32_16x16x32_f16 a[172:175], v[54:57], v[114:117], a[172:175]\n" \
  "v_mfma_f32_16x16x32_f16 a[172:175], v[58:61], v[146:149], a[172:175]\n" \
  "v_mfma_f32_16x16x32_f16 a[140:143], v[66:69], v[146:149], a[140:143]\n" \
  "ds_read_b128 v[146:149], v246 offset:60416\n" \
  "v_mfma_f32_16x16x32_f16 a[140:143], v[62:65], v[114:117], a[140:143]\n" \
  "ds_read_b128 v[114:117], v246 offset:52224\n" \
  "v_mfma_f32_16x16x32_f16 a[144:147], v[62:65], v[118:121], a[144:147]\n" \
  "v_mfma_f32_16x16x32_f16 a[144:147], v[66:69], v[150:153], a[144:147]\n" \
  "v_mfma_f32_16x16x32_f16 a[176:179], v[58:61], v[150:153], a[176:179]\n" \
  "ds_read_b128 v[150:153], v246 offset:61440\n" \
  "v_mfma_f32_16x16x32_f16 a[176:179], v[54:57], v[118:121], a[176:179]\n" \
  "ds_read_b128 v[118:121], v246 offset:53248\n" \
  "v_mfma_f32_16x16x32_f16 a[180:183], v[54:57], v[122:125], a[180:183]\n" \
  "s_add_u32 s72, s74, s70\n" \
  "v_mfma_f32_16x16x32_f16 a[180:183], v[58:61], v[154:157], a[180:183]\n" \
  "s_waitcnt vmcnt(18)\n" \
  "v_mfma_f32_16x16x32_f16 a[148:151], v[66:69], v[154:157], a[148:151]\n" \
  "ds_read_b128 v[154:157], v246 offset:62464\n" \
  "v_cvt_pk_f16_f32 v217, v174, v178\n" \
  "v_mfma_f32_16x16x32_f16 a[148:151], v[62:65], v[122:125], a[148:151]\n" \
  "ds_read_b128 v[122:125], v246 offset:54272\n" \
  "v_cvt_pk_f16_f32 v221, v175, v179\n" \
  "v_mfma_f32_16x16x32_f16 a[152:155], v[62:65], v[126:129], a[152:155]\n" \
  "v_cvt_pk_f16_f32 v225, v176, v180\n" \
  "v_cvt_pk_f16_f32 v229, v177, v181\n" \
  "v_mfma_f32_16x16x32_f16 a[152:155], v[66:69], v[158:161], a[152:155]\n" \
  "ds_write_b128 v242, v[214:217] offset:32768\n" \
  "v_mfma_f32_16x16x32_f16 a[184:187], v[58:61], v[158:161], a[184:187]\n" \
  "ds_read_b128 v[158:161], v246 offset:63488\n" \
  "v_mfma_f32_16x16x32_f16 a[184:187], v[54:57], v[126:129], a[184:187]\n" \
  "ds_read_b128 v[126:129], v246 offset:55296\n" \
  "v_mfma_f32_16x16x32_f16 a[188:191], v[54:57], v[130:133], a[188:191]\n" \
  "buffer_load_dwordx4 v[18:21], %[bxr], s[40:43], s72 offen offset:0 sc0 sc1\n" \
  "v_mfma_f32_16x16x32_f16 a[188:191], v[58:61], v[162:165], a[188:191]\n" \
  "ds_write_b128 v242, v[218:221] offset:33792\n" \
  "v_mfma_f32_16x16x32_f16 a[156:159], v[66:69], v[162:165], a[156:159]\n" \
  "s_add_u32 s62, s75, 0x20000\n" \
  "ds_read_b128 v[162:165], v246 offset:64512\n" \
  "s_add_u32 s63, s75, 0x30000\n" \
  "v_mfma_f32_16x16x32_f16 a[156:159], v[62:65], v[130:133], a[156:159]\n" \
  "s_waitcnt vmcnt(11)\n" \
  "ds_read_b128 v[130:133], v246 offset:56320\n" \
  "s_waitcnt lgkmcnt(14)\n" \
    \
  "v_mfma_f32_16x16x32_f16 a[64:67], v[78:81], v[102:105], a[64:67]\n" \
  "buffer_load_dwordx4 v[14:17], %[bxr], s[40:43], s72 offen offset:1024 sc0 sc1\n" \
  "v_mfma_f32_16x16x32_f16 a[64:67], v[82:85], v[134:137], a[64:67]\n" \
  "ds_write_b128 v242, v[222:225] offset:34816\n" \
  "v_mfma_f32_16x16x32_f16 a[96:99], v[74:77], v[134:137], a[96:99]\n" \
  "s_add_u32 s64, s75, 0x100000\n" \
  "s_add_u32 s65, s75, 0x110000\n" \
  "s_add_u32 s66, s75, 0x120000\n" \
  "v_mfma_f32_16x16x32_f16 a[96:99], v[70:73], v[102:105], a[96:99]\n" \
  "s_waitcnt lgkmcnt(14)\n" \
  "s_add_u32 s67, s75, 0x130000\n" \
  "v_mfma_f32_16x16x32_f16 a[100:103], v[70:73], v[106:109], a[100:103]\n" \
  "buffer_load_dwordx4 v[22:25], %[bxr], s[40:43], s72 offen offset:2048 sc0 sc1\n" \
  "v_mfma_f32_16x16x32_f16 a[100:103], v[74:77], v[138:141], a[100:103]\n" \
  "ds_write_b128 v242, v[226:229] offset:35840\n" \
  "v_mfma_f32_16x16x32_f16 a[68:71], v[82:85], v[138:141], a[68:71]\n" \
  "s_add_u32 s73, s74, s71\n" \
  "v_mfma_f32_16x16x32_f16 a[68:71], v[78:81], v[106:109], a[68:71]\n" \
  "s_waitcnt lgkmcnt(14)\n" \
  "v_mfma_f32_16x16x32_f16 a[72:75], v[78:81], v[110:113], a[72:75]\n" \
  "buffer_load_dwordx4 v[26:29], %[bxr], s[40:43], s72 offen offset:3072 sc0 sc1\n" \
  "v_mfma_f32_16x16x32_f16 a[72:75], v[82:85], v[142:145], a[72:75]\n" \
  "v_mfma_f32_16x16x32_f16 a[104:107], v[74:77], v[142:145], a[104:107]\n" \
  "s_waitcnt lgkmcnt(12)\n" \
  "v_cvt_pk_f16_f32 v214, v182, v186\n" \
  "v_cvt_pk_f16_f32 v218, v183, v187\n" \
  "v_mfma_f32_16x16x32_f16 a[104:107], v[70:73], v[110:113], a[104:107]\n" \
  "v_cvt_pk_f16_f32 v222, v184, v188\n" \
  "v_cvt_pk_f16_f32 v226, v185, v189\n" \
  "v_mfma_f32_16x16x32_f16 a[108:111], v[70:73], v[114:117], a[108:111]\n" \
  "v_cvt_pk_f16_f32 v215, v190, v194\n" \
  "v_cvt_pk_f16_f32 v219, v191, v195\n" \
  "v_mfma_f32_16x16x32_f16 a[108:111], v[74:77], v[146:149], a[108:111]\n" \
  "s_waitcnt vmcnt(11)\n" \
  "v_cvt_pk_f16_f32 v223, v192, v196\n" \
  "v_cvt_pk_f16_f32 v227, v193, v197\n" \
  "v_mfma_f32_16x16x32_f16 a[76:79], v[82:85], v[146:149], a[76:79]\n" \
  "buffer_load_dwordx4 v[34:37], %[bxr], s[40:43], s73 offen offset:0 sc0 sc1\n" \
  "v_mfma_f32_16x16x32_f16 a[76:79], v[78:81], v[114:117], a[76:79]\n" \
  "s_waitcnt lgkmcnt(10)\n" \
  "v_min3_u32 v248, v238, v239, v240\n" \
  "v_min_u32 v248, v248, v241\n" \
  "v_mfma_f32_16x16x32_f16 a[80:83], v[78:81], v[118:121], a[80:83]\n" \
  "v_cmp_le_u32 s[34:35], s76, v248\n" \
  "v_mfma_f32_16x16x32_f16 a[80:83], v[82:85], v[150:153], a[80:83]\n" \
  "v_mfma_f32_16x16x32_f16 a[112:115], v[74:77], v[150:153], a[112:115]\n" \
  "buffer_load_dwordx4 v[30:33], %[bxr], s[40:43], s73 offen offset:1024 sc0 sc1\n" \
  "v_mfma_f32_16x16x32_f16 a[112:115], v[70:73], v[118:121], a[112:115]\n" \
  "s_waitcnt lgkmcnt(8)\n" \
  "v_mfma_f32_16x16x32_f16 a[116:119], v[70:73], v[122:125], a[116:119]\n" \
  "v_mfma_f32_16x16x32_f16 a[116:119], v[74:77], v[154:157], a[116:119]\n" \
  "v_mfma_f32_16x16x32_f16 a[84:87], v[82:85], v[154:157], a[84:87]\n" \
  "buffer_load_dwordx4 v[38:41], %[bxr], s[40:43], s73 offen offset:2048 sc0 sc1\n" \
  "v_mfma_f32_16x16x32_f16 a[84:87], v[78:81], v[122:125], a[84:87]\n" \
  "s_waitcnt lgkmcnt(5)\n" \
  "v_mfma_f32_16x16x32_f16 a[88:91], v[78:81], v[126:129], a[88:91]\n" \
  "v_mfma_f32_16x16x32_f16 a[88:91], v[82:85], v[158:161], a[88:91]\n" \
  "s_waitcnt lgkmcnt(2)\n" \
  "s_barrier\n" \
  "v_mfma_f32_16x16x32_f16 a[120:123], v[74:77], v[158:161], a[120:123]\n" \
  "buffer_load_dwordx4 v[174:177], %[voffW], s[36:39], s62 offen nt\n" \
  "v_mfma_f32_16x16x32_f16 a[120:123], v[70:73], v[126:129], a[120:123]\n" \
  "buffer_load_dwordx4 v[42:45], %[bxr], s[40:43], s73 offen offset:3072 sc0 sc1\n" \
  "v_mfma_f32_16x16x32_f16 a[124:127], v[70:73], v[130:133], a[124:127]\n" \
  "buffer_load_dwordx4 v[178:181], %[voffW], s[36:39], s63 offen nt\n" \
  "v_mfma_f32_16x16x32_f16 a[124:127], v[74:77], v[162:165], a[124:127]\n" \
  "v_mfma_f32_16x16x32_f16 a[92:95], v[82:85], v[162:165], a[92:95]\n" \
  "buffer_load_dwordx4 v[182:185], %[voffW], s[36:39], s64 offen nt\n" \
  "v_mfma_f32_16x16x32_f16 a[92:95], v[78:81], v[130:133], a[92:95]\n" \
  "s_waitcnt vmcnt(17)\n" \
  "v_mfma_f32_16x16x32_f16 a[192:195], v[94:97], v[102:105], a[192:195]\n" \
  "buffer_load_dwordx4 v[186:189], %[voffW], s[36:39], s65 offen nt\n" \
  "v_mfma_f32_16x16x32_f16 a[192:195], v[98:101], v[134:137], a[192:195]\n" \
  "s_memrealtime s[58:59]\n" \
  "v_mfma_f32_16x16x32_f16 a[224:227], v[90:93], v[134:137], a[224:227]\n" \
  "buffer_load_dwordx4 v[190:193], %[voffW], s[36:39], s66 offen nt\n" \
  "v_mfma_f32_16x16x32_f16 a[224:227], v[86:89], v[102:105], a[224:227]\n" \
  "v_mfma_f32_16x16x32_f16 a[228:231], v[86:89], v[106:109], a[228:231]\n" \
  "buffer_load_dwordx4 v[194:197], %[voffW], s[36:39], s67 offen nt\n" \
  "v_mfma_f32_16x16x32_f16 a[228:231], v[90:93], v[138:141], a[228:231]\n" \
  "s_cmp_eq_u32 s77, 13\n" \
  "s_cbranch_scc1 L_s90_%=\n" \
  "L_b90_%=:\n" \
  "v_mfma_f32_16x16x32_f16 a[196:199], v[98:101], v[138:141], a[196:199]\n" \
  "v_mfma_f32_16x16x32_f16 a[196:199], v[94:97], v[106:109], a[196:199]\n" \
  "v_mfma_f32_16x16x32_f16 a[200:203], v[94:97], v[110:113], a[200:203]\n" \
  "v_mfma_f32_16x16x32_f16 a[200:203], v[98:101], v[142:145], a[200:203]\n" \
  "s_waitcnt lgkmcnt(0)\n" \
  "v_mfma_f32_16x16x32_f16 a[232:235], v[90:93], v[142:145], a[232:235]\n" \
  "s_sub_u32 s82, s81, s58\n" \
  "s_branch L_s29_%=\n" \
  "L_b29_%=:\n" \
  "v_mfma_f32_16x16x32_f16 a[232:235], v[86:89], v[110:113], a[232:235]\n" \
  "s_max_i32 s82, s82, -150\n" \
  "s_add_u32 s81, s58, s82\n" \
  "s_add_u32 s81, s81, s87\n" \
  "v_mfma_f32_16x16x32_f16 a[236:239], v[86:89], v[114:117], a[236:239]\n" \
  "v_mfma_f32_16x16x32_f16 a[236:239], v[90:93], v[146:149], a[236:239]\n" \
  "v_mfma_f32_16x16x32_f16 a[204:207], v[98:101], v[146:149], a[204:207]\n" \
  "v_mfma_f32_16x16x32_f16 a[204:207], v[94:97], v[114:117], a[204:207]\n" \
  "v_mfma_f32_16x16x32_f16 a[208:211], v[94:97], v[118:121], a[208:211]\n" \
  "v_mfma_f32_16x16x32_f16 a[208:211], v[98:101], v[150:153], a[208:211]\n" \
  "v_mfma_f32_16x16x32_f16 a[240:243], v[90:93], v[150:153], a[240:243]\n" \
  "v_mfma_f32_16x16x32_f16 a[240:243], v[86:89], v[118:121], a[240:243]\n" \
  "v_readlane_b32 s74, %[vtabA], 8\n" \
  "v_mfma_f32_16x16x32_f16 a[244:247], v[86:89], v[122:125], a[244:247]\n" \
  "s_bitcmp1_b32 s34, 8\n" \
  "s_cbranch_scc0 L_s53_%=\n" \
  "L_b53_%=:\n" \
  "v_mfma_f32_16x16x32_f16 a[244:247], v[90:93], v[154:157], a[244:247]\n" \
  "s_waitcnt vmcnt(18)\n" \
  "s_add_u32 s72, s74, s68\n" \
  "v_mfma_f32_16x16x32_f16 a[212:215], v[98:101], v[154:157], a[212:215]\n" \
  "v_cvt_pk_f16_f32 v216, v198, v202\n" \
  "v_cvt_pk_f16_f32 v220, v199, v203\n" \
  "v_mfma_f32_16x16x32_f16 a[212:215], v[94:97], v[122:125], a[212:215]\n" \
  "s_waitcnt vmcnt(16)\n" \
  "v_cvt_pk_f16_f32 v224, v200, v204\n" \
  "v_cvt_pk_f16_f32 v228, v201, v205\n" \
  "v_mfma_f32_16x16x32_f16 a[216:219], v[94:97], v[126:129], a[216:219]\n" \
  "v_cvt_pk_f16_f32 v217, v206, v210\n" \
  "v_cvt_pk_f16_f32 v221, v207, v211\n" \
  "v_mfma_f32_16x16x32_f16 a[216:219], v[98:101], v[158:161], a[216:219]\n" \
  "v_cvt_pk_f16_f32 v225, v208, v212\n" \
  "v_cvt_pk_f16_f32 v229, v209, v213\n" \
  "v_mfma_f32_16x16x32_f16 a[248:251], v[90:93], v[158:161], a[248:251]\n" \
  "ds_write_b128 v242, v[214:217] offset:49152\n" \
  "v_mfma_f32_16x16x32_f16 a[248:251], v[86:89], v[126:129], a[248:251]\n" \
  "buffer_load_dwordx4 v[50:53], %[bxr], s[40:43], s72 offen offset:0 sc0 sc1\n" \
  "v_mfma_f32_16x16x32_f16 a[252:255], v[86:89], v[130:133], a[252:255]\n" \
  "v_readlane_b32 s75, %[vtabW], 13\n" \
  "v_mfma_f32_16x16x32_f16 a[252:255], v[90:93], v[162:165], a[252:255]\n" \
  "s_waitcnt vmcnt(11)\n" \
  "v_mfma_f32_16x16x32_f16 a[220:223], v[98:101], v[162:165], a[220:223]\n" \
  "ds_write_b128 v242, v[218:221] offset:50176\n" \
  "v_mfma_f32_16x16x32_f16 a[220:223], v[94:97], v[130:133], a[220:223]\n" \
  "buffer_load_dwordx4 v[46:49], %[bxr], s[40:43], s72 offen offset:1024 sc0 sc1\n" \
  "v_mfma_f32_16x16x32_f16 a[0:3], v[18:21], v[102:105], a[0:3]\n" \
  "s_add_u32 s60, s75, 0x0\n" \
  "s_add_u32 s61, s75, 0x10000\n" \
  "s_add_u32 s62, s75, 0x20000\n" \
  "v_mfma_f32_16x16x32_f16 a[0:3], v[14:17], v[134:137], a[0:3]\n" \
  "s_add_u32 s63, s75, 0x30000\n" \
  "s_add_u32 s64, s75, 0x100000\n" \
  "s_add_u32 s65, s75, 0x110000\n" \
  "v_mfma_f32_16x16x32_f16 a[32:35], v[26:29], v[134:137], a[32:35]\n" \
  "ds_write_b128 v242, v[222:225] offset:51200\n" \
  "v_mfma_f32_16x16x32_f16 a[32:35], v[22:25], v[102:105], a[32:35]\n" \
  "buffer_load_dwordx4 v[54:57], %[bxr], s[40:43], s72 offen offset:2048 sc0 sc1\n" \
  "v_mfma_f32_16x16x32_f16 a[36:39], v[22:25], v[106:109], a[36:39]\n" \
  "v_mfma_f32_16x16x32_f16 a[36:39], v[26:29], v[138:141], a[36:39]\n" \
  "s_add_u32 s73, s74, s69\n" \
  "v_mfma_f32_16x16x32_f16 a[4:7], v[14:17], v[138:141], a[4:7]\n" \
  "ds_write_b128 v242, v[226:229] offset:52224\n" \
  "v_mfma_f32_16x16x32_f16 a[4:7], v[18:21], v[106:109], a[4:7]\n" \
  "buffer_load_dwordx4 v[58:61], %[bxr], s[40:43], s72 offen offset:3072 sc0 sc1\n" \
  "v_mfma_f32_16x16x32_f16 a[8:11], v[18:21], v[110:113], a[8:11]\n" \
  "v_mfma_f32_16x16x32_f16 a[8:11], v[14:17], v[142:145], a[8:11]\n" \
  "v_mfma_f32_16x16x32_f16 a[40:43], v[26:29], v[142:145], a[40:43]\n" \
  "v_mfma_f32_16x16x32_f16 a[40:43], v[22:25], v[110:113], a[40:43]\n" \
  "v_cvt_pk_f16_f32 v214, v166, v170\n" \
  "v_cvt_pk_f16_f32 v218, v167, v171\n" \
  "v_mfma_f32_16x16x32_f16 a[44:47], v[22:25], v[114:117], a[44:47]\n" \
  "v_cvt_pk_f16_f32 v222, v168, v172\n" \
  "v_cvt_pk_f16_f32 v226, v169, v173\n" \
  "v_mfma_f32_16x16x32_f16 a[44:47], v[26:29], v[146:149], a[44:47]\n" \
  "buffer_load_dwordx4 v[66:69], %[bxr], s[40:43], s73 offen offset:0 sc0 sc1\n" \
  "v_mfma_f32_16x16x32_f16 a[12:15], v[14:17], v[146:149], a[12:15]\n" \
  "v_mfma_f32_16x16x32_f16 a[12:15], v[18:21], v[114:117], a[12:15]\n" \
  "v_mfma_f32_16x16x32_f16 a[16:19], v[18:21], v[118:121], a[16:19]\n" \
  "v_mfma_f32_16x16x32_f16 a[16:19], v[14:17], v[150:153], a[16:19]\n" \
  "buffer_load_dwordx4 v[62:65], %[bxr], s[40:43], s73 offen offset:1024 sc0 sc1\n" \
  "v_mfma_f32_16x16x32_f16 a[48:51], v[26:29], v[150:153], a[48:51]\n" \
  "v_mfma_f32_16x16x32_f16 a[48:51], v[22:25], v[118:121], a[48:51]\n" \
  "v_mfma_f32_16x16x32_f16 a[52:55], v[22:25], v[122:125], a[52:55]\n" \
  "v_mfma_f32_16x16x32_f16 a[52:55], v[26:29], v[154:157], a[52:55]\n" \
  "buffer_load_dwordx4 v[70:73], %[bxr], s[40:43], s73 offen offset:2048 sc0 sc1\n" \
  "v_mfma_f32_16x16x32_f16 a[20:23], v[14:17], v[154:157], a[20:23]\n" \
  "v_mfma_f32_16x16x32_f16 a[20:23], v[18:21], v[122:125], a[20:23]\n" \
  "s_andn2_b32 s79, 0xfff, s34\n" \
  "s_cbranch_scc1 L_pl8_%=\n" \
  "L_pb8_%=:\n" \
  "v_mfma_f32_16x16x32_f16 a[24:27], v[18:21], v[126:129], a[24:27]\n" \
  "s_barrier\n" \
  "v_mfma_f32_16x16x32_f16 a[24:27], v[14:17], v[158:161], a[24:27]\n" \
  "buffer_load_dwordx4 v[74:77], %[bxr], s[40:43], s73 offen offset:3072 sc0 sc1\n" \
  "v_mfma_f32_16x16x32_f16 a[56:59], v[26:29], v[158:161], a[56:59]\n" \
  "buffer_load_dwordx4 v[198:201], %[voffW], s[36:39], s60 offen nt\n" \
  "v_mfma_f32_16x16x32_f16 a[56:59], v[22:25], v[126:129], a[56:59]\n" \
  "v_mfma_f32_16x16x32_f16 a[60:63], v[22:25], v[130:133], a[60:63]\n" \
  "buffer_load_dwordx4 v[202:205], %[voffW], s[36:39], s61 offen nt\n" \
  "v_mfma_f32_16x16x32_f16 a[60:63], v[26:29], v[162:165], a[60:63]\n" \
  "v_mfma_f32_16x16x32_f16 a[28:31], v[14:17], v[162:165], a[28:31]\n" \
  "buffer_load_dwordx4 v[206:209], %[voffW], s[36:39], s62 offen nt\n" \
  "v_mfma_f32_16x16x32_f16 a[28:31], v[18:21], v[130:133], a[28:31]\n" \
  "s_waitcnt vmcnt(16)\n" \
  "v_mfma_f32_16x16x32_f16 a[128:131], v[34:37], v[102:105], a[128:131]\n" \
  "buffer_load_dwordx4 v[210:213], %[voffW], s[36:39], s63 offen nt\n" \
  "v_mfma_f32_16x16x32_f16 a[128:131], v[30:33], v[134:137], a[128:131]\n" \
  "v_mfma_f32_16x16x32_f16 a[160:163], v[42:45], v[134:137], a[160:163]\n" \
  "ds_read_b128 v[134:137], v245 offset:8192\n" \
  "v_mfma_f32_16x16x32_f16 a[160:163], v[38:41], v[102:105], a[160:163]\n" \
  "ds_read_b128 v[102:105], v245 offset:0\n" \
  "v_mfma_f32_16x16x32_f16 a[164:167], v[38:41], v[106:109], a[164:167]\n" \
  "buffer_load_dwordx4 v[166:169], %[voffW], s[36:39], s64 offen nt\n" \
  "v_mfma_f32_16x16x32_f16 a[164:167], v[42:45], v[138:141], a[164:167]\n" \
  "v_mfma_f32_16x16x32_f16 a[132:135], v[30:33], v[138:141], a[132:135]\n" \
  "ds_read_b128 v[138:141], v245 offset:9216\n" \
  "v_mfma_f32_16x16x32_f16 a[132:135], v[34:37], v[106:109], a[132:135]\n" \
  "ds_read_b128 v[106:109], v245 offset:1024\n" \
  "v_mfma_f32_16x16x32_f16 a[136:139], v[34:37], v[110:113], a[136:139]\n" \
  "buffer_load_dwordx4 v[170:173], %[voffW], s[36:39], s65 offen nt\n" \
  "v_mfma_f32_16x16x32_f16 a[136:139], v[30:33], v[142:145], a[136:139]\n" \
  "v_mfma_f32_16x16x32_f16 a[168:171], v[42:45], v[142:145], a[168:171]\n" \
  "ds_read_b128 v[142:145], v245 offset:10240\n" \
  "v_mfma_f32_16x16x32_f16 a[168:171], v[38:41], v[110:113], a[168:171]\n" \
  "ds_read_b128 v[110:113], v245 offset:2048\n" \
  "v_mfma_f32_16x16x32_f16 a[172:175], v[38:41], v[114:117], a[172:175]\n" \
  "v_mfma_f32_16x16x32_f16 a[172:175], v[42:45], v[146:149], a[172:175]\n" \
  "v_mfma_f32_16x16x32_f16 a[140:143], v[30:33], v[146:149], a[140:143]\n" \
  "ds_read_b128 v[146:149], v245 offset:11264\n" \
  "v_mfma_f32_16x16x32_f16 a[140:143], v[34:37], v[114:117], a[140:143]\n" \
  "ds_read_b128 v[114:117], v245 offset:3072\n" \
  "v_mfma_f32_16x16x32_f16 a[144:147], v[34:37], v[118:121], a[144:147]\n" \
  "s_add_u32 s72, s74, s70\n" \
  "v_mfma_f32_16x16x32_f16 a[144:147], v[30:33], v[150:153], a[144:147]\n" \
  "v_mfma_f32_16x16x32_f16 a[176:179], v[42:45], v[150:153], a[176:179]\n" \
  "ds_read_b128 v[150:153], v245 offset:12288\n" \
  "v_mfma_f32_16x16x32_f16 a[176:179], v[38:41], v[118:121], a[176:179]\n" \
  "ds_read_b128 v[118:121], v245 offset:4096\n" \
  "v_mfma_f32_16x16x32_f16 a[180:183], v[38:41], v[122:125], a[180:183]\n" \
  "v_mfma_f32_16x16x32_f16 a[180:183], v[42:45], v[154:157], a[180:183]\n" \
  "v_mfma_f32_16x16x32_f16 a[148:151], v[30:33], v[154:157], a[148:151]\n" \
  "ds_read_b128 v[154:157], v245 offset:13312\n" \
  "v_mfma_f32_16x16x32_f16 a[148:151], v[34:37], v[122:125], a[148:151]\n" \
  "ds_read_b128 v[122:125], v245 offset:5120\n" \
  "v_mfma_f32_16x16x32_f16 a[152:155], v[34:37], v[126:129], a[152:155]\n" \
  "buffer_load_dwordx4 v[82:85], %[bxr], s[40:43], s72 offen offset:0 sc0 sc1\n" \
  "v_mfma_f32_16x16x32_f16 a[152:155], v[30:33], v[158:161], a[152:155]\n" \
  "v_mfma_f32_16x16x32_f16 a[184:187], v[42:45], v[158:161], a[184:187]\n" \
  "ds_read_b128 v[158:161], v245 offset:14336\n" \
  "v_mfma_f32_16x16x32_f16 a[184:187], v[38:41], v[126:129], a[184:187]\n" \
  "ds_read_b128 v[126:129], v245 offset:6144\n" \
  "v_mfma_f32_16x16x32_f16 a[188:191], v[38:41], v[130:133], a[188:191]\n" \
  "buffer_load_dwordx4 v[78:81], %[bxr], s[40:43], s72 offen offset:1024 sc0 sc1\n" \
  "v_mfma_f32_16x16x32_f16 a[188:191], v[42:45], v[162:165], a[188:191]\n" \
  "s_add_u32 s66, s75, 0x120000\n" \
  "s_add_u32 s67, s75, 0x130000\n" \
  "v_mfma_f32_16x16x32_f16 a[156:159], v[30:33], v[162:165], a[156:159]\n" \
  "ds_read_b128 v[162:165], v245 offset:15360\n" \
  "v_readlane_b32 s75, %[vtabW], 14\n" \
  "v_mfma_f32_16x16x32_f16 a[156:159], v[34:37], v[130:133], a[156:159]\n" \
  "s_waitcnt vmcnt(12)\n" \
  "ds_read_b128 v[130:133], v245 offset:7168\n" \
  "s_waitcnt lgkmcnt(14)\n" \
    \
  "v_mfma_f32_16x16x32_f16 a[64:67], v[50:53], v[102:105], a[64:67]\n" \
  "buffer_load_dwordx4 v[86:89], %[bxr], s[40:43], s72 offen offset:2048 sc0 sc1\n" \
  "v_mfma_f32_16x16x32_f16 a[64:67], v[46:49], v[134:137], a[64:67]\n" \
  "s_add_u32 s60, s75, 0x0\n" \
  "s_add_u32 s61, s75, 0x10000\n" \
  "s_add_u32 s62, s75, 0x20000\n" \
  "v_mfma_f32_16x16x32_f16 a[96:99], v[58:61], v[134:137], a[96:99]\n" \
  "s_add_u32 s63, s75, 0x30000\n" \
  "s_add_u32 s73, s74, s71\n" \
  "v_mfma_f32_16x16x32_f16 a[96:99], v[54:57], v[102:105], a[96:99]\n" \
  "s_waitcnt lgkmcnt(12)\n" \
  "v_mfma_f32_16x16x32_f16 a[100:103], v[54:57], v[106:109], a[100:103]\n" \
  "buffer_load_dwordx4 v[90:93], %[bxr], s[40:43], s72 offen offset:3072 sc0 sc1\n" \
  "v_mfma_f32_16x16x32_f16 a[100:103], v[58:61], v[138:141], a[100:103]\n" \
  "v_mfma_f32_16x16x32_f16 a[68:71], v[46:49], v[138:141], a[68:71]\n" \
  "v_mfma_f32_16x16x32_f16 a[68:71], v[50:53], v[106:109], a[68:71]\n" \
  "s_waitcnt lgkmcnt(10)\n" \
  "v_mfma_f32_16x16x32_f16 a[72:75], v[50:53], v[110:113], a[72:75]\n" \
  "buffer_load_dwordx4 v[98:101], %[bxr], s[40:43], s73 offen offset:0 sc0 sc1\n" \
  "v_mfma_f32_16x16x32_f16 a[72:75], v[46:49], v[142:145], a[72:75]\n" \
  "v_mfma_f32_16x16x32_f16 a[104:107], v[58:61], v[142:145], a[104:107]\n" \
  "v_mfma_f32_16x16x32_f16 a[104:107], v[54:57], v[110:113], a[104:107]\n" \
  "s_waitcnt lgkmcnt(8)\n" \
  "v_cvt_pk_f16_f32 v215, v174, v178\n" \
  "v_cvt_pk_f16_f32 v219, v175, v179\n" \
  "v_mfma_f32_16x16x32_f16 a[108:111], v[54:57], v[114:117], a[108:111]\n" \
  "v_cvt_pk_f16_f32 v223, v176, v180\n" \
  "v_cvt_pk_f16_f32 v227, v177, v181\n" \
  "v_mfma_f32_16x16x32_f16 a[108:111], v[58:61], v[146:149], a[108:111]\n" \
  "s_waitcnt vmcnt(12)\n" \
  "v_cvt_pk_f16_f32 v216, v182, v186\n" \
  "v_cvt_pk_f16_f32 v220, v183, v187\n" \
  "v_mfma_f32_16x16x32_f16 a[76:79], v[46:49], v[146:149], a[76:79]\n" \
  "v_cvt_pk_f16_f32 v224, v184, v188\n" \
  "v_cvt_pk_f16_f32 v228, v185, v189\n" \
  "v_mfma_f32_16x16x32_f16 a[76:79], v[50:53], v[114:117], a[76:79]\n" \
  "s_waitcnt lgkmcnt(6)\n" \
  "v_cvt_pk_f16_f32 v217, v190, v194\n" \
  "v_cvt_pk_f16_f32 v221, v191, v195\n" \
  "v_mfma_f32_16x16x32_f16 a[80:83], v[50:53], v[118:121], a[80:83]\n" \
  "v_cvt_pk_f16_f32 v225, v192, v196\n" \
  "v_cvt_pk_f16_f32 v229, v193, v197\n" \
  "v_mfma_f32_16x16x32_f16 a[80:83], v[46:49], v[150:153], a[80:83]\n" \
  "ds_write_b128 v243, v[214:217] offset:0\n" \
  "v_mfma_f32_16x16x32_f16 a[112:115], v[58:61], v[150:153], a[112:115]\n" \
  "buffer_load_dwordx4 v[94:97], %[bxr], s[40:43], s73 offen offset:1024 sc0 sc1\n" \
  "v_mfma_f32_16x16x32_f16 a[112:115], v[54:57], v[118:121], a[112:115]\n" \
  "s_waitcnt lgkmcnt(5)\n" \
  "v_min3_u32 v248, v238, v239, v240\n" \
  "v_min_u32 v248, v248, v241\n" \
  "v_mfma_f32_16x16x32_f16 a[116:119], v[54:57], v[122:125], a[116:119]\n" \
  "v_cmp_le_u32 s[34:35], s76, v248\n" \
  "v_mfma_f32_16x16x32_f16 a[116:119], v[58:61], v[154:157], a[116:119]\n" \
  "ds_write_b128 v243, v[218:221] offset:1024\n" \
  "v_mfma_f32_16x16x32_f16 a[84:87], v[46:49], v[154:157], a[84:87]\n" \
  "buffer_load_dwordx4 v[22:25], %[bxr], s[40:43], s73 offen offset:2048 sc0 sc1\n" \
  "v_mfma_f32_16x16x32_f16 a[84:87], v[50:53], v[122:125], a[84:87]\n" \
  "s_waitcnt lgkmcnt(4)\n" \
  "v_mfma_f32_16x16x32_f16 a[88:91], v[50:53], v[126:129], a[88:91]\n" \
  "s_waitcnt lgkmcnt(2)\n" \
  "s_barrier\n" \
  "v_mfma_f32_16x16x32_f16 a[88:91], v[46:49], v[158:161], a[88:91]\n" \
  "ds_write_b128 v243, v[222:225] offset:2048\n" \
  "v_mfma_f32_16x16x32_f16 a[120:123], v[58:61], v[158:161], a[120:123]\n" \
  "buffer_load_dwordx4 v[174:177], %[voffW], s[36:39], s66 offen nt\n" \
  "v_mfma_f32_16x16x32_f16 a[120:123], v[54:57], v[126:129], a[120:123]\n" \
  "buffer_load_dwordx4 v[26:29], %[bxr], s[40:43], s73 offen offset:3072 sc0 sc1\n" \
  "v_mfma_f32_16x16x32_f16 a[124:127], v[54:57], v[130:133], a[124:127]\n" \
  "buffer_load_dwordx4 v[178:181], %[voffW], s[36:39], s67 offen nt\n" \
  "v_mfma_f32_16x16x32_f16 a[124:127], v[58:61], v[162:165], a[124:127]\n" \
  "ds_write_b128 v243, v[226:229] offset:3072\n" \
  "v_mfma_f32_16x16x32_f16 a[92:95], v[46:49], v[162:165], a[92:95]\n" \
  "buffer_load_dwordx4 v[182:185], %[voffW], s[36:39], s60 offen nt\n" \
  "v_mfma_f32_16x16x32_f16 a[92:95], v[50:53], v[130:133], a[92:95]\n" \
  "s_waitcnt vmcnt(17)\n" \
  "v_mfma_f32_16x16x32_f16 a[192:195], v[66:69], v[102:105], a[192:195]\n" \
  "buffer_load_dwordx4 v[186:189], %[voffW], s[36:39], s61 offen nt\n" \
  "v_mfma_f32_16x16x32_f16 a[192:195], v[62:65], v[134:137], a[192:195]\n" \
  "s_memrealtime s[58:59]\n" \
  "v_mfma_f32_16x16x32_f16 a[224:227], v[74:77], v[134:137], a[224:227]\n" \
  "buffer_load_dwordx4 v[190:193], %[voffW], s[36:39], s62 offen nt\n" \
  "v_mfma_f32_16x16x32_f16 a[224:227], v[70:73], v[102:105], a[224:227]\n" \
  "v_mfma_f32_16x16x32_f16 a[228:231], v[70:73], v[106:109], a[228:231]\n" \
  "buffer_load_dwordx4 v[194:197], %[voffW], s[36:39], s63 offen nt\n" \
  "v_mfma_f32_16x16x32_f16 a[228:231], v[74:77], v[138:141], a[228:231]\n" \
  "v_mfma_f32_16x16x32_f16 a[196:199], v[62:65], v[138:141], a[196:199]\n" \
  "v_mfma_f32_16x16x32_f16 a[196:199], v[66:69], v[106:109], a[196:199]\n" \
  "v_mfma_f32_16x16x32_f16 a[200:203], v[66:69], v[110:113], a[200:203]\n" \
  "v_mfma_f32_16x16x32_f16 a[200:203], v[62:65], v[142:145], a[200:203]\n" \
  "s_waitcnt lgkmcnt(0)\n" \
  "v_mfma_f32_16x16x32_f16 a[232:235], v[74:77], v[142:145], a[232:235]\n" \
  "s_sub_u32 s82, s81, s58\n" \
  "s_branch L_s30_%=\n" \
  "L_b30_%=:\n" \
  "v_mfma_f32_16x16x32_f16 a[232:235], v[70:73], v[110:113], a[232:235]\n" \
  "s_max_i32 s82, s82, -150\n" \
  "s_add_u32 s81, s58, s82\n" \
  "s_add_u32 s81, s81, s87\n" \
  "v_mfma_f32_16x16x32_f16 a[236:239], v[70:73], v[114:117], a[236:239]\n" \
  "v_mfma_f32_16x16x32_f16 a[236:239], v[74:77], v[146:149], a[236:239]\n" \
  "v_mfma_f32_16x16x32_f16 a[204:207], v[62:65], v[146:149], a[204:207]\n" \
  "v_readlane_b32 s74, %[vtabA], 9\n" \
  "v_mfma_f32_16x16x32_f16 a[204:207], v[66:69], v[114:117], a[204:207]\n" \
  "s_bitcmp1_b32 s34, 9\n" \
  "s_cbranch_scc0 L_s54_%=\n" \
  "L_b54_%=:\n" \
  "v_mfma_f32_16x16x32_f16 a[208:211], v[66:69], v[118:121], a[208:211]\n" \
  "s_add_u32 s72, s74, s68\n" \
  "v_mfma_f32_16x16x32_f16 a[208:211], v[62:65], v[150:153], a[208:211]\n" \
  "v_mfma_f32_16x16x32_f16 a[240:243], v[74:77], v[150:153], a[240:243]\n" \
  "v_mfma_f32_16x16x32_f16 a[240:243], v[70:73], v[118:121], a[240:243]\n" \
  "v_mfma_f32_16x16x32_f16 a[244:247], v[70:73], v[122:125], a[244:247]\n" \
  "v_mfma_f32_16x16x32_f16 a[244:247], v[74:77], v[154:157], a[244:247]\n" \
  "v_mfma_f32_16x16x32_f16 a[212:215], v[62:65], v[154:157], a[212:215]\n" \
  "buffer_load_dwordx4 v[14:17], %[bxr], s[40:43], s72 offen offset:0 sc0 sc1\n" \
  "v_mfma_f32_16x16x32_f16 a[212:215], v[66:69], v[122:125], a[212:215]\n" \
  "v_mfma_f32_16x16x32_f16 a[216:219], v[66:69], v[126:129], a[216:219]\n" \
  "v_mfma_f32_16x16x32_f16 a[216:219], v[62:65], v[158:161], a[216:219]\n" \
  "v_mfma_f32_16x16x32_f16 a[248:251], v[74:77], v[158:161], a[248:251]\n" \
  "buffer_load_dwordx4 v[18:21], %[bxr], s[40:43], s72 offen offset:1024 sc0 sc1\n" \
  "v_mfma_f32_16x16x32_f16 a[248:251], v[70:73], v[126:129], a[248:251]\n" \
  "v_mfma_f32_16x16x32_f16 a[252:255], v[70:73], v[130:133], a[252:255]\n" \
  "s_add_u32 s64, s75, 0x100000\n" \
  "s_add_u32 s65, s75, 0x110000\n" \
  "s_add_u32 s66, s75, 0x120000\n" \
  "v_mfma_f32_16x16x32_f16 a[252:255], v[74:77], v[162:165], a[252:255]\n" \
  "s_add_u32 s67, s75, 0x130000\n" \
  "v_mfma_f32_16x16x32_f16 a[220:223], v[62:65], v[162:165], a[220:223]\n" \
  "buffer_load_dwordx4 v[38:41], %[bxr], s[40:43], s72 offen offset:2048 sc0 sc1\n" \
  "v_mfma_f32_16x16x32_f16 a[220:223], v[66:69], v[130:133], a[220:223]\n" \
  "s_waitcnt vmcnt(13)\n" \
  "v_readlane_b32 s75, %[vtabW], 15\n" \
  "s_add_u32 s73, s74, s69\n" \
  "v_mfma_f32_16x16x32_f16 a[0:3], v[82:85], v[102:105], a[0:3]\n" \
  "v_mfma_f32_16x16x32_f16 a[0:3], v[78:81], v[134:137], a[0:3]\n" \
  "s_add_u32 s60, s75, 0x0\n" \
  "s_add_u32 s61, s75, 0x10000\n" \
  "v_mfma_f32_16x16x32_f16 a[32:35], v[90:93], v[134:137], a[32:35]\n" \
  "buffer_load_dwordx4 v[42:45], %[bxr], s[40:43], s72 offen offset:3072 sc0 sc1\n" \
  "v_mfma_f32_16x16x32_f16 a[32:35], v[86:89], v[102:105], a[32:35]\n" \
  "v_mfma_f32_16x16x32_f16 a[36:39], v[86:89], v[106:109], a[36:39]\n" \
  "v_mfma_f32_16x16x32_f16 a[36:39], v[90:93], v[138:141], a[36:39]\n" \
  "v_mfma_f32_16x16x32_f16 a[4:7], v[78:81], v[138:141], a[4:7]\n" \
  "buffer_load_dwordx4 v[30:33], %[bxr], s[40:43], s73 offen offset:0 sc0 sc1\n" \
  "v_mfma_f32_16x16x32_f16 a[4:7], v[82:85], v[106:109], a[4:7]\n" \
  "v_mfma_f32_16x16x32_f16 a[8:11], v[82:85], v[110:113], a[8:11]\n" \
  "v_mfma_f32_16x16x32_f16 a[8:11], v[78:81], v[142:145], a[8:11]\n" \
  "v_mfma_f32_16x16x32_f16 a[40:43], v[90:93], v[142:145], a[40:43]\n" \
  "v_mfma_f32_16x16x32_f16 a[40:43], v[86:89], v[110:113], a[40:43]\n" \
  "v_cvt_pk_f16_f32 v214, v198, v202\n" \
  "v_cvt_pk_f16_f32 v218, v199, v203\n" \
  "v_mfma_f32_16x16x32_f16 a[44:47], v[86:89], v[114:117], a[44:47]\n" \
  "v_cvt_pk_f16_f32 v222, v200, v204\n" \
  "v_cvt_pk_f16_f32 v226, v201, v205\n" \
  "v_mfma_f32_16x16x32_f16 a[44:47], v[90:93], v[146:149], a[44:47]\n" \
  "v_cvt_pk_f16_f32 v215, v206, v210\n" \
  "v_cvt_pk_f16_f32 v219, v207, v211\n" \
  "v_mfma_f32_16x16x32_f16 a[12:15], v[78:81], v[146:149], a[12:15]\n" \
  "v_cvt_pk_f16_f32 v223, v208, v212\n" \
  "v_cvt_pk_f16_f32 v227, v209, v213\n" \
  "v_mfma_f32_16x16x32_f16 a[12:15], v[82:85], v[114:117], a[12:15]\n" \
  "v_cvt_pk_f16_f32 v216, v166, v170\n" \
  "v_cvt_pk_f16_f32 v220, v167, v171\n" \
  "v_mfma_f32_16x16x32_f16 a[16:19], v[82:85], v[118:121], a[16:19]\n" \
  "v_cvt_pk_f16_f32 v224, v168, v172\n" \
  "v_cvt_pk_f16_f32 v228, v169, v173\n" \
  "v_mfma_f32_16x16x32_f16 a[16:19], v[78:81], v[150:153], a[16:19]\n" \
  "buffer_load_dwordx4 v[34:37], %[bxr], s[40:43], s73 offen offset:1024 sc0 sc1\n" \
  "v_mfma_f32_16x16x32_f16 a[48:51], v[90:93], v[150:153], a[48:51]\n" \
  "v_mfma_f32_16x16x32_f16 a[48:51], v[86:89], v[118:121], a[48:51]\n" \
  "v_mfma_f32_16x16x32_f16 a[52:55], v[86:89], v[122:125], a[52:55]\n" \
  "v_mfma_f32_16x16x32_f16 a[52:55], v[90:93], v[154:157], a[52:55]\n" \
  "buffer_load_dwordx4 v[54:57], %[bxr], s[40:43], s73 offen offset:2048 sc0 sc1\n" \
  "v_mfma_f32_16x16x32_f16 a[20:23], v[78:81], v[154:157], a[20:23]\n" \
  "v_mfma_f32_16x16x32_f16 a[20:23], v[82:85], v[122:125], a[20:23]\n" \
  "s_andn2_b32 s79, 0x1fff, s34\n" \
  "s_cbranch_scc1 L_pl9_%=\n" \
  "L_pb9_%=:\n" \
  "v_mfma_f32_16x16x32_f16 a[24:27], v[82:85], v[126:129], a[24:27]\n" \
  "s_barrier\n" \
  "v_mfma_f32_16x16x32_f16 a[24:27], v[78:81], v[158:161], a[24:27]\n" \
  "buffer_load_dwordx4 v[58:61], %[bxr], s[40:43], s73 offen offset:3072 sc0 sc1\n" \
  "v_mfma_f32_16x16x32_f16 a[56:59], v[90:93], v[158:161], a[56:59]\n" \
  "buffer_load_dwordx4 v[198:201], %[voffW], s[36:39], s64 offen nt\n" \
  "v_mfma_f32_16x16x32_f16 a[56:59], v[86:89], v[126:129], a[56:59]\n" \
  "v_mfma_f32_16x16x32_f16 a[60:63], v[86:89], v[130:133], a[60:63]\n" \
  "buffer_load_dwordx4 v[202:205], %[voffW], s[36:39], s65 offen nt\n" \
  "v_mfma_f32_16x16x32_f16 a[60:63], v[90:93], v[162:165], a[60:63]\n" \
  "v_mfma_f32_16x16x32_f16 a[28:31], v[78:81], v[162:165], a[28:31]\n" \
  "buffer_load_dwordx4 v[206:209], %[voffW], s[36:39], s66 offen nt\n" \
  "v_mfma_f32_16x16x32_f16 a[28:31], v[82:85], v[130:133], a[28:31]\n" \
  "s_waitcnt vmcnt(16)\n" \
  "v_mfma_f32_16x16x32_f16 a[128:131], v[98:101], v[102:105], a[128:131]\n" \
  "buffer_load_dwordx4 v[210:213], %[voffW], s[36:39], s67 offen nt\n" \
  "v_mfma_f32_16x16x32_f16 a[128:131], v[94:97], v[134:137], a[128:131]\n" \
  "v_mfma_f32_16x16x32_f16 a[160:163], v[26:29], v[134:137], a[160:163]\n" \
  "ds_read_b128 v[134:137], v245 offset:24576\n" \
  "v_mfma_f32_16x16x32_f16 a[160:163], v[22:25], v[102:105], a[160:163]\n" \
  "ds_read_b128 v[102:105], v245 offset:16384\n" \
  "v_mfma_f32_16x16x32_f16 a[164:167], v[22:25], v[106:109], a[164:167]\n" \
  "buffer_load_dwordx4 v[166:169], %[voffW], s[36:39], s60 offen nt\n" \
  "v_mfma_f32_16x16x32_f16 a[164:167], v[26:29], v[138:141], a[164:167]\n" \
  "v_mfma_f32_16x16x32_f16 a[132:135], v[94:97], v[138:141], a[132:135]\n" \
  "ds_read_b128 v[138:141], v245 offset:25600\n" \
  "v_mfma_f32_16x16x32_f16 a[132:135], v[98:101], v[106:109], a[132:135]\n" \
  "ds_read_b128 v[106:109], v245 offset:17408\n" \
  "v_mfma_f32_16x16x32_f16 a[136:139], v[98:101], v[110:113], a[136:139]\n" \
  "buffer_load_dwordx4 v[170:173], %[voffW], s[36:39], s61 offen nt\n" \
  "v_mfma_f32_16x16x32_f16 a[136:139], v[94:97], v[142:145], a[136:139]\n" \
  "v_mfma_f32_16x16x32_f16 a[168:171], v[26:29], v[142:145], a[168:171]\n" \
  "ds_read_b128 v[142:145], v245 offset:26624\n" \
  "v_mfma_f32_16x16x32_f16 a[168:171], v[22:25], v[110:113], a[168:171]\n" \
  "ds_read_b128 v[110:113], v245 offset:18432\n" \
  "v_mfma_f32_16x16x32_f16 a[172:175], v[22:25], v[114:117], a[172:175]\n" \
  "v_mfma_f32_16x16x32_f16 a[172:175], v[26:29], v[146:149], a[172:175]\n" \
  "v_mfma_f32_16x16x32_f16 a[140:143], v[94:97], v[146:149], a[140:143]\n" \
  "ds_read_b128 v[146:149], v245 offset:27648\n" \
  "v_mfma_f32_16x16x32_f16 a[140:143], v[98:101], v[114:117], a[140:143]\n" \
  "ds_read_b128 v[114:117], v245 offset:19456\n" \
  "v_mfma_f32_16x16x32_f16 a[144:147], v[98:101], v[118:121], a[144:147]\n" \
  "v_mfma_f32_16x16x32_f16 a[144:147], v[94:97], v[150:153], a[144:147]\n" \
  "v_mfma_f32_16x16x32_f16 a[176:179], v[26:29], v[150:153], a[176:179]\n" \
  "ds_read_b128 v[150:153], v245 offset:28672\n" \
  "v_mfma_f32_16x16x32_f16 a[176:179], v[22:25], v[118:121], a[176:179]\n" \
  "ds_read_b128 v[118:121], v245 offset:20480\n" \
  "v_mfma_f32_16x16x32_f16 a[180:183], v[22:25], v[122:125], a[180:183]\n" \
  "s_add_u32 s72, s74, s70\n" \
  "v_mfma_f32_16x16x32_f16 a[180:183], v[26:29], v[154:157], a[180:183]\n" \
  "s_waitcnt vmcnt(18)\n" \
  "v_mfma_f32_16x16x32_f16 a[148:151], v[94:97], v[154:157], a[148:151]\n" \
  "ds_read_b128 v[154:157], v245 offset:29696\n" \
  "v_cvt_pk_f16_f32 v217, v174, v178\n" \
  "v_mfma_f32_16x16x32_f16 a[148:151], v[98:101], v[122:125], a[148:151]\n" \
  "ds_read_b128 v[122:125], v245 offset:21504\n" \
  "v_cvt_pk_f16_f32 v221, v175, v179\n" \
  "v_mfma_f32_16x16x32_f16 a[152:155], v[98:101], v[126:129], a[152:155]\n" \
  "v_cvt_pk_f16_f32 v225, v176, v180\n" \
  "v_cvt_pk_f16_f32 v229, v177, v181\n" \
  "v_mfma_f32_16x16x32_f16 a[152:155], v[94:97], v[158:161], a[152:155]\n" \
  "ds_write_b128 v243, v[214:217] offset:16384\n" \
  "v_mfma_f32_16x16x32_f16 a[184:187], v[26:29], v[158:161], a[184:187]\n" \
  "ds_read_b128 v[158:161], v245 offset:30720\n" \
  "v_mfma_f32_16x16x32_f16 a[184:187], v[22:25], v[126:129], a[184:187]\n" \
  "ds_read_b128 v[126:129], v245 offset:22528\n" \
  "v_mfma_f32_16x16x32_f16 a[188:191], v[22:25], v[130:133], a[188:191]\n" \
  "buffer_load_dwordx4 v[46:49], %[bxr], s[40:43], s72 offen offset:0 sc0 sc1\n" \
  "v_mfma_f32_16x16x32_f16 a[188:191], v[26:29], v[162:165], a[188:191]\n" \
  "ds_write_b128 v243, v[218:221] offset:17408\n" \
  "v_mfma_f32_16x16x32_f16 a[156:159], v[94:97], v[162:165], a[156:159]\n" \
  "s_add_u32 s62, s75, 0x20000\n" \
  "ds_read_b128 v[162:165], v245 offset:31744\n" \
  "s_add_u32 s63, s75, 0x30000\n" \
  "v_mfma_f32_16x16x32_f16 a[156:159], v[98:101], v[130:133], a[156:159]\n" \
  "s_waitcnt vmcnt(11)\n" \
  "ds_read_b128 v[130:133], v245 offset:23552\n" \
  "s_waitcnt lgkmcnt(14)\n" \
    \
  "v_mfma_f32_16x16x32_f16 a[64:67], v[14:17], v[102:105], a[64:67]\n" \
  "buffer_load_dwordx4 v[50:53], %[bxr], s[40:43], s72 offen offset:1024 sc0 sc1\n" \
  "v_mfma_f32_16x16x32_f16 a[64:67], v[18:21], v[134:137], a[64:67]\n" \
  "ds_write_b128 v243, v[222:225] offset:18432\n" \
  "v_mfma_f32_16x16x32_f16 a[96:99], v[42:45], v[134:137], a[96:99]\n" \
  "s_add_u32 s64, s75, 0x100000\n" \
  "s_add_u32 s65, s75, 0x110000\n" \
  "s_add_u32 s66, s75, 0x120000\n" \
  "v_mfma_f32_16x16x32_f16 a[96:99], v[38:41], v[102:105], a[96:99]\n" \
  "s_waitcnt lgkmcnt(14)\n" \
  "s_add_u32 s67, s75, 0x130000\n" \
  "v_mfma_f32_16x16x32_f16 a[100:103], v[38:41], v[106:109], a[100:103]\n" \
  "buffer_load_dwordx4 v[70:73], %[bxr], s[40:43], s72 offen offset:2048 sc0 sc1\n" \
  "v_mfma_f32_16x16x32_f16 a[100:103], v[42:45], v[138:141], a[100:103]\n" \
  "ds_write_b128 v243, v[226:229] offset:19456\n" \
  "v_mfma_f32_16x16x32_f16 a[68:71], v[18:21], v[138:141], a[68:71]\n" \
  "s_add_u32 s73, s74, s71\n" \
  "v_mfma_f32_16x16x32_f16 a[68:71], v[14:17], v[106:109], a[68:71]\n" \
  "s_waitcnt lgkmcnt(14)\n" \
  "v_mfma_f32_16x16x32_f16 a[72:75], v[14:17], v[110:113], a[72:75]\n" \
  "buffer_load_dwordx4 v[74:77], %[bxr], s[40:43], s72 offen offset:3072 sc0 sc1\n" \
  "v_mfma_f32_16x16x32_f16 a[72:75], v[18:21], v[142:145], a[72:75]\n" \
  "v_mfma_f32_16x16x32_f16 a[104:107], v[42:45], v[142:145], a[104:107]\n" \
  "s_waitcnt lgkmcnt(12)\n" \
  "v_cvt_pk_f16_f32 v214, v182, v186\n" \
  "v_cvt_pk_f16_f32 v218, v183, v187\n" \
  "v_mfma_f32_16x16x32_f16 a[104:107], v[38:41], v[110:113], a[104:107]\n" \
  "v_cvt_pk_f16_f32 v222, v184, v188\n" \
  "v_cvt_pk_f16_f32 v226, v185, v189\n" \
  "v_mfma_f32_16x16x32_f16 a[108:111], v[38:41], v[114:117], a[108:111]\n" \
  "v_cvt_pk_f16_f32 v215, v190, v194\n" \
  "v_cvt_pk_f16_f32 v219, v191, v195\n" \
  "v_mfma_f32_16x16x32_f16 a[108:111], v[42:45], v[146:149], a[108:111]\n" \
  "s_waitcnt vmcnt(11)\n" \
  "v_cvt_pk_f16_f32 v223, v192, v196\n" \
  "v_cvt_pk_f16_f32 v227, v193, v197\n" \
  "v_mfma_f32_16x16x32_f16 a[76:79], v[18:21], v[146:149], a[76:79]\n" \
  "buffer_load_dwordx4 v[62:65], %[bxr], s[40:43], s73 offen offset:0 sc0 sc1\n" \
  "v_mfma_f32_16x16x32_f16 a[76:79], v[14:17], v[114:117], a[76:79]\n" \
  "s_waitcnt lgkmcnt(10)\n" \
  "v_min3_u32 v248, v238, v239, v240\n" \
  "v_min_u32 v248, v248, v241\n" \
  "v_mfma_f32_16x16x32_f16 a[80:83], v[14:17], v[118:121], a[80:83]\n" \
  "v_cmp_le_u32 s[34:35], s76, v248\n" \
  "v_mfma_f32_16x16x32_f16 a[80:83], v[18:21], v[150:153], a[80:83]\n" \
  "v_mfma_f32_16x16x32_f16 a[112:115], v[42:45], v[150:153], a[112:115]\n" \
  "buffer_load_dwordx4 v[66:69], %[bxr], s[40:43], s73 offen offset:1024 sc0 sc1\n" \
  "v_mfma_f32_16x16x32_f16 a[112:115], v[38:41], v[118:121], a[112:115]\n" \
  "s_waitcnt lgkmcnt(8)\n" \
  "v_mfma_f32_16x16x32_f16 a[116:119], v[38:41], v[122:125], a[116:119]\n" \
  "v_mfma_f32_16x16x32_f16 a[116:119], v[42:45], v[154:157], a[116:119]\n" \
  "v_mfma_f32_16x16x32_f16 a[84:87], v[18:21], v[154:157], a[84:87]\n" \
  "buffer_load_dwordx4 v[86:89], %[bxr], s[40:43], s73 offen offset:2048 sc0 sc1\n" \
  "v_mfma_f32_16x16x32_f16 a[84:87], v[14:17], v[122:125], a[84:87]\n" \
  "s_waitcnt lgkmcnt(5)\n" \
  "v_mfma_f32_16x16x32_f16 a[88:91], v[14:17], v[126:129], a[88:91]\n" \
  "v_mfma_f32_16x16x32_f16 a[88:91], v[18:21], v[158:161], a[88:91]\n" \
  "s_waitcnt lgkmcnt(2)\n" \
  "s_barrier\n" \
  "v_mfma_f32_16x16x32_f16 a[120:123], v[42:45], v[158:161], a[120:123]\n" \
  "buffer_load_dwordx4 v[174:177], %[voffW], s[36:39], s62 offen nt\n" \
  "v_mfma_f32_16x16x32_f16 a[120:123], v[38:41], v[126:129], a[120:123]\n" \
  "buffer_load_dwordx4 v[90:93], %[bxr], s[40:43], s73 offen offset:3072 sc0 sc1\n" \
  "v_mfma_f32_16x16x32_f16 a[124:127], v[38:41], v[130:133], a[124:127]\n" \
  "buffer_load_dwordx4 v[178:181], %[voffW], s[36:39], s63 offen nt\n" \
  "v_mfma_f32_16x16x32_f16 a[124:127], v[42:45], v[162:165], a[124:127]\n" \
  "v_mfma_f32_16x16x32_f16 a[92:95], v[18:21], v[162:165], a[92:95]\n" \
  "buffer_load_dwordx4 v[182:185], %[voffW], s[36:39], s64 offen nt\n" \
  "v_mfma_f32_16x16x32_f16 a[92:95], v[14:17], v[130:133], a[92:95]\n" \
  "s_waitcnt vmcnt(17)\n" \
  "v_mfma_f32_16x16x32_f16 a[192:195], v[30:33], v[102:105], a[192:195]\n" \
  "buffer_load_dwordx4 v[186:189], %[voffW], s[36:39], s65 offen nt\n" \
  "v_mfma_f32_16x16x32_f16 a[192:195], v[34:37], v[134:137], a[192:195]\n" \
  "s_memrealtime s[58:59]\n" \
  "v_mfma_f32_16x16x32_f16 a[224:227], v[58:61], v[134:137], a[224:227]\n" \
  "buffer_load_dwordx4 v[190:193], %[voffW], s[36:39], s66 offen nt\n" \
  "v_mfma_f32_16x16x32_f16 a[224:227], v[54:57], v[102:105], a[224:227]\n" \
  "v_mfma_f32_16x16x32_f16 a[228:231], v[54:57], v[106:109], a[228:231]\n" \
  "buffer_load_dwordx4 v[194:197], %[voffW], s[36:39], s67 offen nt\n" \
  "v_mfma_f32_16x16x32_f16 a[228:231], v[58:61], v[138:141], a[228:231]\n" \
  "v_mfma_f32_16x16x32_f16 a[196:199], v[34:37], v[138:141], a[196:199]\n" \
  "v_mfma_f32_16x16x32_f16 a[196:199], v[30:33], v[106:109], a[196:199]\n" \
  "v_mfma_f32_16x16x32_f16 a[200:203], v[30:33], v[110:113], a[200:203]\n" \
  "v_mfma_f32_16x16x32_f16 a[200:203], v[34:37], v[142:145], a[200:203]\n" \
  "s_waitcnt lgkmcnt(0)\n" \
  "v_mfma_f32_16x16x32_f16 a[232:235], v[58:61], v[142:145], a[232:235]\n" \
  "s_sub_u32 s82, s81, s58\n" \
  "s_branch L_s31_%=\n" \
  "L_b31_%=:\n" \
  "v_mfma_f32_16x16x32_f16 a[232:235], v[54:57], v[110:113], a[232:235]\n" \
  "s_max_i32 s82, s82, -150\n" \
  "s_add_u32 s81, s58, s82\n" \
  "s_add_u32 s81, s81, s87\n" \
  "v_mfma_f32_16x16x32_f16 a[236:239], v[54:57], v[114:117], a[236:239]\n" \
  "v_mfma_f32_16x16x32_f16 a[236:239], v[58:61], v[146:149], a[236:239]\n" \
  "v_mfma_f32_16x16x32_f16 a[204:207], v[34:37], v[146:149], a[204:207]\n" \
  "v_mfma_f32_16x16x32_f16 a[204:207], v[30:33], v[114:117], a[204:207]\n" \
  "v_mfma_f32_16x16x32_f16 a[208:211], v[30:33], v[118:121], a[208:211]\n" \
  "v_mfma_f32_16x16x32_f16 a[208:211], v[34:37], v[150:153], a[208:211]\n" \
  "v_mfma_f32_16x16x32_f16 a[240:243], v[58:61], v[150:153], a[240:243]\n" \
  "v_mfma_f32_16x16x32_f16 a[240:243], v[54:57], v[118:121], a[240:243]\n" \
  "v_readlane_b32 s74, %[vtabA], 10\n" \
  "v_mfma_f32_16x16x32_f16 a[244:247], v[54:57], v[122:125], a[244:247]\n" \
  "s_bitcmp1_b32 s34, 10\n" \
  "s_cbranch_scc0 L_s55_%=\n" \
  "L_b55_%=:\n" \
  "v_mfma_f32_16x16x32_f16 a[244:247], v[58:61], v[154:157], a[244:247]\n" \
  "s_waitcnt vmcnt(18)\n" \
  "s_add_u32 s72, s74, s68\n" \
  "v_mfma_f32_16x16x32_f16 a[212:215], v[34:37], v[154:157], a[212:215]\n" \
  "v_cvt_pk_f16_f32 v216, v198, v202\n" \
  "v_cvt_pk_f16_f32 v220, v199, v203\n" \
  "v_mfma_f32_16x16x32_f16 a[212:215], v[30:33], v[122:125], a[212:215]\n" \
  "s_waitcnt vmcnt(16)\n" \
  "v_cvt_pk_f16_f32 v224, v200, v204\n" \
  "v_cvt_pk_f16_f32 v228, v201, v205\n" \
  "v_mfma_f32_16x16x32_f16 a[216:219], v[30:33], v[126:129], a[216:219]\n" \
  "v_cvt_pk_f16_f32 v217, v206, v210\n" \
  "v_cvt_pk_f16_f32 v221, v207, v211\n" \
  "v_mfma_f32_16x16x32_f16 a[216:219], v[34:37], v[158:161], a[216:219]\n" \
  "v_cvt_pk_f16_f32 v225, v208, v212\n" \
  "v_cvt_pk_f16_f32 v229, v209, v213\n" \
  "v_mfma_f32_16x16x32_f16 a[248:251], v[58:61], v[158:161], a[248:251]\n" \
  "ds_write_b128 v243, v[214:217] offset:32768\n" \
  "v_mfma_f32_16x16x32_f16 a[248:251], v[54:57], v[126:129], a[248:251]\n" \
  "buffer_load_dwordx4 v[78:81], %[bxr], s[40:43], s72 offen offset:0 sc0 sc1\n" \
  "v_mfma_f32_16x16x32_f16 a[252:255], v[54:57], v[130:133], a[252:255]\n" \
  "v_readlane_b32 s75, %[vtabW], 16\n" \
  "v_mfma_f32_16x16x32_f16 a[252:255], v[58:61], v[162:165], a[252:255]\n" \
  "s_waitcnt vmcnt(11)\n" \
  "v_mfma_f32_16x16x32_f16 a[220:223], v[34:37], v[162:165], a[220:223]\n" \
  "ds_write_b128 v243, v[218:221] offset:33792\n" \
  "v_mfma_f32_16x16x32_f16 a[220:223], v[30:33], v[130:133], a[220:223]\n" \
  "buffer_load_dwordx4 v[82:85], %[bxr], s[40:43], s72 offen offset:1024 sc0 sc1\n" \
  "v_mfma_f32_16x16x32_f16 a[0:3], v[46:49], v[102:105], a[0:3]\n" \
  "s_add_u32 s60, s75, 0x0\n" \
  "s_add_u32 s61, s75, 0x10000\n" \
  "s_add_u32 s62, s75, 0x20000\n" \
  "v_mfma_f32_16x16x32_f16 a[0:3], v[50:53], v[134:137], a[0:3]\n" \
  "s_add_u32 s63, s75, 0x30000\n" \
  "s_add_u32 s64, s75, 0x100000\n" \
  "s_add_u32 s65, s75, 0x110000\n" \
  "v_mfma_f32_16x16x32_f16 a[32:35], v[74:77], v[134:137], a[32:35]\n" \
  "ds_write_b128 v243, v[222:225] offset:34816\n" \
  "v_mfma_f32_16x16x32_f16 a[32:35], v[70:73], v[102:105], a[32:35]\n" \
  "buffer_load_dwordx4 v[22:25], %[bxr], s[40:43], s72 offen offset:2048 sc0 sc1\n" \
  "v_mfma_f32_16x16x32_f16 a[36:39], v[70:73], v[106:109], a[36:39]\n" \
  "v_mfma_f32_16x16x32_f16 a[36:39], v[74:77], v[138:141], a[36:39]\n" \
  "s_add_u32 s73, s74, s69\n" \
  "v_mfma_f32_16x16x32_f16 a[4:7], v[50:53], v[138:141], a[4:7]\n" \
  "ds_write_b128 v243, v[226:229] offset:35840\n" \
  "v_mfma_f32_16x16x32_f16 a[4:7], v[46:49], v[106:109], a[4:7]\n" \
  "buffer_load_dwordx4 v[26:29], %[bxr], s[40:43], s72 offen offset:3072 sc0 sc1\n" \
  "v_mfma_f32_16x16x32_f16 a[8:11], v[46:49], v[110:113], a[8:11]\n" \
  "v_mfma_f32_16x16x32_f16 a[8:11], v[50:53], v[142:145], a[8:11]\n" \
  "v_mfma_f32_16x16x32_f16 a[40:43], v[74:77], v[142:145], a[40:43]\n" \
  "v_mfma_f32_16x16x32_f16 a[40:43], v[70:73], v[110:113], a[40:43]\n" \
  "v_cvt_pk_f16_f32 v214, v166, v170\n" \
  "v_cvt_pk_f16_f32 v218, v167, v171\n" \
  "v_mfma_f32_16x16x32_f16 a[44:47], v[70:73], v[114:117], a[44:47]\n" \
  "v_cvt_pk_f16_f32 v222, v168, v172\n" \
  "v_cvt_pk_f16_f32 v226, v169, v173\n" \
  "v_mfma_f32_16x16x32_f16 a[44:47], v[74:77], v[146:149], a[44:47]\n" \
  "buffer_load_dwordx4 v[94:97], %[bxr], s[40:43], s73 offen offset:0 sc0 sc1\n" \
  "v_mfma_f32_16x16x32_f16 a[12:15], v[50:53], v[146:149], a[12:15]\n" \
  "v_mfma_f32_16x16x32_f16 a[12:15], v[46:49], v[114:117], a[12:15]\n" \
  "v_mfma_f32_16x16x32_f16 a[16:19], v[46:49], v[118:121], a[16:19]\n" \
  "v_mfma_f32_16x16x32_f16 a[16:19], v[50:53], v[150:153], a[16:19]\n" \
  "buffer_load_dwordx4 v[98:101], %[bxr], s[40:43], s73 offen offset:1024 sc0 sc1\n" \
  "v_mfma_f32_16x16x32_f16 a[48:51], v[74:77], v[150:153], a[48:51]\n" \
  "v_mfma_f32_16x16x32_f16 a[48:51], v[70:73], v[118:121], a[48:51]\n" \
  "v_mfma_f32_16x16x32_f16 a[52:55], v[70:73], v[122:125], a[52:55]\n" \
  "v_mfma_f32_16x16x32_f16 a[52:55], v[74:77], v[154:157], a[52:55]\n" \
  "buffer_load_dwordx4 v[38:41], %[bxr], s[40:43], s73 offen offset:2048 sc0 sc1\n" \
  "v_mfma_f32_16x16x32_f16 a[20:23], v[50:53], v[154:157], a[20:23]\n" \
  "v_mfma_f32_16x16x32_f16 a[20:23], v[46:49], v[122:125], a[20:23]\n" \
  "s_andn2_b32 s79, 0x3fff, s34\n" \
  "s_cbranch_scc1 L_pl10_%=\n" \
  "L_pb10_%=:\n" \
  "v_mfma_f32_16x16x32_f16 a[24:27], v[46:49], v[126:129], a[24:27]\n" \
  "s_barrier\n" \
  "v_mfma_f32_16x16x32_f16 a[24:27], v[50:53], v[158:161], a[24:27]\n" \
  "buffer_load_dwordx4 v[42:45], %[bxr], s[40:43], s73 offen offset:3072 sc0 sc1\n" \
  "v_mfma_f32_16x16x32_f16 a[56:59], v[74:77], v[158:161], a[56:59]\n" \
  "buffer_load_dwordx4 v[198:201], %[voffW], s[36:39], s60 offen nt\n" \
  "v_mfma_f32_16x16x32_f16 a[56:59], v[70:73], v[126:129], a[56:59]\n" \
  "v_mfma_f32_16x16x32_f16 a[60:63], v[70:73], v[130:133], a[60:63]\n" \
  "buffer_load_dwordx4 v[202:205], %[voffW], s[36:39], s61 offen nt\n" \
  "v_mfma_f32_16x16x32_f16 a[60:63], v[74:77], v[162:165], a[60:63]\n" \
  "v_mfma_f32_16x16x32_f16 a[28:31], v[50:53], v[162:165], a[28:31]\n" \
  "buffer_load_dwordx4 v[206:209], %[voffW], s[36:39], s62 offen nt\n" \
  "v_mfma_f32_16x16x32_f16 a[28:31], v[46:49], v[130:133], a[28:31]\n" \
  "s_waitcnt vmcnt(16)\n" \
  "v_mfma_f32_16x16x32_f16 a[128:131], v[62:65], v[102:105], a[128:131]\n" \
  "buffer_load_dwordx4 v[210:213], %[voffW], s[36:39], s63 offen nt\n" \
  "v_mfma_f32_16x16x32_f16 a[128:131], v[66:69], v[134:137], a[128:131]\n" \
  "v_mfma_f32_16x16x32_f16 a[160:163], v[90:93], v[134:137], a[160:163]\n" \
  "ds_read_b128 v[134:137], v245 offset:40960\n" \
  "v_mfma_f32_16x16x32_f16 a[160:163], v[86:89], v[102:105], a[160:163]\n" \
  "ds_read_b128 v[102:105], v245 offset:32768\n" \
  "v_mfma_f32_16x16x32_f16 a[164:167], v[86:89], v[106:109], a[164:167]\n" \
  "buffer_load_dwordx4 v[166:169], %[voffW], s[36:39], s64 offen nt\n" \
  "v_mfma_f32_16x16x32_f16 a[164:167], v[90:93], v[138:141], a[164:167]\n" \
  "v_mfma_f32_16x16x32_f16 a[132:135], v[66:69], v[138:141], a[132:135]\n" \
  "ds_read_b128 v[138:141], v245 offset:41984\n" \
  "v_mfma_f32_16x16x32_f16 a[132:135], v[62:65], v[106:109], a[132:135]\n" \
  "ds_read_b128 v[106:109], v245 offset:33792\n" \
  "v_mfma_f32_16x16x32_f16 a[136:139], v[62:65], v[110:113], a[136:139]\n" \
  "buffer_load_dwordx4 v[170:173], %[voffW], s[36:39], s65 offen nt\n" \
  "v_mfma_f32_16x16x32_f16 a[136:139], v[66:69], v[142:145], a[136:139]\n" \
  "v_mfma_f32_16x16x32_f16 a[168:171], v[90:93], v[142:145], a[168:171]\n" \
  "ds_read_b128 v[142:145], v245 offset:43008\n" \
  "v_mfma_f32_16x16x32_f16 a[168:171], v[86:89], v[110:113], a[168:171]\n" \
  "ds_read_b128 v[110:113], v245 offset:34816\n" \
  "v_mfma_f32_16x16x32_f16 a[172:175], v[86:89], v[114:117], a[172:175]\n" \
  "v_mfma_f32_16x16x32_f16 a[172:175], v[90:93], v[146:149], a[172:175]\n" \
  "v_mfma_f32_16x16x32_f16 a[140:143], v[66:69], v[146:149], a[140:143]\n" \
  "ds_read_b128 v[146:149], v245 offset:44032\n" \
  "v_mfma_f32_16x16x32_f16 a[140:143], v[62:65], v[114:117], a[140:143]\n" \
  "ds_read_b128 v[114:117], v245 offset:35840\n" \
  "v_mfma_f32_16x16x32_f16 a[144:147], v[62:65], v[118:121], a[144:147]\n" \
  "s_add_u32 s72, s74, s70\n" \
  "v_mfma_f32_16x16x32_f16 a[144:147], v[66:69], v[150:153], a[144:147]\n" \
  "v_mfma_f32_16x16x32_f16 a[176:179], v[90:93], v[150:153], a[176:179]\n" \
  "ds_read_b128 v[150:153], v245 offset:45056\n" \
  "v_mfma_f32_16x16x32_f16 a[176:179], v[86:89], v[118:121], a[176:179]\n" \
  "ds_read_b128 v[118:121], v245 offset:36864\n" \
  "v_mfma_f32_16x16x32_f16 a[180:183], v[86:89], v[122:125], a[180:183]\n" \
  "v_mfma_f32_16x16x32_f16 a[180:183], v[90:93], v[154:157], a[180:183]\n" \
  "v_mfma_f32_16x16x32_f16 a[148:151], v[66:69], v[154:157], a[148:151]\n" \
  "ds_read_b128 v[154:157], v245 offset:46080\n" \
  "v_mfma_f32_16x16x32_f16 a[148:151], v[62:65], v[122:125], a[148:151]\n" \
  "ds_read_b128 v[122:125], v245 offset:37888\n" \
  "v_mfma_f32_16x16x32_f16 a[152:155], v[62:65], v[126:129], a[152:155]\n" \
  "buffer_load_dwordx4 v[18:21], %[bxr], s[40:43], s72 offen offset:0 sc0 sc1\n" \
  "v_mfma_f32_16x16x32_f16 a[152:155], v[66:69], v[158:161], a[152:155]\n" \
  "v_mfma_f32_16x16x32_f16 a[184:187], v[90:93], v[158:161], a[184:187]\n" \
  "ds_read_b128 v[158:161], v245 offset:47104\n" \
  "v_mfma_f32_16x16x32_f16 a[184:187], v[86:89], v[126:129], a[184:187]\n" \
  "ds_read_b128 v[126:129], v245 offset:38912\n" \
  "v_mfma_f32_16x16x32_f16 a[188:191], v[86:89], v[130:133], a[188:191]\n" \
  "buffer_load_dwordx4 v[14:17], %[bxr], s[40:43], s72 offen offset:1024 sc0 sc1\n" \
  "v_mfma_f32_16x16x32_f16 a[188:191], v[90:93], v[162:165], a[188:191]\n" \
  "s_add_u32 s66, s75, 0x120000\n" \
  "s_add_u32 s67, s75, 0x130000\n" \
  "v_mfma_f32_16x16x32_f16 a[156:159], v[66:69], v[162:165], a[156:159]\n" \
  "ds_read_b128 v[162:165], v245 offset:48128\n" \
  "v_readlane_b32 s75, %[vtabW], 17\n" \
  "v_mfma_f32_16x16x32_f16 a[156:159], v[62:65], v[130:133], a[156:159]\n" \
  "s_waitcnt vmcnt(12)\n" \
  "ds_read_b128 v[130:133], v245 offset:39936\n" \
  "s_waitcnt lgkmcnt(14)\n" \
    \
  "v_mfma_f32_16x16x32_f16 a[64:67], v[78:81], v[102:105], a[64:67]\n" \
  "buffer_load_dwordx4 v[54:57], %[bxr], s[40:43], s72 offen offset:2048 sc0 sc1\n" \
  "v_mfma_f32_16x16x32_f16 a[64:67], v[82:85], v[134:137], a[64:67]\n" \
  "s_add_u32 s60, s75, 0x0\n" \
  "s_add_u32 s61, s75, 0x10000\n" \
  "s_add_u32 s62, s75, 0x20000\n" \
  "v_mfma_f32_16x16x32_f16 a[96:99], v[26:29], v[134:137], a[96:99]\n" \
  "s_add_u32 s63, s75, 0x30000\n" \
  "s_add_u32 s73, s74, s71\n" \
  "v_mfma_f32_16x16x32_f16 a[96:99], v[22:25], v[102:105], a[96:99]\n" \
  "s_waitcnt lgkmcnt(12)\n" \
  "v_mfma_f32_16x16x32_f16 a[100:103], v[22:25], v[106:109], a[100:103]\n" \
  "buffer_load_dwordx4 v[58:61], %[bxr], s[40:43], s72 offen offset:3072 sc0 sc1\n" \
  "v_mfma_f32_16x16x32_f16 a[100:103], v[26:29], v[138:141], a[100:103]\n" \
  "v_mfma_f32_16x16x32_f16 a[68:71], v[82:85], v[138:141], a[68:71]\n" \
  "v_mfma_f32_16x16x32_f16 a[68:71], v[78:81], v[106:109], a[68:71]\n" \
  "s_waitcnt lgkmcnt(10)\n" \
  "v_mfma_f32_16x16x32_f16 a[72:75], v[78:81], v[110:113], a[72:75]\n" \
  "buffer_load_dwordx4 v[34:37], %[bxr], s[40:43], s73 offen offset:0 sc0 sc1\n" \
  "v_mfma_f32_16x16x32_f16 a[72:75], v[82:85], v[142:145], a[72:75]\n" \
  "v_mfma_f32_16x16x32_f16 a[104:107], v[26:29], v[142:145], a[104:107]\n" \
  "v_mfma_f32_16x16x32_f16 a[104:107], v[22:25], v[110:113], a[104:107]\n" \
  "s_waitcnt lgkmcnt(8)\n" \
  "v_cvt_pk_f16_f32 v215, v174, v178\n" \
  "v_cvt_pk_f16_f32 v219, v175, v179\n" \
  "v_mfma_f32_16x16x32_f16 a[108:111], v[22:25], v[114:117], a[108:111]\n" \
  "v_cvt_pk_f16_f32 v223, v176, v180\n" \
  "v_cvt_pk_f16_f32 v227, v177, v181\n" \
  "v_mfma_f32_16x16x32_f16 a[108:111], v[26:29], v[146:149], a[108:111]\n" \
  "s_waitcnt vmcnt(12)\n" \
  "v_cvt_pk_f16_f32 v216, v182, v186\n" \
  "v_cvt_pk_f16_f32 v220, v183, v187\n" \
  "v_mfma_f32_16x16x32_f16 a[76:79], v[82:85], v[146:149], a[76:79]\n" \
  "v_cvt_pk_f16_f32 v224, v184, v188\n" \
  "v_cvt_pk_f16_f32 v228, v185, v189\n" \
  "v_mfma_f32_16x16x32_f16 a[76:79], v[78:81], v[114:117], a[76:79]\n" \
  "s_waitcnt lgkmcnt(6)\n" \
  "v_cvt_pk_f16_f32 v217, v190, v194\n" \
  "v_cvt_pk_f16_f32 v221, v191, v195\n" \
  "v_mfma_f32_16x16x32_f16 a[80:83], v[78:81], v[118:121], a[80:83]\n" \
  "v_cvt_pk_f16_f32 v225, v192, v196\n" \
  "v_cvt_pk_f16_f32 v229, v193, v197\n" \
  "v_mfma_f32_16x16x32_f16 a[80:83], v[82:85], v[150:153], a[80:83]\n" \
  "ds_write_b128 v243, v[214:217] offset:49152\n" \
  "v_mfma_f32_16x16x32_f16 a[112:115], v[26:29], v[150:153], a[112:115]\n" \
  "buffer_load_dwordx4 v[30:33], %[bxr], s[40:43], s73 offen offset:1024 sc0 sc1\n" \
  "v_mfma_f32_16x16x32_f16 a[112:115], v[22:25], v[118:121], a[112:115]\n" \
  "s_waitcnt lgkmcnt(5)\n" \
  "v_min3_u32 v248, v238, v239, v240\n" \
  "v_min_u32 v248, v248, v241\n" \
  "v_mfma_f32_16x16x32_f16 a[116:119], v[22:25], v[122:125], a[116:119]\n" \
  "v_cmp_le_u32 s[34:35], s76, v248\n" \
  "v_mfma_f32_16x16x32_f16 a[116:119], v[26:29], v[154:157], a[116:119]\n" \
  "ds_write_b128 v243, v[218:221] offset:50176\n" \
  "v_mfma_f32_16x16x32_f16 a[84:87], v[82:85], v[154:157], a[84:87]\n" \
  "buffer_load_dwordx4 v[70:73], %[bxr], s[40:43], s73 offen offset:2048 sc0 sc1\n" \
  "v_mfma_f32_16x16x32_f16 a[84:87], v[78:81], v[122:125], a[84:87]\n" \
  "s_waitcnt lgkmcnt(4)\n" \
  "v_mfma_f32_16x16x32_f16 a[88:91], v[78:81], v[126:129], a[88:91]\n" \
  "s_waitcnt lgkmcnt(2)\n" \
  "s_barrier\n" \
  "v_mfma_f32_16x16x32_f16 a[88:91], v[82:85], v[158:161], a[88:91]\n" \
  "ds_write_b128 v243, v[222:225] offset:51200\n" \
  "v_mfma_f32_16x16x32_f16 a[120:123], v[26:29], v[158:161], a[120:123]\n" \
  "buffer_load_dwordx4 v[174:177], %[voffW], s[36:39], s66 offen nt\n" \
  "v_mfma_f32_16x16x32_f16 a[120:123], v[22:25], v[126:129], a[120:123]\n" \
  "buffer_load_dwordx4 v[74:77], %[bxr], s[40:43], s73 offen offset:3072 sc0 sc1\n" \
  "v_mfma_f32_16x16x32_f16 a[124:127], v[22:25], v[130:133], a[124:127]\n" \
  "buffer_load_dwordx4 v[178:181], %[voffW], s[36:39], s67 offen nt\n" \
  "v_mfma_f32_16x16x32_f16 a[124:127], v[26:29], v[162:165], a[124:127]\n" \
  "ds_write_b128 v243, v[226:229] offset:52224\n" \
  "v_mfma_f32_16x16x32_f16 a[92:95], v[82:85], v[162:165], a[92:95]\n" \
  "buffer_load_dwordx4 v[182:185], %[voffW], s[36:39], s60 offen nt\n" \
  "v_mfma_f32_16x16x32_f16 a[92:95], v[78:81], v[130:133], a[92:95]\n" \
  "s_waitcnt vmcnt(17)\n" \
  "v_mfma_f32_16x16x32_f16 a[192:195], v[94:97], v[102:105], a[192:195]\n" \
  "buffer_load_dwordx4 v[186:189], %[voffW], s[36:39], s61 offen nt\n" \
  "v_mfma_f32_16x16x32_f16 a[192:195], v[98:101], v[134:137], a[192:195]\n" \
  "s_memrealtime s[58:59]\n" \
  "v_mfma_f32_16x16x32_f16 a[224:227], v[42:45], v[134:137], a[224:227]\n" \
  "buffer_load_dwordx4 v[190:193], %[voffW], s[36:39], s62 offen nt\n" \
  "v_mfma_f32_16x16x32_f16 a[224:227], v[38:41], v[102:105], a[224:227]\n" \
  "v_mfma_f32_16x16x32_f16 a[228:231], v[38:41], v[106:109], a[228:231]\n" \
  "buffer_load_dwordx4 v[194:197], %[voffW], s[36:39], s63 offen nt\n" \
  "v_mfma_f32_16x16x32_f16 a[228:231], v[42:45], v[138:141], a[228:231]\n" \
  "v_mfma_f32_16x16x32_f16 a[196:199], v[98:101], v[138:141], a[196:199]\n" \
  "v_mfma_f32_16x16x32_f16 a[196:199], v[94:97], v[106:109], a[196:199]\n" \
  "v_mfma_f32_16x16x32_f16 a[200:203], v[94:97], v[110:113], a[200:203]\n" \
  "v_mfma_f32_16x16x32_f16 a[200:203], v[98:101], v[142:145], a[200:203]\n" \
  "s_waitcnt lgkmcnt(0)\n" \
  "v_mfma_f32_16x16x32_f16 a[232:235], v[42:45], v[142:145], a[232:235]\n" \
  "s_sub_u32 s82, s81, s58\n" \
  "s_branch L_s32_%=\n" \
  "L_b32_%=:\n" \
  "v_mfma_f32_16x16x32_f16 a[232:235], v[38:41], v[110:113], a[232:235]\n" \
  "s_max_i32 s82, s82, -150\n" \
  "s_add_u32 s81, s58, s82\n" \
  "s_add_u32 s81, s81, s87\n" \
  "v_mfma_f32_16x16x32_f16 a[236:239], v[38:41], v[114:117], a[236:239]\n" \
  "v_mfma_f32_16x16x32_f16 a[236:239], v[42:45], v[146:149], a[236:239]\n" \
  "v_mfma_f32_16x16x32_f16 a[204:207], v[98:101], v[146:149], a[204:207]\n" \
  "v_readlane_b32 s74, %[vtabA], 11\n" \
  "v_mfma_f32_16x16x32_f16 a[204:207], v[94:97], v[114:117], a[204:207]\n" \
  "s_bitcmp1_b32 s34, 11\n" \
  "s_cbranch_scc0 L_s56_%=\n" \
  "L_b56_%=:\n" \
  "v_mfma_f32_16x16x32_f16 a[208:211], v[94:97], v[118:121], a[208:211]\n" \
  "s_add_u32 s72, s74, s68\n" \
  "v_mfma_f32_16x16x32_f16 a[208:211], v[98:101], v[150:153], a[208:211]\n" \
  "v_mfma_f32_16x16x32_f16 a[240:243], v[42:45], v[150:153], a[240:243]\n" \
  "v_mfma_f32_16x16x32_f16 a[240:243], v[38:41], v[118:121], a[240:243]\n" \
  "v_mfma_f32_16x16x32_f16 a[244:247], v[38:41], v[122:125], a[244:247]\n" \
  "v_mfma_f32_16x16x32_f16 a[244:247], v[42:45], v[154:157], a[244:247]\n" \
  "v_mfma_f32_16x16x32_f16 a[212:215], v[98:101], v[154:157], a[212:215]\n" \
  "buffer_load_dwordx4 v[50:53], %[bxr], s[40:43], s72 offen offset:0 sc0 sc1\n" \
  "v_mfma_f32_16x16x32_f16 a[212:215], v[94:97], v[122:125], a[212:215]\n" \
  "v_mfma_f32_16x16x32_f16 a[216:219], v[94:97], v[126:129], a[216:219]\n" \
  "v_mfma_f32_16x16x32_f16 a[216:219], v[98:101], v[158:161], a[216:219]\n" \
  "v_mfma_f32_16x16x32_f16 a[248:251], v[42:45], v[158:161], a[248:251]\n" \
  "buffer_load_dwordx4 v[46:49], %[bxr], s[40:43], s72 offen offset:1024 sc0 sc1\n" \
  "v_mfma_f32_16x16x32_f16 a[248:251], v[38:41], v[126:129], a[248:251]\n" \
  "v_mfma_f32_16x16x32_f16 a[252:255], v[38:41], v[130:133], a[252:255]\n" \
  "s_add_u32 s64, s75, 0x100000\n" \
  "s_add_u32 s65, s75, 0x110000\n" \
  "s_add_u32 s66, s75, 0x120000\n" \
  "v_mfma_f32_16x16x32_f16 a[252:255], v[42:45], v[162:165], a[252:255]\n" \
  "s_add_u32 s67, s75, 0x130000\n" \
  "v_mfma_f32_16x16x32_f16 a[220:223], v[98:101], v[162:165], a[220:223]\n" \
  "buffer_load_dwordx4 v[86:89], %[bxr], s[40:43], s72 offen offset:2048 sc0 sc1\n" \
  "v_mfma_f32_16x16x32_f16 a[220:223], v[94:97], v[130:133], a[220:223]\n" \
  "s_waitcnt vmcnt(13)\n" \
  "v_readlane_b32 s75, %[vtabW], 18\n" \
  "s_add_u32 s73, s74, s69\n" \
  "v_mfma_f32_16x16x32_f16 a[0:3], v[18:21], v[102:105], a[0:3]\n" \
  "v_mfma_f32_16x16x32_f16 a[0:3], v[14:17], v[134:137], a[0:3]\n" \
  "s_add_u32 s60, s75, 0x0\n" \
  "s_add_u32 s61, s75, 0x10000\n" \
  "v_mfma_f32_16x16x32_f16 a[32:35], v[58:61], v[134:137], a[32:35]\n" \
  "buffer_load_dwordx4 v[90:93], %[bxr], s[40:43], s72 offen offset:3072 sc0 sc1\n" \
  "v_mfma_f32_16x16x32_f16 a[32:35], v[54:57], v[102:105], a[32:35]\n" \
  "v_mfma_f32_16x16x32_f16 a[36:39], v[54:57], v[106:109], a[36:39]\n" \
  "v_mfma_f32_16x16x32_f16 a[36:39], v[58:61], v[138:141], a[36:39]\n" \
  "v_mfma_f32_16x16x32_f16 a[4:7], v[14:17], v[138:141], a[4:7]\n" \
  "buffer_load_dwordx4 v[66:69], %[bxr], s[40:43], s73 offen offset:0 sc0 sc1\n" \
  "v_mfma_f32_16x16x32_f16 a[4:7], v[18:21], v[106:109], a[4:7]\n" \
  "v_mfma_f32_16x16x32_f16 a[8:11], v[18:21], v[110:113], a[8:11]\n" \
  "v_mfma_f32_16x16x32_f16 a[8:11], v[14:17], v[142:145], a[8:11]\n" \
  "v_mfma_f32_16x16x32_f16 a[40:43], v[58:61], v[142:145], a[40:43]\n" \
  "v_mfma_f32_16x16x32_f16 a[40:43], v[54:57], v[110:113], a[40:43]\n" \
  "v_cvt_pk_f16_f32 v214, v198, v202\n" \
  "v_cvt_pk_f16_f32 v218, v199, v203\n" \
  "v_mfma_f32_16x16x32_f16 a[44:47], v[54:57], v[114:117], a[44:47]\n" \
  "v_cvt_pk_f16_f32 v222, v200, v204\n" \
  "v_cvt_pk_f16_f32 v226, v201, v205\n" \
  "v_mfma_f32_16x16x32_f16 a[44:47], v[58:61], v[146:149], a[44:47]\n" \
  "v_cvt_pk_f16_f32 v215, v206, v210\n" \
  "v_cvt_pk_f16_f32 v219, v207, v211\n" \
  "v_mfma_f32_16x16x32_f16 a[12:15], v[14:17], v[146:149], a[12:15]\n" \
  "v_cvt_pk_f16_f32 v223, v208, v212\n" \
  "v_cvt_pk_f16_f32 v227, v209, v213\n" \
  "v_mfma_f32_16x16x32_f16 a[12:15], v[18:21], v[114:117], a[12:15]\n" \
  "v_cvt_pk_f16_f32 v216, v166, v170\n" \
  "v_cvt_pk_f16_f32 v220, v167, v171\n" \
  "v_mfma_f32_16x16x32_f16 a[16:19], v[18:21], v[118:121], a[16:19]\n" \
  "v_cvt_pk_f16_f32 v224, v168, v172\n" \
  "v_cvt_pk_f16_f32 v228, v169, v173\n" \
  "v_mfma_f32_16x16x32_f16 a[16:19], v[14:17], v[150:153], a[16:19]\n" \
  "buffer_load_dwordx4 v[62:65], %[bxr], s[40:43], s73 offen offset:1024 sc0 sc1\n" \
  "v_mfma_f32_16x16x32_f16 a[48:51], v[58:61], v[150:153], a[48:51]\n" \
  "v_mfma_f32_16x16x32_f16 a[48:51], v[54:57], v[118:121], a[48:51]\n" \
  "v_mfma_f32_16x16x32_f16 a[52:55], v[54:57], v[122:125], a[52:55]\n" \
  "v_mfma_f32_16x16x32_f16 a[52:55], v[58:61], v[154:157], a[52:55]\n" \
  "buffer_load_dwordx4 v[22:25], %[bxr], s[40:43], s73 offen offset:2048 sc0 sc1\n" \
  "v_mfma_f32_16x16x32_f16 a[20:23], v[14:17], v[154:157], a[20:23]\n" \
  "v_mfma_f32_16x16x32_f16 a[20:23], v[18:21], v[122:125], a[20:23]\n" \
  "s_andn2_b32 s79, 0x7fff, s34\n" \
  "s_cbranch_scc1 L_pl11_%=\n" \
  "L_pb11_%=:\n" \
  "v_mfma_f32_16x16x32_f16 a[24:27], v[18:21], v[126:129], a[24:27]\n" \
  "s_barrier\n" \
  "v_mfma_f32_16x16x32_f16 a[24:27], v[14:17], v[158:161], a[24:27]\n" \
  "buffer_load_dwordx4 v[26:29], %[bxr], s[40:43], s73 offen offset:3072 sc0 sc1\n" \
  "v_mfma_f32_16x16x32_f16 a[56:59], v[58:61], v[158:161], a[56:59]\n" \
  "buffer_load_dwordx4 v[198:201], %[voffW], s[36:39], s64 offen nt\n" \
  "v_mfma_f32_16x16x32_f16 a[56:59], v[54:57], v[126:129], a[56:59]\n" \
  "v_mfma_f32_16x16x32_f16 a[60:63], v[54:57], v[130:133], a[60:63]\n" \
  "buffer_load_dwordx4 v[202:205], %[voffW], s[36:39], s65 offen nt\n" \
  "v_mfma_f32_16x16x32_f16 a[60:63], v[58:61], v[162:165], a[60:63]\n" \
  "v_mfma_f32_16x16x32_f16 a[28:31], v[14:17], v[162:165], a[28:31]\n" \
  "buffer_load_dwordx4 v[206:209], %[voffW], s[36:39], s66 offen nt\n" \
  "v_mfma_f32_16x16x32_f16 a[28:31], v[18:21], v[130:133], a[28:31]\n" \
  "s_waitcnt vmcnt(16)\n" \
  "v_mfma_f32_16x16x32_f16 a[128:131], v[34:37], v[102:105], a[128:131]\n" \
  "buffer_load_dwordx4 v[210:213], %[voffW], s[36:39], s67 offen nt\n" \
  "v_mfma_f32_16x16x32_f16 a[128:131], v[30:33], v[134:137], a[128:131]\n" \
  "v_mfma_f32_16x16x32_f16 a[160:163], v[74:77], v[134:137], a[160:163]\n" \
  "ds_read_b128 v[134:137], v245 offset:57344\n" \
  "v_mfma_f32_16x16x32_f16 a[160:163], v[70:73], v[102:105], a[160:163]\n" \
  "ds_read_b128 v[102:105], v245 offset:49152\n" \
  "v_mfma_f32_16x16x32_f16 a[164:167], v[70:73], v[106:109], a[164:167]\n" \
  "buffer_load_dwordx4 v[166:169], %[voffW], s[36:39], s60 offen nt\n" \
  "v_mfma_f32_16x16x32_f16 a[164:167], v[74:77], v[138:141], a[164:167]\n" \
  "v_mfma_f32_16x16x32_f16 a[132:135], v[30:33], v[138:141], a[132:135]\n" \
  "ds_read_b128 v[138:141], v245 offset:58368\n" \
  "v_mfma_f32_16x16x32_f16 a[132:135], v[34:37], v[106:109], a[132:135]\n" \
  "ds_read_b128 v[106:109], v245 offset:50176\n" \
  "v_mfma_f32_16x16x32_f16 a[136:139], v[34:37], v[110:113], a[136:139]\n" \
  "buffer_load_dwordx4 v[170:173], %[voffW], s[36:39], s61 offen nt\n" \
  "v_mfma_f32_16x16x32_f16 a[136:139], v[30:33], v[142:145], a[136:139]\n" \
  "v_mfma_f32_16x16x32_f16 a[168:171], v[74:77], v[142:145], a[168:171]\n" \
  "ds_read_b128 v[142:145], v245 offset:59392\n" \
  "v_mfma_f32_16x16x32_f16 a[168:171], v[70:73], v[110:113], a[168:171]\n" \
  "ds_read_b128 v[110:113], v245 offset:51200\n" \
  "v_mfma_f32_16x16x32_f16 a[172:175], v[70:73], v[114:117], a[172:175]\n" \
  "v_mfma_f32_16x16x32_f16 a[172:175], v[74:77], v[146:149], a[172:175]\n" \
  "v_mfma_f32_16x16x32_f16 a[140:143], v[30:33], v[146:149], a[140:143]\n" \
  "ds_read_b128 v[146:149], v245 offset:60416\n" \
  "v_mfma_f32_16x16x32_f16 a[140:143], v[34:37], v[114:117], a[140:143]\n" \
  "ds_read_b128 v[114:117], v245 offset:52224\n" \
  "v_mfma_f32_16x16x32_f16 a[144:147], v[34:37], v[118:121], a[144:147]\n" \
  "v_mfma_f32_16x16x32_f16 a[144:147], v[30:33], v[150:153], a[144:147]\n" \
  "v_mfma_f32_16x16x32_f16 a[176:179], v[74:77], v[150:153], a[176:179]\n" \
  "ds_read_b128 v[150:153], v245 offset:61440\n" \
  "v_mfma_f32_16x16x32_f16 a[176:179], v[70:73], v[118:121], a[176:179]\n" \
  "ds_read_b128 v[118:121], v245 offset:53248\n" \
  "v_mfma_f32_16x16x32_f16 a[180:183], v[70:73], v[122:125], a[180:183]\n" \
  "s_add_u32 s72, s74, s70\n" \
  "v_mfma_f32_16x16x32_f16 a[180:183], v[74:77], v[154:157], a[180:183]\n" \
  "s_waitcnt vmcnt(18)\n" \
  "v_mfma_f32_16x16x32_f16 a[148:151], v[30:33], v[154:157], a[148:151]\n" \
  "ds_read_b128 v[154:157], v245 offset:62464\n" \
  "v_cvt_pk_f16_f32 v217, v174, v178\n" \
  "v_mfma_f32_16x16x32_f16 a[148:151], v[34:37], v[122:125], a[148:151]\n" \
  "ds_read_b128 v[122:125], v245 offset:54272\n" \
  "v_cvt_pk_f16_f32 v221, v175, v179\n" \
  "v_mfma_f32_16x16x32_f16 a[152:155], v[34:37], v[126:129], a[152:155]\n" \
  "v_cvt_pk_f16_f32 v225, v176, v180\n" \
  "v_cvt_pk_f16_f32 v229, v177, v181\n" \
  "v_mfma_f32_16x16x32_f16 a[152:155], v[30:33], v[158:161], a[152:155]\n" \
  "ds_write_b128 v244, v[214:217] offset:0\n" \
  "v_mfma_f32_16x16x32_f16 a[184:187], v[74:77], v[158:161], a[184:187]\n" \
  "ds_read_b128 v[158:161], v245 offset:63488\n" \
  "v_mfma_f32_16x16x32_f16 a[184:187], v[70:73], v[126:129], a[184:187]\n" \
  "ds_read_b128 v[126:129], v245 offset:55296\n" \
  "v_mfma_f32_16x16x32_f16 a[188:191], v[70:73], v[130:133], a[188:191]\n" \
  "buffer_load_dwordx4 v[82:85], %[bxr], s[40:43], s72 offen offset:0 sc0 sc1\n" \
  "v_mfma_f32_16x16x32_f16 a[188:191], v[74:77], v[162:165], a[188:191]\n" \
  "ds_write_b128 v244, v[218:221] offset:1024\n" \
  "v_mfma_f32_16x16x32_f16 a[156:159], v[30:33], v[162:165], a[156:159]\n" \
  "s_add_u32 s62, s75, 0x20000\n" \
  "ds_read_b128 v[162:165], v245 offset:64512\n" \
  "s_add_u32 s63, s75, 0x30000\n" \
  "v_mfma_f32_16x16x32_f16 a[156:159], v[34:37], v[130:133], a[156:159]\n" \
  "s_waitcnt vmcnt(11)\n" \
  "ds_read_b128 v[130:133], v245 offset:56320\n" \
  "s_waitcnt lgkmcnt(14)\n" \
    \
  "v_mfma_f32_16x16x32_f16 a[64:67], v[50:53], v[102:105], a[64:67]\n" \
  "buffer_load_dwordx4 v[78:81], %[bxr], s[40:43], s72 offen offset:1024 sc0 sc1\n" \
  "v_mfma_f32_16x16x32_f16 a[64:67], v[46:49], v[134:137], a[64:67]\n" \
  "ds_write_b128 v244, v[222:225] offset:2048\n" \
  "v_mfma_f32_16x16x32_f16 a[96:99], v[90:93], v[134:137], a[96:99]\n" \
  "s_add_u32 s64, s75, 0x100000\n" \
  "s_add_u32 s65, s75, 0x110000\n" \
  "s_add_u32 s66, s75, 0x120000\n" \
  "v_mfma_f32_16x16x32_f16 a[96:99], v[86:89], v[102:105], a[96:99]\n" \
  "s_waitcnt lgkmcnt(14)\n" \
  "s_add_u32 s67, s75, 0x130000\n" \
  "v_mfma_f32_16x16x32_f16 a[100:103], v[86:89], v[106:109], a[100:103]\n" \
  "buffer_load_dwordx4 v[38:41], %[bxr], s[40:43], s72 offen offset:2048 sc0 sc1\n" \
  "v_mfma_f32_16x16x32_f16 a[100:103], v[90:93], v[138:141], a[100:103]\n" \
  "ds_write_b128 v244, v[226:229] offset:3072\n" \
  "v_mfma_f32_16x16x32_f16 a[68:71], v[46:49], v[138:141], a[68:71]\n" \
  "s_add_u32 s73, s74, s71\n" \
  "v_mfma_f32_16x16x32_f16 a[68:71], v[50:53], v[106:109], a[68:71]\n" \
  "s_waitcnt lgkmcnt(14)\n" \
  "v_mfma_f32_16x16x32_f16 a[72:75], v[50:53], v[110:113], a[72:75]\n" \
  "buffer_load_dwordx4 v[42:45], %[bxr], s[40:43], s72 offen offset:3072 sc0 sc1\n" \
  "v_mfma_f32_16x16x32_f16 a[72:75], v[46:49], v[142:145], a[72:75]\n" \
  "v_mfma_f32_16x16x32_f16 a[104:107], v[90:93], v[142:145], a[104:107]\n" \
  "s_waitcnt lgkmcnt(12)\n" \
  "v_cvt_pk_f16_f32 v214, v182, v186\n" \
  "v_cvt_pk_f16_f32 v218, v183, v187\n" \
  "v_mfma_f32_16x16x32_f16 a[104:107], v[86:89], v[110:113], a[104:107]\n" \
  "v_cvt_pk_f16_f32 v222, v184, v188\n" \
  "v_cvt_pk_f16_f32 v226, v185, v189\n" \
  "v_mfma_f32_16x16x32_f16 a[108:111], v[86:89], v[114:117], a[108:111]\n" \
  "v_cvt_pk_f16_f32 v215, v190, v194\n" \
  "v_cvt_pk_f16_f32 v219, v191, v195\n" \
  "v_mfma_f32_16x16x32_f16 a[108:111], v[90:93], v[146:149], a[108:111]\n" \
  "s_waitcnt vmcnt(11)\n" \
  "v_cvt_pk_f16_f32 v223, v192, v196\n" \
  "v_cvt_pk_f16_f32 v227, v193, v197\n" \
  "v_mfma_f32_16x16x32_f16 a[76:79], v[46:49], v[146:149], a[76:79]\n" \
  "buffer_load_dwordx4 v[98:101], %[bxr], s[40:43], s73 offen offset:0 sc0 sc1\n" \
  "v_mfma_f32_16x16x32_f16 a[76:79], v[50:53], v[114:117], a[76:79]\n" \
  "s_waitcnt lgkmcnt(10)\n" \
  "v_min3_u32 v248, v238, v239, v240\n" \
  "v_min_u32 v248, v248, v241\n" \
  "v_mfma_f32_16x16x32_f16 a[80:83], v[50:53], v[118:121], a[80:83]\n" \
  "v_cmp_le_u32 s[34:35], s76, v248\n" \
  "v_mfma_f32_16x16x32_f16 a[80:83], v[46:49], v[150:153], a[80:83]\n" \
  "v_mfma_f32_16x16x32_f16 a[112:115], v[90:93], v[150:153], a[112:115]\n" \
  "buffer_load_dwordx4 v[94:97], %[bxr], s[40:43], s73 offen offset:1024 sc0 sc1\n" \
  "v_mfma_f32_16x16x32_f16 a[112:115], v[86:89], v[118:121], a[112:115]\n" \
  "s_waitcnt lgkmcnt(8)\n" \
  "v_mfma_f32_16x16x32_f16 a[116:119], v[86:89], v[122:125], a[116:119]\n" \
  "v_mfma_f32_16x16x32_f16 a[116:119], v[90:93], v[154:157], a[116:119]\n" \
  "v_mfma_f32_16x16x32_f16 a[84:87], v[46:49], v[154:157], a[84:87]\n" \
  "buffer_load_dwordx4 v[54:57], %[bxr], s[40:43], s73 offen offset:2048 sc0 sc1\n" \
  "v_mfma_f32_16x16x32_f16 a[84:87], v[50:53], v[122:125], a[84:87]\n" \
  "s_waitcnt lgkmcnt(5)\n" \
  "v_mfma_f32_16x16x32_f16 a[88:91], v[50:53], v[126:129], a[88:91]\n" \
  "v_mfma_f32_16x16x32_f16 a[88:91], v[46:49], v[158:161], a[88:91]\n" \
  "s_waitcnt lgkmcnt(2)\n" \
  "s_barrier\n" \
  "v_mfma_f32_16x16x32_f16 a[120:123], v[90:93], v[158:161], a[120:123]\n" \
  "buffer_load_dwordx4 v[174:177], %[voffW], s[36:39], s62 offen nt\n" \
  "v_mfma_f32_16x16x32_f16 a[120:123], v[86:89], v[126:129], a[120:123]\n" \
  "buffer_load_dwordx4 v[58:61], %[bxr], s[40:43], s73 offen offset:3072 sc0 sc1\n" \
  "v_mfma_f32_16x16x32_f16 a[124:127], v[86:89], v[130:133], a[124:127]\n" \
  "buffer_load_dwordx4 v[178:181], %[voffW], s[36:39], s63 offen nt\n" \
  "v_mfma_f32_16x16x32_f16 a[124:127], v[90:93], v[162:165], a[124:127]\n" \
  "v_mfma_f32_16x16x32_f16 a[92:95], v[46:49], v[162:165], a[92:95]\n" \
  "buffer_load_dwordx4 v[182:185], %[voffW], s[36:39], s64 offen nt\n" \
  "v_mfma_f32_16x16x32_f16 a[92:95], v[50:53], v[130:133], a[92:95]\n" \
  "s_waitcnt vmcnt(17)\n" \
  "v_mfma_f32_16x16x32_f16 a[192:195], v[66:69], v[102:105], a[192:195]\n" \
  "buffer_load_dwordx4 v[186:189], %[voffW], s[36:39], s65 offen nt\n" \
  "v_mfma_f32_16x16x32_f16 a[192:195], v[62:65], v[134:137], a[192:195]\n" \
  "s_memrealtime s[58:59]\n" \
  "v_mfma_f32_16x16x32_f16 a[224:227], v[26:29], v[134:137], a[224:227]\n" \
  "buffer_load_dwordx4 v[190:193], %[voffW], s[36:39], s66 offen nt\n" \
  "v_mfma_f32_16x16x32_f16 a[224:227], v[22:25], v[102:105], a[224:227]\n" \
  "v_mfma_f32_16x16x32_f16 a[228:231], v[22:25], v[106:109], a[228:231]\n" \
  "buffer_load_dwordx4 v[194:197], %[voffW], s[36:39], s67 offen nt\n" \
  "v_mfma_f32_16x16x32_f16 a[228:231], v[26:29], v[138:141], a[228:231]\n" \
  "v_mfma_f32_16x16x32_f16 a[196:199], v[62:65], v[138:141], a[196:199]\n" \
  "v_mfma_f32_16x16x32_f16 a[196:199], v[66:69], v[106:109], a[196:199]\n" \
  "v_mfma_f32_16x16x32_f16 a[200:203], v[66:69], v[110:113], a[200:203]\n" \
  "v_mfma_f32_16x16x32_f16 a[200:203], v[62:65], v[142:145], a[200:203]\n" \
  "s_waitcnt lgkmcnt(0)\n" \
  "v_mfma_f32_16x16x32_f16 a[232:235], v[26:29], v[142:145], a[232:235]\n" \
  "s_sub_u32 s82, s81, s58\n" \
  "s_branch L_s33_%=\n" \
  "L_b33_%=:\n" \
  "v_mfma_f32_16x16x32_f16 a[232:235], v[22:25], v[110:113], a[232:235]\n" \
  "s_max_i32 s82, s82, -150\n" \
  "s_add_u32 s82, s82, %[phic]\n" \
  "s_add_u32 s81, s58, s82\n" \
  "v_mfma_f32_16x16x32_f16 a[236:239], v[22:25], v[114:117], a[236:239]\n" \
  "s_add_u32 s81, s81, s87\n" \
  "v_mfma_f32_16x16x32_f16 a[236:239], v[26:29], v[146:149], a[236:239]\n" \
  "v_mfma_f32_16x16x32_f16 a[204:207], v[62:65], v[146:149], a[204:207]\n" \
  "v_mfma_f32_16x16x32_f16 a[204:207], v[66:69], v[114:117], a[204:207]\n" \
  "v_mfma_f32_16x16x32_f16 a[208:211], v[66:69], v[118:121], a[208:211]\n" \
  "v_mfma_f32_16x16x32_f16 a[208:211], v[62:65], v[150:153], a[208:211]\n" \
  "v_mfma_f32_16x16x32_f16 a[240:243], v[26:29], v[150:153], a[240:243]\n" \
  "v_mfma_f32_16x16x32_f16 a[240:243], v[22:25], v[118:121], a[240:243]\n" \
  "v_readlane_b32 s74, %[vtabA], 12\n" \
  "v_mfma_f32_16x16x32_f16 a[244:247], v[22:25], v[122:125], a[244:247]\n" \
  "s_bitcmp1_b32 s34, 12\n" \
  "s_cbranch_scc0 L_s57_%=\n" \
  "L_b57_%=:\n" \
  "v_mfma_f32_16x16x32_f16 a[244:247], v[26:29], v[154:157], a[244:247]\n" \
  "s_waitcnt vmcnt(18)\n" \
  "s_add_u32 s72, s74, s68\n" \
  "v_mfma_f32_16x16x32_f16 a[212:215], v[62:65], v[154:157], a[212:215]\n" \
  "v_cvt_pk_f16_f32 v216, v198, v202\n" \
  "v_cvt_pk_f16_f32 v220, v199, v203\n" \
  "v_mfma_f32_16x16x32_f16 a[212:215], v[66:69], v[122:125], a[212:215]\n" \
  "s_waitcnt vmcnt(16)\n" \
  "v_cvt_pk_f16_f32 v224, v200, v204\n" \
  "v_cvt_pk_f16_f32 v228, v201, v205\n" \
  "v_mfma_f32_16x16x32_f16 a[216:219], v[66:69], v[126:129], a[216:219]\n" \
  "v_cvt_pk_f16_f32 v217, v206, v210\n" \
  "v_cvt_pk_f16_f32 v221, v207, v211\n" \
  "v_mfma_f32_16x16x32_f16 a[216:219], v[62:65], v[158:161], a[216:219]\n" \
  "v_cvt_pk_f16_f32 v225, v208, v212\n" \
  "v_cvt_pk_f16_f32 v229, v209, v213\n" \
  "v_mfma_f32_16x16x32_f16 a[248:251], v[26:29], v[158:161], a[248:251]\n" \
  "ds_write_b128 v244, v[214:217] offset:16384\n" \
  "v_mfma_f32_16x16x32_f16 a[248:251], v[22:25], v[126:129], a[248:251]\n" \
  "buffer_load_dwordx4 v[14:17], %[bxr], s[40:43], s72 offen offset:0 sc0 sc1\n" \
  "v_mfma_f32_16x16x32_f16 a[252:255], v[22:25], v[130:133], a[252:255]\n" \
  "v_readlane_b32 s75, %[vtabW], 19\n" \
  "v_mfma_f32_16x16x32_f16 a[252:255], v[26:29], v[162:165], a[252:255]\n" \
  "s_waitcnt vmcnt(11)\n" \
  "v_mfma_f32_16x16x32_f16 a[220:223], v[62:65], v[162:165], a[220:223]\n" \
  "ds_write_b128 v244, v[218:221] offset:17408\n" \
  "v_mfma_f32_16x16x32_f16 a[220:223], v[66:69], v[130:133], a[220:223]\n" \
  "buffer_load_dwordx4 v[18:21], %[bxr], s[40:43], s72 offen offset:1024 sc0 sc1\n" \
  "v_mfma_f32_16x16x32_f16 a[0:3], v[82:85], v[102:105], a[0:3]\n" \
  "s_add_u32 s60, s75, 0x0\n" \
  "s_add_u32 s61, s75, 0x10000\n" \
  "s_add_u32 s62, s75, 0x20000\n" \
  "v_mfma_f32_16x16x32_f16 a[0:3], v[78:81], v[134:137], a[0:3]\n" \
  "s_add_u32 s63, s75, 0x30000\n" \
  "s_add_u32 s64, s75, 0x100000\n" \
  "s_add_u32 s65, s75, 0x110000\n" \
  "v_mfma_f32_16x16x32_f16 a[32:35], v[42:45], v[134:137], a[32:35]\n" \
  "ds_write_b128 v244, v[222:225] offset:18432\n" \
  "v_mfma_f32_16x16x32_f16 a[32:35], v[38:41], v[102:105], a[32:35]\n" \
  "buffer_load_dwordx4 v[70:73], %[bxr], s[40:43], s72 offen offset:2048 sc0 sc1\n" \
  "v_mfma_f32_16x16x32_f16 a[36:39], v[38:41], v[106:109], a[36:39]\n" \
  "v_mfma_f32_16x16x32_f16 a[36:39], v[42:45], v[138:141], a[36:39]\n" \
  "s_add_u32 s73, s74, s69\n" \
  "v_mfma_f32_16x16x32_f16 a[4:7], v[78:81], v[138:141], a[4:7]\n" \
  "ds_write_b128 v244, v[226:229] offset:19456\n" \
  "v_mfma_f32_16x16x32_f16 a[4:7], v[82:85], v[106:109], a[4:7]\n" \
  "buffer_load_dwordx4 v[74:77], %[bxr], s[40:43], s72 offen offset:3072 sc0 sc1\n" \
  "v_mfma_f32_16x16x32_f16 a[8:11], v[82:85], v[110:113], a[8:11]\n" \
  "v_mfma_f32_16x16x32_f16 a[8:11], v[78:81], v[142:145], a[8:11]\n" \
  "v_mfma_f32_16x16x32_f16 a[40:43], v[42:45], v[142:145], a[40:43]\n" \
  "v_mfma_f32_16x16x32_f16 a[40:43], v[38:41], v[110:113], a[40:43]\n" \
  "v_cvt_pk_f16_f32 v214, v166, v170\n" \
  "v_cvt_pk_f16_f32 v218, v167, v171\n" \
  "v_mfma_f32_16x16x32_f16 a[44:47], v[38:41], v[114:117], a[44:47]\n" \
  "v_cvt_pk_f16_f32 v222, v168, v172\n" \
  "v_cvt_pk_f16_f32 v226, v169, v173\n" \
  "v_mfma_f32_16x16x32_f16 a[44:47], v[42:45], v[146:149], a[44:47]\n" \
  "buffer_load_dwordx4 v[30:33], %[bxr], s[40:43], s73 offen offset:0 sc0 sc1\n" \
  "v_mfma_f32_16x16x32_f16 a[12:15], v[78:81], v[146:149], a[12:15]\n" \
  "v_mfma_f32_16x16x32_f16 a[12:15], v[82:85], v[114:117], a[12:15]\n" \
  "v_mfma_f32_16x16x32_f16 a[16:19], v[82:85], v[118:121], a[16:19]\n" \
  "v_mfma_f32_16x16x32_f16 a[16:19], v[78:81], v[150:153], a[16:19]\n" \
  "buffer_load_dwordx4 v[34:37], %[bxr], s[40:43], s73 offen offset:1024 sc0 sc1\n" \
  "v_mfma_f32_16x16x32_f16 a[48:51], v[42:45], v[150:153], a[48:51]\n" \
  "v_mfma_f32_16x16x32_f16 a[48:51], v[38:41], v[118:121], a[48:51]\n" \
  "v_mfma_f32_16x16x32_f16 a[52:55], v[38:41], v[122:125], a[52:55]\n" \
  "v_mfma_f32_16x16x32_f16 a[52:55], v[42:45], v[154:157], a[52:55]\n" \
  "buffer_load_dwordx4 v[86:89], %[bxr], s[40:43], s73 offen offset:2048 sc0 sc1\n" \
  "v_mfma_f32_16x16x32_f16 a[20:23], v[78:81], v[154:157], a[20:23]\n" \
  "v_mfma_f32_16x16x32_f16 a[20:23], v[82:85], v[122:125], a[20:23]\n" \
  "s_andn2_b32 s79, 0xffff, s34\n" \
  "s_cbranch_scc1 L_pl12_%=\n" \
  "L_pb12_%=:\n" \
  "v_mfma_f32_16x16x32_f16 a[24:27], v[82:85], v[126:129], a[24:27]\n" \
  "s_barrier\n" \
  "v_mfma_f32_16x16x32_f16 a[24:27], v[78:81], v[158:161], a[24:27]\n" \
  "buffer_load_dwordx4 v[90:93], %[bxr], s[40:43], s73 offen offset:3072 sc0 sc1\n" \
  "v_mfma_f32_16x16x32_f16 a[56:59], v[42:45], v[158:161], a[56:59]\n" \
  "buffer_load_dwordx4 v[198:201], %[voffW], s[36:39], s60 offen nt\n" \
  "v_mfma_f32_16x16x32_f16 a[56:59], v[38:41], v[126:129], a[56:59]\n" \
  "v_mfma_f32_16x16x32_f16 a[60:63], v[38:41], v[130:133], a[60:63]\n" \
  "buffer_load_dwordx4 v[202:205], %[voffW], s[36:39], s61 offen nt\n" \
  "v_mfma_f32_16x16x32_f16 a[60:63], v[42:45], v[162:165], a[60:63]\n" \
  "v_mfma_f32_16x16x32_f16 a[28:31], v[78:81], v[162:165], a[28:31]\n" \
  "buffer_load_dwordx4 v[206:209], %[voffW], s[36:39], s62 offen nt\n" \
  "v_mfma_f32_16x16x32_f16 a[28:31], v[82:85], v[130:133], a[28:31]\n" \
  "s_waitcnt vmcnt(16)\n" \
  "v_mfma_f32_16x16x32_f16 a[128:131], v[98:101], v[102:105], a[128:131]\n" \
  "buffer_load_dwordx4 v[210:213], %[voffW], s[36:39], s63 offen nt\n" \
  "v_mfma_f32_16x16x32_f16 a[128:131], v[94:97], v[134:137], a[128:131]\n" \
  "v_mfma_f32_16x16x32_f16 a[160:163], v[58:61], v[134:137], a[160:163]\n" \
  "ds_read_b128 v[134:137], v246 offset:8192\n" \
  "v_mfma_f32_16x16x32_f16 a[160:163], v[54:57], v[102:105], a[160:163]\n" \
  "ds_read_b128 v[102:105], v246 offset:0\n" \
  "v_mfma_f32_16x16x32_f16 a[164:167], v[54:57], v[106:109], a[164:167]\n" \
  "buffer_load_dwordx4 v[166:169], %[voffW], s[36:39], s64 offen nt\n" \
  "v_mfma_f32_16x16x32_f16 a[164:167], v[58:61], v[138:141], a[164:167]\n" \
  "v_mfma_f32_16x16x32_f16 a[132:135], v[94:97], v[138:141], a[132:135]\n" \
  "ds_read_b128 v[138:141], v246 offset:9216\n" \
  "v_mfma_f32_16x16x32_f16 a[132:135], v[98:101], v[106:109], a[132:135]\n" \
  "ds_read_b128 v[106:109], v246 offset:1024\n" \
  "v_mfma_f32_16x16x32_f16 a[136:139], v[98:101], v[110:113], a[136:139]\n" \
  "buffer_load_dwordx4 v[170:173], %[voffW], s[36:39], s65 offen nt\n" \
  "v_mfma_f32_16x16x32_f16 a[136:139], v[94:97], v[142:145], a[136:139]\n" \
  "v_mfma_f32_16x16x32_f16 a[168:171], v[58:61], v[142:145], a[168:171]\n" \
  "ds_read_b128 v[142:145], v246 offset:10240\n" \
  "v_mfma_f32_16x16x32_f16 a[168:171], v[54:57], v[110:113], a[168:171]\n" \
  "ds_read_b128 v[110:113], v246 offset:2048\n" \
  "v_mfma_f32_16x16x32_f16 a[172:175], v[54:57], v[114:117], a[172:175]\n" \
  "v_mfma_f32_16x16x32_f16 a[172:175], v[58:61], v[146:149], a[172:175]\n" \
  "v_mfma_f32_16x16x32_f16 a[140:143], v[94:97], v[146:149], a[140:143]\n" \
  "ds_read_b128 v[146:149], v246 offset:11264\n" \
  "v_mfma_f32_16x16x32_f16 a[140:143], v[98:101], v[114:117], a[140:143]\n" \
  "ds_read_b128 v[114:117], v246 offset:3072\n" \
  "v_mfma_f32_16x16x32_f16 a[144:147], v[98:101], v[118:121], a[144:147]\n" \
  "s_add_u32 s72, s74, s70\n" \
  "v_mfma_f32_16x16x32_f16 a[144:147], v[94:97], v[150:153], a[144:147]\n" \
  "v_mfma_f32_16x16x32_f16 a[176:179], v[58:61], v[150:153], a[176:179]\n" \
  "ds_read_b128 v[150:153], v246 offset:12288\n" \
  "v_mfma_f32_16x16x32_f16 a[176:179], v[54:57], v[118:121], a[176:179]\n" \
  "ds_read_b128 v[118:121], v246 offset:4096\n" \
  "v_mfma_f32_16x16x32_f16 a[180:183], v[54:57], v[122:125], a[180:183]\n" \
  "v_mfma_f32_16x16x32_f16 a[180:183], v[58:61], v[154:157], a[180:183]\n" \
  "v_mfma_f32_16x16x32_f16 a[148:151], v[94:97], v[154:157], a[148:151]\n" \
  "ds_read_b128 v[154:157], v246 offset:13312\n" \
  "v_mfma_f32_16x16x32_f16 a[148:151], v[98:101], v[122:125], a[148:151]\n" \
  "ds_read_b128 v[122:125], v246 offset:5120\n" \
  "v_mfma_f32_16x16x32_f16 a[152:155], v[98:101], v[126:129], a[152:155]\n" \
  "buffer_load_dwordx4 v[46:49], %[bxr], s[40:43], s72 offen offset:0 sc0 sc1\n" \
  "v_mfma_f32_16x16x32_f16 a[152:155], v[94:97], v[158:161], a[152:155]\n" \
  "v_mfma_f32_16x16x32_f16 a[184:187], v[58:61], v[158:161], a[184:187]\n" \
  "ds_read_b128 v[158:161], v246 offset:14336\n" \
  "v_mfma_f32_16x16x32_f16 a[184:187], v[54:57], v[126:129], a[184:187]\n" \
  "ds_read_b128 v[126:129], v246 offset:6144\n" \
  "v_mfma_f32_16x16x32_f16 a[188:191], v[54:57], v[130:133], a[188:191]\n" \
  "buffer_load_dwordx4 v[50:53], %[bxr], s[40:43], s72 offen offset:1024 sc0 sc1\n" \
  "v_mfma_f32_16x16x32_f16 a[188:191], v[58:61], v[162:165], a[188:191]\n" \
  "s_add_u32 s66, s75, 0x120000\n" \
  "s_add_u32 s67, s75, 0x130000\n" \
  "v_mfma_f32_16x16x32_f16 a[156:159], v[94:97], v[162:165], a[156:159]\n" \
  "ds_read_b128 v[162:165], v246 offset:15360\n" \
  "v_readlane_b32 s75, %[vtabW], 20\n" \
  "v_mfma_f32_16x16x32_f16 a[156:159], v[98:101], v[130:133], a[156:159]\n" \
  "s_waitcnt vmcnt(12)\n" \
  "ds_read_b128 v[130:133], v246 offset:7168\n" \
  "s_waitcnt lgkmcnt(14)\n" \
    \
  "v_mfma_f32_16x16x32_f16 a[64:67], v[14:17], v[102:105], a[64:67]\n" \
  "buffer_load_dwordx4 v[22:25], %[bxr], s[40:43], s72 offen offset:2048 sc0 sc1\n" \
  "v_mfma_f32_16x16x32_f16 a[64:67], v[18:21], v[134:137], a[64:67]\n" \
  "s_add_u32 s60, s75, 0x0\n" \
  "s_add_u32 s61, s75, 0x10000\n" \
  "s_add_u32 s62, s75, 0x20000\n" \
  "v_mfma_f32_16x16x32_f16 a[96:99], v[74:77], v[134:137], a[96:99]\n" \
  "s_add_u32 s63, s75, 0x30000\n" \
  "s_add_u32 s73, s74, s71\n" \
  "v_mfma_f32_16x16x32_f16 a[96:99], v[70:73], v[102:105], a[96:99]\n" \
  "s_waitcnt lgkmcnt(12)\n" \
  "v_mfma_f32_16x16x32_f16 a[100:103], v[70:73], v[106:109], a[100:103]\n" \
  "buffer_load_dwordx4 v[26:29], %[bxr], s[40:43], s72 offen offset:3072 sc0 sc1\n" \
  "v_mfma_f32_16x16x32_f16 a[100:103], v[74:77], v[138:141], a[100:103]\n" \
  "v_mfma_f32_16x16x32_f16 a[68:71], v[18:21], v[138:141], a[68:71]\n" \
  "v_mfma_f32_16x16x32_f16 a[68:71], v[14:17], v[106:109], a[68:71]\n" \
  "s_waitcnt lgkmcnt(10)\n" \
  "v_mfma_f32_16x16x32_f16 a[72:75], v[14:17], v[110:113], a[72:75]\n" \
  "buffer_load_dwordx4 v[62:65], %[bxr], s[40:43], s73 offen offset:0 sc0 sc1\n" \
  "v_mfma_f32_16x16x32_f16 a[72:75], v[18:21], v[142:145], a[72:75]\n" \
  "v_mfma_f32_16x16x32_f16 a[104:107], v[74:77], v[142:145], a[104:107]\n" \
  "v_mfma_f32_16x16x32_f16 a[104:107], v[70:73], v[110:113], a[104:107]\n" \
  "s_waitcnt lgkmcnt(8)\n" \
  "v_cvt_pk_f16_f32 v215, v174, v178\n" \
  "v_cvt_pk_f16_f32 v219, v175, v179\n" \
  "v_mfma_f32_16x16x32_f16 a[108:111], v[70:73], v[114:117], a[108:111]\n" \
  "v_cvt_pk_f16_f32 v223, v176, v180\n" \
  "v_cvt_pk_f16_f32 v227, v177, v181\n" \
  "v_mfma_f32_16x16x32_f16 a[108:111], v[74:77], v[146:149], a[108:111]\n" \
  "s_waitcnt vmcnt(12)\n" \
  "v_cvt_pk_f16_f32 v216, v182, v186\n" \
  "v_cvt_pk_f16_f32 v220, v183, v187\n" \
  "v_mfma_f32_16x16x32_f16 a[76:79], v[18:21], v[146:149], a[76:79]\n" \
  "v_cvt_pk_f16_f32 v224, v184, v188\n" \
  "v_cvt_pk_f16_f32 v228, v185, v189\n" \
  "v_mfma_f32_16x16x32_f16 a[76:79], v[14:17], v[114:117], a[76:79]\n" \
  "s_waitcnt lgkmcnt(6)\n" \
  "v_cvt_pk_f16_f32 v217, v190, v194\n" \
  "v_cvt_pk_f16_f32 v221, v191, v195\n" \
  "v_mfma_f32_16x16x32_f16 a[80:83], v[14:17], v[118:121], a[80:83]\n" \
  "v_cvt_pk_f16_f32 v225, v192, v196\n" \
  "v_cvt_pk_f16_f32 v229, v193, v197\n" \
  "v_mfma_f32_16x16x32_f16 a[80:83], v[18:21], v[150:153], a[80:83]\n" \
  "ds_write_b128 v242, v[214:217] offset:0\n" \
  "v_mfma_f32_16x16x32_f16 a[112:115], v[74:77], v[150:153], a[112:115]\n" \
  "buffer_load_dwordx4 v[66:69], %[bxr], s[40:43], s73 offen offset:1024 sc0 sc1\n" \
  "v_mfma_f32_16x16x32_f16 a[112:115], v[70:73], v[118:121], a[112:115]\n" \
  "s_waitcnt lgkmcnt(5)\n" \
  "v_min3_u32 v248, v238, v239, v240\n" \
  "v_min_u32 v248, v248, v241\n" \
  "v_mfma_f32_16x16x32_f16 a[116:119], v[70:73], v[122:125], a[116:119]\n" \
  "v_cmp_le_u32 s[34:35], s76, v248\n" \
  "v_mfma_f32_16x16x32_f16 a[116:119], v[74:77], v[154:157], a[116:119]\n" \
  "ds_write_b128 v242, v[218:221] offset:1024\n" \
  "v_mfma_f32_16x16x32_f16 a[84:87], v[18:21], v[154:157], a[84:87]\n" \
  "buffer_load_dwordx4 v[38:41], %[bxr], s[40:43], s73 offen offset:2048 sc0 sc1\n" \
  "v_mfma_f32_16x16x32_f16 a[84:87], v[14:17], v[122:125], a[84:87]\n" \
  "s_waitcnt lgkmcnt(4)\n" \
  "v_mfma_f32_16x16x32_f16 a[88:91], v[14:17], v[126:129], a[88:91]\n" \
  "s_waitcnt lgkmcnt(2)\n" \
  "s_barrier\n" \
  "v_mfma_f32_16x16x32_f16 a[88:91], v[18:21], v[158:161], a[88:91]\n" \
  "ds_write_b128 v242, v[222:225] offset:2048\n" \
  "v_mfma_f32_16x16x32_f16 a[120:123], v[74:77], v[158:161], a[120:123]\n" \
  "buffer_load_dwordx4 v[174:177], %[voffW], s[36:39], s66 offen nt\n" \
  "v_mfma_f32_16x16x32_f16 a[120:123], v[70:73], v[126:129], a[120:123]\n" \
  "buffer_load_dwordx4 v[42:45], %[bxr], s[40:43], s73 offen offset:3072 sc0 sc1\n" \
  "v_mfma_f32_16x16x32_f16 a[124:127], v[70:73], v[130:133], a[124:127]\n" \
  "buffer_load_dwordx4 v[178:181], %[voffW], s[36:39], s67 offen nt\n" \
  "v_mfma_f32_16x16x32_f16 a[124:127], v[74:77], v[162:165], a[124:127]\n" \
  "ds_write_b128 v242, v[226:229] offset:3072\n" \
  "v_mfma_f32_16x16x32_f16 a[92:95], v[18:21], v[162:165], a[92:95]\n" \
  "buffer_load_dwordx4 v[182:185], %[voffW], s[36:39], s60 offen nt\n" \
  "v_mfma_f32_16x16x32_f16 a[92:95], v[14:17], v[130:133], a[92:95]\n" \
  "s_waitcnt vmcnt(17)\n" \
  "v_mfma_f32_16x16x32_f16 a[192:195], v[30:33], v[102:105], a[192:195]\n" \
  "buffer_load_dwordx4 v[186:189], %[voffW], s[36:39], s61 offen nt\n" \
  "v_mfma_f32_16x16x32_f16 a[192:195], v[34:37], v[134:137], a[192:195]\n" \
  "s_memrealtime s[58:59]\n" \
  "v_mfma_f32_16x16x32_f16 a[224:227], v[90:93], v[134:137], a[224:227]\n" \
  "buffer_load_dwordx4 v[190:193], %[voffW], s[36:39], s62 offen nt\n" \
  "v_mfma_f32_16x16x32_f16 a[224:227], v[86:89], v[102:105], a[224:227]\n" \
  "v_mfma_f32_16x16x32_f16 a[228:231], v[86:89], v[106:109], a[228:231]\n" \
  "buffer_load_dwordx4 v[194:197], %[voffW], s[36:39], s63 offen nt\n" \
  "v_mfma_f32_16x16x32_f16 a[228:231], v[90:93], v[138:141], a[228:231]\n" \
  "v_mfma_f32_16x16x32_f16 a[196:199], v[34:37], v[138:141], a[196:199]\n" \
  "v_mfma_f32_16x16x32_f16 a[196:199], v[30:33], v[106:109], a[196:199]\n" \
  "v_mfma_f32_16x16x32_f16 a[200:203], v[30:33], v[110:113], a[200:203]\n" \
  "v_mfma_f32_16x16x32_f16 a[200:203], v[34:37], v[142:145], a[200:203]\n" \
  "s_waitcnt lgkmcnt(0)\n" \
  "v_mfma_f32_16x16x32_f16 a[232:235], v[90:93], v[142:145], a[232:235]\n" \
  "s_sub_u32 s82, s81, s58\n" \
  "s_branch L_s34_%=\n" \
  "L_b34_%=:\n" \
  "v_mfma_f32_16x16x32_f16 a[232:235], v[86:89], v[110:113], a[232:235]\n" \
  "s_max_i32 s82, s82, -150\n" \
  "s_add_u32 s82, s82, %[phic]\n" \
  "s_add_u32 s81, s58, s82\n" \
  "v_mfma_f32_16x16x32_f16 a[236:239], v[86:89], v[114:117], a[236:239]\n" \
  "s_add_u32 s81, s81, s87\n" \
  "v_mfma_f32_16x16x32_f16 a[236:239], v[90:93], v[146:149], a[236:239]\n" \
  "v_mfma_f32_16x16x32_f16 a[204:207], v[34:37], v[146:149], a[204:207]\n" \
  "v_readlane_b32 s74, %[vtabA], 13\n" \
  "v_mfma_f32_16x16x32_f16 a[204:207], v[30:33], v[114:117], a[204:207]\n" \
  "s_bitcmp1_b32 s34, 13\n" \
  "s_cbranch_scc0 L_s58_%=\n" \
  "L_b58_%=:\n" \
  "v_mfma_f32_16x16x32_f16 a[208:211], v[30:33], v[118:121], a[208:211]\n" \
  "s_add_u32 s72, s74, s68\n" \
  "v_mfma_f32_16x16x32_f16 a[208:211], v[34:37], v[150:153], a[208:211]\n" \
  "v_mfma_f32_16x16x32_f16 a[240:243], v[90:93], v[150:153], a[240:243]\n" \
  "v_mfma_f32_16x16x32_f16 a[240:243], v[86:89], v[118:121], a[240:243]\n" \
  "v_mfma_f32_16x16x32_f16 a[244:247], v[86:89], v[122:125], a[244:247]\n" \
  "v_mfma_f32_16x16x32_f16 a[244:247], v[90:93], v[154:157], a[244:247]\n" \
  "v_mfma_f32_16x16x32_f16 a[212:215], v[34:37], v[154:157], a[212:215]\n" \
  "buffer_load_dwordx4 v[78:81], %[bxr], s[40:43], s72 offen offset:0 sc0 sc1\n" \
  "v_mfma_f32_16x16x32_f16 a[212:215], v[30:33], v[122:125], a[212:215]\n" \
  "v_mfma_f32_16x16x32_f16 a[216:219], v[30:33], v[126:129], a[216:219]\n" \
  "v_mfma_f32_16x16x32_f16 a[216:219], v[34:37], v[158:161], a[216:219]\n" \
  "v_mfma_f32_16x16x32_f16 a[248:251], v[90:93], v[158:161], a[248:251]\n" \
  "buffer_load_dwordx4 v[82:85], %[bxr], s[40:43], s72 offen offset:1024 sc0 sc1\n" \
  "v_mfma_f32_16x16x32_f16 a[248:251], v[86:89], v[126:129], a[248:251]\n" \
  "v_mfma_f32_16x16x32_f16 a[252:255], v[86:89], v[130:133], a[252:255]\n" \
  "s_add_u32 s64, s75, 0x100000\n" \
  "s_add_u32 s65, s75, 0x110000\n" \
  "s_add_u32 s66, s75, 0x120000\n" \
  "v_mfma_f32_16x16x32_f16 a[252:255], v[90:93], v[162:165], a[252:255]\n" \
  "s_add_u32 s67, s75, 0x130000\n" \
  "v_mfma_f32_16x16x32_f16 a[220:223], v[34:37], v[162:165], a[220:223]\n" \
  "buffer_load_dwordx4 v[54:57], %[bxr], s[40:43], s72 offen offset:2048 sc0 sc1\n" \
  "v_mfma_f32_16x16x32_f16 a[220:223], v[30:33], v[130:133], a[220:223]\n" \
  "s_waitcnt vmcnt(13)\n" \
  "v_readlane_b32 s75, %[vtabW], 21\n" \
  "s_add_u32 s73, s74, s69\n" \
  "v_mfma_f32_16x16x32_f16 a[0:3], v[46:49], v[102:105], a[0:3]\n" \
  "v_mfma_f32_16x16x32_f16 a[0:3], v[50:53], v[134:137], a[0:3]\n" \
  "s_add_u32 s60, s75, 0x0\n" \
  "s_add_u32 s61, s75, 0x10000\n" \
  "v_mfma_f32_16x16x32_f16 a[32:35], v[26:29], v[134:137], a[32:35]\n" \
  "buffer_load_dwordx4 v[58:61], %[bxr], s[40:43], s72 offen offset:3072 sc0 sc1\n" \
  "v_mfma_f32_16x16x32_f16 a[32:35], v[22:25], v[102:105], a[32:35]\n" \
  "v_mfma_f32_16x16x32_f16 a[36:39], v[22:25], v[106:109], a[36:39]\n" \
  "v_mfma_f32_16x16x32_f16 a[36:39], v[26:29], v[138:141], a[36:39]\n" \
  "v_mfma_f32_16x16x32_f16 a[4:7], v[50:53], v[138:141], a[4:7]\n" \
  "buffer_load_dwordx4 v[94:97], %[bxr], s[40:43], s73 offen offset:0 sc0 sc1\n" \
  "v_mfma_f32_16x16x32_f16 a[4:7], v[46:49], v[106:109], a[4:7]\n" \
  "v_mfma_f32_16x16x32_f16 a[8:11], v[46:49], v[110:113], a[8:11]\n" \
  "v_mfma_f32_16x16x32_f16 a[8:11], v[50:53], v[142:145], a[8:11]\n" \
  "v_mfma_f32_16x16x32_f16 a[40:43], v[26:29], v[142:145], a[40:43]\n" \
  "v_mfma_f32_16x16x32_f16 a[40:43], v[22:25], v[110:113], a[40:43]\n" \
  "v_cvt_pk_f16_f32 v214, v198, v202\n" \
  "v_cvt_pk_f16_f32 v218, v199, v203\n" \
  "v_mfma_f32_16x16x32_f16 a[44:47], v[22:25], v[114:117], a[44:47]\n" \
  "v_cvt_pk_f16_f32 v222, v200, v204\n" \
  "v_cvt_pk_f16_f32 v226, v201, v205\n" \
  "v_mfma_f32_16x16x32_f16 a[44:47], v[26:29], v[146:149], a[44:47]\n" \
  "v_cvt_pk_f16_f32 v215, v206, v210\n" \
  "v_cvt_pk_f16_f32 v219, v207, v211\n" \
  "v_mfma_f32_16x16x32_f16 a[12:15], v[50:53], v[146:149], a[12:15]\n" \
  "v_cvt_pk_f16_f32 v223, v208, v212\n" \
  "v_cvt_pk_f16_f32 v227, v209, v213\n" \
  "v_mfma_f32_16x16x32_f16 a[12:15], v[46:49], v[114:117], a[12:15]\n" \
  "v_cvt_pk_f16_f32 v216, v166, v170\n" \
  "v_cvt_pk_f16_f32 v220, v167, v171\n" \
  "v_mfma_f32_16x16x32_f16 a[16:19], v[46:49], v[118:121], a[16:19]\n" \
  "v_cvt_pk_f16_f32 v224, v168, v172\n" \
  "v_cvt_pk_f16_f32 v228, v169, v173\n" \
  "v_mfma_f32_16x16x32_f16 a[16:19], v[50:53], v[150:153], a[16:19]\n" \
  "buffer_load_dwordx4 v[98:101], %[bxr], s[40:43], s73 offen offset:1024 sc0 sc1\n" \
  "v_mfma_f32_16x16x32_f16 a[48:51], v[26:29], v[150:153], a[48:51]\n" \
  "v_mfma_f32_16x16x32_f16 a[48:51], v[22:25], v[118:121], a[48:51]\n" \
  "v_mfma_f32_16x16x32_f16 a[52:55], v[22:25], v[122:125], a[52:55]\n" \
  "v_mfma_f32_16x16x32_f16 a[52:55], v[26:29], v[154:157], a[52:55]\n" \
  "buffer_load_dwordx4 v[70:73], %[bxr], s[40:43], s73 offen offset:2048 sc0 sc1\n" \
  "v_mfma_f32_16x16x32_f16 a[20:23], v[50:53], v[154:157], a[20:23]\n" \
  "v_mfma_f32_16x16x32_f16 a[20:23], v[46:49], v[122:125], a[20:23]\n" \
  "s_andn2_b32 s79, 0x1ffff, s34\n" \
  "s_cbranch_scc1 L_pl13_%=\n" \
  "L_pb13_%=:\n" \
  "v_mfma_f32_16x16x32_f16 a[24:27], v[46:49], v[126:129], a[24:27]\n" \
  "s_barrier\n" \
  "v_mfma_f32_16x16x32_f16 a[24:27], v[50:53], v[158:161], a[24:27]\n" \
  "buffer_load_dwordx4 v[74:77], %[bxr], s[40:43], s73 offen offset:3072 sc0 sc1\n" \
  "v_mfma_f32_16x16x32_f16 a[56:59], v[26:29], v[158:161], a[56:59]\n" \
  "buffer_load_dwordx4 v[198:201], %[voffW], s[36:39], s64 offen nt\n" \
  "v_mfma_f32_16x16x32_f16 a[56:59], v[22:25], v[126:129], a[56:59]\n" \
  "v_mfma_f32_16x16x32_f16 a[60:63], v[22:25], v[130:133], a[60:63]\n" \
  "buffer_load_dwordx4 v[202:205], %[voffW], s[36:39], s65 offen nt\n" \
  "v_mfma_f32_16x16x32_f16 a[60:63], v[26:29], v[162:165], a[60:63]\n" \
  "v_mfma_f32_16x16x32_f16 a[28:31], v[50:53], v[162:165], a[28:31]\n" \
  "buffer_load_dwordx4 v[206:209], %[voffW], s[36:39], s66 offen nt\n" \
  "v_mfma_f32_16x16x32_f16 a[28:31], v[46:49], v[130:133], a[28:31]\n" \
  "s_waitcnt vmcnt(16)\n" \
  "v_mfma_f32_16x16x32_f16 a[128:131], v[62:65], v[102:105], a[128:131]\n" \
  "buffer_load_dwordx4 v[210:213], %[voffW], s[36:39], s67 offen nt\n" \
  "v_mfma_f32_16x16x32_f16 a[128:131], v[66:69], v[134:137], a[128:131]\n" \
  "v_mfma_f32_16x16x32_f16 a[160:163], v[42:45], v[134:137], a[160:163]\n" \
  "ds_read_b128 v[134:137], v246 offset:24576\n" \
  "v_mfma_f32_16x16x32_f16 a[160:163], v[38:41], v[102:105], a[160:163]\n" \
  "ds_read_b128 v[102:105], v246 offset:16384\n" \
  "v_mfma_f32_16x16x32_f16 a[164:167], v[38:41], v[106:109], a[164:167]\n" \
  "buffer_load_dwordx4 v[166:169], %[voffW], s[36:39], s60 offen nt\n" \
  "v_mfma_f32_16x16x32_f16 a[164:167], v[42:45], v[138:141], a[164:167]\n" \
  "v_mfma_f32_16x16x32_f16 a[132:135], v[66:69], v[138:141], a[132:135]\n" \
  "ds_read_b128 v[138:141], v246 offset:25600\n" \
  "v_mfma_f32_16x16x32_f16 a[132:135], v[62:65], v[106:109], a[132:135]\n" \
  "ds_read_b128 v[106:109], v246 offset:17408\n" \
  "v_mfma_f32_16x16x32_f16 a[136:139], v[62:65], v[110:113], a[136:139]\n" \
  "buffer_load_dwordx4 v[170:173], %[voffW], s[36:39], s61 offen nt\n" \
  "v_mfma_f32_16x16x32_f16 a[136:139], v[66:69], v[142:145], a[136:139]\n" \
  "v_mfma_f32_16x16x32_f16 a[168:171], v[42:45], v[142:145], a[168:171]\n" \
  "ds_read_b128 v[142:145], v246 offset:26624\n" \
  "v_mfma_f32_16x16x32_f16 a[168:171], v[38:41], v[110:113], a[168:171]\n" \
  "ds_read_b128 v[110:113], v246 offset:18432\n" \
  "v_mfma_f32_16x16x32_f16 a[172:175], v[38:41], v[114:117], a[172:175]\n" \
  "v_mfma_f32_16x16x32_f16 a[172:175], v[42:45], v[146:149], a[172:175]\n" \
  "v_mfma_f32_16x16x32_f16 a[140:143], v[66:69], v[146:149], a[140:143]\n" \
  "ds_read_b128 v[146:149], v246 offset:27648\n" \
  "v_mfma_f32_16x16x32_f16 a[140:143], v[62:65], v[114:117], a[140:143]\n" \
  "ds_read_b128 v[114:117], v246 offset:19456\n" \
  "v_mfma_f32_16x16x32_f16 a[144:147], v[62:65], v[118:121], a[144:147]\n" \
  "v_mfma_f32_16x16x32_f16 a[144:147], v[66:69], v[150:153], a[144:147]\n" \
  "v_mfma_f32_16x16x32_f16 a[176:179], v[42:45], v[150:153], a[176:179]\n" \
  "ds_read_b128 v[150:153], v246 offset:28672\n" \
  "v_mfma_f32_16x16x32_f16 a[176:179], v[38:41], v[118:121], a[176:179]\n" \
  "ds_read_b128 v[118:121], v246 offset:20480\n" \
  "v_mfma_f32_16x16x32_f16 a[180:183], v[38:41], v[122:125], a[180:183]\n" \
  "s_add_u32 s72, s74, s70\n" \
  "v_mfma_f32_16x16x32_f16 a[180:183], v[42:45], v[154:157], a[180:183]\n" \
  "s_waitcnt vmcnt(18)\n" \
  "v_mfma_f32_16x16x32_f16 a[148:151], v[66:69], v[154:157], a[148:151]\n" \
  "ds_read_b128 v[154:157], v246 offset:29696\n" \
  "v_cvt_pk_f16_f32 v217, v174, v178\n" \
  "v_mfma_f32_16x16x32_f16 a[148:151], v[62:65], v[122:125], a[148:151]\n" \
  "ds_read_b128 v[122:125], v246 offset:21504\n" \
  "v_cvt_pk_f16_f32 v221, v175, v179\n" \
  "v_mfma_f32_16x16x32_f16 a[152:155], v[62:65], v[126:129], a[152:155]\n" \
  "v_cvt_pk_f16_f32 v225, v176, v180\n" \
  "v_cvt_pk_f16_f32 v229, v177, v181\n" \
  "v_mfma_f32_16x16x32_f16 a[152:155], v[66:69], v[158:161], a[152:155]\n" \
  "ds_write_b128 v242, v[214:217] offset:16384\n" \
  "v_mfma_f32_16x16x32_f16 a[184:187], v[42:45], v[158:161], a[184:187]\n" \
  "ds_read_b128 v[158:161], v246 offset:30720\n" \
  "v_mfma_f32_16x16x32_f16 a[184:187], v[38:41], v[126:129], a[184:187]\n" \
  "ds_read_b128 v[126:129], v246 offset:22528\n" \
  "v_mfma_f32_16x16x32_f16 a[188:191], v[38:41], v[130:133], a[188:191]\n" \
  "buffer_load_dwordx4 v[18:21], %[bxr], s[40:43], s72 offen offset:0 sc0 sc1\n" \
  "v_mfma_f32_16x16x32_f16 a[188:191], v[42:45], v[162:165], a[188:191]\n" \
  "ds_write_b128 v242, v[218:221] offset:17408\n" \
  "v_mfma_f32_16x16x32_f16 a[156:159], v[66:69], v[162:165], a[156:159]\n" \
  "s_add_u32 s62, s75, 0x20000\n" \
  "ds_read_b128 v[162:165], v246 offset:31744\n" \
  "s_add_u32 s63, s75, 0x30000\n" \
  "v_mfma_f32_16x16x32_f16 a[156:159], v[62:65], v[130:133], a[156:159]\n" \
  "s_waitcnt vmcnt(11)\n" \
  "ds_read_b128 v[130:133], v246 offset:23552\n" \
  "s_waitcnt lgkmcnt(14)\n" \
    \
  "v_mfma_f32_16x16x32_f16 a[64:67], v[78:81], v[102:105], a[64:67]\n" \
  "buffer_load_dwordx4 v[14:17], %[bxr], s[40:43], s72 offen offset:1024 sc0 sc1\n" \
  "v_mfma_f32_16x16x32_f16 a[64:67], v[82:85], v[134:137], a[64:67]\n" \
  "ds_write_b128 v242, v[222:225] offset:18432\n" \
  "v_mfma_f32_16x16x32_f16 a[96:99], v[58:61], v[134:137], a[96:99]\n" \
  "s_add_u32 s64, s75, 0x100000\n" \
  "s_add_u32 s65, s75, 0x110000\n" \
  "s_add_u32 s66, s75, 0x120000\n" \
  "v_mfma_f32_16x16x32_f16 a[96:99], v[54:57], v[102:105], a[96:99]\n" \
  "s_waitcnt lgkmcnt(14)\n" \
  "s_add_u32 s67, s75, 0x130000\n" \
  "v_mfma_f32_16x16x32_f16 a[100:103], v[54:57], v[106:109], a[100:103]\n" \
  "buffer_load_dwordx4 v[86:89], %[bxr], s[40:43], s72 offen offset:2048 sc0 sc1\n" \
  "v_mfma_f32_16x16x32_f16 a[100:103], v[58:61], v[138:141], a[100:103]\n" \
  "ds_write_b128 v242, v[226:229] offset:19456\n" \
  "v_mfma_f32_16x16x32_f16 a[68:71], v[82:85], v[138:141], a[68:71]\n" \
  "s_add_u32 s73, s74, s71\n" \
  "v_mfma_f32_16x16x32_f16 a[68:71], v[78:81], v[106:109], a[68:71]\n" \
  "s_waitcnt lgkmcnt(14)\n" \
  "v_mfma_f32_16x16x32_f16 a[72:75], v[78:81], v[110:113], a[72:75]\n" \
  "buffer_load_dwordx4 v[90:93], %[bxr], s[40:43], s72 offen offset:3072 sc0 sc1\n" \
  "v_mfma_f32_16x16x32_f16 a[72:75], v[82:85], v[142:145], a[72:75]\n" \
  "v_mfma_f32_16x16x32_f16 a[104:107], v[58:61], v[142:145], a[104:107]\n" \
  "s_waitcnt lgkmcnt(12)\n" \
  "v_cvt_pk_f16_f32 v214, v182, v186\n" \
  "v_cvt_pk_f16_f32 v218, v183, v187\n" \
  "v_mfma_f32_16x16x32_f16 a[104:107], v[54:57], v[110:113], a[104:107]\n" \
  "v_cvt_pk_f16_f32 v222, v184, v188\n" \
  "v_cvt_pk_f16_f32 v226, v185, v189\n" \
  "v_mfma_f32_16x16x32_f16 a[108:111], v[54:57], v[114:117], a[108:111]\n" \
  "v_cvt_pk_f16_f32 v215, v190, v194\n" \
  "v_cvt_pk_f16_f32 v219, v191, v195\n" \
  "v_mfma_f32_16x16x32_f16 a[108:111], v[58:61], v[146:149], a[108:111]\n" \
  "s_waitcnt vmcnt(11)\n" \
  "v_cvt_pk_f16_f32 v223, v192, v196\n" \
  "v_cvt_pk_f16_f32 v227, v193, v197\n" \
  "v_mfma_f32_16x16x32_f16 a[76:79], v[82:85], v[146:149], a[76:79]\n" \
  "buffer_load_dwordx4 v[34:37], %[bxr], s[40:43], s73 offen offset:0 sc0 sc1\n" \
  "v_mfma_f32_16x16x32_f16 a[76:79], v[78:81], v[114:117], a[76:79]\n" \
  "s_waitcnt lgkmcnt(10)\n" \
  "v_min3_u32 v248, v238, v239, v240\n" \
  "v_min_u32 v248, v248, v241\n" \
  "v_mfma_f32_16x16x32_f16 a[80:83], v[78:81], v[118:121], a[80:83]\n" \
  "v_cmp_le_u32 s[34:35], s76, v248\n" \
  "v_mfma_f32_16x16x32_f16 a[80:83], v[82:85], v[150:153], a[80:83]\n" \
  "v_mfma_f32_16x16x32_f16 a[112:115], v[58:61], v[150:153], a[112:115]\n" \
  "buffer_load_dwordx4 v[30:33], %[bxr], s[40:43], s73 offen offset:1024 sc0 sc1\n" \
  "v_mfma_f32_16x16x32_f16 a[112:115], v[54:57], v[118:121], a[112:115]\n" \
  "s_waitcnt lgkmcnt(8)\n" \
  "v_mfma_f32_16x16x32_f16 a[116:119], v[54:57], v[122:125], a[116:119]\n" \
  "v_mfma_f32_16x16x32_f16 a[116:119], v[58:61], v[154:157], a[116:119]\n" \
  "v_mfma_f32_16x16x32_f16 a[84:87], v[82:85], v[154:157], a[84:87]\n" \
  "buffer_load_dwordx4 v[22:25], %[bxr], s[40:43], s73 offen offset:2048 sc0 sc1\n" \
  "v_mfma_f32_16x16x32_f16 a[84:87], v[78:81], v[122:125], a[84:87]\n" \
  "s_waitcnt lgkmcnt(5)\n" \
  "v_mfma_f32_16x16x32_f16 a[88:91], v[78:81], v[126:129], a[88:91]\n" \
  "v_mfma_f32_16x16x32_f16 a[88:91], v[82:85], v[158:161], a[88:91]\n" \
  "s_waitcnt lgkmcnt(2)\n" \
  "s_barrier\n" \
  "v_mfma_f32_16x16x32_f16 a[120:123], v[58:61], v[158:161], a[120:123]\n" \
  "buffer_load_dwordx4 v[174:177], %[voffW], s[36:39], s62 offen nt\n" \
  "v_mfma_f32_16x16x32_f16 a[120:123], v[54:57], v[126:129], a[120:123]\n" \
  "buffer_load_dwordx4 v[26:29], %[bxr], s[40:43], s73 offen offset:3072 sc0 sc1\n" \
  "v_mfma_f32_16x16x32_f16 a[124:127], v[54:57], v[130:133], a[124:127]\n" \
  "buffer_load_dwordx4 v[178:181], %[voffW], s[36:39], s63 offen nt\n" \
  "v_mfma_f32_16x16x32_f16 a[124:127], v[58:61], v[162:165], a[124:127]\n" \
  "v_mfma_f32_16x16x32_f16 a[92:95], v[82:85], v[162:165], a[92:95]\n" \
  "buffer_load_dwordx4 v[182:185], %[voffW], s[36:39], s64 offen nt\n" \
  "v_mfma_f32_16x16x32_f16 a[92:95], v[78:81], v[130:133], a[92:95]\n" \
  "s_waitcnt vmcnt(17)\n" \
  "v_mfma_f32_16x16x32_f16 a[192:195], v[94:97], v[102:105], a[192:195]\n" \
  "buffer_load_dwordx4 v[186:189], %[voffW], s[36:39], s65 offen nt\n" \
  "v_mfma_f32_16x16x32_f16 a[192:195], v[98:101], v[134:137], a[192:195]\n" \
  "s_memrealtime s[58:59]\n" \
  "v_mfma_f32_16x16x32_f16 a[224:227], v[74:77], v[134:137], a[224:227]\n" \
  "buffer_load_dwordx4 v[190:193], %[voffW], s[36:39], s66 offen nt\n" \
  "v_mfma_f32_16x16x32_f16 a[224:227], v[70:73], v[102:105], a[224:227]\n" \
  "v_mfma_f32_16x16x32_f16 a[228:231], v[70:73], v[106:109], a[228:231]\n" \
  "buffer_load_dwordx4 v[194:197], %[voffW], s[36:39], s67 offen nt\n" \
  "v_mfma_f32_16x16x32_f16 a[228:231], v[74:77], v[138:141], a[228:231]\n" \
  "v_mfma_f32_16x16x32_f16 a[196:199], v[98:101], v[138:141], a[196:199]\n" \
  "v_mfma_f32_16x16x32_f16 a[196:199], v[94:97], v[106:109], a[196:199]\n" \
  "v_mfma_f32_16x16x32_f16 a[200:203], v[94:97], v[110:113], a[200:203]\n" \
  "v_mfma_f32_16x16x32_f16 a[200:203], v[98:101], v[142:145], a[200:203]\n" \
  "s_waitcnt lgkmcnt(0)\n" \
  "v_mfma_f32_16x16x32_f16 a[232:235], v[74:77], v[142:145], a[232:235]\n" \
  "s_sub_u32 s82, s81, s58\n" \
  "s_branch L_s35_%=\n" \
  "L_b35_%=:\n" \
  "v_mfma_f32_16x16x32_f16 a[232:235], v[70:73], v[110:113], a[232:235]\n" \
  "s_max_i32 s82, s82, -150\n" \
  "s_add_u32 s82, s82, %[phic]\n" \
  "s_add_u32 s81, s58, s82\n" \
  "v_mfma_f32_16x16x32_f16 a[236:239], v[70:73], v[114:117], a[236:239]\n" \
  "s_add_u32 s81, s81, s86\n" \
  "v_mfma_f32_16x16x32_f16 a[236:239], v[74:77], v[146:149], a[236:239]\n" \
  "v_mfma_f32_16x16x32_f16 a[204:207], v[98:101], v[146:149], a[204:207]\n" \
  "v_mfma_f32_16x16x32_f16 a[204:207], v[94:97], v[114:117], a[204:207]\n" \
  "v_mfma_f32_16x16x32_f16 a[208:211], v[94:97], v[118:121], a[208:211]\n" \
  "v_mfma_f32_16x16x32_f16 a[208:211], v[98:101], v[150:153], a[208:211]\n" \
  "v_mfma_f32_16x16x32_f16 a[240:243], v[74:77], v[150:153], a[240:243]\n" \
  "v_mfma_f32_16x16x32_f16 a[240:243], v[70:73], v[118:121], a[240:243]\n" \
  "v_readlane_b32 s74, %[vtabA], 14\n" \
  "v_mfma_f32_16x16x32_f16 a[244:247], v[70:73], v[122:125], a[244:247]\n" \
  "s_bitcmp1_b32 s34, 14\n" \
  "s_cbranch_scc0 L_s59_%=\n" \
  "L_b59_%=:\n" \
  "v_mfma_f32_16x16x32_f16 a[244:247], v[74:77], v[154:157], a[244:247]\n" \
  "s_waitcnt vmcnt(18)\n" \
  "s_add_u32 s72, s74, s68\n" \
  "v_mfma_f32_16x16x32_f16 a[212:215], v[98:101], v[154:157], a[212:215]\n" \
  "v_cvt_pk_f16_f32 v216, v198, v202\n" \
  "v_cvt_pk_f16_f32 v220, v199, v203\n" \
  "v_mfma_f32_16x16x32_f16 a[212:215], v[94:97], v[122:125], a[212:215]\n" \
  "s_waitcnt vmcnt(16)\n" \
  "v_cvt_pk_f16_f32 v224, v200, v204\n" \
  "v_cvt_pk_f16_f32 v228, v201, v205\n" \
  "v_mfma_f32_16x16x32_f16 a[216:219], v[94:97], v[126:129], a[216:219]\n" \
  "v_cvt_pk_f16_f32 v217, v206, v210\n" \
  "v_cvt_pk_f16_f32 v221, v207, v211\n" \
  "v_mfma_f32_16x16x32_f16 a[216:219], v[98:101], v[158:161], a[216:219]\n" \
  "v_cvt_pk_f16_f32 v225, v208, v212\n" \
  "v_cvt_pk_f16_f32 v229, v209, v213\n" \
  "v_mfma_f32_16x16x32_f16 a[248:251], v[74:77], v[158:161], a[248:251]\n" \
  "ds_write_b128 v242, v[214:217] offset:32768\n" \
  "v_mfma_f32_16x16x32_f16 a[248:251], v[70:73], v[126:129], a[248:251]\n" \
  "buffer_load_dwordx4 v[50:53], %[bxr], s[40:43], s72 offen offset:0 sc0 sc1\n" \
  "v_mfma_f32_16x16x32_f16 a[252:255], v[70:73], v[130:133], a[252:255]\n" \
  "v_readlane_b32 s75, %[vtabW], 22\n" \
  "v_mfma_f32_16x16x32_f16 a[252:255], v[74:77], v[162:165], a[252:255]\n" \
  "s_waitcnt vmcnt(11)\n" \
  "v_mfma_f32_16x16x32_f16 a[220:223], v[98:101], v[162:165], a[220:223]\n" \
  "ds_write_b128 v242, v[218:221] offset:33792\n" \
  "v_mfma_f32_16x16x32_f16 a[220:223], v[94:97], v[130:133], a[220:223]\n" \
  "buffer_load_dwordx4 v[46:49], %[bxr], s[40:43], s72 offen offset:1024 sc0 sc1\n" \
  "v_mfma_f32_16x16x32_f16 a[0:3], v[18:21], v[102:105], a[0:3]\n" \
  "s_add_u32 s60, s75, 0x0\n" \
  "s_add_u32 s61, s75, 0x10000\n" \
  "s_add_u32 s62, s75, 0x20000\n" \
  "v_mfma_f32_16x16x32_f16 a[0:3], v[14:17], v[134:137], a[0:3]\n" \
  "s_add_u32 s63, s75, 0x30000\n" \
  "s_add_u32 s64, s75, 0x100000\n" \
  "s_add_u32 s65, s75, 0x110000\n" \
  "v_mfma_f32_16x16x32_f16 a[32:35], v[90:93], v[134:137], a[32:35]\n" \
  "ds_write_b128 v242, v[222:225] offset:34816\n" \
  "v_mfma_f32_16x16x32_f16 a[32:35], v[86:89], v[102:105], a[32:35]\n" \
  "buffer_load_dwordx4 v[38:41], %[bxr], s[40:43], s72 offen offset:2048 sc0 sc1\n" \
  "v_mfma_f32_16x16x32_f16 a[36:39], v[86:89], v[106:109], a[36:39]\n" \
  "v_mfma_f32_16x16x32_f16 a[36:39], v[90:93], v[138:141], a[36:39]\n" \
  "s_add_u32 s73, s74, s69\n" \
  "v_mfma_f32_16x16x32_f16 a[4:7], v[14:17], v[138:141], a[4:7]\n" \
  "ds_write_b128 v242, v[226:229] offset:35840\n" \
  "v_mfma_f32_16x16x32_f16 a[4:7], v[18:21], v[106:109], a[4:7]\n" \
  "buffer_load_dwordx4 v[42:45], %[bxr], s[40:43], s72 offen offset:3072 sc0 sc1\n" \
  "v_mfma_f32_16x16x32_f16 a[8:11], v[18:21], v[110:113], a[8:11]\n" \
  "v_mfma_f32_16x16x32_f16 a[8:11], v[14:17], v[142:145], a[8:11]\n" \
  "v_mfma_f32_16x16x32_f16 a[40:43], v[90:93], v[142:145], a[40:43]\n" \
  "v_mfma_f32_16x16x32_f16 a[40:43], v[86:89], v[110:113], a[40:43]\n" \
  "v_cvt_pk_f16_f32 v214, v166, v170\n" \
  "v_cvt_pk_f16_f32 v218, v167, v171\n" \
  "v_mfma_f32_16x16x32_f16 a[44:47], v[86:89], v[114:117], a[44:47]\n" \
  "v_cvt_pk_f16_f32 v222, v168, v172\n" \
  "v_cvt_pk_f16_f32 v226, v169, v173\n" \
  "v_mfma_f32_16x16x32_f16 a[44:47], v[90:93], v[146:149], a[44:47]\n" \
  "buffer_load_dwordx4 v[66:69], %[bxr], s[40:43], s73 offen offset:0 sc0 sc1\n" \
  "v_mfma_f32_16x16x32_f16 a[12:15], v[14:17], v[146:149], a[12:15]\n" \
  "v_mfma_f32_16x16x32_f16 a[12:15], v[18:21], v[114:117], a[12:15]\n" \
  "v_mfma_f32_16x16x32_f16 a[16:19], v[18:21], v[118:121], a[16:19]\n" \
  "v_mfma_f32_16x16x32_f16 a[16:19], v[14:17], v[150:153], a[16:19]\n" \
  "buffer_load_dwordx4 v[62:65], %[bxr], s[40:43], s73 offen offset:1024 sc0 sc1\n" \
  "v_mfma_f32_16x16x32_f16 a[48:51], v[90:93], v[150:153], a[48:51]\n" \
  "v_mfma_f32_16x16x32_f16 a[48:51], v[86:89], v[118:121], a[48:51]\n" \
  "v_mfma_f32_16x16x32_f16 a[52:55], v[86:89], v[122:125], a[52:55]\n" \
  "v_mfma_f32_16x16x32_f16 a[52:55], v[90:93], v[154:157], a[52:55]\n" \
  "buffer_load_dwordx4 v[54:57], %[bxr], s[40:43], s73 offen offset:2048 sc0 sc1\n" \
  "v_mfma_f32_16x16x32_f16 a[20:23], v[14:17], v[154:157], a[20:23]\n" \
  "v_mfma_f32_16x16x32_f16 a[20:23], v[18:21], v[122:125], a[20:23]\n" \
  "s_andn2_b32 s79, 0x7ffff, s34\n" \
  "s_cbranch_scc1 L_pl14_%=\n" \
  "L_pb14_%=:\n" \
  "v_mfma_f32_16x16x32_f16 a[24:27], v[18:21], v[126:129], a[24:27]\n" \
  "s_barrier\n" \
  "v_mfma_f32_16x16x32_f16 a[24:27], v[14:17], v[158:161], a[24:27]\n" \
  "buffer_load_dwordx4 v[58:61], %[bxr], s[40:43], s73 offen offset:3072 sc0 sc1\n" \
  "v_mfma_f32_16x16x32_f16 a[56:59], v[90:93], v[158:161], a[56:59]\n" \
  "buffer_load_dwordx4 v[198:201], %[voffW], s[36:39], s60 offen nt\n" \
  "v_mfma_f32_16x16x32_f16 a[56:59], v[86:89], v[126:129], a[56:59]\n" \
  "v_mfma_f32_16x16x32_f16 a[60:63], v[86:89], v[130:133], a[60:63]\n" \
  "buffer_load_dwordx4 v[202:205], %[voffW], s[36:39], s61 offen nt\n" \
  "v_mfma_f32_16x16x32_f16 a[60:63], v[90:93], v[162:165], a[60:63]\n" \
  "v_mfma_f32_16x16x32_f16 a[28:31], v[14:17], v[162:165], a[28:31]\n" \
  "buffer_load_dwordx4 v[206:209], %[voffW], s[36:39], s62 offen nt\n" \
  "v_mfma_f32_16x16x32_f16 a[28:31], v[18:21], v[130:133], a[28:31]\n" \
  "s_waitcnt vmcnt(16)\n" \
  "v_mfma_f32_16x16x32_f16 a[128:131], v[34:37], v[102:105], a[128:131]\n" \
  "buffer_load_dwordx4 v[210:213], %[voffW], s[36:39], s63 offen nt\n" \
  "v_mfma_f32_16x16x32_f16 a[128:131], v[30:33], v[134:137], a[128:131]\n" \
  "v_mfma_f32_16x16x32_f16 a[160:163], v[26:29], v[134:137], a[160:163]\n" \
  "ds_read_b128 v[134:137], v246 offset:40960\n" \
  "v_mfma_f32_16x16x32_f16 a[160:163], v[22:25], v[102:105], a[160:163]\n" \
  "ds_read_b128 v[102:105], v246 offset:32768\n" \
  "v_mfma_f32_16x16x32_f16 a[164:167], v[22:25], v[106:109], a[164:167]\n" \
  "buffer_load_dwordx4 v[166:169], %[voffW], s[36:39], s64 offen nt\n" \
  "v_mfma_f32_16x16x32_f16 a[164:167], v[26:29], v[138:141], a[164:167]\n" \
  "v_mfma_f32_16x16x32_f16 a[132:135], v[30:33], v[138:141], a[132:135]\n" \
  "ds_read_b128 v[138:141], v246 offset:41984\n" \
  "v_mfma_f32_16x16x32_f16 a[132:135], v[34:37], v[106:109], a[132:135]\n" \
  "ds_read_b128 v[106:109], v246 offset:33792\n" \
  "v_mfma_f32_16x16x32_f16 a[136:139], v[34:37], v[110:113], a[136:139]\n" \
  "buffer_load_dwordx4 v[170:173], %[voffW], s[36:39], s65 offen nt\n" \
  "v_mfma_f32_16x16x32_f16 a[136:139], v[30:33], v[142:145], a[136:139]\n" \
  "v_mfma_f32_16x16x32_f16 a[168:171], v[26:29], v[142:145], a[168:171]\n" \
  "ds_read_b128 v[142:145], v246 offset:43008\n" \
  "v_mfma_f32_16x16x32_f16 a[168:171], v[22:25], v[110:113], a[168:171]\n" \
  "ds_read_b128 v[110:113], v246 offset:34816\n" \
  "v_mfma_f32_16x16x32_f16 a[172:175], v[22:25], v[114:117], a[172:175]\n" \
  "v_mfma_f32_16x16x32_f16 a[172:175], v[26:29], v[146:149], a[172:175]\n" \
  "v_mfma_f32_16x16x32_f16 a[140:143], v[30:33], v[146:149], a[140:143]\n" \
  "ds_read_b128 v[146:149], v246 offset:44032\n" \
  "v_mfma_f32_16x16x32_f16 a[140:143], v[34:37], v[114:117], a[140:143]\n" \
  "ds_read_b128 v[114:117], v246 offset:35840\n" \
  "v_mfma_f32_16x16x32_f16 a[144:147], v[34:37], v[118:121], a[144:147]\n" \
  "s_add_u32 s72, s74, s70\n" \
  "v_mfma_f32_16x16x32_f16 a[144:147], v[30:33], v[150:153], a[144:147]\n" \
  "v_mfma_f32_16x16x32_f16 a[176:179], v[26:29], v[150:153], a[176:179]\n" \
  "ds_read_b128 v[150:153], v246 offset:45056\n" \
  "v_mfma_f32_16x16x32_f16 a[176:179], v[22:25], v[118:121], a[176:179]\n" \
  "ds_read_b128 v[118:121], v246 offset:36864\n" \
  "v_mfma_f32_16x16x32_f16 a[180:183], v[22:25], v[122:125], a[180:183]\n" \
  "v_mfma_f32_16x16x32_f16 a[180:183], v[26:29], v[154:157], a[180:183]\n" \
  "v_mfma_f32_16x16x32_f16 a[148:151], v[30:33], v[154:157], a[148:151]\n" \
  "ds_read_b128 v[154:157], v246 offset:46080\n" \
  "v_mfma_f32_16x16x32_f16 a[148:151], v[34:37], v[122:125], a[148:151]\n" \
  "ds_read_b128 v[122:125], v246 offset:37888\n" \
  "v_mfma_f32_16x16x32_f16 a[152:155], v[34:37], v[126:129], a[152:155]\n" \
  "buffer_load_dwordx4 v[82:85], %[bxr], s[40:43], s72 offen offset:0 sc0 sc1\n" \
  "v_mfma_f32_16x16x32_f16 a[152:155], v[30:33], v[158:161], a[152:155]\n" \
  "v_mfma_f32_16x16x32_f16 a[184:187], v[26:29], v[158:161], a[184:187]\n" \
  "ds_read_b128 v[158:161], v246 offset:47104\n" \
  "v_mfma_f32_16x16x32_f16 a[184:187], v[22:25], v[126:129], a[184:187]\n" \
  "ds_read_b128 v[126:129], v246 offset:38912\n" \
  "v_mfma_f32_16x16x32_f16 a[188:191], v[22:25], v[130:133], a[188:191]\n" \
  "buffer_load_dwordx4 v[78:81], %[bxr], s[40:43], s72 offen offset:1024 sc0 sc1\n" \
  "v_mfma_f32_16x16x32_f16 a[188:191], v[26:29], v[162:165], a[188:191]\n" \
  "s_add_u32 s66, s75, 0x120000\n" \
  "s_add_u32 s67, s75, 0x130000\n" \
  "v_mfma_f32_16x16x32_f16 a[156:159], v[30:33], v[162:165], a[156:159]\n" \
  "ds_read_b128 v[162:165], v246 offset:48128\n" \
  "v_readlane_b32 s75, %[vtabW], 23\n" \
  "v_mfma_f32_16x16x32_f16 a[156:159], v[34:37], v[130:133], a[156:159]\n" \
  "s_waitcnt vmcnt(12)\n" \
  "ds_read_b128 v[130:133], v246 offset:39936\n" \
  "s_waitcnt lgkmcnt(14)\n" \
    \
  "v_mfma_f32_16x16x32_f16 a[64:67], v[50:53], v[102:105], a[64:67]\n" \
  "buffer_load_dwordx4 v[70:73], %[bxr], s[40:43], s72 offen offset:2048 sc0 sc1\n" \
  "v_mfma_f32_16x16x32_f16 a[64:67], v[46:49], v[134:137], a[64:67]\n" \
  "s_add_u32 s60, s75, 0x0\n" \
  "s_add_u32 s61, s75, 0x10000\n" \
  "s_add_u32 s62, s75, 0x20000\n" \
  "v_mfma_f32_16x16x32_f16 a[96:99], v[42:45], v[134:137], a[96:99]\n" \
  "s_add_u32 s63, s75, 0x30000\n" \
  "s_add_u32 s73, s74, s71\n" \
  "v_mfma_f32_16x16x32_f16 a[96:99], v[38:41], v[102:105], a[96:99]\n" \
  "s_waitcnt lgkmcnt(12)\n" \
  "v_mfma_f32_16x16x32_f16 a[100:103], v[38:41], v[106:109], a[100:103]\n" \
  "buffer_load_dwordx4 v[74:77], %[bxr], s[40:43], s72 offen offset:3072 sc0 sc1\n" \
  "v_mfma_f32_16x16x32_f16 a[100:103], v[42:45], v[138:141], a[100:103]\n" \
  "v_mfma_f32_16x16x32_f16 a[68:71], v[46:49], v[138:141], a[68:71]\n" \
  "v_mfma_f32_16x16x32_f16 a[68:71], v[50:53], v[106:109], a[68:71]\n" \
  "s_waitcnt lgkmcnt(10)\n" \
  "v_mfma_f32_16x16x32_f16 a[72:75], v[50:53], v[110:113], a[72:75]\n" \
  "buffer_load_dwordx4 v[98:101], %[bxr], s[40:43], s73 offen offset:0 sc0 sc1\n" \
  "v_mfma_f32_16x16x32_f16 a[72:75], v[46:49], v[142:145], a[72:75]\n" \
  "v_mfma_f32_16x16x32_f16 a[104:107], v[42:45], v[142:145], a[104:107]\n" \
  "v_mfma_f32_16x16x32_f16 a[104:107], v[38:41], v[110:113], a[104:107]\n" \
  "s_waitcnt lgkmcnt(8)\n" \
  "v_cvt_pk_f16_f32 v215, v174, v178\n" \
  "v_cvt_pk_f16_f32 v219, v175, v179\n" \
  "v_mfma_f32_16x16x32_f16 a[108:111], v[38:41], v[114:117], a[108:111]\n" \
  "v_cvt_pk_f16_f32 v223, v176, v180\n" \
  "v_cvt_pk_f16_f32 v227, v177, v181\n" \
  "v_mfma_f32_16x16x32_f16 a[108:111], v[42:45], v[146:149], a[108:111]\n" \
  "s_waitcnt vmcnt(12)\n" \
  "v_cvt_pk_f16_f32 v216, v182, v186\n" \
  "v_cvt_pk_f16_f32 v220, v183, v187\n" \
  "v_mfma_f32_16x16x32_f16 a[76:79], v[46:49], v[146:149], a[76:79]\n" \
  "v_cvt_pk_f16_f32 v224, v184, v188\n" \
  "v_cvt_pk_f16_f32 v228, v185, v189\n" \
  "v_mfma_f32_16x16x32_f16 a[76:79], v[50:53], v[114:117], a[76:79]\n" \
  "s_waitcnt lgkmcnt(6)\n" \
  "v_cvt_pk_f16_f32 v217, v190, v194\n" \
  "v_cvt_pk_f16_f32 v221, v191, v195\n" \
  "v_mfma_f32_16x16x32_f16 a[80:83], v[50:53], v[118:121], a[80:83]\n" \
  "v_cvt_pk_f16_f32 v225, v192, v196\n" \
  "v_cvt_pk_f16_f32 v229, v193, v197\n" \
  "v_mfma_f32_16x16x32_f16 a[80:83], v[46:49], v[150:153], a[80:83]\n" \
  "ds_write_b128 v242, v[214:217] offset:49152\n" \
  "v_mfma_f32_16x16x32_f16 a[112:115], v[42:45], v[150:153], a[112:115]\n" \
  "buffer_load_dwordx4 v[94:97], %[bxr], s[40:43], s73 offen offset:1024 sc0 sc1\n" \
  "v_mfma_f32_16x16x32_f16 a[112:115], v[38:41], v[118:121], a[112:115]\n" \
  "s_waitcnt lgkmcnt(5)\n" \
  "v_readlane_b32 s74, %[vtabA], 15\n" \
  "v_min3_u32 v248, v238, v239, v240\n" \
  "v_mfma_f32_16x16x32_f16 a[116:119], v[38:41], v[122:125], a[116:119]\n" \
  "s_bitcmp1_b32 s34, 15\n" \
  "s_cbranch_scc0 L_s60_%=\n" \
  "L_b60_%=:\n" \
  "v_min_u32 v248, v248, v241\n" \
  "v_mfma_f32_16x16x32_f16 a[116:119], v[42:45], v[154:157], a[116:119]\n" \
  "ds_write_b128 v242, v[218:221] offset:50176\n" \
  "v_mfma_f32_16x16x32_f16 a[84:87], v[46:49], v[154:157], a[84:87]\n" \
  "buffer_load_dwordx4 v[86:89], %[bxr], s[40:43], s73 offen offset:2048 sc0 sc1\n" \
  "v_mfma_f32_16x16x32_f16 a[84:87], v[50:53], v[122:125], a[84:87]\n" \
  "s_waitcnt lgkmcnt(4)\n" \
  "v_cmp_le_u32 s[34:35], s76, v248\n" \
  "s_add_u32 s72, s74, s68\n" \
  "v_mfma_f32_16x16x32_f16 a[88:91], v[50:53], v[126:129], a[88:91]\n" \
  "s_waitcnt vmcnt(13)\n" \
  "s_waitcnt lgkmcnt(2)\n" \
  "s_barrier\n" \
  "v_mfma_f32_16x16x32_f16 a[88:91], v[46:49], v[158:161], a[88:91]\n" \
  "ds_write_b128 v242, v[222:225] offset:51200\n" \
  "v_mfma_f32_16x16x32_f16 a[120:123], v[42:45], v[158:161], a[120:123]\n" \
  "buffer_load_dwordx4 v[174:177], %[voffW], s[36:39], s66 offen nt\n" \
  "v_mfma_f32_16x16x32_f16 a[120:123], v[38:41], v[126:129], a[120:123]\n" \
  "buffer_load_dwordx4 v[90:93], %[bxr], s[40:43], s73 offen offset:3072 sc0 sc1\n" \
  "v_mfma_f32_16x16x32_f16 a[124:127], v[38:41], v[130:133], a[124:127]\n" \
  "buffer_load_dwordx4 v[178:181], %[voffW], s[36:39], s67 offen nt\n" \
  "v_mfma_f32_16x16x32_f16 a[124:127], v[42:45], v[162:165], a[124:127]\n" \
  "ds_write_b128 v242, v[226:229] offset:52224\n" \
  "v_mfma_f32_16x16x32_f16 a[92:95], v[46:49], v[162:165], a[92:95]\n" \
  "buffer_load_dwordx4 v[182:185], %[voffW], s[36:39], s60 offen nt\n" \
  "v_mfma_f32_16x16x32_f16 a[92:95], v[50:53], v[130:133], a[92:95]\n" \
  "buffer_load_dwordx4 v[14:17], %[bxr], s[40:43], s72 offen offset:0 sc0 sc1\n" \
  "v_mfma_f32_16x16x32_f16 a[192:195], v[66:69], v[102:105], a[192:195]\n" \
  "buffer_load_dwordx4 v[186:189], %[voffW], s[36:39], s61 offen nt\n" \
  "v_mfma_f32_16x16x32_f16 a[192:195], v[62:65], v[134:137], a[192:195]\n" \
  "s_memrealtime s[58:59]\n" \
  "v_mfma_f32_16x16x32_f16 a[224:227], v[58:61], v[134:137], a[224:227]\n" \
  "buffer_load_dwordx4 v[190:193], %[voffW], s[36:39], s62 offen nt\n" \
  "v_mfma_f32_16x16x32_f16 a[224:227], v[54:57], v[102:105], a[224:227]\n" \
  "buffer_load_dwordx4 v[18:21], %[bxr], s[40:43], s72 offen offset:1024 sc0 sc1\n" \
  "v_mfma_f32_16x16x32_f16 a[228:231], v[54:57], v[106:109], a[228:231]\n" \
  "buffer_load_dwordx4 v[194:197], %[voffW], s[36:39], s63 offen nt\n" \
  "v_mfma_f32_16x16x32_f16 a[228:231], v[58:61], v[138:141], a[228:231]\n" \
  "v_mfma_f32_16x16x32_f16 a[196:199], v[62:65], v[138:141], a[196:199]\n" \
  "v_mfma_f32_16x16x32_f16 a[196:199], v[66:69], v[106:109], a[196:199]\n" \
  "buffer_load_dwordx4 v[22:25], %[bxr], s[40:43], s72 offen offset:2048 sc0 sc1\n" \
  "v_mfma_f32_16x16x32_f16 a[200:203], v[66:69], v[110:113], a[200:203]\n" \
  "v_mfma_f32_16x16x32_f16 a[200:203], v[62:65], v[142:145], a[200:203]\n" \
  "s_waitcnt lgkmcnt(0)\n" \
  "v_mfma_f32_16x16x32_f16 a[232:235], v[58:61], v[142:145], a[232:235]\n" \
  "s_sub_u32 s82, s81, s58\n" \
  "s_branch L_s36_%=\n" \
  "L_b36_%=:\n" \
  "v_mfma_f32_16x16x32_f16 a[232:235], v[54:57], v[110:113], a[232:235]\n" \
  "buffer_load_dwordx4 v[26:29], %[bxr], s[40:43], s72 offen offset:3072 sc0 sc1\n" \
  "v_mfma_f32_16x16x32_f16 a[236:239], v[54:57], v[114:117], a[236:239]\n" \
  "s_max_i32 s82, s82, -150\n" \
  "s_add_u32 s82, s82, %[phic]\n" \
  "s_add_u32 s81, s58, s82\n" \
  "v_mfma_f32_16x16x32_f16 a[236:239], v[58:61], v[146:149], a[236:239]\n" \
  "s_add_u32 s81, s81, s89\n" \
  "v_mfma_f32_16x16x32_f16 a[204:207], v[62:65], v[146:149], a[204:207]\n" \
  "v_mfma_f32_16x16x32_f16 a[204:207], v[66:69], v[114:117], a[204:207]\n" \
  "v_mfma_f32_16x16x32_f16 a[208:211], v[66:69], v[118:121], a[208:211]\n" \
  "v_mfma_f32_16x16x32_f16 a[208:211], v[62:65], v[150:153], a[208:211]\n" \
  "v_mfma_f32_16x16x32_f16 a[240:243], v[58:61], v[150:153], a[240:243]\n" \
  "v_mfma_f32_16x16x32_f16 a[240:243], v[54:57], v[118:121], a[240:243]\n" \
  "v_mfma_f32_16x16x32_f16 a[244:247], v[54:57], v[122:125], a[244:247]\n" \
  "v_mfma_f32_16x16x32_f16 a[244:247], v[58:61], v[154:157], a[244:247]\n" \
  "v_mfma_f32_16x16x32_f16 a[212:215], v[62:65], v[154:157], a[212:215]\n" \
  "v_mfma_f32_16x16x32_f16 a[212:215], v[66:69], v[122:125], a[212:215]\n" \
  "v_mfma_f32_16x16x32_f16 a[216:219], v[66:69], v[126:129], a[216:219]\n" \
  "v_mfma_f32_16x16x32_f16 a[216:219], v[62:65], v[158:161], a[216:219]\n" \
  "v_mfma_f32_16x16x32_f16 a[248:251], v[58:61], v[158:161], a[248:251]\n" \
  "v_mfma_f32_16x16x32_f16 a[248:251], v[54:57], v[126:129], a[248:251]\n" \
  "v_mfma_f32_16x16x32_f16 a[252:255], v[54:57], v[130:133], a[252:255]\n" \
  "s_add_u32 s73, s74, s69\n" \
  "v_mfma_f32_16x16x32_f16 a[252:255], v[58:61], v[162:165], a[252:255]\n" \
  "v_mfma_f32_16x16x32_f16 a[220:223], v[62:65], v[162:165], a[220:223]\n" \
  "v_mfma_f32_16x16x32_f16 a[220:223], v[66:69], v[130:133], a[220:223]\n" \
  "s_waitcnt vmcnt(14)\n" \
  "v_mfma_f32_16x16x32_f16 a[0:3], v[82:85], v[102:105], a[0:3]\n" \
  "v_mfma_f32_16x16x32_f16 a[0:3], v[78:81], v[134:137], a[0:3]\n" \
  "v_mfma_f32_16x16x32_f16 a[32:35], v[74:77], v[134:137], a[32:35]\n" \
  "v_mfma_f32_16x16x32_f16 a[32:35], v[70:73], v[102:105], a[32:35]\n" \
  "v_mfma_f32_16x16x32_f16 a[36:39], v[70:73], v[106:109], a[36:39]\n" \
  "buffer_load_dwordx4 v[30:33], %[bxr], s[40:43], s73 offen offset:0 sc0 sc1\n" \
  "v_mfma_f32_16x16x32_f16 a[36:39], v[74:77], v[138:141], a[36:39]\n" \
  "v_mfma_f32_16x16x32_f16 a[4:7], v[78:81], v[138:141], a[4:7]\n" \
  "v_mfma_f32_16x16x32_f16 a[4:7], v[82:85], v[106:109], a[4:7]\n" \
  "v_mfma_f32_16x16x32_f16 a[8:11], v[82:85], v[110:113], a[8:11]\n" \
  "buffer_load_dwordx4 v[34:37], %[bxr], s[40:43], s73 offen offset:1024 sc0 sc1\n" \
  "v_mfma_f32_16x16x32_f16 a[8:11], v[78:81], v[142:145], a[8:11]\n" \
  "v_mfma_f32_16x16x32_f16 a[40:43], v[74:77], v[142:145], a[40:43]\n" \
  "v_mfma_f32_16x16x32_f16 a[40:43], v[70:73], v[110:113], a[40:43]\n" \
  "v_mfma_f32_16x16x32_f16 a[44:47], v[70:73], v[114:117], a[44:47]\n" \
  "buffer_load_dwordx4 v[38:41], %[bxr], s[40:43], s73 offen offset:2048 sc0 sc1\n" \
  "v_mfma_f32_16x16x32_f16 a[44:47], v[74:77], v[146:149], a[44:47]\n" \
  "s_add_u32 s72, s74, s70\n" \
  "v_mfma_f32_16x16x32_f16 a[12:15], v[78:81], v[146:149], a[12:15]\n" \
  "v_mfma_f32_16x16x32_f16 a[12:15], v[82:85], v[114:117], a[12:15]\n" \
  "v_mfma_f32_16x16x32_f16 a[16:19], v[82:85], v[118:121], a[16:19]\n" \
  "buffer_load_dwordx4 v[42:45], %[bxr], s[40:43], s73 offen offset:3072 sc0 sc1\n" \
  "v_mfma_f32_16x16x32_f16 a[16:19], v[78:81], v[150:153], a[16:19]\n" \
  "v_mfma_f32_16x16x32_f16 a[48:51], v[74:77], v[150:153], a[48:51]\n" \
  "v_mfma_f32_16x16x32_f16 a[48:51], v[70:73], v[118:121], a[48:51]\n" \
  "v_mfma_f32_16x16x32_f16 a[52:55], v[70:73], v[122:125], a[52:55]\n" \
  "buffer_load_dwordx4 v[46:49], %[bxr], s[40:43], s72 offen offset:0 sc0 sc1\n" \
  "v_mfma_f32_16x16x32_f16 a[52:55], v[74:77], v[154:157], a[52:55]\n" \
  "v_mfma_f32_16x16x32_f16 a[20:23], v[78:81], v[154:157], a[20:23]\n" \
  "v_mfma_f32_16x16x32_f16 a[20:23], v[82:85], v[122:125], a[20:23]\n" \
  "v_mfma_f32_16x16x32_f16 a[24:27], v[82:85], v[126:129], a[24:27]\n" \
  "buffer_load_dwordx4 v[50:53], %[bxr], s[40:43], s72 offen offset:1024 sc0 sc1\n" \
  "v_mfma_f32_16x16x32_f16 a[24:27], v[78:81], v[158:161], a[24:27]\n" \
  "v_mfma_f32_16x16x32_f16 a[56:59], v[74:77], v[158:161], a[56:59]\n" \
  "s_add_u32 s64, s75, 0x100000\n" \
  "s_add_u32 s65, s75, 0x110000\n" \
  "s_add_u32 s66, s75, 0x120000\n" \
  "v_mfma_f32_16x16x32_f16 a[56:59], v[70:73], v[126:129], a[56:59]\n" \
  "s_add_u32 s67, s75, 0x130000\n" \
  "v_mfma_f32_16x16x32_f16 a[60:63], v[70:73], v[130:133], a[60:63]\n" \
  "buffer_load_dwordx4 v[54:57], %[bxr], s[40:43], s72 offen offset:2048 sc0 sc1\n" \
  "v_mfma_f32_16x16x32_f16 a[60:63], v[74:77], v[162:165], a[60:63]\n" \
  "s_add_u32 s73, s74, s71\n" \
  "v_readlane_b32 s75, %[vtabW], 24\n" \
  "v_mfma_f32_16x16x32_f16 a[28:31], v[78:81], v[162:165], a[28:31]\n" \
  "v_mfma_f32_16x16x32_f16 a[28:31], v[82:85], v[130:133], a[28:31]\n" \
  "s_waitcnt vmcnt(16)\n" \
  "s_add_u32 s60, s75, 0x0\n" \
  "s_add_u32 s61, s75, 0x10000\n" \
  "v_mfma_f32_16x16x32_f16 a[128:131], v[98:101], v[102:105], a[128:131]\n" \
  "buffer_load_dwordx4 v[58:61], %[bxr], s[40:43], s72 offen offset:3072 sc0 sc1\n" \
  "v_mfma_f32_16x16x32_f16 a[128:131], v[94:97], v[134:137], a[128:131]\n" \
  "v_mfma_f32_16x16x32_f16 a[160:163], v[90:93], v[134:137], a[160:163]\n" \
  "ds_read_b128 v[134:137], v246 offset:57344\n" \
  "v_mfma_f32_16x16x32_f16 a[160:163], v[86:89], v[102:105], a[160:163]\n" \
  "ds_read_b128 v[102:105], v246 offset:49152\n" \
  "v_mfma_f32_16x16x32_f16 a[164:167], v[86:89], v[106:109], a[164:167]\n" \
  "buffer_load_dwordx4 v[62:65], %[bxr], s[40:43], s73 offen offset:0 sc0 sc1\n" \
  "v_mfma_f32_16x16x32_f16 a[164:167], v[90:93], v[138:141], a[164:167]\n" \
  "v_mfma_f32_16x16x32_f16 a[132:135], v[94:97], v[138:141], a[132:135]\n" \
  "ds_read_b128 v[138:141], v246 offset:58368\n" \
  "v_mfma_f32_16x16x32_f16 a[132:135], v[98:101], v[106:109], a[132:135]\n" \
  "ds_read_b128 v[106:109], v246 offset:50176\n" \
  "v_mfma_f32_16x16x32_f16 a[136:139], v[98:101], v[110:113], a[136:139]\n" \
  "v_mfma_f32_16x16x32_f16 a[136:139], v[94:97], v[142:145], a[136:139]\n" \
  "v_cvt_pk_f16_f32 v214, v198, v202\n" \
  "v_cvt_pk_f16_f32 v218, v199, v203\n" \
  "v_mfma_f32_16x16x32_f16 a[168:171], v[90:93], v[142:145], a[168:171]\n" \
  "ds_read_b128 v[142:145], v246 offset:59392\n" \
  "v_cvt_pk_f16_f32 v222, v200, v204\n" \
  "v_mfma_f32_16x16x32_f16 a[168:171], v[86:89], v[110:113], a[168:171]\n" \
  "ds_read_b128 v[110:113], v246 offset:51200\n" \
  "v_cvt_pk_f16_f32 v226, v201, v205\n" \
  "v_mfma_f32_16x16x32_f16 a[172:175], v[86:89], v[114:117], a[172:175]\n" \
  "v_cvt_pk_f16_f32 v215, v206, v210\n" \
  "v_cvt_pk_f16_f32 v219, v207, v211\n" \
  "v_mfma_f32_16x16x32_f16 a[172:175], v[90:93], v[146:149], a[172:175]\n" \
  "v_cvt_pk_f16_f32 v223, v208, v212\n" \
  "v_cvt_pk_f16_f32 v227, v209, v213\n" \
  "v_mfma_f32_16x16x32_f16 a[140:143], v[94:97], v[146:149], a[140:143]\n" \
  "ds_read_b128 v[146:149], v246 offset:60416\n" \
  "v_cvt_pk_f16_f32 v216, v166, v170\n" \
  "v_mfma_f32_16x16x32_f16 a[140:143], v[98:101], v[114:117], a[140:143]\n" \
  "ds_read_b128 v[114:117], v246 offset:52224\n" \
  "v_cvt_pk_f16_f32 v220, v167, v171\n" \
  "v_mfma_f32_16x16x32_f16 a[144:147], v[98:101], v[118:121], a[144:147]\n" \
  "v_cvt_pk_f16_f32 v224, v168, v172\n" \
  "v_cvt_pk_f16_f32 v228, v169, v173\n" \
  "v_mfma_f32_16x16x32_f16 a[144:147], v[94:97], v[150:153], a[144:147]\n" \
  "buffer_load_dwordx4 v[66:69], %[bxr], s[40:43], s73 offen offset:1024 sc0 sc1\n" \
  "v_mfma_f32_16x16x32_f16 a[176:179], v[90:93], v[150:153], a[176:179]\n" \
  "ds_read_b128 v[150:153], v246 offset:61440\n" \
  "v_mfma_f32_16x16x32_f16 a[176:179], v[86:89], v[118:121], a[176:179]\n" \
  "ds_read_b128 v[118:121], v246 offset:53248\n" \
  "v_mfma_f32_16x16x32_f16 a[180:183], v[86:89], v[122:125], a[180:183]\n" \
  "s_andn2_b32 s79, 0xfffff, s34\n" \
  "s_cbranch_scc1 L_pl15_%=\n" \
  "L_pb15_%=:\n" \
  "v_mfma_f32_16x16x32_f16 a[180:183], v[90:93], v[154:157], a[180:183]\n" \
  "buffer_load_dwordx4 v[70:73], %[bxr], s[40:43], s73 offen offset:2048 sc0 sc1\n" \
  "v_mfma_f32_16x16x32_f16 a[148:151], v[94:97], v[154:157], a[148:151]\n" \
  "ds_read_b128 v[154:157], v246 offset:62464\n" \
  "v_mfma_f32_16x16x32_f16 a[148:151], v[98:101], v[122:125], a[148:151]\n" \
  "s_barrier\n" \
  "ds_read_b128 v[122:125], v246 offset:54272\n" \
  "v_mfma_f32_16x16x32_f16 a[152:155], v[98:101], v[126:129], a[152:155]\n" \
  "buffer_load_dwordx4 v[198:201], %[voffW], s[36:39], s64 offen nt\n" \
  "v_mfma_f32_16x16x32_f16 a[152:155], v[94:97], v[158:161], a[152:155]\n" \
  "buffer_load_dwordx4 v[74:77], %[bxr], s[40:43], s73 offen offset:3072 sc0 sc1\n" \
  "v_mfma_f32_16x16x32_f16 a[184:187], v[90:93], v[158:161], a[184:187]\n" \
  "ds_read_b128 v[158:161], v246 offset:63488\n" \
  "v_mfma_f32_16x16x32_f16 a[184:187], v[86:89], v[126:129], a[184:187]\n" \
  "ds_read_b128 v[126:129], v246 offset:55296\n" \
  "v_mfma_f32_16x16x32_f16 a[188:191], v[86:89], v[130:133], a[188:191]\n" \
  "buffer_load_dwordx4 v[202:205], %[voffW], s[36:39], s65 offen nt\n" \
  "v_mfma_f32_16x16x32_f16 a[188:191], v[90:93], v[162:165], a[188:191]\n" \
  "v_mfma_f32_16x16x32_f16 a[156:159], v[94:97], v[162:165], a[156:159]\n" \
  "ds_read_b128 v[162:165], v246 offset:64512\n" \
  "v_mfma_f32_16x16x32_f16 a[156:159], v[98:101], v[130:133], a[156:159]\n" \
  "s_waitcnt vmcnt(14)\n" \
  "ds_read_b128 v[130:133], v246 offset:56320\n" \
  "s_waitcnt lgkmcnt(14)\n" \
    \
  "v_mfma_f32_16x16x32_f16 a[64:67], v[14:17], v[102:105], a[64:67]\n" \
  "buffer_load_dwordx4 v[206:209], %[voffW], s[36:39], s66 offen nt\n" \
  "v_mfma_f32_16x16x32_f16 a[64:67], v[18:21], v[134:137], a[64:67]\n" \
  "v_mfma_f32_16x16x32_f16 a[96:99], v[26:29], v[134:137], a[96:99]\n" \
  "buffer_load_dwordx4 v[210:213], %[voffW], s[36:39], s67 offen nt\n" \
  "v_mfma_f32_16x16x32_f16 a[96:99], v[22:25], v[102:105], a[96:99]\n" \
  "s_waitcnt lgkmcnt(12)\n" \
  "v_mfma_f32_16x16x32_f16 a[100:103], v[22:25], v[106:109], a[100:103]\n" \
  "buffer_load_dwordx4 v[166:169], %[voffW], s[36:39], s60 offen nt\n" \
  "v_mfma_f32_16x16x32_f16 a[100:103], v[26:29], v[138:141], a[100:103]\n" \
  "v_mfma_f32_16x16x32_f16 a[68:71], v[18:21], v[138:141], a[68:71]\n" \
  "buffer_load_dwordx4 v[170:173], %[voffW], s[36:39], s61 offen nt\n" \
  "v_mfma_f32_16x16x32_f16 a[68:71], v[14:17], v[106:109], a[68:71]\n" \
  "s_waitcnt lgkmcnt(10)\n" \
  "v_mfma_f32_16x16x32_f16 a[72:75], v[14:17], v[110:113], a[72:75]\n" \
  "v_mfma_f32_16x16x32_f16 a[72:75], v[18:21], v[142:145], a[72:75]\n" \
  "v_mfma_f32_16x16x32_f16 a[104:107], v[26:29], v[142:145], a[104:107]\n" \
  "v_mfma_f32_16x16x32_f16 a[104:107], v[22:25], v[110:113], a[104:107]\n" \
  "s_waitcnt lgkmcnt(8)\n" \
  "v_mfma_f32_16x16x32_f16 a[108:111], v[22:25], v[114:117], a[108:111]\n" \
  "v_mfma_f32_16x16x32_f16 a[108:111], v[26:29], v[146:149], a[108:111]\n" \
  "v_mfma_f32_16x16x32_f16 a[76:79], v[18:21], v[146:149], a[76:79]\n" \
  "v_mfma_f32_16x16x32_f16 a[76:79], v[14:17], v[114:117], a[76:79]\n" \
  "s_waitcnt lgkmcnt(6)\n" \
  "v_mfma_f32_16x16x32_f16 a[80:83], v[14:17], v[118:121], a[80:83]\n" \
  "v_mfma_f32_16x16x32_f16 a[80:83], v[18:21], v[150:153], a[80:83]\n" \
  "v_mfma_f32_16x16x32_f16 a[112:115], v[26:29], v[150:153], a[112:115]\n" \
  "v_mfma_f32_16x16x32_f16 a[112:115], v[22:25], v[118:121], a[112:115]\n" \
  "s_waitcnt lgkmcnt(4)\n" \
  "v_mfma_f32_16x16x32_f16 a[116:119], v[22:25], v[122:125], a[116:119]\n" \
  "v_mfma_f32_16x16x32_f16 a[116:119], v[26:29], v[154:157], a[116:119]\n" \
  "v_mfma_f32_16x16x32_f16 a[84:87], v[18:21], v[154:157], a[84:87]\n" \
  "v_mfma_f32_16x16x32_f16 a[84:87], v[14:17], v[122:125], a[84:87]\n" \
  "s_waitcnt lgkmcnt(2)\n" \
  "v_mfma_f32_16x16x32_f16 a[88:91], v[14:17], v[126:129], a[88:91]\n" \
  "v_mfma_f32_16x16x32_f16 a[88:91], v[18:21], v[158:161], a[88:91]\n" \
  "v_mfma_f32_16x16x32_f16 a[120:123], v[26:29], v[158:161], a[120:123]\n" \
  "v_readlane_b32 s74, %[vtabA], 16\n" \
  "v_mfma_f32_16x16x32_f16 a[120:123], v[22:25], v[126:129], a[120:123]\n" \
  "s_waitcnt lgkmcnt(0)\n" \
  "s_bitcmp1_b32 s34, 16\n" \
  "s_cbranch_scc0 L_s61_%=\n" \
  "L_b61_%=:\n" \
  "v_mfma_f32_16x16x32_f16 a[124:127], v[22:25], v[130:133], a[124:127]\n" \
  "s_add_u32 s72, s74, s68\n" \
  "v_mfma_f32_16x16x32_f16 a[124:127], v[26:29], v[162:165], a[124:127]\n" \
  "v_mfma_f32_16x16x32_f16 a[92:95], v[18:21], v[162:165], a[92:95]\n" \
  "v_mfma_f32_16x16x32_f16 a[92:95], v[14:17], v[130:133], a[92:95]\n" \
  "s_waitcnt vmcnt(14)\n" \
  "v_mfma_f32_16x16x32_f16 a[192:195], v[30:33], v[102:105], a[192:195]\n" \
  "v_mfma_f32_16x16x32_f16 a[192:195], v[34:37], v[134:137], a[192:195]\n" \
  "s_memrealtime s[58:59]\n" \
  "v_mfma_f32_16x16x32_f16 a[224:227], v[42:45], v[134:137], a[224:227]\n" \
  "buffer_load_dwordx4 v[78:81], %[bxr], s[40:43], s72 offen offset:0 sc0 sc1\n" \
  "v_mfma_f32_16x16x32_f16 a[224:227], v[38:41], v[102:105], a[224:227]\n" \
  "v_mfma_f32_16x16x32_f16 a[228:231], v[38:41], v[106:109], a[228:231]\n" \
  "v_mfma_f32_16x16x32_f16 a[228:231], v[42:45], v[138:141], a[228:231]\n" \
  "v_mfma_f32_16x16x32_f16 a[196:199], v[34:37], v[138:141], a[196:199]\n" \
  "buffer_load_dwordx4 v[82:85], %[bxr], s[40:43], s72 offen offset:1024 sc0 sc1\n" \
  "v_mfma_f32_16x16x32_f16 a[196:199], v[30:33], v[106:109], a[196:199]\n" \
  "v_mfma_f32_16x16x32_f16 a[200:203], v[30:33], v[110:113], a[200:203]\n" \
  "v_mfma_f32_16x16x32_f16 a[200:203], v[34:37], v[142:145], a[200:203]\n" \
  "s_waitcnt lgkmcnt(0)\n" \
  "s_waitcnt vmcnt(10)\n" \
  "v_mfma_f32_16x16x32_f16 a[232:235], v[42:45], v[142:145], a[232:235]\n" \
  "buffer_load_dwordx4 v[86:89], %[bxr], s[40:43], s72 offen offset:2048 sc0 sc1\n" \
  "v_mfma_f32_16x16x32_f16 a[232:235], v[38:41], v[110:113], a[232:235]\n" \
  "s_sub_u32 s82, s81, s58\n" \
  "s_branch L_s37_%=\n" \
  "L_b37_%=:\n" \
  "v_mfma_f32_16x16x32_f16 a[236:239], v[38:41], v[114:117], a[236:239]\n" \
  "s_max_i32 s82, s82, -150\n" \
  "s_add_u32 s81, s58, s82\n" \
  "s_add_u32 s81, s81, s88\n" \
  "v_mfma_f32_16x16x32_f16 a[236:239], v[42:45], v[146:149], a[236:239]\n" \
  "s_add_u32 s73, s74, s69\n" \
  "v_min3_u32 v248, v238, v239, v240\n" \
  "v_min_u32 v248, v248, v241\n" \
  "v_mfma_f32_16x16x32_f16 a[204:207], v[34:37], v[146:149], a[204:207]\n" \
  "buffer_load_dwordx4 v[90:93], %[bxr], s[40:43], s72 offen offset:3072 sc0 sc1\n" \
  "v_mfma_f32_16x16x32_f16 a[204:207], v[30:33], v[114:117], a[204:207]\n" \
  "v_cmp_le_u32 s[34:35], s76, v248\n" \
  "v_mfma_f32_16x16x32_f16 a[208:211], v[30:33], v[118:121], a[208:211]\n" \
  "v_cvt_pk_f16_f32 v217, v174, v178\n" \
  "v_cvt_pk_f16_f32 v221, v175, v179\n" \
  "v_mfma_f32_16x16x32_f16 a[208:211], v[34:37], v[150:153], a[208:211]\n" \
  "v_cvt_pk_f16_f32 v225, v176, v180\n" \
  "v_cvt_pk_f16_f32 v229, v177, v181\n" \
  "v_mfma_f32_16x16x32_f16 a[240:243], v[42:45], v[150:153], a[240:243]\n" \
  "ds_write_b128 v243, v[214:217] offset:0\n" \
  "v_mfma_f32_16x16x32_f16 a[240:243], v[38:41], v[118:121], a[240:243]\n" \
  "buffer_load_dwordx4 v[94:97], %[bxr], s[40:43], s73 offen offset:0 sc0 sc1\n" \
  "v_mfma_f32_16x16x32_f16 a[244:247], v[38:41], v[122:125], a[244:247]\n" \
  "v_mfma_f32_16x16x32_f16 a[244:247], v[42:45], v[154:157], a[244:247]\n" \
  "v_mfma_f32_16x16x32_f16 a[212:215], v[34:37], v[154:157], a[212:215]\n" \
  "ds_write_b128 v243, v[218:221] offset:1024\n" \
  "v_mfma_f32_16x16x32_f16 a[212:215], v[30:33], v[122:125], a[212:215]\n" \
  "buffer_load_dwordx4 v[98:101], %[bxr], s[40:43], s73 offen offset:1024 sc0 sc1\n" \
  "v_mfma_f32_16x16x32_f16 a[216:219], v[30:33], v[126:129], a[216:219]\n" \
  "s_add_u32 s62, s75, 0x20000\n" \
  "s_add_u32 s63, s75, 0x30000\n" \
  "s_add_u32 s64, s75, 0x100000\n" \
  "v_mfma_f32_16x16x32_f16 a[216:219], v[34:37], v[158:161], a[216:219]\n" \
  "s_add_u32 s65, s75, 0x110000\n" \
  "s_add_u32 s66, s75, 0x120000\n" \
  "s_add_u32 s67, s75, 0x130000\n" \
  "v_mfma_f32_16x16x32_f16 a[248:251], v[42:45], v[158:161], a[248:251]\n" \
  "ds_write_b128 v243, v[222:225] offset:2048\n" \
  "v_mfma_f32_16x16x32_f16 a[248:251], v[38:41], v[126:129], a[248:251]\n" \
  "buffer_load_dwordx4 v[22:25], %[bxr], s[40:43], s73 offen offset:2048 sc0 sc1\n" \
  "v_mfma_f32_16x16x32_f16 a[252:255], v[38:41], v[130:133], a[252:255]\n" \
  "v_mfma_f32_16x16x32_f16 a[252:255], v[42:45], v[162:165], a[252:255]\n" \
  "v_mfma_f32_16x16x32_f16 a[220:223], v[34:37], v[162:165], a[220:223]\n" \
  "ds_write_b128 v243, v[226:229] offset:3072\n" \
  "v_mfma_f32_16x16x32_f16 a[220:223], v[30:33], v[130:133], a[220:223]\n" \
  "buffer_load_dwordx4 v[26:29], %[bxr], s[40:43], s73 offen offset:3072 sc0 sc1\n" \
  "v_mfma_f32_16x16x32_f16 a[0:3], v[46:49], v[102:105], a[0:3]\n" \
  "s_add_u32 s72, s74, s70\n" \
  "v_mfma_f32_16x16x32_f16 a[0:3], v[50:53], v[134:137], a[0:3]\n" \
  "v_mfma_f32_16x16x32_f16 a[32:35], v[58:61], v[134:137], a[32:35]\n" \
  "v_mfma_f32_16x16x32_f16 a[32:35], v[54:57], v[102:105], a[32:35]\n" \
  "v_cvt_pk_f16_f32 v214, v182, v186\n" \
  "v_cvt_pk_f16_f32 v218, v183, v187\n" \
  "v_mfma_f32_16x16x32_f16 a[36:39], v[54:57], v[106:109], a[36:39]\n" \
  "v_cvt_pk_f16_f32 v222, v184, v188\n" \
  "v_cvt_pk_f16_f32 v226, v185, v189\n" \
  "v_mfma_f32_16x16x32_f16 a[36:39], v[58:61], v[138:141], a[36:39]\n" \
  "v_cvt_pk_f16_f32 v215, v190, v194\n" \
  "v_cvt_pk_f16_f32 v219, v191, v195\n" \
  "v_mfma_f32_16x16x32_f16 a[4:7], v[50:53], v[138:141], a[4:7]\n" \
  "v_cvt_pk_f16_f32 v223, v192, v196\n" \
  "v_cvt_pk_f16_f32 v227, v193, v197\n" \
  "v_mfma_f32_16x16x32_f16 a[4:7], v[46:49], v[106:109], a[4:7]\n" \
  "buffer_load_dwordx4 v[18:21], %[bxr], s[40:43], s72 offen offset:0 sc0 sc1\n" \
  "v_mfma_f32_16x16x32_f16 a[8:11], v[46:49], v[110:113], a[8:11]\n" \
  "v_mfma_f32_16x16x32_f16 a[8:11], v[50:53], v[142:145], a[8:11]\n" \
  "v_mfma_f32_16x16x32_f16 a[40:43], v[58:61], v[142:145], a[40:43]\n" \
  "v_mfma_f32_16x16x32_f16 a[40:43], v[54:57], v[110:113], a[40:43]\n" \
  "buffer_load_dwordx4 v[14:17], %[bxr], s[40:43], s72 offen offset:1024 sc0 sc1\n" \
  "v_mfma_f32_16x16x32_f16 a[44:47], v[54:57], v[114:117], a[44:47]\n" \
  "v_mfma_f32_16x16x32_f16 a[44:47], v[58:61], v[146:149], a[44:47]\n" \
  "v_mfma_f32_16x16x32_f16 a[12:15], v[50:53], v[146:149], a[12:15]\n" \
  "v_mfma_f32_16x16x32_f16 a[12:15], v[46:49], v[114:117], a[12:15]\n" \
  "buffer_load_dwordx4 v[38:41], %[bxr], s[40:43], s72 offen offset:2048 sc0 sc1\n" \
  "v_mfma_f32_16x16x32_f16 a[16:19], v[46:49], v[118:121], a[16:19]\n" \
  "v_mfma_f32_16x16x32_f16 a[16:19], v[50:53], v[150:153], a[16:19]\n" \
  "v_mfma_f32_16x16x32_f16 a[48:51], v[58:61], v[150:153], a[48:51]\n" \
  "v_mfma_f32_16x16x32_f16 a[48:51], v[54:57], v[118:121], a[48:51]\n" \
  "buffer_load_dwordx4 v[42:45], %[bxr], s[40:43], s72 offen offset:3072 sc0 sc1\n" \
  "v_mfma_f32_16x16x32_f16 a[52:55], v[54:57], v[122:125], a[52:55]\n" \
  "v_mfma_f32_16x16x32_f16 a[52:55], v[58:61], v[154:157], a[52:55]\n" \
  "s_barrier\n" \
  "v_mfma_f32_16x16x32_f16 a[20:23], v[50:53], v[154:157], a[20:23]\n" \
  "buffer_load_dwordx4 v[174:177], %[voffW], s[36:39], s62 offen nt\n" \
  "v_mfma_f32_16x16x32_f16 a[20:23], v[46:49], v[122:125], a[20:23]\n" \
  "v_mfma_f32_16x16x32_f16 a[24:27], v[46:49], v[126:129], a[24:27]\n" \
  "buffer_load_dwordx4 v[178:181], %[voffW], s[36:39], s63 offen nt\n" \
  "v_mfma_f32_16x16x32_f16 a[24:27], v[50:53], v[158:161], a[24:27]\n" \
  "v_mfma_f32_16x16x32_f16 a[56:59], v[58:61], v[158:161], a[56:59]\n" \
  "buffer_load_dwordx4 v[182:185], %[voffW], s[36:39], s64 offen nt\n" \
  "v_mfma_f32_16x16x32_f16 a[56:59], v[54:57], v[126:129], a[56:59]\n" \
  "v_mfma_f32_16x16x32_f16 a[60:63], v[54:57], v[130:133], a[60:63]\n" \
  "buffer_load_dwordx4 v[186:189], %[voffW], s[36:39], s65 offen nt\n" \
  "v_mfma_f32_16x16x32_f16 a[60:63], v[58:61], v[162:165], a[60:63]\n" \
  "v_mfma_f32_16x16x32_f16 a[28:31], v[50:53], v[162:165], a[28:31]\n" \
  "buffer_load_dwordx4 v[190:193], %[voffW], s[36:39], s66 offen nt\n" \
  "v_mfma_f32_16x16x32_f16 a[28:31], v[46:49], v[130:133], a[28:31]\n" \
  "s_waitcnt vmcnt(22)\n" \
  "v_mfma_f32_16x16x32_f16 a[128:131], v[62:65], v[102:105], a[128:131]\n" \
  "buffer_load_dwordx4 v[194:197], %[voffW], s[36:39], s67 offen nt\n" \
  "v_mfma_f32_16x16x32_f16 a[128:131], v[66:69], v[134:137], a[128:131]\n" \
  "v_mfma_f32_16x16x32_f16 a[160:163], v[74:77], v[134:137], a[160:163]\n" \
  "ds_read_b128 v[134:137], v247 offset:8192\n" \
  "v_mfma_f32_16x16x32_f16 a[160:163], v[70:73], v[102:105], a[160:163]\n" \
  "ds_read_b128 v[102:105], v247 offset:0\n" \
  "v_mfma_f32_16x16x32_f16 a[164:167], v[70:73], v[106:109], a[164:167]\n" \
  "v_mfma_f32_16x16x32_f16 a[164:167], v[74:77], v[138:141], a[164:167]\n" \
  "v_mfma_f32_16x16x32_f16 a[132:135], v[66:69], v[138:141], a[132:135]\n" \
  "ds_read_b128 v[138:141], v247 offset:9216\n" \
  "v_mfma_f32_16x16x32_f16 a[132:135], v[62:65], v[106:109], a[132:135]\n" \
  "ds_read_b128 v[106:109], v247 offset:1024\n" \
  "v_mfma_f32_16x16x32_f16 a[136:139], v[62:65], v[110:113], a[136:139]\n" \
  "v_mfma_f32_16x16x32_f16 a[136:139], v[66:69], v[142:145], a[136:139]\n" \
  "v_mfma_f32_16x16x32_f16 a[168:171], v[74:77], v[142:145], a[168:171]\n" \
  "ds_read_b128 v[142:145], v247 offset:10240\n" \
  "v_mfma_f32_16x16x32_f16 a[168:171], v[70:73], v[110:113], a[168:171]\n" \
  "ds_read_b128 v[110:113], v247 offset:2048\n" \
  "v_mfma_f32_16x16x32_f16 a[172:175], v[70:73], v[114:117], a[172:175]\n" \
  "v_mfma_f32_16x16x32_f16 a[172:175], v[74:77], v[146:149], a[172:175]\n" \
  "v_mfma_f32_16x16x32_f16 a[140:143], v[66:69], v[146:149], a[140:143]\n" \
  "ds_read_b128 v[146:149], v247 offset:11264\n" \
  "v_mfma_f32_16x16x32_f16 a[140:143], v[62:65], v[114:117], a[140:143]\n" \
  "ds_read_b128 v[114:117], v247 offset:3072\n" \
  "v_mfma_f32_16x16x32_f16 a[144:147], v[62:65], v[118:121], a[144:147]\n" \
  "v_mfma_f32_16x16x32_f16 a[144:147], v[66:69], v[150:153], a[144:147]\n" \
  "v_mfma_f32_16x16x32_f16 a[176:179], v[74:77], v[150:153], a[176:179]\n" \
  "ds_read_b128 v[150:153], v247 offset:12288\n" \
  "v_mfma_f32_16x16x32_f16 a[176:179], v[70:73], v[118:121], a[176:179]\n" \
  "ds_read_b128 v[118:121], v247 offset:4096\n" \
  "v_mfma_f32_16x16x32_f16 a[180:183], v[70:73], v[122:125], a[180:183]\n" \
  "v_mfma_f32_16x16x32_f16 a[180:183], v[74:77], v[154:157], a[180:183]\n" \
  "v_mfma_f32_16x16x32_f16 a[148:151], v[66:69], v[154:157], a[148:151]\n" \
  "ds_read_b128 v[154:157], v247 offset:13312\n" \
  "v_mfma_f32_16x16x32_f16 a[148:151], v[62:65], v[122:125], a[148:151]\n" \
  "ds_read_b128 v[122:125], v247 offset:5120\n" \
  "v_mfma_f32_16x16x32_f16 a[152:155], v[62:65], v[126:129], a[152:155]\n" \
  "s_add_u32 s73, s74, s71\n" \
  "v_mfma_f32_16x16x32_f16 a[152:155], v[66:69], v[158:161], a[152:155]\n" \
  "v_mfma_f32_16x16x32_f16 a[184:187], v[74:77], v[158:161], a[184:187]\n" \
  "ds_read_b128 v[158:161], v247 offset:14336\n" \
  "v_mfma_f32_16x16x32_f16 a[184:187], v[70:73], v[126:129], a[184:187]\n" \
  "ds_read_b128 v[126:129], v247 offset:6144\n" \
  "v_mfma_f32_16x16x32_f16 a[188:191], v[70:73], v[130:133], a[188:191]\n" \
  "v_mfma_f32_16x16x32_f16 a[188:191], v[74:77], v[162:165], a[188:191]\n" \
  "v_mfma_f32_16x16x32_f16 a[156:159], v[66:69], v[162:165], a[156:159]\n" \
  "ds_read_b128 v[162:165], v247 offset:15360\n" \
  "v_mfma_f32_16x16x32_f16 a[156:159], v[62:65], v[130:133], a[156:159]\n" \
  "s_waitcnt vmcnt(14)\n" \
  "ds_read_b128 v[130:133], v247 offset:7168\n" \
  "s_waitcnt lgkmcnt(14)\n" \
    \
  "v_mfma_f32_16x16x32_f16 a[64:67], v[78:81], v[102:105], a[64:67]\n" \
  "buffer_load_dwordx4 v[34:37], %[bxr], s[40:43], s73 offen offset:0 sc0 sc1\n" \
  "v_mfma_f32_16x16x32_f16 a[64:67], v[82:85], v[134:137], a[64:67]\n" \
  "v_mfma_f32_16x16x32_f16 a[96:99], v[90:93], v[134:137], a[96:99]\n" \
  "v_mfma_f32_16x16x32_f16 a[96:99], v[86:89], v[102:105], a[96:99]\n" \
  "s_waitcnt lgkmcnt(12)\n" \
  "v_mfma_f32_16x16x32_f16 a[100:103], v[86:89], v[106:109], a[100:103]\n" \
  "buffer_load_dwordx4 v[30:33], %[bxr], s[40:43], s73 offen offset:1024 sc0 sc1\n" \
  "v_mfma_f32_16x16x32_f16 a[100:103], v[90:93], v[138:141], a[100:103]\n" \
  "v_mfma_f32_16x16x32_f16 a[68:71], v[82:85], v[138:141], a[68:71]\n" \
  "v_mfma_f32_16x16x32_f16 a[68:71], v[78:81], v[106:109], a[68:71]\n" \
  "s_waitcnt lgkmcnt(10)\n" \
  "v_mfma_f32_16x16x32_f16 a[72:75], v[78:81], v[110:113], a[72:75]\n" \
  "buffer_load_dwordx4 v[54:57], %[bxr], s[40:43], s73 offen offset:2048 sc0 sc1\n" \
  "v_mfma_f32_16x16x32_f16 a[72:75], v[82:85], v[142:145], a[72:75]\n" \
  "v_mfma_f32_16x16x32_f16 a[104:107], v[90:93], v[142:145], a[104:107]\n" \
  "v_mfma_f32_16x16x32_f16 a[104:107], v[86:89], v[110:113], a[104:107]\n" \
  "s_waitcnt lgkmcnt(8)\n" \
  "v_readlane_b32 s74, %[vtabA], 17\n" \
  "v_mfma_f32_16x16x32_f16 a[108:111], v[86:89], v[114:117], a[108:111]\n" \
  "buffer_load_dwordx4 v[58:61], %[bxr], s[40:43], s73 offen offset:3072 sc0 sc1\n" \
  "v_mfma_f32_16x16x32_f16 a[108:111], v[90:93], v[146:149], a[108:111]\n" \
  "s_bitcmp1_b32 s34, 17\n" \
  "s_cbranch_scc0 L_s62_%=\n" \
  "L_b62_%=:\n" \
  "v_mfma_f32_16x16x32_f16 a[76:79], v[82:85], v[146:149], a[76:79]\n" \
  "s_waitcnt lgkmcnt(6)\n" \
  "v_cvt_pk_f16_f32 v216, v198, v202\n" \
  "v_cvt_pk_f16_f32 v220, v199, v203\n" \
  "v_mfma_f32_16x16x32_f16 a[76:79], v[78:81], v[114:117], a[76:79]\n" \
  "v_cvt_pk_f16_f32 v224, v200, v204\n" \
  "v_cvt_pk_f16_f32 v228, v201, v205\n" \
  "v_mfma_f32_16x16x32_f16 a[80:83], v[78:81], v[118:121], a[80:83]\n" \
  "s_add_u32 s72, s74, s68\n" \
  "v_cvt_pk_f16_f32 v217, v206, v210\n" \
  "v_cvt_pk_f16_f32 v221, v207, v211\n" \
  "v_mfma_f32_16x16x32_f16 a[80:83], v[82:85], v[150:153], a[80:83]\n" \
  "s_waitcnt lgkmcnt(4)\n" \
  "v_cvt_pk_f16_f32 v225, v208, v212\n" \
  "v_cvt_pk_f16_f32 v229, v209, v213\n" \
  "v_mfma_f32_16x16x32_f16 a[112:115], v[90:93], v[150:153], a[112:115]\n" \
  "ds_write_b128 v243, v[214:217] offset:16384\n" \
  "v_mfma_f32_16x16x32_f16 a[112:115], v[86:89], v[118:121], a[112:115]\n" \
  "buffer_load_dwordx4 v[50:53], %[bxr], s[40:43], s72 offen offset:0 sc0 sc1\n" \
  "v_mfma_f32_16x16x32_f16 a[116:119], v[86:89], v[122:125], a[116:119]\n" \
  "v_mfma_f32_16x16x32_f16 a[116:119], v[90:93], v[154:157], a[116:119]\n" \
  "s_waitcnt lgkmcnt(3)\n" \
  "v_mfma_f32_16x16x32_f16 a[84:87], v[82:85], v[154:157], a[84:87]\n" \
  "ds_write_b128 v243, v[218:221] offset:17408\n" \
  "v_mfma_f32_16x16x32_f16 a[84:87], v[78:81], v[122:125], a[84:87]\n" \
  "buffer_load_dwordx4 v[46:49], %[bxr], s[40:43], s72 offen offset:1024 sc0 sc1\n" \
  "v_mfma_f32_16x16x32_f16 a[88:91], v[78:81], v[126:129], a[88:91]\n" \
  "v_readlane_b32 s75, %[vtabW], 25\n" \
  "v_mfma_f32_16x16x32_f16 a[88:91], v[82:85], v[158:161], a[88:91]\n" \
  "s_waitcnt lgkmcnt(2)\n" \
  "v_mfma_f32_16x16x32_f16 a[120:123], v[90:93], v[158:161], a[120:123]\n" \
  "ds_write_b128 v243, v[222:225] offset:18432\n" \
  "v_mfma_f32_16x16x32_f16 a[120:123], v[86:89], v[126:129], a[120:123]\n" \
  "buffer_load_dwordx4 v[70:73], %[bxr], s[40:43], s72 offen offset:2048 sc0 sc1\n" \
  "v_mfma_f32_16x16x32_f16 a[124:127], v[86:89], v[130:133], a[124:127]\n" \
  "s_add_u32 s60, s75, 0x0\n" \
  "s_add_u32 s61, s75, 0x10000\n" \
  "s_add_u32 s62, s75, 0x20000\n" \
  "v_mfma_f32_16x16x32_f16 a[124:127], v[90:93], v[162:165], a[124:127]\n" \
  "s_waitcnt vmcnt(17)\n" \
  "s_add_u32 s63, s75, 0x30000\n" \
  "s_add_u32 s64, s75, 0x100000\n" \
  "v_mfma_f32_16x16x32_f16 a[92:95], v[82:85], v[162:165], a[92:95]\n" \
  "ds_write_b128 v243, v[226:229] offset:19456\n" \
  "v_mfma_f32_16x16x32_f16 a[92:95], v[78:81], v[130:133], a[92:95]\n" \
  "buffer_load_dwordx4 v[74:77], %[bxr], s[40:43], s72 offen offset:3072 sc0 sc1\n" \
  "v_mfma_f32_16x16x32_f16 a[192:195], v[94:97], v[102:105], a[192:195]\n" \
  "s_add_u32 s65, s75, 0x110000\n" \
  "s_add_u32 s73, s74, s69\n" \
  "v_mfma_f32_16x16x32_f16 a[192:195], v[98:101], v[134:137], a[192:195]\n" \
  "s_memrealtime s[58:59]\n" \
  "v_mfma_f32_16x16x32_f16 a[224:227], v[26:29], v[134:137], a[224:227]\n" \
  "v_mfma_f32_16x16x32_f16 a[224:227], v[22:25], v[102:105], a[224:227]\n" \
  "v_cvt_pk_f16_f32 v214, v166, v170\n" \
  "v_cvt_pk_f16_f32 v218, v167, v171\n" \
  "v_mfma_f32_16x16x32_f16 a[228:231], v[22:25], v[106:109], a[228:231]\n" \
  "v_cvt_pk_f16_f32 v222, v168, v172\n" \
  "v_cvt_pk_f16_f32 v226, v169, v173\n" \
  "v_mfma_f32_16x16x32_f16 a[228:231], v[26:29], v[138:141], a[228:231]\n" \
  "buffer_load_dwordx4 v[66:69], %[bxr], s[40:43], s73 offen offset:0 sc0 sc1\n" \
  "v_mfma_f32_16x16x32_f16 a[196:199], v[98:101], v[138:141], a[196:199]\n" \
  "v_mfma_f32_16x16x32_f16 a[196:199], v[94:97], v[106:109], a[196:199]\n" \
  "v_mfma_f32_16x16x32_f16 a[200:203], v[94:97], v[110:113], a[200:203]\n" \
  "v_mfma_f32_16x16x32_f16 a[200:203], v[98:101], v[142:145], a[200:203]\n" \
  "buffer_load_dwordx4 v[62:65], %[bxr], s[40:43], s73 offen offset:1024 sc0 sc1\n" \
  "v_mfma_f32_16x16x32_f16 a[232:235], v[26:29], v[142:145], a[232:235]\n" \
  "s_waitcnt lgkmcnt(0)\n" \
  "v_mfma_f32_16x16x32_f16 a[232:235], v[22:25], v[110:113], a[232:235]\n" \
  "s_sub_u32 s82, s81, s58\n" \
  "s_cmp_gt_i32 s82, 0\n" \
  "s_cbranch_scc1 L_s38_%=\n" \
  "L_b38_%=:\n" \
  "v_mfma_f32_16x16x32_f16 a[236:239], v[22:25], v[114:117], a[236:239]\n" \
  "s_max_i32 s82, s82, -150\n" \
  "s_add_u32 s81, s58, s82\n" \
  "s_add_u32 s81, s81, s88\n" \
  "v_mfma_f32_16x16x32_f16 a[236:239], v[26:29], v[146:149], a[236:239]\n" \
  "buffer_load_dwordx4 v[86:89], %[bxr], s[40:43], s73 offen offset:2048 sc0 sc1\n" \
  "v_mfma_f32_16x16x32_f16 a[204:207], v[98:101], v[146:149], a[204:207]\n" \
  "v_mfma_f32_16x16x32_f16 a[204:207], v[94:97], v[114:117], a[204:207]\n" \
  "v_mfma_f32_16x16x32_f16 a[208:211], v[94:97], v[118:121], a[208:211]\n" \
  "s_andn2_b32 s79, 0x1fffff, s34\n" \
  "s_cbranch_scc1 L_pl16_%=\n" \
  "L_pb16_%=:\n" \
  "v_mfma_f32_16x16x32_f16 a[208:211], v[98:101], v[150:153], a[208:211]\n" \
  "buffer_load_dwordx4 v[90:93], %[bxr], s[40:43], s73 offen offset:3072 sc0 sc1\n" \
  "v_mfma_f32_16x16x32_f16 a[240:243], v[26:29], v[150:153], a[240:243]\n" \
  "v_mfma_f32_16x16x32_f16 a[240:243], v[22:25], v[118:121], a[240:243]\n" \
  "s_barrier\n" \
  "v_mfma_f32_16x16x32_f16 a[244:247], v[22:25], v[122:125], a[244:247]\n" \
  "buffer_load_dwordx4 v[198:201], %[voffW], s[36:39], s60 offen nt\n" \
  "v_mfma_f32_16x16x32_f16 a[244:247], v[26:29], v[154:157], a[244:247]\n" \
  "v_mfma_f32_16x16x32_f16 a[212:215], v[98:101], v[154:157], a[212:215]\n" \
  "buffer_load_dwordx4 v[202:205], %[voffW], s[36:39], s61 offen nt\n" \
  "v_mfma_f32_16x16x32_f16 a[212:215], v[94:97], v[122:125], a[212:215]\n" \
  "v_mfma_f32_16x16x32_f16 a[216:219], v[94:97], v[126:129], a[216:219]\n" \
  "buffer_load_dwordx4 v[206:209], %[voffW], s[36:39], s62 offen nt\n" \
  "v_mfma_f32_16x16x32_f16 a[216:219], v[98:101], v[158:161], a[216:219]\n" \
  "v_mfma_f32_16x16x32_f16 a[248:251], v[26:29], v[158:161], a[248:251]\n" \
  "buffer_load_dwordx4 v[210:213], %[voffW], s[36:39], s63 offen nt\n" \
  "v_mfma_f32_16x16x32_f16 a[248:251], v[22:25], v[126:129], a[248:251]\n" \
  "v_mfma_f32_16x16x32_f16 a[252:255], v[22:25], v[130:133], a[252:255]\n" \
  "buffer_load_dwordx4 v[166:169], %[voffW], s[36:39], s64 offen nt\n" \
  "v_mfma_f32_16x16x32_f16 a[252:255], v[26:29], v[162:165], a[252:255]\n" \
  "v_mfma_f32_16x16x32_f16 a[220:223], v[98:101], v[162:165], a[220:223]\n" \
  "buffer_load_dwordx4 v[170:173], %[voffW], s[36:39], s65 offen nt\n" \
  "v_mfma_f32_16x16x32_f16 a[220:223], v[94:97], v[130:133], a[220:223]\n" \
  "s_waitcnt vmcnt(24)\n" \
  "v_mfma_f32_16x16x32_f16 a[0:3], v[18:21], v[102:105], a[0:3]\n" \
  "v_mfma_f32_16x16x32_f16 a[0:3], v[14:17], v[134:137], a[0:3]\n" \
  "v_mfma_f32_16x16x32_f16 a[32:35], v[42:45], v[134:137], a[32:35]\n" \
  "v_mfma_f32_16x16x32_f16 a[32:35], v[38:41], v[102:105], a[32:35]\n" \
  "v_mfma_f32_16x16x32_f16 a[36:39], v[38:41], v[106:109], a[36:39]\n" \
  "v_mfma_f32_16x16x32_f16 a[36:39], v[42:45], v[138:141], a[36:39]\n" \
  "v_mfma_f32_16x16x32_f16 a[4:7], v[14:17], v[138:141], a[4:7]\n" \
  "v_mfma_f32_16x16x32_f16 a[4:7], v[18:21], v[106:109], a[4:7]\n" \
  "v_mfma_f32_16x16x32_f16 a[8:11], v[18:21], v[110:113], a[8:11]\n" \
  "v_mfma_f32_16x16x32_f16 a[8:11], v[14:17], v[142:145], a[8:11]\n" \
  "v_mfma_f32_16x16x32_f16 a[40:43], v[42:45], v[142:145], a[40:43]\n" \
  "v_mfma_f32_16x16x32_f16 a[40:43], v[38:41], v[110:113], a[40:43]\n" \
  "v_mfma_f32_16x16x32_f16 a[44:47], v[38:41], v[114:117], a[44:47]\n" \
  "v_mfma_f32_16x16x32_f16 a[44:47], v[42:45], v[146:149], a[44:47]\n" \
  "v_mfma_f32_16x16x32_f16 a[12:15], v[14:17], v[146:149], a[12:15]\n" \
  "v_mfma_f32_16x16x32_f16 a[12:15], v[18:21], v[114:117], a[12:15]\n" \
  "v_mfma_f32_16x16x32_f16 a[16:19], v[18:21], v[118:121], a[16:19]\n" \
  "v_mfma_f32_16x16x32_f16 a[16:19], v[14:17], v[150:153], a[16:19]\n" \
  "v_mfma_f32_16x16x32_f16 a[48:51], v[42:45], v[150:153], a[48:51]\n" \
  "v_mfma_f32_16x16x32_f16 a[48:51], v[38:41], v[118:121], a[48:51]\n" \
  "v_mfma_f32_16x16x32_f16 a[52:55], v[38:41], v[122:125], a[52:55]\n" \
  "v_mfma_f32_16x16x32_f16 a[52:55], v[42:45], v[154:157], a[52:55]\n" \
  "v_mfma_f32_16x16x32_f16 a[20:23], v[14:17], v[154:157], a[20:23]\n" \
  "s_add_u32 s72, s74, s70\n" \
  "v_mfma_f32_16x16x32_f16 a[20:23], v[18:21], v[122:125], a[20:23]\n" \
  "v_mfma_f32_16x16x32_f16 a[24:27], v[18:21], v[126:129], a[24:27]\n" \
  "v_mfma_f32_16x16x32_f16 a[24:27], v[14:17], v[158:161], a[24:27]\n" \
  "v_mfma_f32_16x16x32_f16 a[56:59], v[42:45], v[158:161], a[56:59]\n" \
  "v_mfma_f32_16x16x32_f16 a[56:59], v[38:41], v[126:129], a[56:59]\n" \
  "v_mfma_f32_16x16x32_f16 a[60:63], v[38:41], v[130:133], a[60:63]\n" \
  "v_mfma_f32_16x16x32_f16 a[60:63], v[42:45], v[162:165], a[60:63]\n" \
  "v_mfma_f32_16x16x32_f16 a[28:31], v[14:17], v[162:165], a[28:31]\n" \
  "buffer_load_dwordx4 v[82:85], %[bxr], s[40:43], s72 offen offset:0 sc0 sc1\n" \
  "v_mfma_f32_16x16x32_f16 a[28:31], v[18:21], v[130:133], a[28:31]\n" \
  "s_waitcnt vmcnt(15)\n" \
  "v_mfma_f32_16x16x32_f16 a[128:131], v[34:37], v[102:105], a[128:131]\n" \
  "v_mfma_f32_16x16x32_f16 a[128:131], v[30:33], v[134:137], a[128:131]\n" \
  "v_mfma_f32_16x16x32_f16 a[160:163], v[58:61], v[134:137], a[160:163]\n" \
  "ds_read_b128 v[134:137], v247 offset:24576\n" \
  "v_mfma_f32_16x16x32_f16 a[160:163], v[54:57], v[102:105], a[160:163]\n" \
  "ds_read_b128 v[102:105], v247 offset:16384\n" \
  "v_mfma_f32_16x16x32_f16 a[164:167], v[54:57], v[106:109], a[164:167]\n" \
  "buffer_load_dwordx4 v[78:81], %[bxr], s[40:43], s72 offen offset:1024 sc0 sc1\n" \
  "v_mfma_f32_16x16x32_f16 a[164:167], v[58:61], v[138:141], a[164:167]\n" \
  "v_mfma_f32_16x16x32_f16 a[132:135], v[30:33], v[138:141], a[132:135]\n" \
  "ds_read_b128 v[138:141], v247 offset:25600\n" \
  "v_mfma_f32_16x16x32_f16 a[132:135], v[34:37], v[106:109], a[132:135]\n" \
  "s_waitcnt vmcnt(9)\n" \
  "ds_read_b128 v[106:109], v247 offset:17408\n" \
  "v_mfma_f32_16x16x32_f16 a[136:139], v[34:37], v[110:113], a[136:139]\n" \
  "buffer_load_dwordx4 v[22:25], %[bxr], s[40:43], s72 offen offset:2048 sc0 sc1\n" \
  "v_mfma_f32_16x16x32_f16 a[136:139], v[30:33], v[142:145], a[136:139]\n" \
  "v_min3_u32 v248, v238, v239, v240\n" \
  "v_min_u32 v248, v248, v241\n" \
  "v_mfma_f32_16x16x32_f16 a[168:171], v[58:61], v[142:145], a[168:171]\n" \
  "ds_read_b128 v[142:145], v247 offset:26624\n" \
  "v_cmp_le_u32 s[34:35], s76, v248\n" \
  "v_mfma_f32_16x16x32_f16 a[168:171], v[54:57], v[110:113], a[168:171]\n" \
  "ds_read_b128 v[110:113], v247 offset:18432\n" \
  "v_mfma_f32_16x16x32_f16 a[172:175], v[54:57], v[114:117], a[172:175]\n" \
  "v_cvt_pk_f16_f32 v215, v174, v178\n" \
  "v_cvt_pk_f16_f32 v219, v175, v179\n" \
  "v_mfma_f32_16x16x32_f16 a[172:175], v[58:61], v[146:149], a[172:175]\n" \
  "v_cvt_pk_f16_f32 v223, v176, v180\n" \
  "v_cvt_pk_f16_f32 v227, v177, v181\n" \
  "v_mfma_f32_16x16x32_f16 a[140:143], v[30:33], v[146:149], a[140:143]\n" \
  "ds_read_b128 v[146:149], v247 offset:27648\n" \
  "v_cvt_pk_f16_f32 v216, v182, v186\n" \
  "v_mfma_f32_16x16x32_f16 a[140:143], v[34:37], v[114:117], a[140:143]\n" \
  "ds_read_b128 v[114:117], v247 offset:19456\n" \
  "v_cvt_pk_f16_f32 v220, v183, v187\n" \
  "v_mfma_f32_16x16x32_f16 a[144:147], v[34:37], v[118:121], a[144:147]\n" \
  "v_cvt_pk_f16_f32 v224, v184, v188\n" \
  "v_cvt_pk_f16_f32 v228, v185, v189\n" \
  "v_mfma_f32_16x16x32_f16 a[144:147], v[30:33], v[150:153], a[144:147]\n" \
  "s_add_u32 s73, s74, s71\n" \
  "v_cvt_pk_f16_f32 v217, v190, v194\n" \
  "v_cvt_pk_f16_f32 v221, v191, v195\n" \
  "v_mfma_f32_16x16x32_f16 a[176:179], v[58:61], v[150:153], a[176:179]\n" \
  "ds_read_b128 v[150:153], v247 offset:28672\n" \
  "v_cvt_pk_f16_f32 v225, v192, v196\n" \
  "v_mfma_f32_16x16x32_f16 a[176:179], v[54:57], v[118:121], a[176:179]\n" \
  "ds_read_b128 v[118:121], v247 offset:20480\n" \
  "v_cvt_pk_f16_f32 v229, v193, v197\n" \
  "v_mfma_f32_16x16x32_f16 a[180:183], v[54:57], v[122:125], a[180:183]\n" \
  "ds_write_b128 v243, v[214:217] offset:32768\n" \
  "v_mfma_f32_16x16x32_f16 a[180:183], v[58:61], v[154:157], a[180:183]\n" \
  "buffer_load_dwordx4 v[26:29], %[bxr], s[40:43], s72 offen offset:3072 sc0 sc1\n" \
  "v_mfma_f32_16x16x32_f16 a[148:151], v[30:33], v[154:157], a[148:151]\n" \
  "s_add_u32 s66, s75, 0x120000\n" \
  "ds_read_b128 v[154:157], v247 offset:29696\n" \
  "s_add_u32 s67, s75, 0x130000\n" \
  "v_mfma_f32_16x16x32_f16 a[148:151], v[34:37], v[122:125], a[148:151]\n" \
  "ds_read_b128 v[122:125], v247 offset:21504\n" \
  "v_readlane_b32 s75, %[vtabW], 26\n" \
  "v_mfma_f32_16x16x32_f16 a[152:155], v[34:37], v[126:129], a[152:155]\n" \
  "ds_write_b128 v243, v[218:221] offset:33792\n" \
  "v_mfma_f32_16x16x32_f16 a[152:155], v[30:33], v[158:161], a[152:155]\n" \
  "buffer_load_dwordx4 v[98:101], %[bxr], s[40:43], s73 offen offset:0 sc0 sc1\n" \
  "v_mfma_f32_16x16x32_f16 a[184:187], v[58:61], v[158:161], a[184:187]\n" \
  "s_add_u32 s60, s75, 0x0\n" \
  "ds_read_b128 v[158:161], v247 offset:30720\n" \
  "s_add_u32 s61, s75, 0x10000\n" \
  "v_mfma_f32_16x16x32_f16 a[184:187], v[54:57], v[126:129], a[184:187]\n" \
  "s_add_u32 s62, s75, 0x20000\n" \
  "ds_read_b128 v[126:129], v247 offset:22528\n" \
  "s_add_u32 s63, s75, 0x30000\n" \
  "v_mfma_f32_16x16x32_f16 a[188:191], v[54:57], v[130:133], a[188:191]\n" \
  "ds_write_b128 v243, v[222:225] offset:34816\n" \
  "v_mfma_f32_16x16x32_f16 a[188:191], v[58:61], v[162:165], a[188:191]\n" \
  "buffer_load_dwordx4 v[94:97], %[bxr], s[40:43], s73 offen offset:1024 sc0 sc1\n" \
  "v_mfma_f32_16x16x32_f16 a[156:159], v[30:33], v[162:165], a[156:159]\n" \
  "ds_read_b128 v[162:165], v247 offset:31744\n" \
  "v_mfma_f32_16x16x32_f16 a[156:159], v[34:37], v[130:133], a[156:159]\n" \
  "ds_read_b128 v[130:133], v247 offset:23552\n" \
  "s_waitcnt lgkmcnt(14)\n" \
    \
  "v_mfma_f32_16x16x32_f16 a[64:67], v[50:53], v[102:105], a[64:67]\n" \
  "ds_write_b128 v243, v[226:229] offset:35840\n" \
  "v_mfma_f32_16x16x32_f16 a[64:67], v[46:49], v[134:137], a[64:67]\n" \
  "buffer_load_dwordx4 v[38:41], %[bxr], s[40:43], s73 offen offset:2048 sc0 sc1\n" \
  "v_mfma_f32_16x16x32_f16 a[96:99], v[74:77], v[134:137], a[96:99]\n" \
  "v_readlane_b32 s74, %[vtabA], 18\n" \
  "v_mfma_f32_16x16x32_f16 a[96:99], v[70:73], v[102:105], a[96:99]\n" \
  "s_waitcnt lgkmcnt(14)\n" \
  "s_bitcmp1_b32 s34, 18\n" \
  "s_cbranch_scc0 L_s63_%=\n" \
  "L_b63_%=:\n" \
  "v_mfma_f32_16x16x32_f16 a[100:103], v[70:73], v[106:109], a[100:103]\n" \
  "s_add_u32 s72, s74, s68\n" \
  "v_mfma_f32_16x16x32_f16 a[100:103], v[74:77], v[138:141], a[100:103]\n" \
  "buffer_load_dwordx4 v[42:45], %[bxr], s[40:43], s73 offen offset:3072 sc0 sc1\n" \
  "v_mfma_f32_16x16x32_f16 a[68:71], v[46:49], v[138:141], a[68:71]\n" \
  "v_mfma_f32_16x16x32_f16 a[68:71], v[50:53], v[106:109], a[68:71]\n" \
  "s_waitcnt lgkmcnt(14)\n" \
  "v_mfma_f32_16x16x32_f16 a[72:75], v[50:53], v[110:113], a[72:75]\n" \
  "v_mfma_f32_16x16x32_f16 a[72:75], v[46:49], v[142:145], a[72:75]\n" \
  "buffer_load_dwordx4 v[14:17], %[bxr], s[40:43], s72 offen offset:0 sc0 sc1\n" \
  "v_mfma_f32_16x16x32_f16 a[104:107], v[74:77], v[142:145], a[104:107]\n" \
  "v_mfma_f32_16x16x32_f16 a[104:107], v[70:73], v[110:113], a[104:107]\n" \
  "s_waitcnt lgkmcnt(12)\n" \
  "v_mfma_f32_16x16x32_f16 a[108:111], v[70:73], v[114:117], a[108:111]\n" \
  "v_mfma_f32_16x16x32_f16 a[108:111], v[74:77], v[146:149], a[108:111]\n" \
  "buffer_load_dwordx4 v[18:21], %[bxr], s[40:43], s72 offen offset:1024 sc0 sc1\n" \
  "v_mfma_f32_16x16x32_f16 a[76:79], v[46:49], v[146:149], a[76:79]\n" \
  "v_mfma_f32_16x16x32_f16 a[76:79], v[50:53], v[114:117], a[76:79]\n" \
  "s_waitcnt lgkmcnt(10)\n" \
  "v_mfma_f32_16x16x32_f16 a[80:83], v[50:53], v[118:121], a[80:83]\n" \
  "s_barrier\n" \
  "v_mfma_f32_16x16x32_f16 a[80:83], v[46:49], v[150:153], a[80:83]\n" \
  "buffer_load_dwordx4 v[54:57], %[bxr], s[40:43], s72 offen offset:2048 sc0 sc1\n" \
  "v_mfma_f32_16x16x32_f16 a[112:115], v[74:77], v[150:153], a[112:115]\n" \
  "buffer_load_dwordx4 v[174:177], %[voffW], s[36:39], s66 offen nt\n" \
  "v_mfma_f32_16x16x32_f16 a[112:115], v[70:73], v[118:121], a[112:115]\n" \
  "s_waitcnt lgkmcnt(7)\n" \
  "v_mfma_f32_16x16x32_f16 a[116:119], v[70:73], v[122:125], a[116:119]\n" \
  "buffer_load_dwordx4 v[178:181], %[voffW], s[36:39], s67 offen nt\n" \
  "v_mfma_f32_16x16x32_f16 a[116:119], v[74:77], v[154:157], a[116:119]\n" \
  "buffer_load_dwordx4 v[58:61], %[bxr], s[40:43], s72 offen offset:3072 sc0 sc1\n" \
  "v_mfma_f32_16x16x32_f16 a[84:87], v[46:49], v[154:157], a[84:87]\n" \
  "buffer_load_dwordx4 v[182:185], %[voffW], s[36:39], s60 offen nt\n" \
  "v_mfma_f32_16x16x32_f16 a[84:87], v[50:53], v[122:125], a[84:87]\n" \
  "s_waitcnt lgkmcnt(4)\n" \
  "v_mfma_f32_16x16x32_f16 a[88:91], v[50:53], v[126:129], a[88:91]\n" \
  "buffer_load_dwordx4 v[186:189], %[voffW], s[36:39], s61 offen nt\n" \
  "v_mfma_f32_16x16x32_f16 a[88:91], v[46:49], v[158:161], a[88:91]\n" \
  "v_mfma_f32_16x16x32_f16 a[120:123], v[74:77], v[158:161], a[120:123]\n" \
  "buffer_load_dwordx4 v[190:193], %[voffW], s[36:39], s62 offen nt\n" \
  "v_mfma_f32_16x16x32_f16 a[120:123], v[70:73], v[126:129], a[120:123]\n" \
  "s_waitcnt lgkmcnt(1)\n" \
  "v_mfma_f32_16x16x32_f16 a[124:127], v[70:73], v[130:133], a[124:127]\n" \
  "buffer_load_dwordx4 v[194:197], %[voffW], s[36:39], s63 offen nt\n" \
  "v_mfma_f32_16x16x32_f16 a[124:127], v[74:77], v[162:165], a[124:127]\n" \
  "v_mfma_f32_16x16x32_f16 a[92:95], v[46:49], v[162:165], a[92:95]\n" \
  "v_mfma_f32_16x16x32_f16 a[92:95], v[50:53], v[130:133], a[92:95]\n" \
  "s_waitcnt vmcnt(24)\n" \
  "v_mfma_f32_16x16x32_f16 a[192:195], v[66:69], v[102:105], a[192:195]\n" \
  "v_mfma_f32_16x16x32_f16 a[192:195], v[62:65], v[134:137], a[192:195]\n" \
  "s_memrealtime s[58:59]\n" \
  "v_mfma_f32_16x16x32_f16 a[224:227], v[90:93], v[134:137], a[224:227]\n" \
  "v_mfma_f32_16x16x32_f16 a[224:227], v[86:89], v[102:105], a[224:227]\n" \
  "v_mfma_f32_16x16x32_f16 a[228:231], v[86:89], v[106:109], a[228:231]\n" \
  "v_mfma_f32_16x16x32_f16 a[228:231], v[90:93], v[138:141], a[228:231]\n" \
  "v_mfma_f32_16x16x32_f16 a[196:199], v[62:65], v[138:141], a[196:199]\n" \
  "v_mfma_f32_16x16x32_f16 a[196:199], v[66:69], v[106:109], a[196:199]\n" \
  "v_mfma_f32_16x16x32_f16 a[200:203], v[66:69], v[110:113], a[200:203]\n" \
  "v_mfma_f32_16x16x32_f16 a[200:203], v[62:65], v[142:145], a[200:203]\n" \
  "s_waitcnt lgkmcnt(0)\n" \
  "v_mfma_f32_16x16x32_f16 a[232:235], v[90:93], v[142:145], a[232:235]\n" \
  "s_sub_u32 s82, s81, s58\n" \
  "s_cmp_gt_i32 s82, 0\n" \
  "s_cbranch_scc1 L_s39_%=\n" \
  "L_b39_%=:\n" \
  "v_mfma_f32_16x16x32_f16 a[232:235], v[86:89], v[110:113], a[232:235]\n" \
  "s_max_i32 s82, s82, -150\n" \
  "s_add_u32 s81, s58, s82\n" \
  "s_add_u32 s81, s81, s88\n" \
  "v_mfma_f32_16x16x32_f16 a[236:239], v[86:89], v[114:117], a[236:239]\n" \
  "v_mfma_f32_16x16x32_f16 a[236:239], v[90:93], v[146:149], a[236:239]\n" \
  "v_mfma_f32_16x16x32_f16 a[204:207], v[62:65], v[146:149], a[204:207]\n" \
  "v_mfma_f32_16x16x32_f16 a[204:207], v[66:69], v[114:117], a[204:207]\n" \
  "v_mfma_f32_16x16x32_f16 a[208:211], v[66:69], v[118:121], a[208:211]\n" \
  "v_mfma_f32_16x16x32_f16 a[208:211], v[62:65], v[150:153], a[208:211]\n" \
  "v_mfma_f32_16x16x32_f16 a[240:243], v[90:93], v[150:153], a[240:243]\n" \
  "v_mfma_f32_16x16x32_f16 a[240:243], v[86:89], v[118:121], a[240:243]\n" \
  "v_mfma_f32_16x16x32_f16 a[244:247], v[86:89], v[122:125], a[244:247]\n" \
  "s_add_u32 s73, s74, s69\n" \
  "v_mfma_f32_16x16x32_f16 a[244:247], v[90:93], v[154:157], a[244:247]\n" \
  "v_mfma_f32_16x16x32_f16 a[212:215], v[62:65], v[154:157], a[212:215]\n" \
  "v_mfma_f32_16x16x32_f16 a[212:215], v[66:69], v[122:125], a[212:215]\n" \
  "v_mfma_f32_16x16x32_f16 a[216:219], v[66:69], v[126:129], a[216:219]\n" \
  "v_mfma_f32_16x16x32_f16 a[216:219], v[62:65], v[158:161], a[216:219]\n" \
  "v_mfma_f32_16x16x32_f16 a[248:251], v[90:93], v[158:161], a[248:251]\n" \
  "v_mfma_f32_16x16x32_f16 a[248:251], v[86:89], v[126:129], a[248:251]\n" \
  "v_mfma_f32_16x16x32_f16 a[252:255], v[86:89], v[130:133], a[252:255]\n" \
  "buffer_load_dwordx4 v[30:33], %[bxr], s[40:43], s73 offen offset:0 sc0 sc1\n" \
  "v_mfma_f32_16x16x32_f16 a[252:255], v[90:93], v[162:165], a[252:255]\n" \
  "v_mfma_f32_16x16x32_f16 a[220:223], v[62:65], v[162:165], a[220:223]\n" \
  "v_mfma_f32_16x16x32_f16 a[220:223], v[66:69], v[130:133], a[220:223]\n" \
  "s_waitcnt vmcnt(15)\n" \
  "v_mfma_f32_16x16x32_f16 a[0:3], v[82:85], v[102:105], a[0:3]\n" \
  "buffer_load_dwordx4 v[34:37], %[bxr], s[40:43], s73 offen offset:1024 sc0 sc1\n" \
  "v_mfma_f32_16x16x32_f16 a[0:3], v[78:81], v[134:137], a[0:3]\n" \
  "v_mfma_f32_16x16x32_f16 a[32:35], v[26:29], v[134:137], a[32:35]\n" \
  "v_mfma_f32_16x16x32_f16 a[32:35], v[22:25], v[102:105], a[32:35]\n" \
  "v_mfma_f32_16x16x32_f16 a[36:39], v[22:25], v[106:109], a[36:39]\n" \
  "buffer_load_dwordx4 v[70:73], %[bxr], s[40:43], s73 offen offset:2048 sc0 sc1\n" \
  "v_mfma_f32_16x16x32_f16 a[36:39], v[26:29], v[138:141], a[36:39]\n" \
  "v_mfma_f32_16x16x32_f16 a[4:7], v[78:81], v[138:141], a[4:7]\n" \
  "v_mfma_f32_16x16x32_f16 a[4:7], v[82:85], v[106:109], a[4:7]\n" \
  "v_mfma_f32_16x16x32_f16 a[8:11], v[82:85], v[110:113], a[8:11]\n" \
  "buffer_load_dwordx4 v[74:77], %[bxr], s[40:43], s73 offen offset:3072 sc0 sc1\n" \
  "v_mfma_f32_16x16x32_f16 a[8:11], v[78:81], v[142:145], a[8:11]\n" \
  "s_add_u32 s72, s74, s70\n" \
  "v_mfma_f32_16x16x32_f16 a[40:43], v[26:29], v[142:145], a[40:43]\n" \
  "v_cvt_pk_f16_f32 v214, v198, v202\n" \
  "v_cvt_pk_f16_f32 v218, v199, v203\n" \
  "v_mfma_f32_16x16x32_f16 a[40:43], v[22:25], v[110:113], a[40:43]\n" \
  "v_cvt_pk_f16_f32 v222, v200, v204\n" \
  "v_cvt_pk_f16_f32 v226, v201, v205\n" \
  "v_mfma_f32_16x16x32_f16 a[44:47], v[22:25], v[114:117], a[44:47]\n" \
  "v_cvt_pk_f16_f32 v215, v206, v210\n" \
  "v_cvt_pk_f16_f32 v219, v207, v211\n" \
  "v_mfma_f32_16x16x32_f16 a[44:47], v[26:29], v[146:149], a[44:47]\n" \
  "v_cvt_pk_f16_f32 v223, v208, v212\n" \
  "v_cvt_pk_f16_f32 v227, v209, v213\n" \
  "v_mfma_f32_16x16x32_f16 a[12:15], v[78:81], v[146:149], a[12:15]\n" \
  "v_cvt_pk_f16_f32 v216, v166, v170\n" \
  "v_cvt_pk_f16_f32 v220, v167, v171\n" \
  "v_mfma_f32_16x16x32_f16 a[12:15], v[82:85], v[114:117], a[12:15]\n" \
  "v_cvt_pk_f16_f32 v224, v168, v172\n" \
  "v_cvt_pk_f16_f32 v228, v169, v173\n" \
  "v_mfma_f32_16x16x32_f16 a[16:19], v[82:85], v[118:121], a[16:19]\n" \
  "buffer_load_dwordx4 v[46:49], %[bxr], s[40:43], s72 offen offset:0 sc0 sc1\n" \
  "v_mfma_f32_16x16x32_f16 a[16:19], v[78:81], v[150:153], a[16:19]\n" \
  "v_mfma_f32_16x16x32_f16 a[48:51], v[26:29], v[150:153], a[48:51]\n" \
  "s_add_u32 s64, s75, 0x100000\n" \
  "s_add_u32 s65, s75, 0x110000\n" \
  "s_add_u32 s66, s75, 0x120000\n" \
  "v_mfma_f32_16x16x32_f16 a[48:51], v[22:25], v[118:121], a[48:51]\n" \
  "s_add_u32 s67, s75, 0x130000\n" \
  "v_mfma_f32_16x16x32_f16 a[52:55], v[22:25], v[122:125], a[52:55]\n" \
  "buffer_load_dwordx4 v[50:53], %[bxr], s[40:43], s72 offen offset:1024 sc0 sc1\n" \
  "v_mfma_f32_16x16x32_f16 a[52:55], v[26:29], v[154:157], a[52:55]\n" \
  "v_readlane_b32 s75, %[vtabW], 27\n" \
  "v_mfma_f32_16x16x32_f16 a[20:23], v[78:81], v[154:157], a[20:23]\n" \
  "v_mfma_f32_16x16x32_f16 a[20:23], v[82:85], v[122:125], a[20:23]\n" \
  "s_add_u32 s60, s75, 0x0\n" \
  "s_add_u32 s61, s75, 0x10000\n" \
  "v_mfma_f32_16x16x32_f16 a[24:27], v[82:85], v[126:129], a[24:27]\n" \
  "buffer_load_dwordx4 v[86:89], %[bxr], s[40:43], s72 offen offset:2048 sc0 sc1\n" \
  "v_mfma_f32_16x16x32_f16 a[24:27], v[78:81], v[158:161], a[24:27]\n" \
  "s_add_u32 s73, s74, s71\n" \
  "v_mfma_f32_16x16x32_f16 a[56:59], v[26:29], v[158:161], a[56:59]\n" \
  "v_mfma_f32_16x16x32_f16 a[56:59], v[22:25], v[126:129], a[56:59]\n" \
  "v_mfma_f32_16x16x32_f16 a[60:63], v[22:25], v[130:133], a[60:63]\n" \
  "buffer_load_dwordx4 v[90:93], %[bxr], s[40:43], s72 offen offset:3072 sc0 sc1\n" \
  "v_mfma_f32_16x16x32_f16 a[60:63], v[26:29], v[162:165], a[60:63]\n" \
  "v_mfma_f32_16x16x32_f16 a[28:31], v[78:81], v[162:165], a[28:31]\n" \
  "v_mfma_f32_16x16x32_f16 a[28:31], v[82:85], v[130:133], a[28:31]\n" \
  "s_waitcnt vmcnt(18)\n" \
  "v_mfma_f32_16x16x32_f16 a[128:131], v[98:101], v[102:105], a[128:131]\n" \
  "buffer_load_dwordx4 v[62:65], %[bxr], s[40:43], s73 offen offset:0 sc0 sc1\n" \
  "v_mfma_f32_16x16x32_f16 a[128:131], v[94:97], v[134:137], a[128:131]\n" \
  "v_mfma_f32_16x16x32_f16 a[160:163], v[42:45], v[134:137], a[160:163]\n" \
  "ds_read_b128 v[134:137], v245 offset:8192\n" \
  "v_mfma_f32_16x16x32_f16 a[160:163], v[38:41], v[102:105], a[160:163]\n" \
  "ds_read_b128 v[102:105], v245 offset:0\n" \
  "v_mfma_f32_16x16x32_f16 a[164:167], v[38:41], v[106:109], a[164:167]\n" \
  "buffer_load_dwordx4 v[66:69], %[bxr], s[40:43], s73 offen offset:1024 sc0 sc1\n" \
  "v_mfma_f32_16x16x32_f16 a[164:167], v[42:45], v[138:141], a[164:167]\n" \
  "v_mfma_f32_16x16x32_f16 a[132:135], v[94:97], v[138:141], a[132:135]\n" \
  "ds_read_b128 v[138:141], v245 offset:9216\n" \
  "v_mfma_f32_16x16x32_f16 a[132:135], v[98:101], v[106:109], a[132:135]\n" \
  "ds_read_b128 v[106:109], v245 offset:1024\n" \
  "v_mfma_f32_16x16x32_f16 a[136:139], v[98:101], v[110:113], a[136:139]\n" \
  "buffer_load_dwordx4 v[22:25], %[bxr], s[40:43], s73 offen offset:2048 sc0 sc1\n" \
  "v_mfma_f32_16x16x32_f16 a[136:139], v[94:97], v[142:145], a[136:139]\n" \
  "v_mfma_f32_16x16x32_f16 a[168:171], v[42:45], v[142:145], a[168:171]\n" \
  "ds_read_b128 v[142:145], v245 offset:10240\n" \
  "v_mfma_f32_16x16x32_f16 a[168:171], v[38:41], v[110:113], a[168:171]\n" \
  "ds_read_b128 v[110:113], v245 offset:2048\n" \
  "v_mfma_f32_16x16x32_f16 a[172:175], v[38:41], v[114:117], a[172:175]\n" \
  "buffer_load_dwordx4 v[26:29], %[bxr], s[40:43], s73 offen offset:3072 sc0 sc1\n" \
  "v_mfma_f32_16x16x32_f16 a[172:175], v[42:45], v[146:149], a[172:175]\n" \
  "s_andn2_b32 s79, 0x7fffff, s34\n" \
  "s_cbranch_scc1 L_pl17_%=\n" \
  "L_pb17_%=:\n" \
  "v_mfma_f32_16x16x32_f16 a[140:143], v[94:97], v[146:149], a[140:143]\n" \
  "ds_read_b128 v[146:149], v245 offset:11264\n" \
  "v_mfma_f32_16x16x32_f16 a[140:143], v[98:101], v[114:117], a[140:143]\n" \
  "s_barrier\n" \
  "ds_read_b128 v[114:117], v245 offset:3072\n" \
  "v_mfma_f32_16x16x32_f16 a[144:147], v[98:101], v[118:121], a[144:147]\n" \
  "buffer_load_dwordx4 v[198:201], %[voffW], s[36:39], s64 offen nt\n" \
  "v_mfma_f32_16x16x32_f16 a[144:147], v[94:97], v[150:153], a[144:147]\n" \
  "v_mfma_f32_16x16x32_f16 a[176:179], v[42:45], v[150:153], a[176:179]\n" \
  "ds_read_b128 v[150:153], v245 offset:12288\n" \
  "v_mfma_f32_16x16x32_f16 a[176:179], v[38:41], v[118:121], a[176:179]\n" \
  "ds_read_b128 v[118:121], v245 offset:4096\n" \
  "v_mfma_f32_16x16x32_f16 a[180:183], v[38:41], v[122:125], a[180:183]\n" \
  "buffer_load_dwordx4 v[202:205], %[voffW], s[36:39], s65 offen nt\n" \
  "v_mfma_f32_16x16x32_f16 a[180:183], v[42:45], v[154:157], a[180:183]\n" \
  "v_mfma_f32_16x16x32_f16 a[148:151], v[94:97], v[154:157], a[148:151]\n" \
  "ds_read_b128 v[154:157], v245 offset:13312\n" \
  "v_mfma_f32_16x16x32_f16 a[148:151], v[98:101], v[122:125], a[148:151]\n" \
  "ds_read_b128 v[122:125], v245 offset:5120\n" \
  "v_mfma_f32_16x16x32_f16 a[152:155], v[98:101], v[126:129], a[152:155]\n" \
  "buffer_load_dwordx4 v[206:209], %[voffW], s[36:39], s66 offen nt\n" \
  "v_mfma_f32_16x16x32_f16 a[152:155], v[94:97], v[158:161], a[152:155]\n" \
  "v_mfma_f32_16x16x32_f16 a[184:187], v[42:45], v[158:161], a[184:187]\n" \
  "ds_read_b128 v[158:161], v245 offset:14336\n" \
  "v_mfma_f32_16x16x32_f16 a[184:187], v[38:41], v[126:129], a[184:187]\n" \
  "ds_read_b128 v[126:129], v245 offset:6144\n" \
  "v_mfma_f32_16x16x32_f16 a[188:191], v[38:41], v[130:133], a[188:191]\n" \
  "buffer_load_dwordx4 v[210:213], %[voffW], s[36:39], s67 offen nt\n" \
  "s_getpc_b64 s[48:49]\n" \
  "L_cw2_%=:\n" \
  "v_min_u32 v230, L_done_%=-L_cw2_%=, %[voffC]\n" \
  "global_load_dword v231, v230, s[48:49]\n" \
  "v_mfma_f32_16x16x32_f16 a[188:191], v[42:45], v[162:165], a[188:191]\n" \
  "v_mfma_f32_16x16x32_f16 a[156:159], v[94:97], v[162:165], a[156:159]\n" \
  "ds_read_b128 v[162:165], v245 offset:15360\n" \
  "v_mfma_f32_16x16x32_f16 a[156:159], v[98:101], v[130:133], a[156:159]\n" \
  "s_waitcnt vmcnt(20)\n" \
  "ds_read_b128 v[130:133], v245 offset:7168\n" \
  "s_waitcnt lgkmcnt(14)\n" \
    \
  "v_mfma_f32_16x16x32_f16 a[64:67], v[14:17], v[102:105], a[64:67]\n" \
  "buffer_load_dwordx4 v[166:169], %[voffW], s[36:39], s60 offen nt\n" \
  "v_mfma_f32_16x16x32_f16 a[64:67], v[18:21], v[134:137], a[64:67]\n" \
  "v_mfma_f32_16x16x32_f16 a[96:99], v[58:61], v[134:137], a[96:99]\n" \
  "buffer_load_dwordx4 v[170:173], %[voffW], s[36:39], s61 offen nt\n" \
  "v_mfma_f32_16x16x32_f16 a[96:99], v[54:57], v[102:105], a[96:99]\n" \
  "s_waitcnt lgkmcnt(12)\n" \
  "v_mfma_f32_16x16x32_f16 a[100:103], v[54:57], v[106:109], a[100:103]\n" \
  "v_mfma_f32_16x16x32_f16 a[100:103], v[58:61], v[138:141], a[100:103]\n" \
  "v_mfma_f32_16x16x32_f16 a[68:71], v[18:21], v[138:141], a[68:71]\n" \
  "v_mfma_f32_16x16x32_f16 a[68:71], v[14:17], v[106:109], a[68:71]\n" \
  "s_waitcnt lgkmcnt(10)\n" \
  "v_mfma_f32_16x16x32_f16 a[72:75], v[14:17], v[110:113], a[72:75]\n" \
  "v_mfma_f32_16x16x32_f16 a[72:75], v[18:21], v[142:145], a[72:75]\n" \
  "v_mfma_f32_16x16x32_f16 a[104:107], v[58:61], v[142:145], a[104:107]\n" \
  "v_mfma_f32_16x16x32_f16 a[104:107], v[54:57], v[110:113], a[104:107]\n" \
  "s_waitcnt lgkmcnt(8)\n" \
  "v_mfma_f32_16x16x32_f16 a[108:111], v[54:57], v[114:117], a[108:111]\n" \
  "v_mfma_f32_16x16x32_f16 a[108:111], v[58:61], v[146:149], a[108:111]\n" \
  "v_mfma_f32_16x16x32_f16 a[76:79], v[18:21], v[146:149], a[76:79]\n" \
  "v_mfma_f32_16x16x32_f16 a[76:79], v[14:17], v[114:117], a[76:79]\n" \
  "s_waitcnt lgkmcnt(6)\n" \
  "v_mfma_f32_16x16x32_f16 a[80:83], v[14:17], v[118:121], a[80:83]\n" \
  "v_mfma_f32_16x16x32_f16 a[80:83], v[18:21], v[150:153], a[80:83]\n" \
  "v_mfma_f32_16x16x32_f16 a[112:115], v[58:61], v[150:153], a[112:115]\n" \
  "v_readlane_b32 s74, %[vtabA], 19\n" \
  "v_mfma_f32_16x16x32_f16 a[112:115], v[54:57], v[118:121], a[112:115]\n" \
  "s_waitcnt lgkmcnt(4)\n" \
  "s_bitcmp1_b32 s34, 19\n" \
  "s_cbranch_scc0 L_s64_%=\n" \
  "L_b64_%=:\n" \
  "v_mfma_f32_16x16x32_f16 a[116:119], v[54:57], v[122:125], a[116:119]\n" \
  "s_add_u32 s72, s74, s68\n" \
  "v_mfma_f32_16x16x32_f16 a[116:119], v[58:61], v[154:157], a[116:119]\n" \
  "v_mfma_f32_16x16x32_f16 a[84:87], v[18:21], v[154:157], a[84:87]\n" \
  "v_mfma_f32_16x16x32_f16 a[84:87], v[14:17], v[122:125], a[84:87]\n" \
  "s_waitcnt lgkmcnt(2)\n" \
  "v_mfma_f32_16x16x32_f16 a[88:91], v[14:17], v[126:129], a[88:91]\n" \
  "v_mfma_f32_16x16x32_f16 a[88:91], v[18:21], v[158:161], a[88:91]\n" \
  "v_mfma_f32_16x16x32_f16 a[120:123], v[58:61], v[158:161], a[120:123]\n" \
  "buffer_load_dwordx4 v[78:81], %[bxr], s[40:43], s72 offen offset:0 sc0 sc1\n" \
  "v_mfma_f32_16x16x32_f16 a[120:123], v[54:57], v[126:129], a[120:123]\n" \
  "s_waitcnt lgkmcnt(0)\n" \
  "v_mfma_f32_16x16x32_f16 a[124:127], v[54:57], v[130:133], a[124:127]\n" \
  "v_mfma_f32_16x16x32_f16 a[124:127], v[58:61], v[162:165], a[124:127]\n" \
  "v_mfma_f32_16x16x32_f16 a[92:95], v[18:21], v[162:165], a[92:95]\n" \
  "buffer_load_dwordx4 v[82:85], %[bxr], s[40:43], s72 offen offset:1024 sc0 sc1\n" \
  "v_mfma_f32_16x16x32_f16 a[92:95], v[14:17], v[130:133], a[92:95]\n" \
  "s_waitcnt vmcnt(16)\n" \
  "v_mfma_f32_16x16x32_f16 a[192:195], v[30:33], v[102:105], a[192:195]\n" \
  "v_mfma_f32_16x16x32_f16 a[192:195], v[34:37], v[134:137], a[192:195]\n" \
  "s_memrealtime s[58:59]\n" \
  "v_mfma_f32_16x16x32_f16 a[224:227], v[74:77], v[134:137], a[224:227]\n" \
  "buffer_load_dwordx4 v[38:41], %[bxr], s[40:43], s72 offen offset:2048 sc0 sc1\n" \
  "v_mfma_f32_16x16x32_f16 a[224:227], v[70:73], v[102:105], a[224:227]\n" \
  "s_add_u32 s73, s74, s69\n" \
  "v_mfma_f32_16x16x32_f16 a[228:231], v[70:73], v[106:109], a[228:231]\n" \
  "s_waitcnt vmcnt(9)\n" \
  "v_mfma_f32_16x16x32_f16 a[228:231], v[74:77], v[138:141], a[228:231]\n" \
  "v_min3_u32 v248, v238, v239, v240\n" \
  "v_min_u32 v248, v248, v241\n" \
  "v_mfma_f32_16x16x32_f16 a[196:199], v[34:37], v[138:141], a[196:199]\n" \
  "buffer_load_dwordx4 v[42:45], %[bxr], s[40:43], s72 offen offset:3072 sc0 sc1\n" \
  "v_mfma_f32_16x16x32_f16 a[196:199], v[30:33], v[106:109], a[196:199]\n" \
  "v_cmp_le_u32 s[34:35], s76, v248\n" \
  "v_mfma_f32_16x16x32_f16 a[200:203], v[30:33], v[110:113], a[200:203]\n" \
  "v_cvt_pk_f16_f32 v217, v174, v178\n" \
  "v_cvt_pk_f16_f32 v221, v175, v179\n" \
  "v_mfma_f32_16x16x32_f16 a[200:203], v[34:37], v[142:145], a[200:203]\n" \
  "s_waitcnt lgkmcnt(0)\n" \
  "v_cvt_pk_f16_f32 v225, v176, v180\n" \
  "v_cvt_pk_f16_f32 v229, v177, v181\n" \
  "v_mfma_f32_16x16x32_f16 a[232:235], v[74:77], v[142:145], a[232:235]\n" \
  "ds_write_b128 v243, v[214:217] offset:49152\n" \
  "v_mfma_f32_16x16x32_f16 a[232:235], v[70:73], v[110:113], a[232:235]\n" \
  "buffer_load_dwordx4 v[94:97], %[bxr], s[40:43], s73 offen offset:0 sc0 sc1\n" \
  "v_mfma_f32_16x16x32_f16 a[236:239], v[70:73], v[114:117], a[236:239]\n" \
  "s_sub_u32 s82, s81, s58\n" \
  "s_cmp_gt_i32 s82, 0\n" \
  "s_cbranch_scc1 L_s40_%=\n" \
  "L_b40_%=:\n" \
  "v_mfma_f32_16x16x32_f16 a[236:239], v[74:77], v[146:149], a[236:239]\n" \
  "s_max_i32 s82, s82, -150\n" \
  "s_add_u32 s81, s58, s82\n" \
  "s_add_u32 s81, s81, s88\n" \
  "v_mfma_f32_16x16x32_f16 a[204:207], v[34:37], v[146:149], a[204:207]\n" \
  "ds_write_b128 v243, v[218:221] offset:50176\n" \
  "v_mfma_f32_16x16x32_f16 a[204:207], v[30:33], v[114:117], a[204:207]\n" \
  "buffer_load_dwordx4 v[98:101], %[bxr], s[40:43], s73 offen offset:1024 sc0 sc1\n" \
  "v_mfma_f32_16x16x32_f16 a[208:211], v[30:33], v[118:121], a[208:211]\n" \
  "s_add_u32 s62, s75, 0x20000\n" \
  "s_add_u32 s63, s75, 0x30000\n" \
  "s_add_u32 s64, s75, 0x100000\n" \
  "v_mfma_f32_16x16x32_f16 a[208:211], v[34:37], v[150:153], a[208:211]\n" \
  "s_add_u32 s65, s75, 0x110000\n" \
  "s_add_u32 s66, s75, 0x120000\n" \
  "s_add_u32 s67, s75, 0x130000\n" \
  "v_mfma_f32_16x16x32_f16 a[240:243], v[74:77], v[150:153], a[240:243]\n" \
  "ds_write_b128 v243, v[222:225] offset:51200\n" \
  "v_mfma_f32_16x16x32_f16 a[240:243], v[70:73], v[118:121], a[240:243]\n" \
  "buffer_load_dwordx4 v[54:57], %[bxr], s[40:43], s73 offen offset:2048 sc0 sc1\n" \
  "v_mfma_f32_16x16x32_f16 a[244:247], v[70:73], v[122:125], a[244:247]\n" \
  "v_mfma_f32_16x16x32_f16 a[244:247], v[74:77], v[154:157], a[244:247]\n" \
  "v_mfma_f32_16x16x32_f16 a[212:215], v[34:37], v[154:157], a[212:215]\n" \
  "ds_write_b128 v243, v[226:229] offset:52224\n" \
  "v_mfma_f32_16x16x32_f16 a[212:215], v[30:33], v[122:125], a[212:215]\n" \
  "buffer_load_dwordx4 v[58:61], %[bxr], s[40:43], s73 offen offset:3072 sc0 sc1\n" \
  "v_mfma_f32_16x16x32_f16 a[216:219], v[30:33], v[126:129], a[216:219]\n" \
  "s_add_u32 s72, s74, s70\n" \
  "v_mfma_f32_16x16x32_f16 a[216:219], v[34:37], v[158:161], a[216:219]\n" \
  "v_mfma_f32_16x16x32_f16 a[248:251], v[74:77], v[158:161], a[248:251]\n" \
  "v_mfma_f32_16x16x32_f16 a[248:251], v[70:73], v[126:129], a[248:251]\n" \
  "v_cvt_pk_f16_f32 v214, v182, v186\n" \
  "v_cvt_pk_f16_f32 v218, v183, v187\n" \
  "v_mfma_f32_16x16x32_f16 a[252:255], v[70:73], v[130:133], a[252:255]\n" \
  "v_cvt_pk_f16_f32 v222, v184, v188\n" \
  "v_cvt_pk_f16_f32 v226, v185, v189\n" \
  "v_mfma_f32_16x16x32_f16 a[252:255], v[74:77], v[162:165], a[252:255]\n" \
  "v_cvt_pk_f16_f32 v215, v190, v194\n" \
  "v_cvt_pk_f16_f32 v219, v191, v195\n" \
  "v_mfma_f32_16x16x32_f16 a[220:223], v[34:37], v[162:165], a[220:223]\n" \
  "v_cvt_pk_f16_f32 v223, v192, v196\n" \
  "v_cvt_pk_f16_f32 v227, v193, v197\n" \
  "v_mfma_f32_16x16x32_f16 a[220:223], v[30:33], v[130:133], a[220:223]\n" \
  "buffer_load_dwordx4 v[18:21], %[bxr], s[40:43], s72 offen offset:0 sc0 sc1\n" \
  "v_mfma_f32_16x16x32_f16 a[0:3], v[46:49], v[102:105], a[0:3]\n" \
  "v_mfma_f32_16x16x32_f16 a[0:3], v[50:53], v[134:137], a[0:3]\n" \
  "v_mfma_f32_16x16x32_f16 a[32:35], v[90:93], v[134:137], a[32:35]\n" \
  "v_mfma_f32_16x16x32_f16 a[32:35], v[86:89], v[102:105], a[32:35]\n" \
  "buffer_load_dwordx4 v[14:17], %[bxr], s[40:43], s72 offen offset:1024 sc0 sc1\n" \
  "v_mfma_f32_16x16x32_f16 a[36:39], v[86:89], v[106:109], a[36:39]\n" \
  "v_mfma_f32_16x16x32_f16 a[36:39], v[90:93], v[138:141], a[36:39]\n" \
  "v_mfma_f32_16x16x32_f16 a[4:7], v[50:53], v[138:141], a[4:7]\n" \
  "v_mfma_f32_16x16x32_f16 a[4:7], v[46:49], v[106:109], a[4:7]\n" \
  "buffer_load_dwordx4 v[70:73], %[bxr], s[40:43], s72 offen offset:2048 sc0 sc1\n" \
  "v_mfma_f32_16x16x32_f16 a[8:11], v[46:49], v[110:113], a[8:11]\n" \
  "v_mfma_f32_16x16x32_f16 a[8:11], v[50:53], v[142:145], a[8:11]\n" \
  "v_mfma_f32_16x16x32_f16 a[40:43], v[90:93], v[142:145], a[40:43]\n" \
  "v_mfma_f32_16x16x32_f16 a[40:43], v[86:89], v[110:113], a[40:43]\n" \
  "buffer_load_dwordx4 v[74:77], %[bxr], s[40:43], s72 offen offset:3072 sc0 sc1\n" \
  "v_mfma_f32_16x16x32_f16 a[44:47], v[86:89], v[114:117], a[44:47]\n" \
  "v_mfma_f32_16x16x32_f16 a[44:47], v[90:93], v[146:149], a[44:47]\n" \
  "s_barrier\n" \
  "v_mfma_f32_16x16x32_f16 a[12:15], v[50:53], v[146:149], a[12:15]\n" \
  "buffer_load_dwordx4 v[174:177], %[voffW], s[36:39], s62 offen nt\n" \
  "v_mfma_f32_16x16x32_f16 a[12:15], v[46:49], v[114:117], a[12:15]\n" \
  "v_mfma_f32_16x16x32_f16 a[16:19], v[46:49], v[118:121], a[16:19]\n" \
  "buffer_load_dwordx4 v[178:181], %[voffW], s[36:39], s63 offen nt\n" \
  "v_mfma_f32_16x16x32_f16 a[16:19], v[50:53], v[150:153], a[16:19]\n" \
  "v_mfma_f32_16x16x32_f16 a[48:51], v[90:93], v[150:153], a[48:51]\n" \
  "buffer_load_dwordx4 v[182:185], %[voffW], s[36:39], s64 offen nt\n" \
  "v_mfma_f32_16x16x32_f16 a[48:51], v[86:89], v[118:121], a[48:51]\n" \
  "v_mfma_f32_16x16x32_f16 a[52:55], v[86:89], v[122:125], a[52:55]\n" \
  "buffer_load_dwordx4 v[186:189], %[voffW], s[36:39], s65 offen nt\n" \
  "v_mfma_f32_16x16x32_f16 a[52:55], v[90:93], v[154:157], a[52:55]\n" \
  "v_mfma_f32_16x16x32_f16 a[20:23], v[50:53], v[154:157], a[20:23]\n" \
  "buffer_load_dwordx4 v[190:193], %[voffW], s[36:39], s66 offen nt\n" \
  "v_mfma_f32_16x16x32_f16 a[20:23], v[46:49], v[122:125], a[20:23]\n" \
  "v_mfma_f32_16x16x32_f16 a[24:27], v[46:49], v[126:129], a[24:27]\n" \
  "buffer_load_dwordx4 v[194:197], %[voffW], s[36:39], s67 offen nt\n" \
  "v_mfma_f32_16x16x32_f16 a[24:27], v[50:53], v[158:161], a[24:27]\n" \
  "v_mfma_f32_16x16x32_f16 a[56:59], v[90:93], v[158:161], a[56:59]\n" \
  "v_mfma_f32_16x16x32_f16 a[56:59], v[86:89], v[126:129], a[56:59]\n" \
  "v_mfma_f32_16x16x32_f16 a[60:63], v[86:89], v[130:133], a[60:63]\n" \
  "v_mfma_f32_16x16x32_f16 a[60:63], v[90:93], v[162:165], a[60:63]\n" \
  "v_mfma_f32_16x16x32_f16 a[28:31], v[50:53], v[162:165], a[28:31]\n" \
  "v_mfma_f32_16x16x32_f16 a[28:31], v[46:49], v[130:133], a[28:31]\n" \
  "v_mfma_f32_16x16x32_f16 a[128:131], v[62:65], v[102:105], a[128:131]\n" \
  "v_mfma_f32_16x16x32_f16 a[128:131], v[66:69], v[134:137], a[128:131]\n" \
  "v_mfma_f32_16x16x32_f16 a[160:163], v[26:29], v[134:137], a[160:163]\n" \
  "ds_read_b128 v[134:137], v245 offset:24576\n" \
  "v_mfma_f32_16x16x32_f16 a[160:163], v[22:25], v[102:105], a[160:163]\n" \
  "ds_read_b128 v[102:105], v245 offset:16384\n" \
  "v_mfma_f32_16x16x32_f16 a[164:167], v[22:25], v[106:109], a[164:167]\n" \
  "v_mfma_f32_16x16x32_f16 a[164:167], v[26:29], v[138:141], a[164:167]\n" \
  "v_mfma_f32_16x16x32_f16 a[132:135], v[66:69], v[138:141], a[132:135]\n" \
  "ds_read_b128 v[138:141], v245 offset:25600\n" \
  "v_mfma_f32_16x16x32_f16 a[132:135], v[62:65], v[106:109], a[132:135]\n" \
  "ds_read_b128 v[106:109], v245 offset:17408\n" \
  "v_mfma_f32_16x16x32_f16 a[136:139], v[62:65], v[110:113], a[136:139]\n" \
  "v_mfma_f32_16x16x32_f16 a[136:139], v[66:69], v[142:145], a[136:139]\n" \
  "v_mfma_f32_16x16x32_f16 a[168:171], v[26:29], v[142:145], a[168:171]\n" \
  "ds_read_b128 v[142:145], v245 offset:26624\n" \
  "v_mfma_f32_16x16x32_f16 a[168:171], v[22:25], v[110:113], a[168:171]\n" \
  "ds_read_b128 v[110:113], v245 offset:18432\n" \
  "v_mfma_f32_16x16x32_f16 a[172:175], v[22:25], v[114:117], a[172:175]\n" \
  "v_mfma_f32_16x16x32_f16 a[172:175], v[26:29], v[146:149], a[172:175]\n" \
  "v_mfma_f32_16x16x32_f16 a[140:143], v[66:69], v[146:149], a[140:143]\n" \
  "ds_read_b128 v[146:149], v245 offset:27648\n" \
  "v_mfma_f32_16x16x32_f16 a[140:143], v[62:65], v[114:117], a[140:143]\n" \
  "ds_read_b128 v[114:117], v245 offset:19456\n" \
  "v_mfma_f32_16x16x32_f16 a[144:147], v[62:65], v[118:121], a[144:147]\n" \
  "s_add_u32 s73, s74, s71\n" \
  "v_mfma_f32_16x16x32_f16 a[144:147], v[66:69], v[150:153], a[144:147]\n" \
  "v_mfma_f32_16x16x32_f16 a[176:179], v[26:29], v[150:153], a[176:179]\n" \
  "ds_read_b128 v[150:153], v245 offset:28672\n" \
  "v_mfma_f32_16x16x32_f16 a[176:179], v[22:25], v[118:121], a[176:179]\n" \
  "ds_read_b128 v[118:121], v245 offset:20480\n" \
  "v_mfma_f32_16x16x32_f16 a[180:183], v[22:25], v[122:125], a[180:183]\n" \
  "v_mfma_f32_16x16x32_f16 a[180:183], v[26:29], v[154:157], a[180:183]\n" \
  "v_mfma_f32_16x16x32_f16 a[148:151], v[66:69], v[154:157], a[148:151]\n" \
  "ds_read_b128 v[154:157], v245 offset:29696\n" \
  "v_mfma_f32_16x16x32_f16 a[148:151], v[62:65], v[122:125], a[148:151]\n" \
  "ds_read_b128 v[122:125], v245 offset:21504\n" \
  "v_mfma_f32_16x16x32_f16 a[152:155], v[62:65], v[126:129], a[152:155]\n" \
  "buffer_load_dwordx4 v[34:37], %[bxr], s[40:43], s73 offen offset:0 sc0 sc1\n" \
  "v_mfma_f32_16x16x32_f16 a[152:155], v[66:69], v[158:161], a[152:155]\n" \
  "v_mfma_f32_16x16x32_f16 a[184:187], v[26:29], v[158:161], a[184:187]\n" \
  "ds_read_b128 v[158:161], v245 offset:30720\n" \
  "v_mfma_f32_16x16x32_f16 a[184:187], v[22:25], v[126:129], a[184:187]\n" \
  "ds_read_b128 v[126:129], v245 offset:22528\n" \
  "v_mfma_f32_16x16x32_f16 a[188:191], v[22:25], v[130:133], a[188:191]\n" \
  "buffer_load_dwordx4 v[30:33], %[bxr], s[40:43], s73 offen offset:1024 sc0 sc1\n" \
  "v_mfma_f32_16x16x32_f16 a[188:191], v[26:29], v[162:165], a[188:191]\n" \
  "v_mfma_f32_16x16x32_f16 a[156:159], v[66:69], v[162:165], a[156:159]\n" \
  "ds_read_b128 v[162:165], v245 offset:31744\n" \
  "v_mfma_f32_16x16x32_f16 a[156:159], v[62:65], v[130:133], a[156:159]\n" \
  "s_waitcnt vmcnt(16)\n" \
  "ds_read_b128 v[130:133], v245 offset:23552\n" \
  "s_waitcnt lgkmcnt(14)\n" \
    \
  "v_mfma_f32_16x16x32_f16 a[64:67], v[78:81], v[102:105], a[64:67]\n" \
  "buffer_load_dwordx4 v[86:89], %[bxr], s[40:43], s73 offen offset:2048 sc0 sc1\n" \
  "v_mfma_f32_16x16x32_f16 a[64:67], v[82:85], v[134:137], a[64:67]\n" \
  "v_mfma_f32_16x16x32_f16 a[96:99], v[42:45], v[134:137], a[96:99]\n" \
  "v_mfma_f32_16x16x32_f16 a[96:99], v[38:41], v[102:105], a[96:99]\n" \
  "s_waitcnt lgkmcnt(12)\n" \
  "v_readlane_b32 s74, %[vtabA], 20\n" \
  "v_mfma_f32_16x16x32_f16 a[100:103], v[38:41], v[106:109], a[100:103]\n" \
  "buffer_load_dwordx4 v[90:93], %[bxr], s[40:43], s73 offen offset:3072 sc0 sc1\n" \
  "v_mfma_f32_16x16x32_f16 a[100:103], v[42:45], v[138:141], a[100:103]\n" \
  "s_bitcmp1_b32 s34, 20\n" \
  "s_cbranch_scc0 L_s65_%=\n" \
  "L_b65_%=:\n" \
  "v_mfma_f32_16x16x32_f16 a[68:71], v[82:85], v[138:141], a[68:71]\n" \
  "s_waitcnt lgkmcnt(10)\n" \
  "v_cvt_pk_f16_f32 v216, v198, v202\n" \
  "v_cvt_pk_f16_f32 v220, v199, v203\n" \
  "v_mfma_f32_16x16x32_f16 a[68:71], v[78:81], v[106:109], a[68:71]\n" \
  "v_cvt_pk_f16_f32 v224, v200, v204\n" \
  "v_cvt_pk_f16_f32 v228, v201, v205\n" \
  "v_mfma_f32_16x16x32_f16 a[72:75], v[78:81], v[110:113], a[72:75]\n" \
  "s_add_u32 s72, s74, s68\n" \
  "v_cvt_pk_f16_f32 v217, v206, v210\n" \
  "v_cvt_pk_f16_f32 v221, v207, v211\n" \
  "v_mfma_f32_16x16x32_f16 a[72:75], v[82:85], v[142:145], a[72:75]\n" \
  "s_waitcnt lgkmcnt(8)\n" \
  "v_cvt_pk_f16_f32 v225, v208, v212\n" \
  "v_cvt_pk_f16_f32 v229, v209, v213\n" \
  "v_mfma_f32_16x16x32_f16 a[104:107], v[42:45], v[142:145], a[104:107]\n" \
  "ds_write_b128 v244, v[214:217] offset:0\n" \
  "v_mfma_f32_16x16x32_f16 a[104:107], v[38:41], v[110:113], a[104:107]\n" \
  "buffer_load_dwordx4 v[50:53], %[bxr], s[40:43], s72 offen offset:0 sc0 sc1\n" \
  "v_mfma_f32_16x16x32_f16 a[108:111], v[38:41], v[114:117], a[108:111]\n" \
  "v_mfma_f32_16x16x32_f16 a[108:111], v[42:45], v[146:149], a[108:111]\n" \
  "s_waitcnt lgkmcnt(7)\n" \
  "v_mfma_f32_16x16x32_f16 a[76:79], v[82:85], v[146:149], a[76:79]\n" \
  "ds_write_b128 v244, v[218:221] offset:1024\n" \
  "v_mfma_f32_16x16x32_f16 a[76:79], v[78:81], v[114:117], a[76:79]\n" \
  "buffer_load_dwordx4 v[46:49], %[bxr], s[40:43], s72 offen offset:1024 sc0 sc1\n" \
  "v_mfma_f32_16x16x32_f16 a[80:83], v[78:81], v[118:121], a[80:83]\n" \
  "v_readlane_b32 s75, %[vtabW], 28\n" \
  "v_mfma_f32_16x16x32_f16 a[80:83], v[82:85], v[150:153], a[80:83]\n" \
  "s_waitcnt lgkmcnt(6)\n" \
  "v_mfma_f32_16x16x32_f16 a[112:115], v[42:45], v[150:153], a[112:115]\n" \
  "ds_write_b128 v244, v[222:225] offset:2048\n" \
  "v_mfma_f32_16x16x32_f16 a[112:115], v[38:41], v[118:121], a[112:115]\n" \
  "buffer_load_dwordx4 v[22:25], %[bxr], s[40:43], s72 offen offset:2048 sc0 sc1\n" \
  "v_mfma_f32_16x16x32_f16 a[116:119], v[38:41], v[122:125], a[116:119]\n" \
  "s_add_u32 s60, s75, 0x0\n" \
  "s_add_u32 s61, s75, 0x10000\n" \
  "s_add_u32 s62, s75, 0x20000\n" \
  "v_mfma_f32_16x16x32_f16 a[116:119], v[42:45], v[154:157], a[116:119]\n" \
  "s_waitcnt lgkmcnt(5)\n" \
  "s_add_u32 s63, s75, 0x30000\n" \
  "s_add_u32 s64, s75, 0x100000\n" \
  "v_mfma_f32_16x16x32_f16 a[84:87], v[82:85], v[154:157], a[84:87]\n" \
  "ds_write_b128 v244, v[226:229] offset:3072\n" \
  "v_mfma_f32_16x16x32_f16 a[84:87], v[78:81], v[122:125], a[84:87]\n" \
  "buffer_load_dwordx4 v[26:29], %[bxr], s[40:43], s72 offen offset:3072 sc0 sc1\n" \
  "v_mfma_f32_16x16x32_f16 a[88:91], v[78:81], v[126:129], a[88:91]\n" \
  "s_add_u32 s65, s75, 0x110000\n" \
  "s_add_u32 s73, s74, s69\n" \
  "v_mfma_f32_16x16x32_f16 a[88:91], v[82:85], v[158:161], a[88:91]\n" \
  "v_mfma_f32_16x16x32_f16 a[120:123], v[42:45], v[158:161], a[120:123]\n" \
  "v_mfma_f32_16x16x32_f16 a[120:123], v[38:41], v[126:129], a[120:123]\n" \
  "s_waitcnt lgkmcnt(4)\n" \
  "v_cvt_pk_f16_f32 v214, v166, v170\n" \
  "v_cvt_pk_f16_f32 v218, v167, v171\n" \
  "v_mfma_f32_16x16x32_f16 a[124:127], v[38:41], v[130:133], a[124:127]\n" \
  "v_cvt_pk_f16_f32 v222, v168, v172\n" \
  "v_cvt_pk_f16_f32 v226, v169, v173\n" \
  "v_mfma_f32_16x16x32_f16 a[124:127], v[42:45], v[162:165], a[124:127]\n" \
  "buffer_load_dwordx4 v[66:69], %[bxr], s[40:43], s73 offen offset:0 sc0 sc1\n" \
  "v_mfma_f32_16x16x32_f16 a[92:95], v[82:85], v[162:165], a[92:95]\n" \
  "v_mfma_f32_16x16x32_f16 a[92:95], v[78:81], v[130:133], a[92:95]\n" \
  "s_waitcnt vmcnt(19)\n" \
  "v_mfma_f32_16x16x32_f16 a[192:195], v[94:97], v[102:105], a[192:195]\n" \
  "v_mfma_f32_16x16x32_f16 a[192:195], v[98:101], v[134:137], a[192:195]\n" \
  "buffer_load_dwordx4 v[62:65], %[bxr], s[40:43], s73 offen offset:1024 sc0 sc1\n" \
  "v_mfma_f32_16x16x32_f16 a[224:227], v[58:61], v[134:137], a[224:227]\n" \
  "s_memrealtime s[58:59]\n" \
  "v_mfma_f32_16x16x32_f16 a[224:227], v[54:57], v[102:105], a[224:227]\n" \
  "v_mfma_f32_16x16x32_f16 a[228:231], v[54:57], v[106:109], a[228:231]\n" \
  "v_mfma_f32_16x16x32_f16 a[228:231], v[58:61], v[138:141], a[228:231]\n" \
  "buffer_load_dwordx4 v[38:41], %[bxr], s[40:43], s73 offen offset:2048 sc0 sc1\n" \
  "v_mfma_f32_16x16x32_f16 a[196:199], v[98:101], v[138:141], a[196:199]\n" \
  "v_mfma_f32_16x16x32_f16 a[196:199], v[94:97], v[106:109], a[196:199]\n" \
  "v_mfma_f32_16x16x32_f16 a[200:203], v[94:97], v[110:113], a[200:203]\n" \
  "s_andn2_b32 s79, 0x3ffffff, s34\n" \
  "s_cbranch_scc1 L_pl18_%=\n" \
  "L_pb18_%=:\n" \
  "v_mfma_f32_16x16x32_f16 a[200:203], v[98:101], v[142:145], a[200:203]\n" \
  "buffer_load_dwordx4 v[42:45], %[bxr], s[40:43], s73 offen offset:3072 sc0 sc1\n" \
  "v_mfma_f32_16x16x32_f16 a[232:235], v[58:61], v[142:145], a[232:235]\n" \
  "s_waitcnt lgkmcnt(0)\n" \
  "s_barrier\n" \
  "v_mfma_f32_16x16x32_f16 a[232:235], v[54:57], v[110:113], a[232:235]\n" \
  "s_sub_u32 s82, s81, s58\n" \
  "s_cmp_gt_i32 s82, 0\n" \
  "s_cbranch_scc1 L_s41_%=\n" \
  "L_b41_%=:\n" \
  "v_mfma_f32_16x16x32_f16 a[236:239], v[54:57], v[114:117], a[236:239]\n" \
  "buffer_load_dwordx4 v[198:201], %[voffW], s[36:39], s60 offen nt\n" \
  "v_mfma_f32_16x16x32_f16 a[236:239], v[58:61], v[146:149], a[236:239]\n" \
  "s_max_i32 s82, s82, -150\n" \
  "s_add_u32 s81, s58, s82\n" \
  "s_add_u32 s81, s81, s88\n" \
  "v_mfma_f32_16x16x32_f16 a[204:207], v[98:101], v[146:149], a[204:207]\n" \
  "buffer_load_dwordx4 v[202:205], %[voffW], s[36:39], s61 offen nt\n" \
  "v_mfma_f32_16x16x32_f16 a[204:207], v[94:97], v[114:117], a[204:207]\n" \
  "v_mfma_f32_16x16x32_f16 a[208:211], v[94:97], v[118:121], a[208:211]\n" \
  "buffer_load_dwordx4 v[206:209], %[voffW], s[36:39], s62 offen nt\n" \
  "v_mfma_f32_16x16x32_f16 a[208:211], v[98:101], v[150:153], a[208:211]\n" \
  "v_mfma_f32_16x16x32_f16 a[240:243], v[58:61], v[150:153], a[240:243]\n" \
  "buffer_load_dwordx4 v[210:213], %[voffW], s[36:39], s63 offen nt\n" \
  "v_mfma_f32_16x16x32_f16 a[240:243], v[54:57], v[118:121], a[240:243]\n" \
  "v_mfma_f32_16x16x32_f16 a[244:247], v[54:57], v[122:125], a[244:247]\n" \
  "buffer_load_dwordx4 v[166:169], %[voffW], s[36:39], s64 offen nt\n" \
  "v_mfma_f32_16x16x32_f16 a[244:247], v[58:61], v[154:157], a[244:247]\n" \
  "v_mfma_f32_16x16x32_f16 a[212:215], v[98:101], v[154:157], a[212:215]\n" \
  "buffer_load_dwordx4 v[170:173], %[voffW], s[36:39], s65 offen nt\n" \
  "v_mfma_f32_16x16x32_f16 a[212:215], v[94:97], v[122:125], a[212:215]\n" \
  "v_mfma_f32_16x16x32_f16 a[216:219], v[94:97], v[126:129], a[216:219]\n" \
  "v_mfma_f32_16x16x32_f16 a[216:219], v[98:101], v[158:161], a[216:219]\n" \
  "v_mfma_f32_16x16x32_f16 a[248:251], v[58:61], v[158:161], a[248:251]\n" \
  "v_mfma_f32_16x16x32_f16 a[248:251], v[54:57], v[126:129], a[248:251]\n" \
  "v_mfma_f32_16x16x32_f16 a[252:255], v[54:57], v[130:133], a[252:255]\n" \
  "v_mfma_f32_16x16x32_f16 a[252:255], v[58:61], v[162:165], a[252:255]\n" \
  "v_mfma_f32_16x16x32_f16 a[220:223], v[98:101], v[162:165], a[220:223]\n" \
  "v_mfma_f32_16x16x32_f16 a[220:223], v[94:97], v[130:133], a[220:223]\n" \
  "s_waitcnt vmcnt(24)\n" \
  "v_mfma_f32_16x16x32_f16 a[0:3], v[18:21], v[102:105], a[0:3]\n" \
  "v_mfma_f32_16x16x32_f16 a[0:3], v[14:17], v[134:137], a[0:3]\n" \
  "v_mfma_f32_16x16x32_f16 a[32:35], v[74:77], v[134:137], a[32:35]\n" \
  "v_mfma_f32_16x16x32_f16 a[32:35], v[70:73], v[102:105], a[32:35]\n" \
  "v_mfma_f32_16x16x32_f16 a[36:39], v[70:73], v[106:109], a[36:39]\n" \
  "v_mfma_f32_16x16x32_f16 a[36:39], v[74:77], v[138:141], a[36:39]\n" \
  "v_mfma_f32_16x16x32_f16 a[4:7], v[14:17], v[138:141], a[4:7]\n" \
  "v_mfma_f32_16x16x32_f16 a[4:7], v[18:21], v[106:109], a[4:7]\n" \
  "v_mfma_f32_16x16x32_f16 a[8:11], v[18:21], v[110:113], a[8:11]\n" \
  "v_mfma_f32_16x16x32_f16 a[8:11], v[14:17], v[142:145], a[8:11]\n" \
  "v_mfma_f32_16x16x32_f16 a[40:43], v[74:77], v[142:145], a[40:43]\n" \
  "v_mfma_f32_16x16x32_f16 a[40:43], v[70:73], v[110:113], a[40:43]\n" \
  "v_mfma_f32_16x16x32_f16 a[44:47], v[70:73], v[114:117], a[44:47]\n" \
  "v_mfma_f32_16x16x32_f16 a[44:47], v[74:77], v[146:149], a[44:47]\n" \
  "v_mfma_f32_16x16x32_f16 a[12:15], v[14:17], v[146:149], a[12:15]\n" \
  "s_add_u32 s72, s74, s70\n" \
  "v_mfma_f32_16x16x32_f16 a[12:15], v[18:21], v[114:117], a[12:15]\n" \
  "v_mfma_f32_16x16x32_f16 a[16:19], v[18:21], v[118:121], a[16:19]\n" \
  "v_mfma_f32_16x16x32_f16 a[16:19], v[14:17], v[150:153], a[16:19]\n" \
  "v_mfma_f32_16x16x32_f16 a[48:51], v[74:77], v[150:153], a[48:51]\n" \
  "v_mfma_f32_16x16x32_f16 a[48:51], v[70:73], v[118:121], a[48:51]\n" \
  "v_mfma_f32_16x16x32_f16 a[52:55], v[70:73], v[122:125], a[52:55]\n" \
  "v_mfma_f32_16x16x32_f16 a[52:55], v[74:77], v[154:157], a[52:55]\n" \
  "v_mfma_f32_16x16x32_f16 a[20:23], v[14:17], v[154:157], a[20:23]\n" \
  "buffer_load_dwordx4 v[82:85], %[bxr], s[40:43], s72 offen offset:0 sc0 sc1\n" \
  "v_mfma_f32_16x16x32_f16 a[20:23], v[18:21], v[122:125], a[20:23]\n" \
  "v_mfma_f32_16x16x32_f16 a[24:27], v[18:21], v[126:129], a[24:27]\n" \
  "v_mfma_f32_16x16x32_f16 a[24:27], v[14:17], v[158:161], a[24:27]\n" \
  "v_mfma_f32_16x16x32_f16 a[56:59], v[74:77], v[158:161], a[56:59]\n" \
  "buffer_load_dwordx4 v[78:81], %[bxr], s[40:43], s72 offen offset:1024 sc0 sc1\n" \
  "v_mfma_f32_16x16x32_f16 a[56:59], v[70:73], v[126:129], a[56:59]\n" \
  "v_mfma_f32_16x16x32_f16 a[60:63], v[70:73], v[130:133], a[60:63]\n" \
  "v_mfma_f32_16x16x32_f16 a[60:63], v[74:77], v[162:165], a[60:63]\n" \
  "v_mfma_f32_16x16x32_f16 a[28:31], v[14:17], v[162:165], a[28:31]\n" \
  "buffer_load_dwordx4 v[54:57], %[bxr], s[40:43], s72 offen offset:2048 sc0 sc1\n" \
  "v_mfma_f32_16x16x32_f16 a[28:31], v[18:21], v[130:133], a[28:31]\n" \
  "s_waitcnt vmcnt(17)\n" \
  "s_waitcnt vmcnt(10)\n" \
  "v_mfma_f32_16x16x32_f16 a[128:131], v[34:37], v[102:105], a[128:131]\n" \
  "v_min3_u32 v248, v238, v239, v240\n" \
  "v_min_u32 v248, v248, v241\n" \
  "v_mfma_f32_16x16x32_f16 a[128:131], v[30:33], v[134:137], a[128:131]\n" \
  "v_cmp_le_u32 s[34:35], s76, v248\n" \
  "v_mfma_f32_16x16x32_f16 a[160:163], v[90:93], v[134:137], a[160:163]\n" \
  "ds_read_b128 v[134:137], v245 offset:40960\n" \
  "v_mfma_f32_16x16x32_f16 a[160:163], v[86:89], v[102:105], a[160:163]\n" \
  "ds_read_b128 v[102:105], v245 offset:32768\n" \
  "v_mfma_f32_16x16x32_f16 a[164:167], v[86:89], v[106:109], a[164:167]\n" \
  "v_cvt_pk_f16_f32 v215, v174, v178\n" \
  "v_cvt_pk_f16_f32 v219, v175, v179\n" \
  "v_mfma_f32_16x16x32_f16 a[164:167], v[90:93], v[138:141], a[164:167]\n" \
  "v_cvt_pk_f16_f32 v223, v176, v180\n" \
  "v_cvt_pk_f16_f32 v227, v177, v181\n" \
  "v_mfma_f32_16x16x32_f16 a[132:135], v[30:33], v[138:141], a[132:135]\n" \
  "ds_read_b128 v[138:141], v245 offset:41984\n" \
  "v_cvt_pk_f16_f32 v216, v182, v186\n" \
  "v_mfma_f32_16x16x32_f16 a[132:135], v[34:37], v[106:109], a[132:135]\n" \
  "ds_read_b128 v[106:109], v245 offset:33792\n" \
  "v_cvt_pk_f16_f32 v220, v183, v187\n" \
  "v_mfma_f32_16x16x32_f16 a[136:139], v[34:37], v[110:113], a[136:139]\n" \
  "v_cvt_pk_f16_f32 v224, v184, v188\n" \
  "v_cvt_pk_f16_f32 v228, v185, v189\n" \
  "v_mfma_f32_16x16x32_f16 a[136:139], v[30:33], v[142:145], a[136:139]\n" \
  "s_add_u32 s73, s74, s71\n" \
  "v_cvt_pk_f16_f32 v217, v190, v194\n" \
  "v_cvt_pk_f16_f32 v221, v191, v195\n" \
  "v_mfma_f32_16x16x32_f16 a[168:171], v[90:93], v[142:145], a[168:171]\n" \
  "ds_read_b128 v[142:145], v245 offset:43008\n" \
  "v_cvt_pk_f16_f32 v225, v192, v196\n" \
  "v_mfma_f32_16x16x32_f16 a[168:171], v[86:89], v[110:113], a[168:171]\n" \
  "ds_read_b128 v[110:113], v245 offset:34816\n" \
  "v_cvt_pk_f16_f32 v229, v193, v197\n" \
  "v_mfma_f32_16x16x32_f16 a[172:175], v[86:89], v[114:117], a[172:175]\n" \
  "ds_write_b128 v244, v[214:217] offset:16384\n" \
  "v_mfma_f32_16x16x32_f16 a[172:175], v[90:93], v[146:149], a[172:175]\n" \
  "buffer_load_dwordx4 v[58:61], %[bxr], s[40:43], s72 offen offset:3072 sc0 sc1\n" \
  "v_mfma_f32_16x16x32_f16 a[140:143], v[30:33], v[146:149], a[140:143]\n" \
  "s_add_u32 s66, s75, 0x120000\n" \
  "ds_read_b128 v[146:149], v245 offset:44032\n" \
  "s_add_u32 s67, s75, 0x130000\n" \
  "v_mfma_f32_16x16x32_f16 a[140:143], v[34:37], v[114:117], a[140:143]\n" \
  "ds_read_b128 v[114:117], v245 offset:35840\n" \
  "v_readlane_b32 s75, %[vtabW], 29\n" \
  "v_mfma_f32_16x16x32_f16 a[144:147], v[34:37], v[118:121], a[144:147]\n" \
  "ds_write_b128 v244, v[218:221] offset:17408\n" \
  "v_mfma_f32_16x16x32_f16 a[144:147], v[30:33], v[150:153], a[144:147]\n" \
  "buffer_load_dwordx4 v[98:101], %[bxr], s[40:43], s73 offen offset:0 sc0 sc1\n" \
  "v_mfma_f32_16x16x32_f16 a[176:179], v[90:93], v[150:153], a[176:179]\n" \
  "s_add_u32 s60, s75, 0x0\n" \
  "ds_read_b128 v[150:153], v245 offset:45056\n" \
  "s_add_u32 s61, s75, 0x10000\n" \
  "v_mfma_f32_16x16x32_f16 a[176:179], v[86:89], v[118:121], a[176:179]\n" \
  "s_add_u32 s62, s75, 0x20000\n" \
  "ds_read_b128 v[118:121], v245 offset:36864\n" \
  "s_add_u32 s63, s75, 0x30000\n" \
  "v_mfma_f32_16x16x32_f16 a[180:183], v[86:89], v[122:125], a[180:183]\n" \
  "ds_write_b128 v244, v[222:225] offset:18432\n" \
  "v_mfma_f32_16x16x32_f16 a[180:183], v[90:93], v[154:157], a[180:183]\n" \
  "buffer_load_dwordx4 v[94:97], %[bxr], s[40:43], s73 offen offset:1024 sc0 sc1\n" \
  "v_mfma_f32_16x16x32_f16 a[148:151], v[30:33], v[154:157], a[148:151]\n" \
  "ds_read_b128 v[154:157], v245 offset:46080\n" \
  "v_mfma_f32_16x16x32_f16 a[148:151], v[34:37], v[122:125], a[148:151]\n" \
  "ds_read_b128 v[122:125], v245 offset:37888\n" \
  "v_mfma_f32_16x16x32_f16 a[152:155], v[34:37], v[126:129], a[152:155]\n" \
  "ds_write_b128 v244, v[226:229] offset:19456\n" \
  "v_mfma_f32_16x16x32_f16 a[152:155], v[30:33], v[158:161], a[152:155]\n" \
  "buffer_load_dwordx4 v[70:73], %[bxr], s[40:43], s73 offen offset:2048 sc0 sc1\n" \
  "v_mfma_f32_16x16x32_f16 a[184:187], v[90:93], v[158:161], a[184:187]\n" \
  "ds_read_b128 v[158:161], v245 offset:47104\n" \
  "v_readlane_b32 s74, %[vtabA], 21\n" \
  "v_mfma_f32_16x16x32_f16 a[184:187], v[86:89], v[126:129], a[184:187]\n" \
  "s_bitcmp1_b32 s34, 21\n" \
  "s_cbranch_scc0 L_s66_%=\n" \
  "L_b66_%=:\n" \
  "ds_read_b128 v[126:129], v245 offset:38912\n" \
  "v_mfma_f32_16x16x32_f16 a[188:191], v[86:89], v[130:133], a[188:191]\n" \
  "s_add_u32 s72, s74, s68\n" \
  "v_mfma_f32_16x16x32_f16 a[188:191], v[90:93], v[162:165], a[188:191]\n" \
  "buffer_load_dwordx4 v[74:77], %[bxr], s[40:43], s73 offen offset:3072 sc0 sc1\n" \
  "v_mfma_f32_16x16x32_f16 a[156:159], v[30:33], v[162:165], a[156:159]\n" \
  "ds_read_b128 v[162:165], v245 offset:48128\n" \
  "v_mfma_f32_16x16x32_f16 a[156:159], v[34:37], v[130:133], a[156:159]\n" \
  "ds_read_b128 v[130:133], v245 offset:39936\n" \
  "s_waitcnt lgkmcnt(14)\n" \
    \
  "v_mfma_f32_16x16x32_f16 a[64:67], v[50:53], v[102:105], a[64:67]\n" \
  "v_mfma_f32_16x16x32_f16 a[64:67], v[46:49], v[134:137], a[64:67]\n" \
  "buffer_load_dwordx4 v[14:17], %[bxr], s[40:43], s72 offen offset:0 sc0 sc1\n" \
  "v_mfma_f32_16x16x32_f16 a[96:99], v[26:29], v[134:137], a[96:99]\n" \
  "v_mfma_f32_16x16x32_f16 a[96:99], v[22:25], v[102:105], a[96:99]\n" \
  "s_waitcnt lgkmcnt(14)\n" \
  "v_mfma_f32_16x16x32_f16 a[100:103], v[22:25], v[106:109], a[100:103]\n" \
  "v_mfma_f32_16x16x32_f16 a[100:103], v[26:29], v[138:141], a[100:103]\n" \
  "buffer_load_dwordx4 v[18:21], %[bxr], s[40:43], s72 offen offset:1024 sc0 sc1\n" \
  "v_mfma_f32_16x16x32_f16 a[68:71], v[46:49], v[138:141], a[68:71]\n" \
  "v_mfma_f32_16x16x32_f16 a[68:71], v[50:53], v[106:109], a[68:71]\n" \
  "s_waitcnt lgkmcnt(14)\n" \
  "v_mfma_f32_16x16x32_f16 a[72:75], v[50:53], v[110:113], a[72:75]\n" \
  "v_mfma_f32_16x16x32_f16 a[72:75], v[46:49], v[142:145], a[72:75]\n" \
  "s_barrier\n" \
  "v_mfma_f32_16x16x32_f16 a[104:107], v[26:29], v[142:145], a[104:107]\n" \
  "buffer_load_dwordx4 v[174:177], %[voffW], s[36:39], s66 offen nt\n" \
  "v_mfma_f32_16x16x32_f16 a[104:107], v[22:25], v[110:113], a[104:107]\n" \
  "s_waitcnt lgkmcnt(11)\n" \
  "v_mfma_f32_16x16x32_f16 a[108:111], v[22:25], v[114:117], a[108:111]\n" \
  "buffer_load_dwordx4 v[178:181], %[voffW], s[36:39], s67 offen nt\n" \
  "v_mfma_f32_16x16x32_f16 a[108:111], v[26:29], v[146:149], a[108:111]\n" \
  "v_mfma_f32_16x16x32_f16 a[76:79], v[46:49], v[146:149], a[76:79]\n" \
  "buffer_load_dwordx4 v[182:185], %[voffW], s[36:39], s60 offen nt\n" \
  "v_mfma_f32_16x16x32_f16 a[76:79], v[50:53], v[114:117], a[76:79]\n" \
  "s_waitcnt lgkmcnt(8)\n" \
  "v_mfma_f32_16x16x32_f16 a[80:83], v[50:53], v[118:121], a[80:83]\n" \
  "buffer_load_dwordx4 v[186:189], %[voffW], s[36:39], s61 offen nt\n" \
  "v_mfma_f32_16x16x32_f16 a[80:83], v[46:49], v[150:153], a[80:83]\n" \
  "v_mfma_f32_16x16x32_f16 a[112:115], v[26:29], v[150:153], a[112:115]\n" \
  "buffer_load_dwordx4 v[190:193], %[voffW], s[36:39], s62 offen nt\n" \
  "v_mfma_f32_16x16x32_f16 a[112:115], v[22:25], v[118:121], a[112:115]\n" \
  "s_waitcnt lgkmcnt(5)\n" \
  "v_mfma_f32_16x16x32_f16 a[116:119], v[22:25], v[122:125], a[116:119]\n" \
  "buffer_load_dwordx4 v[194:197], %[voffW], s[36:39], s63 offen nt\n" \
  "v_mfma_f32_16x16x32_f16 a[116:119], v[26:29], v[154:157], a[116:119]\n" \
  "v_mfma_f32_16x16x32_f16 a[84:87], v[46:49], v[154:157], a[84:87]\n" \
  "v_mfma_f32_16x16x32_f16 a[84:87], v[50:53], v[122:125], a[84:87]\n" \
  "s_waitcnt lgkmcnt(2)\n" \
  "v_mfma_f32_16x16x32_f16 a[88:91], v[50:53], v[126:129], a[88:91]\n" \
  "v_mfma_f32_16x16x32_f16 a[88:91], v[46:49], v[158:161], a[88:91]\n" \
  "v_mfma_f32_16x16x32_f16 a[120:123], v[26:29], v[158:161], a[120:123]\n" \
  "v_mfma_f32_16x16x32_f16 a[120:123], v[22:25], v[126:129], a[120:123]\n" \
  "s_waitcnt lgkmcnt(0)\n" \
  "v_mfma_f32_16x16x32_f16 a[124:127], v[22:25], v[130:133], a[124:127]\n" \
  "v_mfma_f32_16x16x32_f16 a[124:127], v[26:29], v[162:165], a[124:127]\n" \
  "v_mfma_f32_16x16x32_f16 a[92:95], v[46:49], v[162:165], a[92:95]\n" \
  "v_mfma_f32_16x16x32_f16 a[92:95], v[50:53], v[130:133], a[92:95]\n" \
  "s_waitcnt vmcnt(22)\n" \
  "v_mfma_f32_16x16x32_f16 a[192:195], v[66:69], v[102:105], a[192:195]\n" \
  "v_mfma_f32_16x16x32_f16 a[192:195], v[62:65], v[134:137], a[192:195]\n" \
  "s_memrealtime s[58:59]\n" \
  "v_mfma_f32_16x16x32_f16 a[224:227], v[42:45], v[134:137], a[224:227]\n" \
  "v_mfma_f32_16x16x32_f16 a[224:227], v[38:41], v[102:105], a[224:227]\n" \
  "v_mfma_f32_16x16x32_f16 a[228:231], v[38:41], v[106:109], a[228:231]\n" \
  "v_mfma_f32_16x16x32_f16 a[228:231], v[42:45], v[138:141], a[228:231]\n" \
  "v_mfma_f32_16x16x32_f16 a[196:199], v[62:65], v[138:141], a[196:199]\n" \
  "v_mfma_f32_16x16x32_f16 a[196:199], v[66:69], v[106:109], a[196:199]\n" \
  "v_mfma_f32_16x16x32_f16 a[200:203], v[66:69], v[110:113], a[200:203]\n" \
  "v_mfma_f32_16x16x32_f16 a[200:203], v[62:65], v[142:145], a[200:203]\n" \
  "s_waitcnt lgkmcnt(0)\n" \
  "v_mfma_f32_16x16x32_f16 a[232:235], v[42:45], v[142:145], a[232:235]\n" \
  "s_sub_u32 s82, s81, s58\n" \
  "s_cmp_gt_i32 s82, 0\n" \
  "s_cbranch_scc1 L_s42_%=\n" \
  "L_b42_%=:\n" \
  "v_mfma_f32_16x16x32_f16 a[232:235], v[38:41], v[110:113], a[232:235]\n" \
  "s_max_i32 s82, s82, -150\n" \
  "s_add_u32 s81, s58, s82\n" \
  "s_add_u32 s81, s81, s88\n" \
  "v_mfma_f32_16x16x32_f16 a[236:239], v[38:41], v[114:117], a[236:239]\n" \
  "v_mfma_f32_16x16x32_f16 a[236:239], v[42:45], v[146:149], a[236:239]\n" \
  "v_mfma_f32_16x16x32_f16 a[204:207], v[62:65], v[146:149], a[204:207]\n" \
  "v_mfma_f32_16x16x32_f16 a[204:207], v[66:69], v[114:117], a[204:207]\n" \
  "v_mfma_f32_16x16x32_f16 a[208:211], v[66:69], v[118:121], a[208:211]\n" \
  "v_mfma_f32_16x16x32_f16 a[208:211], v[62:65], v[150:153], a[208:211]\n" \
  "v_mfma_f32_16x16x32_f16 a[240:243], v[42:45], v[150:153], a[240:243]\n" \
  "v_mfma_f32_16x16x32_f16 a[240:243], v[38:41], v[118:121], a[240:243]\n" \
  "v_mfma_f32_16x16x32_f16 a[244:247], v[38:41], v[122:125], a[244:247]\n" \
  "buffer_load_dwordx4 v[86:89], %[bxr], s[40:43], s72 offen offset:2048 sc0 sc1\n" \
  "v_mfma_f32_16x16x32_f16 a[244:247], v[42:45], v[154:157], a[244:247]\n" \
  "s_add_u32 s73, s74, s69\n" \
  "v_mfma_f32_16x16x32_f16 a[212:215], v[62:65], v[154:157], a[212:215]\n" \
  "v_mfma_f32_16x16x32_f16 a[212:215], v[66:69], v[122:125], a[212:215]\n" \
  "v_mfma_f32_16x16x32_f16 a[216:219], v[66:69], v[126:129], a[216:219]\n" \
  "buffer_load_dwordx4 v[90:93], %[bxr], s[40:43], s72 offen offset:3072 sc0 sc1\n" \
  "v_mfma_f32_16x16x32_f16 a[216:219], v[62:65], v[158:161], a[216:219]\n" \
  "v_mfma_f32_16x16x32_f16 a[248:251], v[42:45], v[158:161], a[248:251]\n" \
  "v_mfma_f32_16x16x32_f16 a[248:251], v[38:41], v[126:129], a[248:251]\n" \
  "v_mfma_f32_16x16x32_f16 a[252:255], v[38:41], v[130:133], a[252:255]\n" \
  "buffer_load_dwordx4 v[30:33], %[bxr], s[40:43], s73 offen offset:0 sc0 sc1\n" \
  "v_mfma_f32_16x16x32_f16 a[252:255], v[42:45], v[162:165], a[252:255]\n" \
  "v_mfma_f32_16x16x32_f16 a[220:223], v[62:65], v[162:165], a[220:223]\n" \
  "v_mfma_f32_16x16x32_f16 a[220:223], v[66:69], v[130:133], a[220:223]\n" \
  "s_waitcnt vmcnt(15)\n" \
  "v_mfma_f32_16x16x32_f16 a[0:3], v[82:85], v[102:105], a[0:3]\n" \
  "buffer_load_dwordx4 v[34:37], %[bxr], s[40:43], s73 offen offset:1024 sc0 sc1\n" \
  "v_mfma_f32_16x16x32_f16 a[0:3], v[78:81], v[134:137], a[0:3]\n" \
  "v_mfma_f32_16x16x32_f16 a[32:35], v[58:61], v[134:137], a[32:35]\n" \
  "v_cvt_pk_f16_f32 v214, v198, v202\n" \
  "v_cvt_pk_f16_f32 v218, v199, v203\n" \
  "v_mfma_f32_16x16x32_f16 a[32:35], v[54:57], v[102:105], a[32:35]\n" \
  "v_cvt_pk_f16_f32 v222, v200, v204\n" \
  "v_cvt_pk_f16_f32 v226, v201, v205\n" \
  "v_mfma_f32_16x16x32_f16 a[36:39], v[54:57], v[106:109], a[36:39]\n" \
  "v_cvt_pk_f16_f32 v215, v206, v210\n" \
  "v_cvt_pk_f16_f32 v219, v207, v211\n" \
  "v_mfma_f32_16x16x32_f16 a[36:39], v[58:61], v[138:141], a[36:39]\n" \
  "v_cvt_pk_f16_f32 v223, v208, v212\n" \
  "v_cvt_pk_f16_f32 v227, v209, v213\n" \
  "v_mfma_f32_16x16x32_f16 a[4:7], v[78:81], v[138:141], a[4:7]\n" \
  "v_cvt_pk_f16_f32 v216, v166, v170\n" \
  "v_cvt_pk_f16_f32 v220, v167, v171\n" \
  "v_mfma_f32_16x16x32_f16 a[4:7], v[82:85], v[106:109], a[4:7]\n" \
  "v_cvt_pk_f16_f32 v224, v168, v172\n" \
  "v_cvt_pk_f16_f32 v228, v169, v173\n" \
  "v_mfma_f32_16x16x32_f16 a[8:11], v[82:85], v[110:113], a[8:11]\n" \
  "buffer_load_dwordx4 v[22:25], %[bxr], s[40:43], s73 offen offset:2048 sc0 sc1\n" \
  "v_mfma_f32_16x16x32_f16 a[8:11], v[78:81], v[142:145], a[8:11]\n" \
  "s_add_u32 s72, s74, s70\n" \
  "v_mfma_f32_16x16x32_f16 a[40:43], v[58:61], v[142:145], a[40:43]\n" \
  "s_add_u32 s64, s75, 0x100000\n" \
  "s_add_u32 s65, s75, 0x110000\n" \
  "s_add_u32 s66, s75, 0x120000\n" \
  "v_mfma_f32_16x16x32_f16 a[40:43], v[54:57], v[110:113], a[40:43]\n" \
  "s_add_u32 s67, s75, 0x130000\n" \
  "v_mfma_f32_16x16x32_f16 a[44:47], v[54:57], v[114:117], a[44:47]\n" \
  "buffer_load_dwordx4 v[26:29], %[bxr], s[40:43], s73 offen offset:3072 sc0 sc1\n" \
  "v_mfma_f32_16x16x32_f16 a[44:47], v[58:61], v[146:149], a[44:47]\n" \
  "v_readlane_b32 s75, %[vtabW], 30\n" \
  "v_mfma_f32_16x16x32_f16 a[12:15], v[78:81], v[146:149], a[12:15]\n" \
  "v_mfma_f32_16x16x32_f16 a[12:15], v[82:85], v[114:117], a[12:15]\n" \
  "s_add_u32 s60, s75, 0x0\n" \
  "s_add_u32 s61, s75, 0x10000\n" \
  "v_mfma_f32_16x16x32_f16 a[16:19], v[82:85], v[118:121], a[16:19]\n" \
  "buffer_load_dwordx4 v[46:49], %[bxr], s[40:43], s72 offen offset:0 sc0 sc1\n" \
  "v_mfma_f32_16x16x32_f16 a[16:19], v[78:81], v[150:153], a[16:19]\n" \
  "v_mfma_f32_16x16x32_f16 a[48:51], v[58:61], v[150:153], a[48:51]\n" \
  "v_mfma_f32_16x16x32_f16 a[48:51], v[54:57], v[118:121], a[48:51]\n" \
  "v_mfma_f32_16x16x32_f16 a[52:55], v[54:57], v[122:125], a[52:55]\n" \
  "buffer_load_dwordx4 v[50:53], %[bxr], s[40:43], s72 offen offset:1024 sc0 sc1\n" \
  "v_mfma_f32_16x16x32_f16 a[52:55], v[58:61], v[154:157], a[52:55]\n" \
  "v_mfma_f32_16x16x32_f16 a[20:23], v[78:81], v[154:157], a[20:23]\n" \
  "v_mfma_f32_16x16x32_f16 a[20:23], v[82:85], v[122:125], a[20:23]\n" \
  "v_mfma_f32_16x16x32_f16 a[24:27], v[82:85], v[126:129], a[24:27]\n" \
  "buffer_load_dwordx4 v[38:41], %[bxr], s[40:43], s72 offen offset:2048 sc0 sc1\n" \
  "v_mfma_f32_16x16x32_f16 a[24:27], v[78:81], v[158:161], a[24:27]\n" \
  "s_add_u32 s73, s74, s71\n" \
  "v_mfma_f32_16x16x32_f16 a[56:59], v[58:61], v[158:161], a[56:59]\n" \
  "v_mfma_f32_16x16x32_f16 a[56:59], v[54:57], v[126:129], a[56:59]\n" \
  "v_mfma_f32_16x16x32_f16 a[60:63], v[54:57], v[130:133], a[60:63]\n" \
  "buffer_load_dwordx4 v[42:45], %[bxr], s[40:43], s72 offen offset:3072 sc0 sc1\n" \
  "v_mfma_f32_16x16x32_f16 a[60:63], v[58:61], v[162:165], a[60:63]\n" \
  "v_mfma_f32_16x16x32_f16 a[28:31], v[78:81], v[162:165], a[28:31]\n" \
  "v_mfma_f32_16x16x32_f16 a[28:31], v[82:85], v[130:133], a[28:31]\n" \
  "s_waitcnt vmcnt(18)\n" \
  "v_mfma_f32_16x16x32_f16 a[128:131], v[98:101], v[102:105], a[128:131]\n" \
  "buffer_load_dwordx4 v[62:65], %[bxr], s[40:43], s73 offen offset:0 sc0 sc1\n" \
  "v_mfma_f32_16x16x32_f16 a[128:131], v[94:97], v[134:137], a[128:131]\n" \
  "v_mfma_f32_16x16x32_f16 a[160:163], v[74:77], v[134:137], a[160:163]\n" \
  "ds_read_b128 v[134:137], v245 offset:57344\n" \
  "v_mfma_f32_16x16x32_f16 a[160:163], v[70:73], v[102:105], a[160:163]\n" \
  "ds_read_b128 v[102:105], v245 offset:49152\n" \
  "v_mfma_f32_16x16x32_f16 a[164:167], v[70:73], v[106:109], a[164:167]\n" \
  "buffer_load_dwordx4 v[66:69], %[bxr], s[40:43], s73 offen offset:1024 sc0 sc1\n" \
  "v_mfma_f32_16x16x32_f16 a[164:167], v[74:77], v[138:141], a[164:167]\n" \
  "s_andn2_b32 s79, 0xffffffff, s34\n" \
  "s_cbranch_scc1 L_pl19_%=\n" \
  "L_pb19_%=:\n" \
  "v_mfma_f32_16x16x32_f16 a[132:135], v[94:97], v[138:141], a[132:135]\n" \
  "ds_read_b128 v[138:141], v245 offset:58368\n" \
  "v_mfma_f32_16x16x32_f16 a[132:135], v[98:101], v[106:109], a[132:135]\n" \
  "s_barrier\n" \
  "ds_read_b128 v[106:109], v245 offset:50176\n" \
  "v_mfma_f32_16x16x32_f16 a[136:139], v[98:101], v[110:113], a[136:139]\n" \
  "buffer_load_dwordx4 v[198:201], %[voffW], s[36:39], s64 offen nt\n" \
  "v_mfma_f32_16x16x32_f16 a[136:139], v[94:97], v[142:145], a[136:139]\n" \
  "v_mfma_f32_16x16x32_f16 a[168:171], v[74:77], v[142:145], a[168:171]\n" \
  "ds_read_b128 v[142:145], v245 offset:59392\n" \
  "v_mfma_f32_16x16x32_f16 a[168:171], v[70:73], v[110:113], a[168:171]\n" \
  "ds_read_b128 v[110:113], v245 offset:51200\n" \
  "v_mfma_f32_16x16x32_f16 a[172:175], v[70:73], v[114:117], a[172:175]\n" \
  "buffer_load_dwordx4 v[202:205], %[voffW], s[36:39], s65 offen nt\n" \
  "v_mfma_f32_16x16x32_f16 a[172:175], v[74:77], v[146:149], a[172:175]\n" \
  "v_mfma_f32_16x16x32_f16 a[140:143], v[94:97], v[146:149], a[140:143]\n" \
  "ds_read_b128 v[146:149], v245 offset:60416\n" \
  "v_mfma_f32_16x16x32_f16 a[140:143], v[98:101], v[114:117], a[140:143]\n" \
  "ds_read_b128 v[114:117], v245 offset:52224\n" \
  "v_mfma_f32_16x16x32_f16 a[144:147], v[98:101], v[118:121], a[144:147]\n" \
  "buffer_load_dwordx4 v[206:209], %[voffW], s[36:39], s66 offen nt\n" \
  "v_mfma_f32_16x16x32_f16 a[144:147], v[94:97], v[150:153], a[144:147]\n" \
  "v_mfma_f32_16x16x32_f16 a[176:179], v[74:77], v[150:153], a[176:179]\n" \
  "ds_read_b128 v[150:153], v245 offset:61440\n" \
  "v_mfma_f32_16x16x32_f16 a[176:179], v[70:73], v[118:121], a[176:179]\n" \
  "ds_read_b128 v[118:121], v245 offset:53248\n" \
  "v_mfma_f32_16x16x32_f16 a[180:183], v[70:73], v[122:125], a[180:183]\n" \
  "buffer_load_dwordx4 v[210:213], %[voffW], s[36:39], s67 offen nt\n" \
  "v_mfma_f32_16x16x32_f16 a[180:183], v[74:77], v[154:157], a[180:183]\n" \
  "v_mfma_f32_16x16x32_f16 a[148:151], v[94:97], v[154:157], a[148:151]\n" \
  "ds_read_b128 v[154:157], v245 offset:62464\n" \
  "v_mfma_f32_16x16x32_f16 a[148:151], v[98:101], v[122:125], a[148:151]\n" \
  "ds_read_b128 v[122:125], v245 offset:54272\n" \
  "v_mfma_f32_16x16x32_f16 a[152:155], v[98:101], v[126:129], a[152:155]\n" \
  "buffer_load_dwordx4 v[166:169], %[voffW], s[36:39], s60 offen nt\n" \
  "v_mfma_f32_16x16x32_f16 a[152:155], v[94:97], v[158:161], a[152:155]\n" \
  "v_mfma_f32_16x16x32_f16 a[184:187], v[74:77], v[158:161], a[184:187]\n" \
  "ds_read_b128 v[158:161], v245 offset:63488\n" \
  "v_mfma_f32_16x16x32_f16 a[184:187], v[70:73], v[126:129], a[184:187]\n" \
  "ds_read_b128 v[126:129], v245 offset:55296\n" \
  "v_mfma_f32_16x16x32_f16 a[188:191], v[70:73], v[130:133], a[188:191]\n" \
  "buffer_load_dwordx4 v[170:173], %[voffW], s[36:39], s61 offen nt\n" \
  "v_mfma_f32_16x16x32_f16 a[188:191], v[74:77], v[162:165], a[188:191]\n" \
  "v_mfma_f32_16x16x32_f16 a[156:159], v[94:97], v[162:165], a[156:159]\n" \
  "ds_read_b128 v[162:165], v245 offset:64512\n" \
  "v_mfma_f32_16x16x32_f16 a[156:159], v[98:101], v[130:133], a[156:159]\n" \
  "s_waitcnt vmcnt(16)\n" \
  "ds_read_b128 v[130:133], v245 offset:56320\n" \
  "s_waitcnt lgkmcnt(14)\n" \
    \
  "v_mfma_f32_16x16x32_f16 a[64:67], v[14:17], v[102:105], a[64:67]\n" \
  "v_mfma_f32_16x16x32_f16 a[64:67], v[18:21], v[134:137], a[64:67]\n" \
  "v_mfma_f32_16x16x32_f16 a[96:99], v[90:93], v[134:137], a[96:99]\n" \
  "v_mfma_f32_16x16x32_f16 a[96:99], v[86:89], v[102:105], a[96:99]\n" \
  "s_waitcnt lgkmcnt(12)\n" \
  "v_mfma_f32_16x16x32_f16 a[100:103], v[86:89], v[106:109], a[100:103]\n" \
  "v_mfma_f32_16x16x32_f16 a[100:103], v[90:93], v[138:141], a[100:103]\n" \
  "v_mfma_f32_16x16x32_f16 a[68:71], v[18:21], v[138:141], a[68:71]\n" \
  "v_mfma_f32_16x16x32_f16 a[68:71], v[14:17], v[106:109], a[68:71]\n" \
  "s_waitcnt lgkmcnt(10)\n" \
  "v_mfma_f32_16x16x32_f16 a[72:75], v[14:17], v[110:113], a[72:75]\n" \
  "v_mfma_f32_16x16x32_f16 a[72:75], v[18:21], v[142:145], a[72:75]\n" \
  "v_mfma_f32_16x16x32_f16 a[104:107], v[90:93], v[142:145], a[104:107]\n" \
  "v_mfma_f32_16x16x32_f16 a[104:107], v[86:89], v[110:113], a[104:107]\n" \
  "s_waitcnt lgkmcnt(8)\n" \
  "v_mfma_f32_16x16x32_f16 a[108:111], v[86:89], v[114:117], a[108:111]\n" \
  "v_mfma_f32_16x16x32_f16 a[108:111], v[90:93], v[146:149], a[108:111]\n" \
  "v_mfma_f32_16x16x32_f16 a[76:79], v[18:21], v[146:149], a[76:79]\n" \
  "v_mfma_f32_16x16x32_f16 a[76:79], v[14:17], v[114:117], a[76:79]\n" \
  "s_waitcnt lgkmcnt(6)\n" \
  "v_mfma_f32_16x16x32_f16 a[80:83], v[14:17], v[118:121], a[80:83]\n" \
  "v_mfma_f32_16x16x32_f16 a[80:83], v[18:21], v[150:153], a[80:83]\n" \
  "v_mfma_f32_16x16x32_f16 a[112:115], v[90:93], v[150:153], a[112:115]\n" \
  "buffer_load_dwordx4 v[54:57], %[bxr], s[40:43], s73 offen offset:2048 sc0 sc1\n" \
  "v_mfma_f32_16x16x32_f16 a[112:115], v[86:89], v[118:121], a[112:115]\n" \
  "s_waitcnt lgkmcnt(4)\n" \
  "v_readlane_b32 s74, %[vtabA], 22\n" \
  "v_mfma_f32_16x16x32_f16 a[116:119], v[86:89], v[122:125], a[116:119]\n" \
  "s_bitcmp1_b32 s34, 22\n" \
  "s_cbranch_scc0 L_s67_%=\n" \
  "L_b67_%=:\n" \
  "v_mfma_f32_16x16x32_f16 a[116:119], v[90:93], v[154:157], a[116:119]\n" \
  "s_add_u32 s72, s74, s68\n" \
  "v_mfma_f32_16x16x32_f16 a[84:87], v[18:21], v[154:157], a[84:87]\n" \
  "buffer_load_dwordx4 v[58:61], %[bxr], s[40:43], s73 offen offset:3072 sc0 sc1\n" \
  "v_mfma_f32_16x16x32_f16 a[84:87], v[14:17], v[122:125], a[84:87]\n" \
  "s_waitcnt lgkmcnt(2)\n" \
  "v_mfma_f32_16x16x32_f16 a[88:91], v[14:17], v[126:129], a[88:91]\n" \
  "v_mfma_f32_16x16x32_f16 a[88:91], v[18:21], v[158:161], a[88:91]\n" \
  "v_mfma_f32_16x16x32_f16 a[120:123], v[90:93], v[158:161], a[120:123]\n" \
  "buffer_load_dwordx4 v[78:81], %[bxr], s[40:43], s72 offen offset:0 sc0 sc1\n" \
  "v_mfma_f32_16x16x32_f16 a[120:123], v[86:89], v[126:129], a[120:123]\n" \
  "s_waitcnt lgkmcnt(0)\n" \
  "v_mfma_f32_16x16x32_f16 a[124:127], v[86:89], v[130:133], a[124:127]\n" \
  "s_waitcnt vmcnt(9)\n" \
  "v_mfma_f32_16x16x32_f16 a[124:127], v[90:93], v[162:165], a[124:127]\n" \
  "v_min3_u32 v248, v238, v239, v240\n" \
  "v_min_u32 v248, v248, v241\n" \
  "v_mfma_f32_16x16x32_f16 a[92:95], v[18:21], v[162:165], a[92:95]\n" \
  "buffer_load_dwordx4 v[82:85], %[bxr], s[40:43], s72 offen offset:1024 sc0 sc1\n" \
  "v_mfma_f32_16x16x32_f16 a[92:95], v[14:17], v[130:133], a[92:95]\n" \
  "v_cmp_le_u32 s[34:35], s76, v248\n" \
  "v_mfma_f32_16x16x32_f16 a[192:195], v[30:33], v[102:105], a[192:195]\n" \
  "v_cvt_pk_f16_f32 v217, v174, v178\n" \
  "v_cvt_pk_f16_f32 v221, v175, v179\n" \
  "v_mfma_f32_16x16x32_f16 a[192:195], v[34:37], v[134:137], a[192:195]\n" \
  "s_memrealtime s[58:59]\n" \
  "v_cvt_pk_f16_f32 v225, v176, v180\n" \
  "v_cvt_pk_f16_f32 v229, v177, v181\n" \
  "v_mfma_f32_16x16x32_f16 a[224:227], v[26:29], v[134:137], a[224:227]\n" \
  "ds_write_b128 v242, v[214:217] offset:0\n" \
  "v_mfma_f32_16x16x32_f16 a[224:227], v[22:25], v[102:105], a[224:227]\n" \
  "buffer_load_dwordx4 v[70:73], %[bxr], s[40:43], s72 offen offset:2048 sc0 sc1\n" \
  "v_mfma_f32_16x16x32_f16 a[228:231], v[22:25], v[106:109], a[228:231]\n" \
  "s_add_u32 s73, s74, s69\n" \
  "v_mfma_f32_16x16x32_f16 a[228:231], v[26:29], v[138:141], a[228:231]\n" \
  "v_mfma_f32_16x16x32_f16 a[196:199], v[34:37], v[138:141], a[196:199]\n" \
  "ds_write_b128 v242, v[218:221] offset:1024\n" \
  "v_mfma_f32_16x16x32_f16 a[196:199], v[30:33], v[106:109], a[196:199]\n" \
  "buffer_load_dwordx4 v[74:77], %[bxr], s[40:43], s72 offen offset:3072 sc0 sc1\n" \
  "v_mfma_f32_16x16x32_f16 a[200:203], v[30:33], v[110:113], a[200:203]\n" \
  "s_add_u32 s62, s75, 0x20000\n" \
  "s_add_u32 s63, s75, 0x30000\n" \
  "s_add_u32 s64, s75, 0x100000\n" \
  "v_mfma_f32_16x16x32_f16 a[200:203], v[34:37], v[142:145], a[200:203]\n" \
  "s_waitcnt lgkmcnt(0)\n" \
  "s_add_u32 s65, s75, 0x110000\n" \
  "s_add_u32 s66, s75, 0x120000\n" \
  "v_mfma_f32_16x16x32_f16 a[232:235], v[26:29], v[142:145], a[232:235]\n" \
  "ds_write_b128 v242, v[222:225] offset:2048\n" \
  "v_mfma_f32_16x16x32_f16 a[232:235], v[22:25], v[110:113], a[232:235]\n" \
  "buffer_load_dwordx4 v[94:97], %[bxr], s[40:43], s73 offen offset:0 sc0 sc1\n" \
  "v_mfma_f32_16x16x32_f16 a[236:239], v[22:25], v[114:117], a[236:239]\n" \
  "s_sub_u32 s82, s81, s58\n" \
  "s_cmp_gt_i32 s82, 0\n" \
  "s_cbranch_scc1 L_s43_%=\n" \
  "L_b43_%=:\n" \
  "v_mfma_f32_16x16x32_f16 a[236:239], v[26:29], v[146:149], a[236:239]\n" \
  "s_max_i32 s82, s82, -150\n" \
  "s_add_u32 s81, s58, s82\n" \
  "s_add_u32 s81, s81, s88\n" \
  "v_mfma_f32_16x16x32_f16 a[204:207], v[34:37], v[146:149], a[204:207]\n" \
  "ds_write_b128 v242, v[226:229] offset:3072\n" \
  "v_mfma_f32_16x16x32_f16 a[204:207], v[30:33], v[114:117], a[204:207]\n" \
  "buffer_load_dwordx4 v[98:101], %[bxr], s[40:43], s73 offen offset:1024 sc0 sc1\n" \
  "v_mfma_f32_16x16x32_f16 a[208:211], v[30:33], v[118:121], a[208:211]\n" \
  "s_add_u32 s67, s75, 0x130000\n" \
  "v_mfma_f32_16x16x32_f16 a[208:211], v[34:37], v[150:153], a[208:211]\n" \
  "v_mfma_f32_16x16x32_f16 a[240:243], v[26:29], v[150:153], a[240:243]\n" \
  "v_mfma_f32_16x16x32_f16 a[240:243], v[22:25], v[118:121], a[240:243]\n" \
  "v_cvt_pk_f16_f32 v214, v182, v186\n" \
  "v_cvt_pk_f16_f32 v218, v183, v187\n" \
  "v_mfma_f32_16x16x32_f16 a[244:247], v[22:25], v[122:125], a[244:247]\n" \
  "v_cvt_pk_f16_f32 v222, v184, v188\n" \
  "v_cvt_pk_f16_f32 v226, v185, v189\n" \
  "v_mfma_f32_16x16x32_f16 a[244:247], v[26:29], v[154:157], a[244:247]\n" \
  "v_cvt_pk_f16_f32 v215, v190, v194\n" \
  "v_cvt_pk_f16_f32 v219, v191, v195\n" \
  "v_mfma_f32_16x16x32_f16 a[212:215], v[34:37], v[154:157], a[212:215]\n" \
  "v_cvt_pk_f16_f32 v223, v192, v196\n" \
  "v_cvt_pk_f16_f32 v227, v193, v197\n" \
  "v_mfma_f32_16x16x32_f16 a[212:215], v[30:33], v[122:125], a[212:215]\n" \
  "buffer_load_dwordx4 v[86:89], %[bxr], s[40:43], s73 offen offset:2048 sc0 sc1\n" \
  "v_mfma_f32_16x16x32_f16 a[216:219], v[30:33], v[126:129], a[216:219]\n" \
  "s_add_u32 s72, s74, s70\n" \
  "v_mfma_f32_16x16x32_f16 a[216:219], v[34:37], v[158:161], a[216:219]\n" \
  "v_mfma_f32_16x16x32_f16 a[248:251], v[26:29], v[158:161], a[248:251]\n" \
  "v_mfma_f32_16x16x32_f16 a[248:251], v[22:25], v[126:129], a[248:251]\n" \
  "buffer_load_dwordx4 v[90:93], %[bxr], s[40:43], s73 offen offset:3072 sc0 sc1\n" \
  "v_mfma_f32_16x16x32_f16 a[252:255], v[22:25], v[130:133], a[252:255]\n" \
  "v_mfma_f32_16x16x32_f16 a[252:255], v[26:29], v[162:165], a[252:255]\n" \
  "v_mfma_f32_16x16x32_f16 a[220:223], v[34:37], v[162:165], a[220:223]\n" \
  "v_mfma_f32_16x16x32_f16 a[220:223], v[30:33], v[130:133], a[220:223]\n" \
  "buffer_load_dwordx4 v[18:21], %[bxr], s[40:43], s72 offen offset:0 sc0 sc1\n" \
  "v_mfma_f32_16x16x32_f16 a[0:3], v[46:49], v[102:105], a[0:3]\n" \
  "v_mfma_f32_16x16x32_f16 a[0:3], v[50:53], v[134:137], a[0:3]\n" \
  "v_mfma_f32_16x16x32_f16 a[32:35], v[42:45], v[134:137], a[32:35]\n" \
  "v_mfma_f32_16x16x32_f16 a[32:35], v[38:41], v[102:105], a[32:35]\n" \
  "buffer_load_dwordx4 v[14:17], %[bxr], s[40:43], s72 offen offset:1024 sc0 sc1\n" \
  "v_mfma_f32_16x16x32_f16 a[36:39], v[38:41], v[106:109], a[36:39]\n" \
  "v_mfma_f32_16x16x32_f16 a[36:39], v[42:45], v[138:141], a[36:39]\n" \
  "s_barrier\n" \
  "v_mfma_f32_16x16x32_f16 a[4:7], v[50:53], v[138:141], a[4:7]\n" \
  "buffer_load_dwordx4 v[174:177], %[voffW], s[36:39], s62 offen nt\n" \
  "v_mfma_f32_16x16x32_f16 a[4:7], v[46:49], v[106:109], a[4:7]\n" \
  "v_mfma_f32_16x16x32_f16 a[8:11], v[46:49], v[110:113], a[8:11]\n" \
  "buffer_load_dwordx4 v[178:181], %[voffW], s[36:39], s63 offen nt\n" \
  "v_mfma_f32_16x16x32_f16 a[8:11], v[50:53], v[142:145], a[8:11]\n" \
  "v_mfma_f32_16x16x32_f16 a[40:43], v[42:45], v[142:145], a[40:43]\n" \
  "buffer_load_dwordx4 v[182:185], %[voffW], s[36:39], s64 offen nt\n" \
  "v_mfma_f32_16x16x32_f16 a[40:43], v[38:41], v[110:113], a[40:43]\n" \
  "v_mfma_f32_16x16x32_f16 a[44:47], v[38:41], v[114:117], a[44:47]\n" \
  "buffer_load_dwordx4 v[186:189], %[voffW], s[36:39], s65 offen nt\n" \
  "v_mfma_f32_16x16x32_f16 a[44:47], v[42:45], v[146:149], a[44:47]\n" \
  "v_mfma_f32_16x16x32_f16 a[12:15], v[50:53], v[146:149], a[12:15]\n" \
  "buffer_load_dwordx4 v[190:193], %[voffW], s[36:39], s66 offen nt\n" \
  "v_mfma_f32_16x16x32_f16 a[12:15], v[46:49], v[114:117], a[12:15]\n" \
  "v_mfma_f32_16x16x32_f16 a[16:19], v[46:49], v[118:121], a[16:19]\n" \
  "buffer_load_dwordx4 v[194:197], %[voffW], s[36:39], s67 offen nt\n" \
  "v_mfma_f32_16x16x32_f16 a[16:19], v[50:53], v[150:153], a[16:19]\n" \
  "v_mfma_f32_16x16x32_f16 a[48:51], v[42:45], v[150:153], a[48:51]\n" \
  "v_mfma_f32_16x16x32_f16 a[48:51], v[38:41], v[118:121], a[48:51]\n" \
  "v_mfma_f32_16x16x32_f16 a[52:55], v[38:41], v[122:125], a[52:55]\n" \
  "v_mfma_f32_16x16x32_f16 a[52:55], v[42:45], v[154:157], a[52:55]\n" \
  "v_mfma_f32_16x16x32_f16 a[20:23], v[50:53], v[154:157], a[20:23]\n" \
  "v_mfma_f32_16x16x32_f16 a[20:23], v[46:49], v[122:125], a[20:23]\n" \
  "v_mfma_f32_16x16x32_f16 a[24:27], v[46:49], v[126:129], a[24:27]\n" \
  "v_mfma_f32_16x16x32_f16 a[24:27], v[50:53], v[158:161], a[24:27]\n" \
  "v_mfma_f32_16x16x32_f16 a[56:59], v[42:45], v[158:161], a[56:59]\n" \
  "v_mfma_f32_16x16x32_f16 a[56:59], v[38:41], v[126:129], a[56:59]\n" \
  "v_mfma_f32_16x16x32_f16 a[60:63], v[38:41], v[130:133], a[60:63]\n" \
  "v_mfma_f32_16x16x32_f16 a[60:63], v[42:45], v[162:165], a[60:63]\n" \
  "v_mfma_f32_16x16x32_f16 a[28:31], v[50:53], v[162:165], a[28:31]\n" \
  "v_mfma_f32_16x16x32_f16 a[28:31], v[46:49], v[130:133], a[28:31]\n" \
  "s_waitcnt vmcnt(16)\n" \
  "v_mfma_f32_16x16x32_f16 a[128:131], v[62:65], v[102:105], a[128:131]\n" \
  "v_mfma_f32_16x16x32_f16 a[128:131], v[66:69], v[134:137], a[128:131]\n" \
  "v_mfma_f32_16x16x32_f16 a[160:163], v[58:61], v[134:137], a[160:163]\n" \
  "ds_read_b128 v[134:137], v246 offset:8192\n" \
  "v_mfma_f32_16x16x32_f16 a[160:163], v[54:57], v[102:105], a[160:163]\n" \
  "ds_read_b128 v[102:105], v246 offset:0\n" \
  "v_mfma_f32_16x16x32_f16 a[164:167], v[54:57], v[106:109], a[164:167]\n" \
  "v_mfma_f32_16x16x32_f16 a[164:167], v[58:61], v[138:141], a[164:167]\n" \
  "v_mfma_f32_16x16x32_f16 a[132:135], v[66:69], v[138:141], a[132:135]\n" \
  "ds_read_b128 v[138:141], v246 offset:9216\n" \
  "v_mfma_f32_16x16x32_f16 a[132:135], v[62:65], v[106:109], a[132:135]\n" \
  "ds_read_b128 v[106:109], v246 offset:1024\n" \
  "v_mfma_f32_16x16x32_f16 a[136:139], v[62:65], v[110:113], a[136:139]\n" \
  "v_mfma_f32_16x16x32_f16 a[136:139], v[66:69], v[142:145], a[136:139]\n" \
  "v_mfma_f32_16x16x32_f16 a[168:171], v[58:61], v[142:145], a[168:171]\n" \
  "ds_read_b128 v[142:145], v246 offset:10240\n" \
  "v_mfma_f32_16x16x32_f16 a[168:171], v[54:57], v[110:113], a[168:171]\n" \
  "ds_read_b128 v[110:113], v246 offset:2048\n" \
  "v_mfma_f32_16x16x32_f16 a[172:175], v[54:57], v[114:117], a[172:175]\n" \
  "v_mfma_f32_16x16x32_f16 a[172:175], v[58:61], v[146:149], a[172:175]\n" \
  "v_mfma_f32_16x16x32_f16 a[140:143], v[66:69], v[146:149], a[140:143]\n" \
  "ds_read_b128 v[146:149], v246 offset:11264\n" \
  "v_mfma_f32_16x16x32_f16 a[140:143], v[62:65], v[114:117], a[140:143]\n" \
  "ds_read_b128 v[114:117], v246 offset:3072\n" \
  "v_mfma_f32_16x16x32_f16 a[144:147], v[62:65], v[118:121], a[144:147]\n" \
  "buffer_load_dwordx4 v[22:25], %[bxr], s[40:43], s72 offen offset:2048 sc0 sc1\n" \
  "v_mfma_f32_16x16x32_f16 a[144:147], v[66:69], v[150:153], a[144:147]\n" \
  "s_add_u32 s73, s74, s71\n" \
  "v_mfma_f32_16x16x32_f16 a[176:179], v[58:61], v[150:153], a[176:179]\n" \
  "ds_read_b128 v[150:153], v246 offset:12288\n" \
  "v_mfma_f32_16x16x32_f16 a[176:179], v[54:57], v[118:121], a[176:179]\n" \
  "ds_read_b128 v[118:121], v246 offset:4096\n" \
  "v_mfma_f32_16x16x32_f16 a[180:183], v[54:57], v[122:125], a[180:183]\n" \
  "buffer_load_dwordx4 v[26:29], %[bxr], s[40:43], s72 offen offset:3072 sc0 sc1\n" \
  "v_mfma_f32_16x16x32_f16 a[180:183], v[58:61], v[154:157], a[180:183]\n" \
  "v_mfma_f32_16x16x32_f16 a[148:151], v[66:69], v[154:157], a[148:151]\n" \
  "ds_read_b128 v[154:157], v246 offset:13312\n" \
  "v_mfma_f32_16x16x32_f16 a[148:151], v[62:65], v[122:125], a[148:151]\n" \
  "ds_read_b128 v[122:125], v246 offset:5120\n" \
  "v_mfma_f32_16x16x32_f16 a[152:155], v[62:65], v[126:129], a[152:155]\n" \
  "buffer_load_dwordx4 v[34:37], %[bxr], s[40:43], s73 offen offset:0 sc0 sc1\n" \
  "v_mfma_f32_16x16x32_f16 a[152:155], v[66:69], v[158:161], a[152:155]\n" \
  "v_mfma_f32_16x16x32_f16 a[184:187], v[58:61], v[158:161], a[184:187]\n" \
  "ds_read_b128 v[158:161], v246 offset:14336\n" \
  "v_mfma_f32_16x16x32_f16 a[184:187], v[54:57], v[126:129], a[184:187]\n" \
  "ds_read_b128 v[126:129], v246 offset:6144\n" \
  "v_readlane_b32 s74, %[vtabA], 23\n" \
  "v_mfma_f32_16x16x32_f16 a[188:191], v[54:57], v[130:133], a[188:191]\n" \
  "buffer_load_dwordx4 v[30:33], %[bxr], s[40:43], s73 offen offset:1024 sc0 sc1\n" \
  "v_mfma_f32_16x16x32_f16 a[188:191], v[58:61], v[162:165], a[188:191]\n" \
  "s_bitcmp1_b32 s34, 23\n" \
  "s_cbranch_scc0 L_s68_%=\n" \
  "L_b68_%=:\n" \
  "v_mfma_f32_16x16x32_f16 a[156:159], v[66:69], v[162:165], a[156:159]\n" \
  "s_waitcnt lgkmcnt(12)\n" \
  "ds_read_b128 v[162:165], v246 offset:15360\n" \
  "v_cvt_pk_f16_f32 v216, v198, v202\n" \
  "v_mfma_f32_16x16x32_f16 a[156:159], v[62:65], v[130:133], a[156:159]\n" \
  "s_waitcnt vmcnt(16)\n" \
  "ds_read_b128 v[130:133], v246 offset:7168\n" \
  "v_cvt_pk_f16_f32 v220, v199, v203\n" \
    \
  "v_mfma_f32_16x16x32_f16 a[64:67], v[78:81], v[102:105], a[64:67]\n" \
  "v_cvt_pk_f16_f32 v224, v200, v204\n" \
  "v_cvt_pk_f16_f32 v228, v201, v205\n" \
  "v_mfma_f32_16x16x32_f16 a[64:67], v[82:85], v[134:137], a[64:67]\n" \
  "s_xor_b32 s68, s68, s92\n" \
  "v_cvt_pk_f16_f32 v217, v206, v210\n" \
  "v_cvt_pk_f16_f32 v221, v207, v211\n" \
  "v_mfma_f32_16x16x32_f16 a[96:99], v[74:77], v[134:137], a[96:99]\n" \
  "s_waitcnt lgkmcnt(12)\n" \
  "v_cvt_pk_f16_f32 v225, v208, v212\n" \
  "v_cvt_pk_f16_f32 v229, v209, v213\n" \
  "v_mfma_f32_16x16x32_f16 a[96:99], v[70:73], v[102:105], a[96:99]\n" \
  "ds_write_b128 v242, v[214:217] offset:16384\n" \
  "v_mfma_f32_16x16x32_f16 a[100:103], v[70:73], v[106:109], a[100:103]\n" \
  "buffer_load_dwordx4 v[38:41], %[bxr], s[40:43], s73 offen offset:2048 sc0 sc1\n" \
  "v_mfma_f32_16x16x32_f16 a[100:103], v[74:77], v[138:141], a[100:103]\n" \
  "s_xor_b32 s69, s69, s92\n" \
  "s_xor_b32 s70, s70, s92\n" \
  "s_xor_b32 s71, s71, s92\n" \
  "v_mfma_f32_16x16x32_f16 a[68:71], v[82:85], v[138:141], a[68:71]\n" \
  "s_waitcnt lgkmcnt(11)\n" \
  "v_readlane_b32 s75, %[vtabW], 31\n" \
  "s_add_u32 s72, s74, s68\n" \
  "v_mfma_f32_16x16x32_f16 a[68:71], v[78:81], v[106:109], a[68:71]\n" \
  "ds_write_b128 v242, v[218:221] offset:17408\n" \
  "v_mfma_f32_16x16x32_f16 a[72:75], v[78:81], v[110:113], a[72:75]\n" \
  "buffer_load_dwordx4 v[42:45], %[bxr], s[40:43], s73 offen offset:3072 sc0 sc1\n" \
  "v_mfma_f32_16x16x32_f16 a[72:75], v[82:85], v[142:145], a[72:75]\n" \
  "s_add_u32 s60, s75, 0x0\n" \
  "s_add_u32 s61, s75, 0x10000\n" \
  "s_add_u32 s62, s75, 0x20000\n" \
  "v_mfma_f32_16x16x32_f16 a[104:107], v[74:77], v[142:145], a[104:107]\n" \
  "s_waitcnt lgkmcnt(10)\n" \
  "s_add_u32 s63, s75, 0x30000\n" \
  "s_add_u32 s64, s75, 0x100000\n" \
  "v_mfma_f32_16x16x32_f16 a[104:107], v[70:73], v[110:113], a[104:107]\n" \
  "ds_write_b128 v242, v[222:225] offset:18432\n" \
  "v_mfma_f32_16x16x32_f16 a[108:111], v[70:73], v[114:117], a[108:111]\n" \
  "buffer_load_dwordx4 v[50:53], %[bxr], s[40:43], s72 offen offset:0 sc0 sc1\n" \
  "v_mfma_f32_16x16x32_f16 a[108:111], v[74:77], v[146:149], a[108:111]\n" \
  "s_add_u32 s65, s75, 0x110000\n" \
  "v_mfma_f32_16x16x32_f16 a[76:79], v[82:85], v[146:149], a[76:79]\n" \
  "s_waitcnt lgkmcnt(9)\n" \
  "v_mfma_f32_16x16x32_f16 a[76:79], v[78:81], v[114:117], a[76:79]\n" \
  "ds_write_b128 v242, v[226:229] offset:19456\n" \
  "v_mfma_f32_16x16x32_f16 a[80:83], v[78:81], v[118:121], a[80:83]\n" \
  "buffer_load_dwordx4 v[46:49], %[bxr], s[40:43], s72 offen offset:1024 sc0 sc1\n" \
  "v_mfma_f32_16x16x32_f16 a[80:83], v[82:85], v[150:153], a[80:83]\n" \
  "v_mfma_f32_16x16x32_f16 a[112:115], v[74:77], v[150:153], a[112:115]\n" \
  "v_mfma_f32_16x16x32_f16 a[112:115], v[70:73], v[118:121], a[112:115]\n" \
  "s_waitcnt lgkmcnt(8)\n" \
  "v_mfma_f32_16x16x32_f16 a[116:119], v[70:73], v[122:125], a[116:119]\n" \
  "v_cvt_pk_f16_f32 v214, v166, v170\n" \
  "v_cvt_pk_f16_f32 v218, v167, v171\n" \
  "v_mfma_f32_16x16x32_f16 a[116:119], v[74:77], v[154:157], a[116:119]\n" \
  "v_cvt_pk_f16_f32 v222, v168, v172\n" \
  "v_cvt_pk_f16_f32 v226, v169, v173\n" \
  "v_mfma_f32_16x16x32_f16 a[84:87], v[82:85], v[154:157], a[84:87]\n" \
  "buffer_load_dwordx4 v[54:57], %[bxr], s[40:43], s72 offen offset:2048 sc0 sc1\n" \
  "v_mfma_f32_16x16x32_f16 a[84:87], v[78:81], v[122:125], a[84:87]\n" \
  "s_waitcnt lgkmcnt(6)\n" \
  "v_readlane_b32 s74, %[vtabA], 24\n" \
  "v_mfma_f32_16x16x32_f16 a[88:91], v[78:81], v[126:129], a[88:91]\n" \
  "s_bitcmp1_b32 s34, 24\n" \
  "s_cbranch_scc0 L_s69_%=\n" \
  "L_b69_%=:\n" \
  "v_mfma_f32_16x16x32_f16 a[88:91], v[82:85], v[158:161], a[88:91]\n" \
  "s_add_u32 s73, s74, s68\n" \
  "v_mfma_f32_16x16x32_f16 a[120:123], v[74:77], v[158:161], a[120:123]\n" \
  "buffer_load_dwordx4 v[58:61], %[bxr], s[40:43], s72 offen offset:3072 sc0 sc1\n" \
  "v_mfma_f32_16x16x32_f16 a[120:123], v[70:73], v[126:129], a[120:123]\n" \
  "s_waitcnt lgkmcnt(4)\n" \
  "v_mfma_f32_16x16x32_f16 a[124:127], v[70:73], v[130:133], a[124:127]\n" \
  "v_mfma_f32_16x16x32_f16 a[124:127], v[74:77], v[162:165], a[124:127]\n" \
  "v_mfma_f32_16x16x32_f16 a[92:95], v[82:85], v[162:165], a[92:95]\n" \
  "buffer_load_dwordx4 v[66:69], %[bxr], s[40:43], s73 offen offset:0 sc0 sc1\n" \
  "v_mfma_f32_16x16x32_f16 a[92:95], v[78:81], v[130:133], a[92:95]\n" \
  "s_waitcnt vmcnt(19)\n" \
  "v_mfma_f32_16x16x32_f16 a[192:195], v[94:97], v[102:105], a[192:195]\n" \
  "v_mfma_f32_16x16x32_f16 a[192:195], v[98:101], v[134:137], a[192:195]\n" \
  "s_andn2_b32 s79, 0xffffffff, s34\n" \
  "s_cbranch_scc1 L_pl20_%=\n" \
  "L_pb20_%=:\n" \
  "v_mfma_f32_16x16x32_f16 a[224:227], v[90:93], v[134:137], a[224:227]\n" \
  "buffer_load_dwordx4 v[62:65], %[bxr], s[40:43], s73 offen offset:1024 sc0 sc1\n" \
  "v_mfma_f32_16x16x32_f16 a[224:227], v[86:89], v[102:105], a[224:227]\n" \
  "s_memrealtime s[58:59]\n" \
  "s_barrier\n" \
  "v_mfma_f32_16x16x32_f16 a[228:231], v[86:89], v[106:109], a[228:231]\n" \
  "buffer_load_dwordx4 v[198:201], %[voffW], s[36:39], s60 offen nt\n" \
  "v_mfma_f32_16x16x32_f16 a[228:231], v[90:93], v[138:141], a[228:231]\n" \
  "v_mfma_f32_16x16x32_f16 a[196:199], v[98:101], v[138:141], a[196:199]\n" \
  "buffer_load_dwordx4 v[202:205], %[voffW], s[36:39], s61 offen nt\n" \
  "v_mfma_f32_16x16x32_f16 a[196:199], v[94:97], v[106:109], a[196:199]\n" \
  "v_mfma_f32_16x16x32_f16 a[200:203], v[94:97], v[110:113], a[200:203]\n" \
  "buffer_load_dwordx4 v[206:209], %[voffW], s[36:39], s62 offen nt\n" \
  "v_mfma_f32_16x16x32_f16 a[200:203], v[98:101], v[142:145], a[200:203]\n" \
  "v_mfma_f32_16x16x32_f16 a[232:235], v[90:93], v[142:145], a[232:235]\n" \
  "buffer_load_dwordx4 v[210:213], %[voffW], s[36:39], s63 offen nt\n" \
  "v_mfma_f32_16x16x32_f16 a[232:235], v[86:89], v[110:113], a[232:235]\n" \
  "s_waitcnt lgkmcnt(0)\n" \
  "v_mfma_f32_16x16x32_f16 a[236:239], v[86:89], v[114:117], a[236:239]\n" \
  "buffer_load_dwordx4 v[166:169], %[voffW], s[36:39], s64 offen nt\n" \
  "v_mfma_f32_16x16x32_f16 a[236:239], v[90:93], v[146:149], a[236:239]\n" \
  "s_sub_u32 s82, s81, s58\n" \
  "s_cmp_gt_i32 s82, 0\n" \
  "s_cbranch_scc1 L_s44_%=\n" \
  "L_b44_%=:\n" \
  "v_mfma_f32_16x16x32_f16 a[204:207], v[98:101], v[146:149], a[204:207]\n" \
  "buffer_load_dwordx4 v[170:173], %[voffW], s[36:39], s65 offen nt\n" \
  "v_mfma_f32_16x16x32_f16 a[204:207], v[94:97], v[114:117], a[204:207]\n" \
  "s_max_i32 s82, s82, -150\n" \
  "s_add_u32 s81, s58, s82\n" \
  "s_add_u32 s81, s81, s88\n" \
  "v_mfma_f32_16x16x32_f16 a[208:211], v[94:97], v[118:121], a[208:211]\n" \
  "v_mfma_f32_16x16x32_f16 a[208:211], v[98:101], v[150:153], a[208:211]\n" \
  "v_mfma_f32_16x16x32_f16 a[240:243], v[90:93], v[150:153], a[240:243]\n" \
  "v_mfma_f32_16x16x32_f16 a[240:243], v[86:89], v[118:121], a[240:243]\n" \
  "v_mfma_f32_16x16x32_f16 a[244:247], v[86:89], v[122:125], a[244:247]\n" \
  "v_mfma_f32_16x16x32_f16 a[244:247], v[90:93], v[154:157], a[244:247]\n" \
  "v_mfma_f32_16x16x32_f16 a[212:215], v[98:101], v[154:157], a[212:215]\n" \
  "v_mfma_f32_16x16x32_f16 a[212:215], v[94:97], v[122:125], a[212:215]\n" \
  "v_mfma_f32_16x16x32_f16 a[216:219], v[94:97], v[126:129], a[216:219]\n" \
  "v_mfma_f32_16x16x32_f16 a[216:219], v[98:101], v[158:161], a[216:219]\n" \
  "v_mfma_f32_16x16x32_f16 a[248:251], v[90:93], v[158:161], a[248:251]\n" \
  "v_mfma_f32_16x16x32_f16 a[248:251], v[86:89], v[126:129], a[248:251]\n" \
  "v_mfma_f32_16x16x32_f16 a[252:255], v[86:89], v[130:133], a[252:255]\n" \
  "v_mfma_f32_16x16x32_f16 a[252:255], v[90:93], v[162:165], a[252:255]\n" \
  "v_mfma_f32_16x16x32_f16 a[220:223], v[98:101], v[162:165], a[220:223]\n" \
  "v_mfma_f32_16x16x32_f16 a[220:223], v[94:97], v[130:133], a[220:223]\n" \
  "s_waitcnt vmcnt(16)\n" \
  "v_mfma_f32_16x16x32_f16 a[0:3], v[18:21], v[102:105], a[0:3]\n" \
  "v_mfma_f32_16x16x32_f16 a[0:3], v[14:17], v[134:137], a[0:3]\n" \
  "v_mfma_f32_16x16x32_f16 a[32:35], v[26:29], v[134:137], a[32:35]\n" \
  "v_mfma_f32_16x16x32_f16 a[32:35], v[22:25], v[102:105], a[32:35]\n" \
  "v_mfma_f32_16x16x32_f16 a[36:39], v[22:25], v[106:109], a[36:39]\n" \
  "v_mfma_f32_16x16x32_f16 a[36:39], v[26:29], v[138:141], a[36:39]\n" \
  "v_mfma_f32_16x16x32_f16 a[4:7], v[14:17], v[138:141], a[4:7]\n" \
  "v_mfma_f32_16x16x32_f16 a[4:7], v[18:21], v[106:109], a[4:7]\n" \
  "v_mfma_f32_16x16x32_f16 a[8:11], v[18:21], v[110:113], a[8:11]\n" \
  "v_mfma_f32_16x16x32_f16 a[8:11], v[14:17], v[142:145], a[8:11]\n" \
  "v_mfma_f32_16x16x32_f16 a[40:43], v[26:29], v[142:145], a[40:43]\n" \
  "v_mfma_f32_16x16x32_f16 a[40:43], v[22:25], v[110:113], a[40:43]\n" \
  "v_mfma_f32_16x16x32_f16 a[44:47], v[22:25], v[114:117], a[44:47]\n" \
  "v_mfma_f32_16x16x32_f16 a[44:47], v[26:29], v[146:149], a[44:47]\n" \
  "v_mfma_f32_16x16x32_f16 a[12:15], v[14:17], v[146:149], a[12:15]\n" \
  "buffer_load_dwordx4 v[70:73], %[bxr], s[40:43], s73 offen offset:2048 sc0 sc1\n" \
  "v_mfma_f32_16x16x32_f16 a[12:15], v[18:21], v[114:117], a[12:15]\n" \
  "v_readlane_b32 s74, %[vtabA], 25\n" \
  "v_mfma_f32_16x16x32_f16 a[16:19], v[18:21], v[118:121], a[16:19]\n" \
  "s_bitcmp1_b32 s34, 25\n" \
  "s_cbranch_scc0 L_s70_%=\n" \
  "L_b70_%=:\n" \
  "v_mfma_f32_16x16x32_f16 a[16:19], v[14:17], v[150:153], a[16:19]\n" \
  "s_add_u32 s72, s74, s68\n" \
  "v_mfma_f32_16x16x32_f16 a[48:51], v[26:29], v[150:153], a[48:51]\n" \
  "buffer_load_dwordx4 v[74:77], %[bxr], s[40:43], s73 offen offset:3072 sc0 sc1\n" \
  "v_mfma_f32_16x16x32_f16 a[48:51], v[22:25], v[118:121], a[48:51]\n" \
  "v_mfma_f32_16x16x32_f16 a[52:55], v[22:25], v[122:125], a[52:55]\n" \
  "v_mfma_f32_16x16x32_f16 a[52:55], v[26:29], v[154:157], a[52:55]\n" \
  "v_mfma_f32_16x16x32_f16 a[20:23], v[14:17], v[154:157], a[20:23]\n" \
  "buffer_load_dwordx4 v[82:85], %[bxr], s[40:43], s72 offen offset:0 sc0 sc1\n" \
  "v_mfma_f32_16x16x32_f16 a[20:23], v[18:21], v[122:125], a[20:23]\n" \
  "v_mfma_f32_16x16x32_f16 a[24:27], v[18:21], v[126:129], a[24:27]\n" \
  "s_waitcnt vmcnt(10)\n" \
  "v_mfma_f32_16x16x32_f16 a[24:27], v[14:17], v[158:161], a[24:27]\n" \
  "v_min3_u32 v248, v238, v239, v240\n" \
  "v_min_u32 v248, v248, v241\n" \
  "v_mfma_f32_16x16x32_f16 a[56:59], v[26:29], v[158:161], a[56:59]\n" \
  "buffer_load_dwordx4 v[78:81], %[bxr], s[40:43], s72 offen offset:1024 sc0 sc1\n" \
  "v_mfma_f32_16x16x32_f16 a[56:59], v[22:25], v[126:129], a[56:59]\n" \
  "v_cmp_le_u32 s[34:35], s76, v248\n" \
  "v_mfma_f32_16x16x32_f16 a[60:63], v[22:25], v[130:133], a[60:63]\n" \
  "v_cvt_pk_f16_f32 v215, v174, v178\n" \
  "v_cvt_pk_f16_f32 v219, v175, v179\n" \
  "v_mfma_f32_16x16x32_f16 a[60:63], v[26:29], v[162:165], a[60:63]\n" \
  "v_cvt_pk_f16_f32 v223, v176, v180\n" \
  "v_cvt_pk_f16_f32 v227, v177, v181\n" \
  "v_mfma_f32_16x16x32_f16 a[28:31], v[14:17], v[162:165], a[28:31]\n" \
  "v_cvt_pk_f16_f32 v216, v182, v186\n" \
  "v_cvt_pk_f16_f32 v220, v183, v187\n" \
  "v_mfma_f32_16x16x32_f16 a[28:31], v[18:21], v[130:133], a[28:31]\n" \
  "v_cvt_pk_f16_f32 v224, v184, v188\n" \
  "v_cvt_pk_f16_f32 v228, v185, v189\n" \
  "v_mfma_f32_16x16x32_f16 a[128:131], v[34:37], v[102:105], a[128:131]\n" \
  "v_cvt_pk_f16_f32 v217, v190, v194\n" \
  "v_cvt_pk_f16_f32 v221, v191, v195\n" \
  "v_mfma_f32_16x16x32_f16 a[128:131], v[30:33], v[134:137], a[128:131]\n" \
  "v_cvt_pk_f16_f32 v225, v192, v196\n" \
  "v_cvt_pk_f16_f32 v229, v193, v197\n" \
  "v_mfma_f32_16x16x32_f16 a[160:163], v[42:45], v[134:137], a[160:163]\n" \
  "ds_read_b128 v[134:137], v246 offset:24576\n" \
  "v_mfma_f32_16x16x32_f16 a[160:163], v[38:41], v[102:105], a[160:163]\n" \
  "ds_read_b128 v[102:105], v246 offset:16384\n" \
  "v_mfma_f32_16x16x32_f16 a[164:167], v[38:41], v[106:109], a[164:167]\n" \
  "ds_write_b128 v242, v[214:217] offset:32768\n" \
  "v_mfma_f32_16x16x32_f16 a[164:167], v[42:45], v[138:141], a[164:167]\n" \
  "buffer_load_dwordx4 v[86:89], %[bxr], s[40:43], s72 offen offset:2048 sc0 sc1\n" \
  "v_mfma_f32_16x16x32_f16 a[132:135], v[30:33], v[138:141], a[132:135]\n" \
  "s_add_u32 s66, s75, 0x120000\n" \
  "ds_read_b128 v[138:141], v246 offset:25600\n" \
  "s_add_u32 s67, s75, 0x130000\n" \
  "v_mfma_f32_16x16x32_f16 a[132:135], v[34:37], v[106:109], a[132:135]\n" \
  "ds_read_b128 v[106:109], v246 offset:17408\n" \
  "v_readlane_b32 s74, %[vtabA], 26\n" \
  "v_mfma_f32_16x16x32_f16 a[136:139], v[34:37], v[110:113], a[136:139]\n" \
  "ds_write_b128 v242, v[218:221] offset:33792\n" \
  "v_mfma_f32_16x16x32_f16 a[136:139], v[30:33], v[142:145], a[136:139]\n" \
  "buffer_load_dwordx4 v[90:93], %[bxr], s[40:43], s72 offen offset:3072 sc0 sc1\n" \
  "v_mfma_f32_16x16x32_f16 a[168:171], v[42:45], v[142:145], a[168:171]\n" \
  "s_bitcmp1_b32 s34, 26\n" \
  "s_cbranch_scc0 L_s71_%=\n" \
  "L_b71_%=:\n" \
  "ds_read_b128 v[142:145], v246 offset:26624\n" \
  "v_mfma_f32_16x16x32_f16 a[168:171], v[38:41], v[110:113], a[168:171]\n" \
  "s_add_u32 s73, s74, s68\n" \
  "ds_read_b128 v[110:113], v246 offset:18432\n" \
  "v_mfma_f32_16x16x32_f16 a[172:175], v[38:41], v[114:117], a[172:175]\n" \
  "ds_write_b128 v242, v[222:225] offset:34816\n" \
  "v_mfma_f32_16x16x32_f16 a[172:175], v[42:45], v[146:149], a[172:175]\n" \
  "buffer_load_dwordx4 v[98:101], %[bxr], s[40:43], s73 offen offset:0 sc0 sc1\n" \
  "v_mfma_f32_16x16x32_f16 a[140:143], v[30:33], v[146:149], a[140:143]\n" \
  "ds_read_b128 v[146:149], v246 offset:27648\n" \
  "v_mfma_f32_16x16x32_f16 a[140:143], v[34:37], v[114:117], a[140:143]\n" \
  "ds_read_b128 v[114:117], v246 offset:19456\n" \
  "v_mfma_f32_16x16x32_f16 a[144:147], v[34:37], v[118:121], a[144:147]\n" \
  "ds_write_b128 v242, v[226:229] offset:35840\n" \
  "v_mfma_f32_16x16x32_f16 a[144:147], v[30:33], v[150:153], a[144:147]\n" \
  "buffer_load_dwordx4 v[94:97], %[bxr], s[40:43], s73 offen offset:1024 sc0 sc1\n" \
  "v_mfma_f32_16x16x32_f16 a[176:179], v[42:45], v[150:153], a[176:179]\n" \
  "ds_read_b128 v[150:153], v246 offset:28672\n" \
  "v_mfma_f32_16x16x32_f16 a[176:179], v[38:41], v[118:121], a[176:179]\n" \
  "ds_read_b128 v[118:121], v246 offset:20480\n" \
  "v_mfma_f32_16x16x32_f16 a[180:183], v[38:41], v[122:125], a[180:183]\n" \
  "v_mfma_f32_16x16x32_f16 a[180:183], v[42:45], v[154:157], a[180:183]\n" \
  "buffer_load_dwordx4 v[22:25], %[bxr], s[40:43], s73 offen offset:2048 sc0 sc1\n" \
  "v_mfma_f32_16x16x32_f16 a[148:151], v[30:33], v[154:157], a[148:151]\n" \
  "ds_read_b128 v[154:157], v246 offset:29696\n" \
  "v_readlane_b32 s74, %[vtabA], 27\n" \
  "v_mfma_f32_16x16x32_f16 a[148:151], v[34:37], v[122:125], a[148:151]\n" \
  "s_bitcmp1_b32 s34, 27\n" \
  "s_cbranch_scc0 L_s72_%=\n" \
  "L_b72_%=:\n" \
  "ds_read_b128 v[122:125], v246 offset:21504\n" \
  "v_mfma_f32_16x16x32_f16 a[152:155], v[34:37], v[126:129], a[152:155]\n" \
  "s_add_u32 s72, s74, s68\n" \
  "v_mfma_f32_16x16x32_f16 a[152:155], v[30:33], v[158:161], a[152:155]\n" \
  "buffer_load_dwordx4 v[26:29], %[bxr], s[40:43], s73 offen offset:3072 sc0 sc1\n" \
  "v_mfma_f32_16x16x32_f16 a[184:187], v[42:45], v[158:161], a[184:187]\n" \
  "ds_read_b128 v[158:161], v246 offset:30720\n" \
  "v_mfma_f32_16x16x32_f16 a[184:187], v[38:41], v[126:129], a[184:187]\n" \
  "ds_read_b128 v[126:129], v246 offset:22528\n" \
  "v_mfma_f32_16x16x32_f16 a[188:191], v[38:41], v[130:133], a[188:191]\n" \
  "v_mfma_f32_16x16x32_f16 a[188:191], v[42:45], v[162:165], a[188:191]\n" \
  "buffer_load_dwordx4 v[14:17], %[bxr], s[40:43], s72 offen offset:0 sc0 sc1\n" \
  "v_mfma_f32_16x16x32_f16 a[156:159], v[30:33], v[162:165], a[156:159]\n" \
  "ds_read_b128 v[162:165], v246 offset:31744\n" \
  "v_mfma_f32_16x16x32_f16 a[156:159], v[34:37], v[130:133], a[156:159]\n" \
  "ds_read_b128 v[130:133], v246 offset:23552\n" \
  "s_waitcnt lgkmcnt(14)\n" \
  "s_cmp_lg_u32 s92, 0\n" \
  "s_cbranch_scc1 L_tail1_%=\n" \
    \
  "v_mfma_f32_16x16x32_f16 a[64:67], v[50:53], v[102:105], a[64:67]\n" \
  "v_mfma_f32_16x16x32_f16 a[64:67], v[46:49], v[134:137], a[64:67]\n" \
  "s_nop 7\n" \
  "v_mfma_f32_16x16x32_f16 a[96:99], v[58:61], v[134:137], a[96:99]\n" \
  "ds_read_b128 v[134:137], v246 offset:40960\n" \
  "v_mfma_f32_16x16x32_f16 a[96:99], v[54:57], v[102:105], a[96:99]\n" \
  "s_waitcnt lgkmcnt(14)\n" \
  "ds_read_b128 v[102:105], v246 offset:32768\n" \
  "s_barrier\n" \
  "v_mfma_f32_16x16x32_f16 a[100:103], v[54:57], v[106:109], a[100:103]\n" \
  "buffer_load_dwordx4 v[174:177], %[voffW], s[36:39], s66 offen nt\n" \
  "v_mfma_f32_16x16x32_f16 a[100:103], v[58:61], v[138:141], a[100:103]\n" \
  "v_mfma_f32_16x16x32_f16 a[68:71], v[46:49], v[138:141], a[68:71]\n" \
  "ds_read_b128 v[138:141], v246 offset:41984\n" \
  "v_mfma_f32_16x16x32_f16 a[68:71], v[50:53], v[106:109], a[68:71]\n" \
  "s_waitcnt lgkmcnt(14)\n" \
  "ds_read_b128 v[106:109], v246 offset:33792\n" \
  "v_mfma_f32_16x16x32_f16 a[72:75], v[50:53], v[110:113], a[72:75]\n" \
  "buffer_load_dwordx4 v[178:181], %[voffW], s[36:39], s67 offen nt\n" \
  "v_mfma_f32_16x16x32_f16 a[72:75], v[46:49], v[142:145], a[72:75]\n" \
  "v_mfma_f32_16x16x32_f16 a[104:107], v[58:61], v[142:145], a[104:107]\n" \
  "ds_read_b128 v[142:145], v246 offset:43008\n" \
  "v_mfma_f32_16x16x32_f16 a[104:107], v[54:57], v[110:113], a[104:107]\n" \
  "s_waitcnt lgkmcnt(14)\n" \
  "ds_read_b128 v[110:113], v246 offset:34816\n" \
  "v_mfma_f32_16x16x32_f16 a[108:111], v[54:57], v[114:117], a[108:111]\n" \
  "v_mfma_f32_16x16x32_f16 a[108:111], v[58:61], v[146:149], a[108:111]\n" \
  "v_mfma_f32_16x16x32_f16 a[76:79], v[46:49], v[146:149], a[76:79]\n" \
  "ds_read_b128 v[146:149], v246 offset:44032\n" \
  "v_mfma_f32_16x16x32_f16 a[76:79], v[50:53], v[114:117], a[76:79]\n" \
  "s_waitcnt lgkmcnt(13)\n" \
  "ds_read_b128 v[114:117], v246 offset:35840\n" \
  "v_mfma_f32_16x16x32_f16 a[80:83], v[50:53], v[118:121], a[80:83]\n" \
  "v_mfma_f32_16x16x32_f16 a[80:83], v[46:49], v[150:153], a[80:83]\n" \
  "s_nop 7\n" \
  "v_mfma_f32_16x16x32_f16 a[112:115], v[58:61], v[150:153], a[112:115]\n" \
  "ds_read_b128 v[150:153], v246 offset:45056\n" \
  "v_mfma_f32_16x16x32_f16 a[112:115], v[54:57], v[118:121], a[112:115]\n" \
  "s_waitcnt lgkmcnt(13)\n" \
  "ds_read_b128 v[118:121], v246 offset:36864\n" \
  "v_mfma_f32_16x16x32_f16 a[116:119], v[54:57], v[122:125], a[116:119]\n" \
  "v_mfma_f32_16x16x32_f16 a[116:119], v[58:61], v[154:157], a[116:119]\n" \
  "v_mfma_f32_16x16x32_f16 a[84:87], v[46:49], v[154:157], a[84:87]\n" \
  "ds_read_b128 v[154:157], v246 offset:46080\n" \
  "v_mfma_f32_16x16x32_f16 a[84:87], v[50:53], v[122:125], a[84:87]\n" \
  "s_waitcnt lgkmcnt(13)\n" \
  "ds_read_b128 v[122:125], v246 offset:37888\n" \
  "v_mfma_f32_16x16x32_f16 a[88:91], v[50:53], v[126:129], a[88:91]\n" \
  "v_mfma_f32_16x16x32_f16 a[88:91], v[46:49], v[158:161], a[88:91]\n" \
  "v_mfma_f32_16x16x32_f16 a[120:123], v[58:61], v[158:161], a[120:123]\n" \
  "ds_read_b128 v[158:161], v246 offset:47104\n" \
  "v_mfma_f32_16x16x32_f16 a[120:123], v[54:57], v[126:129], a[120:123]\n" \
  "s_waitcnt lgkmcnt(13)\n" \
  "ds_read_b128 v[126:129], v246 offset:38912\n" \
  "v_mfma_f32_16x16x32_f16 a[124:127], v[54:57], v[130:133], a[124:127]\n" \
  "v_mfma_f32_16x16x32_f16 a[124:127], v[58:61], v[162:165], a[124:127]\n" \
  "v_mfma_f32_16x16x32_f16 a[92:95], v[46:49], v[162:165], a[92:95]\n" \
  "ds_read_b128 v[162:165], v246 offset:48128\n" \
  "v_mfma_f32_16x16x32_f16 a[92:95], v[50:53], v[130:133], a[92:95]\n" \
  "s_waitcnt vmcnt(11)\n" \
  "ds_read_b128 v[130:133], v246 offset:39936\n" \
  "s_waitcnt lgkmcnt(14)\n" \
  "v_mfma_f32_16x16x32_f16 a[64:67], v[66:69], v[102:105], a[64:67]\n" \
  "v_mfma_f32_16x16x32_f16 a[64:67], v[62:65], v[134:137], a[64:67]\n" \
  "s_nop 7\n" \
  "v_mfma_f32_16x16x32_f16 a[96:99], v[74:77], v[134:137], a[96:99]\n" \
  "ds_read_b128 v[134:137], v246 offset:57344\n" \
  "v_mfma_f32_16x16x32_f16 a[96:99], v[70:73], v[102:105], a[96:99]\n" \
  "s_waitcnt lgkmcnt(13)\n" \
  "ds_read_b128 v[102:105], v246 offset:49152\n" \
  "v_mfma_f32_16x16x32_f16 a[100:103], v[70:73], v[106:109], a[100:103]\n" \
  "v_mfma_f32_16x16x32_f16 a[100:103], v[74:77], v[138:141], a[100:103]\n" \
  "v_mfma_f32_16x16x32_f16 a[68:71], v[62:65], v[138:141], a[68:71]\n" \
  "ds_read_b128 v[138:141], v246 offset:58368\n" \
  "v_mfma_f32_16x16x32_f16 a[68:71], v[66:69], v[106:109], a[68:71]\n" \
  "s_waitcnt lgkmcnt(13)\n" \
  "ds_read_b128 v[106:109], v246 offset:50176\n" \
  "v_mfma_f32_16x16x32_f16 a[72:75], v[66:69], v[110:113], a[72:75]\n" \
  "s_andn2_b32 s79, 0xffffffff, s34\n" \
  "s_cbranch_scc1 L_pl21_%=\n" \
  "L_pb21_%=:\n" \
  "v_mfma_f32_16x16x32_f16 a[72:75], v[62:65], v[142:145], a[72:75]\n" \
  "v_mfma_f32_16x16x32_f16 a[104:107], v[74:77], v[142:145], a[104:107]\n" \
  "ds_read_b128 v[142:145], v246 offset:59392\n" \
  "v_mfma_f32_16x16x32_f16 a[104:107], v[70:73], v[110:113], a[104:107]\n" \
  "s_waitcnt lgkmcnt(13)\n" \
  "ds_read_b128 v[110:113], v246 offset:51200\n" \
  "v_mfma_f32_16x16x32_f16 a[108:111], v[70:73], v[114:117], a[108:111]\n" \
  "buffer_load_dwordx4 v[18:21], %[bxr], s[40:43], s72 offen offset:1024 sc0 sc1\n" \
  "v_mfma_f32_16x16x32_f16 a[108:111], v[74:77], v[146:149], a[108:111]\n" \
  "v_mfma_f32_16x16x32_f16 a[76:79], v[62:65], v[146:149], a[76:79]\n" \
  "ds_read_b128 v[146:149], v246 offset:60416\n" \
  "v_mfma_f32_16x16x32_f16 a[76:79], v[66:69], v[114:117], a[76:79]\n" \
  "s_waitcnt lgkmcnt(13)\n" \
  "ds_read_b128 v[114:117], v246 offset:52224\n" \
  "v_mfma_f32_16x16x32_f16 a[80:83], v[66:69], v[118:121], a[80:83]\n" \
  "buffer_load_dwordx4 v[38:41], %[bxr], s[40:43], s72 offen offset:2048 sc0 sc1\n" \
  "v_mfma_f32_16x16x32_f16 a[80:83], v[62:65], v[150:153], a[80:83]\n" \
  "v_readlane_b32 s74, %[vtabA], 28\n" \
  "s_nop 7\n" \
  "v_mfma_f32_16x16x32_f16 a[112:115], v[74:77], v[150:153], a[112:115]\n" \
  "s_bitcmp1_b32 s34, 28\n" \
  "s_cbranch_scc0 L_s73_%=\n" \
  "L_b73_%=:\n" \
  "ds_read_b128 v[150:153], v246 offset:61440\n" \
  "v_mfma_f32_16x16x32_f16 a[112:115], v[70:73], v[118:121], a[112:115]\n" \
  "s_waitcnt lgkmcnt(13)\n" \
  "ds_read_b128 v[118:121], v246 offset:53248\n" \
  "s_add_u32 s73, s74, s68\n" \
  "v_mfma_f32_16x16x32_f16 a[116:119], v[70:73], v[122:125], a[116:119]\n" \
  "buffer_load_dwordx4 v[42:45], %[bxr], s[40:43], s72 offen offset:3072 sc0 sc1\n" \
  "v_mfma_f32_16x16x32_f16 a[116:119], v[74:77], v[154:157], a[116:119]\n" \
  "v_mfma_f32_16x16x32_f16 a[84:87], v[62:65], v[154:157], a[84:87]\n" \
  "ds_read_b128 v[154:157], v246 offset:62464\n" \
  "v_mfma_f32_16x16x32_f16 a[84:87], v[66:69], v[122:125], a[84:87]\n" \
  "s_waitcnt lgkmcnt(13)\n" \
  "ds_read_b128 v[122:125], v246 offset:54272\n" \
  "v_mfma_f32_16x16x32_f16 a[88:91], v[66:69], v[126:129], a[88:91]\n" \
  "buffer_load_dwordx4 v[30:33], %[bxr], s[40:43], s73 offen offset:0 sc0 sc1\n" \
  "v_mfma_f32_16x16x32_f16 a[88:91], v[62:65], v[158:161], a[88:91]\n" \
  "v_mfma_f32_16x16x32_f16 a[120:123], v[74:77], v[158:161], a[120:123]\n" \
  "ds_read_b128 v[158:161], v246 offset:63488\n" \
  "v_cvt_pk_f16_f32 v214, v198, v202\n" \
  "v_mfma_f32_16x16x32_f16 a[120:123], v[70:73], v[126:129], a[120:123]\n" \
  "s_waitcnt lgkmcnt(13)\n" \
  "ds_read_b128 v[126:129], v246 offset:55296\n" \
  "v_cvt_pk_f16_f32 v218, v199, v203\n" \
  "v_mfma_f32_16x16x32_f16 a[124:127], v[70:73], v[130:133], a[124:127]\n" \
  "v_cvt_pk_f16_f32 v222, v200, v204\n" \
  "v_cvt_pk_f16_f32 v226, v201, v205\n" \
  "v_mfma_f32_16x16x32_f16 a[124:127], v[74:77], v[162:165], a[124:127]\n" \
  "s_waitcnt lgkmcnt(12)\n" \
  "v_cvt_pk_f16_f32 v215, v206, v210\n" \
  "v_cvt_pk_f16_f32 v219, v207, v211\n" \
  "v_mfma_f32_16x16x32_f16 a[92:95], v[62:65], v[162:165], a[92:95]\n" \
  "s_waitcnt vmcnt(11)\n" \
  "ds_read_b128 v[162:165], v246 offset:64512\n" \
  "v_cvt_pk_f16_f32 v223, v208, v212\n" \
  "v_mfma_f32_16x16x32_f16 a[92:95], v[66:69], v[130:133], a[92:95]\n" \
  "ds_read_b128 v[130:133], v246 offset:56320\n" \
  "v_cvt_pk_f16_f32 v227, v209, v213\n" \
  "v_mfma_f32_16x16x32_f16 a[64:67], v[82:85], v[102:105], a[64:67]\n" \
  "v_cvt_pk_f16_f32 v216, v166, v170\n" \
  "v_cvt_pk_f16_f32 v220, v167, v171\n" \
  "v_mfma_f32_16x16x32_f16 a[64:67], v[78:81], v[134:137], a[64:67]\n" \
  "v_cvt_pk_f16_f32 v224, v168, v172\n" \
  "v_cvt_pk_f16_f32 v228, v169, v173\n" \
  "s_nop 7\n" \
  "v_mfma_f32_16x16x32_f16 a[96:99], v[90:93], v[134:137], a[96:99]\n" \
  "ds_read_b128 v[134:137], v247 offset:8192\n" \
  "v_mfma_f32_16x16x32_f16 a[96:99], v[86:89], v[102:105], a[96:99]\n" \
  "s_waitcnt lgkmcnt(13)\n" \
  "ds_read_b128 v[102:105], v247 offset:0\n" \
  "v_mfma_f32_16x16x32_f16 a[100:103], v[86:89], v[106:109], a[100:103]\n" \
  "buffer_load_dwordx4 v[34:37], %[bxr], s[40:43], s73 offen offset:1024 sc0 sc1\n" \
  "v_mfma_f32_16x16x32_f16 a[100:103], v[90:93], v[138:141], a[100:103]\n" \
  "v_mfma_f32_16x16x32_f16 a[68:71], v[78:81], v[138:141], a[68:71]\n" \
  "ds_read_b128 v[138:141], v247 offset:9216\n" \
  "v_mfma_f32_16x16x32_f16 a[68:71], v[82:85], v[106:109], a[68:71]\n" \
  "s_waitcnt lgkmcnt(13)\n" \
  "ds_read_b128 v[106:109], v247 offset:1024\n" \
  "v_mfma_f32_16x16x32_f16 a[72:75], v[82:85], v[110:113], a[72:75]\n" \
  "buffer_load_dwordx4 v[54:57], %[bxr], s[40:43], s73 offen offset:2048 sc0 sc1\n" \
  "v_mfma_f32_16x16x32_f16 a[72:75], v[78:81], v[142:145], a[72:75]\n" \
  "v_readlane_b32 s74, %[vtabA], 29\n" \
  "v_mfma_f32_16x16x32_f16 a[104:107], v[90:93], v[142:145], a[104:107]\n" \
  "s_bitcmp1_b32 s34, 29\n" \
  "s_cbranch_scc0 L_s74_%=\n" \
  "L_b74_%=:\n" \
  "ds_read_b128 v[142:145], v247 offset:10240\n" \
  "v_mfma_f32_16x16x32_f16 a[104:107], v[86:89], v[110:113], a[104:107]\n" \
  "s_waitcnt lgkmcnt(13)\n" \
  "ds_read_b128 v[110:113], v247 offset:2048\n" \
  "s_add_u32 s72, s74, s68\n" \
  "v_mfma_f32_16x16x32_f16 a[108:111], v[86:89], v[114:117], a[108:111]\n" \
  "buffer_load_dwordx4 v[58:61], %[bxr], s[40:43], s73 offen offset:3072 sc0 sc1\n" \
  "v_mfma_f32_16x16x32_f16 a[108:111], v[90:93], v[146:149], a[108:111]\n" \
  "v_mfma_f32_16x16x32_f16 a[76:79], v[78:81], v[146:149], a[76:79]\n" \
  "ds_read_b128 v[146:149], v247 offset:11264\n" \
  "v_mfma_f32_16x16x32_f16 a[76:79], v[82:85], v[114:117], a[76:79]\n" \
  "s_waitcnt lgkmcnt(13)\n" \
  "ds_read_b128 v[114:117], v247 offset:3072\n" \
  "v_mfma_f32_16x16x32_f16 a[80:83], v[82:85], v[118:121], a[80:83]\n" \
  "buffer_load_dwordx4 v[46:49], %[bxr], s[40:43], s72 offen offset:0 sc0 sc1\n" \
  "v_mfma_f32_16x16x32_f16 a[80:83], v[78:81], v[150:153], a[80:83]\n" \
  "s_nop 7\n" \
  "v_mfma_f32_16x16x32_f16 a[112:115], v[90:93], v[150:153], a[112:115]\n" \
  "ds_read_b128 v[150:153], v247 offset:12288\n" \
  "v_mfma_f32_16x16x32_f16 a[112:115], v[86:89], v[118:121], a[112:115]\n" \
  "s_waitcnt lgkmcnt(13)\n" \
  "ds_read_b128 v[118:121], v247 offset:4096\n" \
  "v_mfma_f32_16x16x32_f16 a[116:119], v[86:89], v[122:125], a[116:119]\n" \
  "buffer_load_dwordx4 v[50:53], %[bxr], s[40:43], s72 offen offset:1024 sc0 sc1\n" \
  "v_mfma_f32_16x16x32_f16 a[116:119], v[90:93], v[154:157], a[116:119]\n" \
  "v_mfma_f32_16x16x32_f16 a[84:87], v[78:81], v[154:157], a[84:87]\n" \
  "ds_read_b128 v[154:157], v247 offset:13312\n" \
  "v_mfma_f32_16x16x32_f16 a[84:87], v[82:85], v[122:125], a[84:87]\n" \
  "s_waitcnt lgkmcnt(13)\n" \
  "ds_read_b128 v[122:125], v247 offset:5120\n" \
  "v_mfma_f32_16x16x32_f16 a[88:91], v[82:85], v[126:129], a[88:91]\n" \
  "buffer_load_dwordx4 v[70:73], %[bxr], s[40:43], s72 offen offset:2048 sc0 sc1\n" \
  "v_mfma_f32_16x16x32_f16 a[88:91], v[78:81], v[158:161], a[88:91]\n" \
  "v_readlane_b32 s74, %[vtabA], 30\n" \
  "v_mfma_f32_16x16x32_f16 a[120:123], v[90:93], v[158:161], a[120:123]\n" \
  "s_bitcmp1_b32 s34, 30\n" \
  "s_cbranch_scc0 L_s75_%=\n" \
  "L_b75_%=:\n" \
  "ds_read_b128 v[158:161], v247 offset:14336\n" \
  "v_mfma_f32_16x16x32_f16 a[120:123], v[86:89], v[126:129], a[120:123]\n" \
  "s_waitcnt lgkmcnt(13)\n" \
  "ds_read_b128 v[126:129], v247 offset:6144\n" \
  "s_add_u32 s73, s74, s68\n" \
  "v_mfma_f32_16x16x32_f16 a[124:127], v[86:89], v[130:133], a[124:127]\n" \
  "buffer_load_dwordx4 v[74:77], %[bxr], s[40:43], s72 offen offset:3072 sc0 sc1\n" \
  "v_mfma_f32_16x16x32_f16 a[124:127], v[90:93], v[162:165], a[124:127]\n" \
  "v_mfma_f32_16x16x32_f16 a[92:95], v[78:81], v[162:165], a[92:95]\n" \
  "s_waitcnt vmcnt(11)\n" \
  "ds_read_b128 v[162:165], v247 offset:15360\n" \
  "v_mfma_f32_16x16x32_f16 a[92:95], v[82:85], v[130:133], a[92:95]\n" \
  "ds_read_b128 v[130:133], v247 offset:7168\n" \
  "s_waitcnt lgkmcnt(14)\n" \
  "v_mfma_f32_16x16x32_f16 a[64:67], v[98:101], v[102:105], a[64:67]\n" \
  "buffer_load_dwordx4 v[62:65], %[bxr], s[40:43], s73 offen offset:0 sc0 sc1\n" \
  "v_mfma_f32_16x16x32_f16 a[64:67], v[94:97], v[134:137], a[64:67]\n" \
  "v_min3_u32 v248, v238, v239, v240\n" \
  "v_min_u32 v248, v248, v241\n" \
  "s_nop 7\n" \
  "v_mfma_f32_16x16x32_f16 a[96:99], v[26:29], v[134:137], a[96:99]\n" \
  "ds_read_b128 v[134:137], v247 offset:24576\n" \
  "v_cmp_le_u32 s[34:35], s76, v248\n" \
  "v_mfma_f32_16x16x32_f16 a[96:99], v[22:25], v[102:105], a[96:99]\n" \
  "s_waitcnt lgkmcnt(13)\n" \
  "ds_read_b128 v[102:105], v247 offset:16384\n" \
  "v_mfma_f32_16x16x32_f16 a[100:103], v[22:25], v[106:109], a[100:103]\n" \
  "buffer_load_dwordx4 v[66:69], %[bxr], s[40:43], s73 offen offset:1024 sc0 sc1\n" \
  "v_mfma_f32_16x16x32_f16 a[100:103], v[26:29], v[138:141], a[100:103]\n" \
  "v_mfma_f32_16x16x32_f16 a[68:71], v[94:97], v[138:141], a[68:71]\n" \
  "ds_read_b128 v[138:141], v247 offset:25600\n" \
  "v_mfma_f32_16x16x32_f16 a[68:71], v[98:101], v[106:109], a[68:71]\n" \
  "s_waitcnt lgkmcnt(13)\n" \
  "ds_read_b128 v[106:109], v247 offset:17408\n" \
  "v_mfma_f32_16x16x32_f16 a[72:75], v[98:101], v[110:113], a[72:75]\n" \
  "buffer_load_dwordx4 v[86:89], %[bxr], s[40:43], s73 offen offset:2048 sc0 sc1\n" \
  "v_mfma_f32_16x16x32_f16 a[72:75], v[94:97], v[142:145], a[72:75]\n" \
  "v_readlane_b32 s74, %[vtabA], 31\n" \
  "v_mfma_f32_16x16x32_f16 a[104:107], v[26:29], v[142:145], a[104:107]\n" \
  "s_bitcmp1_b32 s34, 31\n" \
  "s_cbranch_scc0 L_s76_%=\n" \
  "L_b76_%=:\n" \
  "ds_read_b128 v[142:145], v247 offset:26624\n" \
  "v_mfma_f32_16x16x32_f16 a[104:107], v[22:25], v[110:113], a[104:107]\n" \
  "s_waitcnt lgkmcnt(13)\n" \
  "ds_read_b128 v[110:113], v247 offset:18432\n" \
  "s_add_u32 s72, s74, s68\n" \
  "v_mfma_f32_16x16x32_f16 a[108:111], v[22:25], v[114:117], a[108:111]\n" \
  "buffer_load_dwordx4 v[90:93], %[bxr], s[40:43], s73 offen offset:3072 sc0 sc1\n" \
  "v_mfma_f32_16x16x32_f16 a[108:111], v[26:29], v[146:149], a[108:111]\n" \
  "v_mfma_f32_16x16x32_f16 a[76:79], v[94:97], v[146:149], a[76:79]\n" \
  "ds_read_b128 v[146:149], v247 offset:27648\n" \
  "v_mfma_f32_16x16x32_f16 a[76:79], v[98:101], v[114:117], a[76:79]\n" \
  "s_waitcnt lgkmcnt(13)\n" \
  "ds_read_b128 v[114:117], v247 offset:19456\n" \
  "v_mfma_f32_16x16x32_f16 a[80:83], v[98:101], v[118:121], a[80:83]\n" \
  "buffer_load_dwordx4 v[78:81], %[bxr], s[40:43], s72 offen offset:0 sc0 sc1\n" \
  "v_mfma_f32_16x16x32_f16 a[80:83], v[94:97], v[150:153], a[80:83]\n" \
  "s_nop 7\n" \
  "v_mfma_f32_16x16x32_f16 a[112:115], v[26:29], v[150:153], a[112:115]\n" \
  "ds_read_b128 v[150:153], v247 offset:28672\n" \
  "v_mfma_f32_16x16x32_f16 a[112:115], v[22:25], v[118:121], a[112:115]\n" \
  "s_waitcnt lgkmcnt(13)\n" \
  "ds_read_b128 v[118:121], v247 offset:20480\n" \
  "v_mfma_f32_16x16x32_f16 a[116:119], v[22:25], v[122:125], a[116:119]\n" \
  "buffer_load_dwordx4 v[82:85], %[bxr], s[40:43], s72 offen offset:1024 sc0 sc1\n" \
  "v_mfma_f32_16x16x32_f16 a[116:119], v[26:29], v[154:157], a[116:119]\n" \
  "v_mfma_f32_16x16x32_f16 a[84:87], v[94:97], v[154:157], a[84:87]\n" \
  "ds_read_b128 v[154:157], v247 offset:29696\n" \
  "v_mfma_f32_16x16x32_f16 a[84:87], v[98:101], v[122:125], a[84:87]\n" \
  "s_waitcnt lgkmcnt(13)\n" \
  "ds_read_b128 v[122:125], v247 offset:21504\n" \
  "v_mfma_f32_16x16x32_f16 a[88:91], v[98:101], v[126:129], a[88:91]\n" \
  "v_mfma_f32_16x16x32_f16 a[88:91], v[94:97], v[158:161], a[88:91]\n" \
  "v_mfma_f32_16x16x32_f16 a[120:123], v[26:29], v[158:161], a[120:123]\n" \
  "ds_read_b128 v[158:161], v247 offset:30720\n" \
  "v_mfma_f32_16x16x32_f16 a[120:123], v[22:25], v[126:129], a[120:123]\n" \
  "s_waitcnt lgkmcnt(13)\n" \
  "ds_read_b128 v[126:129], v247 offset:22528\n" \
  "v_mfma_f32_16x16x32_f16 a[124:127], v[22:25], v[130:133], a[124:127]\n" \
  "buffer_load_dwordx4 v[22:25], %[bxr], s[40:43], s72 offen offset:2048 sc0 sc1\n" \
  "v_mfma_f32_16x16x32_f16 a[124:127], v[26:29], v[162:165], a[124:127]\n" \
  "v_readlane_b32 s74, %[vtabA], 23\n" \
  "v_mfma_f32_16x16x32_f16 a[92:95], v[94:97], v[162:165], a[92:95]\n" \
  "ds_read_b128 v[162:165], v247 offset:31744\n" \
  "v_mfma_f32_16x16x32_f16 a[92:95], v[98:101], v[130:133], a[92:95]\n" \
  "s_waitcnt vmcnt(15)\n" \
  "ds_read_b128 v[130:133], v247 offset:23552\n" \
  "s_waitcnt lgkmcnt(14)\n" \
  "v_mfma_f32_16x16x32_f16 a[64:67], v[14:17], v[102:105], a[64:67]\n" \
  "buffer_load_dwordx4 v[26:29], %[bxr], s[40:43], s72 offen offset:3072 sc0 sc1\n" \
  "v_mfma_f32_16x16x32_f16 a[64:67], v[18:21], v[134:137], a[64:67]\n" \
  "s_add_u32 s73, s74, s69\n" \
  "s_nop 7\n" \
  "v_mfma_f32_16x16x32_f16 a[96:99], v[42:45], v[134:137], a[96:99]\n" \
  "ds_read_b128 v[134:137], v245 offset:8192\n" \
  "v_mfma_f32_16x16x32_f16 a[96:99], v[38:41], v[102:105], a[96:99]\n" \
  "s_waitcnt lgkmcnt(13)\n" \
  "ds_read_b128 v[102:105], v245 offset:0\n" \
  "v_mfma_f32_16x16x32_f16 a[100:103], v[38:41], v[106:109], a[100:103]\n" \
  "buffer_load_dwordx4 v[94:97], %[bxr], s[40:43], s73 offen offset:0 sc0 sc1\n" \
  "v_mfma_f32_16x16x32_f16 a[100:103], v[42:45], v[138:141], a[100:103]\n" \
  "v_mfma_f32_16x16x32_f16 a[68:71], v[18:21], v[138:141], a[68:71]\n" \
  "ds_read_b128 v[138:141], v245 offset:9216\n" \
  "v_mfma_f32_16x16x32_f16 a[68:71], v[14:17], v[106:109], a[68:71]\n" \
  "s_waitcnt lgkmcnt(13)\n" \
  "ds_read_b128 v[106:109], v245 offset:1024\n" \
  "v_mfma_f32_16x16x32_f16 a[72:75], v[14:17], v[110:113], a[72:75]\n" \
  "buffer_load_dwordx4 v[98:101], %[bxr], s[40:43], s73 offen offset:1024 sc0 sc1\n" \
  "v_mfma_f32_16x16x32_f16 a[72:75], v[18:21], v[142:145], a[72:75]\n" \
  "v_mfma_f32_16x16x32_f16 a[104:107], v[42:45], v[142:145], a[104:107]\n" \
  "ds_read_b128 v[142:145], v245 offset:10240\n" \
  "v_mfma_f32_16x16x32_f16 a[104:107], v[38:41], v[110:113], a[104:107]\n" \
  "s_waitcnt lgkmcnt(13)\n" \
  "ds_read_b128 v[110:113], v245 offset:2048\n" \
  "v_mfma_f32_16x16x32_f16 a[108:111], v[38:41], v[114:117], a[108:111]\n" \
  "v_mfma_f32_16x16x32_f16 a[108:111], v[42:45], v[146:149], a[108:111]\n" \
  "v_mfma_f32_16x16x32_f16 a[76:79], v[18:21], v[146:149], a[76:79]\n" \
  "ds_read_b128 v[146:149], v245 offset:11264\n" \
  "v_mfma_f32_16x16x32_f16 a[76:79], v[14:17], v[114:117], a[76:79]\n" \
  "s_waitcnt lgkmcnt(13)\n" \
  "ds_read_b128 v[114:117], v245 offset:3072\n" \
  "v_mfma_f32_16x16x32_f16 a[80:83], v[14:17], v[118:121], a[80:83]\n" \
  "v_mfma_f32_16x16x32_f16 a[80:83], v[18:21], v[150:153], a[80:83]\n" \
  "s_nop 7\n" \
  "v_mfma_f32_16x16x32_f16 a[112:115], v[42:45], v[150:153], a[112:115]\n" \
  "ds_read_b128 v[150:153], v245 offset:12288\n" \
  "v_mfma_f32_16x16x32_f16 a[112:115], v[38:41], v[118:121], a[112:115]\n" \
  "s_waitcnt lgkmcnt(13)\n" \
  "ds_read_b128 v[118:121], v245 offset:4096\n" \
  "v_mfma_f32_16x16x32_f16 a[116:119], v[38:41], v[122:125], a[116:119]\n" \
  "v_mfma_f32_16x16x32_f16 a[116:119], v[42:45], v[154:157], a[116:119]\n" \
  "v_mfma_f32_16x16x32_f16 a[84:87], v[18:21], v[154:157], a[84:87]\n" \
  "ds_read_b128 v[154:157], v245 offset:13312\n" \
  "v_mfma_f32_16x16x32_f16 a[84:87], v[14:17], v[122:125], a[84:87]\n" \
  "ds_read_b128 v[122:125], v245 offset:5120\n" \
  "s_waitcnt lgkmcnt(14)\n" \
  "v_mfma_f32_16x16x32_f16 a[88:91], v[14:17], v[126:129], a[88:91]\n" \
  "v_cvt_pk_f16_f32 v217, v174, v178\n" \
  "v_cvt_pk_f16_f32 v221, v175, v179\n" \
  "v_mfma_f32_16x16x32_f16 a[88:91], v[18:21], v[158:161], a[88:91]\n" \
  "v_cvt_pk_f16_f32 v225, v176, v180\n" \
  "v_cvt_pk_f16_f32 v229, v177, v181\n" \
  "v_mfma_f32_16x16x32_f16 a[120:123], v[42:45], v[158:161], a[120:123]\n" \
  "ds_read_b128 v[158:161], v245 offset:14336\n" \
  "v_mfma_f32_16x16x32_f16 a[120:123], v[38:41], v[126:129], a[120:123]\n" \
  "s_waitcnt lgkmcnt(13)\n" \
  "ds_read_b128 v[126:129], v245 offset:6144\n" \
  "v_mfma_f32_16x16x32_f16 a[124:127], v[38:41], v[130:133], a[124:127]\n" \
  "ds_write_b128 v242, v[214:217] offset:49152\n" \
  "v_mfma_f32_16x16x32_f16 a[124:127], v[42:45], v[162:165], a[124:127]\n" \
  "buffer_load_dwordx4 v[38:41], %[bxr], s[40:43], s73 offen offset:2048 sc0 sc1\n" \
  "v_mfma_f32_16x16x32_f16 a[92:95], v[18:21], v[162:165], a[92:95]\n" \
  "ds_read_b128 v[162:165], v245 offset:15360\n" \
  "v_readlane_b32 s74, %[vtabA], 24\n" \
  "v_mfma_f32_16x16x32_f16 a[92:95], v[14:17], v[130:133], a[92:95]\n" \
  "s_waitcnt vmcnt(15)\n" \
  "ds_read_b128 v[130:133], v245 offset:7168\n" \
  "s_waitcnt lgkmcnt(14)\n" \
  "v_mfma_f32_16x16x32_f16 a[64:67], v[30:33], v[102:105], a[64:67]\n" \
  "ds_write_b128 v242, v[218:221] offset:50176\n" \
  "v_mfma_f32_16x16x32_f16 a[64:67], v[34:37], v[134:137], a[64:67]\n" \
  "buffer_load_dwordx4 v[42:45], %[bxr], s[40:43], s73 offen offset:3072 sc0 sc1\n" \
  "s_nop 7\n" \
  "v_mfma_f32_16x16x32_f16 a[96:99], v[58:61], v[134:137], a[96:99]\n" \
  "s_add_u32 s72, s74, s69\n" \
  "ds_read_b128 v[134:137], v245 offset:24576\n" \
  "v_mfma_f32_16x16x32_f16 a[96:99], v[54:57], v[102:105], a[96:99]\n" \
  "s_waitcnt lgkmcnt(14)\n" \
  "ds_read_b128 v[102:105], v245 offset:16384\n" \
  "v_mfma_f32_16x16x32_f16 a[100:103], v[54:57], v[106:109], a[100:103]\n" \
  "ds_write_b128 v242, v[222:225] offset:51200\n" \
  "v_mfma_f32_16x16x32_f16 a[100:103], v[58:61], v[138:141], a[100:103]\n" \
  "buffer_load_dwordx4 v[18:21], %[bxr], s[40:43], s72 offen offset:0 sc0 sc1\n" \
  "v_mfma_f32_16x16x32_f16 a[68:71], v[34:37], v[138:141], a[68:71]\n" \
  "ds_read_b128 v[138:141], v245 offset:25600\n" \
  "v_mfma_f32_16x16x32_f16 a[68:71], v[30:33], v[106:109], a[68:71]\n" \
  "s_waitcnt lgkmcnt(14)\n" \
  "ds_read_b128 v[106:109], v245 offset:17408\n" \
  "v_mfma_f32_16x16x32_f16 a[72:75], v[30:33], v[110:113], a[72:75]\n" \
  "ds_write_b128 v242, v[226:229] offset:52224\n" \
  "v_mfma_f32_16x16x32_f16 a[72:75], v[34:37], v[142:145], a[72:75]\n" \
  "buffer_load_dwordx4 v[14:17], %[bxr], s[40:43], s72 offen offset:1024 sc0 sc1\n" \
  "v_mfma_f32_16x16x32_f16 a[104:107], v[58:61], v[142:145], a[104:107]\n" \
  "ds_read_b128 v[142:145], v245 offset:26624\n" \
  "v_mfma_f32_16x16x32_f16 a[104:107], v[54:57], v[110:113], a[104:107]\n" \
  "s_waitcnt lgkmcnt(14)\n" \
  "ds_read_b128 v[110:113], v245 offset:18432\n" \
  "v_mfma_f32_16x16x32_f16 a[108:111], v[54:57], v[114:117], a[108:111]\n" \
  "v_mfma_f32_16x16x32_f16 a[108:111], v[58:61], v[146:149], a[108:111]\n" \
  "v_mfma_f32_16x16x32_f16 a[76:79], v[34:37], v[146:149], a[76:79]\n" \
  "ds_read_b128 v[146:149], v245 offset:27648\n" \
  "v_mfma_f32_16x16x32_f16 a[76:79], v[30:33], v[114:117], a[76:79]\n" \
  "s_waitcnt lgkmcnt(14)\n" \
  "ds_read_b128 v[114:117], v245 offset:19456\n" \
  "v_mfma_f32_16x16x32_f16 a[80:83], v[30:33], v[118:121], a[80:83]\n" \
  "v_mfma_f32_16x16x32_f16 a[80:83], v[34:37], v[150:153], a[80:83]\n" \
  "s_nop 7\n" \
  "v_mfma_f32_16x16x32_f16 a[112:115], v[58:61], v[150:153], a[112:115]\n" \
  "ds_read_b128 v[150:153], v245 offset:28672\n" \
  "v_mfma_f32_16x16x32_f16 a[112:115], v[54:57], v[118:121], a[112:115]\n" \
  "s_waitcnt lgkmcnt(14)\n" \
  "ds_read_b128 v[118:121], v245 offset:20480\n" \
  "v_mfma_f32_16x16x32_f16 a[116:119], v[54:57], v[122:125], a[116:119]\n" \
  "v_mfma_f32_16x16x32_f16 a[116:119], v[58:61], v[154:157], a[116:119]\n" \
  "v_mfma_f32_16x16x32_f16 a[84:87], v[34:37], v[154:157], a[84:87]\n" \
  "ds_read_b128 v[154:157], v245 offset:29696\n" \
  "v_mfma_f32_16x16x32_f16 a[84:87], v[30:33], v[122:125], a[84:87]\n" \
  "s_waitcnt lgkmcnt(14)\n" \
  "ds_read_b128 v[122:125], v245 offset:21504\n" \
  "v_mfma_f32_16x16x32_f16 a[88:91], v[30:33], v[126:129], a[88:91]\n" \
  "v_mfma_f32_16x16x32_f16 a[88:91], v[34:37], v[158:161], a[88:91]\n" \
  "v_mfma_f32_16x16x32_f16 a[120:123], v[58:61], v[158:161], a[120:123]\n" \
  "ds_read_b128 v[158:161], v245 offset:30720\n" \
  "v_mfma_f32_16x16x32_f16 a[120:123], v[54:57], v[126:129], a[120:123]\n" \
  "s_waitcnt lgkmcnt(14)\n" \
  "ds_read_b128 v[126:129], v245 offset:22528\n" \
  "v_mfma_f32_16x16x32_f16 a[124:127], v[54:57], v[130:133], a[124:127]\n" \
  "buffer_load_dwordx4 v[54:57], %[bxr], s[40:43], s72 offen offset:2048 sc0 sc1\n" \
  "v_mfma_f32_16x16x32_f16 a[124:127], v[58:61], v[162:165], a[124:127]\n" \
  "v_readlane_b32 s74, %[vtabA], 25\n" \
  "v_mfma_f32_16x16x32_f16 a[92:95], v[34:37], v[162:165], a[92:95]\n" \
  "ds_read_b128 v[162:165], v245 offset:31744\n" \
  "v_mfma_f32_16x16x32_f16 a[92:95], v[30:33], v[130:133], a[92:95]\n" \
  "s_waitcnt vmcnt(15)\n" \
  "ds_read_b128 v[130:133], v245 offset:23552\n" \
  "s_waitcnt lgkmcnt(14)\n" \
  "v_mfma_f32_16x16x32_f16 a[64:67], v[46:49], v[102:105], a[64:67]\n" \
  "buffer_load_dwordx4 v[58:61], %[bxr], s[40:43], s72 offen offset:3072 sc0 sc1\n" \
  "v_mfma_f32_16x16x32_f16 a[64:67], v[50:53], v[134:137], a[64:67]\n" \
  "s_waitcnt lgkmcnt(0)\n" \
  "s_barrier\n" \
  "s_add_u32 s73, s74, s69\n" \
  "s_nop 7\n" \
  "v_mfma_f32_16x16x32_f16 a[96:99], v[74:77], v[134:137], a[96:99]\n" \
  "ds_read_b128 v[134:137], v245 offset:40960\n" \
  "v_mfma_f32_16x16x32_f16 a[96:99], v[70:73], v[102:105], a[96:99]\n" \
  "ds_read_b128 v[102:105], v245 offset:32768\n" \
  "v_mfma_f32_16x16x32_f16 a[100:103], v[70:73], v[106:109], a[100:103]\n" \
  "buffer_load_dwordx4 v[34:37], %[bxr], s[40:43], s73 offen offset:0 sc0 sc1\n" \
  "v_mfma_f32_16x16x32_f16 a[100:103], v[74:77], v[138:141], a[100:103]\n" \
  "v_mfma_f32_16x16x32_f16 a[68:71], v[50:53], v[138:141], a[68:71]\n" \
  "ds_read_b128 v[138:141], v245 offset:41984\n" \
  "v_mfma_f32_16x16x32_f16 a[68:71], v[46:49], v[106:109], a[68:71]\n" \
  "ds_read_b128 v[106:109], v245 offset:33792\n" \
  "v_mfma_f32_16x16x32_f16 a[72:75], v[46:49], v[110:113], a[72:75]\n" \
  "buffer_load_dwordx4 v[30:33], %[bxr], s[40:43], s73 offen offset:1024 sc0 sc1\n" \
  "v_mfma_f32_16x16x32_f16 a[72:75], v[50:53], v[142:145], a[72:75]\n" \
  "v_mfma_f32_16x16x32_f16 a[104:107], v[74:77], v[142:145], a[104:107]\n" \
  "ds_read_b128 v[142:145], v245 offset:43008\n" \
  "v_mfma_f32_16x16x32_f16 a[104:107], v[70:73], v[110:113], a[104:107]\n" \
  "ds_read_b128 v[110:113], v245 offset:34816\n" \
  "v_mfma_f32_16x16x32_f16 a[108:111], v[70:73], v[114:117], a[108:111]\n" \
  "v_mfma_f32_16x16x32_f16 a[108:111], v[74:77], v[146:149], a[108:111]\n" \
  "v_mfma_f32_16x16x32_f16 a[76:79], v[50:53], v[146:149], a[76:79]\n" \
  "ds_read_b128 v[146:149], v245 offset:44032\n" \
  "v_mfma_f32_16x16x32_f16 a[76:79], v[46:49], v[114:117], a[76:79]\n" \
  "ds_read_b128 v[114:117], v245 offset:35840\n" \
  "v_mfma_f32_16x16x32_f16 a[80:83], v[46:49], v[118:121], a[80:83]\n" \
  "v_mfma_f32_16x16x32_f16 a[80:83], v[50:53], v[150:153], a[80:83]\n" \
  "s_nop 7\n" \
  "v_mfma_f32_16x16x32_f16 a[112:115], v[74:77], v[150:153], a[112:115]\n" \
  "ds_read_b128 v[150:153], v245 offset:45056\n" \
  "v_mfma_f32_16x16x32_f16 a[112:115], v[70:73], v[118:121], a[112:115]\n" \
  "ds_read_b128 v[118:121], v245 offset:36864\n" \
  "v_mfma_f32_16x16x32_f16 a[116:119], v[70:73], v[122:125], a[116:119]\n" \
  "v_mfma_f32_16x16x32_f16 a[116:119], v[74:77], v[154:157], a[116:119]\n" \
  "v_mfma_f32_16x16x32_f16 a[84:87], v[50:53], v[154:157], a[84:87]\n" \
  "ds_read_b128 v[154:157], v245 offset:46080\n" \
  "v_mfma_f32_16x16x32_f16 a[84:87], v[46:49], v[122:125], a[84:87]\n" \
  "ds_read_b128 v[122:125], v245 offset:37888\n" \
  "v_mfma_f32_16x16x32_f16 a[88:91], v[46:49], v[126:129], a[88:91]\n" \
  "v_mfma_f32_16x16x32_f16 a[88:91], v[50:53], v[158:161], a[88:91]\n" \
  "v_mfma_f32_16x16x32_f16 a[120:123], v[74:77], v[158:161], a[120:123]\n" \
  "ds_read_b128 v[158:161], v245 offset:47104\n" \
  "v_mfma_f32_16x16x32_f16 a[120:123], v[70:73], v[126:129], a[120:123]\n" \
  "ds_read_b128 v[126:129], v245 offset:38912\n" \
  "v_mfma_f32_16x16x32_f16 a[124:127], v[70:73], v[130:133], a[124:127]\n" \
  "buffer_load_dwordx4 v[70:73], %[bxr], s[40:43], s73 offen offset:2048 sc0 sc1\n" \
  "v_mfma_f32_16x16x32_f16 a[124:127], v[74:77], v[162:165], a[124:127]\n" \
  "v_readlane_b32 s74, %[vtabA], 26\n" \
  "v_mfma_f32_16x16x32_f16 a[92:95], v[50:53], v[162:165], a[92:95]\n" \
  "ds_read_b128 v[162:165], v245 offset:48128\n" \
  "v_mfma_f32_16x16x32_f16 a[92:95], v[46:49], v[130:133], a[92:95]\n" \
  "s_waitcnt vmcnt(15)\n" \
  "ds_read_b128 v[130:133], v245 offset:39936\n" \
  "s_waitcnt lgkmcnt(14)\n" \
  "v_mfma_f32_16x16x32_f16 a[64:67], v[62:65], v[102:105], a[64:67]\n" \
  "buffer_load_dwordx4 v[74:77], %[bxr], s[40:43], s73 offen offset:3072 sc0 sc1\n" \
  "v_mfma_f32_16x16x32_f16 a[64:67], v[66:69], v[134:137], a[64:67]\n" \
  "s_add_u32 s72, s74, s69\n" \
  "s_nop 7\n" \
  "v_mfma_f32_16x16x32_f16 a[96:99], v[90:93], v[134:137], a[96:99]\n" \
  "ds_read_b128 v[134:137], v245 offset:57344\n" \
  "v_mfma_f32_16x16x32_f16 a[96:99], v[86:89], v[102:105], a[96:99]\n" \
  "s_waitcnt lgkmcnt(13)\n" \
  "ds_read_b128 v[102:105], v245 offset:49152\n" \
  "v_mfma_f32_16x16x32_f16 a[100:103], v[86:89], v[106:109], a[100:103]\n" \
  "buffer_load_dwordx4 v[50:53], %[bxr], s[40:43], s72 offen offset:0 sc0 sc1\n" \
  "v_mfma_f32_16x16x32_f16 a[100:103], v[90:93], v[138:141], a[100:103]\n" \
  "v_mfma_f32_16x16x32_f16 a[68:71], v[66:69], v[138:141], a[68:71]\n" \
  "ds_read_b128 v[138:141], v245 offset:58368\n" \
  "v_mfma_f32_16x16x32_f16 a[68:71], v[62:65], v[106:109], a[68:71]\n" \
  "s_waitcnt lgkmcnt(13)\n" \
  "ds_read_b128 v[106:109], v245 offset:50176\n" \
  "v_mfma_f32_16x16x32_f16 a[72:75], v[62:65], v[110:113], a[72:75]\n" \
  "buffer_load_dwordx4 v[46:49], %[bxr], s[40:43], s72 offen offset:1024 sc0 sc1\n" \
  "v_mfma_f32_16x16x32_f16 a[72:75], v[66:69], v[142:145], a[72:75]\n" \
  "v_mfma_f32_16x16x32_f16 a[104:107], v[90:93], v[142:145], a[104:107]\n" \
  "ds_read_b128 v[142:145], v245 offset:59392\n" \
  "v_mfma_f32_16x16x32_f16 a[104:107], v[86:89], v[110:113], a[104:107]\n" \
  "s_waitcnt lgkmcnt(13)\n" \
  "ds_read_b128 v[110:113], v245 offset:51200\n" \
  "v_mfma_f32_16x16x32_f16 a[108:111], v[86:89], v[114:117], a[108:111]\n" \
  "v_mfma_f32_16x16x32_f16 a[108:111], v[90:93], v[146:149], a[108:111]\n" \
  "v_mfma_f32_16x16x32_f16 a[76:79], v[66:69], v[146:149], a[76:79]\n" \
  "ds_read_b128 v[146:149], v245 offset:60416\n" \
  "v_mfma_f32_16x16x32_f16 a[76:79], v[62:65], v[114:117], a[76:79]\n" \
  "s_waitcnt lgkmcnt(13)\n" \
  "ds_read_b128 v[114:117], v245 offset:52224\n" \
  "v_mfma_f32_16x16x32_f16 a[80:83], v[62:65], v[118:121], a[80:83]\n" \
  "v_mfma_f32_16x16x32_f16 a[80:83], v[66:69], v[150:153], a[80:83]\n" \
  "s_nop 7\n" \
  "v_mfma_f32_16x16x32_f16 a[112:115], v[90:93], v[150:153], a[112:115]\n" \
  "ds_read_b128 v[150:153], v245 offset:61440\n" \
  "v_mfma_f32_16x16x32_f16 a[112:115], v[86:89], v[118:121], a[112:115]\n" \
  "s_waitcnt lgkmcnt(13)\n" \
  "ds_read_b128 v[118:121], v245 offset:53248\n" \
  "v_mfma_f32_16x16x32_f16 a[116:119], v[86:89], v[122:125], a[116:119]\n" \
  "v_mfma_f32_16x16x32_f16 a[116:119], v[90:93], v[154:157], a[116:119]\n" \
  "v_mfma_f32_16x16x32_f16 a[84:87], v[66:69], v[154:157], a[84:87]\n" \
  "ds_read_b128 v[154:157], v245 offset:62464\n" \
  "v_mfma_f32_16x16x32_f16 a[84:87], v[62:65], v[122:125], a[84:87]\n" \
  "s_waitcnt lgkmcnt(13)\n" \
  "ds_read_b128 v[122:125], v245 offset:54272\n" \
  "v_mfma_f32_16x16x32_f16 a[88:91], v[62:65], v[126:129], a[88:91]\n" \
  "v_mfma_f32_16x16x32_f16 a[88:91], v[66:69], v[158:161], a[88:91]\n" \
  "v_mfma_f32_16x16x32_f16 a[120:123], v[90:93], v[158:161], a[120:123]\n" \
  "ds_read_b128 v[158:161], v245 offset:63488\n" \
  "v_mfma_f32_16x16x32_f16 a[120:123], v[86:89], v[126:129], a[120:123]\n" \
  "s_waitcnt lgkmcnt(13)\n" \
  "ds_read_b128 v[126:129], v245 offset:55296\n" \
  "v_mfma_f32_16x16x32_f16 a[124:127], v[86:89], v[130:133], a[124:127]\n" \
  "buffer_load_dwordx4 v[86:89], %[bxr], s[40:43], s72 offen offset:2048 sc0 sc1\n" \
  "v_mfma_f32_16x16x32_f16 a[124:127], v[90:93], v[162:165], a[124:127]\n" \
  "v_readlane_b32 s74, %[vtabA], 27\n" \
  "v_mfma_f32_16x16x32_f16 a[92:95], v[66:69], v[162:165], a[92:95]\n" \
  "ds_read_b128 v[162:165], v245 offset:64512\n" \
  "v_mfma_f32_16x16x32_f16 a[92:95], v[62:65], v[130:133], a[92:95]\n" \
  "s_waitcnt vmcnt(15)\n" \
  "ds_read_b128 v[130:133], v245 offset:56320\n" \
  "s_waitcnt lgkmcnt(14)\n" \
  "v_mfma_f32_16x16x32_f16 a[64:67], v[78:81], v[102:105], a[64:67]\n" \
  "buffer_load_dwordx4 v[90:93], %[bxr], s[40:43], s72 offen offset:3072 sc0 sc1\n" \
  "v_mfma_f32_16x16x32_f16 a[64:67], v[82:85], v[134:137], a[64:67]\n" \
  "s_add_u32 s73, s74, s69\n" \
  "s_nop 7\n" \
  "v_mfma_f32_16x16x32_f16 a[96:99], v[26:29], v[134:137], a[96:99]\n" \
  "ds_read_b128 v[134:137], v246 offset:24576\n" \
  "v_mfma_f32_16x16x32_f16 a[96:99], v[22:25], v[102:105], a[96:99]\n" \
  "s_waitcnt lgkmcnt(13)\n" \
  "ds_read_b128 v[102:105], v246 offset:16384\n" \
  "v_mfma_f32_16x16x32_f16 a[100:103], v[22:25], v[106:109], a[100:103]\n" \
  "buffer_load_dwordx4 v[66:69], %[bxr], s[40:43], s73 offen offset:0 sc0 sc1\n" \
  "v_mfma_f32_16x16x32_f16 a[100:103], v[26:29], v[138:141], a[100:103]\n" \
  "v_mfma_f32_16x16x32_f16 a[68:71], v[82:85], v[138:141], a[68:71]\n" \
  "ds_read_b128 v[138:141], v246 offset:25600\n" \
  "v_mfma_f32_16x16x32_f16 a[68:71], v[78:81], v[106:109], a[68:71]\n" \
  "s_waitcnt lgkmcnt(13)\n" \
  "ds_read_b128 v[106:109], v246 offset:17408\n" \
  "v_mfma_f32_16x16x32_f16 a[72:75], v[78:81], v[110:113], a[72:75]\n" \
  "buffer_load_dwordx4 v[62:65], %[bxr], s[40:43], s73 offen offset:1024 sc0 sc1\n" \
  "v_mfma_f32_16x16x32_f16 a[72:75], v[82:85], v[142:145], a[72:75]\n" \
  "v_mfma_f32_16x16x32_f16 a[104:107], v[26:29], v[142:145], a[104:107]\n" \
  "ds_read_b128 v[142:145], v246 offset:26624\n" \
  "v_mfma_f32_16x16x32_f16 a[104:107], v[22:25], v[110:113], a[104:107]\n" \
  "s_waitcnt lgkmcnt(13)\n" \
  "ds_read_b128 v[110:113], v246 offset:18432\n" \
  "v_mfma_f32_16x16x32_f16 a[108:111], v[22:25], v[114:117], a[108:111]\n" \
  "v_mfma_f32_16x16x32_f16 a[108:111], v[26:29], v[146:149], a[108:111]\n" \
  "v_mfma_f32_16x16x32_f16 a[76:79], v[82:85], v[146:149], a[76:79]\n" \
  "ds_read_b128 v[146:149], v246 offset:27648\n" \
  "v_mfma_f32_16x16x32_f16 a[76:79], v[78:81], v[114:117], a[76:79]\n" \
  "s_waitcnt lgkmcnt(13)\n" \
  "ds_read_b128 v[114:117], v246 offset:19456\n" \
  "v_mfma_f32_16x16x32_f16 a[80:83], v[78:81], v[118:121], a[80:83]\n" \
  "v_mfma_f32_16x16x32_f16 a[80:83], v[82:85], v[150:153], a[80:83]\n" \
  "s_nop 7\n" \
  "v_mfma_f32_16x16x32_f16 a[112:115], v[26:29], v[150:153], a[112:115]\n" \
  "ds_read_b128 v[150:153], v246 offset:28672\n" \
  "v_mfma_f32_16x16x32_f16 a[112:115], v[22:25], v[118:121], a[112:115]\n" \
  "s_waitcnt lgkmcnt(13)\n" \
  "ds_read_b128 v[118:121], v246 offset:20480\n" \
  "v_mfma_f32_16x16x32_f16 a[116:119], v[22:25], v[122:125], a[116:119]\n" \
  "v_mfma_f32_16x16x32_f16 a[116:119], v[26:29], v[154:157], a[116:119]\n" \
  "v_mfma_f32_16x16x32_f16 a[84:87], v[82:85], v[154:157], a[84:87]\n" \
  "ds_read_b128 v[154:157], v246 offset:29696\n" \
  "v_mfma_f32_16x16x32_f16 a[84:87], v[78:81], v[122:125], a[84:87]\n" \
  "s_waitcnt lgkmcnt(13)\n" \
  "ds_read_b128 v[122:125], v246 offset:21504\n" \
  "v_mfma_f32_16x16x32_f16 a[88:91], v[78:81], v[126:129], a[88:91]\n" \
  "v_mfma_f32_16x16x32_f16 a[88:91], v[82:85], v[158:161], a[88:91]\n" \
  "s_add_u32 s83, s90, 0x0\n" \
  "v_mfma_f32_16x16x32_f16 a[120:123], v[26:29], v[158:161], a[120:123]\n" \
  "ds_read_b128 v[158:161], v246 offset:30720\n" \
  "v_mfma_f32_16x16x32_f16 a[120:123], v[22:25], v[126:129], a[120:123]\n" \
  "s_waitcnt lgkmcnt(13)\n" \
  "ds_read_b128 v[126:129], v246 offset:22528\n" \
  "v_mfma_f32_16x16x32_f16 a[124:127], v[22:25], v[130:133], a[124:127]\n" \
  "buffer_load_dwordx4 v[22:25], %[bxr], s[40:43], s73 offen offset:2048 sc0 sc1\n" \
  "v_mfma_f32_16x16x32_f16 a[124:127], v[26:29], v[162:165], a[124:127]\n" \
  "v_readlane_b32 s74, %[vtabA], 28\n" \
  "v_mfma_f32_16x16x32_f16 a[92:95], v[82:85], v[162:165], a[92:95]\n" \
  "ds_read_b128 v[162:165], v246 offset:31744\n" \
  "v_mfma_f32_16x16x32_f16 a[92:95], v[78:81], v[130:133], a[92:95]\n" \
  "s_waitcnt vmcnt(15)\n" \
  "ds_read_b128 v[130:133], v246 offset:23552\n" \
  "s_waitcnt lgkmcnt(14)\n" \
  "v_mfma_f32_16x16x32_f16 a[192:195], v[94:97], v[102:105], a[192:195]\n" \
  "buffer_load_dwordx4 v[26:29], %[bxr], s[40:43], s73 offen offset:3072 sc0 sc1\n" \
  "v_mfma_f32_16x16x32_f16 a[192:195], v[98:101], v[134:137], a[192:195]\n" \
  "buffer_store_dwordx4 a[64:67], %[voffO], s[44:47], s83 offen offset:0 sc1\n" \
  "s_nop 7\n" \
  "v_mfma_f32_16x16x32_f16 a[224:227], v[42:45], v[134:137], a[224:227]\n" \
  "s_add_u32 s72, s74, s69\n" \
  "ds_read_b128 v[134:137], v246 offset:40960\n" \
  "v_mfma_f32_16x16x32_f16 a[224:227], v[38:41], v[102:105], a[224:227]\n" \
  "s_waitcnt lgkmcnt(13)\n" \
  "ds_read_b128 v[102:105], v246 offset:32768\n" \
  "v_mfma_f32_16x16x32_f16 a[228:231], v[38:41], v[106:109], a[228:231]\n" \
  "buffer_load_dwordx4 v[82:85], %[bxr], s[40:43], s72 offen offset:0 sc0 sc1\n" \
  "v_mfma_f32_16x16x32_f16 a[228:231], v[42:45], v[138:141], a[228:231]\n" \
  "v_mfma_f32_16x16x32_f16 a[196:199], v[98:101], v[138:141], a[196:199]\n" \
  "ds_read_b128 v[138:141], v246 offset:41984\n" \
  "v_mfma_f32_16x16x32_f16 a[196:199], v[94:97], v[106:109], a[196:199]\n" \
  "s_waitcnt lgkmcnt(13)\n" \
  "ds_read_b128 v[106:109], v246 offset:33792\n" \
  "v_mfma_f32_16x16x32_f16 a[200:203], v[94:97], v[110:113], a[200:203]\n" \
  "buffer_load_dwordx4 v[78:81], %[bxr], s[40:43], s72 offen offset:1024 sc0 sc1\n" \
  "v_mfma_f32_16x16x32_f16 a[200:203], v[98:101], v[142:145], a[200:203]\n" \
  "v_mfma_f32_16x16x32_f16 a[232:235], v[42:45], v[142:145], a[232:235]\n" \
  "ds_read_b128 v[142:145], v246 offset:43008\n" \
  "v_mfma_f32_16x16x32_f16 a[232:235], v[38:41], v[110:113], a[232:235]\n" \
  "s_waitcnt lgkmcnt(13)\n" \
  "ds_read_b128 v[110:113], v246 offset:34816\n" \
  "v_mfma_f32_16x16x32_f16 a[236:239], v[38:41], v[114:117], a[236:239]\n" \
  "v_mfma_f32_16x16x32_f16 a[236:239], v[42:45], v[146:149], a[236:239]\n" \
  "v_mfma_f32_16x16x32_f16 a[204:207], v[98:101], v[146:149], a[204:207]\n" \
  "ds_read_b128 v[146:149], v246 offset:44032\n" \
  "v_mfma_f32_16x16x32_f16 a[204:207], v[94:97], v[114:117], a[204:207]\n" \
  "s_waitcnt lgkmcnt(13)\n" \
  "ds_read_b128 v[114:117], v246 offset:35840\n" \
  "v_mfma_f32_16x16x32_f16 a[208:211], v[94:97], v[118:121], a[208:211]\n" \
  "v_mfma_f32_16x16x32_f16 a[208:211], v[98:101], v[150:153], a[208:211]\n" \
  "s_nop 7\n" \
  "v_mfma_f32_16x16x32_f16 a[240:243], v[42:45], v[150:153], a[240:243]\n" \
  "ds_read_b128 v[150:153], v246 offset:45056\n" \
  "v_mfma_f32_16x16x32_f16 a[240:243], v[38:41], v[118:121], a[240:243]\n" \
  "s_waitcnt lgkmcnt(13)\n" \
  "ds_read_b128 v[118:121], v246 offset:36864\n" \
  "v_mfma_f32_16x16x32_f16 a[244:247], v[38:41], v[122:125], a[244:247]\n" \
  "v_mfma_f32_16x16x32_f16 a[244:247], v[42:45], v[154:157], a[244:247]\n" \
  "v_mfma_f32_16x16x32_f16 a[212:215], v[98:101], v[154:157], a[212:215]\n" \
  "ds_read_b128 v[154:157], v246 offset:46080\n" \
  "v_mfma_f32_16x16x32_f16 a[212:215], v[94:97], v[122:125], a[212:215]\n" \
  "s_waitcnt lgkmcnt(13)\n" \
  "ds_read_b128 v[122:125], v246 offset:37888\n" \
  "v_mfma_f32_16x16x32_f16 a[216:219], v[94:97], v[126:129], a[216:219]\n" \
  "v_mfma_f32_16x16x32_f16 a[216:219], v[98:101], v[158:161], a[216:219]\n" \
  "buffer_store_dwordx4 a[80:83], %[voffO], s[44:47], s83 offen offset:256 sc1\n" \
  "v_mfma_f32_16x16x32_f16 a[248:251], v[42:45], v[158:161], a[248:251]\n" \
  "ds_read_b128 v[158:161], v246 offset:47104\n" \
  "v_mfma_f32_16x16x32_f16 a[248:251], v[38:41], v[126:129], a[248:251]\n" \
  "s_waitcnt lgkmcnt(13)\n" \
  "ds_read_b128 v[126:129], v246 offset:38912\n" \
  "v_mfma_f32_16x16x32_f16 a[252:255], v[38:41], v[130:133], a[252:255]\n" \
  "buffer_load_dwordx4 v[38:41], %[bxr], s[40:43], s72 offen offset:2048 sc0 sc1\n" \
  "v_mfma_f32_16x16x32_f16 a[252:255], v[42:45], v[162:165], a[252:255]\n" \
  "v_readlane_b32 s74, %[vtabA], 29\n" \
  "v_mfma_f32_16x16x32_f16 a[220:223], v[98:101], v[162:165], a[220:223]\n" \
  "ds_read_b128 v[162:165], v246 offset:48128\n" \
  "v_mfma_f32_16x16x32_f16 a[220:223], v[94:97], v[130:133], a[220:223]\n" \
  "s_waitcnt vmcnt(17)\n" \
  "ds_read_b128 v[130:133], v246 offset:39936\n" \
  "s_waitcnt lgkmcnt(14)\n" \
  "v_mfma_f32_16x16x32_f16 a[192:195], v[18:21], v[102:105], a[192:195]\n" \
  "buffer_load_dwordx4 v[42:45], %[bxr], s[40:43], s72 offen offset:3072 sc0 sc1\n" \
  "v_mfma_f32_16x16x32_f16 a[192:195], v[14:17], v[134:137], a[192:195]\n" \
  "s_add_u32 s73, s74, s69\n" \
  "s_nop 7\n" \
  "v_mfma_f32_16x16x32_f16 a[224:227], v[58:61], v[134:137], a[224:227]\n" \
  "ds_read_b128 v[134:137], v246 offset:57344\n" \
  "v_mfma_f32_16x16x32_f16 a[224:227], v[54:57], v[102:105], a[224:227]\n" \
  "s_waitcnt lgkmcnt(13)\n" \
  "ds_read_b128 v[102:105], v246 offset:49152\n" \
  "v_mfma_f32_16x16x32_f16 a[228:231], v[54:57], v[106:109], a[228:231]\n" \
  "buffer_load_dwordx4 v[98:101], %[bxr], s[40:43], s73 offen offset:0 sc0 sc1\n" \
  "v_mfma_f32_16x16x32_f16 a[228:231], v[58:61], v[138:141], a[228:231]\n" \
  "v_mfma_f32_16x16x32_f16 a[196:199], v[14:17], v[138:141], a[196:199]\n" \
  "ds_read_b128 v[138:141], v246 offset:58368\n" \
  "v_mfma_f32_16x16x32_f16 a[196:199], v[18:21], v[106:109], a[196:199]\n" \
  "s_waitcnt lgkmcnt(13)\n" \
  "ds_read_b128 v[106:109], v246 offset:50176\n" \
  "v_mfma_f32_16x16x32_f16 a[200:203], v[18:21], v[110:113], a[200:203]\n" \
  "buffer_load_dwordx4 v[94:97], %[bxr], s[40:43], s73 offen offset:1024 sc0 sc1\n" \
  "v_mfma_f32_16x16x32_f16 a[200:203], v[14:17], v[142:145], a[200:203]\n" \
  "s_add_u32 s84, s90, 0x10000\n" \
  "v_mfma_f32_16x16x32_f16 a[232:235], v[58:61], v[142:145], a[232:235]\n" \
  "ds_read_b128 v[142:145], v246 offset:59392\n" \
  "v_mfma_f32_16x16x32_f16 a[232:235], v[54:57], v[110:113], a[232:235]\n" \
  "s_waitcnt lgkmcnt(13)\n" \
  "ds_read_b128 v[110:113], v246 offset:51200\n" \
  "v_mfma_f32_16x16x32_f16 a[236:239], v[54:57], v[114:117], a[236:239]\n" \
  "v_mfma_f32_16x16x32_f16 a[236:239], v[58:61], v[146:149], a[236:239]\n" \
  "v_mfma_f32_16x16x32_f16 a[204:207], v[14:17], v[146:149], a[204:207]\n" \
  "ds_read_b128 v[146:149], v246 offset:60416\n" \
  "v_mfma_f32_16x16x32_f16 a[204:207], v[18:21], v[114:117], a[204:207]\n" \
  "s_waitcnt lgkmcnt(13)\n" \
  "ds_read_b128 v[114:117], v246 offset:52224\n" \
  "v_mfma_f32_16x16x32_f16 a[208:211], v[18:21], v[118:121], a[208:211]\n" \
  "v_mfma_f32_16x16x32_f16 a[208:211], v[14:17], v[150:153], a[208:211]\n" \
  "buffer_store_dwordx4 a[68:71], %[voffO], s[44:47], s84 offen offset:0 sc1\n" \
  "s_nop 7\n" \
  "v_mfma_f32_16x16x32_f16 a[240:243], v[58:61], v[150:153], a[240:243]\n" \
  "ds_read_b128 v[150:153], v246 offset:61440\n" \
  "v_mfma_f32_16x16x32_f16 a[240:243], v[54:57], v[118:121], a[240:243]\n" \
  "s_waitcnt lgkmcnt(13)\n" \
  "ds_read_b128 v[118:121], v246 offset:53248\n" \
  "v_mfma_f32_16x16x32_f16 a[244:247], v[54:57], v[122:125], a[244:247]\n" \
  "v_mfma_f32_16x16x32_f16 a[244:247], v[58:61], v[154:157], a[244:247]\n" \
  "v_mfma_f32_16x16x32_f16 a[212:215], v[14:17], v[154:157], a[212:215]\n" \
  "ds_read_b128 v[154:157], v246 offset:62464\n" \
  "v_mfma_f32_16x16x32_f16 a[212:215], v[18:21], v[122:125], a[212:215]\n" \
  "s_waitcnt lgkmcnt(13)\n" \
  "ds_read_b128 v[122:125], v246 offset:54272\n" \
  "v_mfma_f32_16x16x32_f16 a[216:219], v[18:21], v[126:129], a[216:219]\n" \
  "v_mfma_f32_16x16x32_f16 a[216:219], v[14:17], v[158:161], a[216:219]\n" \
  "v_mfma_f32_16x16x32_f16 a[248:251], v[58:61], v[158:161], a[248:251]\n" \
  "ds_read_b128 v[158:161], v246 offset:63488\n" \
  "v_mfma_f32_16x16x32_f16 a[248:251], v[54:57], v[126:129], a[248:251]\n" \
  "s_waitcnt lgkmcnt(13)\n" \
  "ds_read_b128 v[126:129], v246 offset:55296\n" \
  "v_mfma_f32_16x16x32_f16 a[252:255], v[54:57], v[130:133], a[252:255]\n" \
  "buffer_load_dwordx4 v[54:57], %[bxr], s[40:43], s73 offen offset:2048 sc0 sc1\n" \
  "v_mfma_f32_16x16x32_f16 a[252:255], v[58:61], v[162:165], a[252:255]\n" \
  "v_readlane_b32 s74, %[vtabA], 30\n" \
  "v_mfma_f32_16x16x32_f16 a[220:223], v[14:17], v[162:165], a[220:223]\n" \
  "ds_read_b128 v[162:165], v246 offset:64512\n" \
  "v_mfma_f32_16x16x32_f16 a[220:223], v[18:21], v[130:133], a[220:223]\n" \
  "s_waitcnt vmcnt(18)\n" \
  "ds_read_b128 v[130:133], v246 offset:56320\n" \
  "s_waitcnt lgkmcnt(14)\n" \
  "v_mfma_f32_16x16x32_f16 a[192:195], v[34:37], v[102:105], a[192:195]\n" \
  "buffer_load_dwordx4 v[58:61], %[bxr], s[40:43], s73 offen offset:3072 sc0 sc1\n" \
  "v_mfma_f32_16x16x32_f16 a[192:195], v[30:33], v[134:137], a[192:195]\n" \
  "s_add_u32 s72, s74, s69\n" \
  "s_nop 7\n" \
  "v_mfma_f32_16x16x32_f16 a[224:227], v[74:77], v[134:137], a[224:227]\n" \
  "ds_read_b128 v[134:137], v247 offset:8192\n" \
  "v_mfma_f32_16x16x32_f16 a[224:227], v[70:73], v[102:105], a[224:227]\n" \
  "s_waitcnt lgkmcnt(13)\n" \
  "ds_read_b128 v[102:105], v247 offset:0\n" \
  "v_mfma_f32_16x16x32_f16 a[228:231], v[70:73], v[106:109], a[228:231]\n" \
  "buffer_load_dwordx4 v[14:17], %[bxr], s[40:43], s72 offen offset:0 sc0 sc1\n" \
  "v_mfma_f32_16x16x32_f16 a[228:231], v[74:77], v[138:141], a[228:231]\n" \
  "v_mfma_f32_16x16x32_f16 a[196:199], v[30:33], v[138:141], a[196:199]\n" \
  "ds_read_b128 v[138:141], v247 offset:9216\n" \
  "v_mfma_f32_16x16x32_f16 a[196:199], v[34:37], v[106:109], a[196:199]\n" \
  "s_waitcnt lgkmcnt(13)\n" \
  "ds_read_b128 v[106:109], v247 offset:1024\n" \
  "v_mfma_f32_16x16x32_f16 a[200:203], v[34:37], v[110:113], a[200:203]\n" \
  "buffer_load_dwordx4 v[18:21], %[bxr], s[40:43], s72 offen offset:1024 sc0 sc1\n" \
  "v_mfma_f32_16x16x32_f16 a[200:203], v[30:33], v[142:145], a[200:203]\n" \
  "buffer_store_dwordx4 a[84:87], %[voffO], s[44:47], s84 offen offset:256 sc1\n" \
  "v_mfma_f32_16x16x32_f16 a[232:235], v[74:77], v[142:145], a[232:235]\n" \
  "ds_read_b128 v[142:145], v247 offset:10240\n" \
  "v_mfma_f32_16x16x32_f16 a[232:235], v[70:73], v[110:113], a[232:235]\n" \
  "s_waitcnt lgkmcnt(13)\n" \
  "ds_read_b128 v[110:113], v247 offset:2048\n" \
  "v_mfma_f32_16x16x32_f16 a[236:239], v[70:73], v[114:117], a[236:239]\n" \
  "v_mfma_f32_16x16x32_f16 a[236:239], v[74:77], v[146:149], a[236:239]\n" \
  "v_mfma_f32_16x16x32_f16 a[204:207], v[30:33], v[146:149], a[204:207]\n" \
  "ds_read_b128 v[146:149], v247 offset:11264\n" \
  "v_mfma_f32_16x16x32_f16 a[204:207], v[34:37], v[114:117], a[204:207]\n" \
  "s_waitcnt lgkmcnt(13)\n" \
  "ds_read_b128 v[114:117], v247 offset:3072\n" \
  "v_mfma_f32_16x16x32_f16 a[208:211], v[34:37], v[118:121], a[208:211]\n" \
  "v_mfma_f32_16x16x32_f16 a[208:211], v[30:33], v[150:153], a[208:211]\n" \
  "s_nop 7\n" \
  "v_mfma_f32_16x16x32_f16 a[240:243], v[74:77], v[150:153], a[240:243]\n" \
  "ds_read_b128 v[150:153], v247 offset:12288\n" \
  "v_mfma_f32_16x16x32_f16 a[240:243], v[70:73], v[118:121], a[240:243]\n" \
  "s_waitcnt lgkmcnt(13)\n" \
  "ds_read_b128 v[118:121], v247 offset:4096\n" \
  "v_mfma_f32_16x16x32_f16 a[244:247], v[70:73], v[122:125], a[244:247]\n" \
  "v_mfma_f32_16x16x32_f16 a[244:247], v[74:77], v[154:157], a[244:247]\n" \
  "v_mfma_f32_16x16x32_f16 a[212:215], v[30:33], v[154:157], a[212:215]\n" \
  "ds_read_b128 v[154:157], v247 offset:13312\n" \
  "v_mfma_f32_16x16x32_f16 a[212:215], v[34:37], v[122:125], a[212:215]\n" \
  "s_waitcnt lgkmcnt(13)\n" \
  "ds_read_b128 v[122:125], v247 offset:5120\n" \
  "v_mfma_f32_16x16x32_f16 a[216:219], v[34:37], v[126:129], a[216:219]\n" \
  "v_mfma_f32_16x16x32_f16 a[216:219], v[30:33], v[158:161], a[216:219]\n" \
  "s_add_u32 s83, s90, 0x20000\n" \
  "v_mfma_f32_16x16x32_f16 a[248:251], v[74:77], v[158:161], a[248:251]\n" \
  "ds_read_b128 v[158:161], v247 offset:14336\n" \
  "v_mfma_f32_16x16x32_f16 a[248:251], v[70:73], v[126:129], a[248:251]\n" \
  "s_waitcnt lgkmcnt(13)\n" \
  "ds_read_b128 v[126:129], v247 offset:6144\n" \
  "v_mfma_f32_16x16x32_f16 a[252:255], v[70:73], v[130:133], a[252:255]\n" \
  "buffer_load_dwordx4 v[70:73], %[bxr], s[40:43], s72 offen offset:2048 sc0 sc1\n" \
  "v_mfma_f32_16x16x32_f16 a[252:255], v[74:77], v[162:165], a[252:255]\n" \
  "v_readlane_b32 s74, %[vtabA], 31\n" \
  "v_mfma_f32_16x16x32_f16 a[220:223], v[30:33], v[162:165], a[220:223]\n" \
  "ds_read_b128 v[162:165], v247 offset:15360\n" \
  "v_mfma_f32_16x16x32_f16 a[220:223], v[34:37], v[130:133], a[220:223]\n" \
  "s_waitcnt vmcnt(19)\n" \
  "ds_read_b128 v[130:133], v247 offset:7168\n" \
  "s_waitcnt lgkmcnt(14)\n" \
  "v_mfma_f32_16x16x32_f16 a[192:195], v[50:53], v[102:105], a[192:195]\n" \
  "buffer_load_dwordx4 v[74:77], %[bxr], s[40:43], s72 offen offset:3072 sc0 sc1\n" \
  "v_mfma_f32_16x16x32_f16 a[192:195], v[46:49], v[134:137], a[192:195]\n" \
  "buffer_store_dwordx4 a[72:75], %[voffO], s[44:47], s83 offen offset:0 sc1\n" \
  "s_nop 7\n" \
  "v_mfma_f32_16x16x32_f16 a[224:227], v[90:93], v[134:137], a[224:227]\n" \
  "s_add_u32 s73, s74, s69\n" \
  "ds_read_b128 v[134:137], v247 offset:24576\n" \
  "v_mfma_f32_16x16x32_f16 a[224:227], v[86:89], v[102:105], a[224:227]\n" \
  "s_waitcnt lgkmcnt(13)\n" \
  "ds_read_b128 v[102:105], v247 offset:16384\n" \
  "v_mfma_f32_16x16x32_f16 a[228:231], v[86:89], v[106:109], a[228:231]\n" \
  "buffer_load_dwordx4 v[30:33], %[bxr], s[40:43], s73 offen offset:0 sc0 sc1\n" \
  "v_mfma_f32_16x16x32_f16 a[228:231], v[90:93], v[138:141], a[228:231]\n" \
  "v_mfma_f32_16x16x32_f16 a[196:199], v[46:49], v[138:141], a[196:199]\n" \
  "ds_read_b128 v[138:141], v247 offset:25600\n" \
  "v_mfma_f32_16x16x32_f16 a[196:199], v[50:53], v[106:109], a[196:199]\n" \
  "s_waitcnt lgkmcnt(13)\n" \
  "ds_read_b128 v[106:109], v247 offset:17408\n" \
  "v_mfma_f32_16x16x32_f16 a[200:203], v[50:53], v[110:113], a[200:203]\n" \
  "buffer_load_dwordx4 v[34:37], %[bxr], s[40:43], s73 offen offset:1024 sc0 sc1\n" \
  "v_mfma_f32_16x16x32_f16 a[200:203], v[46:49], v[142:145], a[200:203]\n" \
  "v_mfma_f32_16x16x32_f16 a[232:235], v[90:93], v[142:145], a[232:235]\n" \
  "ds_read_b128 v[142:145], v247 offset:26624\n" \
  "v_mfma_f32_16x16x32_f16 a[232:235], v[86:89], v[110:113], a[232:235]\n" \
  "s_waitcnt lgkmcnt(13)\n" \
  "ds_read_b128 v[110:113], v247 offset:18432\n" \
  "v_mfma_f32_16x16x32_f16 a[236:239], v[86:89], v[114:117], a[236:239]\n" \
  "v_mfma_f32_16x16x32_f16 a[236:239], v[90:93], v[146:149], a[236:239]\n" \
  "v_mfma_f32_16x16x32_f16 a[204:207], v[46:49], v[146:149], a[204:207]\n" \
  "ds_read_b128 v[146:149], v247 offset:27648\n" \
  "v_mfma_f32_16x16x32_f16 a[204:207], v[50:53], v[114:117], a[204:207]\n" \
  "s_waitcnt lgkmcnt(13)\n" \
  "ds_read_b128 v[114:117], v247 offset:19456\n" \
  "v_mfma_f32_16x16x32_f16 a[208:211], v[50:53], v[118:121], a[208:211]\n" \
  "v_mfma_f32_16x16x32_f16 a[208:211], v[46:49], v[150:153], a[208:211]\n" \
  "s_nop 7\n" \
  "v_mfma_f32_16x16x32_f16 a[240:243], v[90:93], v[150:153], a[240:243]\n" \
  "ds_read_b128 v[150:153], v247 offset:28672\n" \
  "v_mfma_f32_16x16x32_f16 a[240:243], v[86:89], v[118:121], a[240:243]\n" \
  "s_waitcnt lgkmcnt(13)\n" \
  "ds_read_b128 v[118:121], v247 offset:20480\n" \
  "v_mfma_f32_16x16x32_f16 a[244:247], v[86:89], v[122:125], a[244:247]\n" \
  "v_mfma_f32_16x16x32_f16 a[244:247], v[90:93], v[154:157], a[244:247]\n" \
  "v_mfma_f32_16x16x32_f16 a[212:215], v[46:49], v[154:157], a[212:215]\n" \
  "ds_read_b128 v[154:157], v247 offset:29696\n" \
  "v_mfma_f32_16x16x32_f16 a[212:215], v[50:53], v[122:125], a[212:215]\n" \
  "s_waitcnt lgkmcnt(13)\n" \
  "ds_read_b128 v[122:125], v247 offset:21504\n" \
  "v_mfma_f32_16x16x32_f16 a[216:219], v[50:53], v[126:129], a[216:219]\n" \
  "v_mfma_f32_16x16x32_f16 a[216:219], v[46:49], v[158:161], a[216:219]\n" \
  "buffer_store_dwordx4 a[88:91], %[voffO], s[44:47], s83 offen offset:256 sc1\n" \
  "v_mfma_f32_16x16x32_f16 a[248:251], v[90:93], v[158:161], a[248:251]\n" \
  "ds_read_b128 v[158:161], v247 offset:30720\n" \
  "v_mfma_f32_16x16x32_f16 a[248:251], v[86:89], v[126:129], a[248:251]\n" \
  "s_waitcnt lgkmcnt(13)\n" \
  "ds_read_b128 v[126:129], v247 offset:22528\n" \
  "v_mfma_f32_16x16x32_f16 a[252:255], v[86:89], v[130:133], a[252:255]\n" \
  "buffer_load_dwordx4 v[86:89], %[bxr], s[40:43], s73 offen offset:2048 sc0 sc1\n" \
  "v_mfma_f32_16x16x32_f16 a[252:255], v[90:93], v[162:165], a[252:255]\n" \
  "v_readlane_b32 s74, %[vtabA], 23\n" \
  "v_mfma_f32_16x16x32_f16 a[220:223], v[46:49], v[162:165], a[220:223]\n" \
  "ds_read_b128 v[162:165], v247 offset:31744\n" \
  "v_mfma_f32_16x16x32_f16 a[220:223], v[50:53], v[130:133], a[220:223]\n" \
  "s_waitcnt vmcnt(21)\n" \
  "ds_read_b128 v[130:133], v247 offset:23552\n" \
  "s_waitcnt lgkmcnt(14)\n" \
  "v_mfma_f32_16x16x32_f16 a[192:195], v[66:69], v[102:105], a[192:195]\n" \
  "buffer_load_dwordx4 v[90:93], %[bxr], s[40:43], s73 offen offset:3072 sc0 sc1\n" \
  "v_mfma_f32_16x16x32_f16 a[192:195], v[62:65], v[134:137], a[192:195]\n" \
  "s_add_u32 s72, s74, s70\n" \
  "s_nop 7\n" \
  "v_mfma_f32_16x16x32_f16 a[224:227], v[26:29], v[134:137], a[224:227]\n" \
  "ds_read_b128 v[134:137], v245 offset:8192\n" \
  "v_mfma_f32_16x16x32_f16 a[224:227], v[22:25], v[102:105], a[224:227]\n" \
  "s_waitcnt lgkmcnt(13)\n" \
  "ds_read_b128 v[102:105], v245 offset:0\n" \
  "v_mfma_f32_16x16x32_f16 a[228:231], v[22:25], v[106:109], a[228:231]\n" \
  "buffer_load_dwordx4 v[46:49], %[bxr], s[40:43], s72 offen offset:0 sc0 sc1\n" \
  "v_mfma_f32_16x16x32_f16 a[228:231], v[26:29], v[138:141], a[228:231]\n" \
  "v_mfma_f32_16x16x32_f16 a[196:199], v[62:65], v[138:141], a[196:199]\n" \
  "ds_read_b128 v[138:141], v245 offset:9216\n" \
  "v_mfma_f32_16x16x32_f16 a[196:199], v[66:69], v[106:109], a[196:199]\n" \
  "s_waitcnt lgkmcnt(13)\n" \
  "ds_read_b128 v[106:109], v245 offset:1024\n" \
  "v_mfma_f32_16x16x32_f16 a[200:203], v[66:69], v[110:113], a[200:203]\n" \
  "buffer_load_dwordx4 v[50:53], %[bxr], s[40:43], s72 offen offset:1024 sc0 sc1\n" \
  "v_mfma_f32_16x16x32_f16 a[200:203], v[62:65], v[142:145], a[200:203]\n" \
  "s_add_u32 s84, s90, 0x30000\n" \
  "v_mfma_f32_16x16x32_f16 a[232:235], v[26:29], v[142:145], a[232:235]\n" \
  "ds_read_b128 v[142:145], v245 offset:10240\n" \
  "v_mfma_f32_16x16x32_f16 a[232:235], v[22:25], v[110:113], a[232:235]\n" \
  "s_waitcnt lgkmcnt(13)\n" \
  "ds_read_b128 v[110:113], v245 offset:2048\n" \
  "v_mfma_f32_16x16x32_f16 a[236:239], v[22:25], v[114:117], a[236:239]\n" \
  "v_mfma_f32_16x16x32_f16 a[236:239], v[26:29], v[146:149], a[236:239]\n" \
  "v_mfma_f32_16x16x32_f16 a[204:207], v[62:65], v[146:149], a[204:207]\n" \
  "ds_read_b128 v[146:149], v245 offset:11264\n" \
  "v_mfma_f32_16x16x32_f16 a[204:207], v[66:69], v[114:117], a[204:207]\n" \
  "s_waitcnt lgkmcnt(13)\n" \
  "ds_read_b128 v[114:117], v245 offset:3072\n" \
  "v_mfma_f32_16x16x32_f16 a[208:211], v[66:69], v[118:121], a[208:211]\n" \
  "v_mfma_f32_16x16x32_f16 a[208:211], v[62:65], v[150:153], a[208:211]\n" \
  "buffer_store_dwordx4 a[76:79], %[voffO], s[44:47], s84 offen offset:0 sc1\n" \
  "s_nop 7\n" \
  "v_mfma_f32_16x16x32_f16 a[240:243], v[26:29], v[150:153], a[240:243]\n" \
  "ds_read_b128 v[150:153], v245 offset:12288\n" \
  "v_mfma_f32_16x16x32_f16 a[240:243], v[22:25], v[118:121], a[240:243]\n" \
  "s_waitcnt lgkmcnt(13)\n" \
  "ds_read_b128 v[118:121], v245 offset:4096\n" \
  "v_mfma_f32_16x16x32_f16 a[244:247], v[22:25], v[122:125], a[244:247]\n" \
  "v_mfma_f32_16x16x32_f16 a[244:247], v[26:29], v[154:157], a[244:247]\n" \
  "v_mfma_f32_16x16x32_f16 a[212:215], v[62:65], v[154:157], a[212:215]\n" \
  "ds_read_b128 v[154:157], v245 offset:13312\n" \
  "v_mfma_f32_16x16x32_f16 a[212:215], v[66:69], v[122:125], a[212:215]\n" \
  "s_waitcnt lgkmcnt(13)\n" \
  "ds_read_b128 v[122:125], v245 offset:5120\n" \
  "v_mfma_f32_16x16x32_f16 a[216:219], v[66:69], v[126:129], a[216:219]\n" \
  "v_mfma_f32_16x16x32_f16 a[216:219], v[62:65], v[158:161], a[216:219]\n" \
  "v_mfma_f32_16x16x32_f16 a[248:251], v[26:29], v[158:161], a[248:251]\n" \
  "ds_read_b128 v[158:161], v245 offset:14336\n" \
  "v_mfma_f32_16x16x32_f16 a[248:251], v[22:25], v[126:129], a[248:251]\n" \
  "s_waitcnt lgkmcnt(13)\n" \
  "ds_read_b128 v[126:129], v245 offset:6144\n" \
  "v_mfma_f32_16x16x32_f16 a[252:255], v[22:25], v[130:133], a[252:255]\n" \
  "buffer_load_dwordx4 v[22:25], %[bxr], s[40:43], s72 offen offset:2048 sc0 sc1\n" \
  "v_mfma_f32_16x16x32_f16 a[252:255], v[26:29], v[162:165], a[252:255]\n" \
  "v_readlane_b32 s74, %[vtabA], 24\n" \
  "v_mfma_f32_16x16x32_f16 a[220:223], v[62:65], v[162:165], a[220:223]\n" \
  "ds_read_b128 v[162:165], v245 offset:15360\n" \
  "v_mfma_f32_16x16x32_f16 a[220:223], v[66:69], v[130:133], a[220:223]\n" \
  "s_waitcnt vmcnt(20)\n" \
  "ds_read_b128 v[130:133], v245 offset:7168\n" \
  "s_waitcnt lgkmcnt(14)\n" \
  "v_mfma_f32_16x16x32_f16 a[192:195], v[82:85], v[102:105], a[192:195]\n" \
  "buffer_load_dwordx4 v[26:29], %[bxr], s[40:43], s72 offen offset:3072 sc0 sc1\n" \
  "v_mfma_f32_16x16x32_f16 a[192:195], v[78:81], v[134:137], a[192:195]\n" \
  "s_add_u32 s73, s74, s70\n" \
  "s_nop 7\n" \
  "v_mfma_f32_16x16x32_f16 a[224:227], v[42:45], v[134:137], a[224:227]\n" \
  "ds_read_b128 v[134:137], v245 offset:24576\n" \
  "v_mfma_f32_16x16x32_f16 a[224:227], v[38:41], v[102:105], a[224:227]\n" \
  "s_waitcnt lgkmcnt(13)\n" \
  "ds_read_b128 v[102:105], v245 offset:16384\n" \
  "v_mfma_f32_16x16x32_f16 a[228:231], v[38:41], v[106:109], a[228:231]\n" \
  "buffer_load_dwordx4 v[62:65], %[bxr], s[40:43], s73 offen offset:0 sc0 sc1\n" \
  "v_mfma_f32_16x16x32_f16 a[228:231], v[42:45], v[138:141], a[228:231]\n" \
  "v_mfma_f32_16x16x32_f16 a[196:199], v[78:81], v[138:141], a[196:199]\n" \
  "ds_read_b128 v[138:141], v245 offset:25600\n" \
  "v_mfma_f32_16x16x32_f16 a[196:199], v[82:85], v[106:109], a[196:199]\n" \
  "s_waitcnt lgkmcnt(13)\n" \
  "ds_read_b128 v[106:109], v245 offset:17408\n" \
  "v_mfma_f32_16x16x32_f16 a[200:203], v[82:85], v[110:113], a[200:203]\n" \
  "buffer_load_dwordx4 v[66:69], %[bxr], s[40:43], s73 offen offset:1024 sc0 sc1\n" \
  "v_mfma_f32_16x16x32_f16 a[200:203], v[78:81], v[142:145], a[200:203]\n" \
  "buffer_store_dwordx4 a[92:95], %[voffO], s[44:47], s84 offen offset:256 sc1\n" \
  "v_mfma_f32_16x16x32_f16 a[232:235], v[42:45], v[142:145], a[232:235]\n" \
  "ds_read_b128 v[142:145], v245 offset:26624\n" \
  "v_mfma_f32_16x16x32_f16 a[232:235], v[38:41], v[110:113], a[232:235]\n" \
  "s_waitcnt lgkmcnt(13)\n" \
  "ds_read_b128 v[110:113], v245 offset:18432\n" \
  "v_mfma_f32_16x16x32_f16 a[236:239], v[38:41], v[114:117], a[236:239]\n" \
  "v_mfma_f32_16x16x32_f16 a[236:239], v[42:45], v[146:149], a[236:239]\n" \
  "v_mfma_f32_16x16x32_f16 a[204:207], v[78:81], v[146:149], a[204:207]\n" \
  "ds_read_b128 v[146:149], v245 offset:27648\n" \
  "v_mfma_f32_16x16x32_f16 a[204:207], v[82:85], v[114:117], a[204:207]\n" \
  "s_waitcnt lgkmcnt(13)\n" \
  "ds_read_b128 v[114:117], v245 offset:19456\n" \
  "v_mfma_f32_16x16x32_f16 a[208:211], v[82:85], v[118:121], a[208:211]\n" \
  "v_mfma_f32_16x16x32_f16 a[208:211], v[78:81], v[150:153], a[208:211]\n" \
  "s_nop 7\n" \
  "v_mfma_f32_16x16x32_f16 a[240:243], v[42:45], v[150:153], a[240:243]\n" \
  "ds_read_b128 v[150:153], v245 offset:28672\n" \
  "v_mfma_f32_16x16x32_f16 a[240:243], v[38:41], v[118:121], a[240:243]\n" \
  "s_waitcnt lgkmcnt(13)\n" \
  "ds_read_b128 v[118:121], v245 offset:20480\n" \
  "v_mfma_f32_16x16x32_f16 a[244:247], v[38:41], v[122:125], a[244:247]\n" \
  "v_mfma_f32_16x16x32_f16 a[244:247], v[42:45], v[154:157], a[244:247]\n" \
  "v_mfma_f32_16x16x32_f16 a[212:215], v[78:81], v[154:157], a[212:215]\n" \
  "ds_read_b128 v[154:157], v245 offset:29696\n" \
  "v_mfma_f32_16x16x32_f16 a[212:215], v[82:85], v[122:125], a[212:215]\n" \
  "s_waitcnt lgkmcnt(13)\n" \
  "ds_read_b128 v[122:125], v245 offset:21504\n" \
  "v_mfma_f32_16x16x32_f16 a[216:219], v[82:85], v[126:129], a[216:219]\n" \
  "v_mfma_f32_16x16x32_f16 a[216:219], v[78:81], v[158:161], a[216:219]\n" \
  "s_add_u32 s83, s90, 0x100000\n" \
  "v_mfma_f32_16x16x32_f16 a[248:251], v[42:45], v[158:161], a[248:251]\n" \
  "ds_read_b128 v[158:161], v245 offset:30720\n" \
  "v_mfma_f32_16x16x32_f16 a[248:251], v[38:41], v[126:129], a[248:251]\n" \
  "s_waitcnt lgkmcnt(13)\n" \
  "ds_read_b128 v[126:129], v245 offset:22528\n" \
  "v_mfma_f32_16x16x32_f16 a[252:255], v[38:41], v[130:133], a[252:255]\n" \
  "buffer_load_dwordx4 v[38:41], %[bxr], s[40:43], s73 offen offset:2048 sc0 sc1\n" \
  "v_mfma_f32_16x16x32_f16 a[252:255], v[42:45], v[162:165], a[252:255]\n" \
  "v_readlane_b32 s74, %[vtabA], 25\n" \
  "v_mfma_f32_16x16x32_f16 a[220:223], v[78:81], v[162:165], a[220:223]\n" \
  "ds_read_b128 v[162:165], v245 offset:31744\n" \
  "v_mfma_f32_16x16x32_f16 a[220:223], v[82:85], v[130:133], a[220:223]\n" \
  "s_waitcnt vmcnt(20)\n" \
  "ds_read_b128 v[130:133], v245 offset:23552\n" \
  "s_waitcnt lgkmcnt(14)\n" \
  "v_mfma_f32_16x16x32_f16 a[192:195], v[98:101], v[102:105], a[192:195]\n" \
  "buffer_load_dwordx4 v[42:45], %[bxr], s[40:43], s73 offen offset:3072 sc0 sc1\n" \
  "v_mfma_f32_16x16x32_f16 a[192:195], v[94:97], v[134:137], a[192:195]\n" \
  "buffer_store_dwordx4 a[96:99], %[voffO], s[44:47], s83 offen offset:0 sc1\n" \
  "s_nop 7\n" \
  "v_mfma_f32_16x16x32_f16 a[224:227], v[58:61], v[134:137], a[224:227]\n" \
  "s_add_u32 s72, s74, s70\n" \
  "ds_read_b128 v[134:137], v245 offset:40960\n" \
  "v_mfma_f32_16x16x32_f16 a[224:227], v[54:57], v[102:105], a[224:227]\n" \
  "s_waitcnt lgkmcnt(13)\n" \
  "ds_read_b128 v[102:105], v245 offset:32768\n" \
  "v_mfma_f32_16x16x32_f16 a[228:231], v[54:57], v[106:109], a[228:231]\n" \
  "buffer_load_dwordx4 v[78:81], %[bxr], s[40:43], s72 offen offset:0 sc0 sc1\n" \
  "v_mfma_f32_16x16x32_f16 a[228:231], v[58:61], v[138:141], a[228:231]\n" \
  "v_mfma_f32_16x16x32_f16 a[196:199], v[94:97], v[138:141], a[196:199]\n" \
  "ds_read_b128 v[138:141], v245 offset:41984\n" \
  "v_mfma_f32_16x16x32_f16 a[196:199], v[98:101], v[106:109], a[196:199]\n" \
  "s_waitcnt lgkmcnt(13)\n" \
  "ds_read_b128 v[106:109], v245 offset:33792\n" \
  "v_mfma_f32_16x16x32_f16 a[200:203], v[98:101], v[110:113], a[200:203]\n" \
  "buffer_load_dwordx4 v[82:85], %[bxr], s[40:43], s72 offen offset:1024 sc0 sc1\n" \
  "v_mfma_f32_16x16x32_f16 a[200:203], v[94:97], v[142:145], a[200:203]\n" \
  "v_mfma_f32_16x16x32_f16 a[232:235], v[58:61], v[142:145], a[232:235]\n" \
  "ds_read_b128 v[142:145], v245 offset:43008\n" \
  "v_mfma_f32_16x16x32_f16 a[232:235], v[54:57], v[110:113], a[232:235]\n" \
  "s_waitcnt lgkmcnt(13)\n" \
  "ds_read_b128 v[110:113], v245 offset:34816\n" \
  "v_mfma_f32_16x16x32_f16 a[236:239], v[54:57], v[114:117], a[236:239]\n" \
  "v_mfma_f32_16x16x32_f16 a[236:239], v[58:61], v[146:149], a[236:239]\n" \
  "v_mfma_f32_16x16x32_f16 a[204:207], v[94:97], v[146:149], a[204:207]\n" \
  "ds_read_b128 v[146:149], v245 offset:44032\n" \
  "v_mfma_f32_16x16x32_f16 a[204:207], v[98:101], v[114:117], a[204:207]\n" \
  "s_waitcnt lgkmcnt(13)\n" \
  "ds_read_b128 v[114:117], v245 offset:35840\n" \
  "v_mfma_f32_16x16x32_f16 a[208:211], v[98:101], v[118:121], a[208:211]\n" \
  "v_mfma_f32_16x16x32_f16 a[208:211], v[94:97], v[150:153], a[208:211]\n" \
  "s_nop 7\n" \
  "v_mfma_f32_16x16x32_f16 a[240:243], v[58:61], v[150:153], a[240:243]\n" \
  "ds_read_b128 v[150:153], v245 offset:45056\n" \
  "v_mfma_f32_16x16x32_f16 a[240:243], v[54:57], v[118:121], a[240:243]\n" \
  "s_waitcnt lgkmcnt(13)\n" \
  "ds_read_b128 v[118:121], v245 offset:36864\n" \
  "v_mfma_f32_16x16x32_f16 a[244:247], v[54:57], v[122:125], a[244:247]\n" \
  "v_mfma_f32_16x16x32_f16 a[244:247], v[58:61], v[154:157], a[244:247]\n" \
  "v_mfma_f32_16x16x32_f16 a[212:215], v[94:97], v[154:157], a[212:215]\n" \
  "ds_read_b128 v[154:157], v245 offset:46080\n" \
  "v_mfma_f32_16x16x32_f16 a[212:215], v[98:101], v[122:125], a[212:215]\n" \
  "s_waitcnt lgkmcnt(13)\n" \
  "ds_read_b128 v[122:125], v245 offset:37888\n" \
  "v_mfma_f32_16x16x32_f16 a[216:219], v[98:101], v[126:129], a[216:219]\n" \
  "v_mfma_f32_16x16x32_f16 a[216:219], v[94:97], v[158:161], a[216:219]\n" \
  "buffer_store_dwordx4 a[112:115], %[voffO], s[44:47], s83 offen offset:256 sc1\n" \
  "v_mfma_f32_16x16x32_f16 a[248:251], v[58:61], v[158:161], a[248:251]\n" \
  "ds_read_b128 v[158:161], v245 offset:47104\n" \
  "v_mfma_f32_16x16x32_f16 a[248:251], v[54:57], v[126:129], a[248:251]\n" \
  "s_waitcnt lgkmcnt(13)\n" \
  "ds_read_b128 v[126:129], v245 offset:38912\n" \
  "v_mfma_f32_16x16x32_f16 a[252:255], v[54:57], v[130:133], a[252:255]\n" \
  "buffer_load_dwordx4 v[54:57], %[bxr], s[40:43], s72 offen offset:2048 sc0 sc1\n" \
  "v_mfma_f32_16x16x32_f16 a[252:255], v[58:61], v[162:165], a[252:255]\n" \
  "v_readlane_b32 s74, %[vtabA], 26\n" \
  "v_mfma_f32_16x16x32_f16 a[220:223], v[94:97], v[162:165], a[220:223]\n" \
  "ds_read_b128 v[162:165], v245 offset:48128\n" \
  "v_mfma_f32_16x16x32_f16 a[220:223], v[98:101], v[130:133], a[220:223]\n" \
  "s_waitcnt vmcnt(21)\n" \
  "ds_read_b128 v[130:133], v245 offset:39936\n" \
  "s_waitcnt lgkmcnt(14)\n" \
  "v_mfma_f32_16x16x32_f16 a[192:195], v[14:17], v[102:105], a[192:195]\n" \
  "buffer_load_dwordx4 v[58:61], %[bxr], s[40:43], s72 offen offset:3072 sc0 sc1\n" \
  "v_mfma_f32_16x16x32_f16 a[192:195], v[18:21], v[134:137], a[192:195]\n" \
  "s_add_u32 s73, s74, s70\n" \
  "s_nop 7\n" \
  "v_mfma_f32_16x16x32_f16 a[224:227], v[74:77], v[134:137], a[224:227]\n" \
  "ds_read_b128 v[134:137], v245 offset:57344\n" \
  "v_mfma_f32_16x16x32_f16 a[224:227], v[70:73], v[102:105], a[224:227]\n" \
  "s_waitcnt lgkmcnt(13)\n" \
  "ds_read_b128 v[102:105], v245 offset:49152\n" \
  "v_mfma_f32_16x16x32_f16 a[228:231], v[70:73], v[106:109], a[228:231]\n" \
  "buffer_load_dwordx4 v[94:97], %[bxr], s[40:43], s73 offen offset:0 sc0 sc1\n" \
  "v_mfma_f32_16x16x32_f16 a[228:231], v[74:77], v[138:141], a[228:231]\n" \
  "v_mfma_f32_16x16x32_f16 a[196:199], v[18:21], v[138:141], a[196:199]\n" \
  "ds_read_b128 v[138:141], v245 offset:58368\n" \
  "v_mfma_f32_16x16x32_f16 a[196:199], v[14:17], v[106:109], a[196:199]\n" \
  "s_waitcnt lgkmcnt(13)\n" \
  "ds_read_b128 v[106:109], v245 offset:50176\n" \
  "v_mfma_f32_16x16x32_f16 a[200:203], v[14:17], v[110:113], a[200:203]\n" \
  "buffer_load_dwordx4 v[98:101], %[bxr], s[40:43], s73 offen offset:1024 sc0 sc1\n" \
  "v_mfma_f32_16x16x32_f16 a[200:203], v[18:21], v[142:145], a[200:203]\n" \
  "s_add_u32 s84, s90, 0x110000\n" \
  "v_mfma_f32_16x16x32_f16 a[232:235], v[74:77], v[142:145], a[232:235]\n" \
  "ds_read_b128 v[142:145], v245 offset:59392\n" \
  "v_mfma_f32_16x16x32_f16 a[232:235], v[70:73], v[110:113], a[232:235]\n" \
  "s_waitcnt lgkmcnt(13)\n" \
  "ds_read_b128 v[110:113], v245 offset:51200\n" \
  "v_mfma_f32_16x16x32_f16 a[236:239], v[70:73], v[114:117], a[236:239]\n" \
  "v_mfma_f32_16x16x32_f16 a[236:239], v[74:77], v[146:149], a[236:239]\n" \
  "v_mfma_f32_16x16x32_f16 a[204:207], v[18:21], v[146:149], a[204:207]\n" \
  "ds_read_b128 v[146:149], v245 offset:60416\n" \
  "v_mfma_f32_16x16x32_f16 a[204:207], v[14:17], v[114:117], a[204:207]\n" \
  "s_waitcnt lgkmcnt(13)\n" \
  "ds_read_b128 v[114:117], v245 offset:52224\n" \
  "v_mfma_f32_16x16x32_f16 a[208:211], v[14:17], v[118:121], a[208:211]\n" \
  "v_mfma_f32_16x16x32_f16 a[208:211], v[18:21], v[150:153], a[208:211]\n" \
  "buffer_store_dwordx4 a[100:103], %[voffO], s[44:47], s84 offen offset:0 sc1\n" \
  "s_nop 7\n" \
  "v_mfma_f32_16x16x32_f16 a[240:243], v[74:77], v[150:153], a[240:243]\n" \
  "ds_read_b128 v[150:153], v245 offset:61440\n" \
  "v_mfma_f32_16x16x32_f16 a[240:243], v[70:73], v[118:121], a[240:243]\n" \
  "s_waitcnt lgkmcnt(13)\n" \
  "ds_read_b128 v[118:121], v245 offset:53248\n" \
  "v_mfma_f32_16x16x32_f16 a[244:247], v[70:73], v[122:125], a[244:247]\n" \
  "v_mfma_f32_16x16x32_f16 a[244:247], v[74:77], v[154:157], a[244:247]\n" \
  "v_mfma_f32_16x16x32_f16 a[212:215], v[18:21], v[154:157], a[212:215]\n" \
  "ds_read_b128 v[154:157], v245 offset:62464\n" \
  "v_mfma_f32_16x16x32_f16 a[212:215], v[14:17], v[122:125], a[212:215]\n" \
  "s_waitcnt lgkmcnt(13)\n" \
  "ds_read_b128 v[122:125], v245 offset:54272\n" \
  "v_mfma_f32_16x16x32_f16 a[216:219], v[14:17], v[126:129], a[216:219]\n" \
  "v_mfma_f32_16x16x32_f16 a[216:219], v[18:21], v[158:161], a[216:219]\n" \
  "v_mfma_f32_16x16x32_f16 a[248:251], v[74:77], v[158:161], a[248:251]\n" \
  "ds_read_b128 v[158:161], v245 offset:63488\n" \
  "v_mfma_f32_16x16x32_f16 a[248:251], v[70:73], v[126:129], a[248:251]\n" \
  "s_waitcnt lgkmcnt(13)\n" \
  "ds_read_b128 v[126:129], v245 offset:55296\n" \
  "v_mfma_f32_16x16x32_f16 a[252:255], v[70:73], v[130:133], a[252:255]\n" \
  "buffer_load_dwordx4 v[70:73], %[bxr], s[40:43], s73 offen offset:2048 sc0 sc1\n" \
  "v_mfma_f32_16x16x32_f16 a[252:255], v[74:77], v[162:165], a[252:255]\n" \
  "v_readlane_b32 s74, %[vtabA], 27\n" \
  "v_mfma_f32_16x16x32_f16 a[220:223], v[18:21], v[162:165], a[220:223]\n" \
  "ds_read_b128 v[162:165], v245 offset:64512\n" \
  "v_mfma_f32_16x16x32_f16 a[220:223], v[14:17], v[130:133], a[220:223]\n" \
  "s_waitcnt vmcnt(20)\n" \
  "ds_read_b128 v[130:133], v245 offset:56320\n" \
  "s_waitcnt lgkmcnt(14)\n" \
  "v_mfma_f32_16x16x32_f16 a[192:195], v[30:33], v[102:105], a[192:195]\n" \
  "buffer_load_dwordx4 v[74:77], %[bxr], s[40:43], s73 offen offset:3072 sc0 sc1\n" \
  "v_mfma_f32_16x16x32_f16 a[192:195], v[34:37], v[134:137], a[192:195]\n" \
  "s_add_u32 s72, s74, s70\n" \
  "s_nop 7\n" \
  "v_mfma_f32_16x16x32_f16 a[224:227], v[90:93], v[134:137], a[224:227]\n" \
  "ds_read_b128 v[134:137], v246 offset:24576\n" \
  "v_mfma_f32_16x16x32_f16 a[224:227], v[86:89], v[102:105], a[224:227]\n" \
  "s_waitcnt lgkmcnt(13)\n" \
  "ds_read_b128 v[102:105], v246 offset:16384\n" \
  "v_mfma_f32_16x16x32_f16 a[228:231], v[86:89], v[106:109], a[228:231]\n" \
  "buffer_load_dwordx4 v[18:21], %[bxr], s[40:43], s72 offen offset:0 sc0 sc1\n" \
  "v_mfma_f32_16x16x32_f16 a[228:231], v[90:93], v[138:141], a[228:231]\n" \
  "v_mfma_f32_16x16x32_f16 a[196:199], v[34:37], v[138:141], a[196:199]\n" \
  "ds_read_b128 v[138:141], v246 offset:25600\n" \
  "v_mfma_f32_16x16x32_f16 a[196:199], v[30:33], v[106:109], a[196:199]\n" \
  "s_waitcnt lgkmcnt(13)\n" \
  "ds_read_b128 v[106:109], v246 offset:17408\n" \
  "v_mfma_f32_16x16x32_f16 a[200:203], v[30:33], v[110:113], a[200:203]\n" \
  "buffer_load_dwordx4 v[14:17], %[bxr], s[40:43], s72 offen offset:1024 sc0 sc1\n" \
  "v_mfma_f32_16x16x32_f16 a[200:203], v[34:37], v[142:145], a[200:203]\n" \
  "buffer_store_dwordx4 a[116:119], %[voffO], s[44:47], s84 offen offset:256 sc1\n" \
  "v_mfma_f32_16x16x32_f16 a[232:235], v[90:93], v[142:145], a[232:235]\n" \
  "ds_read_b128 v[142:145], v246 offset:26624\n" \
  "v_mfma_f32_16x16x32_f16 a[232:235], v[86:89], v[110:113], a[232:235]\n" \
  "s_waitcnt lgkmcnt(13)\n" \
  "ds_read_b128 v[110:113], v246 offset:18432\n" \
  "v_mfma_f32_16x16x32_f16 a[236:239], v[86:89], v[114:117], a[236:239]\n" \
  "v_mfma_f32_16x16x32_f16 a[236:239], v[90:93], v[146:149], a[236:239]\n" \
  "s_waitcnt vmcnt(16)\n" \
  "v_mov_b32 v250, s76\n" \
  "s_nop 0\n" \
  "global_store_dword v249, v250, %[xfo] offset:0 sc1\n" \
  "v_mfma_f32_16x16x32_f16 a[204:207], v[34:37], v[146:149], a[204:207]\n" \
  "ds_read_b128 v[146:149], v246 offset:27648\n" \
  "v_mfma_f32_16x16x32_f16 a[204:207], v[30:33], v[114:117], a[204:207]\n" \
  "s_waitcnt lgkmcnt(13)\n" \
  "ds_read_b128 v[114:117], v246 offset:19456\n" \
  "v_mfma_f32_16x16x32_f16 a[208:211], v[30:33], v[118:121], a[208:211]\n" \
  "v_mfma_f32_16x16x32_f16 a[208:211], v[34:37], v[150:153], a[208:211]\n" \
  "s_nop 7\n" \
  "v_mfma_f32_16x16x32_f16 a[240:243], v[90:93], v[150:153], a[240:243]\n" \
  "ds_read_b128 v[150:153], v246 offset:28672\n" \
  "v_mfma_f32_16x16x32_f16 a[240:243], v[86:89], v[118:121], a[240:243]\n" \
  "s_waitcnt lgkmcnt(13)\n" \
  "ds_read_b128 v[118:121], v246 offset:20480\n" \
  "v_mfma_f32_16x16x32_f16 a[244:247], v[86:89], v[122:125], a[244:247]\n" \
  "v_mfma_f32_16x16x32_f16 a[244:247], v[90:93], v[154:157], a[244:247]\n" \
  "v_mfma_f32_16x16x32_f16 a[212:215], v[34:37], v[154:157], a[212:215]\n" \
  "ds_read_b128 v[154:157], v246 offset:29696\n" \
  "v_mfma_f32_16x16x32_f16 a[212:215], v[30:33], v[122:125], a[212:215]\n" \
  "s_waitcnt lgkmcnt(13)\n" \
  "ds_read_b128 v[122:125], v246 offset:21504\n" \
  "v_mfma_f32_16x16x32_f16 a[216:219], v[30:33], v[126:129], a[216:219]\n" \
  "v_mfma_f32_16x16x32_f16 a[216:219], v[34:37], v[158:161], a[216:219]\n" \
  "s_add_u32 s83, s90, 0x120000\n" \
  "v_mfma_f32_16x16x32_f16 a[248:251], v[90:93], v[158:161], a[248:251]\n" \
  "ds_read_b128 v[158:161], v246 offset:30720\n" \
  "v_mfma_f32_16x16x32_f16 a[248:251], v[86:89], v[126:129], a[248:251]\n" \
  "s_waitcnt lgkmcnt(13)\n" \
  "ds_read_b128 v[126:129], v246 offset:22528\n" \
  "v_mfma_f32_16x16x32_f16 a[252:255], v[86:89], v[130:133], a[252:255]\n" \
  "buffer_load_dwordx4 v[86:89], %[bxr], s[40:43], s72 offen offset:2048 sc0 sc1\n" \
  "v_mfma_f32_16x16x32_f16 a[252:255], v[90:93], v[162:165], a[252:255]\n" \
  "v_readlane_b32 s74, %[vtabA], 28\n" \
  "v_mfma_f32_16x16x32_f16 a[220:223], v[34:37], v[162:165], a[220:223]\n" \
  "ds_read_b128 v[162:165], v246 offset:31744\n" \
  "v_mfma_f32_16x16x32_f16 a[220:223], v[30:33], v[130:133], a[220:223]\n" \
  "ds_read_b128 v[130:133], v246 offset:23552\n" \
  "s_waitcnt lgkmcnt(14)\n" \
  "v_mfma_f32_16x16x32_f16 a[0:3], v[46:49], v[102:105], a[0:3]\n" \
  "buffer_load_dwordx4 v[90:93], %[bxr], s[40:43], s72 offen offset:3072 sc0 sc1\n" \
  "v_mfma_f32_16x16x32_f16 a[0:3], v[50:53], v[134:137], a[0:3]\n" \
  "buffer_store_dwordx4 a[104:107], %[voffO], s[44:47], s83 offen offset:0 sc1\n" \
  "s_nop 7\n" \
  "v_mfma_f32_16x16x32_f16 a[32:35], v[26:29], v[134:137], a[32:35]\n" \
  "s_add_u32 s73, s74, s70\n" \
  "ds_read_b128 v[134:137], v246 offset:40960\n" \
  "v_mfma_f32_16x16x32_f16 a[32:35], v[22:25], v[102:105], a[32:35]\n" \
  "s_waitcnt lgkmcnt(13)\n" \
  "ds_read_b128 v[102:105], v246 offset:32768\n" \
  "v_mfma_f32_16x16x32_f16 a[36:39], v[22:25], v[106:109], a[36:39]\n" \
  "buffer_load_dwordx4 v[34:37], %[bxr], s[40:43], s73 offen offset:0 sc0 sc1\n" \
  "v_mfma_f32_16x16x32_f16 a[36:39], v[26:29], v[138:141], a[36:39]\n" \
  "v_mfma_f32_16x16x32_f16 a[4:7], v[50:53], v[138:141], a[4:7]\n" \
  "ds_read_b128 v[138:141], v246 offset:41984\n" \
  "v_mfma_f32_16x16x32_f16 a[4:7], v[46:49], v[106:109], a[4:7]\n" \
  "s_waitcnt lgkmcnt(13)\n" \
  "ds_read_b128 v[106:109], v246 offset:33792\n" \
  "v_mfma_f32_16x16x32_f16 a[8:11], v[46:49], v[110:113], a[8:11]\n" \
  "buffer_load_dwordx4 v[30:33], %[bxr], s[40:43], s73 offen offset:1024 sc0 sc1\n" \
  "v_mfma_f32_16x16x32_f16 a[8:11], v[50:53], v[142:145], a[8:11]\n" \
  "v_mfma_f32_16x16x32_f16 a[40:43], v[26:29], v[142:145], a[40:43]\n" \
  "ds_read_b128 v[142:145], v246 offset:43008\n" \
  "v_mfma_f32_16x16x32_f16 a[40:43], v[22:25], v[110:113], a[40:43]\n" \
  "s_waitcnt lgkmcnt(13)\n" \
  "ds_read_b128 v[110:113], v246 offset:34816\n" \
  "v_mfma_f32_16x16x32_f16 a[44:47], v[22:25], v[114:117], a[44:47]\n" \
  "v_mfma_f32_16x16x32_f16 a[44:47], v[26:29], v[146:149], a[44:47]\n" \
  "v_mfma_f32_16x16x32_f16 a[12:15], v[50:53], v[146:149], a[12:15]\n" \
  "ds_read_b128 v[146:149], v246 offset:44032\n" \
  "v_mfma_f32_16x16x32_f16 a[12:15], v[46:49], v[114:117], a[12:15]\n" \
  "s_waitcnt lgkmcnt(13)\n" \
  "ds_read_b128 v[114:117], v246 offset:35840\n" \
  "v_mfma_f32_16x16x32_f16 a[16:19], v[46:49], v[118:121], a[16:19]\n" \
  "v_mfma_f32_16x16x32_f16 a[16:19], v[50:53], v[150:153], a[16:19]\n" \
  "s_nop 7\n" \
  "v_mfma_f32_16x16x32_f16 a[48:51], v[26:29], v[150:153], a[48:51]\n" \
  "ds_read_b128 v[150:153], v246 offset:45056\n" \
  "v_mfma_f32_16x16x32_f16 a[48:51], v[22:25], v[118:121], a[48:51]\n" \
  "s_waitcnt lgkmcnt(13)\n" \
  "ds_read_b128 v[118:121], v246 offset:36864\n" \
  "v_mfma_f32_16x16x32_f16 a[52:55], v[22:25], v[122:125], a[52:55]\n" \
  "v_mfma_f32_16x16x32_f16 a[52:55], v[26:29], v[154:157], a[52:55]\n" \
  "v_mfma_f32_16x16x32_f16 a[20:23], v[50:53], v[154:157], a[20:23]\n" \
  "ds_read_b128 v[154:157], v246 offset:46080\n" \
  "v_mfma_f32_16x16x32_f16 a[20:23], v[46:49], v[122:125], a[20:23]\n" \
  "s_waitcnt lgkmcnt(13)\n" \
  "ds_read_b128 v[122:125], v246 offset:37888\n" \
  "v_mfma_f32_16x16x32_f16 a[24:27], v[46:49], v[126:129], a[24:27]\n" \
  "v_mfma_f32_16x16x32_f16 a[24:27], v[50:53], v[158:161], a[24:27]\n" \
  "buffer_store_dwordx4 a[120:123], %[voffO], s[44:47], s83 offen offset:256 sc1\n" \
  "v_mfma_f32_16x16x32_f16 a[56:59], v[26:29], v[158:161], a[56:59]\n" \
  "ds_read_b128 v[158:161], v246 offset:47104\n" \
  "v_mfma_f32_16x16x32_f16 a[56:59], v[22:25], v[126:129], a[56:59]\n" \
  "s_waitcnt lgkmcnt(13)\n" \
  "ds_read_b128 v[126:129], v246 offset:38912\n" \
  "v_mfma_f32_16x16x32_f16 a[60:63], v[22:25], v[130:133], a[60:63]\n" \
  "buffer_load_dwordx4 v[22:25], %[bxr], s[40:43], s73 offen offset:2048 sc0 sc1\n" \
  "v_mfma_f32_16x16x32_f16 a[60:63], v[26:29], v[162:165], a[60:63]\n" \
  "v_readlane_b32 s74, %[vtabA], 29\n" \
  "v_mfma_f32_16x16x32_f16 a[28:31], v[50:53], v[162:165], a[28:31]\n" \
  "ds_read_b128 v[162:165], v246 offset:48128\n" \
  "v_mfma_f32_16x16x32_f16 a[28:31], v[46:49], v[130:133], a[28:31]\n" \
  "s_waitcnt vmcnt(21)\n" \
  "ds_read_b128 v[130:133], v246 offset:39936\n" \
  "s_waitcnt lgkmcnt(14)\n" \
  "v_mfma_f32_16x16x32_f16 a[0:3], v[62:65], v[102:105], a[0:3]\n" \
  "buffer_load_dwordx4 v[26:29], %[bxr], s[40:43], s73 offen offset:3072 sc0 sc1\n" \
  "v_mfma_f32_16x16x32_f16 a[0:3], v[66:69], v[134:137], a[0:3]\n" \
  "s_add_u32 s72, s74, s70\n" \
  "s_nop 7\n" \
  "v_mfma_f32_16x16x32_f16 a[32:35], v[42:45], v[134:137], a[32:35]\n" \
  "ds_read_b128 v[134:137], v246 offset:57344\n" \
  "v_mfma_f32_16x16x32_f16 a[32:35], v[38:41], v[102:105], a[32:35]\n" \
  "s_waitcnt lgkmcnt(13)\n" \
  "ds_read_b128 v[102:105], v246 offset:49152\n" \
  "v_mfma_f32_16x16x32_f16 a[36:39], v[38:41], v[106:109], a[36:39]\n" \
  "buffer_load_dwordx4 v[50:53], %[bxr], s[40:43], s72 offen offset:0 sc0 sc1\n" \
  "v_mfma_f32_16x16x32_f16 a[36:39], v[42:45], v[138:141], a[36:39]\n" \
  "v_mfma_f32_16x16x32_f16 a[4:7], v[66:69], v[138:141], a[4:7]\n" \
  "ds_read_b128 v[138:141], v246 offset:58368\n" \
  "v_mfma_f32_16x16x32_f16 a[4:7], v[62:65], v[106:109], a[4:7]\n" \
  "s_waitcnt lgkmcnt(13)\n" \
  "ds_read_b128 v[106:109], v246 offset:50176\n" \
  "v_mfma_f32_16x16x32_f16 a[8:11], v[62:65], v[110:113], a[8:11]\n" \
  "buffer_load_dwordx4 v[46:49], %[bxr], s[40:43], s72 offen offset:1024 sc0 sc1\n" \
  "v_mfma_f32_16x16x32_f16 a[8:11], v[66:69], v[142:145], a[8:11]\n" \
  "s_add_u32 s84, s90, 0x130000\n" \
  "v_mfma_f32_16x16x32_f16 a[40:43], v[42:45], v[142:145], a[40:43]\n" \
  "ds_read_b128 v[142:145], v246 offset:59392\n" \
  "v_mfma_f32_16x16x32_f16 a[40:43], v[38:41], v[110:113], a[40:43]\n" \
  "s_waitcnt lgkmcnt(13)\n" \
  "ds_read_b128 v[110:113], v246 offset:51200\n" \
  "v_mfma_f32_16x16x32_f16 a[44:47], v[38:41], v[114:117], a[44:47]\n" \
  "v_mfma_f32_16x16x32_f16 a[44:47], v[42:45], v[146:149], a[44:47]\n" \
  "v_mfma_f32_16x16x32_f16 a[12:15], v[66:69], v[146:149], a[12:15]\n" \
  "ds_read_b128 v[146:149], v246 offset:60416\n" \
  "v_mfma_f32_16x16x32_f16 a[12:15], v[62:65], v[114:117], a[12:15]\n" \
  "s_waitcnt lgkmcnt(13)\n" \
  "ds_read_b128 v[114:117], v246 offset:52224\n" \
  "v_mfma_f32_16x16x32_f16 a[16:19], v[62:65], v[118:121], a[16:19]\n" \
  "v_mfma_f32_16x16x32_f16 a[16:19], v[66:69], v[150:153], a[16:19]\n" \
  "buffer_store_dwordx4 a[108:111], %[voffO], s[44:47], s84 offen offset:0 sc1\n" \
  "s_nop 7\n" \
  "v_mfma_f32_16x16x32_f16 a[48:51], v[42:45], v[150:153], a[48:51]\n" \
  "ds_read_b128 v[150:153], v246 offset:61440\n" \
  "v_mfma_f32_16x16x32_f16 a[48:51], v[38:41], v[118:121], a[48:51]\n" \
  "s_waitcnt lgkmcnt(13)\n" \
  "ds_read_b128 v[118:121], v246 offset:53248\n" \
  "v_mfma_f32_16x16x32_f16 a[52:55], v[38:41], v[122:125], a[52:55]\n" \
  "v_mfma_f32_16x16x32_f16 a[52:55], v[42:45], v[154:157], a[52:55]\n" \
  "v_mfma_f32_16x16x32_f16 a[20:23], v[66:69], v[154:157], a[20:23]\n" \
  "ds_read_b128 v[154:157], v246 offset:62464\n" \
  "v_mfma_f32_16x16x32_f16 a[20:23], v[62:65], v[122:125], a[20:23]\n" \
  "s_waitcnt lgkmcnt(13)\n" \
  "ds_read_b128 v[122:125], v246 offset:54272\n" \
  "v_mfma_f32_16x16x32_f16 a[24:27], v[62:65], v[126:129], a[24:27]\n" \
  "v_mfma_f32_16x16x32_f16 a[24:27], v[66:69], v[158:161], a[24:27]\n" \
  "v_mfma_f32_16x16x32_f16 a[56:59], v[42:45], v[158:161], a[56:59]\n" \
  "ds_read_b128 v[158:161], v246 offset:63488\n" \
  "v_mfma_f32_16x16x32_f16 a[56:59], v[38:41], v[126:129], a[56:59]\n" \
  "s_waitcnt lgkmcnt(13)\n" \
  "ds_read_b128 v[126:129], v246 offset:55296\n" \
  "v_mfma_f32_16x16x32_f16 a[60:63], v[38:41], v[130:133], a[60:63]\n" \
  "buffer_load_dwordx4 v[38:41], %[bxr], s[40:43], s72 offen offset:2048 sc0 sc1\n" \
  "v_mfma_f32_16x16x32_f16 a[60:63], v[42:45], v[162:165], a[60:63]\n" \
  "v_readlane_b32 s74, %[vtabA], 30\n" \
  "v_mfma_f32_16x16x32_f16 a[28:31], v[66:69], v[162:165], a[28:31]\n" \
  "ds_read_b128 v[162:165], v246 offset:64512\n" \
  "v_mfma_f32_16x16x32_f16 a[28:31], v[62:65], v[130:133], a[28:31]\n" \
  "s_waitcnt vmcnt(20)\n" \
  "ds_read_b128 v[130:133], v246 offset:56320\n" \
  "s_waitcnt lgkmcnt(14)\n" \
  "v_mfma_f32_16x16x32_f16 a[0:3], v[78:81], v[102:105], a[0:3]\n" \
  "buffer_load_dwordx4 v[42:45], %[bxr], s[40:43], s72 offen offset:3072 sc0 sc1\n" \
  "v_mfma_f32_16x16x32_f16 a[0:3], v[82:85], v[134:137], a[0:3]\n" \
  "s_add_u32 s73, s74, s70\n" \
  "s_nop 7\n" \
  "v_mfma_f32_16x16x32_f16 a[32:35], v[58:61], v[134:137], a[32:35]\n" \
  "ds_read_b128 v[134:137], v247 offset:8192\n" \
  "v_mfma_f32_16x16x32_f16 a[32:35], v[54:57], v[102:105], a[32:35]\n" \
  "s_waitcnt lgkmcnt(13)\n" \
  "ds_read_b128 v[102:105], v247 offset:0\n" \
  "v_mfma_f32_16x16x32_f16 a[36:39], v[54:57], v[106:109], a[36:39]\n" \
  "buffer_load_dwordx4 v[66:69], %[bxr], s[40:43], s73 offen offset:0 sc0 sc1\n" \
  "v_mfma_f32_16x16x32_f16 a[36:39], v[58:61], v[138:141], a[36:39]\n" \
  "global_load_dwordx4 v[238:241], v249, %[xfi] sc1\n" \
  "v_mfma_f32_16x16x32_f16 a[4:7], v[82:85], v[138:141], a[4:7]\n" \
  "ds_read_b128 v[138:141], v247 offset:9216\n" \
  "v_mfma_f32_16x16x32_f16 a[4:7], v[78:81], v[106:109], a[4:7]\n" \
  "s_waitcnt lgkmcnt(13)\n" \
  "ds_read_b128 v[106:109], v247 offset:1024\n" \
  "v_mfma_f32_16x16x32_f16 a[8:11], v[78:81], v[110:113], a[8:11]\n" \
  "buffer_load_dwordx4 v[62:65], %[bxr], s[40:43], s73 offen offset:1024 sc0 sc1\n" \
  "v_mfma_f32_16x16x32_f16 a[8:11], v[82:85], v[142:145], a[8:11]\n" \
  "buffer_store_dwordx4 a[124:127], %[voffO], s[44:47], s84 offen offset:256 sc1\n" \
  "v_mfma_f32_16x16x32_f16 a[40:43], v[58:61], v[142:145], a[40:43]\n" \
  "ds_read_b128 v[142:145], v247 offset:10240\n" \
  "v_mfma_f32_16x16x32_f16 a[40:43], v[54:57], v[110:113], a[40:43]\n" \
  "s_waitcnt lgkmcnt(13)\n" \
  "ds_read_b128 v[110:113], v247 offset:2048\n" \
  "v_mfma_f32_16x16x32_f16 a[44:47], v[54:57], v[114:117], a[44:47]\n" \
  "v_mfma_f32_16x16x32_f16 a[44:47], v[58:61], v[146:149], a[44:47]\n" \
  "v_mfma_f32_16x16x32_f16 a[12:15], v[82:85], v[146:149], a[12:15]\n" \
  "ds_read_b128 v[146:149], v247 offset:11264\n" \
  "v_mfma_f32_16x16x32_f16 a[12:15], v[78:81], v[114:117], a[12:15]\n" \
  "s_waitcnt lgkmcnt(13)\n" \
  "ds_read_b128 v[114:117], v247 offset:3072\n" \
  "v_mfma_f32_16x16x32_f16 a[16:19], v[78:81], v[118:121], a[16:19]\n" \
  "v_mfma_f32_16x16x32_f16 a[16:19], v[82:85], v[150:153], a[16:19]\n" \
  "s_nop 7\n" \
  "v_mfma_f32_16x16x32_f16 a[48:51], v[58:61], v[150:153], a[48:51]\n" \
  "ds_read_b128 v[150:153], v247 offset:12288\n" \
  "v_mfma_f32_16x16x32_f16 a[48:51], v[54:57], v[118:121], a[48:51]\n" \
  "s_waitcnt lgkmcnt(13)\n" \
  "ds_read_b128 v[118:121], v247 offset:4096\n" \
  "v_mfma_f32_16x16x32_f16 a[52:55], v[54:57], v[122:125], a[52:55]\n" \
  "v_mfma_f32_16x16x32_f16 a[52:55], v[58:61], v[154:157], a[52:55]\n" \
  "v_mfma_f32_16x16x32_f16 a[20:23], v[82:85], v[154:157], a[20:23]\n" \
  "ds_read_b128 v[154:157], v247 offset:13312\n" \
  "v_mfma_f32_16x16x32_f16 a[20:23], v[78:81], v[122:125], a[20:23]\n" \
  "s_waitcnt lgkmcnt(13)\n" \
  "ds_read_b128 v[122:125], v247 offset:5120\n" \
  "v_mfma_f32_16x16x32_f16 a[24:27], v[78:81], v[126:129], a[24:27]\n" \
  "v_mfma_f32_16x16x32_f16 a[24:27], v[82:85], v[158:161], a[24:27]\n" \
  "s_add_u32 s83, s90, 0x400000\n" \
  "v_mfma_f32_16x16x32_f16 a[56:59], v[58:61], v[158:161], a[56:59]\n" \
  "ds_read_b128 v[158:161], v247 offset:14336\n" \
  "v_mfma_f32_16x16x32_f16 a[56:59], v[54:57], v[126:129], a[56:59]\n" \
  "s_waitcnt lgkmcnt(13)\n" \
  "ds_read_b128 v[126:129], v247 offset:6144\n" \
  "v_mfma_f32_16x16x32_f16 a[60:63], v[54:57], v[130:133], a[60:63]\n" \
  "buffer_load_dwordx4 v[54:57], %[bxr], s[40:43], s73 offen offset:2048 sc0 sc1\n" \
  "v_mfma_f32_16x16x32_f16 a[60:63], v[58:61], v[162:165], a[60:63]\n" \
  "v_readlane_b32 s74, %[vtabA], 31\n" \
  "v_mfma_f32_16x16x32_f16 a[28:31], v[82:85], v[162:165], a[28:31]\n" \
  "ds_read_b128 v[162:165], v247 offset:15360\n" \
  "v_mfma_f32_16x16x32_f16 a[28:31], v[78:81], v[130:133], a[28:31]\n" \
  "s_waitcnt vmcnt(20)\n" \
  "ds_read_b128 v[130:133], v247 offset:7168\n" \
  "s_waitcnt lgkmcnt(14)\n" \
  "v_mfma_f32_16x16x32_f16 a[0:3], v[94:97], v[102:105], a[0:3]\n" \
  "buffer_load_dwordx4 v[58:61], %[bxr], s[40:43], s73 offen offset:3072 sc0 sc1\n" \
  "v_mfma_f32_16x16x32_f16 a[0:3], v[98:101], v[134:137], a[0:3]\n" \
  "buffer_store_dwordx4 a[192:195], %[voffO], s[44:47], s83 offen offset:0 sc1\n" \
  "s_nop 7\n" \
  "v_mfma_f32_16x16x32_f16 a[32:35], v[74:77], v[134:137], a[32:35]\n" \
  "s_add_u32 s72, s74, s70\n" \
  "ds_read_b128 v[134:137], v247 offset:24576\n" \
  "v_mfma_f32_16x16x32_f16 a[32:35], v[70:73], v[102:105], a[32:35]\n" \
  "s_waitcnt lgkmcnt(13)\n" \
  "ds_read_b128 v[102:105], v247 offset:16384\n" \
  "v_mfma_f32_16x16x32_f16 a[36:39], v[70:73], v[106:109], a[36:39]\n" \
  "buffer_load_dwordx4 v[82:85], %[bxr], s[40:43], s72 offen offset:0 sc0 sc1\n" \
  "v_mfma_f32_16x16x32_f16 a[36:39], v[74:77], v[138:141], a[36:39]\n" \
  "v_mfma_f32_16x16x32_f16 a[4:7], v[98:101], v[138:141], a[4:7]\n" \
  "ds_read_b128 v[138:141], v247 offset:25600\n" \
  "v_mfma_f32_16x16x32_f16 a[4:7], v[94:97], v[106:109], a[4:7]\n" \
  "s_waitcnt lgkmcnt(13)\n" \
  "ds_read_b128 v[106:109], v247 offset:17408\n" \
  "v_mfma_f32_16x16x32_f16 a[8:11], v[94:97], v[110:113], a[8:11]\n" \
  "buffer_load_dwordx4 v[78:81], %[bxr], s[40:43], s72 offen offset:1024 sc0 sc1\n" \
  "v_mfma_f32_16x16x32_f16 a[8:11], v[98:101], v[142:145], a[8:11]\n" \
  "v_mfma_f32_16x16x32_f16 a[40:43], v[74:77], v[142:145], a[40:43]\n" \
  "ds_read_b128 v[142:145], v247 offset:26624\n" \
  "v_mfma_f32_16x16x32_f16 a[40:43], v[70:73], v[110:113], a[40:43]\n" \
  "s_waitcnt lgkmcnt(13)\n" \
  "ds_read_b128 v[110:113], v247 offset:18432\n" \
  "v_mfma_f32_16x16x32_f16 a[44:47], v[70:73], v[114:117], a[44:47]\n" \
  "v_mfma_f32_16x16x32_f16 a[44:47], v[74:77], v[146:149], a[44:47]\n" \
  "v_mfma_f32_16x16x32_f16 a[12:15], v[98:101], v[146:149], a[12:15]\n" \
  "ds_read_b128 v[146:149], v247 offset:27648\n" \
  "v_mfma_f32_16x16x32_f16 a[12:15], v[94:97], v[114:117], a[12:15]\n" \
  "s_waitcnt lgkmcnt(13)\n" \
  "ds_read_b128 v[114:117], v247 offset:19456\n" \
  "v_mfma_f32_16x16x32_f16 a[16:19], v[94:97], v[118:121], a[16:19]\n" \
  "v_mfma_f32_16x16x32_f16 a[16:19], v[98:101], v[150:153], a[16:19]\n" \
  "s_nop 7\n" \
  "v_mfma_f32_16x16x32_f16 a[48:51], v[74:77], v[150:153], a[48:51]\n" \
  "ds_read_b128 v[150:153], v247 offset:28672\n" \
  "v_mfma_f32_16x16x32_f16 a[48:51], v[70:73], v[118:121], a[48:51]\n" \
  "s_waitcnt lgkmcnt(13)\n" \
  "ds_read_b128 v[118:121], v247 offset:20480\n" \
  "v_mfma_f32_16x16x32_f16 a[52:55], v[70:73], v[122:125], a[52:55]\n" \
  "v_mfma_f32_16x16x32_f16 a[52:55], v[74:77], v[154:157], a[52:55]\n" \
  "v_mfma_f32_16x16x32_f16 a[20:23], v[98:101], v[154:157], a[20:23]\n" \
  "ds_read_b128 v[154:157], v247 offset:29696\n" \
  "v_mfma_f32_16x16x32_f16 a[20:23], v[94:97], v[122:125], a[20:23]\n" \
  "s_waitcnt lgkmcnt(13)\n" \
  "ds_read_b128 v[122:125], v247 offset:21504\n" \
  "v_mfma_f32_16x16x32_f16 a[24:27], v[94:97], v[126:129], a[24:27]\n" \
  "v_mfma_f32_16x16x32_f16 a[24:27], v[98:101], v[158:161], a[24:27]\n" \
  "buffer_store_dwordx4 a[208:211], %[voffO], s[44:47], s83 offen offset:256 sc1\n" \
  "v_mfma_f32_16x16x32_f16 a[56:59], v[74:77], v[158:161], a[56:59]\n" \
  "ds_read_b128 v[158:161], v247 offset:30720\n" \
  "v_mfma_f32_16x16x32_f16 a[56:59], v[70:73], v[126:129], a[56:59]\n" \
  "s_waitcnt lgkmcnt(13)\n" \
  "ds_read_b128 v[126:129], v247 offset:22528\n" \
  "v_mfma_f32_16x16x32_f16 a[60:63], v[70:73], v[130:133], a[60:63]\n" \
  "buffer_load_dwordx4 v[70:73], %[bxr], s[40:43], s72 offen offset:2048 sc0 sc1\n" \
  "v_mfma_f32_16x16x32_f16 a[60:63], v[74:77], v[162:165], a[60:63]\n" \
  "v_readlane_b32 s74, %[vtabA], 23\n" \
  "v_mfma_f32_16x16x32_f16 a[28:31], v[98:101], v[162:165], a[28:31]\n" \
  "ds_read_b128 v[162:165], v247 offset:31744\n" \
  "v_mfma_f32_16x16x32_f16 a[28:31], v[94:97], v[130:133], a[28:31]\n" \
  "s_waitcnt vmcnt(21)\n" \
  "ds_read_b128 v[130:133], v247 offset:23552\n" \
  "s_waitcnt lgkmcnt(14)\n" \
  "v_mfma_f32_16x16x32_f16 a[0:3], v[18:21], v[102:105], a[0:3]\n" \
  "buffer_load_dwordx4 v[74:77], %[bxr], s[40:43], s72 offen offset:3072 sc0 sc1\n" \
  "v_mfma_f32_16x16x32_f16 a[0:3], v[14:17], v[134:137], a[0:3]\n" \
  "s_add_u32 s73, s74, s71\n" \
  "s_nop 7\n" \
  "v_mfma_f32_16x16x32_f16 a[32:35], v[90:93], v[134:137], a[32:35]\n" \
  "ds_read_b128 v[134:137], v245 offset:8192\n" \
  "v_mfma_f32_16x16x32_f16 a[32:35], v[86:89], v[102:105], a[32:35]\n" \
  "s_waitcnt lgkmcnt(13)\n" \
  "ds_read_b128 v[102:105], v245 offset:0\n" \
  "v_mfma_f32_16x16x32_f16 a[36:39], v[86:89], v[106:109], a[36:39]\n" \
  "buffer_load_dwordx4 v[98:101], %[bxr], s[40:43], s73 offen offset:0 sc0 sc1\n" \
  "v_mfma_f32_16x16x32_f16 a[36:39], v[90:93], v[138:141], a[36:39]\n" \
  "v_mfma_f32_16x16x32_f16 a[4:7], v[14:17], v[138:141], a[4:7]\n" \
  "ds_read_b128 v[138:141], v245 offset:9216\n" \
  "v_mfma_f32_16x16x32_f16 a[4:7], v[18:21], v[106:109], a[4:7]\n" \
  "s_waitcnt lgkmcnt(13)\n" \
  "ds_read_b128 v[106:109], v245 offset:1024\n" \
  "v_mfma_f32_16x16x32_f16 a[8:11], v[18:21], v[110:113], a[8:11]\n" \
  "buffer_load_dwordx4 v[94:97], %[bxr], s[40:43], s73 offen offset:1024 sc0 sc1\n" \
  "v_mfma_f32_16x16x32_f16 a[8:11], v[14:17], v[142:145], a[8:11]\n" \
  "s_add_u32 s84, s90, 0x410000\n" \
  "v_mfma_f32_16x16x32_f16 a[40:43], v[90:93], v[142:145], a[40:43]\n" \
  "ds_read_b128 v[142:145], v245 offset:10240\n" \
  "v_mfma_f32_16x16x32_f16 a[40:43], v[86:89], v[110:113], a[40:43]\n" \
  "s_waitcnt lgkmcnt(13)\n" \
  "ds_read_b128 v[110:113], v245 offset:2048\n" \
  "v_mfma_f32_16x16x32_f16 a[44:47], v[86:89], v[114:117], a[44:47]\n" \
  "v_mfma_f32_16x16x32_f16 a[44:47], v[90:93], v[146:149], a[44:47]\n" \
  "v_mfma_f32_16x16x32_f16 a[12:15], v[14:17], v[146:149], a[12:15]\n" \
  "ds_read_b128 v[146:149], v245 offset:11264\n" \
  "v_mfma_f32_16x16x32_f16 a[12:15], v[18:21], v[114:117], a[12:15]\n" \
  "s_waitcnt lgkmcnt(13)\n" \
  "ds_read_b128 v[114:117], v245 offset:3072\n" \
  "v_mfma_f32_16x16x32_f16 a[16:19], v[18:21], v[118:121], a[16:19]\n" \
  "v_mfma_f32_16x16x32_f16 a[16:19], v[14:17], v[150:153], a[16:19]\n" \
  "buffer_store_dwordx4 a[196:199], %[voffO], s[44:47], s84 offen offset:0 sc1\n" \
  "s_nop 7\n" \
  "v_mfma_f32_16x16x32_f16 a[48:51], v[90:93], v[150:153], a[48:51]\n" \
  "ds_read_b128 v[150:153], v245 offset:12288\n" \
  "v_mfma_f32_16x16x32_f16 a[48:51], v[86:89], v[118:121], a[48:51]\n" \
  "s_waitcnt lgkmcnt(13)\n" \
  "ds_read_b128 v[118:121], v245 offset:4096\n" \
  "v_mfma_f32_16x16x32_f16 a[52:55], v[86:89], v[122:125], a[52:55]\n" \
  "v_mfma_f32_16x16x32_f16 a[52:55], v[90:93], v[154:157], a[52:55]\n" \
  "v_mfma_f32_16x16x32_f16 a[20:23], v[14:17], v[154:157], a[20:23]\n" \
  "ds_read_b128 v[154:157], v245 offset:13312\n" \
  "v_mfma_f32_16x16x32_f16 a[20:23], v[18:21], v[122:125], a[20:23]\n" \
  "s_waitcnt lgkmcnt(13)\n" \
  "ds_read_b128 v[122:125], v245 offset:5120\n" \
  "v_mfma_f32_16x16x32_f16 a[24:27], v[18:21], v[126:129], a[24:27]\n" \
  "v_mfma_f32_16x16x32_f16 a[24:27], v[14:17], v[158:161], a[24:27]\n" \
  "v_mfma_f32_16x16x32_f16 a[56:59], v[90:93], v[158:161], a[56:59]\n" \
  "ds_read_b128 v[158:161], v245 offset:14336\n" \
  "v_mfma_f32_16x16x32_f16 a[56:59], v[86:89], v[126:129], a[56:59]\n" \
  "s_waitcnt lgkmcnt(13)\n" \
  "ds_read_b128 v[126:129], v245 offset:6144\n" \
  "v_mfma_f32_16x16x32_f16 a[60:63], v[86:89], v[130:133], a[60:63]\n" \
  "buffer_load_dwordx4 v[86:89], %[bxr], s[40:43], s73 offen offset:2048 sc0 sc1\n" \
  "v_mfma_f32_16x16x32_f16 a[60:63], v[90:93], v[162:165], a[60:63]\n" \
  "v_readlane_b32 s74, %[vtabA], 24\n" \
  "v_mfma_f32_16x16x32_f16 a[28:31], v[14:17], v[162:165], a[28:31]\n" \
  "ds_read_b128 v[162:165], v245 offset:15360\n" \
  "v_mfma_f32_16x16x32_f16 a[28:31], v[18:21], v[130:133], a[28:31]\n" \
  "s_waitcnt vmcnt(20)\n" \
  "ds_read_b128 v[130:133], v245 offset:7168\n" \
  "s_waitcnt lgkmcnt(14)\n" \
  "v_mfma_f32_16x16x32_f16 a[0:3], v[34:37], v[102:105], a[0:3]\n" \
  "buffer_load_dwordx4 v[90:93], %[bxr], s[40:43], s73 offen offset:3072 sc0 sc1\n" \
  "v_mfma_f32_16x16x32_f16 a[0:3], v[30:33], v[134:137], a[0:3]\n" \
  "s_add_u32 s72, s74, s71\n" \
  "s_nop 7\n" \
  "v_mfma_f32_16x16x32_f16 a[32:35], v[26:29], v[134:137], a[32:35]\n" \
  "ds_read_b128 v[134:137], v245 offset:24576\n" \
  "v_mfma_f32_16x16x32_f16 a[32:35], v[22:25], v[102:105], a[32:35]\n" \
  "s_waitcnt lgkmcnt(13)\n" \
  "ds_read_b128 v[102:105], v245 offset:16384\n" \
  "v_mfma_f32_16x16x32_f16 a[36:39], v[22:25], v[106:109], a[36:39]\n" \
  "buffer_load_dwordx4 v[14:17], %[bxr], s[40:43], s72 offen offset:0 sc0 sc1\n" \
  "v_mfma_f32_16x16x32_f16 a[36:39], v[26:29], v[138:141], a[36:39]\n" \
  "v_mfma_f32_16x16x32_f16 a[4:7], v[30:33], v[138:141], a[4:7]\n" \
  "ds_read_b128 v[138:141], v245 offset:25600\n" \
  "v_mfma_f32_16x16x32_f16 a[4:7], v[34:37], v[106:109], a[4:7]\n" \
  "s_waitcnt lgkmcnt(13)\n" \
  "ds_read_b128 v[106:109], v245 offset:17408\n" \
  "v_mfma_f32_16x16x32_f16 a[8:11], v[34:37], v[110:113], a[8:11]\n" \
  "buffer_load_dwordx4 v[18:21], %[bxr], s[40:43], s72 offen offset:1024 sc0 sc1\n" \
  "v_mfma_f32_16x16x32_f16 a[8:11], v[30:33], v[142:145], a[8:11]\n" \
  "buffer_store_dwordx4 a[212:215], %[voffO], s[44:47], s84 offen offset:256 sc1\n" \
  "v_mfma_f32_16x16x32_f16 a[40:43], v[26:29], v[142:145], a[40:43]\n" \
  "ds_read_b128 v[142:145], v245 offset:26624\n" \
  "v_mfma_f32_16x16x32_f16 a[40:43], v[22:25], v[110:113], a[40:43]\n" \
  "s_waitcnt lgkmcnt(13)\n" \
  "ds_read_b128 v[110:113], v245 offset:18432\n" \
  "v_mfma_f32_16x16x32_f16 a[44:47], v[22:25], v[114:117], a[44:47]\n" \
  "v_mfma_f32_16x16x32_f16 a[44:47], v[26:29], v[146:149], a[44:47]\n" \
  "s_waitcnt vmcnt(16)\n" \
  "v_mov_b32 v250, s76\n" \
  "s_nop 0\n" \
  "global_store_dword v249, v250, %[xfo] offset:4 sc1\n" \
  "v_mfma_f32_16x16x32_f16 a[12:15], v[30:33], v[146:149], a[12:15]\n" \
  "ds_read_b128 v[146:149], v245 offset:27648\n" \
  "v_mfma_f32_16x16x32_f16 a[12:15], v[34:37], v[114:117], a[12:15]\n" \
  "s_waitcnt lgkmcnt(13)\n" \
  "ds_read_b128 v[114:117], v245 offset:19456\n" \
  "v_mfma_f32_16x16x32_f16 a[16:19], v[34:37], v[118:121], a[16:19]\n" \
  "s_add_u32 s60, s91, 0x0\n" \
  "v_readfirstlane_b32 s79, v238\n" \
  "s_nop 0\n" \
  "s_cmp_ge_u32 s79, s76\n" \
  "s_cbranch_scc0 L_s91_%=\n" \
  "L_b91_%=:\n" \
  "global_load_dword v251, v249, %[wsp] sc1\n" \
  "v_mfma_f32_16x16x32_f16 a[16:19], v[30:33], v[150:153], a[16:19]\n" \
  "s_nop 7\n" \
  "v_mfma_f32_16x16x32_f16 a[48:51], v[26:29], v[150:153], a[48:51]\n" \
  "ds_read_b128 v[150:153], v245 offset:28672\n" \
  "v_mfma_f32_16x16x32_f16 a[48:51], v[22:25], v[118:121], a[48:51]\n" \
  "s_waitcnt lgkmcnt(13)\n" \
  "ds_read_b128 v[118:121], v245 offset:20480\n" \
  "v_mfma_f32_16x16x32_f16 a[52:55], v[22:25], v[122:125], a[52:55]\n" \
  "buffer_load_dwordx4 a[64:67], %[voffO], s[44:47], s60 offen offset:0 sc1\n" \
  "v_mfma_f32_16x16x32_f16 a[52:55], v[26:29], v[154:157], a[52:55]\n" \
  "v_mfma_f32_16x16x32_f16 a[20:23], v[30:33], v[154:157], a[20:23]\n" \
  "ds_read_b128 v[154:157], v245 offset:29696\n" \
  "v_mfma_f32_16x16x32_f16 a[20:23], v[34:37], v[122:125], a[20:23]\n" \
  "s_waitcnt lgkmcnt(13)\n" \
  "ds_read_b128 v[122:125], v245 offset:21504\n" \
  "v_mfma_f32_16x16x32_f16 a[24:27], v[34:37], v[126:129], a[24:27]\n" \
  "buffer_load_dwordx4 a[80:83], %[voffO], s[44:47], s60 offen offset:256 sc1\n" \
  "v_mfma_f32_16x16x32_f16 a[24:27], v[30:33], v[158:161], a[24:27]\n" \
  "s_add_u32 s83, s90, 0x420000\n" \
  "s_add_u32 s61, s91, 0x10000\n" \
  "v_mfma_f32_16x16x32_f16 a[56:59], v[26:29], v[158:161], a[56:59]\n" \
  "ds_read_b128 v[158:161], v245 offset:30720\n" \
  "v_mfma_f32_16x16x32_f16 a[56:59], v[22:25], v[126:129], a[56:59]\n" \
  "s_waitcnt lgkmcnt(13)\n" \
  "ds_read_b128 v[126:129], v245 offset:22528\n" \
  "v_mfma_f32_16x16x32_f16 a[60:63], v[22:25], v[130:133], a[60:63]\n" \
  "buffer_load_dwordx4 a[68:71], %[voffO], s[44:47], s61 offen offset:0 sc1\n" \
  "v_mfma_f32_16x16x32_f16 a[60:63], v[26:29], v[162:165], a[60:63]\n" \
  "buffer_load_dwordx4 v[22:25], %[bxr], s[40:43], s72 offen offset:2048 sc0 sc1\n" \
  "v_mfma_f32_16x16x32_f16 a[28:31], v[30:33], v[162:165], a[28:31]\n" \
  "ds_read_b128 v[162:165], v245 offset:31744\n" \
  "v_mfma_f32_16x16x32_f16 a[28:31], v[34:37], v[130:133], a[28:31]\n" \
  "ds_read_b128 v[130:133], v245 offset:23552\n" \
  "s_waitcnt lgkmcnt(14)\n" \
  "v_mfma_f32_16x16x32_f16 a[0:3], v[50:53], v[102:105], a[0:3]\n" \
  "buffer_load_dwordx4 a[84:87], %[voffO], s[44:47], s61 offen offset:256 sc1\n" \
  "v_mfma_f32_16x16x32_f16 a[0:3], v[46:49], v[134:137], a[0:3]\n" \
  "buffer_store_dwordx4 a[200:203], %[voffO], s[44:47], s83 offen offset:0 sc1\n" \
  "s_nop 7\n" \
  "v_mfma_f32_16x16x32_f16 a[32:35], v[42:45], v[134:137], a[32:35]\n" \
  "s_add_u32 s60, s91, 0x20000\n" \
  "ds_read_b128 v[134:137], v245 offset:40960\n" \
  "v_readlane_b32 s74, %[vtabA], 25\n" \
  "v_mfma_f32_16x16x32_f16 a[32:35], v[38:41], v[102:105], a[32:35]\n" \
  "s_waitcnt lgkmcnt(13)\n" \
  "ds_read_b128 v[102:105], v245 offset:32768\n" \
  "v_mfma_f32_16x16x32_f16 a[36:39], v[38:41], v[106:109], a[36:39]\n" \
  "buffer_load_dwordx4 a[72:75], %[voffO], s[44:47], s60 offen offset:0 sc1\n" \
  "v_mfma_f32_16x16x32_f16 a[36:39], v[42:45], v[138:141], a[36:39]\n" \
  "buffer_load_dwordx4 v[26:29], %[bxr], s[40:43], s72 offen offset:3072 sc0 sc1\n" \
  "v_mfma_f32_16x16x32_f16 a[4:7], v[46:49], v[138:141], a[4:7]\n" \
  "s_add_u32 s73, s74, s71\n" \
  "ds_read_b128 v[138:141], v245 offset:41984\n" \
  "v_mfma_f32_16x16x32_f16 a[4:7], v[50:53], v[106:109], a[4:7]\n" \
  "s_waitcnt lgkmcnt(13)\n" \
  "ds_read_b128 v[106:109], v245 offset:33792\n" \
  "v_mfma_f32_16x16x32_f16 a[8:11], v[50:53], v[110:113], a[8:11]\n" \
  "buffer_load_dwordx4 a[88:91], %[voffO], s[44:47], s60 offen offset:256 sc1\n" \
  "v_mfma_f32_16x16x32_f16 a[8:11], v[46:49], v[142:145], a[8:11]\n" \
  "buffer_load_dwordx4 v[30:33], %[bxr], s[40:43], s73 offen offset:0 sc0 sc1\n" \
  "v_mfma_f32_16x16x32_f16 a[40:43], v[42:45], v[142:145], a[40:43]\n" \
  "s_add_u32 s61, s91, 0x30000\n" \
  "ds_read_b128 v[142:145], v245 offset:43008\n" \
  "v_mfma_f32_16x16x32_f16 a[40:43], v[38:41], v[110:113], a[40:43]\n" \
  "s_waitcnt lgkmcnt(13)\n" \
  "ds_read_b128 v[110:113], v245 offset:34816\n" \
  "v_mfma_f32_16x16x32_f16 a[44:47], v[38:41], v[114:117], a[44:47]\n" \
  "buffer_load_dwordx4 a[76:79], %[voffO], s[44:47], s61 offen offset:0 sc1\n" \
  "v_mfma_f32_16x16x32_f16 a[44:47], v[42:45], v[146:149], a[44:47]\n" \
  "buffer_load_dwordx4 v[34:37], %[bxr], s[40:43], s73 offen offset:1024 sc0 sc1\n" \
  "v_mfma_f32_16x16x32_f16 a[12:15], v[46:49], v[146:149], a[12:15]\n" \
  "ds_read_b128 v[146:149], v245 offset:44032\n" \
  "v_mfma_f32_16x16x32_f16 a[12:15], v[50:53], v[114:117], a[12:15]\n" \
  "s_waitcnt lgkmcnt(13)\n" \
  "ds_read_b128 v[114:117], v245 offset:35840\n" \
  "v_mfma_f32_16x16x32_f16 a[16:19], v[50:53], v[118:121], a[16:19]\n" \
  "buffer_load_dwordx4 a[92:95], %[voffO], s[44:47], s61 offen offset:256 sc1\n" \
  "v_mfma_f32_16x16x32_f16 a[16:19], v[46:49], v[150:153], a[16:19]\n" \
  "s_nop 7\n" \
  "v_mfma_f32_16x16x32_f16 a[48:51], v[42:45], v[150:153], a[48:51]\n" \
  "ds_read_b128 v[150:153], v245 offset:45056\n" \
  "v_mfma_f32_16x16x32_f16 a[48:51], v[38:41], v[118:121], a[48:51]\n" \
  "s_waitcnt lgkmcnt(13)\n" \
  "ds_read_b128 v[118:121], v245 offset:36864\n" \
  "v_mfma_f32_16x16x32_f16 a[52:55], v[38:41], v[122:125], a[52:55]\n" \
  "v_mfma_f32_16x16x32_f16 a[52:55], v[42:45], v[154:157], a[52:55]\n" \
  "v_mfma_f32_16x16x32_f16 a[20:23], v[46:49], v[154:157], a[20:23]\n" \
  "ds_read_b128 v[154:157], v245 offset:46080\n" \
  "v_mfma_f32_16x16x32_f16 a[20:23], v[50:53], v[122:125], a[20:23]\n" \
  "s_waitcnt lgkmcnt(13)\n" \
  "ds_read_b128 v[122:125], v245 offset:37888\n" \
  "v_mfma_f32_16x16x32_f16 a[24:27], v[50:53], v[126:129], a[24:27]\n" \
  "v_mfma_f32_16x16x32_f16 a[24:27], v[46:49], v[158:161], a[24:27]\n" \
  "buffer_store_dwordx4 a[216:219], %[voffO], s[44:47], s83 offen offset:256 sc1\n" \
  "v_mfma_f32_16x16x32_f16 a[56:59], v[42:45], v[158:161], a[56:59]\n" \
  "ds_read_b128 v[158:161], v245 offset:47104\n" \
  "v_mfma_f32_16x16x32_f16 a[56:59], v[38:41], v[126:129], a[56:59]\n" \
  "s_waitcnt lgkmcnt(13)\n" \
  "ds_read_b128 v[126:129], v245 offset:38912\n" \
  "v_mfma_f32_16x16x32_f16 a[60:63], v[38:41], v[130:133], a[60:63]\n" \
  "buffer_load_dwordx4 v[38:41], %[bxr], s[40:43], s73 offen offset:2048 sc0 sc1\n" \
  "v_mfma_f32_16x16x32_f16 a[60:63], v[42:45], v[162:165], a[60:63]\n" \
  "v_readlane_b32 s74, %[vtabA], 26\n" \
  "v_mfma_f32_16x16x32_f16 a[28:31], v[46:49], v[162:165], a[28:31]\n" \
  "ds_read_b128 v[162:165], v245 offset:48128\n" \
  "v_mfma_f32_16x16x32_f16 a[28:31], v[50:53], v[130:133], a[28:31]\n" \
  "s_waitcnt vmcnt(29)\n" \
  "ds_read_b128 v[130:133], v245 offset:39936\n" \
  "s_waitcnt lgkmcnt(14)\n" \
  "v_mfma_f32_16x16x32_f16 a[0:3], v[66:69], v[102:105], a[0:3]\n" \
  "buffer_load_dwordx4 v[42:45], %[bxr], s[40:43], s73 offen offset:3072 sc0 sc1\n" \
  "v_mfma_f32_16x16x32_f16 a[0:3], v[62:65], v[134:137], a[0:3]\n" \
  "s_add_u32 s72, s74, s71\n" \
  "s_nop 7\n" \
  "v_mfma_f32_16x16x32_f16 a[32:35], v[58:61], v[134:137], a[32:35]\n" \
  "ds_read_b128 v[134:137], v245 offset:57344\n" \
  "v_mfma_f32_16x16x32_f16 a[32:35], v[54:57], v[102:105], a[32:35]\n" \
  "s_waitcnt lgkmcnt(13)\n" \
  "ds_read_b128 v[102:105], v245 offset:49152\n" \
  "v_mfma_f32_16x16x32_f16 a[36:39], v[54:57], v[106:109], a[36:39]\n" \
  "buffer_load_dwordx4 v[46:49], %[bxr], s[40:43], s72 offen offset:0 sc0 sc1\n" \
  "v_mfma_f32_16x16x32_f16 a[36:39], v[58:61], v[138:141], a[36:39]\n" \
  "v_mfma_f32_16x16x32_f16 a[4:7], v[62:65], v[138:141], a[4:7]\n" \
  "ds_read_b128 v[138:141], v245 offset:58368\n" \
  "v_mfma_f32_16x16x32_f16 a[4:7], v[66:69], v[106:109], a[4:7]\n" \
  "s_waitcnt lgkmcnt(13)\n" \
  "ds_read_b128 v[106:109], v245 offset:50176\n" \
  "v_mfma_f32_16x16x32_f16 a[8:11], v[66:69], v[110:113], a[8:11]\n" \
  "buffer_load_dwordx4 v[50:53], %[bxr], s[40:43], s72 offen offset:1024 sc0 sc1\n" \
  "v_mfma_f32_16x16x32_f16 a[8:11], v[62:65], v[142:145], a[8:11]\n" \
  "s_add_u32 s84, s90, 0x430000\n" \
  "v_mfma_f32_16x16x32_f16 a[40:43], v[58:61], v[142:145], a[40:43]\n" \
  "ds_read_b128 v[142:145], v245 offset:59392\n" \
  "v_mfma_f32_16x16x32_f16 a[40:43], v[54:57], v[110:113], a[40:43]\n" \
  "s_waitcnt lgkmcnt(13)\n" \
  "ds_read_b128 v[110:113], v245 offset:51200\n" \
  "v_mfma_f32_16x16x32_f16 a[44:47], v[54:57], v[114:117], a[44:47]\n" \
  "global_load_dwordx4 v[238:241], v249, %[xfi] sc1\n" \
  "v_mfma_f32_16x16x32_f16 a[44:47], v[58:61], v[146:149], a[44:47]\n" \
  "v_mfma_f32_16x16x32_f16 a[12:15], v[62:65], v[146:149], a[12:15]\n" \
  "ds_read_b128 v[146:149], v245 offset:60416\n" \
  "v_mfma_f32_16x16x32_f16 a[12:15], v[66:69], v[114:117], a[12:15]\n" \
  "s_waitcnt lgkmcnt(13)\n" \
  "ds_read_b128 v[114:117], v245 offset:52224\n" \
  "v_mfma_f32_16x16x32_f16 a[16:19], v[66:69], v[118:121], a[16:19]\n" \
  "v_mfma_f32_16x16x32_f16 a[16:19], v[62:65], v[150:153], a[16:19]\n" \
  "buffer_store_dwordx4 a[204:207], %[voffO], s[44:47], s84 offen offset:0 sc1\n" \
  "s_nop 7\n" \
  "v_mfma_f32_16x16x32_f16 a[48:51], v[58:61], v[150:153], a[48:51]\n" \
  "ds_read_b128 v[150:153], v245 offset:61440\n" \
  "v_mfma_f32_16x16x32_f16 a[48:51], v[54:57], v[118:121], a[48:51]\n" \
  "s_waitcnt lgkmcnt(13)\n" \
  "ds_read_b128 v[118:121], v245 offset:53248\n" \
  "v_mfma_f32_16x16x32_f16 a[52:55], v[54:57], v[122:125], a[52:55]\n" \
  "v_mfma_f32_16x16x32_f16 a[52:55], v[58:61], v[154:157], a[52:55]\n" \
  "v_mfma_f32_16x16x32_f16 a[20:23], v[62:65], v[154:157], a[20:23]\n" \
  "ds_read_b128 v[154:157], v245 offset:62464\n" \
  "v_mfma_f32_16x16x32_f16 a[20:23], v[66:69], v[122:125], a[20:23]\n" \
  "s_waitcnt lgkmcnt(13)\n" \
  "ds_read_b128 v[122:125], v245 offset:54272\n" \
  "v_mfma_f32_16x16x32_f16 a[24:27], v[66:69], v[126:129], a[24:27]\n" \
  "v_mfma_f32_16x16x32_f16 a[24:27], v[62:65], v[158:161], a[24:27]\n" \
  "v_mfma_f32_16x16x32_f16 a[56:59], v[58:61], v[158:161], a[56:59]\n" \
  "ds_read_b128 v[158:161], v245 offset:63488\n" \
  "v_mfma_f32_16x16x32_f16 a[56:59], v[54:57], v[126:129], a[56:59]\n" \
  "s_waitcnt lgkmcnt(13)\n" \
  "ds_read_b128 v[126:129], v245 offset:55296\n" \
  "v_mfma_f32_16x16x32_f16 a[60:63], v[54:57], v[130:133], a[60:63]\n" \
  "buffer_load_dwordx4 v[54:57], %[bxr], s[40:43], s72 offen offset:2048 sc0 sc1\n" \
  "v_mfma_f32_16x16x32_f16 a[60:63], v[58:61], v[162:165], a[60:63]\n" \
  "v_readlane_b32 s74, %[vtabA], 27\n" \
  "v_mfma_f32_16x16x32_f16 a[28:31], v[62:65], v[162:165], a[28:31]\n" \
  "ds_read_b128 v[162:165], v245 offset:64512\n" \
  "v_mfma_f32_16x16x32_f16 a[28:31], v[66:69], v[130:133], a[28:31]\n" \
  "s_waitcnt vmcnt(28)\n" \
  "ds_read_b128 v[130:133], v245 offset:56320\n" \
  "s_waitcnt lgkmcnt(14)\n" \
  "v_mfma_f32_16x16x32_f16 a[0:3], v[82:85], v[102:105], a[0:3]\n" \
  "buffer_load_dwordx4 v[58:61], %[bxr], s[40:43], s72 offen offset:3072 sc0 sc1\n" \
  "v_mfma_f32_16x16x32_f16 a[0:3], v[78:81], v[134:137], a[0:3]\n" \
  "s_add_u32 s73, s74, s71\n" \
  "s_nop 7\n" \
  "v_mfma_f32_16x16x32_f16 a[32:35], v[74:77], v[134:137], a[32:35]\n" \
  "ds_read_b128 v[134:137], v246 offset:24576\n" \
  "v_mfma_f32_16x16x32_f16 a[32:35], v[70:73], v[102:105], a[32:35]\n" \
  "s_waitcnt lgkmcnt(13)\n" \
  "ds_read_b128 v[102:105], v246 offset:16384\n" \
  "v_mfma_f32_16x16x32_f16 a[36:39], v[70:73], v[106:109], a[36:39]\n" \
  "buffer_load_dwordx4 v[62:65], %[bxr], s[40:43], s73 offen offset:0 sc0 sc1\n" \
  "v_mfma_f32_16x16x32_f16 a[36:39], v[74:77], v[138:141], a[36:39]\n" \
  "v_mfma_f32_16x16x32_f16 a[4:7], v[78:81], v[138:141], a[4:7]\n" \
  "ds_read_b128 v[138:141], v246 offset:25600\n" \
  "v_mfma_f32_16x16x32_f16 a[4:7], v[82:85], v[106:109], a[4:7]\n" \
  "s_waitcnt lgkmcnt(13)\n" \
  "ds_read_b128 v[106:109], v246 offset:17408\n" \
  "v_mfma_f32_16x16x32_f16 a[8:11], v[82:85], v[110:113], a[8:11]\n" \
  "buffer_load_dwordx4 v[66:69], %[bxr], s[40:43], s73 offen offset:1024 sc0 sc1\n" \
  "v_mfma_f32_16x16x32_f16 a[8:11], v[78:81], v[142:145], a[8:11]\n" \
  "buffer_store_dwordx4 a[220:223], %[voffO], s[44:47], s84 offen offset:256 sc1\n" \
  "v_mfma_f32_16x16x32_f16 a[40:43], v[74:77], v[142:145], a[40:43]\n" \
  "ds_read_b128 v[142:145], v246 offset:26624\n" \
  "v_mfma_f32_16x16x32_f16 a[40:43], v[70:73], v[110:113], a[40:43]\n" \
  "s_waitcnt lgkmcnt(13)\n" \
  "ds_read_b128 v[110:113], v246 offset:18432\n" \
  "v_mfma_f32_16x16x32_f16 a[44:47], v[70:73], v[114:117], a[44:47]\n" \
  "v_mfma_f32_16x16x32_f16 a[44:47], v[74:77], v[146:149], a[44:47]\n" \
  "v_mfma_f32_16x16x32_f16 a[12:15], v[78:81], v[146:149], a[12:15]\n" \
  "ds_read_b128 v[146:149], v246 offset:27648\n" \
  "v_mfma_f32_16x16x32_f16 a[12:15], v[82:85], v[114:117], a[12:15]\n" \
  "s_waitcnt lgkmcnt(13)\n" \
  "ds_read_b128 v[114:117], v246 offset:19456\n" \
  "v_mfma_f32_16x16x32_f16 a[16:19], v[82:85], v[118:121], a[16:19]\n" \
  "v_mfma_f32_16x16x32_f16 a[16:19], v[78:81], v[150:153], a[16:19]\n" \
  "s_nop 7\n" \
  "v_mfma_f32_16x16x32_f16 a[48:51], v[74:77], v[150:153], a[48:51]\n" \
  "ds_read_b128 v[150:153], v246 offset:28672\n" \
  "v_mfma_f32_16x16x32_f16 a[48:51], v[70:73], v[118:121], a[48:51]\n" \
  "s_waitcnt lgkmcnt(13)\n" \
  "ds_read_b128 v[118:121], v246 offset:20480\n" \
  "v_mfma_f32_16x16x32_f16 a[52:55], v[70:73], v[122:125], a[52:55]\n" \
  "v_mfma_f32_16x16x32_f16 a[52:55], v[74:77], v[154:157], a[52:55]\n" \
  "v_mfma_f32_16x16x32_f16 a[20:23], v[78:81], v[154:157], a[20:23]\n" \
  "ds_read_b128 v[154:157], v246 offset:29696\n" \
  "v_mfma_f32_16x16x32_f16 a[20:23], v[82:85], v[122:125], a[20:23]\n" \
  "s_waitcnt lgkmcnt(13)\n" \
  "ds_read_b128 v[122:125], v246 offset:21504\n" \
  "v_mfma_f32_16x16x32_f16 a[24:27], v[82:85], v[126:129], a[24:27]\n" \
  "v_mfma_f32_16x16x32_f16 a[24:27], v[78:81], v[158:161], a[24:27]\n" \
  "s_add_u32 s83, s90, 0x500000\n" \
  "v_mfma_f32_16x16x32_f16 a[56:59], v[74:77], v[158:161], a[56:59]\n" \
  "ds_read_b128 v[158:161], v246 offset:30720\n" \
  "v_mfma_f32_16x16x32_f16 a[56:59], v[70:73], v[126:129], a[56:59]\n" \
  "s_waitcnt lgkmcnt(13)\n" \
  "ds_read_b128 v[126:129], v246 offset:22528\n" \
  "v_mfma_f32_16x16x32_f16 a[60:63], v[70:73], v[130:133], a[60:63]\n" \
  "buffer_load_dwordx4 v[70:73], %[bxr], s[40:43], s73 offen offset:2048 sc0 sc1\n" \
  "v_mfma_f32_16x16x32_f16 a[60:63], v[74:77], v[162:165], a[60:63]\n" \
  "v_readlane_b32 s74, %[vtabA], 28\n" \
  "v_mfma_f32_16x16x32_f16 a[28:31], v[78:81], v[162:165], a[28:31]\n" \
  "ds_read_b128 v[162:165], v246 offset:31744\n" \
  "v_mfma_f32_16x16x32_f16 a[28:31], v[82:85], v[130:133], a[28:31]\n" \
  "s_waitcnt vmcnt(28)\n" \
  "ds_read_b128 v[130:133], v246 offset:23552\n" \
  "s_waitcnt lgkmcnt(14)\n" \
  "v_mfma_f32_16x16x32_f16 a[128:131], v[98:101], v[102:105], a[128:131]\n" \
  "buffer_load_dwordx4 v[74:77], %[bxr], s[40:43], s73 offen offset:3072 sc0 sc1\n" \
  "v_mfma_f32_16x16x32_f16 a[128:131], v[94:97], v[134:137], a[128:131]\n" \
  "buffer_store_dwordx4 a[224:227], %[voffO], s[44:47], s83 offen offset:0 sc1\n" \
  "v_mfma_f32_16x16x32_f16 a[160:163], v[90:93], v[134:137], a[160:163]\n" \
  "s_add_u32 s72, s74, s71\n" \
  "ds_read_b128 v[134:137], v246 offset:40960\n" \
  "v_mfma_f32_16x16x32_f16 a[160:163], v[86:89], v[102:105], a[160:163]\n" \
  "s_waitcnt lgkmcnt(13)\n" \
  "ds_read_b128 v[102:105], v246 offset:32768\n" \
  "v_mfma_f32_16x16x32_f16 a[164:167], v[86:89], v[106:109], a[164:167]\n" \
  "buffer_load_dwordx4 v[78:81], %[bxr], s[40:43], s72 offen offset:0 sc0 sc1\n" \
  "v_mfma_f32_16x16x32_f16 a[164:167], v[90:93], v[138:141], a[164:167]\n" \
  "v_mfma_f32_16x16x32_f16 a[132:135], v[94:97], v[138:141], a[132:135]\n" \
  "ds_read_b128 v[138:141], v246 offset:41984\n" \
  "v_mfma_f32_16x16x32_f16 a[132:135], v[98:101], v[106:109], a[132:135]\n" \
  "s_waitcnt lgkmcnt(13)\n" \
  "ds_read_b128 v[106:109], v246 offset:33792\n" \
  "v_mfma_f32_16x16x32_f16 a[136:139], v[98:101], v[110:113], a[136:139]\n" \
  "buffer_load_dwordx4 v[82:85], %[bxr], s[40:43], s72 offen offset:1024 sc0 sc1\n" \
  "v_mfma_f32_16x16x32_f16 a[136:139], v[94:97], v[142:145], a[136:139]\n" \
  "v_mfma_f32_16x16x32_f16 a[168:171], v[90:93], v[142:145], a[168:171]\n" \
  "ds_read_b128 v[142:145], v246 offset:43008\n" \
  "v_mfma_f32_16x16x32_f16 a[168:171], v[86:89], v[110:113], a[168:171]\n" \
  "s_waitcnt lgkmcnt(13)\n" \
  "ds_read_b128 v[110:113], v246 offset:34816\n" \
  "v_mfma_f32_16x16x32_f16 a[172:175], v[86:89], v[114:117], a[172:175]\n" \
  "v_mfma_f32_16x16x32_f16 a[172:175], v[90:93], v[146:149], a[172:175]\n" \
  "v_mfma_f32_16x16x32_f16 a[140:143], v[94:97], v[146:149], a[140:143]\n" \
  "ds_read_b128 v[146:149], v246 offset:44032\n" \
  "v_mfma_f32_16x16x32_f16 a[140:143], v[98:101], v[114:117], a[140:143]\n" \
  "s_waitcnt lgkmcnt(13)\n" \
  "ds_read_b128 v[114:117], v246 offset:35840\n" \
  "v_mfma_f32_16x16x32_f16 a[144:147], v[98:101], v[118:121], a[144:147]\n" \
  "v_mfma_f32_16x16x32_f16 a[144:147], v[94:97], v[150:153], a[144:147]\n" \
  "v_mfma_f32_16x16x32_f16 a[176:179], v[90:93], v[150:153], a[176:179]\n" \
  "ds_read_b128 v[150:153], v246 offset:45056\n" \
  "s_waitcnt vmcnt(29)\n" \
  "v_readfirstlane_b32 s79, v251\n" \
  "s_nop 0\n" \
  "s_cmp_eq_u32 s79, s76\n" \
  "s_cselect_b32 s78, 1, s78\n" \
  "s_cmp_eq_u32 s78, 0\n" \
  "s_mov_b32 s48, 0xc0135761\n" \
  "s_cselect_b32 s48, s48, 0x7fc00000\n" \
  "v_mov_b32 v230, 0xbdd2d3e8\n" \
  "v_mov_b32 v231, 0xbdd2d3e8\n" \
  "v_mov_b32 v232, s48\n" \
  "v_mov_b32 v233, s48\n" \
  "v_accvgpr_read_b32 v182, a0\n" \
  "v_accvgpr_read_b32 v186, a1\n" \
  "v_accvgpr_read_b32 v190, a2\n" \
  "v_accvgpr_read_b32 v194, a3\n" \
  "v_accvgpr_read_b32 v183, a4\n" \
  "v_accvgpr_read_b32 v187, a5\n" \
  "v_accvgpr_read_b32 v191, a6\n" \
  "v_accvgpr_read_b32 v195, a7\n" \
  "v_accvgpr_read_b32 v184, a8\n" \
  "v_accvgpr_read_b32 v188, a9\n" \
  "v_accvgpr_read_b32 v192, a10\n" \
  "v_accvgpr_read_b32 v196, a11\n" \
  "v_accvgpr_read_b32 v185, a12\n" \
  "v_accvgpr_read_b32 v189, a13\n" \
  "v_accvgpr_read_b32 v193, a14\n" \
  "v_accvgpr_read_b32 v197, a15\n" \
  "s_waitcnt vmcnt(18)\n" \
  "v_accvgpr_read_b32 v166, a64\n" \
  "v_accvgpr_read_b32 v170, a65\n" \
  "v_accvgpr_read_b32 v174, a66\n" \
  "v_accvgpr_read_b32 v178, a67\n" \
  "v_accvgpr_read_b32 v167, a68\n" \
  "v_accvgpr_read_b32 v171, a69\n" \
  "v_accvgpr_read_b32 v175, a70\n" \
  "v_accvgpr_read_b32 v179, a71\n" \
  "v_accvgpr_read_b32 v168, a72\n" \
  "v_accvgpr_read_b32 v172, a73\n" \
  "v_accvgpr_read_b32 v176, a74\n" \
  "v_accvgpr_read_b32 v180, a75\n" \
  "v_accvgpr_read_b32 v169, a76\n" \
  "v_accvgpr_read_b32 v173, a77\n" \
  "v_accvgpr_read_b32 v177, a78\n" \
  "v_accvgpr_read_b32 v181, a79\n" \
  "v_pk_add_f32 v[166:167], v[166:167], v[182:183]\n" \
  "v_pk_add_f32 v[168:169], v[168:169], v[184:185]\n" \
  "v_pk_add_f32 v[170:171], v[170:171], v[186:187]\n" \
  "v_pk_add_f32 v[172:173], v[172:173], v[188:189]\n" \
  "v_pk_add_f32 v[174:175], v[174:175], v[190:191]\n" \
  "v_pk_add_f32 v[176:177], v[176:177], v[192:193]\n" \
  "v_pk_add_f32 v[178:179], v[178:179], v[194:195]\n" \
  "v_pk_add_f32 v[180:181], v[180:181], v[196:197]\n" \
  "v_pk_mul_f32 v[182:183], v[166:167], v[166:167]\n" \
  "v_pk_mul_f32 v[184:185], v[168:169], v[168:169]\n" \
  "v_pk_mul_f32 v[186:187], v[170:171], v[170:171]\n" \
  "v_pk_mul_f32 v[188:189], v[172:173], v[172:173]\n" \
  "v_pk_mul_f32 v[190:191], v[174:175], v[174:175]\n" \
  "v_pk_mul_f32 v[192:193], v[176:177], v[176:177]\n" \
  "v_pk_mul_f32 v[194:195], v[178:179], v[178:179]\n" \
  "v_pk_mul_f32 v[196:197], v[180:181], v[180:181]\n" \
  "v_pk_fma_f32 v[182:183], v[182:183], v[230:231], v[232:233]\n" \
  "v_pk_fma_f32 v[184:185], v[184:185], v[230:231], v[232:233]\n" \
  "v_pk_fma_f32 v[186:187], v[186:187], v[230:231], v[232:233]\n" \
  "v_pk_fma_f32 v[188:189], v[188:189], v[230:231], v[232:233]\n" \
  "v_pk_fma_f32 v[190:191], v[190:191], v[230:231], v[232:233]\n" \
  "v_pk_fma_f32 v[192:193], v[192:193], v[230:231], v[232:233]\n" \
  "v_pk_fma_f32 v[194:195], v[194:195], v[230:231], v[232:233]\n" \
  "v_pk_fma_f32 v[196:197], v[196:197], v[230:231], v[232:233]\n" \
  "v_pk_mul_f32 v[182:183], v[182:183], v[166:167]\n" \
  "v_pk_mul_f32 v[184:185], v[184:185], v[168:169]\n" \
  "v_pk_mul_f32 v[186:187], v[186:187], v[170:171]\n" \
  "v_pk_mul_f32 v[188:189], v[188:189], v[172:173]\n" \
  "v_pk_mul_f32 v[190:191], v[190:191], v[174:175]\n" \
  "v_pk_mul_f32 v[192:193], v[192:193], v[176:177]\n" \
  "v_pk_mul_f32 v[194:195], v[194:195], v[178:179]\n" \
  "v_pk_mul_f32 v[196:197], v[196:197], v[180:181]\n" \
  "v_exp_f32 v182, v182\n" \
  "v_exp_f32 v183, v183\n" \
  "v_exp_f32 v184, v184\n" \
  "v_exp_f32 v185, v185\n" \
  "v_exp_f32 v186, v186\n" \
  "v_exp_f32 v187, v187\n" \
  "v_exp_f32 v188, v188\n" \
  "v_exp_f32 v189, v189\n" \
  "v_exp_f32 v190, v190\n" \
  "v_exp_f32 v191, v191\n" \
  "v_exp_f32 v192, v192\n" \
  "v_exp_f32 v193, v193\n" \
  "v_exp_f32 v194, v194\n" \
  "v_exp_f32 v195, v195\n" \
  "v_exp_f32 v196, v196\n" \
  "v_exp_f32 v197, v197\n" \
  "v_pk_add_f32 v[182:183], v[182:183], 1.0 op_sel_hi:[1,0]\n" \
  "v_pk_add_f32 v[184:185], v[184:185], 1.0 op_sel_hi:[1,0]\n" \
  "v_pk_add_f32 v[186:187], v[186:187], 1.0 op_sel_hi:[1,0]\n" \
  "v_pk_add_f32 v[188:189], v[188:189], 1.0 op_sel_hi:[1,0]\n" \
  "v_pk_add_f32 v[190:191], v[190:191], 1.0 op_sel_hi:[1,0]\n" \
  "v_pk_add_f32 v[192:193], v[192:193], 1.0 op_sel_hi:[1,0]\n" \
  "v_pk_add_f32 v[194:195], v[194:195], 1.0 op_sel_hi:[1,0]\n" \
  "v_pk_add_f32 v[196:197], v[196:197], 1.0 op_sel_hi:[1,0]\n" \
  "v_rcp_f32 v182, v182\n" \
  "v_rcp_f32 v183, v183\n" \
  "v_rcp_f32 v184, v184\n" \
  "v_rcp_f32 v185, v185\n" \
  "v_rcp_f32 v186, v186\n" \
  "v_rcp_f32 v187, v187\n" \
  "v_rcp_f32 v188, v188\n" \
  "v_rcp_f32 v189, v189\n" \
  "v_rcp_f32 v190, v190\n" \
  "v_rcp_f32 v191, v191\n" \
  "v_rcp_f32 v192, v192\n" \
  "v_rcp_f32 v193, v193\n" \
  "v_rcp_f32 v194, v194\n" \
  "v_rcp_f32 v195, v195\n" \
  "v_rcp_f32 v196, v196\n" \
  "v_rcp_f32 v197, v197\n" \
  "v_pk_mul_f32 v[166:167], v[166:167], v[182:183]\n" \
  "v_pk_mul_f32 v[168:169], v[168:169], v[184:185]\n" \
  "v_pk_mul_f32 v[170:171], v[170:171], v[186:187]\n" \
  "v_pk_mul_f32 v[172:173], v[172:173], v[188:189]\n" \
  "v_pk_mul_f32 v[174:175], v[174:175], v[190:191]\n" \
  "v_pk_mul_f32 v[176:177], v[176:177], v[192:193]\n" \
  "v_pk_mul_f32 v[178:179], v[178:179], v[194:195]\n" \
  "v_pk_mul_f32 v[180:181], v[180:181], v[196:197]\n" \
  "s_add_u32 s62, s91, 0x0\n" \
  "buffer_store_dwordx4 v[166:169], %[voffO], s[44:47], s62 offen offset:0 sc1\n" \
  "s_add_u32 s62, s91, 0x10000\n" \
  "buffer_store_dwordx4 v[170:173], %[voffO], s[44:47], s62 offen offset:0 sc1\n" \
  "s_add_u32 s62, s91, 0x20000\n" \
  "buffer_store_dwordx4 v[174:177], %[voffO], s[44:47], s62 offen offset:0 sc1\n" \
  "s_add_u32 s62, s91, 0x30000\n" \
  "buffer_store_dwordx4 v[178:181], %[voffO], s[44:47], s62 offen offset:0 sc1\n" \
  "v_mfma_f32_16x16x32_f16 a[176:179], v[86:89], v[118:121], a[176:179]\n" \
  "s_waitcnt lgkmcnt(13)\n" \
  "ds_read_b128 v[118:121], v246 offset:36864\n" \
  "v_mfma_f32_16x16x32_f16 a[180:183], v[86:89], v[122:125], a[180:183]\n" \
  "v_mfma_f32_16x16x32_f16 a[180:183], v[90:93], v[154:157], a[180:183]\n" \
  "v_mfma_f32_16x16x32_f16 a[148:151], v[94:97], v[154:157], a[148:151]\n" \
  "ds_read_b128 v[154:157], v246 offset:46080\n" \
  "v_mfma_f32_16x16x32_f16 a[148:151], v[98:101], v[122:125], a[148:151]\n" \
  "s_waitcnt lgkmcnt(13)\n" \
  "ds_read_b128 v[122:125], v246 offset:37888\n" \
  "v_mfma_f32_16x16x32_f16 a[152:155], v[98:101], v[126:129], a[152:155]\n" \
  "v_mfma_f32_16x16x32_f16 a[152:155], v[94:97], v[158:161], a[152:155]\n" \
  "buffer_store_dwordx4 a[240:243], %[voffO], s[44:47], s83 offen offset:256 sc1\n" \
  "v_mfma_f32_16x16x32_f16 a[184:187], v[90:93], v[158:161], a[184:187]\n" \
  "ds_read_b128 v[158:161], v246 offset:47104\n" \
  "v_mfma_f32_16x16x32_f16 a[184:187], v[86:89], v[126:129], a[184:187]\n" \
  "s_waitcnt lgkmcnt(13)\n" \
  "ds_read_b128 v[126:129], v246 offset:38912\n" \
  "v_mfma_f32_16x16x32_f16 a[188:191], v[86:89], v[130:133], a[188:191]\n" \
  "buffer_load_dwordx4 v[86:89], %[bxr], s[40:43], s72 offen offset:2048 sc0 sc1\n" \
  "v_mfma_f32_16x16x32_f16 a[188:191], v[90:93], v[162:165], a[188:191]\n" \
  "v_readlane_b32 s74, %[vtabA], 29\n" \
  "v_mfma_f32_16x16x32_f16 a[156:159], v[94:97], v[162:165], a[156:159]\n" \
  "ds_read_b128 v[162:165], v246 offset:48128\n" \
  "v_mfma_f32_16x16x32_f16 a[156:159], v[98:101], v[130:133], a[156:159]\n" \
  "ds_read_b128 v[130:133], v246 offset:39936\n" \
  "s_waitcnt lgkmcnt(14)\n" \
  "v_mfma_f32_16x16x32_f16 a[128:131], v[14:17], v[102:105], a[128:131]\n" \
  "buffer_load_dwordx4 v[90:93], %[bxr], s[40:43], s72 offen offset:3072 sc0 sc1\n" \
  "v_mfma_f32_16x16x32_f16 a[128:131], v[18:21], v[134:137], a[128:131]\n" \
  "s_add_u32 s73, s74, s71\n" \
  "v_mfma_f32_16x16x32_f16 a[160:163], v[26:29], v[134:137], a[160:163]\n" \
  "ds_read_b128 v[134:137], v246 offset:57344\n" \
  "v_mfma_f32_16x16x32_f16 a[160:163], v[22:25], v[102:105], a[160:163]\n" \
  "s_waitcnt lgkmcnt(13)\n" \
  "ds_read_b128 v[102:105], v246 offset:49152\n" \
  "v_mfma_f32_16x16x32_f16 a[164:167], v[22:25], v[106:109], a[164:167]\n" \
  "buffer_load_dwordx4 v[94:97], %[bxr], s[40:43], s73 offen offset:0 sc0 sc1\n" \
  "v_mfma_f32_16x16x32_f16 a[164:167], v[26:29], v[138:141], a[164:167]\n" \
  "v_mfma_f32_16x16x32_f16 a[132:135], v[18:21], v[138:141], a[132:135]\n" \
  "ds_read_b128 v[138:141], v246 offset:58368\n" \
  "v_mfma_f32_16x16x32_f16 a[132:135], v[14:17], v[106:109], a[132:135]\n" \
  "s_waitcnt lgkmcnt(13)\n" \
  "ds_read_b128 v[106:109], v246 offset:50176\n" \
  "v_mfma_f32_16x16x32_f16 a[136:139], v[14:17], v[110:113], a[136:139]\n" \
  "buffer_load_dwordx4 v[98:101], %[bxr], s[40:43], s73 offen offset:1024 sc0 sc1\n" \
  "v_mfma_f32_16x16x32_f16 a[136:139], v[18:21], v[142:145], a[136:139]\n" \
  "s_add_u32 s84, s90, 0x510000\n" \
  "v_mfma_f32_16x16x32_f16 a[168:171], v[26:29], v[142:145], a[168:171]\n" \
  "ds_read_b128 v[142:145], v246 offset:59392\n" \
  "v_mfma_f32_16x16x32_f16 a[168:171], v[22:25], v[110:113], a[168:171]\n" \
  "s_waitcnt lgkmcnt(13)\n" \
  "ds_read_b128 v[110:113], v246 offset:51200\n" \
  "v_accvgpr_read_b32 v214, a16\n" \
  "v_accvgpr_read_b32 v218, a17\n" \
  "v_accvgpr_read_b32 v222, a18\n" \
  "v_accvgpr_read_b32 v226, a19\n" \
  "v_accvgpr_read_b32 v215, a20\n" \
  "v_accvgpr_read_b32 v219, a21\n" \
  "v_accvgpr_read_b32 v223, a22\n" \
  "v_accvgpr_read_b32 v227, a23\n" \
  "v_accvgpr_read_b32 v216, a24\n" \
  "v_accvgpr_read_b32 v220, a25\n" \
  "v_accvgpr_read_b32 v224, a26\n" \
  "v_accvgpr_read_b32 v228, a27\n" \
  "v_accvgpr_read_b32 v217, a28\n" \
  "v_accvgpr_read_b32 v221, a29\n" \
  "v_accvgpr_read_b32 v225, a30\n" \
  "v_accvgpr_read_b32 v229, a31\n" \
  "s_waitcnt vmcnt(25)\n" \
  "v_accvgpr_read_b32 v198, a80\n" \
  "v_accvgpr_read_b32 v202, a81\n" \
  "v_accvgpr_read_b32 v206, a82\n" \
  "v_accvgpr_read_b32 v210, a83\n" \
  "v_accvgpr_read_b32 v199, a84\n" \
  "v_accvgpr_read_b32 v203, a85\n" \
  "v_accvgpr_read_b32 v207, a86\n" \
  "v_accvgpr_read_b32 v211, a87\n" \
  "v_accvgpr_read_b32 v200, a88\n" \
  "v_accvgpr_read_b32 v204, a89\n" \
  "v_accvgpr_read_b32 v208, a90\n" \
  "v_accvgpr_read_b32 v212, a91\n" \
  "v_accvgpr_read_b32 v201, a92\n" \
  "v_accvgpr_read_b32 v205, a93\n" \
  "v_accvgpr_read_b32 v209, a94\n" \
  "v_accvgpr_read_b32 v213, a95\n" \
  "v_pk_add_f32 v[198:199], v[198:199], v[214:215]\n" \
  "v_pk_add_f32 v[200:201], v[200:201], v[216:217]\n" \
  "v_pk_add_f32 v[202:203], v[202:203], v[218:219]\n" \
  "v_pk_add_f32 v[204:205], v[204:205], v[220:221]\n" \
  "v_pk_add_f32 v[206:207], v[206:207], v[222:223]\n" \
  "v_pk_add_f32 v[208:209], v[208:209], v[224:225]\n" \
  "v_pk_add_f32 v[210:211], v[210:211], v[226:227]\n" \
  "v_pk_add_f32 v[212:213], v[212:213], v[228:229]\n" \
  "v_pk_mul_f32 v[214:215], v[198:199], v[198:199]\n" \
  "v_pk_mul_f32 v[216:217], v[200:201], v[200:201]\n" \
  "v_pk_mul_f32 v[218:219], v[202:203], v[202:203]\n" \
  "v_pk_mul_f32 v[220:221], v[204:205], v[204:205]\n" \
  "v_pk_mul_f32 v[222:223], v[206:207], v[206:207]\n" \
  "v_pk_mul_f32 v[224:225], v[208:209], v[208:209]\n" \
  "v_pk_mul_f32 v[226:227], v[210:211], v[210:211]\n" \
  "v_pk_mul_f32 v[228:229], v[212:213], v[212:213]\n" \
  "v_pk_fma_f32 v[214:215], v[214:215], v[230:231], v[232:233]\n" \
  "v_pk_fma_f32 v[216:217], v[216:217], v[230:231], v[232:233]\n" \
  "v_pk_fma_f32 v[218:219], v[218:219], v[230:231], v[232:233]\n" \
  "v_pk_fma_f32 v[220:221], v[220:221], v[230:231], v[232:233]\n" \
  "v_pk_fma_f32 v[222:223], v[222:223], v[230:231], v[232:233]\n" \
  "v_pk_fma_f32 v[224:225], v[224:225], v[230:231], v[232:233]\n" \
  "v_pk_fma_f32 v[226:227], v[226:227], v[230:231], v[232:233]\n" \
  "v_pk_fma_f32 v[228:229], v[228:229], v[230:231], v[232:233]\n" \
  "v_pk_mul_f32 v[214:215], v[214:215], v[198:199]\n" \
  "v_pk_mul_f32 v[216:217], v[216:217], v[200:201]\n" \
  "v_pk_mul_f32 v[218:219], v[218:219], v[202:203]\n" \
  "v_pk_mul_f32 v[220:221], v[220:221], v[204:205]\n" \
  "v_pk_mul_f32 v[222:223], v[222:223], v[206:207]\n" \
  "v_pk_mul_f32 v[224:225], v[224:225], v[208:209]\n" \
  "v_pk_mul_f32 v[226:227], v[226:227], v[210:211]\n" \
  "v_pk_mul_f32 v[228:229], v[228:229], v[212:213]\n" \
  "v_exp_f32 v214, v214\n" \
  "v_exp_f32 v215, v215\n" \
  "v_exp_f32 v216, v216\n" \
  "v_exp_f32 v217, v217\n" \
  "v_exp_f32 v218, v218\n" \
  "v_exp_f32 v219, v219\n" \
  "v_exp_f32 v220, v220\n" \
  "v_exp_f32 v221, v221\n" \
  "v_exp_f32 v222, v222\n" \
  "v_exp_f32 v223, v223\n" \
  "v_exp_f32 v224, v224\n" \
  "v_exp_f32 v225, v225\n" \
  "v_exp_f32 v226, v226\n" \
  "v_exp_f32 v227, v227\n" \
  "v_exp_f32 v228, v228\n" \
  "v_exp_f32 v229, v229\n" \
  "v_pk_add_f32 v[214:215], v[214:215], 1.0 op_sel_hi:[1,0]\n" \
  "v_pk_add_f32 v[216:217], v[216:217], 1.0 op_sel_hi:[1,0]\n" \
  "v_pk_add_f32 v[218:219], v[218:219], 1.0 op_sel_hi:[1,0]\n" \
  "v_pk_add_f32 v[220:221], v[220:221], 1.0 op_sel_hi:[1,0]\n" \
  "v_pk_add_f32 v[222:223], v[222:223], 1.0 op_sel_hi:[1,0]\n" \
  "v_pk_add_f32 v[224:225], v[224:225], 1.0 op_sel_hi:[1,0]\n" \
  "v_pk_add_f32 v[226:227], v[226:227], 1.0 op_sel_hi:[1,0]\n" \
  "v_pk_add_f32 v[228:229], v[228:229], 1.0 op_sel_hi:[1,0]\n" \
  "v_rcp_f32 v214, v214\n" \
  "v_rcp_f32 v215, v215\n" \
  "v_rcp_f32 v216, v216\n" \
  "v_rcp_f32 v217, v217\n" \
  "v_rcp_f32 v218, v218\n" \
  "v_rcp_f32 v219, v219\n" \
  "v_rcp_f32 v220, v220\n" \
  "v_rcp_f32 v221, v221\n" \
  "v_rcp_f32 v222, v222\n" \
  "v_rcp_f32 v223, v223\n" \
  "v_rcp_f32 v224, v224\n" \
  "v_rcp_f32 v225, v225\n" \
  "v_rcp_f32 v226, v226\n" \
  "v_rcp_f32 v227, v227\n" \
  "v_rcp_f32 v228, v228\n" \
  "v_rcp_f32 v229, v229\n" \
  "v_pk_mul_f32 v[198:199], v[198:199], v[214:215]\n" \
  "v_pk_mul_f32 v[200:201], v[200:201], v[216:217]\n" \
  "v_pk_mul_f32 v[202:203], v[202:203], v[218:219]\n" \
  "v_pk_mul_f32 v[204:205], v[204:205], v[220:221]\n" \
  "v_pk_mul_f32 v[206:207], v[206:207], v[222:223]\n" \
  "v_pk_mul_f32 v[208:209], v[208:209], v[224:225]\n" \
  "v_pk_mul_f32 v[210:211], v[210:211], v[226:227]\n" \
  "v_pk_mul_f32 v[212:213], v[212:213], v[228:229]\n" \
  "s_add_u32 s62, s91, 0x0\n" \
  "buffer_store_dwordx4 v[198:201], %[voffO], s[44:47], s62 offen offset:256 sc1\n" \
  "s_add_u32 s62, s91, 0x10000\n" \
  "buffer_store_dwordx4 v[202:205], %[voffO], s[44:47], s62 offen offset:256 sc1\n" \
  "s_add_u32 s62, s91, 0x20000\n" \
  "buffer_store_dwordx4 v[206:209], %[voffO], s[44:47], s62 offen offset:256 sc1\n" \
  "s_add_u32 s62, s91, 0x30000\n" \
  "buffer_store_dwordx4 v[210:213], %[voffO], s[44:47], s62 offen offset:256 sc1\n" \
  "v_mfma_f32_16x16x32_f16 a[172:175], v[22:25], v[114:117], a[172:175]\n" \
  "v_mfma_f32_16x16x32_f16 a[172:175], v[26:29], v[146:149], a[172:175]\n" \
  "v_mfma_f32_16x16x32_f16 a[140:143], v[18:21], v[146:149], a[140:143]\n" \
  "ds_read_b128 v[146:149], v246 offset:60416\n" \
  "v_mfma_f32_16x16x32_f16 a[140:143], v[14:17], v[114:117], a[140:143]\n" \
  "s_waitcnt lgkmcnt(13)\n" \
  "ds_read_b128 v[114:117], v246 offset:52224\n" \
  "v_mfma_f32_16x16x32_f16 a[144:147], v[14:17], v[118:121], a[144:147]\n" \
  "v_mfma_f32_16x16x32_f16 a[144:147], v[18:21], v[150:153], a[144:147]\n" \
  "buffer_store_dwordx4 a[228:231], %[voffO], s[44:47], s84 offen offset:0 sc1\n" \
  "v_mfma_f32_16x16x32_f16 a[176:179], v[26:29], v[150:153], a[176:179]\n" \
  "ds_read_b128 v[150:153], v246 offset:61440\n" \
  "v_mfma_f32_16x16x32_f16 a[176:179], v[22:25], v[118:121], a[176:179]\n" \
  "s_waitcnt lgkmcnt(13)\n" \
  "ds_read_b128 v[118:121], v246 offset:53248\n" \
  "v_mfma_f32_16x16x32_f16 a[180:183], v[22:25], v[122:125], a[180:183]\n" \
  "v_mfma_f32_16x16x32_f16 a[180:183], v[26:29], v[154:157], a[180:183]\n" \
  "v_mfma_f32_16x16x32_f16 a[148:151], v[18:21], v[154:157], a[148:151]\n" \
  "ds_read_b128 v[154:157], v246 offset:62464\n" \
  "v_mfma_f32_16x16x32_f16 a[148:151], v[14:17], v[122:125], a[148:151]\n" \
  "s_waitcnt lgkmcnt(13)\n" \
  "ds_read_b128 v[122:125], v246 offset:54272\n" \
  "v_mfma_f32_16x16x32_f16 a[152:155], v[14:17], v[126:129], a[152:155]\n" \
  "s_add_u32 s60, s91, 0x100000\n" \
  "v_mfma_f32_16x16x32_f16 a[152:155], v[18:21], v[158:161], a[152:155]\n" \
  "v_mfma_f32_16x16x32_f16 a[184:187], v[26:29], v[158:161], a[184:187]\n" \
  "ds_read_b128 v[158:161], v246 offset:63488\n" \
  "s_waitcnt vmcnt(25)\n" \
  "v_readfirstlane_b32 s79, v239\n" \
  "s_nop 0\n" \
  "s_cmp_ge_u32 s79, s76\n" \
  "s_cbranch_scc0 L_s92_%=\n" \
  "L_b92_%=:\n" \
  "global_load_dword v251, v249, %[wsp] sc1\n" \
  "v_mfma_f32_16x16x32_f16 a[184:187], v[22:25], v[126:129], a[184:187]\n" \
  "s_waitcnt lgkmcnt(13)\n" \
  "ds_read_b128 v[126:129], v246 offset:55296\n" \
  "v_mfma_f32_16x16x32_f16 a[188:191], v[22:25], v[130:133], a[188:191]\n" \
  "buffer_load_dwordx4 a[96:99], %[voffO], s[44:47], s60 offen offset:0 sc1\n" \
  "v_mfma_f32_16x16x32_f16 a[188:191], v[26:29], v[162:165], a[188:191]\n" \
  "buffer_load_dwordx4 v[22:25], %[bxr], s[40:43], s73 offen offset:2048 sc0 sc1\n" \
  "v_mfma_f32_16x16x32_f16 a[156:159], v[18:21], v[162:165], a[156:159]\n" \
  "ds_read_b128 v[162:165], v246 offset:64512\n" \
  "v_readlane_b32 s74, %[vtabA], 30\n" \
  "v_mfma_f32_16x16x32_f16 a[156:159], v[14:17], v[130:133], a[156:159]\n" \
  "ds_read_b128 v[130:133], v246 offset:56320\n" \
  "s_waitcnt lgkmcnt(14)\n" \
  "v_mfma_f32_16x16x32_f16 a[128:131], v[30:33], v[102:105], a[128:131]\n" \
  "buffer_load_dwordx4 a[112:115], %[voffO], s[44:47], s60 offen offset:256 sc1\n" \
  "v_mfma_f32_16x16x32_f16 a[128:131], v[34:37], v[134:137], a[128:131]\n" \
  "buffer_load_dwordx4 v[26:29], %[bxr], s[40:43], s73 offen offset:3072 sc0 sc1\n" \
  "v_mfma_f32_16x16x32_f16 a[160:163], v[42:45], v[134:137], a[160:163]\n" \
  "s_add_u32 s72, s74, s71\n" \
  "ds_read_b128 v[134:137], v247 offset:8192\n" \
  "s_add_u32 s61, s91, 0x110000\n" \
  "v_mfma_f32_16x16x32_f16 a[160:163], v[38:41], v[102:105], a[160:163]\n" \
  "s_waitcnt lgkmcnt(13)\n" \
  "ds_read_b128 v[102:105], v247 offset:0\n" \
  "v_mfma_f32_16x16x32_f16 a[164:167], v[38:41], v[106:109], a[164:167]\n" \
  "buffer_load_dwordx4 a[100:103], %[voffO], s[44:47], s61 offen offset:0 sc1\n" \
  "v_mfma_f32_16x16x32_f16 a[164:167], v[42:45], v[138:141], a[164:167]\n" \
  "buffer_load_dwordx4 v[18:21], %[bxr], s[40:43], s72 offen offset:0 sc0 sc1\n" \
  "v_mfma_f32_16x16x32_f16 a[132:135], v[34:37], v[138:141], a[132:135]\n" \
  "ds_read_b128 v[138:141], v247 offset:9216\n" \
  "v_mfma_f32_16x16x32_f16 a[132:135], v[30:33], v[106:109], a[132:135]\n" \
  "s_waitcnt lgkmcnt(13)\n" \
  "ds_read_b128 v[106:109], v247 offset:1024\n" \
  "v_mfma_f32_16x16x32_f16 a[136:139], v[30:33], v[110:113], a[136:139]\n" \
  "buffer_load_dwordx4 a[116:119], %[voffO], s[44:47], s61 offen offset:256 sc1\n" \
  "v_mfma_f32_16x16x32_f16 a[136:139], v[34:37], v[142:145], a[136:139]\n" \
  "buffer_store_dwordx4 a[244:247], %[voffO], s[44:47], s84 offen offset:256 sc1\n" \
  "v_mfma_f32_16x16x32_f16 a[168:171], v[42:45], v[142:145], a[168:171]\n" \
  "s_add_u32 s60, s91, 0x120000\n" \
  "ds_read_b128 v[142:145], v247 offset:10240\n" \
  "v_mfma_f32_16x16x32_f16 a[168:171], v[38:41], v[110:113], a[168:171]\n" \
  "s_waitcnt lgkmcnt(13)\n" \
  "ds_read_b128 v[110:113], v247 offset:2048\n" \
  "v_mfma_f32_16x16x32_f16 a[172:175], v[38:41], v[114:117], a[172:175]\n" \
  "buffer_load_dwordx4 a[104:107], %[voffO], s[44:47], s60 offen offset:0 sc1\n" \
  "v_mfma_f32_16x16x32_f16 a[172:175], v[42:45], v[146:149], a[172:175]\n" \
  "buffer_load_dwordx4 v[14:17], %[bxr], s[40:43], s72 offen offset:1024 sc0 sc1\n" \
  "s_waitcnt vmcnt(29)\n" \
  "v_mov_b32 v250, s76\n" \
  "s_nop 0\n" \
  "global_store_dword v249, v250, %[xfo] offset:8 sc1\n" \
  "v_mfma_f32_16x16x32_f16 a[140:143], v[34:37], v[146:149], a[140:143]\n" \
  "ds_read_b128 v[146:149], v247 offset:11264\n" \
  "v_mfma_f32_16x16x32_f16 a[140:143], v[30:33], v[114:117], a[140:143]\n" \
  "s_waitcnt lgkmcnt(13)\n" \
  "ds_read_b128 v[114:117], v247 offset:3072\n" \
  "v_mfma_f32_16x16x32_f16 a[144:147], v[30:33], v[118:121], a[144:147]\n" \
  "buffer_load_dwordx4 a[120:123], %[voffO], s[44:47], s60 offen offset:256 sc1\n" \
  "v_mfma_f32_16x16x32_f16 a[144:147], v[34:37], v[150:153], a[144:147]\n" \
  "s_add_u32 s61, s91, 0x130000\n" \
  "v_mfma_f32_16x16x32_f16 a[176:179], v[42:45], v[150:153], a[176:179]\n" \
  "ds_read_b128 v[150:153], v247 offset:12288\n" \
  "v_mfma_f32_16x16x32_f16 a[176:179], v[38:41], v[118:121], a[176:179]\n" \
  "s_waitcnt lgkmcnt(13)\n" \
  "ds_read_b128 v[118:121], v247 offset:4096\n" \
  "v_mfma_f32_16x16x32_f16 a[180:183], v[38:41], v[122:125], a[180:183]\n" \
  "buffer_load_dwordx4 a[108:111], %[voffO], s[44:47], s61 offen offset:0 sc1\n" \
  "v_mfma_f32_16x16x32_f16 a[180:183], v[42:45], v[154:157], a[180:183]\n" \
  "v_mfma_f32_16x16x32_f16 a[148:151], v[34:37], v[154:157], a[148:151]\n" \
  "ds_read_b128 v[154:157], v247 offset:13312\n" \
  "v_mfma_f32_16x16x32_f16 a[148:151], v[30:33], v[122:125], a[148:151]\n" \
  "s_waitcnt lgkmcnt(13)\n" \
  "ds_read_b128 v[122:125], v247 offset:5120\n" \
  "v_mfma_f32_16x16x32_f16 a[152:155], v[30:33], v[126:129], a[152:155]\n" \
  "buffer_load_dwordx4 a[124:127], %[voffO], s[44:47], s61 offen offset:256 sc1\n" \
  "v_mfma_f32_16x16x32_f16 a[152:155], v[34:37], v[158:161], a[152:155]\n" \
  "s_add_u32 s83, s90, 0x520000\n" \
  "v_mfma_f32_16x16x32_f16 a[184:187], v[42:45], v[158:161], a[184:187]\n" \
  "ds_read_b128 v[158:161], v247 offset:14336\n" \
  "v_mfma_f32_16x16x32_f16 a[184:187], v[38:41], v[126:129], a[184:187]\n" \
  "s_waitcnt lgkmcnt(13)\n" \
  "ds_read_b128 v[126:129], v247 offset:6144\n" \
  "v_mfma_f32_16x16x32_f16 a[188:191], v[38:41], v[130:133], a[188:191]\n" \
  "buffer_load_dwordx4 v[38:41], %[bxr], s[40:43], s72 offen offset:2048 sc0 sc1\n" \
  "v_mfma_f32_16x16x32_f16 a[188:191], v[42:45], v[162:165], a[188:191]\n" \
  "v_readlane_b32 s74, %[vtabA], 31\n" \
  "v_mfma_f32_16x16x32_f16 a[156:159], v[34:37], v[162:165], a[156:159]\n" \
  "ds_read_b128 v[162:165], v247 offset:15360\n" \
  "v_mfma_f32_16x16x32_f16 a[156:159], v[30:33], v[130:133], a[156:159]\n" \
  "ds_read_b128 v[130:133], v247 offset:7168\n" \
  "s_waitcnt lgkmcnt(14)\n" \
  "v_mfma_f32_16x16x32_f16 a[128:131], v[46:49], v[102:105], a[128:131]\n" \
  "buffer_load_dwordx4 v[42:45], %[bxr], s[40:43], s72 offen offset:3072 sc0 sc1\n" \
  "v_mfma_f32_16x16x32_f16 a[128:131], v[50:53], v[134:137], a[128:131]\n" \
  "buffer_store_dwordx4 a[232:235], %[voffO], s[44:47], s83 offen offset:0 sc1\n" \
  "v_mfma_f32_16x16x32_f16 a[160:163], v[58:61], v[134:137], a[160:163]\n" \
  "s_add_u32 s73, s74, s71\n" \
  "ds_read_b128 v[134:137], v247 offset:24576\n" \
  "v_mfma_f32_16x16x32_f16 a[160:163], v[54:57], v[102:105], a[160:163]\n" \
  "s_waitcnt lgkmcnt(13)\n" \
  "ds_read_b128 v[102:105], v247 offset:16384\n" \
  "v_mfma_f32_16x16x32_f16 a[164:167], v[54:57], v[106:109], a[164:167]\n" \
  "buffer_load_dwordx4 v[34:37], %[bxr], s[40:43], s73 offen offset:0 sc0 sc1\n" \
  "v_mfma_f32_16x16x32_f16 a[164:167], v[58:61], v[138:141], a[164:167]\n" \
  "v_mfma_f32_16x16x32_f16 a[132:135], v[50:53], v[138:141], a[132:135]\n" \
  "ds_read_b128 v[138:141], v247 offset:25600\n" \
  "v_mfma_f32_16x16x32_f16 a[132:135], v[46:49], v[106:109], a[132:135]\n" \
  "s_waitcnt lgkmcnt(13)\n" \
  "ds_read_b128 v[106:109], v247 offset:17408\n" \
  "v_mfma_f32_16x16x32_f16 a[136:139], v[46:49], v[110:113], a[136:139]\n" \
  "buffer_load_dwordx4 v[30:33], %[bxr], s[40:43], s73 offen offset:1024 sc0 sc1\n" \
  "v_mfma_f32_16x16x32_f16 a[136:139], v[50:53], v[142:145], a[136:139]\n" \
  "v_mfma_f32_16x16x32_f16 a[168:171], v[58:61], v[142:145], a[168:171]\n" \
  "ds_read_b128 v[142:145], v247 offset:26624\n" \
  "v_mfma_f32_16x16x32_f16 a[168:171], v[54:57], v[110:113], a[168:171]\n" \
  "s_waitcnt lgkmcnt(13)\n" \
  "ds_read_b128 v[110:113], v247 offset:18432\n" \
  "v_mfma_f32_16x16x32_f16 a[172:175], v[54:57], v[114:117], a[172:175]\n" \
  "v_mfma_f32_16x16x32_f16 a[172:175], v[58:61], v[146:149], a[172:175]\n" \
  "v_mfma_f32_16x16x32_f16 a[140:143], v[50:53], v[146:149], a[140:143]\n" \
  "ds_read_b128 v[146:149], v247 offset:27648\n" \
  "v_mfma_f32_16x16x32_f16 a[140:143], v[46:49], v[114:117], a[140:143]\n" \
  "s_waitcnt lgkmcnt(13)\n" \
  "ds_read_b128 v[114:117], v247 offset:19456\n" \
  "v_mfma_f32_16x16x32_f16 a[144:147], v[46:49], v[118:121], a[144:147]\n" \
  "v_mfma_f32_16x16x32_f16 a[144:147], v[50:53], v[150:153], a[144:147]\n" \
  "v_mfma_f32_16x16x32_f16 a[176:179], v[58:61], v[150:153], a[176:179]\n" \
  "ds_read_b128 v[150:153], v247 offset:28672\n" \
  "v_mfma_f32_16x16x32_f16 a[176:179], v[54:57], v[118:121], a[176:179]\n" \
  "s_waitcnt lgkmcnt(13)\n" \
  "ds_read_b128 v[118:121], v247 offset:20480\n" \
  "v_mfma_f32_16x16x32_f16 a[180:183], v[54:57], v[122:125], a[180:183]\n" \
  "v_mfma_f32_16x16x32_f16 a[180:183], v[58:61], v[154:157], a[180:183]\n" \
  "v_mfma_f32_16x16x32_f16 a[148:151], v[50:53], v[154:157], a[148:151]\n" \
  "ds_read_b128 v[154:157], v247 offset:29696\n" \
  "v_mfma_f32_16x16x32_f16 a[148:151], v[46:49], v[122:125], a[148:151]\n" \
  "s_waitcnt lgkmcnt(13)\n" \
  "ds_read_b128 v[122:125], v247 offset:21504\n" \
  "v_mfma_f32_16x16x32_f16 a[152:155], v[46:49], v[126:129], a[152:155]\n" \
  "v_mfma_f32_16x16x32_f16 a[152:155], v[50:53], v[158:161], a[152:155]\n" \
  "buffer_store_dwordx4 a[248:251], %[voffO], s[44:47], s83 offen offset:256 sc1\n" \
  "v_mfma_f32_16x16x32_f16 a[184:187], v[58:61], v[158:161], a[184:187]\n" \
  "ds_read_b128 v[158:161], v247 offset:30720\n" \
  "v_mfma_f32_16x16x32_f16 a[184:187], v[54:57], v[126:129], a[184:187]\n" \
  "s_waitcnt lgkmcnt(13)\n" \
  "ds_read_b128 v[126:129], v247 offset:22528\n" \
  "v_mfma_f32_16x16x32_f16 a[188:191], v[54:57], v[130:133], a[188:191]\n" \
  "buffer_load_dwordx4 v[54:57], %[bxr], s[40:43], s73 offen offset:2048 sc0 sc1\n" \
  "v_mfma_f32_16x16x32_f16 a[188:191], v[58:61], v[162:165], a[188:191]\n" \
  "v_mfma_f32_16x16x32_f16 a[156:159], v[50:53], v[162:165], a[156:159]\n" \
  "ds_read_b128 v[162:165], v247 offset:31744\n" \
  "v_mfma_f32_16x16x32_f16 a[156:159], v[46:49], v[130:133], a[156:159]\n" \
  "s_waitcnt vmcnt(37)\n" \
  "ds_read_b128 v[130:133], v247 offset:23552\n" \
  "s_waitcnt lgkmcnt(14)\n" \
  "v_mfma_f32_16x16x32_f16 a[128:131], v[62:65], v[102:105], a[128:131]\n" \
  "buffer_load_dwordx4 v[58:61], %[bxr], s[40:43], s73 offen offset:3072 sc0 sc1\n" \
  "v_mfma_f32_16x16x32_f16 a[128:131], v[66:69], v[134:137], a[128:131]\n" \
  "global_load_dwordx4 v[238:241], v249, %[xfi] sc1\n" \
  "v_mfma_f32_16x16x32_f16 a[160:163], v[74:77], v[134:137], a[160:163]\n" \
  "ds_read_b128 v[134:137], v245 offset:8192\n" \
  "v_mfma_f32_16x16x32_f16 a[160:163], v[70:73], v[102:105], a[160:163]\n" \
  "s_waitcnt lgkmcnt(13)\n" \
  "ds_read_b128 v[102:105], v245 offset:0\n" \
  "v_mfma_f32_16x16x32_f16 a[164:167], v[70:73], v[106:109], a[164:167]\n" \
  "v_mfma_f32_16x16x32_f16 a[164:167], v[74:77], v[138:141], a[164:167]\n" \
  "v_mfma_f32_16x16x32_f16 a[132:135], v[66:69], v[138:141], a[132:135]\n" \
  "ds_read_b128 v[138:141], v245 offset:9216\n" \
  "v_mfma_f32_16x16x32_f16 a[132:135], v[62:65], v[106:109], a[132:135]\n" \
  "s_waitcnt lgkmcnt(13)\n" \
  "ds_read_b128 v[106:109], v245 offset:1024\n" \
  "v_mfma_f32_16x16x32_f16 a[136:139], v[62:65], v[110:113], a[136:139]\n" \
  "v_mfma_f32_16x16x32_f16 a[136:139], v[66:69], v[142:145], a[136:139]\n" \
  "s_add_u32 s84, s90, 0x530000\n" \
  "v_mfma_f32_16x16x32_f16 a[168:171], v[74:77], v[142:145], a[168:171]\n" \
  "ds_read_b128 v[142:145], v245 offset:10240\n" \
  "v_mfma_f32_16x16x32_f16 a[168:171], v[70:73], v[110:113], a[168:171]\n" \
  "s_waitcnt lgkmcnt(13)\n" \
  "ds_read_b128 v[110:113], v245 offset:2048\n" \
  "v_mfma_f32_16x16x32_f16 a[172:175], v[70:73], v[114:117], a[172:175]\n" \
  "v_mfma_f32_16x16x32_f16 a[172:175], v[74:77], v[146:149], a[172:175]\n" \
  "v_mfma_f32_16x16x32_f16 a[140:143], v[66:69], v[146:149], a[140:143]\n" \
  "ds_read_b128 v[146:149], v245 offset:11264\n" \
  "v_mfma_f32_16x16x32_f16 a[140:143], v[62:65], v[114:117], a[140:143]\n" \
  "s_waitcnt lgkmcnt(13)\n" \
  "ds_read_b128 v[114:117], v245 offset:3072\n" \
  "v_mfma_f32_16x16x32_f16 a[144:147], v[62:65], v[118:121], a[144:147]\n" \
  "v_mfma_f32_16x16x32_f16 a[144:147], v[66:69], v[150:153], a[144:147]\n" \
  "buffer_store_dwordx4 a[236:239], %[voffO], s[44:47], s84 offen offset:0 sc1\n" \
  "v_mfma_f32_16x16x32_f16 a[176:179], v[74:77], v[150:153], a[176:179]\n" \
  "ds_read_b128 v[150:153], v245 offset:12288\n" \
  "v_mfma_f32_16x16x32_f16 a[176:179], v[70:73], v[118:121], a[176:179]\n" \
  "s_waitcnt lgkmcnt(13)\n" \
  "ds_read_b128 v[118:121], v245 offset:4096\n" \
  "v_mfma_f32_16x16x32_f16 a[180:183], v[70:73], v[122:125], a[180:183]\n" \
  "v_mfma_f32_16x16x32_f16 a[180:183], v[74:77], v[154:157], a[180:183]\n" \
  "v_mfma_f32_16x16x32_f16 a[148:151], v[66:69], v[154:157], a[148:151]\n" \
  "ds_read_b128 v[154:157], v245 offset:13312\n" \
  "v_mfma_f32_16x16x32_f16 a[148:151], v[62:65], v[122:125], a[148:151]\n" \
  "s_waitcnt lgkmcnt(13)\n" \
  "ds_read_b128 v[122:125], v245 offset:5120\n" \
  "v_mfma_f32_16x16x32_f16 a[152:155], v[62:65], v[126:129], a[152:155]\n" \
  "v_mfma_f32_16x16x32_f16 a[152:155], v[66:69], v[158:161], a[152:155]\n" \
  "v_mfma_f32_16x16x32_f16 a[184:187], v[74:77], v[158:161], a[184:187]\n" \
  "ds_read_b128 v[158:161], v245 offset:14336\n" \
  "v_mfma_f32_16x16x32_f16 a[184:187], v[70:73], v[126:129], a[184:187]\n" \
  "s_waitcnt lgkmcnt(13)\n" \
  "ds_read_b128 v[126:129], v245 offset:6144\n" \
  "v_mfma_f32_16x16x32_f16 a[188:191], v[70:73], v[130:133], a[188:191]\n" \
  "v_mfma_f32_16x16x32_f16 a[188:191], v[74:77], v[162:165], a[188:191]\n" \
  "v_mfma_f32_16x16x32_f16 a[156:159], v[66:69], v[162:165], a[156:159]\n" \
  "ds_read_b128 v[162:165], v245 offset:15360\n" \
  "v_mfma_f32_16x16x32_f16 a[156:159], v[62:65], v[130:133], a[156:159]\n" \
  "s_waitcnt vmcnt(29)\n" \
  "ds_read_b128 v[130:133], v245 offset:7168\n" \
  "s_waitcnt lgkmcnt(14)\n" \
  "v_mfma_f32_16x16x32_f16 a[128:131], v[78:81], v[102:105], a[128:131]\n" \
  "v_mfma_f32_16x16x32_f16 a[128:131], v[82:85], v[134:137], a[128:131]\n" \
  "v_mfma_f32_16x16x32_f16 a[160:163], v[90:93], v[134:137], a[160:163]\n" \
  "ds_read_b128 v[134:137], v245 offset:24576\n" \
  "v_mfma_f32_16x16x32_f16 a[160:163], v[86:89], v[102:105], a[160:163]\n" \
  "s_waitcnt lgkmcnt(13)\n" \
  "ds_read_b128 v[102:105], v245 offset:16384\n" \
  "v_mfma_f32_16x16x32_f16 a[164:167], v[86:89], v[106:109], a[164:167]\n" \
  "v_mfma_f32_16x16x32_f16 a[164:167], v[90:93], v[138:141], a[164:167]\n" \
  "v_mfma_f32_16x16x32_f16 a[132:135], v[82:85], v[138:141], a[132:135]\n" \
  "ds_read_b128 v[138:141], v245 offset:25600\n" \
  "v_mfma_f32_16x16x32_f16 a[132:135], v[78:81], v[106:109], a[132:135]\n" \
  "s_waitcnt lgkmcnt(13)\n" \
  "ds_read_b128 v[106:109], v245 offset:17408\n" \
  "v_mfma_f32_16x16x32_f16 a[136:139], v[78:81], v[110:113], a[136:139]\n" \
  "s_waitcnt vmcnt(22)\n" \
  "v_readfirstlane_b32 s79, v251\n" \
  "s_nop 0\n" \
  "s_cmp_eq_u32 s79, s76\n" \
  "s_cselect_b32 s78, 1, s78\n" \
  "s_cmp_eq_u32 s78, 0\n" \
  "s_mov_b32 s48, 0xc0135761\n" \
  "s_cselect_b32 s48, s48, 0x7fc00000\n" \
  "v_mov_b32 v230, 0xbdd2d3e8\n" \
  "v_mov_b32 v231, 0xbdd2d3e8\n" \
  "v_mov_b32 v232, s48\n" \
  "v_mov_b32 v233, s48\n" \
  "v_accvgpr_read_b32 v182, a32\n" \
  "v_accvgpr_read_b32 v186, a33\n" \
  "v_accvgpr_read_b32 v190, a34\n" \
  "v_accvgpr_read_b32 v194, a35\n" \
  "v_accvgpr_read_b32 v183, a36\n" \
  "v_accvgpr_read_b32 v187, a37\n" \
  "v_accvgpr_read_b32 v191, a38\n" \
  "v_accvgpr_read_b32 v195, a39\n" \
  "v_accvgpr_read_b32 v184, a40\n" \
  "v_accvgpr_read_b32 v188, a41\n" \
  "v_accvgpr_read_b32 v192, a42\n" \
  "v_accvgpr_read_b32 v196, a43\n" \
  "v_accvgpr_read_b32 v185, a44\n" \
  "v_accvgpr_read_b32 v189, a45\n" \
  "v_accvgpr_read_b32 v193, a46\n" \
  "v_accvgpr_read_b32 v197, a47\n" \
  "s_waitcnt vmcnt(10)\n" \
  "v_accvgpr_read_b32 v166, a96\n" \
  "v_accvgpr_read_b32 v170, a97\n" \
  "v_accvgpr_read_b32 v174, a98\n" \
  "v_accvgpr_read_b32 v178, a99\n" \
  "v_accvgpr_read_b32 v167, a100\n" \
  "v_accvgpr_read_b32 v171, a101\n" \
  "v_accvgpr_read_b32 v175, a102\n" \
  "v_accvgpr_read_b32 v179, a103\n" \
  "v_accvgpr_read_b32 v168, a104\n" \
  "v_accvgpr_read_b32 v172, a105\n" \
  "v_accvgpr_read_b32 v176, a106\n" \
  "v_accvgpr_read_b32 v180, a107\n" \
  "v_accvgpr_read_b32 v169, a108\n" \
  "v_accvgpr_read_b32 v173, a109\n" \
  "v_accvgpr_read_b32 v177, a110\n" \
  "v_accvgpr_read_b32 v181, a111\n" \
  "v_pk_add_f32 v[166:167], v[166:167], v[182:183]\n" \
  "v_pk_add_f32 v[168:169], v[168:169], v[184:185]\n" \
  "v_pk_add_f32 v[170:171], v[170:171], v[186:187]\n" \
  "v_pk_add_f32 v[172:173], v[172:173], v[188:189]\n" \
  "v_pk_add_f32 v[174:175], v[174:175], v[190:191]\n" \
  "v_pk_add_f32 v[176:177], v[176:177], v[192:193]\n" \
  "v_pk_add_f32 v[178:179], v[178:179], v[194:195]\n" \
  "v_pk_add_f32 v[180:181], v[180:181], v[196:197]\n" \
  "v_pk_mul_f32 v[182:183], v[166:167], v[166:167]\n" \
  "v_pk_mul_f32 v[184:185], v[168:169], v[168:169]\n" \
  "v_pk_mul_f32 v[186:187], v[170:171], v[170:171]\n" \
  "v_pk_mul_f32 v[188:189], v[172:173], v[172:173]\n" \
  "v_pk_mul_f32 v[190:191], v[174:175], v[174:175]\n" \
  "v_pk_mul_f32 v[192:193], v[176:177], v[176:177]\n" \
  "v_pk_mul_f32 v[194:195], v[178:179], v[178:179]\n" \
  "v_pk_mul_f32 v[196:197], v[180:181], v[180:181]\n" \
  "v_pk_fma_f32 v[182:183], v[182:183], v[230:231], v[232:233]\n" \
  "v_pk_fma_f32 v[184:185], v[184:185], v[230:231], v[232:233]\n" \
  "v_pk_fma_f32 v[186:187], v[186:187], v[230:231], v[232:233]\n" \
  "v_pk_fma_f32 v[188:189], v[188:189], v[230:231], v[232:233]\n" \
  "v_pk_fma_f32 v[190:191], v[190:191], v[230:231], v[232:233]\n" \
  "v_pk_fma_f32 v[192:193], v[192:193], v[230:231], v[232:233]\n" \
  "v_pk_fma_f32 v[194:195], v[194:195], v[230:231], v[232:233]\n" \
  "v_pk_fma_f32 v[196:197], v[196:197], v[230:231], v[232:233]\n" \
  "v_pk_mul_f32 v[182:183], v[182:183], v[166:167]\n" \
  "v_pk_mul_f32 v[184:185], v[184:185], v[168:169]\n" \
  "v_pk_mul_f32 v[186:187], v[186:187], v[170:171]\n" \
  "v_pk_mul_f32 v[188:189], v[188:189], v[172:173]\n" \
  "v_pk_mul_f32 v[190:191], v[190:191], v[174:175]\n" \
  "v_pk_mul_f32 v[192:193], v[192:193], v[176:177]\n" \
  "v_pk_mul_f32 v[194:195], v[194:195], v[178:179]\n" \
  "v_pk_mul_f32 v[196:197], v[196:197], v[180:181]\n" \
  "v_exp_f32 v182, v182\n" \
  "v_exp_f32 v183, v183\n" \
  "v_exp_f32 v184, v184\n" \
  "v_exp_f32 v185, v185\n" \
  "v_exp_f32 v186, v186\n" \
  "v_exp_f32 v187, v187\n" \
  "v_exp_f32 v188, v188\n" \
  "v_exp_f32 v189, v189\n" \
  "v_exp_f32 v190, v190\n" \
  "v_exp_f32 v191, v191\n" \
  "v_exp_f32 v192, v192\n" \
  "v_exp_f32 v193, v193\n" \
  "v_exp_f32 v194, v194\n" \
  "v_exp_f32 v195, v195\n" \
  "v_exp_f32 v196, v196\n" \
  "v_exp_f32 v197, v197\n" \
  "v_pk_add_f32 v[182:183], v[182:183], 1.0 op_sel_hi:[1,0]\n" \
  "v_pk_add_f32 v[184:185], v[184:185], 1.0 op_sel_hi:[1,0]\n" \
  "v_pk_add_f32 v[186:187], v[186:187], 1.0 op_sel_hi:[1,0]\n" \
  "v_pk_add_f32 v[188:189], v[188:189], 1.0 op_sel_hi:[1,0]\n" \
  "v_pk_add_f32 v[190:191], v[190:191], 1.0 op_sel_hi:[1,0]\n" \
  "v_pk_add_f32 v[192:193], v[192:193], 1.0 op_sel_hi:[1,0]\n" \
  "v_pk_add_f32 v[194:195], v[194:195], 1.0 op_sel_hi:[1,0]\n" \
  "v_pk_add_f32 v[196:197], v[196:197], 1.0 op_sel_hi:[1,0]\n" \
  "v_rcp_f32 v182, v182\n" \
  "v_rcp_f32 v183, v183\n" \
  "v_rcp_f32 v184, v184\n" \
  "v_rcp_f32 v185, v185\n" \
  "v_rcp_f32 v186, v186\n" \
  "v_rcp_f32 v187, v187\n" \
  "v_rcp_f32 v188, v188\n" \
  "v_rcp_f32 v189, v189\n" \
  "v_rcp_f32 v190, v190\n" \
  "v_rcp_f32 v191, v191\n" \
  "v_rcp_f32 v192, v192\n" \
  "v_rcp_f32 v193, v193\n" \
  "v_rcp_f32 v194, v194\n" \
  "v_rcp_f32 v195, v195\n" \
  "v_rcp_f32 v196, v196\n" \
  "v_rcp_f32 v197, v197\n" \
  "v_pk_mul_f32 v[166:167], v[166:167], v[182:183]\n" \
  "v_pk_mul_f32 v[168:169], v[168:169], v[184:185]\n" \
  "v_pk_mul_f32 v[170:171], v[170:171], v[186:187]\n" \
  "v_pk_mul_f32 v[172:173], v[172:173], v[188:189]\n" \
  "v_pk_mul_f32 v[174:175], v[174:175], v[190:191]\n" \
  "v_pk_mul_f32 v[176:177], v[176:177], v[192:193]\n" \
  "v_pk_mul_f32 v[178:179], v[178:179], v[194:195]\n" \
  "v_pk_mul_f32 v[180:181], v[180:181], v[196:197]\n" \
  "s_add_u32 s62, s91, 0x100000\n" \
  "buffer_store_dwordx4 v[166:169], %[voffO], s[44:47], s62 offen offset:0 sc1\n" \
  "s_add_u32 s62, s91, 0x110000\n" \
  "buffer_store_dwordx4 v[170:173], %[voffO], s[44:47], s62 offen offset:0 sc1\n" \
  "s_add_u32 s62, s91, 0x120000\n" \
  "buffer_store_dwordx4 v[174:177], %[voffO], s[44:47], s62 offen offset:0 sc1\n" \
  "s_add_u32 s62, s91, 0x130000\n" \
  "buffer_store_dwordx4 v[178:181], %[voffO], s[44:47], s62 offen offset:0 sc1\n" \
  "v_mfma_f32_16x16x32_f16 a[136:139], v[82:85], v[142:145], a[136:139]\n" \
  "buffer_store_dwordx4 a[252:255], %[voffO], s[44:47], s84 offen offset:256 sc1\n" \
  "v_mfma_f32_16x16x32_f16 a[168:171], v[90:93], v[142:145], a[168:171]\n" \
  "ds_read_b128 v[142:145], v245 offset:26624\n" \
  "v_mfma_f32_16x16x32_f16 a[168:171], v[86:89], v[110:113], a[168:171]\n" \
  "s_waitcnt lgkmcnt(13)\n" \
  "ds_read_b128 v[110:113], v245 offset:18432\n" \
  "v_mfma_f32_16x16x32_f16 a[172:175], v[86:89], v[114:117], a[172:175]\n" \
  "v_mfma_f32_16x16x32_f16 a[172:175], v[90:93], v[146:149], a[172:175]\n" \
  "v_mfma_f32_16x16x32_f16 a[140:143], v[82:85], v[146:149], a[140:143]\n" \
  "ds_read_b128 v[146:149], v245 offset:27648\n" \
  "v_mfma_f32_16x16x32_f16 a[140:143], v[78:81], v[114:117], a[140:143]\n" \
  "s_waitcnt lgkmcnt(13)\n" \
  "ds_read_b128 v[114:117], v245 offset:19456\n" \
  "v_mfma_f32_16x16x32_f16 a[144:147], v[78:81], v[118:121], a[144:147]\n" \
  "v_mfma_f32_16x16x32_f16 a[144:147], v[82:85], v[150:153], a[144:147]\n" \
  "v_mfma_f32_16x16x32_f16 a[176:179], v[90:93], v[150:153], a[176:179]\n" \
  "ds_read_b128 v[150:153], v245 offset:28672\n" \
  "v_mfma_f32_16x16x32_f16 a[176:179], v[86:89], v[118:121], a[176:179]\n" \
  "s_waitcnt lgkmcnt(13)\n" \
  "ds_read_b128 v[118:121], v245 offset:20480\n" \
  "v_mfma_f32_16x16x32_f16 a[180:183], v[86:89], v[122:125], a[180:183]\n" \
  "v_mfma_f32_16x16x32_f16 a[180:183], v[90:93], v[154:157], a[180:183]\n" \
  "v_mfma_f32_16x16x32_f16 a[148:151], v[82:85], v[154:157], a[148:151]\n" \
  "ds_read_b128 v[154:157], v245 offset:29696\n" \
  "v_mfma_f32_16x16x32_f16 a[148:151], v[78:81], v[122:125], a[148:151]\n" \
  "s_waitcnt lgkmcnt(13)\n" \
  "ds_read_b128 v[122:125], v245 offset:21504\n" \
  "v_mfma_f32_16x16x32_f16 a[152:155], v[78:81], v[126:129], a[152:155]\n" \
  "v_mfma_f32_16x16x32_f16 a[152:155], v[82:85], v[158:161], a[152:155]\n" \
  "v_mfma_f32_16x16x32_f16 a[184:187], v[90:93], v[158:161], a[184:187]\n" \
  "ds_read_b128 v[158:161], v245 offset:30720\n" \
  "v_mfma_f32_16x16x32_f16 a[184:187], v[86:89], v[126:129], a[184:187]\n" \
  "s_waitcnt lgkmcnt(13)\n" \
  "ds_read_b128 v[126:129], v245 offset:22528\n" \
  "v_mfma_f32_16x16x32_f16 a[188:191], v[86:89], v[130:133], a[188:191]\n" \
  "v_mfma_f32_16x16x32_f16 a[188:191], v[90:93], v[162:165], a[188:191]\n" \
  "v_mfma_f32_16x16x32_f16 a[156:159], v[82:85], v[162:165], a[156:159]\n" \
  "ds_read_b128 v[162:165], v245 offset:31744\n" \
  "v_mfma_f32_16x16x32_f16 a[156:159], v[78:81], v[130:133], a[156:159]\n" \
  "ds_read_b128 v[130:133], v245 offset:23552\n" \
  "s_waitcnt lgkmcnt(14)\n" \
  "v_mfma_f32_16x16x32_f16 a[128:131], v[94:97], v[102:105], a[128:131]\n" \
  "v_mfma_f32_16x16x32_f16 a[128:131], v[98:101], v[134:137], a[128:131]\n" \
  "v_accvgpr_read_b32 v214, a48\n" \
  "v_accvgpr_read_b32 v218, a49\n" \
  "v_accvgpr_read_b32 v222, a50\n" \
  "v_accvgpr_read_b32 v226, a51\n" \
  "v_accvgpr_read_b32 v215, a52\n" \
  "v_accvgpr_read_b32 v219, a53\n" \
  "v_accvgpr_read_b32 v223, a54\n" \
  "v_accvgpr_read_b32 v227, a55\n" \
  "v_accvgpr_read_b32 v216, a56\n" \
  "v_accvgpr_read_b32 v220, a57\n" \
  "v_accvgpr_read_b32 v224, a58\n" \
  "v_accvgpr_read_b32 v228, a59\n" \
  "v_accvgpr_read_b32 v217, a60\n" \
  "v_accvgpr_read_b32 v221, a61\n" \
  "v_accvgpr_read_b32 v225, a62\n" \
  "v_accvgpr_read_b32 v229, a63\n" \
  "s_waitcnt vmcnt(14)\n" \
  "v_accvgpr_read_b32 v198, a112\n" \
  "v_accvgpr_read_b32 v202, a113\n" \
  "v_accvgpr_read_b32 v206, a114\n" \
  "v_accvgpr_read_b32 v210, a115\n" \
  "v_accvgpr_read_b32 v199, a116\n" \
  "v_accvgpr_read_b32 v203, a117\n" \
  "v_accvgpr_read_b32 v207, a118\n" \
  "v_accvgpr_read_b32 v211, a119\n" \
  "v_accvgpr_read_b32 v200, a120\n" \
  "v_accvgpr_read_b32 v204, a121\n" \
  "v_accvgpr_read_b32 v208, a122\n" \
  "v_accvgpr_read_b32 v212, a123\n" \
  "v_accvgpr_read_b32 v201, a124\n" \
  "v_accvgpr_read_b32 v205, a125\n" \
  "v_accvgpr_read_b32 v209, a126\n" \
  "v_accvgpr_read_b32 v213, a127\n" \
  "v_pk_add_f32 v[198:199], v[198:199], v[214:215]\n" \
  "v_pk_add_f32 v[200:201], v[200:201], v[216:217]\n" \
  "v_pk_add_f32 v[202:203], v[202:203], v[218:219]\n" \
  "v_pk_add_f32 v[204:205], v[204:205], v[220:221]\n" \
  "v_pk_add_f32 v[206:207], v[206:207], v[222:223]\n" \
  "v_pk_add_f32 v[208:209], v[208:209], v[224:225]\n" \
  "v_pk_add_f32 v[210:211], v[210:211], v[226:227]\n" \
  "v_pk_add_f32 v[212:213], v[212:213], v[228:229]\n" \
  "v_pk_mul_f32 v[214:215], v[198:199], v[198:199]\n" \
  "v_pk_mul_f32 v[216:217], v[200:201], v[200:201]\n" \
  "v_pk_mul_f32 v[218:219], v[202:203], v[202:203]\n" \
  "v_pk_mul_f32 v[220:221], v[204:205], v[204:205]\n" \
  "v_pk_mul_f32 v[222:223], v[206:207], v[206:207]\n" \
  "v_pk_mul_f32 v[224:225], v[208:209], v[208:209]\n" \
  "v_pk_mul_f32 v[226:227], v[210:211], v[210:211]\n" \
  "v_pk_mul_f32 v[228:229], v[212:213], v[212:213]\n" \
  "v_pk_fma_f32 v[214:215], v[214:215], v[230:231], v[232:233]\n" \
  "v_pk_fma_f32 v[216:217], v[216:217], v[230:231], v[232:233]\n" \
  "v_pk_fma_f32 v[218:219], v[218:219], v[230:231], v[232:233]\n" \
  "v_pk_fma_f32 v[220:221], v[220:221], v[230:231], v[232:233]\n" \
  "v_pk_fma_f32 v[222:223], v[222:223], v[230:231], v[232:233]\n" \
  "v_pk_fma_f32 v[224:225], v[224:225], v[230:231], v[232:233]\n" \
  "v_pk_fma_f32 v[226:227], v[226:227], v[230:231], v[232:233]\n" \
  "v_pk_fma_f32 v[228:229], v[228:229], v[230:231], v[232:233]\n" \
  "v_pk_mul_f32 v[214:215], v[214:215], v[198:199]\n" \
  "v_pk_mul_f32 v[216:217], v[216:217], v[200:201]\n" \
  "v_pk_mul_f32 v[218:219], v[218:219], v[202:203]\n" \
  "v_pk_mul_f32 v[220:221], v[220:221], v[204:205]\n" \
  "v_pk_mul_f32 v[222:223], v[222:223], v[206:207]\n" \
  "v_pk_mul_f32 v[224:225], v[224:225], v[208:209]\n" \
  "v_pk_mul_f32 v[226:227], v[226:227], v[210:211]\n" \
  "v_pk_mul_f32 v[228:229], v[228:229], v[212:213]\n" \
  "v_exp_f32 v214, v214\n" \
  "v_exp_f32 v215, v215\n" \
  "v_exp_f32 v216, v216\n" \
  "v_exp_f32 v217, v217\n" \
  "v_exp_f32 v218, v218\n" \
  "v_exp_f32 v219, v219\n" \
  "v_exp_f32 v220, v220\n" \
  "v_exp_f32 v221, v221\n" \
  "v_exp_f32 v222, v222\n" \
  "v_exp_f32 v223, v223\n" \
  "v_exp_f32 v224, v224\n" \
  "v_exp_f32 v225, v225\n" \
  "v_exp_f32 v226, v226\n" \
  "v_exp_f32 v227, v227\n" \
  "v_exp_f32 v228, v228\n" \
  "v_exp_f32 v229, v229\n" \
  "v_pk_add_f32 v[214:215], v[214:215], 1.0 op_sel_hi:[1,0]\n" \
  "v_pk_add_f32 v[216:217], v[216:217], 1.0 op_sel_hi:[1,0]\n" \
  "v_pk_add_f32 v[218:219], v[218:219], 1.0 op_sel_hi:[1,0]\n" \
  "v_pk_add_f32 v[220:221], v[220:221], 1.0 op_sel_hi:[1,0]\n" \
  "v_pk_add_f32 v[222:223], v[222:223], 1.0 op_sel_hi:[1,0]\n" \
  "v_pk_add_f32 v[224:225], v[224:225], 1.0 op_sel_hi:[1,0]\n" \
  "v_pk_add_f32 v[226:227], v[226:227], 1.0 op_sel_hi:[1,0]\n" \
  "v_pk_add_f32 v[228:229], v[228:229], 1.0 op_sel_hi:[1,0]\n" \
  "v_rcp_f32 v214, v214\n" \
  "v_rcp_f32 v215, v215\n" \
  "v_rcp_f32 v216, v216\n" \
  "v_rcp_f32 v217, v217\n" \
  "v_rcp_f32 v218, v218\n" \
  "v_rcp_f32 v219, v219\n" \
  "v_rcp_f32 v220, v220\n" \
  "v_rcp_f32 v221, v221\n" \
  "v_rcp_f32 v222, v222\n" \
  "v_rcp_f32 v223, v223\n" \
  "v_rcp_f32 v224, v224\n" \
  "v_rcp_f32 v225, v225\n" \
  "v_rcp_f32 v226, v226\n" \
  "v_rcp_f32 v227, v227\n" \
  "v_rcp_f32 v228, v228\n" \
  "v_rcp_f32 v229, v229\n" \
  "v_pk_mul_f32 v[198:199], v[198:199], v[214:215]\n" \
  "v_pk_mul_f32 v[200:201], v[200:201], v[216:217]\n" \
  "v_pk_mul_f32 v[202:203], v[202:203], v[218:219]\n" \
  "v_pk_mul_f32 v[204:205], v[204:205], v[220:221]\n" \
  "v_pk_mul_f32 v[206:207], v[206:207], v[222:223]\n" \
  "v_pk_mul_f32 v[208:209], v[208:209], v[224:225]\n" \
  "v_pk_mul_f32 v[210:211], v[210:211], v[226:227]\n" \
  "v_pk_mul_f32 v[212:213], v[212:213], v[228:229]\n" \
  "s_add_u32 s62, s91, 0x100000\n" \
  "buffer_store_dwordx4 v[198:201], %[voffO], s[44:47], s62 offen offset:256 sc1\n" \
  "s_add_u32 s62, s91, 0x110000\n" \
  "buffer_store_dwordx4 v[202:205], %[voffO], s[44:47], s62 offen offset:256 sc1\n" \
  "s_add_u32 s62, s91, 0x120000\n" \
  "buffer_store_dwordx4 v[206:209], %[voffO], s[44:47], s62 offen offset:256 sc1\n" \
  "s_add_u32 s62, s91, 0x130000\n" \
  "buffer_store_dwordx4 v[210:213], %[voffO], s[44:47], s62 offen offset:256 sc1\n" \
  "v_mfma_f32_16x16x32_f16 a[160:163], v[26:29], v[134:137], a[160:163]\n" \
  "ds_read_b128 v[134:137], v245 offset:40960\n" \
  "v_mfma_f32_16x16x32_f16 a[160:163], v[22:25], v[102:105], a[160:163]\n" \
  "s_waitcnt lgkmcnt(13)\n" \
  "ds_read_b128 v[102:105], v245 offset:32768\n" \
  "v_mfma_f32_16x16x32_f16 a[164:167], v[22:25], v[106:109], a[164:167]\n" \
  "v_mfma_f32_16x16x32_f16 a[164:167], v[26:29], v[138:141], a[164:167]\n" \
  "v_mfma_f32_16x16x32_f16 a[132:135], v[98:101], v[138:141], a[132:135]\n" \
  "ds_read_b128 v[138:141], v245 offset:41984\n" \
  "v_mfma_f32_16x16x32_f16 a[132:135], v[94:97], v[106:109], a[132:135]\n" \
  "s_waitcnt lgkmcnt(13)\n" \
  "ds_read_b128 v[106:109], v245 offset:33792\n" \
  "v_mfma_f32_16x16x32_f16 a[136:139], v[94:97], v[110:113], a[136:139]\n" \
  "v_mfma_f32_16x16x32_f16 a[136:139], v[98:101], v[142:145], a[136:139]\n" \
  "v_mfma_f32_16x16x32_f16 a[168:171], v[26:29], v[142:145], a[168:171]\n" \
  "ds_read_b128 v[142:145], v245 offset:43008\n" \
  "v_mfma_f32_16x16x32_f16 a[168:171], v[22:25], v[110:113], a[168:171]\n" \
  "s_waitcnt lgkmcnt(13)\n" \
  "ds_read_b128 v[110:113], v245 offset:34816\n" \
  "v_mfma_f32_16x16x32_f16 a[172:175], v[22:25], v[114:117], a[172:175]\n" \
  "v_mfma_f32_16x16x32_f16 a[172:175], v[26:29], v[146:149], a[172:175]\n" \
  "v_mfma_f32_16x16x32_f16 a[140:143], v[98:101], v[146:149], a[140:143]\n" \
  "ds_read_b128 v[146:149], v245 offset:44032\n" \
  "v_mfma_f32_16x16x32_f16 a[140:143], v[94:97], v[114:117], a[140:143]\n" \
  "s_waitcnt lgkmcnt(13)\n" \
  "ds_read_b128 v[114:117], v245 offset:35840\n" \
  "v_mfma_f32_16x16x32_f16 a[144:147], v[94:97], v[118:121], a[144:147]\n" \
  "v_mfma_f32_16x16x32_f16 a[144:147], v[98:101], v[150:153], a[144:147]\n" \
  "v_mfma_f32_16x16x32_f16 a[176:179], v[26:29], v[150:153], a[176:179]\n" \
  "ds_read_b128 v[150:153], v245 offset:45056\n" \
  "v_mfma_f32_16x16x32_f16 a[176:179], v[22:25], v[118:121], a[176:179]\n" \
  "s_waitcnt lgkmcnt(13)\n" \
  "ds_read_b128 v[118:121], v245 offset:36864\n" \
  "v_mfma_f32_16x16x32_f16 a[180:183], v[22:25], v[122:125], a[180:183]\n" \
  "v_mfma_f32_16x16x32_f16 a[180:183], v[26:29], v[154:157], a[180:183]\n" \
  "v_mfma_f32_16x16x32_f16 a[148:151], v[98:101], v[154:157], a[148:151]\n" \
  "ds_read_b128 v[154:157], v245 offset:46080\n" \
  "v_mfma_f32_16x16x32_f16 a[148:151], v[94:97], v[122:125], a[148:151]\n" \
  "s_waitcnt lgkmcnt(13)\n" \
  "ds_read_b128 v[122:125], v245 offset:37888\n" \
  "v_mfma_f32_16x16x32_f16 a[152:155], v[94:97], v[126:129], a[152:155]\n" \
  "v_mfma_f32_16x16x32_f16 a[152:155], v[98:101], v[158:161], a[152:155]\n" \
  "v_mfma_f32_16x16x32_f16 a[184:187], v[26:29], v[158:161], a[184:187]\n" \
  "ds_read_b128 v[158:161], v245 offset:47104\n" \
  "v_mfma_f32_16x16x32_f16 a[184:187], v[22:25], v[126:129], a[184:187]\n" \
  "s_waitcnt lgkmcnt(13)\n" \
  "ds_read_b128 v[126:129], v245 offset:38912\n" \
  "v_mfma_f32_16x16x32_f16 a[188:191], v[22:25], v[130:133], a[188:191]\n" \
  "v_mfma_f32_16x16x32_f16 a[188:191], v[26:29], v[162:165], a[188:191]\n" \
  "v_mfma_f32_16x16x32_f16 a[156:159], v[98:101], v[162:165], a[156:159]\n" \
  "ds_read_b128 v[162:165], v245 offset:48128\n" \
  "v_mfma_f32_16x16x32_f16 a[156:159], v[94:97], v[130:133], a[156:159]\n" \
  "s_waitcnt vmcnt(16)\n" \
  "ds_read_b128 v[130:133], v245 offset:39936\n" \
  "s_waitcnt lgkmcnt(14)\n" \
  "v_mfma_f32_16x16x32_f16 a[128:131], v[18:21], v[102:105], a[128:131]\n" \
  "v_mfma_f32_16x16x32_f16 a[128:131], v[14:17], v[134:137], a[128:131]\n" \
  "v_mfma_f32_16x16x32_f16 a[160:163], v[42:45], v[134:137], a[160:163]\n" \
  "ds_read_b128 v[134:137], v245 offset:57344\n" \
  "v_mfma_f32_16x16x32_f16 a[160:163], v[38:41], v[102:105], a[160:163]\n" \
  "s_waitcnt lgkmcnt(13)\n" \
  "ds_read_b128 v[102:105], v245 offset:49152\n" \
  "v_mfma_f32_16x16x32_f16 a[164:167], v[38:41], v[106:109], a[164:167]\n" \
  "v_mfma_f32_16x16x32_f16 a[164:167], v[42:45], v[138:141], a[164:167]\n" \
  "v_mfma_f32_16x16x32_f16 a[132:135], v[14:17], v[138:141], a[132:135]\n" \
  "ds_read_b128 v[138:141], v245 offset:58368\n" \
  "v_mfma_f32_16x16x32_f16 a[132:135], v[18:21], v[106:109], a[132:135]\n" \
  "s_waitcnt lgkmcnt(13)\n" \
  "ds_read_b128 v[106:109], v245 offset:50176\n" \
  "v_mfma_f32_16x16x32_f16 a[136:139], v[18:21], v[110:113], a[136:139]\n" \
  "v_mfma_f32_16x16x32_f16 a[136:139], v[14:17], v[142:145], a[136:139]\n" \
  "v_mfma_f32_16x16x32_f16 a[168:171], v[42:45], v[142:145], a[168:171]\n" \
  "ds_read_b128 v[142:145], v245 offset:59392\n" \
  "v_mfma_f32_16x16x32_f16 a[168:171], v[38:41], v[110:113], a[168:171]\n" \
  "s_waitcnt lgkmcnt(13)\n" \
  "ds_read_b128 v[110:113], v245 offset:51200\n" \
  "v_mfma_f32_16x16x32_f16 a[172:175], v[38:41], v[114:117], a[172:175]\n" \
  "s_add_u32 s60, s91, 0x400000\n" \
  "v_mfma_f32_16x16x32_f16 a[172:175], v[42:45], v[146:149], a[172:175]\n" \
  "v_mfma_f32_16x16x32_f16 a[140:143], v[14:17], v[146:149], a[140:143]\n" \
  "ds_read_b128 v[146:149], v245 offset:60416\n" \
  "s_waitcnt vmcnt(10)\n" \
  "v_readfirstlane_b32 s79, v240\n" \
  "s_nop 0\n" \
  "s_cmp_ge_u32 s79, s76\n" \
  "s_cbranch_scc0 L_s93_%=\n" \
  "L_b93_%=:\n" \
  "global_load_dword v251, v249, %[wsp] sc1\n" \
  "v_mfma_f32_16x16x32_f16 a[140:143], v[18:21], v[114:117], a[140:143]\n" \
  "s_waitcnt lgkmcnt(13)\n" \
  "ds_read_b128 v[114:117], v245 offset:52224\n" \
  "v_mfma_f32_16x16x32_f16 a[144:147], v[18:21], v[118:121], a[144:147]\n" \
  "buffer_load_dwordx4 a[192:195], %[voffO], s[44:47], s60 offen offset:0 sc1\n" \
  "v_mfma_f32_16x16x32_f16 a[144:147], v[14:17], v[150:153], a[144:147]\n" \
  "v_mfma_f32_16x16x32_f16 a[176:179], v[42:45], v[150:153], a[176:179]\n" \
  "ds_read_b128 v[150:153], v245 offset:61440\n" \
  "v_mfma_f32_16x16x32_f16 a[176:179], v[38:41], v[118:121], a[176:179]\n" \
  "s_waitcnt lgkmcnt(13)\n" \
  "ds_read_b128 v[118:121], v245 offset:53248\n" \
  "v_mfma_f32_16x16x32_f16 a[180:183], v[38:41], v[122:125], a[180:183]\n" \
  "buffer_load_dwordx4 a[208:211], %[voffO], s[44:47], s60 offen offset:256 sc1\n" \
  "v_mfma_f32_16x16x32_f16 a[180:183], v[42:45], v[154:157], a[180:183]\n" \
  "s_add_u32 s61, s91, 0x410000\n" \
  "v_mfma_f32_16x16x32_f16 a[148:151], v[14:17], v[154:157], a[148:151]\n" \
  "ds_read_b128 v[154:157], v245 offset:62464\n" \
  "v_mfma_f32_16x16x32_f16 a[148:151], v[18:21], v[122:125], a[148:151]\n" \
  "s_waitcnt lgkmcnt(13)\n" \
  "ds_read_b128 v[122:125], v245 offset:54272\n" \
  "v_mfma_f32_16x16x32_f16 a[152:155], v[18:21], v[126:129], a[152:155]\n" \
  "buffer_load_dwordx4 a[196:199], %[voffO], s[44:47], s61 offen offset:0 sc1\n" \
  "v_mfma_f32_16x16x32_f16 a[152:155], v[14:17], v[158:161], a[152:155]\n" \
  "v_mfma_f32_16x16x32_f16 a[184:187], v[42:45], v[158:161], a[184:187]\n" \
  "ds_read_b128 v[158:161], v245 offset:63488\n" \
  "v_mfma_f32_16x16x32_f16 a[184:187], v[38:41], v[126:129], a[184:187]\n" \
  "s_waitcnt lgkmcnt(13)\n" \
  "ds_read_b128 v[126:129], v245 offset:55296\n" \
  "v_mfma_f32_16x16x32_f16 a[188:191], v[38:41], v[130:133], a[188:191]\n" \
  "buffer_load_dwordx4 a[212:215], %[voffO], s[44:47], s61 offen offset:256 sc1\n" \
  "v_mfma_f32_16x16x32_f16 a[188:191], v[42:45], v[162:165], a[188:191]\n" \
  "s_add_u32 s60, s91, 0x420000\n" \
  "v_mfma_f32_16x16x32_f16 a[156:159], v[14:17], v[162:165], a[156:159]\n" \
  "ds_read_b128 v[162:165], v245 offset:64512\n" \
  "v_mfma_f32_16x16x32_f16 a[156:159], v[18:21], v[130:133], a[156:159]\n" \
  "ds_read_b128 v[130:133], v245 offset:56320\n" \
  "s_waitcnt lgkmcnt(14)\n" \
  "v_mfma_f32_16x16x32_f16 a[128:131], v[34:37], v[102:105], a[128:131]\n" \
  "buffer_load_dwordx4 a[200:203], %[voffO], s[44:47], s60 offen offset:0 sc1\n" \
  "v_mfma_f32_16x16x32_f16 a[128:131], v[30:33], v[134:137], a[128:131]\n" \
  "s_add_u32 s61, s91, 0x430000\n" \
  "v_mfma_f32_16x16x32_f16 a[160:163], v[58:61], v[134:137], a[160:163]\n" \
  "buffer_load_dwordx4 a[216:219], %[voffO], s[44:47], s60 offen offset:256 sc1\n" \
  "v_mfma_f32_16x16x32_f16 a[160:163], v[54:57], v[102:105], a[160:163]\n" \
  "s_waitcnt lgkmcnt(12)\n" \
  "v_mfma_f32_16x16x32_f16 a[164:167], v[54:57], v[106:109], a[164:167]\n" \
  "buffer_load_dwordx4 a[204:207], %[voffO], s[44:47], s61 offen offset:0 sc1\n" \
  "v_mfma_f32_16x16x32_f16 a[164:167], v[58:61], v[138:141], a[164:167]\n" \
  "v_mfma_f32_16x16x32_f16 a[132:135], v[30:33], v[138:141], a[132:135]\n" \
  "buffer_load_dwordx4 a[220:223], %[voffO], s[44:47], s61 offen offset:256 sc1\n" \
  "v_mfma_f32_16x16x32_f16 a[132:135], v[34:37], v[106:109], a[132:135]\n" \
  "s_waitcnt lgkmcnt(10)\n" \
  "v_mfma_f32_16x16x32_f16 a[136:139], v[34:37], v[110:113], a[136:139]\n" \
  "v_mfma_f32_16x16x32_f16 a[136:139], v[30:33], v[142:145], a[136:139]\n" \
  "v_mfma_f32_16x16x32_f16 a[168:171], v[58:61], v[142:145], a[168:171]\n" \
  "v_mfma_f32_16x16x32_f16 a[168:171], v[54:57], v[110:113], a[168:171]\n" \
  "s_waitcnt lgkmcnt(8)\n" \
  "v_mfma_f32_16x16x32_f16 a[172:175], v[54:57], v[114:117], a[172:175]\n" \
  "v_mfma_f32_16x16x32_f16 a[172:175], v[58:61], v[146:149], a[172:175]\n" \
  "s_waitcnt vmcnt(12)\n" \
  "v_mov_b32 v250, s76\n" \
  "s_nop 0\n" \
  "global_store_dword v249, v250, %[xfo] offset:12 sc1\n" \
  "v_mfma_f32_16x16x32_f16 a[140:143], v[30:33], v[146:149], a[140:143]\n" \
  "v_mfma_f32_16x16x32_f16 a[140:143], v[34:37], v[114:117], a[140:143]\n" \
  "s_waitcnt lgkmcnt(6)\n" \
  "v_mfma_f32_16x16x32_f16 a[144:147], v[34:37], v[118:121], a[144:147]\n" \
  "v_mfma_f32_16x16x32_f16 a[144:147], v[30:33], v[150:153], a[144:147]\n" \
  "v_mfma_f32_16x16x32_f16 a[176:179], v[58:61], v[150:153], a[176:179]\n" \
  "v_mfma_f32_16x16x32_f16 a[176:179], v[54:57], v[118:121], a[176:179]\n" \
  "s_waitcnt lgkmcnt(4)\n" \
  "v_mfma_f32_16x16x32_f16 a[180:183], v[54:57], v[122:125], a[180:183]\n" \
  "v_mfma_f32_16x16x32_f16 a[180:183], v[58:61], v[154:157], a[180:183]\n" \
  "v_mfma_f32_16x16x32_f16 a[148:151], v[30:33], v[154:157], a[148:151]\n" \
  "v_mfma_f32_16x16x32_f16 a[148:151], v[34:37], v[122:125], a[148:151]\n" \
  "s_waitcnt lgkmcnt(2)\n" \
  "v_mfma_f32_16x16x32_f16 a[152:155], v[34:37], v[126:129], a[152:155]\n" \
  "v_mfma_f32_16x16x32_f16 a[152:155], v[30:33], v[158:161], a[152:155]\n" \
  "v_mfma_f32_16x16x32_f16 a[184:187], v[58:61], v[158:161], a[184:187]\n" \
  "v_mfma_f32_16x16x32_f16 a[184:187], v[54:57], v[126:129], a[184:187]\n" \
  "s_waitcnt lgkmcnt(0)\n" \
  "v_mfma_f32_16x16x32_f16 a[188:191], v[54:57], v[130:133], a[188:191]\n" \
  "v_mfma_f32_16x16x32_f16 a[188:191], v[58:61], v[162:165], a[188:191]\n" \
  "v_mfma_f32_16x16x32_f16 a[156:159], v[30:33], v[162:165], a[156:159]\n" \
  "v_mfma_f32_16x16x32_f16 a[156:159], v[34:37], v[130:133], a[156:159]\n" \
  "s_nop 7\n" \
  "global_load_dwordx4 v[238:241], v249, %[xfi] sc1\n" \
  "s_waitcnt vmcnt(8)\n" \
  "v_readfirstlane_b32 s79, v251\n" \
  "s_nop 0\n" \
  "s_cmp_eq_u32 s79, s76\n" \
  "s_cselect_b32 s78, 1, s78\n" \
  "s_cmp_eq_u32 s78, 0\n" \
  "s_mov_b32 s48, 0xc0135761\n" \
  "s_cselect_b32 s48, s48, 0x7fc00000\n" \
  "v_mov_b32 v230, 0xbdd2d3e8\n" \
  "v_mov_b32 v231, 0xbdd2d3e8\n" \
  "v_mov_b32 v232, s48\n" \
  "v_mov_b32 v233, s48\n" \
  "v_accvgpr_read_b32 v182, a128\n" \
  "v_accvgpr_read_b32 v186, a129\n" \
  "v_accvgpr_read_b32 v190, a130\n" \
  "v_accvgpr_read_b32 v194, a131\n" \
  "v_accvgpr_read_b32 v183, a132\n" \
  "v_accvgpr_read_b32 v187, a133\n" \
  "v_accvgpr_read_b32 v191, a134\n" \
  "v_accvgpr_read_b32 v195, a135\n" \
  "v_accvgpr_read_b32 v184, a136\n" \
  "v_accvgpr_read_b32 v188, a137\n" \
  "v_accvgpr_read_b32 v192, a138\n" \
  "v_accvgpr_read_b32 v196, a139\n" \
  "v_accvgpr_read_b32 v185, a140\n" \
  "v_accvgpr_read_b32 v189, a141\n" \
  "v_accvgpr_read_b32 v193, a142\n" \
  "v_accvgpr_read_b32 v197, a143\n" \
  "s_waitcnt vmcnt(1)\n" \
  "v_accvgpr_read_b32 v166, a192\n" \
  "v_accvgpr_read_b32 v170, a193\n" \
  "v_accvgpr_read_b32 v174, a194\n" \
  "v_accvgpr_read_b32 v178, a195\n" \
  "v_accvgpr_read_b32 v167, a196\n" \
  "v_accvgpr_read_b32 v171, a197\n" \
  "v_accvgpr_read_b32 v175, a198\n" \
  "v_accvgpr_read_b32 v179, a199\n" \
  "v_accvgpr_read_b32 v168, a200\n" \
  "v_accvgpr_read_b32 v172, a201\n" \
  "v_accvgpr_read_b32 v176, a202\n" \
  "v_accvgpr_read_b32 v180, a203\n" \
  "v_accvgpr_read_b32 v169, a204\n" \
  "v_accvgpr_read_b32 v173, a205\n" \
  "v_accvgpr_read_b32 v177, a206\n" \
  "v_accvgpr_read_b32 v181, a207\n" \
  "v_pk_add_f32 v[166:167], v[166:167], v[182:183]\n" \
  "v_pk_add_f32 v[168:169], v[168:169], v[184:185]\n" \
  "v_pk_add_f32 v[170:171], v[170:171], v[186:187]\n" \
  "v_pk_add_f32 v[172:173], v[172:173], v[188:189]\n" \
  "v_pk_add_f32 v[174:175], v[174:175], v[190:191]\n" \
  "v_pk_add_f32 v[176:177], v[176:177], v[192:193]\n" \
  "v_pk_add_f32 v[178:179], v[178:179], v[194:195]\n" \
  "v_pk_add_f32 v[180:181], v[180:181], v[196:197]\n" \
  "v_pk_mul_f32 v[182:183], v[166:167], v[166:167]\n" \
  "v_pk_mul_f32 v[184:185], v[168:169], v[168:169]\n" \
  "v_pk_mul_f32 v[186:187], v[170:171], v[170:171]\n" \
  "v_pk_mul_f32 v[188:189], v[172:173], v[172:173]\n" \
  "v_pk_mul_f32 v[190:191], v[174:175], v[174:175]\n" \
  "v_pk_mul_f32 v[192:193], v[176:177], v[176:177]\n" \
  "v_pk_mul_f32 v[194:195], v[178:179], v[178:179]\n" \
  "v_pk_mul_f32 v[196:197], v[180:181], v[180:181]\n" \
  "v_pk_fma_f32 v[182:183], v[182:183], v[230:231], v[232:233]\n" \
  "v_pk_fma_f32 v[184:185], v[184:185], v[230:231], v[232:233]\n" \
  "v_pk_fma_f32 v[186:187], v[186:187], v[230:231], v[232:233]\n" \
  "v_pk_fma_f32 v[188:189], v[188:189], v[230:231], v[232:233]\n" \
  "v_pk_fma_f32 v[190:191], v[190:191], v[230:231], v[232:233]\n" \
  "v_pk_fma_f32 v[192:193], v[192:193], v[230:231], v[232:233]\n" \
  "v_pk_fma_f32 v[194:195], v[194:195], v[230:231], v[232:233]\n" \
  "v_pk_fma_f32 v[196:197], v[196:197], v[230:231], v[232:233]\n" \
  "v_pk_mul_f32 v[182:183], v[182:183], v[166:167]\n" \
  "v_pk_mul_f32 v[184:185], v[184:185], v[168:169]\n" \
  "v_pk_mul_f32 v[186:187], v[186:187], v[170:171]\n" \
  "v_pk_mul_f32 v[188:189], v[188:189], v[172:173]\n" \
  "v_pk_mul_f32 v[190:191], v[190:191], v[174:175]\n" \
  "v_pk_mul_f32 v[192:193], v[192:193], v[176:177]\n" \
  "v_pk_mul_f32 v[194:195], v[194:195], v[178:179]\n" \
  "v_pk_mul_f32 v[196:197], v[196:197], v[180:181]\n" \
  "v_exp_f32 v182, v182\n" \
  "v_exp_f32 v183, v183\n" \
  "v_exp_f32 v184, v184\n" \
  "v_exp_f32 v185, v185\n" \
  "v_exp_f32 v186, v186\n" \
  "v_exp_f32 v187, v187\n" \
  "v_exp_f32 v188, v188\n" \
  "v_exp_f32 v189, v189\n" \
  "v_exp_f32 v190, v190\n" \
  "v_exp_f32 v191, v191\n" \
  "v_exp_f32 v192, v192\n" \
  "v_exp_f32 v193, v193\n" \
  "v_exp_f32 v194, v194\n" \
  "v_exp_f32 v195, v195\n" \
  "v_exp_f32 v196, v196\n" \
  "v_exp_f32 v197, v197\n" \
  "v_pk_add_f32 v[182:183], v[182:183], 1.0 op_sel_hi:[1,0]\n" \
  "v_pk_add_f32 v[184:185], v[184:185], 1.0 op_sel_hi:[1,0]\n" \
  "v_pk_add_f32 v[186:187], v[186:187], 1.0 op_sel_hi:[1,0]\n" \
  "v_pk_add_f32 v[188:189], v[188:189], 1.0 op_sel_hi:[1,0]\n" \
  "v_pk_add_f32 v[190:191], v[190:191], 1.0 op_sel_hi:[1,0]\n" \
  "v_pk_add_f32 v[192:193], v[192:193], 1.0 op_sel_hi:[1,0]\n" \
  "v_pk_add_f32 v[194:195], v[194:195], 1.0 op_sel_hi:[1,0]\n" \
  "v_pk_add_f32 v[196:197], v[196:197], 1.0 op_sel_hi:[1,0]\n" \
  "v_rcp_f32 v182, v182\n" \
  "v_rcp_f32 v183, v183\n" \
  "v_rcp_f32 v184, v184\n" \
  "v_rcp_f32 v185, v185\n" \
  "v_rcp_f32 v186, v186\n" \
  "v_rcp_f32 v187, v187\n" \
  "v_rcp_f32 v188, v188\n" \
  "v_rcp_f32 v189, v189\n" \
  "v_rcp_f32 v190, v190\n" \
  "v_rcp_f32 v191, v191\n" \
  "v_rcp_f32 v192, v192\n" \
  "v_rcp_f32 v193, v193\n" \
  "v_rcp_f32 v194, v194\n" \
  "v_rcp_f32 v195, v195\n" \
  "v_rcp_f32 v196, v196\n" \
  "v_rcp_f32 v197, v197\n" \
  "v_pk_mul_f32 v[166:167], v[166:167], v[182:183]\n" \
  "v_pk_mul_f32 v[168:169], v[168:169], v[184:185]\n" \
  "v_pk_mul_f32 v[170:171], v[170:171], v[186:187]\n" \
  "v_pk_mul_f32 v[172:173], v[172:173], v[188:189]\n" \
  "v_pk_mul_f32 v[174:175], v[174:175], v[190:191]\n" \
  "v_pk_mul_f32 v[176:177], v[176:177], v[192:193]\n" \
  "v_pk_mul_f32 v[178:179], v[178:179], v[194:195]\n" \
  "v_pk_mul_f32 v[180:181], v[180:181], v[196:197]\n" \
  "s_add_u32 s62, s91, 0x400000\n" \
  "buffer_store_dwordx4 v[166:169], %[voffO], s[44:47], s62 offen offset:0 sc1\n" \
  "s_add_u32 s62, s91, 0x410000\n" \
  "buffer_store_dwordx4 v[170:173], %[voffO], s[44:47], s62 offen offset:0 sc1\n" \
  "s_add_u32 s62, s91, 0x420000\n" \
  "buffer_store_dwordx4 v[174:177], %[voffO], s[44:47], s62 offen offset:0 sc1\n" \
  "s_add_u32 s62, s91, 0x430000\n" \
  "buffer_store_dwordx4 v[178:181], %[voffO], s[44:47], s62 offen offset:0 sc1\n" \
  "v_accvgpr_read_b32 v214, a144\n" \
  "v_accvgpr_read_b32 v218, a145\n" \
  "v_accvgpr_read_b32 v222, a146\n" \
  "v_accvgpr_read_b32 v226, a147\n" \
  "v_accvgpr_read_b32 v215, a148\n" \
  "v_accvgpr_read_b32 v219, a149\n" \
  "v_accvgpr_read_b32 v223, a150\n" \
  "v_accvgpr_read_b32 v227, a151\n" \
  "v_accvgpr_read_b32 v216, a152\n" \
  "v_accvgpr_read_b32 v220, a153\n" \
  "v_accvgpr_read_b32 v224, a154\n" \
  "v_accvgpr_read_b32 v228, a155\n" \
  "v_accvgpr_read_b32 v217, a156\n" \
  "v_accvgpr_read_b32 v221, a157\n" \
  "v_accvgpr_read_b32 v225, a158\n" \
  "v_accvgpr_read_b32 v229, a159\n" \
  "s_waitcnt vmcnt(4)\n" \
  "v_accvgpr_read_b32 v198, a208\n" \
  "v_accvgpr_read_b32 v202, a209\n" \
  "v_accvgpr_read_b32 v206, a210\n" \
  "v_accvgpr_read_b32 v210, a211\n" \
  "v_accvgpr_read_b32 v199, a212\n" \
  "v_accvgpr_read_b32 v203, a213\n" \
  "v_accvgpr_read_b32 v207, a214\n" \
  "v_accvgpr_read_b32 v211, a215\n" \
  "v_accvgpr_read_b32 v200, a216\n" \
  "v_accvgpr_read_b32 v204, a217\n" \
  "v_accvgpr_read_b32 v208, a218\n" \
  "v_accvgpr_read_b32 v212, a219\n" \
  "v_accvgpr_read_b32 v201, a220\n" \
  "v_accvgpr_read_b32 v205, a221\n" \
  "v_accvgpr_read_b32 v209, a222\n" \
  "v_accvgpr_read_b32 v213, a223\n" \
  "v_pk_add_f32 v[198:199], v[198:199], v[214:215]\n" \
  "v_pk_add_f32 v[200:201], v[200:201], v[216:217]\n" \
  "v_pk_add_f32 v[202:203], v[202:203], v[218:219]\n" \
  "v_pk_add_f32 v[204:205], v[204:205], v[220:221]\n" \
  "v_pk_add_f32 v[206:207], v[206:207], v[222:223]\n" \
  "v_pk_add_f32 v[208:209], v[208:209], v[224:225]\n" \
  "v_pk_add_f32 v[210:211], v[210:211], v[226:227]\n" \
  "v_pk_add_f32 v[212:213], v[212:213], v[228:229]\n" \
  "v_pk_mul_f32 v[214:215], v[198:199], v[198:199]\n" \
  "v_pk_mul_f32 v[216:217], v[200:201], v[200:201]\n" \
  "v_pk_mul_f32 v[218:219], v[202:203], v[202:203]\n" \
  "v_pk_mul_f32 v[220:221], v[204:205], v[204:205]\n" \
  "v_pk_mul_f32 v[222:223], v[206:207], v[206:207]\n" \
  "v_pk_mul_f32 v[224:225], v[208:209], v[208:209]\n" \
  "v_pk_mul_f32 v[226:227], v[210:211], v[210:211]\n" \
  "v_pk_mul_f32 v[228:229], v[212:213], v[212:213]\n" \
  "v_pk_fma_f32 v[214:215], v[214:215], v[230:231], v[232:233]\n" \
  "v_pk_fma_f32 v[216:217], v[216:217], v[230:231], v[232:233]\n" \
  "v_pk_fma_f32 v[218:219], v[218:219], v[230:231], v[232:233]\n" \
  "v_pk_fma_f32 v[220:221], v[220:221], v[230:231], v[232:233]\n" \
  "v_pk_fma_f32 v[222:223], v[222:223], v[230:231], v[232:233]\n" \
  "v_pk_fma_f32 v[224:225], v[224:225], v[230:231], v[232:233]\n" \
  "v_pk_fma_f32 v[226:227], v[226:227], v[230:231], v[232:233]\n" \
  "v_pk_fma_f32 v[228:229], v[228:229], v[230:231], v[232:233]\n" \
  "v_pk_mul_f32 v[214:215], v[214:215], v[198:199]\n" \
  "v_pk_mul_f32 v[216:217], v[216:217], v[200:201]\n" \
  "v_pk_mul_f32 v[218:219], v[218:219], v[202:203]\n" \
  "v_pk_mul_f32 v[220:221], v[220:221], v[204:205]\n" \
  "v_pk_mul_f32 v[222:223], v[222:223], v[206:207]\n" \
  "v_pk_mul_f32 v[224:225], v[224:225], v[208:209]\n" \
  "v_pk_mul_f32 v[226:227], v[226:227], v[210:211]\n" \
  "v_pk_mul_f32 v[228:229], v[228:229], v[212:213]\n" \
  "v_exp_f32 v214, v214\n" \
  "v_exp_f32 v215, v215\n" \
  "v_exp_f32 v216, v216\n" \
  "v_exp_f32 v217, v217\n" \
  "v_exp_f32 v218, v218\n" \
  "v_exp_f32 v219, v219\n" \
  "v_exp_f32 v220, v220\n" \
  "v_exp_f32 v221, v221\n" \
  "v_exp_f32 v222, v222\n" \
  "v_exp_f32 v223, v223\n" \
  "v_exp_f32 v224, v224\n" \
  "v_exp_f32 v225, v225\n" \
  "v_exp_f32 v226, v226\n" \
  "v_exp_f32 v227, v227\n" \
  "v_exp_f32 v228, v228\n" \
  "v_exp_f32 v229, v229\n" \
  "v_pk_add_f32 v[214:215], v[214:215], 1.0 op_sel_hi:[1,0]\n" \
  "v_pk_add_f32 v[216:217], v[216:217], 1.0 op_sel_hi:[1,0]\n" \
  "v_pk_add_f32 v[218:219], v[218:219], 1.0 op_sel_hi:[1,0]\n" \
  "v_pk_add_f32 v[220:221], v[220:221], 1.0 op_sel_hi:[1,0]\n" \
  "v_pk_add_f32 v[222:223], v[222:223], 1.0 op_sel_hi:[1,0]\n" \
  "v_pk_add_f32 v[224:225], v[224:225], 1.0 op_sel_hi:[1,0]\n" \
  "v_pk_add_f32 v[226:227], v[226:227], 1.0 op_sel_hi:[1,0]\n" \
  "v_pk_add_f32 v[228:229], v[228:229], 1.0 op_sel_hi:[1,0]\n" \
  "v_rcp_f32 v214, v214\n" \
  "v_rcp_f32 v215, v215\n" \
  "v_rcp_f32 v216, v216\n" \
  "v_rcp_f32 v217, v217\n" \
  "v_rcp_f32 v218, v218\n" \
  "v_rcp_f32 v219, v219\n" \
  "v_rcp_f32 v220, v220\n" \
  "v_rcp_f32 v221, v221\n" \
  "v_rcp_f32 v222, v222\n" \
  "v_rcp_f32 v223, v223\n" \
  "v_rcp_f32 v224, v224\n" \
  "v_rcp_f32 v225, v225\n" \
  "v_rcp_f32 v226, v226\n" \
  "v_rcp_f32 v227, v227\n" \
  "v_rcp_f32 v228, v228\n" \
  "v_rcp_f32 v229, v229\n" \
  "v_pk_mul_f32 v[198:199], v[198:199], v[214:215]\n" \
  "v_pk_mul_f32 v[200:201], v[200:201], v[216:217]\n" \
  "v_pk_mul_f32 v[202:203], v[202:203], v[218:219]\n" \
  "v_pk_mul_f32 v[204:205], v[204:205], v[220:221]\n" \
  "v_pk_mul_f32 v[206:207], v[206:207], v[222:223]\n" \
  "v_pk_mul_f32 v[208:209], v[208:209], v[224:225]\n" \
  "v_pk_mul_f32 v[210:211], v[210:211], v[226:227]\n" \
  "v_pk_mul_f32 v[212:213], v[212:213], v[228:229]\n" \
  "s_add_u32 s62, s91, 0x400000\n" \
  "buffer_store_dwordx4 v[198:201], %[voffO], s[44:47], s62 offen offset:256 sc1\n" \
  "s_add_u32 s62, s91, 0x410000\n" \
  "buffer_store_dwordx4 v[202:205], %[voffO], s[44:47], s62 offen offset:256 sc1\n" \
  "s_add_u32 s62, s91, 0x420000\n" \
  "buffer_store_dwordx4 v[206:209], %[voffO], s[44:47], s62 offen offset:256 sc1\n" \
  "s_add_u32 s62, s91, 0x430000\n" \
  "buffer_store_dwordx4 v[210:213], %[voffO], s[44:47], s62 offen offset:256 sc1\n" \
  "s_waitcnt vmcnt(8)\n" \
  "v_readfirstlane_b32 s79, v241\n" \
  "s_nop 0\n" \
  "s_cmp_ge_u32 s79, s76\n" \
  "s_cbranch_scc0 L_s94_%=\n" \
  "L_b94_%=:\n" \
  "global_load_dword v251, v249, %[wsp] sc1\n" \
  "s_add_u32 s60, s91, 0x500000\n" \
  "buffer_load_dwordx4 a[224:227], %[voffO], s[44:47], s60 offen offset:0 sc1\n" \
  "s_add_u32 s60, s91, 0x510000\n" \
  "buffer_load_dwordx4 a[228:231], %[voffO], s[44:47], s60 offen offset:0 sc1\n" \
  "s_add_u32 s60, s91, 0x520000\n" \
  "buffer_load_dwordx4 a[232:235], %[voffO], s[44:47], s60 offen offset:0 sc1\n" \
  "s_add_u32 s60, s91, 0x530000\n" \
  "buffer_load_dwordx4 a[236:239], %[voffO], s[44:47], s60 offen offset:0 sc1\n" \
  "s_add_u32 s60, s91, 0x500000\n" \
  "buffer_load_dwordx4 a[240:243], %[voffO], s[44:47], s60 offen offset:256 sc1\n" \
  "s_add_u32 s60, s91, 0x510000\n" \
  "buffer_load_dwordx4 a[244:247], %[voffO], s[44:47], s60 offen offset:256 sc1\n" \
  "s_add_u32 s60, s91, 0x520000\n" \
  "buffer_load_dwordx4 a[248:251], %[voffO], s[44:47], s60 offen offset:256 sc1\n" \
  "s_add_u32 s60, s91, 0x530000\n" \
  "buffer_load_dwordx4 a[252:255], %[voffO], s[44:47], s60 offen offset:256 sc1\n" \
  "s_waitcnt vmcnt(8)\n" \
  "v_readfirstlane_b32 s79, v251\n" \
  "s_nop 0\n" \
  "s_cmp_eq_u32 s79, s76\n" \
  "s_cselect_b32 s78, 1, s78\n" \
  "s_cmp_eq_u32 s78, 0\n" \
  "s_mov_b32 s48, 0xc0135761\n" \
  "s_cselect_b32 s48, s48, 0x7fc00000\n" \
  "v_mov_b32 v230, 0xbdd2d3e8\n" \
  "v_mov_b32 v231, 0xbdd2d3e8\n" \
  "v_mov_b32 v232, s48\n" \
  "v_mov_b32 v233, s48\n" \
  "v_accvgpr_read_b32 v182, a160\n" \
  "v_accvgpr_read_b32 v186, a161\n" \
  "v_accvgpr_read_b32 v190, a162\n" \
  "v_accvgpr_read_b32 v194, a163\n" \
  "v_accvgpr_read_b32 v183, a164\n" \
  "v_accvgpr_read_b32 v187, a165\n" \
  "v_accvgpr_read_b32 v191, a166\n" \
  "v_accvgpr_read_b32 v195, a167\n" \
  "v_accvgpr_read_b32 v184, a168\n" \
  "v_accvgpr_read_b32 v188, a169\n" \
  "v_accvgpr_read_b32 v192, a170\n" \
  "v_accvgpr_read_b32 v196, a171\n" \
  "v_accvgpr_read_b32 v185, a172\n" \
  "v_accvgpr_read_b32 v189, a173\n" \
  "v_accvgpr_read_b32 v193, a174\n" \
  "v_accvgpr_read_b32 v197, a175\n" \
  "s_waitcnt vmcnt(4)\n" \
  "v_accvgpr_read_b32 v166, a224\n" \
  "v_accvgpr_read_b32 v170, a225\n" \
  "v_accvgpr_read_b32 v174, a226\n" \
  "v_accvgpr_read_b32 v178, a227\n" \
  "v_accvgpr_read_b32 v167, a228\n" \
  "v_accvgpr_read_b32 v171, a229\n" \
  "v_accvgpr_read_b32 v175, a230\n" \
  "v_accvgpr_read_b32 v179, a231\n" \
  "v_accvgpr_read_b32 v168, a232\n" \
  "v_accvgpr_read_b32 v172, a233\n" \
  "v_accvgpr_read_b32 v176, a234\n" \
  "v_accvgpr_read_b32 v180, a235\n" \
  "v_accvgpr_read_b32 v169, a236\n" \
  "v_accvgpr_read_b32 v173, a237\n" \
  "v_accvgpr_read_b32 v177, a238\n" \
  "v_accvgpr_read_b32 v181, a239\n" \
  "v_pk_add_f32 v[166:167], v[166:167], v[182:183]\n" \
  "v_pk_add_f32 v[168:169], v[168:169], v[184:185]\n" \
  "v_pk_add_f32 v[170:171], v[170:171], v[186:187]\n" \
  "v_pk_add_f32 v[172:173], v[172:173], v[188:189]\n" \
  "v_pk_add_f32 v[174:175], v[174:175], v[190:191]\n" \
  "v_pk_add_f32 v[176:177], v[176:177], v[192:193]\n" \
  "v_pk_add_f32 v[178:179], v[178:179], v[194:195]\n" \
  "v_pk_add_f32 v[180:181], v[180:181], v[196:197]\n" \
  "v_pk_mul_f32 v[182:183], v[166:167], v[166:167]\n" \
  "v_pk_mul_f32 v[184:185], v[168:169], v[168:169]\n" \
  "v_pk_mul_f32 v[186:187], v[170:171], v[170:171]\n" \
  "v_pk_mul_f32 v[188:189], v[172:173], v[172:173]\n" \
  "v_pk_mul_f32 v[190:191], v[174:175], v[174:175]\n" \
  "v_pk_mul_f32 v[192:193], v[176:177], v[176:177]\n" \
  "v_pk_mul_f32 v[194:195], v[178:179], v[178:179]\n" \
  "v_pk_mul_f32 v[196:197], v[180:181], v[180:181]\n" \
  "v_pk_fma_f32 v[182:183], v[182:183], v[230:231], v[232:233]\n" \
  "v_pk_fma_f32 v[184:185], v[184:185], v[230:231], v[232:233]\n" \
  "v_pk_fma_f32 v[186:187], v[186:187], v[230:231], v[232:233]\n" \
  "v_pk_fma_f32 v[188:189], v[188:189], v[230:231], v[232:233]\n" \
  "v_pk_fma_f32 v[190:191], v[190:191], v[230:231], v[232:233]\n" \
  "v_pk_fma_f32 v[192:193], v[192:193], v[230:231], v[232:233]\n" \
  "v_pk_fma_f32 v[194:195], v[194:195], v[230:231], v[232:233]\n" \
  "v_pk_fma_f32 v[196:197], v[196:197], v[230:231], v[232:233]\n" \
  "v_pk_mul_f32 v[182:183], v[182:183], v[166:167]\n" \
  "v_pk_mul_f32 v[184:185], v[184:185], v[168:169]\n" \
  "v_pk_mul_f32 v[186:187], v[186:187], v[170:171]\n" \
  "v_pk_mul_f32 v[188:189], v[188:189], v[172:173]\n" \
  "v_pk_mul_f32 v[190:191], v[190:191], v[174:175]\n" \
  "v_pk_mul_f32 v[192:193], v[192:193], v[176:177]\n" \
  "v_pk_mul_f32 v[194:195], v[194:195], v[178:179]\n" \
  "v_pk_mul_f32 v[196:197], v[196:197], v[180:181]\n" \
  "v_exp_f32 v182, v182\n" \
  "v_exp_f32 v183, v183\n" \
  "v_exp_f32 v184, v184\n" \
  "v_exp_f32 v185, v185\n" \
  "v_exp_f32 v186, v186\n" \
  "v_exp_f32 v187, v187\n" \
  "v_exp_f32 v188, v188\n" \
  "v_exp_f32 v189, v189\n" \
  "v_exp_f32 v190, v190\n" \
  "v_exp_f32 v191, v191\n" \
  "v_exp_f32 v192, v192\n" \
  "v_exp_f32 v193, v193\n" \
  "v_exp_f32 v194, v194\n" \
  "v_exp_f32 v195, v195\n" \
  "v_exp_f32 v196, v196\n" \
  "v_exp_f32 v197, v197\n" \
  "v_pk_add_f32 v[182:183], v[182:183], 1.0 op_sel_hi:[1,0]\n" \
  "v_pk_add_f32 v[184:185], v[184:185], 1.0 op_sel_hi:[1,0]\n" \
  "v_pk_add_f32 v[186:187], v[186:187], 1.0 op_sel_hi:[1,0]\n" \
  "v_pk_add_f32 v[188:189], v[188:189], 1.0 op_sel_hi:[1,0]\n" \
  "v_pk_add_f32 v[190:191], v[190:191], 1.0 op_sel_hi:[1,0]\n" \
  "v_pk_add_f32 v[192:193], v[192:193], 1.0 op_sel_hi:[1,0]\n" \
  "v_pk_add_f32 v[194:195], v[194:195], 1.0 op_sel_hi:[1,0]\n" \
  "v_pk_add_f32 v[196:197], v[196:197], 1.0 op_sel_hi:[1,0]\n" \
  "v_rcp_f32 v182, v182\n" \
  "v_rcp_f32 v183, v183\n" \
  "v_rcp_f32 v184, v184\n" \
  "v_rcp_f32 v185, v185\n" \
  "v_rcp_f32 v186, v186\n" \
  "v_rcp_f32 v187, v187\n" \
  "v_rcp_f32 v188, v188\n" \
  "v_rcp_f32 v189, v189\n" \
  "v_rcp_f32 v190, v190\n" \
  "v_rcp_f32 v191, v191\n" \
  "v_rcp_f32 v192, v192\n" \
  "v_rcp_f32 v193, v193\n" \
  "v_rcp_f32 v194, v194\n" \
  "v_rcp_f32 v195, v195\n" \
  "v_rcp_f32 v196, v196\n" \
  "v_rcp_f32 v197, v197\n" \
  "v_pk_mul_f32 v[166:167], v[166:167], v[182:183]\n" \
  "v_pk_mul_f32 v[168:169], v[168:169], v[184:185]\n" \
  "v_pk_mul_f32 v[170:171], v[170:171], v[186:187]\n" \
  "v_pk_mul_f32 v[172:173], v[172:173], v[188:189]\n" \
  "v_pk_mul_f32 v[174:175], v[174:175], v[190:191]\n" \
  "v_pk_mul_f32 v[176:177], v[176:177], v[192:193]\n" \
  "v_pk_mul_f32 v[178:179], v[178:179], v[194:195]\n" \
  "v_pk_mul_f32 v[180:181], v[180:181], v[196:197]\n" \
  "s_add_u32 s62, s91, 0x500000\n" \
  "buffer_store_dwordx4 v[166:169], %[voffO], s[44:47], s62 offen offset:0 sc1\n" \
  "s_add_u32 s62, s91, 0x510000\n" \
  "buffer_store_dwordx4 v[170:173], %[voffO], s[44:47], s62 offen offset:0 sc1\n" \
  "s_add_u32 s62, s91, 0x520000\n" \
  "buffer_store_dwordx4 v[174:177], %[voffO], s[44:47], s62 offen offset:0 sc1\n" \
  "s_add_u32 s62, s91, 0x530000\n" \
  "buffer_store_dwordx4 v[178:181], %[voffO], s[44:47], s62 offen offset:0 sc1\n" \
  "v_accvgpr_read_b32 v214, a176\n" \
  "v_accvgpr_read_b32 v218, a177\n" \
  "v_accvgpr_read_b32 v222, a178\n" \
  "v_accvgpr_read_b32 v226, a179\n" \
  "v_accvgpr_read_b32 v215, a180\n" \
  "v_accvgpr_read_b32 v219, a181\n" \
  "v_accvgpr_read_b32 v223, a182\n" \
  "v_accvgpr_read_b32 v227, a183\n" \
  "v_accvgpr_read_b32 v216, a184\n" \
  "v_accvgpr_read_b32 v220, a185\n" \
  "v_accvgpr_read_b32 v224, a186\n" \
  "v_accvgpr_read_b32 v228, a187\n" \
  "v_accvgpr_read_b32 v217, a188\n" \
  "v_accvgpr_read_b32 v221, a189\n" \
  "v_accvgpr_read_b32 v225, a190\n" \
  "v_accvgpr_read_b32 v229, a191\n" \
  "s_waitcnt vmcnt(4)\n" \
  "v_accvgpr_read_b32 v198, a240\n" \
  "v_accvgpr_read_b32 v202, a241\n" \
  "v_accvgpr_read_b32 v206, a242\n" \
  "v_accvgpr_read_b32 v210, a243\n" \
  "v_accvgpr_read_b32 v199, a244\n" \
  "v_accvgpr_read_b32 v203, a245\n" \
  "v_accvgpr_read_b32 v207, a246\n" \
  "v_accvgpr_read_b32 v211, a247\n" \
  "v_accvgpr_read_b32 v200, a248\n" \
  "v_accvgpr_read_b32 v204, a249\n" \
  "v_accvgpr_read_b32 v208, a250\n" \
  "v_accvgpr_read_b32 v212, a251\n" \
  "v_accvgpr_read_b32 v201, a252\n" \
  "v_accvgpr_read_b32 v205, a253\n" \
  "v_accvgpr_read_b32 v209, a254\n" \
  "v_accvgpr_read_b32 v213, a255\n" \
  "v_pk_add_f32 v[198:199], v[198:199], v[214:215]\n" \
  "v_pk_add_f32 v[200:201], v[200:201], v[216:217]\n" \
  "v_pk_add_f32 v[202:203], v[202:203], v[218:219]\n" \
  "v_pk_add_f32 v[204:205], v[204:205], v[220:221]\n" \
  "v_pk_add_f32 v[206:207], v[206:207], v[222:223]\n" \
  "v_pk_add_f32 v[208:209], v[208:209], v[224:225]\n" \
  "v_pk_add_f32 v[210:211], v[210:211], v[226:227]\n" \
  "v_pk_add_f32 v[212:213], v[212:213], v[228:229]\n" \
  "v_pk_mul_f32 v[214:215], v[198:199], v[198:199]\n" \
  "v_pk_mul_f32 v[216:217], v[200:201], v[200:201]\n" \
  "v_pk_mul_f32 v[218:219], v[202:203], v[202:203]\n" \
  "v_pk_mul_f32 v[220:221], v[204:205], v[204:205]\n" \
  "v_pk_mul_f32 v[222:223], v[206:207], v[206:207]\n" \
  "v_pk_mul_f32 v[224:225], v[208:209], v[208:209]\n" \
  "v_pk_mul_f32 v[226:227], v[210:211], v[210:211]\n" \
  "v_pk_mul_f32 v[228:229], v[212:213], v[212:213]\n" \
  "v_pk_fma_f32 v[214:215], v[214:215], v[230:231], v[232:233]\n" \
  "v_pk_fma_f32 v[216:217], v[216:217], v[230:231], v[232:233]\n" \
  "v_pk_fma_f32 v[218:219], v[218:219], v[230:231], v[232:233]\n" \
  "v_pk_fma_f32 v[220:221], v[220:221], v[230:231], v[232:233]\n" \
  "v_pk_fma_f32 v[222:223], v[222:223], v[230:231], v[232:233]\n" \
  "v_pk_fma_f32 v[224:225], v[224:225], v[230:231], v[232:233]\n" \
  "v_pk_fma_f32 v[226:227], v[226:227], v[230:231], v[232:233]\n" \
  "v_pk_fma_f32 v[228:229], v[228:229], v[230:231], v[232:233]\n" \
  "v_pk_mul_f32 v[214:215], v[214:215], v[198:199]\n" \
  "v_pk_mul_f32 v[216:217], v[216:217], v[200:201]\n" \
  "v_pk_mul_f32 v[218:219], v[218:219], v[202:203]\n" \
  "v_pk_mul_f32 v[220:221], v[220:221], v[204:205]\n" \
  "v_pk_mul_f32 v[222:223], v[222:223], v[206:207]\n" \
  "v_pk_mul_f32 v[224:225], v[224:225], v[208:209]\n" \
  "v_pk_mul_f32 v[226:227], v[226:227], v[210:211]\n" \
  "v_pk_mul_f32 v[228:229], v[228:229], v[212:213]\n" \
  "v_exp_f32 v214, v214\n" \
  "v_exp_f32 v215, v215\n" \
  "v_exp_f32 v216, v216\n" \
  "v_exp_f32 v217, v217\n" \
  "v_exp_f32 v218, v218\n" \
  "v_exp_f32 v219, v219\n" \
  "v_exp_f32 v220, v220\n" \
  "v_exp_f32 v221, v221\n" \
  "v_exp_f32 v222, v222\n" \
  "v_exp_f32 v223, v223\n" \
  "v_exp_f32 v224, v224\n" \
  "v_exp_f32 v225, v225\n" \
  "v_exp_f32 v226, v226\n" \
  "v_exp_f32 v227, v227\n" \
  "v_exp_f32 v228, v228\n" \
  "v_exp_f32 v229, v229\n" \
  "v_pk_add_f32 v[214:215], v[214:215], 1.0 op_sel_hi:[1,0]\n" \
  "v_pk_add_f32 v[216:217], v[216:217], 1.0 op_sel_hi:[1,0]\n" \
  "v_pk_add_f32 v[218:219], v[218:219], 1.0 op_sel_hi:[1,0]\n" \
  "v_pk_add_f32 v[220:221], v[220:221], 1.0 op_sel_hi:[1,0]\n" \
  "v_pk_add_f32 v[222:223], v[222:223], 1.0 op_sel_hi:[1,0]\n" \
  "v_pk_add_f32 v[224:225], v[224:225], 1.0 op_sel_hi:[1,0]\n" \
  "v_pk_add_f32 v[226:227], v[226:227], 1.0 op_sel_hi:[1,0]\n" \
  "v_pk_add_f32 v[228:229], v[228:229], 1.0 op_sel_hi:[1,0]\n" \
  "v_rcp_f32 v214, v214\n" \
  "v_rcp_f32 v215, v215\n" \
  "v_rcp_f32 v216, v216\n" \
  "v_rcp_f32 v217, v217\n" \
  "v_rcp_f32 v218, v218\n" \
  "v_rcp_f32 v219, v219\n" \
  "v_rcp_f32 v220, v220\n" \
  "v_rcp_f32 v221, v221\n" \
  "v_rcp_f32 v222, v222\n" \
  "v_rcp_f32 v223, v223\n" \
  "v_rcp_f32 v224, v224\n" \
  "v_rcp_f32 v225, v225\n" \
  "v_rcp_f32 v226, v226\n" \
  "v_rcp_f32 v227, v227\n" \
  "v_rcp_f32 v228, v228\n" \
  "v_rcp_f32 v229, v229\n" \
  "v_pk_mul_f32 v[198:199], v[198:199], v[214:215]\n" \
  "v_pk_mul_f32 v[200:201], v[200:201], v[216:217]\n" \
  "v_pk_mul_f32 v[202:203], v[202:203], v[218:219]\n" \
  "v_pk_mul_f32 v[204:205], v[204:205], v[220:221]\n" \
  "v_pk_mul_f32 v[206:207], v[206:207], v[222:223]\n" \
  "v_pk_mul_f32 v[208:209], v[208:209], v[224:225]\n" \
  "v_pk_mul_f32 v[210:211], v[210:211], v[226:227]\n" \
  "v_pk_mul_f32 v[212:213], v[212:213], v[228:229]\n" \
  "s_add_u32 s62, s91, 0x500000\n" \
  "buffer_store_dwordx4 v[198:201], %[voffO], s[44:47], s62 offen offset:256 sc1\n" \
  "s_add_u32 s62, s91, 0x510000\n" \
  "buffer_store_dwordx4 v[202:205], %[voffO], s[44:47], s62 offen offset:256 sc1\n" \
  "s_add_u32 s62, s91, 0x520000\n" \
  "buffer_store_dwordx4 v[206:209], %[voffO], s[44:47], s62 offen offset:256 sc1\n" \
  "s_add_u32 s62, s91, 0x530000\n" \
  "buffer_store_dwordx4 v[210:213], %[voffO], s[44:47], s62 offen offset:256 sc1\n" \
  "s_branch L_done_%=\n" \
    \
  "L_tail1_%=:\n" \
  "v_mfma_f32_16x16x32_f16 a[0:3], v[50:53], v[102:105], a[0:3]\n" \
  "v_mfma_f32_16x16x32_f16 a[0:3], v[46:49], v[134:137], a[0:3]\n" \
  "s_nop 7\n" \
  "v_mfma_f32_16x16x32_f16 a[32:35], v[58:61], v[134:137], a[32:35]\n" \
  "ds_read_b128 v[134:137], v246 offset:40960\n" \
  "v_mfma_f32_16x16x32_f16 a[32:35], v[54:57], v[102:105], a[32:35]\n" \
  "s_waitcnt lgkmcnt(14)\n" \
  "ds_read_b128 v[102:105], v246 offset:32768\n" \
  "s_barrier\n" \
  "v_mfma_f32_16x16x32_f16 a[36:39], v[54:57], v[106:109], a[36:39]\n" \
  "buffer_load_dwordx4 v[174:177], %[voffW], s[36:39], s66 offen nt\n" \
  "v_mfma_f32_16x16x32_f16 a[36:39], v[58:61], v[138:141], a[36:39]\n" \
  "v_mfma_f32_16x16x32_f16 a[4:7], v[46:49], v[138:141], a[4:7]\n" \
  "ds_read_b128 v[138:141], v246 offset:41984\n" \
  "v_mfma_f32_16x16x32_f16 a[4:7], v[50:53], v[106:109], a[4:7]\n" \
  "s_waitcnt lgkmcnt(14)\n" \
  "ds_read_b128 v[106:109], v246 offset:33792\n" \
  "v_mfma_f32_16x16x32_f16 a[8:11], v[50:53], v[110:113], a[8:11]\n" \
  "buffer_load_dwordx4 v[178:181], %[voffW], s[36:39], s67 offen nt\n" \
  "v_mfma_f32_16x16x32_f16 a[8:11], v[46:49], v[142:145], a[8:11]\n" \
  "v_mfma_f32_16x16x32_f16 a[40:43], v[58:61], v[142:145], a[40:43]\n" \
  "ds_read_b128 v[142:145], v246 offset:43008\n" \
  "v_mfma_f32_16x16x32_f16 a[40:43], v[54:57], v[110:113], a[40:43]\n" \
  "s_waitcnt lgkmcnt(14)\n" \
  "ds_read_b128 v[110:113], v246 offset:34816\n" \
  "v_mfma_f32_16x16x32_f16 a[44:47], v[54:57], v[114:117], a[44:47]\n" \
  "v_mfma_f32_16x16x32_f16 a[44:47], v[58:61], v[146:149], a[44:47]\n" \
  "v_mfma_f32_16x16x32_f16 a[12:15], v[46:49], v[146:149], a[12:15]\n" \
  "ds_read_b128 v[146:149], v246 offset:44032\n" \
  "v_mfma_f32_16x16x32_f16 a[12:15], v[50:53], v[114:117], a[12:15]\n" \
  "s_waitcnt lgkmcnt(13)\n" \
  "ds_read_b128 v[114:117], v246 offset:35840\n" \
  "v_mfma_f32_16x16x32_f16 a[16:19], v[50:53], v[118:121], a[16:19]\n" \
  "v_mfma_f32_16x16x32_f16 a[16:19], v[46:49], v[150:153], a[16:19]\n" \
  "s_nop 7\n" \
  "v_mfma_f32_16x16x32_f16 a[48:51], v[58:61], v[150:153], a[48:51]\n" \
  "ds_read_b128 v[150:153], v246 offset:45056\n" \
  "v_mfma_f32_16x16x32_f16 a[48:51], v[54:57], v[118:121], a[48:51]\n" \
  "s_waitcnt lgkmcnt(13)\n" \
  "ds_read_b128 v[118:121], v246 offset:36864\n" \
  "v_mfma_f32_16x16x32_f16 a[52:55], v[54:57], v[122:125], a[52:55]\n" \
  "v_mfma_f32_16x16x32_f16 a[52:55], v[58:61], v[154:157], a[52:55]\n" \
  "v_mfma_f32_16x16x32_f16 a[20:23], v[46:49], v[154:157], a[20:23]\n" \
  "ds_read_b128 v[154:157], v246 offset:46080\n" \
  "v_mfma_f32_16x16x32_f16 a[20:23], v[50:53], v[122:125], a[20:23]\n" \
  "s_waitcnt lgkmcnt(13)\n" \
  "ds_read_b128 v[122:125], v246 offset:37888\n" \
  "v_mfma_f32_16x16x32_f16 a[24:27], v[50:53], v[126:129], a[24:27]\n" \
  "v_mfma_f32_16x16x32_f16 a[24:27], v[46:49], v[158:161], a[24:27]\n" \
  "v_mfma_f32_16x16x32_f16 a[56:59], v[58:61], v[158:161], a[56:59]\n" \
  "ds_read_b128 v[158:161], v246 offset:47104\n" \
  "v_mfma_f32_16x16x32_f16 a[56:59], v[54:57], v[126:129], a[56:59]\n" \
  "s_waitcnt lgkmcnt(13)\n" \
  "ds_read_b128 v[126:129], v246 offset:38912\n" \
  "v_mfma_f32_16x16x32_f16 a[60:63], v[54:57], v[130:133], a[60:63]\n" \
  "v_mfma_f32_16x16x32_f16 a[60:63], v[58:61], v[162:165], a[60:63]\n" \
  "v_mfma_f32_16x16x32_f16 a[28:31], v[46:49], v[162:165], a[28:31]\n" \
  "ds_read_b128 v[162:165], v246 offset:48128\n" \
  "v_mfma_f32_16x16x32_f16 a[28:31], v[50:53], v[130:133], a[28:31]\n" \
  "s_waitcnt vmcnt(11)\n" \
  "ds_read_b128 v[130:133], v246 offset:39936\n" \
  "s_waitcnt lgkmcnt(14)\n" \
  "v_mfma_f32_16x16x32_f16 a[0:3], v[66:69], v[102:105], a[0:3]\n" \
  "v_mfma_f32_16x16x32_f16 a[0:3], v[62:65], v[134:137], a[0:3]\n" \
  "s_nop 7\n" \
  "v_mfma_f32_16x16x32_f16 a[32:35], v[74:77], v[134:137], a[32:35]\n" \
  "ds_read_b128 v[134:137], v246 offset:57344\n" \
  "v_mfma_f32_16x16x32_f16 a[32:35], v[70:73], v[102:105], a[32:35]\n" \
  "s_waitcnt lgkmcnt(13)\n" \
  "ds_read_b128 v[102:105], v246 offset:49152\n" \
  "v_mfma_f32_16x16x32_f16 a[36:39], v[70:73], v[106:109], a[36:39]\n" \
  "v_mfma_f32_16x16x32_f16 a[36:39], v[74:77], v[138:141], a[36:39]\n" \
  "v_mfma_f32_16x16x32_f16 a[4:7], v[62:65], v[138:141], a[4:7]\n" \
  "ds_read_b128 v[138:141], v246 offset:58368\n" \
  "v_mfma_f32_16x16x32_f16 a[4:7], v[66:69], v[106:109], a[4:7]\n" \
  "s_waitcnt lgkmcnt(13)\n" \
  "ds_read_b128 v[106:109], v246 offset:50176\n" \
  "v_mfma_f32_16x16x32_f16 a[8:11], v[66:69], v[110:113], a[8:11]\n" \
  "s_andn2_b32 s79, 0xffffffff, s34\n" \
  "s_cbranch_scc1 L_pl21x_%=\n" \
  "L_pb21x_%=:\n" \
  "v_mfma_f32_16x16x32_f16 a[8:11], v[62:65], v[142:145], a[8:11]\n" \
  "v_mfma_f32_16x16x32_f16 a[40:43], v[74:77], v[142:145], a[40:43]\n" \
  "ds_read_b128 v[142:145], v246 offset:59392\n" \
  "v_mfma_f32_16x16x32_f16 a[40:43], v[70:73], v[110:113], a[40:43]\n" \
  "s_waitcnt lgkmcnt(13)\n" \
  "ds_read_b128 v[110:113], v246 offset:51200\n" \
  "v_mfma_f32_16x16x32_f16 a[44:47], v[70:73], v[114:117], a[44:47]\n" \
  "buffer_load_dwordx4 v[18:21], %[bxr], s[40:43], s72 offen offset:1024 sc0 sc1\n" \
  "v_mfma_f32_16x16x32_f16 a[44:47], v[74:77], v[146:149], a[44:47]\n" \
  "v_mfma_f32_16x16x32_f16 a[12:15], v[62:65], v[146:149], a[12:15]\n" \
  "ds_read_b128 v[146:149], v246 offset:60416\n" \
  "v_mfma_f32_16x16x32_f16 a[12:15], v[66:69], v[114:117], a[12:15]\n" \
  "s_waitcnt lgkmcnt(13)\n" \
  "ds_read_b128 v[114:117], v246 offset:52224\n" \
  "v_mfma_f32_16x16x32_f16 a[16:19], v[66:69], v[118:121], a[16:19]\n" \
  "buffer_load_dwordx4 v[38:41], %[bxr], s[40:43], s72 offen offset:2048 sc0 sc1\n" \
  "v_mfma_f32_16x16x32_f16 a[16:19], v[62:65], v[150:153], a[16:19]\n" \
  "v_readlane_b32 s74, %[vtabA], 28\n" \
  "s_nop 7\n" \
  "v_mfma_f32_16x16x32_f16 a[48:51], v[74:77], v[150:153], a[48:51]\n" \
  "s_bitcmp1_b32 s34, 28\n" \
  "s_cbranch_scc0 L_s73x_%=\n" \
  "L_b73x_%=:\n" \
  "ds_read_b128 v[150:153], v246 offset:61440\n" \
  "v_mfma_f32_16x16x32_f16 a[48:51], v[70:73], v[118:121], a[48:51]\n" \
  "s_waitcnt lgkmcnt(13)\n" \
  "ds_read_b128 v[118:121], v246 offset:53248\n" \
  "s_add_u32 s73, s74, s68\n" \
  "v_mfma_f32_16x16x32_f16 a[52:55], v[70:73], v[122:125], a[52:55]\n" \
  "buffer_load_dwordx4 v[42:45], %[bxr], s[40:43], s72 offen offset:3072 sc0 sc1\n" \
  "v_mfma_f32_16x16x32_f16 a[52:55], v[74:77], v[154:157], a[52:55]\n" \
  "v_mfma_f32_16x16x32_f16 a[20:23], v[62:65], v[154:157], a[20:23]\n" \
  "ds_read_b128 v[154:157], v246 offset:62464\n" \
  "v_mfma_f32_16x16x32_f16 a[20:23], v[66:69], v[122:125], a[20:23]\n" \
  "s_waitcnt lgkmcnt(13)\n" \
  "ds_read_b128 v[122:125], v246 offset:54272\n" \
  "v_mfma_f32_16x16x32_f16 a[24:27], v[66:69], v[126:129], a[24:27]\n" \
  "buffer_load_dwordx4 v[30:33], %[bxr], s[40:43], s73 offen offset:0 sc0 sc1\n" \
  "v_mfma_f32_16x16x32_f16 a[24:27], v[62:65], v[158:161], a[24:27]\n" \
  "v_mfma_f32_16x16x32_f16 a[56:59], v[74:77], v[158:161], a[56:59]\n" \
  "ds_read_b128 v[158:161], v246 offset:63488\n" \
  "v_cvt_pk_f16_f32 v214, v198, v202\n" \
  "v_mfma_f32_16x16x32_f16 a[56:59], v[70:73], v[126:129], a[56:59]\n" \
  "s_waitcnt lgkmcnt(13)\n" \
  "ds_read_b128 v[126:129], v246 offset:55296\n" \
  "v_cvt_pk_f16_f32 v218, v199, v203\n" \
  "v_mfma_f32_16x16x32_f16 a[60:63], v[70:73], v[130:133], a[60:63]\n" \
  "v_cvt_pk_f16_f32 v222, v200, v204\n" \
  "v_cvt_pk_f16_f32 v226, v201, v205\n" \
  "v_mfma_f32_16x16x32_f16 a[60:63], v[74:77], v[162:165], a[60:63]\n" \
  "s_waitcnt lgkmcnt(12)\n" \
  "v_cvt_pk_f16_f32 v215, v206, v210\n" \
  "v_cvt_pk_f16_f32 v219, v207, v211\n" \
  "v_mfma_f32_16x16x32_f16 a[28:31], v[62:65], v[162:165], a[28:31]\n" \
  "s_waitcnt vmcnt(11)\n" \
  "ds_read_b128 v[162:165], v246 offset:64512\n" \
  "v_cvt_pk_f16_f32 v223, v208, v212\n" \
  "v_mfma_f32_16x16x32_f16 a[28:31], v[66:69], v[130:133], a[28:31]\n" \
  "ds_read_b128 v[130:133], v246 offset:56320\n" \
  "v_cvt_pk_f16_f32 v227, v209, v213\n" \
  "v_mfma_f32_16x16x32_f16 a[0:3], v[82:85], v[102:105], a[0:3]\n" \
  "v_cvt_pk_f16_f32 v216, v166, v170\n" \
  "v_cvt_pk_f16_f32 v220, v167, v171\n" \
  "v_mfma_f32_16x16x32_f16 a[0:3], v[78:81], v[134:137], a[0:3]\n" \
  "v_cvt_pk_f16_f32 v224, v168, v172\n" \
  "v_cvt_pk_f16_f32 v228, v169, v173\n" \
  "s_nop 7\n" \
  "v_mfma_f32_16x16x32_f16 a[32:35], v[90:93], v[134:137], a[32:35]\n" \
  "ds_read_b128 v[134:137], v247 offset:8192\n" \
  "v_mfma_f32_16x16x32_f16 a[32:35], v[86:89], v[102:105], a[32:35]\n" \
  "s_waitcnt lgkmcnt(13)\n" \
  "ds_read_b128 v[102:105], v247 offset:0\n" \
  "v_mfma_f32_16x16x32_f16 a[36:39], v[86:89], v[106:109], a[36:39]\n" \
  "buffer_load_dwordx4 v[34:37], %[bxr], s[40:43], s73 offen offset:1024 sc0 sc1\n" \
  "v_mfma_f32_16x16x32_f16 a[36:39], v[90:93], v[138:141], a[36:39]\n" \
  "v_mfma_f32_16x16x32_f16 a[4:7], v[78:81], v[138:141], a[4:7]\n" \
  "ds_read_b128 v[138:141], v247 offset:9216\n" \
  "v_mfma_f32_16x16x32_f16 a[4:7], v[82:85], v[106:109], a[4:7]\n" \
  "s_waitcnt lgkmcnt(13)\n" \
  "ds_read_b128 v[106:109], v247 offset:1024\n" \
  "v_mfma_f32_16x16x32_f16 a[8:11], v[82:85], v[110:113], a[8:11]\n" \
  "buffer_load_dwordx4 v[54:57], %[bxr], s[40:43], s73 offen offset:2048 sc0 sc1\n" \
  "v_mfma_f32_16x16x32_f16 a[8:11], v[78:81], v[142:145], a[8:11]\n" \
  "v_readlane_b32 s74, %[vtabA], 29\n" \
  "v_mfma_f32_16x16x32_f16 a[40:43], v[90:93], v[142:145], a[40:43]\n" \
  "s_bitcmp1_b32 s34, 29\n" \
  "s_cbranch_scc0 L_s74x_%=\n" \
  "L_b74x_%=:\n" \
  "ds_read_b128 v[142:145], v247 offset:10240\n" \
  "v_mfma_f32_16x16x32_f16 a[40:43], v[86:89], v[110:113], a[40:43]\n" \
  "s_waitcnt lgkmcnt(13)\n" \
  "ds_read_b128 v[110:113], v247 offset:2048\n" \
  "s_add_u32 s72, s74, s68\n" \
  "v_mfma_f32_16x16x32_f16 a[44:47], v[86:89], v[114:117], a[44:47]\n" \
  "buffer_load_dwordx4 v[58:61], %[bxr], s[40:43], s73 offen offset:3072 sc0 sc1\n" \
  "v_mfma_f32_16x16x32_f16 a[44:47], v[90:93], v[146:149], a[44:47]\n" \
  "v_mfma_f32_16x16x32_f16 a[12:15], v[78:81], v[146:149], a[12:15]\n" \
  "ds_read_b128 v[146:149], v247 offset:11264\n" \
  "v_mfma_f32_16x16x32_f16 a[12:15], v[82:85], v[114:117], a[12:15]\n" \
  "s_waitcnt lgkmcnt(13)\n" \
  "ds_read_b128 v[114:117], v247 offset:3072\n" \
  "v_mfma_f32_16x16x32_f16 a[16:19], v[82:85], v[118:121], a[16:19]\n" \
  "buffer_load_dwordx4 v[46:49], %[bxr], s[40:43], s72 offen offset:0 sc0 sc1\n" \
  "v_mfma_f32_16x16x32_f16 a[16:19], v[78:81], v[150:153], a[16:19]\n" \
  "s_nop 7\n" \
  "v_mfma_f32_16x16x32_f16 a[48:51], v[90:93], v[150:153], a[48:51]\n" \
  "ds_read_b128 v[150:153], v247 offset:12288\n" \
  "v_mfma_f32_16x16x32_f16 a[48:51], v[86:89], v[118:121], a[48:51]\n" \
  "s_waitcnt lgkmcnt(13)\n" \
  "ds_read_b128 v[118:121], v247 offset:4096\n" \
  "v_mfma_f32_16x16x32_f16 a[52:55], v[86:89], v[122:125], a[52:55]\n" \
  "buffer_load_dwordx4 v[50:53], %[bxr], s[40:43], s72 offen offset:1024 sc0 sc1\n" \
  "v_mfma_f32_16x16x32_f16 a[52:55], v[90:93], v[154:157], a[52:55]\n" \
  "v_mfma_f32_16x16x32_f16 a[20:23], v[78:81], v[154:157], a[20:23]\n" \
  "ds_read_b128 v[154:157], v247 offset:13312\n" \
  "v_mfma_f32_16x16x32_f16 a[20:23], v[82:85], v[122:125], a[20:23]\n" \
  "s_waitcnt lgkmcnt(13)\n" \
  "ds_read_b128 v[122:125], v247 offset:5120\n" \
  "v_mfma_f32_16x16x32_f16 a[24:27], v[82:85], v[126:129], a[24:27]\n" \
  "buffer_load_dwordx4 v[70:73], %[bxr], s[40:43], s72 offen offset:2048 sc0 sc1\n" \
  "v_mfma_f32_16x16x32_f16 a[24:27], v[78:81], v[158:161], a[24:27]\n" \
  "v_readlane_b32 s74, %[vtabA], 30\n" \
  "v_mfma_f32_16x16x32_f16 a[56:59], v[90:93], v[158:161], a[56:59]\n" \
  "s_bitcmp1_b32 s34, 30\n" \
  "s_cbranch_scc0 L_s75x_%=\n" \
  "L_b75x_%=:\n" \
  "ds_read_b128 v[158:161], v247 offset:14336\n" \
  "v_mfma_f32_16x16x32_f16 a[56:59], v[86:89], v[126:129], a[56:59]\n" \
  "s_waitcnt lgkmcnt(13)\n" \
  "ds_read_b128 v[126:129], v247 offset:6144\n" \
  "s_add_u32 s73, s74, s68\n" \
  "v_mfma_f32_16x16x32_f16 a[60:63], v[86:89], v[130:133], a[60:63]\n" \
  "buffer_load_dwordx4 v[74:77], %[bxr], s[40:43], s72 offen offset:3072 sc0 sc1\n" \
  "v_mfma_f32_16x16x32_f16 a[60:63], v[90:93], v[162:165], a[60:63]\n" \
  "v_mfma_f32_16x16x32_f16 a[28:31], v[78:81], v[162:165], a[28:31]\n" \
  "s_waitcnt vmcnt(11)\n" \
  "ds_read_b128 v[162:165], v247 offset:15360\n" \
  "v_mfma_f32_16x16x32_f16 a[28:31], v[82:85], v[130:133], a[28:31]\n" \
  "ds_read_b128 v[130:133], v247 offset:7168\n" \
  "s_waitcnt lgkmcnt(14)\n" \
  "v_mfma_f32_16x16x32_f16 a[0:3], v[98:101], v[102:105], a[0:3]\n" \
  "buffer_load_dwordx4 v[62:65], %[bxr], s[40:43], s73 offen offset:0 sc0 sc1\n" \
  "v_mfma_f32_16x16x32_f16 a[0:3], v[94:97], v[134:137], a[0:3]\n" \
  "v_min3_u32 v248, v238, v239, v240\n" \
  "v_min_u32 v248, v248, v241\n" \
  "s_nop 7\n" \
  "v_mfma_f32_16x16x32_f16 a[32:35], v[26:29], v[134:137], a[32:35]\n" \
  "ds_read_b128 v[134:137], v247 offset:24576\n" \
  "v_cmp_le_u32 s[34:35], s76, v248\n" \
  "v_mfma_f32_16x16x32_f16 a[32:35], v[22:25], v[102:105], a[32:35]\n" \
  "s_waitcnt lgkmcnt(13)\n" \
  "ds_read_b128 v[102:105], v247 offset:16384\n" \
  "v_mfma_f32_16x16x32_f16 a[36:39], v[22:25], v[106:109], a[36:39]\n" \
  "buffer_load_dwordx4 v[66:69], %[bxr], s[40:43], s73 offen offset:1024 sc0 sc1\n" \
  "v_mfma_f32_16x16x32_f16 a[36:39], v[26:29], v[138:141], a[36:39]\n" \
  "v_mfma_f32_16x16x32_f16 a[4:7], v[94:97], v[138:141], a[4:7]\n" \
  "ds_read_b128 v[138:141], v247 offset:25600\n" \
  "v_mfma_f32_16x16x32_f16 a[4:7], v[98:101], v[106:109], a[4:7]\n" \
  "s_waitcnt lgkmcnt(13)\n" \
  "ds_read_b128 v[106:109], v247 offset:17408\n" \
  "v_mfma_f32_16x16x32_f16 a[8:11], v[98:101], v[110:113], a[8:11]\n" \
  "buffer_load_dwordx4 v[86:89], %[bxr], s[40:43], s73 offen offset:2048 sc0 sc1\n" \
  "v_mfma_f32_16x16x32_f16 a[8:11], v[94:97], v[142:145], a[8:11]\n" \
  "v_readlane_b32 s74, %[vtabA], 31\n" \
  "v_mfma_f32_16x16x32_f16 a[40:43], v[26:29], v[142:145], a[40:43]\n" \
  "s_bitcmp1_b32 s34, 31\n" \
  "s_cbranch_scc0 L_s76x_%=\n" \
  "L_b76x_%=:\n" \
  "ds_read_b128 v[142:145], v247 offset:26624\n" \
  "v_mfma_f32_16x16x32_f16 a[40:43], v[22:25], v[110:113], a[40:43]\n" \
  "s_waitcnt lgkmcnt(13)\n" \
  "ds_read_b128 v[110:113], v247 offset:18432\n" \
  "s_add_u32 s72, s74, s68\n" \
  "v_mfma_f32_16x16x32_f16 a[44:47], v[22:25], v[114:117], a[44:47]\n" \
  "buffer_load_dwordx4 v[90:93], %[bxr], s[40:43], s73 offen offset:3072 sc0 sc1\n" \
  "v_mfma_f32_16x16x32_f16 a[44:47], v[26:29], v[146:149], a[44:47]\n" \
  "v_mfma_f32_16x16x32_f16 a[12:15], v[94:97], v[146:149], a[12:15]\n" \
  "ds_read_b128 v[146:149], v247 offset:27648\n" \
  "v_mfma_f32_16x16x32_f16 a[12:15], v[98:101], v[114:117], a[12:15]\n" \
  "s_waitcnt lgkmcnt(13)\n" \
  "ds_read_b128 v[114:117], v247 offset:19456\n" \
  "v_mfma_f32_16x16x32_f16 a[16:19], v[98:101], v[118:121], a[16:19]\n" \
  "buffer_load_dwordx4 v[78:81], %[bxr], s[40:43], s72 offen offset:0 sc0 sc1\n" \
  "v_mfma_f32_16x16x32_f16 a[16:19], v[94:97], v[150:153], a[16:19]\n" \
  "s_nop 7\n" \
  "v_mfma_f32_16x16x32_f16 a[48:51], v[26:29], v[150:153], a[48:51]\n" \
  "ds_read_b128 v[150:153], v247 offset:28672\n" \
  "v_mfma_f32_16x16x32_f16 a[48:51], v[22:25], v[118:121], a[48:51]\n" \
  "s_waitcnt lgkmcnt(13)\n" \
  "ds_read_b128 v[118:121], v247 offset:20480\n" \
  "v_mfma_f32_16x16x32_f16 a[52:55], v[22:25], v[122:125], a[52:55]\n" \
  "buffer_load_dwordx4 v[82:85], %[bxr], s[40:43], s72 offen offset:1024 sc0 sc1\n" \
  "v_mfma_f32_16x16x32_f16 a[52:55], v[26:29], v[154:157], a[52:55]\n" \
  "v_mfma_f32_16x16x32_f16 a[20:23], v[94:97], v[154:157], a[20:23]\n" \
  "ds_read_b128 v[154:157], v247 offset:29696\n" \
  "v_mfma_f32_16x16x32_f16 a[20:23], v[98:101], v[122:125], a[20:23]\n" \
  "s_waitcnt lgkmcnt(13)\n" \
  "ds_read_b128 v[122:125], v247 offset:21504\n" \
  "v_mfma_f32_16x16x32_f16 a[24:27], v[98:101], v[126:129], a[24:27]\n" \
  "v_mfma_f32_16x16x32_f16 a[24:27], v[94:97], v[158:161], a[24:27]\n" \
  "v_mfma_f32_16x16x32_f16 a[56:59], v[26:29], v[158:161], a[56:59]\n" \
  "ds_read_b128 v[158:161], v247 offset:30720\n" \
  "v_mfma_f32_16x16x32_f16 a[56:59], v[22:25], v[126:129], a[56:59]\n" \
  "s_waitcnt lgkmcnt(13)\n" \
  "ds_read_b128 v[126:129], v247 offset:22528\n" \
  "v_mfma_f32_16x16x32_f16 a[60:63], v[22:25], v[130:133], a[60:63]\n" \
  "buffer_load_dwordx4 v[22:25], %[bxr], s[40:43], s72 offen offset:2048 sc0 sc1\n" \
  "v_mfma_f32_16x16x32_f16 a[60:63], v[26:29], v[162:165], a[60:63]\n" \
  "v_readlane_b32 s74, %[vtabA], 23\n" \
  "v_mfma_f32_16x16x32_f16 a[28:31], v[94:97], v[162:165], a[28:31]\n" \
  "ds_read_b128 v[162:165], v247 offset:31744\n" \
  "v_mfma_f32_16x16x32_f16 a[28:31], v[98:101], v[130:133], a[28:31]\n" \
  "s_waitcnt vmcnt(15)\n" \
  "ds_read_b128 v[130:133], v247 offset:23552\n" \
  "s_waitcnt lgkmcnt(14)\n" \
  "v_mfma_f32_16x16x32_f16 a[0:3], v[14:17], v[102:105], a[0:3]\n" \
  "buffer_load_dwordx4 v[26:29], %[bxr], s[40:43], s72 offen offset:3072 sc0 sc1\n" \
  "v_mfma_f32_16x16x32_f16 a[0:3], v[18:21], v[134:137], a[0:3]\n" \
  "s_add_u32 s73, s74, s69\n" \
  "s_nop 7\n" \
  "v_mfma_f32_16x16x32_f16 a[32:35], v[42:45], v[134:137], a[32:35]\n" \
  "ds_read_b128 v[134:137], v245 offset:8192\n" \
  "v_mfma_f32_16x16x32_f16 a[32:35], v[38:41], v[102:105], a[32:35]\n" \
  "s_waitcnt lgkmcnt(13)\n" \
  "ds_read_b128 v[102:105], v245 offset:0\n" \
  "v_mfma_f32_16x16x32_f16 a[36:39], v[38:41], v[106:109], a[36:39]\n" \
  "buffer_load_dwordx4 v[94:97], %[bxr], s[40:43], s73 offen offset:0 sc0 sc1\n" \
  "v_mfma_f32_16x16x32_f16 a[36:39], v[42:45], v[138:141], a[36:39]\n" \
  "v_mfma_f32_16x16x32_f16 a[4:7], v[18:21], v[138:141], a[4:7]\n" \
  "ds_read_b128 v[138:141], v245 offset:9216\n" \
  "v_mfma_f32_16x16x32_f16 a[4:7], v[14:17], v[106:109], a[4:7]\n" \
  "s_waitcnt lgkmcnt(13)\n" \
  "ds_read_b128 v[106:109], v245 offset:1024\n" \
  "v_mfma_f32_16x16x32_f16 a[8:11], v[14:17], v[110:113], a[8:11]\n" \
  "buffer_load_dwordx4 v[98:101], %[bxr], s[40:43], s73 offen offset:1024 sc0 sc1\n" \
  "v_mfma_f32_16x16x32_f16 a[8:11], v[18:21], v[142:145], a[8:11]\n" \
  "v_mfma_f32_16x16x32_f16 a[40:43], v[42:45], v[142:145], a[40:43]\n" \
  "ds_read_b128 v[142:145], v245 offset:10240\n" \
  "v_mfma_f32_16x16x32_f16 a[40:43], v[38:41], v[110:113], a[40:43]\n" \
  "s_waitcnt lgkmcnt(13)\n" \
  "ds_read_b128 v[110:113], v245 offset:2048\n" \
  "v_mfma_f32_16x16x32_f16 a[44:47], v[38:41], v[114:117], a[44:47]\n" \
  "v_mfma_f32_16x16x32_f16 a[44:47], v[42:45], v[146:149], a[44:47]\n" \
  "v_mfma_f32_16x16x32_f16 a[12:15], v[18:21], v[146:149], a[12:15]\n" \
  "ds_read_b128 v[146:149], v245 offset:11264\n" \
  "v_mfma_f32_16x16x32_f16 a[12:15], v[14:17], v[114:117], a[12:15]\n" \
  "s_waitcnt lgkmcnt(13)\n" \
  "ds_read_b128 v[114:117], v245 offset:3072\n" \
  "v_mfma_f32_16x16x32_f16 a[16:19], v[14:17], v[118:121], a[16:19]\n" \
  "v_mfma_f32_16x16x32_f16 a[16:19], v[18:21], v[150:153], a[16:19]\n" \
  "s_nop 7\n" \
  "v_mfma_f32_16x16x32_f16 a[48:51], v[42:45], v[150:153], a[48:51]\n" \
  "ds_read_b128 v[150:153], v245 offset:12288\n" \
  "v_mfma_f32_16x16x32_f16 a[48:51], v[38:41], v[118:121], a[48:51]\n" \
  "s_waitcnt lgkmcnt(13)\n" \
  "ds_read_b128 v[118:121], v245 offset:4096\n" \
  "v_mfma_f32_16x16x32_f16 a[52:55], v[38:41], v[122:125], a[52:55]\n" \
  "v_mfma_f32_16x16x32_f16 a[52:55], v[42:45], v[154:157], a[52:55]\n" \
  "v_mfma_f32_16x16x32_f16 a[20:23], v[18:21], v[154:157], a[20:23]\n" \
  "ds_read_b128 v[154:157], v245 offset:13312\n" \
  "v_mfma_f32_16x16x32_f16 a[20:23], v[14:17], v[122:125], a[20:23]\n" \
  "ds_read_b128 v[122:125], v245 offset:5120\n" \
  "s_waitcnt lgkmcnt(14)\n" \
  "v_mfma_f32_16x16x32_f16 a[24:27], v[14:17], v[126:129], a[24:27]\n" \
  "v_cvt_pk_f16_f32 v217, v174, v178\n" \
  "v_cvt_pk_f16_f32 v221, v175, v179\n" \
  "v_mfma_f32_16x16x32_f16 a[24:27], v[18:21], v[158:161], a[24:27]\n" \
  "v_cvt_pk_f16_f32 v225, v176, v180\n" \
  "v_cvt_pk_f16_f32 v229, v177, v181\n" \
  "v_mfma_f32_16x16x32_f16 a[56:59], v[42:45], v[158:161], a[56:59]\n" \
  "ds_read_b128 v[158:161], v245 offset:14336\n" \
  "v_mfma_f32_16x16x32_f16 a[56:59], v[38:41], v[126:129], a[56:59]\n" \
  "s_waitcnt lgkmcnt(13)\n" \
  "ds_read_b128 v[126:129], v245 offset:6144\n" \
  "v_mfma_f32_16x16x32_f16 a[60:63], v[38:41], v[130:133], a[60:63]\n" \
  "ds_write_b128 v242, v[214:217] offset:49152\n" \
  "v_mfma_f32_16x16x32_f16 a[60:63], v[42:45], v[162:165], a[60:63]\n" \
  "buffer_load_dwordx4 v[38:41], %[bxr], s[40:43], s73 offen offset:2048 sc0 sc1\n" \
  "v_mfma_f32_16x16x32_f16 a[28:31], v[18:21], v[162:165], a[28:31]\n" \
  "ds_read_b128 v[162:165], v245 offset:15360\n" \
  "v_readlane_b32 s74, %[vtabA], 24\n" \
  "v_mfma_f32_16x16x32_f16 a[28:31], v[14:17], v[130:133], a[28:31]\n" \
  "s_waitcnt vmcnt(15)\n" \
  "ds_read_b128 v[130:133], v245 offset:7168\n" \
  "s_waitcnt lgkmcnt(14)\n" \
  "v_mfma_f32_16x16x32_f16 a[0:3], v[30:33], v[102:105], a[0:3]\n" \
  "ds_write_b128 v242, v[218:221] offset:50176\n" \
  "v_mfma_f32_16x16x32_f16 a[0:3], v[34:37], v[134:137], a[0:3]\n" \
  "buffer_load_dwordx4 v[42:45], %[bxr], s[40:43], s73 offen offset:3072 sc0 sc1\n" \
  "s_nop 7\n" \
  "v_mfma_f32_16x16x32_f16 a[32:35], v[58:61], v[134:137], a[32:35]\n" \
  "s_add_u32 s72, s74, s69\n" \
  "ds_read_b128 v[134:137], v245 offset:24576\n" \
  "v_mfma_f32_16x16x32_f16 a[32:35], v[54:57], v[102:105], a[32:35]\n" \
  "s_waitcnt lgkmcnt(14)\n" \
  "ds_read_b128 v[102:105], v245 offset:16384\n" \
  "v_mfma_f32_16x16x32_f16 a[36:39], v[54:57], v[106:109], a[36:39]\n" \
  "ds_write_b128 v242, v[222:225] offset:51200\n" \
  "v_mfma_f32_16x16x32_f16 a[36:39], v[58:61], v[138:141], a[36:39]\n" \
  "buffer_load_dwordx4 v[18:21], %[bxr], s[40:43], s72 offen offset:0 sc0 sc1\n" \
  "v_mfma_f32_16x16x32_f16 a[4:7], v[34:37], v[138:141], a[4:7]\n" \
  "ds_read_b128 v[138:141], v245 offset:25600\n" \
  "v_mfma_f32_16x16x32_f16 a[4:7], v[30:33], v[106:109], a[4:7]\n" \
  "s_waitcnt lgkmcnt(14)\n" \
  "ds_read_b128 v[106:109], v245 offset:17408\n" \
  "v_mfma_f32_16x16x32_f16 a[8:11], v[30:33], v[110:113], a[8:11]\n" \
  "ds_write_b128 v242, v[226:229] offset:52224\n" \
  "v_mfma_f32_16x16x32_f16 a[8:11], v[34:37], v[142:145], a[8:11]\n" \
  "buffer_load_dwordx4 v[14:17], %[bxr], s[40:43], s72 offen offset:1024 sc0 sc1\n" \
  "v_mfma_f32_16x16x32_f16 a[40:43], v[58:61], v[142:145], a[40:43]\n" \
  "ds_read_b128 v[142:145], v245 offset:26624\n" \
  "v_mfma_f32_16x16x32_f16 a[40:43], v[54:57], v[110:113], a[40:43]\n" \
  "s_waitcnt lgkmcnt(14)\n" \
  "ds_read_b128 v[110:113], v245 offset:18432\n" \
  "v_mfma_f32_16x16x32_f16 a[44:47], v[54:57], v[114:117], a[44:47]\n" \
  "v_mfma_f32_16x16x32_f16 a[44:47], v[58:61], v[146:149], a[44:47]\n" \
  "v_mfma_f32_16x16x32_f16 a[12:15], v[34:37], v[146:149], a[12:15]\n" \
  "ds_read_b128 v[146:149], v245 offset:27648\n" \
  "v_mfma_f32_16x16x32_f16 a[12:15], v[30:33], v[114:117], a[12:15]\n" \
  "s_waitcnt lgkmcnt(14)\n" \
  "ds_read_b128 v[114:117], v245 offset:19456\n" \
  "v_mfma_f32_16x16x32_f16 a[16:19], v[30:33], v[118:121], a[16:19]\n" \
  "v_mfma_f32_16x16x32_f16 a[16:19], v[34:37], v[150:153], a[16:19]\n" \
  "s_nop 7\n" \
  "v_mfma_f32_16x16x32_f16 a[48:51], v[58:61], v[150:153], a[48:51]\n" \
  "ds_read_b128 v[150:153], v245 offset:28672\n" \
  "v_mfma_f32_16x16x32_f16 a[48:51], v[54:57], v[118:121], a[48:51]\n" \
  "s_waitcnt lgkmcnt(14)\n" \
  "ds_read_b128 v[118:121], v245 offset:20480\n" \
  "v_mfma_f32_16x16x32_f16 a[52:55], v[54:57], v[122:125], a[52:55]\n" \
  "v_mfma_f32_16x16x32_f16 a[52:55], v[58:61], v[154:157], a[52:55]\n" \
  "v_mfma_f32_16x16x32_f16 a[20:23], v[34:37], v[154:157], a[20:23]\n" \
  "ds_read_b128 v[154:157], v245 offset:29696\n" \
  "v_mfma_f32_16x16x32_f16 a[20:23], v[30:33], v[122:125], a[20:23]\n" \
  "s_waitcnt lgkmcnt(14)\n" \
  "ds_read_b128 v[122:125], v245 offset:21504\n" \
  "v_mfma_f32_16x16x32_f16 a[24:27], v[30:33], v[126:129], a[24:27]\n" \
  "v_mfma_f32_16x16x32_f16 a[24:27], v[34:37], v[158:161], a[24:27]\n" \
  "v_mfma_f32_16x16x32_f16 a[56:59], v[58:61], v[158:161], a[56:59]\n" \
  "ds_read_b128 v[158:161], v245 offset:30720\n" \
  "v_mfma_f32_16x16x32_f16 a[56:59], v[54:57], v[126:129], a[56:59]\n" \
  "s_waitcnt lgkmcnt(14)\n" \
  "ds_read_b128 v[126:129], v245 offset:22528\n" \
  "v_mfma_f32_16x16x32_f16 a[60:63], v[54:57], v[130:133], a[60:63]\n" \
  "buffer_load_dwordx4 v[54:57], %[bxr], s[40:43], s72 offen offset:2048 sc0 sc1\n" \
  "v_mfma_f32_16x16x32_f16 a[60:63], v[58:61], v[162:165], a[60:63]\n" \
  "v_readlane_b32 s74, %[vtabA], 25\n" \
  "v_mfma_f32_16x16x32_f16 a[28:31], v[34:37], v[162:165], a[28:31]\n" \
  "ds_read_b128 v[162:165], v245 offset:31744\n" \
  "v_mfma_f32_16x16x32_f16 a[28:31], v[30:33], v[130:133], a[28:31]\n" \
  "s_waitcnt vmcnt(15)\n" \
  "ds_read_b128 v[130:133], v245 offset:23552\n" \
  "s_waitcnt lgkmcnt(14)\n" \
  "v_mfma_f32_16x16x32_f16 a[0:3], v[46:49], v[102:105], a[0:3]\n" \
  "buffer_load_dwordx4 v[58:61], %[bxr], s[40:43], s72 offen offset:3072 sc0 sc1\n" \
  "v_mfma_f32_16x16x32_f16 a[0:3], v[50:53], v[134:137], a[0:3]\n" \
  "s_waitcnt lgkmcnt(0)\n" \
  "s_barrier\n" \
  "s_add_u32 s73, s74, s69\n" \
  "s_nop 7\n" \
  "v_mfma_f32_16x16x32_f16 a[32:35], v[74:77], v[134:137], a[32:35]\n" \
  "ds_read_b128 v[134:137], v245 offset:40960\n" \
  "v_mfma_f32_16x16x32_f16 a[32:35], v[70:73], v[102:105], a[32:35]\n" \
  "ds_read_b128 v[102:105], v245 offset:32768\n" \
  "v_mfma_f32_16x16x32_f16 a[36:39], v[70:73], v[106:109], a[36:39]\n" \
  "buffer_load_dwordx4 v[34:37], %[bxr], s[40:43], s73 offen offset:0 sc0 sc1\n" \
  "v_mfma_f32_16x16x32_f16 a[36:39], v[74:77], v[138:141], a[36:39]\n" \
  "v_mfma_f32_16x16x32_f16 a[4:7], v[50:53], v[138:141], a[4:7]\n" \
  "ds_read_b128 v[138:141], v245 offset:41984\n" \
  "v_mfma_f32_16x16x32_f16 a[4:7], v[46:49], v[106:109], a[4:7]\n" \
  "ds_read_b128 v[106:109], v245 offset:33792\n" \
  "v_mfma_f32_16x16x32_f16 a[8:11], v[46:49], v[110:113], a[8:11]\n" \
  "buffer_load_dwordx4 v[30:33], %[bxr], s[40:43], s73 offen offset:1024 sc0 sc1\n" \
  "v_mfma_f32_16x16x32_f16 a[8:11], v[50:53], v[142:145], a[8:11]\n" \
  "v_mfma_f32_16x16x32_f16 a[40:43], v[74:77], v[142:145], a[40:43]\n" \
  "ds_read_b128 v[142:145], v245 offset:43008\n" \
  "v_mfma_f32_16x16x32_f16 a[40:43], v[70:73], v[110:113], a[40:43]\n" \
  "ds_read_b128 v[110:113], v245 offset:34816\n" \
  "v_mfma_f32_16x16x32_f16 a[44:47], v[70:73], v[114:117], a[44:47]\n" \
  "v_mfma_f32_16x16x32_f16 a[44:47], v[74:77], v[146:149], a[44:47]\n" \
  "v_mfma_f32_16x16x32_f16 a[12:15], v[50:53], v[146:149], a[12:15]\n" \
  "ds_read_b128 v[146:149], v245 offset:44032\n" \
  "v_mfma_f32_16x16x32_f16 a[12:15], v[46:49], v[114:117], a[12:15]\n" \
  "ds_read_b128 v[114:117], v245 offset:35840\n" \
  "v_mfma_f32_16x16x32_f16 a[16:19], v[46:49], v[118:121], a[16:19]\n" \
  "v_mfma_f32_16x16x32_f16 a[16:19], v[50:53], v[150:153], a[16:19]\n" \
  "s_nop 7\n" \
  "v_mfma_f32_16x16x32_f16 a[48:51], v[74:77], v[150:153], a[48:51]\n" \
  "ds_read_b128 v[150:153], v245 offset:45056\n" \
  "v_mfma_f32_16x16x32_f16 a[48:51], v[70:73], v[118:121], a[48:51]\n" \
  "ds_read_b128 v[118:121], v245 offset:36864\n" \
  "v_mfma_f32_16x16x32_f16 a[52:55], v[70:73], v[122:125], a[52:55]\n" \
  "v_mfma_f32_16x16x32_f16 a[52:55], v[74:77], v[154:157], a[52:55]\n" \
  "v_mfma_f32_16x16x32_f16 a[20:23], v[50:53], v[154:157], a[20:23]\n" \
  "ds_read_b128 v[154:157], v245 offset:46080\n" \
  "v_mfma_f32_16x16x32_f16 a[20:23], v[46:49], v[122:125], a[20:23]\n" \
  "ds_read_b128 v[122:125], v245 offset:37888\n" \
  "v_mfma_f32_16x16x32_f16 a[24:27], v[46:49], v[126:129], a[24:27]\n" \
  "v_mfma_f32_16x16x32_f16 a[24:27], v[50:53], v[158:161], a[24:27]\n" \
  "v_mfma_f32_16x16x32_f16 a[56:59], v[74:77], v[158:161], a[56:59]\n" \
  "ds_read_b128 v[158:161], v245 offset:47104\n" \
  "v_mfma_f32_16x16x32_f16 a[56:59], v[70:73], v[126:129], a[56:59]\n" \
  "ds_read_b128 v[126:129], v245 offset:38912\n" \
  "v_mfma_f32_16x16x32_f16 a[60:63], v[70:73], v[130:133], a[60:63]\n" \
  "buffer_load_dwordx4 v[70:73], %[bxr], s[40:43], s73 offen offset:2048 sc0 sc1\n" \
  "v_mfma_f32_16x16x32_f16 a[60:63], v[74:77], v[162:165], a[60:63]\n" \
  "v_readlane_b32 s74, %[vtabA], 26\n" \
  "v_mfma_f32_16x16x32_f16 a[28:31], v[50:53], v[162:165], a[28:31]\n" \
  "ds_read_b128 v[162:165], v245 offset:48128\n" \
  "v_mfma_f32_16x16x32_f16 a[28:31], v[46:49], v[130:133], a[28:31]\n" \
  "s_waitcnt vmcnt(15)\n" \
  "ds_read_b128 v[130:133], v245 offset:39936\n" \
  "s_waitcnt lgkmcnt(14)\n" \
  "v_mfma_f32_16x16x32_f16 a[0:3], v[62:65], v[102:105], a[0:3]\n" \
  "buffer_load_dwordx4 v[74:77], %[bxr], s[40:43], s73 offen offset:3072 sc0 sc1\n" \
  "v_mfma_f32_16x16x32_f16 a[0:3], v[66:69], v[134:137], a[0:3]\n" \
  "s_add_u32 s72, s74, s69\n" \
  "s_nop 7\n" \
  "v_mfma_f32_16x16x32_f16 a[32:35], v[90:93], v[134:137], a[32:35]\n" \
  "ds_read_b128 v[134:137], v245 offset:57344\n" \
  "v_mfma_f32_16x16x32_f16 a[32:35], v[86:89], v[102:105], a[32:35]\n" \
  "s_waitcnt lgkmcnt(13)\n" \
  "ds_read_b128 v[102:105], v245 offset:49152\n" \
  "v_mfma_f32_16x16x32_f16 a[36:39], v[86:89], v[106:109], a[36:39]\n" \
  "buffer_load_dwordx4 v[50:53], %[bxr], s[40:43], s72 offen offset:0 sc0 sc1\n" \
  "v_mfma_f32_16x16x32_f16 a[36:39], v[90:93], v[138:141], a[36:39]\n" \
  "v_mfma_f32_16x16x32_f16 a[4:7], v[66:69], v[138:141], a[4:7]\n" \
  "ds_read_b128 v[138:141], v245 offset:58368\n" \
  "v_mfma_f32_16x16x32_f16 a[4:7], v[62:65], v[106:109], a[4:7]\n" \
  "s_waitcnt lgkmcnt(13)\n" \
  "ds_read_b128 v[106:109], v245 offset:50176\n" \
  "v_mfma_f32_16x16x32_f16 a[8:11], v[62:65], v[110:113], a[8:11]\n" \
  "buffer_load_dwordx4 v[46:49], %[bxr], s[40:43], s72 offen offset:1024 sc0 sc1\n" \
  "v_mfma_f32_16x16x32_f16 a[8:11], v[66:69], v[142:145], a[8:11]\n" \
  "v_mfma_f32_16x16x32_f16 a[40:43], v[90:93], v[142:145], a[40:43]\n" \
  "ds_read_b128 v[142:145], v245 offset:59392\n" \
  "v_mfma_f32_16x16x32_f16 a[40:43], v[86:89], v[110:113], a[40:43]\n" \
  "s_waitcnt lgkmcnt(13)\n" \
  "ds_read_b128 v[110:113], v245 offset:51200\n" \
  "v_mfma_f32_16x16x32_f16 a[44:47], v[86:89], v[114:117], a[44:47]\n" \
  "v_mfma_f32_16x16x32_f16 a[44:47], v[90:93], v[146:149], a[44:47]\n" \
  "v_mfma_f32_16x16x32_f16 a[12:15], v[66:69], v[146:149], a[12:15]\n" \
  "ds_read_b128 v[146:149], v245 offset:60416\n" \
  "v_mfma_f32_16x16x32_f16 a[12:15], v[62:65], v[114:117], a[12:15]\n" \
  "s_waitcnt lgkmcnt(13)\n" \
  "ds_read_b128 v[114:117], v245 offset:52224\n" \
  "v_mfma_f32_16x16x32_f16 a[16:19], v[62:65], v[118:121], a[16:19]\n" \
  "v_mfma_f32_16x16x32_f16 a[16:19], v[66:69], v[150:153], a[16:19]\n" \
  "s_nop 7\n" \
  "v_mfma_f32_16x16x32_f16 a[48:51], v[90:93], v[150:153], a[48:51]\n" \
  "ds_read_b128 v[150:153], v245 offset:61440\n" \
  "v_mfma_f32_16x16x32_f16 a[48:51], v[86:89], v[118:121], a[48:51]\n" \
  "s_waitcnt lgkmcnt(13)\n" \
  "ds_read_b128 v[118:121], v245 offset:53248\n" \
  "v_mfma_f32_16x16x32_f16 a[52:55], v[86:89], v[122:125], a[52:55]\n" \
  "v_mfma_f32_16x16x32_f16 a[52:55], v[90:93], v[154:157], a[52:55]\n" \
  "v_mfma_f32_16x16x32_f16 a[20:23], v[66:69], v[154:157], a[20:23]\n" \
  "ds_read_b128 v[154:157], v245 offset:62464\n" \
  "v_mfma_f32_16x16x32_f16 a[20:23], v[62:65], v[122:125], a[20:23]\n" \
  "s_waitcnt lgkmcnt(13)\n" \
  "ds_read_b128 v[122:125], v245 offset:54272\n" \
  "v_mfma_f32_16x16x32_f16 a[24:27], v[62:65], v[126:129], a[24:27]\n" \
  "v_mfma_f32_16x16x32_f16 a[24:27], v[66:69], v[158:161], a[24:27]\n" \
  "v_mfma_f32_16x16x32_f16 a[56:59], v[90:93], v[158:161], a[56:59]\n" \
  "ds_read_b128 v[158:161], v245 offset:63488\n" \
  "v_mfma_f32_16x16x32_f16 a[56:59], v[86:89], v[126:129], a[56:59]\n" \
  "s_waitcnt lgkmcnt(13)\n" \
  "ds_read_b128 v[126:129], v245 offset:55296\n" \
  "v_mfma_f32_16x16x32_f16 a[60:63], v[86:89], v[130:133], a[60:63]\n" \
  "buffer_load_dwordx4 v[86:89], %[bxr], s[40:43], s72 offen offset:2048 sc0 sc1\n" \
  "v_mfma_f32_16x16x32_f16 a[60:63], v[90:93], v[162:165], a[60:63]\n" \
  "v_readlane_b32 s74, %[vtabA], 27\n" \
  "v_mfma_f32_16x16x32_f16 a[28:31], v[66:69], v[162:165], a[28:31]\n" \
  "ds_read_b128 v[162:165], v245 offset:64512\n" \
  "v_mfma_f32_16x16x32_f16 a[28:31], v[62:65], v[130:133], a[28:31]\n" \
  "s_waitcnt vmcnt(15)\n" \
  "ds_read_b128 v[130:133], v245 offset:56320\n" \
  "s_waitcnt lgkmcnt(14)\n" \
  "v_mfma_f32_16x16x32_f16 a[0:3], v[78:81], v[102:105], a[0:3]\n" \
  "buffer_load_dwordx4 v[90:93], %[bxr], s[40:43], s72 offen offset:3072 sc0 sc1\n" \
  "v_mfma_f32_16x16x32_f16 a[0:3], v[82:85], v[134:137], a[0:3]\n" \
  "s_add_u32 s73, s74, s69\n" \
  "s_nop 7\n" \
  "v_mfma_f32_16x16x32_f16 a[32:35], v[26:29], v[134:137], a[32:35]\n" \
  "ds_read_b128 v[134:137], v246 offset:24576\n" \
  "v_mfma_f32_16x16x32_f16 a[32:35], v[22:25], v[102:105], a[32:35]\n" \
  "s_waitcnt lgkmcnt(13)\n" \
  "ds_read_b128 v[102:105], v246 offset:16384\n" \
  "v_mfma_f32_16x16x32_f16 a[36:39], v[22:25], v[106:109], a[36:39]\n" \
  "buffer_load_dwordx4 v[66:69], %[bxr], s[40:43], s73 offen offset:0 sc0 sc1\n" \
  "v_mfma_f32_16x16x32_f16 a[36:39], v[26:29], v[138:141], a[36:39]\n" \
  "v_mfma_f32_16x16x32_f16 a[4:7], v[82:85], v[138:141], a[4:7]\n" \
  "ds_read_b128 v[138:141], v246 offset:25600\n" \
  "v_mfma_f32_16x16x32_f16 a[4:7], v[78:81], v[106:109], a[4:7]\n" \
  "s_waitcnt lgkmcnt(13)\n" \
  "ds_read_b128 v[106:109], v246 offset:17408\n" \
  "v_mfma_f32_16x16x32_f16 a[8:11], v[78:81], v[110:113], a[8:11]\n" \
  "buffer_load_dwordx4 v[62:65], %[bxr], s[40:43], s73 offen offset:1024 sc0 sc1\n" \
  "v_mfma_f32_16x16x32_f16 a[8:11], v[82:85], v[142:145], a[8:11]\n" \
  "v_mfma_f32_16x16x32_f16 a[40:43], v[26:29], v[142:145], a[40:43]\n" \
  "ds_read_b128 v[142:145], v246 offset:26624\n" \
  "v_mfma_f32_16x16x32_f16 a[40:43], v[22:25], v[110:113], a[40:43]\n" \
  "s_waitcnt lgkmcnt(13)\n" \
  "ds_read_b128 v[110:113], v246 offset:18432\n" \
  "v_mfma_f32_16x16x32_f16 a[44:47], v[22:25], v[114:117], a[44:47]\n" \
  "v_mfma_f32_16x16x32_f16 a[44:47], v[26:29], v[146:149], a[44:47]\n" \
  "v_mfma_f32_16x16x32_f16 a[12:15], v[82:85], v[146:149], a[12:15]\n" \
  "ds_read_b128 v[146:149], v246 offset:27648\n" \
  "v_mfma_f32_16x16x32_f16 a[12:15], v[78:81], v[114:117], a[12:15]\n" \
  "s_waitcnt lgkmcnt(13)\n" \
  "ds_read_b128 v[114:117], v246 offset:19456\n" \
  "v_mfma_f32_16x16x32_f16 a[16:19], v[78:81], v[118:121], a[16:19]\n" \
  "v_mfma_f32_16x16x32_f16 a[16:19], v[82:85], v[150:153], a[16:19]\n" \
  "s_nop 7\n" \
  "v_mfma_f32_16x16x32_f16 a[48:51], v[26:29], v[150:153], a[48:51]\n" \
  "ds_read_b128 v[150:153], v246 offset:28672\n" \
  "v_mfma_f32_16x16x32_f16 a[48:51], v[22:25], v[118:121], a[48:51]\n" \
  "s_waitcnt lgkmcnt(13)\n" \
  "ds_read_b128 v[118:121], v246 offset:20480\n" \
  "v_mfma_f32_16x16x32_f16 a[52:55], v[22:25], v[122:125], a[52:55]\n" \
  "v_mfma_f32_16x16x32_f16 a[52:55], v[26:29], v[154:157], a[52:55]\n" \
  "v_mfma_f32_16x16x32_f16 a[20:23], v[82:85], v[154:157], a[20:23]\n" \
  "ds_read_b128 v[154:157], v246 offset:29696\n" \
  "v_mfma_f32_16x16x32_f16 a[20:23], v[78:81], v[122:125], a[20:23]\n" \
  "s_waitcnt lgkmcnt(13)\n" \
  "ds_read_b128 v[122:125], v246 offset:21504\n" \
  "v_mfma_f32_16x16x32_f16 a[24:27], v[78:81], v[126:129], a[24:27]\n" \
  "v_mfma_f32_16x16x32_f16 a[24:27], v[82:85], v[158:161], a[24:27]\n" \
  "s_add_u32 s83, s90, 0x0\n" \
  "v_mfma_f32_16x16x32_f16 a[56:59], v[26:29], v[158:161], a[56:59]\n" \
  "ds_read_b128 v[158:161], v246 offset:30720\n" \
  "v_mfma_f32_16x16x32_f16 a[56:59], v[22:25], v[126:129], a[56:59]\n" \
  "s_waitcnt lgkmcnt(13)\n" \
  "ds_read_b128 v[126:129], v246 offset:22528\n" \
  "v_mfma_f32_16x16x32_f16 a[60:63], v[22:25], v[130:133], a[60:63]\n" \
  "buffer_load_dwordx4 v[22:25], %[bxr], s[40:43], s73 offen offset:2048 sc0 sc1\n" \
  "v_mfma_f32_16x16x32_f16 a[60:63], v[26:29], v[162:165], a[60:63]\n" \
  "v_readlane_b32 s74, %[vtabA], 28\n" \
  "v_mfma_f32_16x16x32_f16 a[28:31], v[82:85], v[162:165], a[28:31]\n" \
  "ds_read_b128 v[162:165], v246 offset:31744\n" \
  "v_mfma_f32_16x16x32_f16 a[28:31], v[78:81], v[130:133], a[28:31]\n" \
  "s_waitcnt vmcnt(15)\n" \
  "ds_read_b128 v[130:133], v246 offset:23552\n" \
  "s_waitcnt lgkmcnt(14)\n" \
  "v_mfma_f32_16x16x32_f16 a[128:131], v[94:97], v[102:105], a[128:131]\n" \
  "buffer_load_dwordx4 v[26:29], %[bxr], s[40:43], s73 offen offset:3072 sc0 sc1\n" \
  "v_mfma_f32_16x16x32_f16 a[128:131], v[98:101], v[134:137], a[128:131]\n" \
  "buffer_store_dwordx4 a[0:3], %[voffO], s[44:47], s83 offen offset:0 sc1\n" \
  "s_nop 7\n" \
  "v_mfma_f32_16x16x32_f16 a[160:163], v[42:45], v[134:137], a[160:163]\n" \
  "s_add_u32 s72, s74, s69\n" \
  "ds_read_b128 v[134:137], v246 offset:40960\n" \
  "v_mfma_f32_16x16x32_f16 a[160:163], v[38:41], v[102:105], a[160:163]\n" \
  "s_waitcnt lgkmcnt(13)\n" \
  "ds_read_b128 v[102:105], v246 offset:32768\n" \
  "v_mfma_f32_16x16x32_f16 a[164:167], v[38:41], v[106:109], a[164:167]\n" \
  "buffer_load_dwordx4 v[82:85], %[bxr], s[40:43], s72 offen offset:0 sc0 sc1\n" \
  "v_mfma_f32_16x16x32_f16 a[164:167], v[42:45], v[138:141], a[164:167]\n" \
  "v_mfma_f32_16x16x32_f16 a[132:135], v[98:101], v[138:141], a[132:135]\n" \
  "ds_read_b128 v[138:141], v246 offset:41984\n" \
  "v_mfma_f32_16x16x32_f16 a[132:135], v[94:97], v[106:109], a[132:135]\n" \
  "s_waitcnt lgkmcnt(13)\n" \
  "ds_read_b128 v[106:109], v246 offset:33792\n" \
  "v_mfma_f32_16x16x32_f16 a[136:139], v[94:97], v[110:113], a[136:139]\n" \
  "buffer_load_dwordx4 v[78:81], %[bxr], s[40:43], s72 offen offset:1024 sc0 sc1\n" \
  "v_mfma_f32_16x16x32_f16 a[136:139], v[98:101], v[142:145], a[136:139]\n" \
  "v_mfma_f32_16x16x32_f16 a[168:171], v[42:45], v[142:145], a[168:171]\n" \
  "ds_read_b128 v[142:145], v246 offset:43008\n" \
  "v_mfma_f32_16x16x32_f16 a[168:171], v[38:41], v[110:113], a[168:171]\n" \
  "s_waitcnt lgkmcnt(13)\n" \
  "ds_read_b128 v[110:113], v246 offset:34816\n" \
  "v_mfma_f32_16x16x32_f16 a[172:175], v[38:41], v[114:117], a[172:175]\n" \
  "v_mfma_f32_16x16x32_f16 a[172:175], v[42:45], v[146:149], a[172:175]\n" \
  "v_mfma_f32_16x16x32_f16 a[140:143], v[98:101], v[146:149], a[140:143]\n" \
  "ds_read_b128 v[146:149], v246 offset:44032\n" \
  "v_mfma_f32_16x16x32_f16 a[140:143], v[94:97], v[114:117], a[140:143]\n" \
  "s_waitcnt lgkmcnt(13)\n" \
  "ds_read_b128 v[114:117], v246 offset:35840\n" \
  "v_mfma_f32_16x16x32_f16 a[144:147], v[94:97], v[118:121], a[144:147]\n" \
  "v_mfma_f32_16x16x32_f16 a[144:147], v[98:101], v[150:153], a[144:147]\n" \
  "s_nop 7\n" \
  "v_mfma_f32_16x16x32_f16 a[176:179], v[42:45], v[150:153], a[176:179]\n" \
  "ds_read_b128 v[150:153], v246 offset:45056\n" \
  "v_mfma_f32_16x16x32_f16 a[176:179], v[38:41], v[118:121], a[176:179]\n" \
  "s_waitcnt lgkmcnt(13)\n" \
  "ds_read_b128 v[118:121], v246 offset:36864\n" \
  "v_mfma_f32_16x16x32_f16 a[180:183], v[38:41], v[122:125], a[180:183]\n" \
  "v_mfma_f32_16x16x32_f16 a[180:183], v[42:45], v[154:157], a[180:183]\n" \
  "v_mfma_f32_16x16x32_f16 a[148:151], v[98:101], v[154:157], a[148:151]\n" \
  "ds_read_b128 v[154:157], v246 offset:46080\n" \
  "v_mfma_f32_16x16x32_f16 a[148:151], v[94:97], v[122:125], a[148:151]\n" \
  "s_waitcnt lgkmcnt(13)\n" \
  "ds_read_b128 v[122:125], v246 offset:37888\n" \
  "v_mfma_f32_16x16x32_f16 a[152:155], v[94:97], v[126:129], a[152:155]\n" \
  "v_mfma_f32_16x16x32_f16 a[152:155], v[98:101], v[158:161], a[152:155]\n" \
  "buffer_store_dwordx4 a[16:19], %[voffO], s[44:47], s83 offen offset:256 sc1\n" \
  "v_mfma_f32_16x16x32_f16 a[184:187], v[42:45], v[158:161], a[184:187]\n" \
  "ds_read_b128 v[158:161], v246 offset:47104\n" \
  "v_mfma_f32_16x16x32_f16 a[184:187], v[38:41], v[126:129], a[184:187]\n" \
  "s_waitcnt lgkmcnt(13)\n" \
  "ds_read_b128 v[126:129], v246 offset:38912\n" \
  "v_mfma_f32_16x16x32_f16 a[188:191], v[38:41], v[130:133], a[188:191]\n" \
  "buffer_load_dwordx4 v[38:41], %[bxr], s[40:43], s72 offen offset:2048 sc0 sc1\n" \
  "v_mfma_f32_16x16x32_f16 a[188:191], v[42:45], v[162:165], a[188:191]\n" \
  "v_readlane_b32 s74, %[vtabA], 29\n" \
  "v_mfma_f32_16x16x32_f16 a[156:159], v[98:101], v[162:165], a[156:159]\n" \
  "ds_read_b128 v[162:165], v246 offset:48128\n" \
  "v_mfma_f32_16x16x32_f16 a[156:159], v[94:97], v[130:133], a[156:159]\n" \
  "s_waitcnt vmcnt(17)\n" \
  "ds_read_b128 v[130:133], v246 offset:39936\n" \
  "s_waitcnt lgkmcnt(14)\n" \
  "v_mfma_f32_16x16x32_f16 a[128:131], v[18:21], v[102:105], a[128:131]\n" \
  "buffer_load_dwordx4 v[42:45], %[bxr], s[40:43], s72 offen offset:3072 sc0 sc1\n" \
  "v_mfma_f32_16x16x32_f16 a[128:131], v[14:17], v[134:137], a[128:131]\n" \
  "s_add_u32 s73, s74, s69\n" \
  "s_nop 7\n" \
  "v_mfma_f32_16x16x32_f16 a[160:163], v[58:61], v[134:137], a[160:163]\n" \
  "ds_read_b128 v[134:137], v246 offset:57344\n" \
  "v_mfma_f32_16x16x32_f16 a[160:163], v[54:57], v[102:105], a[160:163]\n" \
  "s_waitcnt lgkmcnt(13)\n" \
  "ds_read_b128 v[102:105], v246 offset:49152\n" \
  "v_mfma_f32_16x16x32_f16 a[164:167], v[54:57], v[106:109], a[164:167]\n" \
  "buffer_load_dwordx4 v[98:101], %[bxr], s[40:43], s73 offen offset:0 sc0 sc1\n" \
  "v_mfma_f32_16x16x32_f16 a[164:167], v[58:61], v[138:141], a[164:167]\n" \
  "v_mfma_f32_16x16x32_f16 a[132:135], v[14:17], v[138:141], a[132:135]\n" \
  "ds_read_b128 v[138:141], v246 offset:58368\n" \
  "v_mfma_f32_16x16x32_f16 a[132:135], v[18:21], v[106:109], a[132:135]\n" \
  "s_waitcnt lgkmcnt(13)\n" \
  "ds_read_b128 v[106:109], v246 offset:50176\n" \
  "v_mfma_f32_16x16x32_f16 a[136:139], v[18:21], v[110:113], a[136:139]\n" \
  "buffer_load_dwordx4 v[94:97], %[bxr], s[40:43], s73 offen offset:1024 sc0 sc1\n" \
  "v_mfma_f32_16x16x32_f16 a[136:139], v[14:17], v[142:145], a[136:139]\n" \
  "s_add_u32 s84, s90, 0x10000\n" \
  "v_mfma_f32_16x16x32_f16 a[168:171], v[58:61], v[142:145], a[168:171]\n" \
  "ds_read_b128 v[142:145], v246 offset:59392\n" \
  "v_mfma_f32_16x16x32_f16 a[168:171], v[54:57], v[110:113], a[168:171]\n" \
  "s_waitcnt lgkmcnt(13)\n" \
  "ds_read_b128 v[110:113], v246 offset:51200\n" \
  "v_mfma_f32_16x16x32_f16 a[172:175], v[54:57], v[114:117], a[172:175]\n" \
  "v_mfma_f32_16x16x32_f16 a[172:175], v[58:61], v[146:149], a[172:175]\n" \
  "v_mfma_f32_16x16x32_f16 a[140:143], v[14:17], v[146:149], a[140:143]\n" \
  "ds_read_b128 v[146:149], v246 offset:60416\n" \
  "v_mfma_f32_16x16x32_f16 a[140:143], v[18:21], v[114:117], a[140:143]\n" \
  "s_waitcnt lgkmcnt(13)\n" \
  "ds_read_b128 v[114:117], v246 offset:52224\n" \
  "v_mfma_f32_16x16x32_f16 a[144:147], v[18:21], v[118:121], a[144:147]\n" \
  "v_mfma_f32_16x16x32_f16 a[144:147], v[14:17], v[150:153], a[144:147]\n" \
  "buffer_store_dwordx4 a[4:7], %[voffO], s[44:47], s84 offen offset:0 sc1\n" \
  "s_nop 7\n" \
  "v_mfma_f32_16x16x32_f16 a[176:179], v[58:61], v[150:153], a[176:179]\n" \
  "ds_read_b128 v[150:153], v246 offset:61440\n" \
  "v_mfma_f32_16x16x32_f16 a[176:179], v[54:57], v[118:121], a[176:179]\n" \
  "s_waitcnt lgkmcnt(13)\n" \
  "ds_read_b128 v[118:121], v246 offset:53248\n" \
  "v_mfma_f32_16x16x32_f16 a[180:183], v[54:57], v[122:125], a[180:183]\n" \
  "v_mfma_f32_16x16x32_f16 a[180:183], v[58:61], v[154:157], a[180:183]\n" \
  "v_mfma_f32_16x16x32_f16 a[148:151], v[14:17], v[154:157], a[148:151]\n" \
  "ds_read_b128 v[154:157], v246 offset:62464\n" \
  "v_mfma_f32_16x16x32_f16 a[148:151], v[18:21], v[122:125], a[148:151]\n" \
  "s_waitcnt lgkmcnt(13)\n" \
  "ds_read_b128 v[122:125], v246 offset:54272\n" \
  "v_mfma_f32_16x16x32_f16 a[152:155], v[18:21], v[126:129], a[152:155]\n" \
  "v_mfma_f32_16x16x32_f16 a[152:155], v[14:17], v[158:161], a[152:155]\n" \
  "v_mfma_f32_16x16x32_f16 a[184:187], v[58:61], v[158:161], a[184:187]\n" \
  "ds_read_b128 v[158:161], v246 offset:63488\n" \
  "v_mfma_f32_16x16x32_f16 a[184:187], v[54:57], v[126:129], a[184:187]\n" \
  "s_waitcnt lgkmcnt(13)\n" \
  "ds_read_b128 v[126:129], v246 offset:55296\n" \
  "v_mfma_f32_16x16x32_f16 a[188:191], v[54:57], v[130:133], a[188:191]\n" \
  "buffer_load_dwordx4 v[54:57], %[bxr], s[40:43], s73 offen offset:2048 sc0 sc1\n" \
  "v_mfma_f32_16x16x32_f16 a[188:191], v[58:61], v[162:165], a[188:191]\n" \
  "v_readlane_b32 s74, %[vtabA], 30\n" \
  "v_mfma_f32_16x16x32_f16 a[156:159], v[14:17], v[162:165], a[156:159]\n" \
  "ds_read_b128 v[162:165], v246 offset:64512\n" \
  "v_mfma_f32_16x16x32_f16 a[156:159], v[18:21], v[130:133], a[156:159]\n" \
  "s_waitcnt vmcnt(18)\n" \
  "ds_read_b128 v[130:133], v246 offset:56320\n" \
  "s_waitcnt lgkmcnt(14)\n" \
  "v_mfma_f32_16x16x32_f16 a[128:131], v[34:37], v[102:105], a[128:131]\n" \
  "buffer_load_dwordx4 v[58:61], %[bxr], s[40:43], s73 offen offset:3072 sc0 sc1\n" \
  "v_mfma_f32_16x16x32_f16 a[128:131], v[30:33], v[134:137], a[128:131]\n" \
  "s_add_u32 s72, s74, s69\n" \
  "s_nop 7\n" \
  "v_mfma_f32_16x16x32_f16 a[160:163], v[74:77], v[134:137], a[160:163]\n" \
  "ds_read_b128 v[134:137], v247 offset:8192\n" \
  "v_mfma_f32_16x16x32_f16 a[160:163], v[70:73], v[102:105], a[160:163]\n" \
  "s_waitcnt lgkmcnt(13)\n" \
  "ds_read_b128 v[102:105], v247 offset:0\n" \
  "v_mfma_f32_16x16x32_f16 a[164:167], v[70:73], v[106:109], a[164:167]\n" \
  "buffer_load_dwordx4 v[14:17], %[bxr], s[40:43], s72 offen offset:0 sc0 sc1\n" \
  "v_mfma_f32_16x16x32_f16 a[164:167], v[74:77], v[138:141], a[164:167]\n" \
  "v_mfma_f32_16x16x32_f16 a[132:135], v[30:33], v[138:141], a[132:135]\n" \
  "ds_read_b128 v[138:141], v247 offset:9216\n" \
  "v_mfma_f32_16x16x32_f16 a[132:135], v[34:37], v[106:109], a[132:135]\n" \
  "s_waitcnt lgkmcnt(13)\n" \
  "ds_read_b128 v[106:109], v247 offset:1024\n" \
  "v_mfma_f32_16x16x32_f16 a[136:139], v[34:37], v[110:113], a[136:139]\n" \
  "buffer_load_dwordx4 v[18:21], %[bxr], s[40:43], s72 offen offset:1024 sc0 sc1\n" \
  "v_mfma_f32_16x16x32_f16 a[136:139], v[30:33], v[142:145], a[136:139]\n" \
  "buffer_store_dwordx4 a[20:23], %[voffO], s[44:47], s84 offen offset:256 sc1\n" \
  "v_mfma_f32_16x16x32_f16 a[168:171], v[74:77], v[142:145], a[168:171]\n" \
  "ds_read_b128 v[142:145], v247 offset:10240\n" \
  "v_mfma_f32_16x16x32_f16 a[168:171], v[70:73], v[110:113], a[168:171]\n" \
  "s_waitcnt lgkmcnt(13)\n" \
  "ds_read_b128 v[110:113], v247 offset:2048\n" \
  "v_mfma_f32_16x16x32_f16 a[172:175], v[70:73], v[114:117], a[172:175]\n" \
  "v_mfma_f32_16x16x32_f16 a[172:175], v[74:77], v[146:149], a[172:175]\n" \
  "v_mfma_f32_16x16x32_f16 a[140:143], v[30:33], v[146:149], a[140:143]\n" \
  "ds_read_b128 v[146:149], v247 offset:11264\n" \
  "v_mfma_f32_16x16x32_f16 a[140:143], v[34:37], v[114:117], a[140:143]\n" \
  "s_waitcnt lgkmcnt(13)\n" \
  "ds_read_b128 v[114:117], v247 offset:3072\n" \
  "v_mfma_f32_16x16x32_f16 a[144:147], v[34:37], v[118:121], a[144:147]\n" \
  "v_mfma_f32_16x16x32_f16 a[144:147], v[30:33], v[150:153], a[144:147]\n" \
  "s_nop 7\n" \
  "v_mfma_f32_16x16x32_f16 a[176:179], v[74:77], v[150:153], a[176:179]\n" \
  "ds_read_b128 v[150:153], v247 offset:12288\n" \
  "v_mfma_f32_16x16x32_f16 a[176:179], v[70:73], v[118:121], a[176:179]\n" \
  "s_waitcnt lgkmcnt(13)\n" \
  "ds_read_b128 v[118:121], v247 offset:4096\n" \
  "v_mfma_f32_16x16x32_f16 a[180:183], v[70:73], v[122:125], a[180:183]\n" \
  "v_mfma_f32_16x16x32_f16 a[180:183], v[74:77], v[154:157], a[180:183]\n" \
  "v_mfma_f32_16x16x32_f16 a[148:151], v[30:33], v[154:157], a[148:151]\n" \
  "ds_read_b128 v[154:157], v247 offset:13312\n" \
  "v_mfma_f32_16x16x32_f16 a[148:151], v[34:37], v[122:125], a[148:151]\n" \
  "s_waitcnt lgkmcnt(13)\n" \
  "ds_read_b128 v[122:125], v247 offset:5120\n" \
  "v_mfma_f32_16x16x32_f16 a[152:155], v[34:37], v[126:129], a[152:155]\n" \
  "v_mfma_f32_16x16x32_f16 a[152:155], v[30:33], v[158:161], a[152:155]\n" \
  "s_add_u32 s83, s90, 0x20000\n" \
  "v_mfma_f32_16x16x32_f16 a[184:187], v[74:77], v[158:161], a[184:187]\n" \
  "ds_read_b128 v[158:161], v247 offset:14336\n" \
  "v_mfma_f32_16x16x32_f16 a[184:187], v[70:73], v[126:129], a[184:187]\n" \
  "s_waitcnt lgkmcnt(13)\n" \
  "ds_read_b128 v[126:129], v247 offset:6144\n" \
  "v_mfma_f32_16x16x32_f16 a[188:191], v[70:73], v[130:133], a[188:191]\n" \
  "buffer_load_dwordx4 v[70:73], %[bxr], s[40:43], s72 offen offset:2048 sc0 sc1\n" \
  "v_mfma_f32_16x16x32_f16 a[188:191], v[74:77], v[162:165], a[188:191]\n" \
  "v_readlane_b32 s74, %[vtabA], 31\n" \
  "v_mfma_f32_16x16x32_f16 a[156:159], v[30:33], v[162:165], a[156:159]\n" \
  "ds_read_b128 v[162:165], v247 offset:15360\n" \
  "v_mfma_f32_16x16x32_f16 a[156:159], v[34:37], v[130:133], a[156:159]\n" \
  "s_waitcnt vmcnt(19)\n" \
  "ds_read_b128 v[130:133], v247 offset:7168\n" \
  "s_waitcnt lgkmcnt(14)\n" \
  "v_mfma_f32_16x16x32_f16 a[128:131], v[50:53], v[102:105], a[128:131]\n" \
  "buffer_load_dwordx4 v[74:77], %[bxr], s[40:43], s72 offen offset:3072 sc0 sc1\n" \
  "v_mfma_f32_16x16x32_f16 a[128:131], v[46:49], v[134:137], a[128:131]\n" \
  "buffer_store_dwordx4 a[8:11], %[voffO], s[44:47], s83 offen offset:0 sc1\n" \
  "s_nop 7\n" \
  "v_mfma_f32_16x16x32_f16 a[160:163], v[90:93], v[134:137], a[160:163]\n" \
  "s_add_u32 s73, s74, s69\n" \
  "ds_read_b128 v[134:137], v247 offset:24576\n" \
  "v_mfma_f32_16x16x32_f16 a[160:163], v[86:89], v[102:105], a[160:163]\n" \
  "s_waitcnt lgkmcnt(13)\n" \
  "ds_read_b128 v[102:105], v247 offset:16384\n" \
  "v_mfma_f32_16x16x32_f16 a[164:167], v[86:89], v[106:109], a[164:167]\n" \
  "buffer_load_dwordx4 v[30:33], %[bxr], s[40:43], s73 offen offset:0 sc0 sc1\n" \
  "v_mfma_f32_16x16x32_f16 a[164:167], v[90:93], v[138:141], a[164:167]\n" \
  "v_mfma_f32_16x16x32_f16 a[132:135], v[46:49], v[138:141], a[132:135]\n" \
  "ds_read_b128 v[138:141], v247 offset:25600\n" \
  "v_mfma_f32_16x16x32_f16 a[132:135], v[50:53], v[106:109], a[132:135]\n" \
  "s_waitcnt lgkmcnt(13)\n" \
  "ds_read_b128 v[106:109], v247 offset:17408\n" \
  "v_mfma_f32_16x16x32_f16 a[136:139], v[50:53], v[110:113], a[136:139]\n" \
  "buffer_load_dwordx4 v[34:37], %[bxr], s[40:43], s73 offen offset:1024 sc0 sc1\n" \
  "v_mfma_f32_16x16x32_f16 a[136:139], v[46:49], v[142:145], a[136:139]\n" \
  "v_mfma_f32_16x16x32_f16 a[168:171], v[90:93], v[142:145], a[168:171]\n" \
  "ds_read_b128 v[142:145], v247 offset:26624\n" \
  "v_mfma_f32_16x16x32_f16 a[168:171], v[86:89], v[110:113], a[168:171]\n" \
  "s_waitcnt lgkmcnt(13)\n" \
  "ds_read_b128 v[110:113], v247 offset:18432\n" \
  "v_mfma_f32_16x16x32_f16 a[172:175], v[86:89], v[114:117], a[172:175]\n" \
  "v_mfma_f32_16x16x32_f16 a[172:175], v[90:93], v[146:149], a[172:175]\n" \
  "v_mfma_f32_16x16x32_f16 a[140:143], v[46:49], v[146:149], a[140:143]\n" \
  "ds_read_b128 v[146:149], v247 offset:27648\n" \
  "v_mfma_f32_16x16x32_f16 a[140:143], v[50:53], v[114:117], a[140:143]\n" \
  "s_waitcnt lgkmcnt(13)\n" \
  "ds_read_b128 v[114:117], v247 offset:19456\n" \
  "v_mfma_f32_16x16x32_f16 a[144:147], v[50:53], v[118:121], a[144:147]\n" \
  "v_mfma_f32_16x16x32_f16 a[144:147], v[46:49], v[150:153], a[144:147]\n" \
  "s_nop 7\n" \
  "v_mfma_f32_16x16x32_f16 a[176:179], v[90:93], v[150:153], a[176:179]\n" \
  "ds_read_b128 v[150:153], v247 offset:28672\n" \
  "v_mfma_f32_16x16x32_f16 a[176:179], v[86:89], v[118:121], a[176:179]\n" \
  "s_waitcnt lgkmcnt(13)\n" \
  "ds_read_b128 v[118:121], v247 offset:20480\n" \
  "v_mfma_f32_16x16x32_f16 a[180:183], v[86:89], v[122:125], a[180:183]\n" \
  "v_mfma_f32_16x16x32_f16 a[180:183], v[90:93], v[154:157], a[180:183]\n" \
  "v_mfma_f32_16x16x32_f16 a[148:151], v[46:49], v[154:157], a[148:151]\n" \
  "ds_read_b128 v[154:157], v247 offset:29696\n" \
  "v_mfma_f32_16x16x32_f16 a[148:151], v[50:53], v[122:125], a[148:151]\n" \
  "s_waitcnt lgkmcnt(13)\n" \
  "ds_read_b128 v[122:125], v247 offset:21504\n" \
  "v_mfma_f32_16x16x32_f16 a[152:155], v[50:53], v[126:129], a[152:155]\n" \
  "v_mfma_f32_16x16x32_f16 a[152:155], v[46:49], v[158:161], a[152:155]\n" \
  "buffer_store_dwordx4 a[24:27], %[voffO], s[44:47], s83 offen offset:256 sc1\n" \
  "v_mfma_f32_16x16x32_f16 a[184:187], v[90:93], v[158:161], a[184:187]\n" \
  "ds_read_b128 v[158:161], v247 offset:30720\n" \
  "v_mfma_f32_16x16x32_f16 a[184:187], v[86:89], v[126:129], a[184:187]\n" \
  "s_waitcnt lgkmcnt(13)\n" \
  "ds_read_b128 v[126:129], v247 offset:22528\n" \
  "v_mfma_f32_16x16x32_f16 a[188:191], v[86:89], v[130:133], a[188:191]\n" \
  "buffer_load_dwordx4 v[86:89], %[bxr], s[40:43], s73 offen offset:2048 sc0 sc1\n" \
  "v_mfma_f32_16x16x32_f16 a[188:191], v[90:93], v[162:165], a[188:191]\n" \
  "v_readlane_b32 s74, %[vtabA], 23\n" \
  "v_mfma_f32_16x16x32_f16 a[156:159], v[46:49], v[162:165], a[156:159]\n" \
  "ds_read_b128 v[162:165], v247 offset:31744\n" \
  "v_mfma_f32_16x16x32_f16 a[156:159], v[50:53], v[130:133], a[156:159]\n" \
  "s_waitcnt vmcnt(21)\n" \
  "ds_read_b128 v[130:133], v247 offset:23552\n" \
  "s_waitcnt lgkmcnt(14)\n" \
  "v_mfma_f32_16x16x32_f16 a[128:131], v[66:69], v[102:105], a[128:131]\n" \
  "buffer_load_dwordx4 v[90:93], %[bxr], s[40:43], s73 offen offset:3072 sc0 sc1\n" \
  "v_mfma_f32_16x16x32_f16 a[128:131], v[62:65], v[134:137], a[128:131]\n" \
  "s_add_u32 s72, s74, s70\n" \
  "s_nop 7\n" \
  "v_mfma_f32_16x16x32_f16 a[160:163], v[26:29], v[134:137], a[160:163]\n" \
  "ds_read_b128 v[134:137], v245 offset:8192\n" \
  "v_mfma_f32_16x16x32_f16 a[160:163], v[22:25], v[102:105], a[160:163]\n" \
  "s_waitcnt lgkmcnt(13)\n" \
  "ds_read_b128 v[102:105], v245 offset:0\n" \
  "v_mfma_f32_16x16x32_f16 a[164:167], v[22:25], v[106:109], a[164:167]\n" \
  "buffer_load_dwordx4 v[46:49], %[bxr], s[40:43], s72 offen offset:0 sc0 sc1\n" \
  "v_mfma_f32_16x16x32_f16 a[164:167], v[26:29], v[138:141], a[164:167]\n" \
  "v_mfma_f32_16x16x32_f16 a[132:135], v[62:65], v[138:141], a[132:135]\n" \
  "ds_read_b128 v[138:141], v245 offset:9216\n" \
  "v_mfma_f32_16x16x32_f16 a[132:135], v[66:69], v[106:109], a[132:135]\n" \
  "s_waitcnt lgkmcnt(13)\n" \
  "ds_read_b128 v[106:109], v245 offset:1024\n" \
  "v_mfma_f32_16x16x32_f16 a[136:139], v[66:69], v[110:113], a[136:139]\n" \
  "buffer_load_dwordx4 v[50:53], %[bxr], s[40:43], s72 offen offset:1024 sc0 sc1\n" \
  "v_mfma_f32_16x16x32_f16 a[136:139], v[62:65], v[142:145], a[136:139]\n" \
  "s_add_u32 s84, s90, 0x30000\n" \
  "v_mfma_f32_16x16x32_f16 a[168:171], v[26:29], v[142:145], a[168:171]\n" \
  "ds_read_b128 v[142:145], v245 offset:10240\n" \
  "v_mfma_f32_16x16x32_f16 a[168:171], v[22:25], v[110:113], a[168:171]\n" \
  "s_waitcnt lgkmcnt(13)\n" \
  "ds_read_b128 v[110:113], v245 offset:2048\n" \
  "v_mfma_f32_16x16x32_f16 a[172:175], v[22:25], v[114:117], a[172:175]\n" \
  "v_mfma_f32_16x16x32_f16 a[172:175], v[26:29], v[146:149], a[172:175]\n" \
  "v_mfma_f32_16x16x32_f16 a[140:143], v[62:65], v[146:149], a[140:143]\n" \
  "ds_read_b128 v[146:149], v245 offset:11264\n" \
  "v_mfma_f32_16x16x32_f16 a[140:143], v[66:69], v[114:117], a[140:143]\n" \
  "s_waitcnt lgkmcnt(13)\n" \
  "ds_read_b128 v[114:117], v245 offset:3072\n" \
  "v_mfma_f32_16x16x32_f16 a[144:147], v[66:69], v[118:121], a[144:147]\n" \
  "v_mfma_f32_16x16x32_f16 a[144:147], v[62:65], v[150:153], a[144:147]\n" \
  "buffer_store_dwordx4 a[12:15], %[voffO], s[44:47], s84 offen offset:0 sc1\n" \
  "s_nop 7\n" \
  "v_mfma_f32_16x16x32_f16 a[176:179], v[26:29], v[150:153], a[176:179]\n" \
  "ds_read_b128 v[150:153], v245 offset:12288\n" \
  "v_mfma_f32_16x16x32_f16 a[176:179], v[22:25], v[118:121], a[176:179]\n" \
  "s_waitcnt lgkmcnt(13)\n" \
  "ds_read_b128 v[118:121], v245 offset:4096\n" \
  "v_mfma_f32_16x16x32_f16 a[180:183], v[22:25], v[122:125], a[180:183]\n" \
  "v_mfma_f32_16x16x32_f16 a[180:183], v[26:29], v[154:157], a[180:183]\n" \
  "v_mfma_f32_16x16x32_f16 a[148:151], v[62:65], v[154:157], a[148:151]\n" \
  "ds_read_b128 v[154:157], v245 offset:13312\n" \
  "v_mfma_f32_16x16x32_f16 a[148:151], v[66:69], v[122:125], a[148:151]\n" \
  "s_waitcnt lgkmcnt(13)\n" \
  "ds_read_b128 v[122:125], v245 offset:5120\n" \
  "v_mfma_f32_16x16x32_f16 a[152:155], v[66:69], v[126:129], a[152:155]\n" \
  "v_mfma_f32_16x16x32_f16 a[152:155], v[62:65], v[158:161], a[152:155]\n" \
  "v_mfma_f32_16x16x32_f16 a[184:187], v[26:29], v[158:161], a[184:187]\n" \
  "ds_read_b128 v[158:161], v245 offset:14336\n" \
  "v_mfma_f32_16x16x32_f16 a[184:187], v[22:25], v[126:129], a[184:187]\n" \
  "s_waitcnt lgkmcnt(13)\n" \
  "ds_read_b128 v[126:129], v245 offset:6144\n" \
  "v_mfma_f32_16x16x32_f16 a[188:191], v[22:25], v[130:133], a[188:191]\n" \
  "buffer_load_dwordx4 v[22:25], %[bxr], s[40:43], s72 offen offset:2048 sc0 sc1\n" \
  "v_mfma_f32_16x16x32_f16 a[188:191], v[26:29], v[162:165], a[188:191]\n" \
  "v_readlane_b32 s74, %[vtabA], 24\n" \
  "v_mfma_f32_16x16x32_f16 a[156:159], v[62:65], v[162:165], a[156:159]\n" \
  "ds_read_b128 v[162:165], v245 offset:15360\n" \
  "v_mfma_f32_16x16x32_f16 a[156:159], v[66:69], v[130:133], a[156:159]\n" \
  "s_waitcnt vmcnt(20)\n" \
  "ds_read_b128 v[130:133], v245 offset:7168\n" \
  "s_waitcnt lgkmcnt(14)\n" \
  "v_mfma_f32_16x16x32_f16 a[128:131], v[82:85], v[102:105], a[128:131]\n" \
  "buffer_load_dwordx4 v[26:29], %[bxr], s[40:43], s72 offen offset:3072 sc0 sc1\n" \
  "v_mfma_f32_16x16x32_f16 a[128:131], v[78:81], v[134:137], a[128:131]\n" \
  "s_add_u32 s73, s74, s70\n" \
  "s_nop 7\n" \
  "v_mfma_f32_16x16x32_f16 a[160:163], v[42:45], v[134:137], a[160:163]\n" \
  "ds_read_b128 v[134:137], v245 offset:24576\n" \
  "v_mfma_f32_16x16x32_f16 a[160:163], v[38:41], v[102:105], a[160:163]\n" \
  "s_waitcnt lgkmcnt(13)\n" \
  "ds_read_b128 v[102:105], v245 offset:16384\n" \
  "v_mfma_f32_16x16x32_f16 a[164:167], v[38:41], v[106:109], a[164:167]\n" \
  "buffer_load_dwordx4 v[62:65], %[bxr], s[40:43], s73 offen offset:0 sc0 sc1\n" \
  "v_mfma_f32_16x16x32_f16 a[164:167], v[42:45], v[138:141], a[164:167]\n" \
  "v_mfma_f32_16x16x32_f16 a[132:135], v[78:81], v[138:141], a[132:135]\n" \
  "ds_read_b128 v[138:141], v245 offset:25600\n" \
  "v_mfma_f32_16x16x32_f16 a[132:135], v[82:85], v[106:109], a[132:135]\n" \
  "s_waitcnt lgkmcnt(13)\n" \
  "ds_read_b128 v[106:109], v245 offset:17408\n" \
  "v_mfma_f32_16x16x32_f16 a[136:139], v[82:85], v[110:113], a[136:139]\n" \
  "buffer_load_dwordx4 v[66:69], %[bxr], s[40:43], s73 offen offset:1024 sc0 sc1\n" \
  "v_mfma_f32_16x16x32_f16 a[136:139], v[78:81], v[142:145], a[136:139]\n" \
  "buffer_store_dwordx4 a[28:31], %[voffO], s[44:47], s84 offen offset:256 sc1\n" \
  "v_mfma_f32_16x16x32_f16 a[168:171], v[42:45], v[142:145], a[168:171]\n" \
  "ds_read_b128 v[142:145], v245 offset:26624\n" \
  "v_mfma_f32_16x16x32_f16 a[168:171], v[38:41], v[110:113], a[168:171]\n" \
  "s_waitcnt lgkmcnt(13)\n" \
  "ds_read_b128 v[110:113], v245 offset:18432\n" \
  "v_mfma_f32_16x16x32_f16 a[172:175], v[38:41], v[114:117], a[172:175]\n" \
  "v_mfma_f32_16x16x32_f16 a[172:175], v[42:45], v[146:149], a[172:175]\n" \
  "v_mfma_f32_16x16x32_f16 a[140:143], v[78:81], v[146:149], a[140:143]\n" \
  "ds_read_b128 v[146:149], v245 offset:27648\n" \
  "v_mfma_f32_16x16x32_f16 a[140:143], v[82:85], v[114:117], a[140:143]\n" \
  "s_waitcnt lgkmcnt(13)\n" \
  "ds_read_b128 v[114:117], v245 offset:19456\n" \
  "v_mfma_f32_16x16x32_f16 a[144:147], v[82:85], v[118:121], a[144:147]\n" \
  "v_mfma_f32_16x16x32_f16 a[144:147], v[78:81], v[150:153], a[144:147]\n" \
  "s_nop 7\n" \
  "v_mfma_f32_16x16x32_f16 a[176:179], v[42:45], v[150:153], a[176:179]\n" \
  "ds_read_b128 v[150:153], v245 offset:28672\n" \
  "v_mfma_f32_16x16x32_f16 a[176:179], v[38:41], v[118:121], a[176:179]\n" \
  "s_waitcnt lgkmcnt(13)\n" \
  "ds_read_b128 v[118:121], v245 offset:20480\n" \
  "v_mfma_f32_16x16x32_f16 a[180:183], v[38:41], v[122:125], a[180:183]\n" \
  "v_mfma_f32_16x16x32_f16 a[180:183], v[42:45], v[154:157], a[180:183]\n" \
  "v_mfma_f32_16x16x32_f16 a[148:151], v[78:81], v[154:157], a[148:151]\n" \
  "ds_read_b128 v[154:157], v245 offset:29696\n" \
  "v_mfma_f32_16x16x32_f16 a[148:151], v[82:85], v[122:125], a[148:151]\n" \
  "s_waitcnt lgkmcnt(13)\n" \
  "ds_read_b128 v[122:125], v245 offset:21504\n" \
  "v_mfma_f32_16x16x32_f16 a[152:155], v[82:85], v[126:129], a[152:155]\n" \
  "v_mfma_f32_16x16x32_f16 a[152:155], v[78:81], v[158:161], a[152:155]\n" \
  "s_add_u32 s83, s90, 0x100000\n" \
  "v_mfma_f32_16x16x32_f16 a[184:187], v[42:45], v[158:161], a[184:187]\n" \
  "ds_read_b128 v[158:161], v245 offset:30720\n" \
  "v_mfma_f32_16x16x32_f16 a[184:187], v[38:41], v[126:129], a[184:187]\n" \
  "s_waitcnt lgkmcnt(13)\n" \
  "ds_read_b128 v[126:129], v245 offset:22528\n" \
  "v_mfma_f32_16x16x32_f16 a[188:191], v[38:41], v[130:133], a[188:191]\n" \
  "buffer_load_dwordx4 v[38:41], %[bxr], s[40:43], s73 offen offset:2048 sc0 sc1\n" \
  "v_mfma_f32_16x16x32_f16 a[188:191], v[42:45], v[162:165], a[188:191]\n" \
  "v_readlane_b32 s74, %[vtabA], 25\n" \
  "v_mfma_f32_16x16x32_f16 a[156:159], v[78:81], v[162:165], a[156:159]\n" \
  "ds_read_b128 v[162:165], v245 offset:31744\n" \
  "v_mfma_f32_16x16x32_f16 a[156:159], v[82:85], v[130:133], a[156:159]\n" \
  "s_waitcnt vmcnt(20)\n" \
  "ds_read_b128 v[130:133], v245 offset:23552\n" \
  "s_waitcnt lgkmcnt(14)\n" \
  "v_mfma_f32_16x16x32_f16 a[128:131], v[98:101], v[102:105], a[128:131]\n" \
  "buffer_load_dwordx4 v[42:45], %[bxr], s[40:43], s73 offen offset:3072 sc0 sc1\n" \
  "v_mfma_f32_16x16x32_f16 a[128:131], v[94:97], v[134:137], a[128:131]\n" \
  "buffer_store_dwordx4 a[32:35], %[voffO], s[44:47], s83 offen offset:0 sc1\n" \
  "s_nop 7\n" \
  "v_mfma_f32_16x16x32_f16 a[160:163], v[58:61], v[134:137], a[160:163]\n" \
  "s_add_u32 s72, s74, s70\n" \
  "ds_read_b128 v[134:137], v245 offset:40960\n" \
  "v_mfma_f32_16x16x32_f16 a[160:163], v[54:57], v[102:105], a[160:163]\n" \
  "s_waitcnt lgkmcnt(13)\n" \
  "ds_read_b128 v[102:105], v245 offset:32768\n" \
  "v_mfma_f32_16x16x32_f16 a[164:167], v[54:57], v[106:109], a[164:167]\n" \
  "buffer_load_dwordx4 v[78:81], %[bxr], s[40:43], s72 offen offset:0 sc0 sc1\n" \
  "v_mfma_f32_16x16x32_f16 a[164:167], v[58:61], v[138:141], a[164:167]\n" \
  "v_mfma_f32_16x16x32_f16 a[132:135], v[94:97], v[138:141], a[132:135]\n" \
  "ds_read_b128 v[138:141], v245 offset:41984\n" \
  "v_mfma_f32_16x16x32_f16 a[132:135], v[98:101], v[106:109], a[132:135]\n" \
  "s_waitcnt lgkmcnt(13)\n" \
  "ds_read_b128 v[106:109], v245 offset:33792\n" \
  "v_mfma_f32_16x16x32_f16 a[136:139], v[98:101], v[110:113], a[136:139]\n" \
  "buffer_load_dwordx4 v[82:85], %[bxr], s[40:43], s72 offen offset:1024 sc0 sc1\n" \
  "v_mfma_f32_16x16x32_f16 a[136:139], v[94:97], v[142:145], a[136:139]\n" \
  "v_mfma_f32_16x16x32_f16 a[168:171], v[58:61], v[142:145], a[168:171]\n" \
  "ds_read_b128 v[142:145], v245 offset:43008\n" \
  "v_mfma_f32_16x16x32_f16 a[168:171], v[54:57], v[110:113], a[168:171]\n" \
  "s_waitcnt lgkmcnt(13)\n" \
  "ds_read_b128 v[110:113], v245 offset:34816\n" \
  "v_mfma_f32_16x16x32_f16 a[172:175], v[54:57], v[114:117], a[172:175]\n" \
  "v_mfma_f32_16x16x32_f16 a[172:175], v[58:61], v[146:149], a[172:175]\n" \
  "v_mfma_f32_16x16x32_f16 a[140:143], v[94:97], v[146:149], a[140:143]\n" \
  "ds_read_b128 v[146:149], v245 offset:44032\n" \
  "v_mfma_f32_16x16x32_f16 a[140:143], v[98:101], v[114:117], a[140:143]\n" \
  "s_waitcnt lgkmcnt(13)\n" \
  "ds_read_b128 v[114:117], v245 offset:35840\n" \
  "v_mfma_f32_16x16x32_f16 a[144:147], v[98:101], v[118:121], a[144:147]\n" \
  "v_mfma_f32_16x16x32_f16 a[144:147], v[94:97], v[150:153], a[144:147]\n" \
  "s_nop 7\n" \
  "v_mfma_f32_16x16x32_f16 a[176:179], v[58:61], v[150:153], a[176:179]\n" \
  "ds_read_b128 v[150:153], v245 offset:45056\n" \
  "v_mfma_f32_16x16x32_f16 a[176:179], v[54:57], v[118:121], a[176:179]\n" \
  "s_waitcnt lgkmcnt(13)\n" \
  "ds_read_b128 v[118:121], v245 offset:36864\n" \
  "v_mfma_f32_16x16x32_f16 a[180:183], v[54:57], v[122:125], a[180:183]\n" \
  "v_mfma_f32_16x16x32_f16 a[180:183], v[58:61], v[154:157], a[180:183]\n" \
  "v_mfma_f32_16x16x32_f16 a[148:151], v[94:97], v[154:157], a[148:151]\n" \
  "ds_read_b128 v[154:157], v245 offset:46080\n" \
  "v_mfma_f32_16x16x32_f16 a[148:151], v[98:101], v[122:125], a[148:151]\n" \
  "s_waitcnt lgkmcnt(13)\n" \
  "ds_read_b128 v[122:125], v245 offset:37888\n" \
  "v_mfma_f32_16x16x32_f16 a[152:155], v[98:101], v[126:129], a[152:155]\n" \
  "v_mfma_f32_16x16x32_f16 a[152:155], v[94:97], v[158:161], a[152:155]\n" \
  "buffer_store_dwordx4 a[48:51], %[voffO], s[44:47], s83 offen offset:256 sc1\n" \
  "v_mfma_f32_16x16x32_f16 a[184:187], v[58:61], v[158:161], a[184:187]\n" \
  "ds_read_b128 v[158:161], v245 offset:47104\n" \
  "v_mfma_f32_16x16x32_f16 a[184:187], v[54:57], v[126:129], a[184:187]\n" \
  "s_waitcnt lgkmcnt(13)\n" \
  "ds_read_b128 v[126:129], v245 offset:38912\n" \
  "v_mfma_f32_16x16x32_f16 a[188:191], v[54:57], v[130:133], a[188:191]\n" \
  "buffer_load_dwordx4 v[54:57], %[bxr], s[40:43], s72 offen offset:2048 sc0 sc1\n" \
  "v_mfma_f32_16x16x32_f16 a[188:191], v[58:61], v[162:165], a[188:191]\n" \
  "v_readlane_b32 s74, %[vtabA], 26\n" \
  "v_mfma_f32_16x16x32_f16 a[156:159], v[94:97], v[162:165], a[156:159]\n" \
  "ds_read_b128 v[162:165], v245 offset:48128\n" \
  "v_mfma_f32_16x16x32_f16 a[156:159], v[98:101], v[130:133], a[156:159]\n" \
  "s_waitcnt vmcnt(21)\n" \
  "ds_read_b128 v[130:133], v245 offset:39936\n" \
  "s_waitcnt lgkmcnt(14)\n" \
  "v_mfma_f32_16x16x32_f16 a[128:131], v[14:17], v[102:105], a[128:131]\n" \
  "buffer_load_dwordx4 v[58:61], %[bxr], s[40:43], s72 offen offset:3072 sc0 sc1\n" \
  "v_mfma_f32_16x16x32_f16 a[128:131], v[18:21], v[134:137], a[128:131]\n" \
  "s_add_u32 s73, s74, s70\n" \
  "s_nop 7\n" \
  "v_mfma_f32_16x16x32_f16 a[160:163], v[74:77], v[134:137], a[160:163]\n" \
  "ds_read_b128 v[134:137], v245 offset:57344\n" \
  "v_mfma_f32_16x16x32_f16 a[160:163], v[70:73], v[102:105], a[160:163]\n" \
  "s_waitcnt lgkmcnt(13)\n" \
  "ds_read_b128 v[102:105], v245 offset:49152\n" \
  "v_mfma_f32_16x16x32_f16 a[164:167], v[70:73], v[106:109], a[164:167]\n" \
  "buffer_load_dwordx4 v[94:97], %[bxr], s[40:43], s73 offen offset:0 sc0 sc1\n" \
  "v_mfma_f32_16x16x32_f16 a[164:167], v[74:77], v[138:141], a[164:167]\n" \
  "v_mfma_f32_16x16x32_f16 a[132:135], v[18:21], v[138:141], a[132:135]\n" \
  "ds_read_b128 v[138:141], v245 offset:58368\n" \
  "v_mfma_f32_16x16x32_f16 a[132:135], v[14:17], v[106:109], a[132:135]\n" \
  "s_waitcnt lgkmcnt(13)\n" \
  "ds_read_b128 v[106:109], v245 offset:50176\n" \
  "v_mfma_f32_16x16x32_f16 a[136:139], v[14:17], v[110:113], a[136:139]\n" \
  "buffer_load_dwordx4 v[98:101], %[bxr], s[40:43], s73 offen offset:1024 sc0 sc1\n" \
  "v_mfma_f32_16x16x32_f16 a[136:139], v[18:21], v[142:145], a[136:139]\n" \
  "s_add_u32 s84, s90, 0x110000\n" \
  "v_mfma_f32_16x16x32_f16 a[168:171], v[74:77], v[142:145], a[168:171]\n" \
  "ds_read_b128 v[142:145], v245 offset:59392\n" \
  "v_mfma_f32_16x16x32_f16 a[168:171], v[70:73], v[110:113], a[168:171]\n" \
  "s_waitcnt lgkmcnt(13)\n" \
  "ds_read_b128 v[110:113], v245 offset:51200\n" \
  "v_mfma_f32_16x16x32_f16 a[172:175], v[70:73], v[114:117], a[172:175]\n" \
  "v_mfma_f32_16x16x32_f16 a[172:175], v[74:77], v[146:149], a[172:175]\n" \
  "v_mfma_f32_16x16x32_f16 a[140:143], v[18:21], v[146:149], a[140:143]\n" \
  "ds_read_b128 v[146:149], v245 offset:60416\n" \
  "v_mfma_f32_16x16x32_f16 a[140:143], v[14:17], v[114:117], a[140:143]\n" \
  "s_waitcnt lgkmcnt(13)\n" \
  "ds_read_b128 v[114:117], v245 offset:52224\n" \
  "v_mfma_f32_16x16x32_f16 a[144:147], v[14:17], v[118:121], a[144:147]\n" \
  "v_mfma_f32_16x16x32_f16 a[144:147], v[18:21], v[150:153], a[144:147]\n" \
  "buffer_store_dwordx4 a[36:39], %[voffO], s[44:47], s84 offen offset:0 sc1\n" \
  "s_nop 7\n" \
  "v_mfma_f32_16x16x32_f16 a[176:179], v[74:77], v[150:153], a[176:179]\n" \
  "ds_read_b128 v[150:153], v245 offset:61440\n" \
  "v_mfma_f32_16x16x32_f16 a[176:179], v[70:73], v[118:121], a[176:179]\n" \
  "s_waitcnt lgkmcnt(13)\n" \
  "ds_read_b128 v[118:121], v245 offset:53248\n" \
  "v_mfma_f32_16x16x32_f16 a[180:183], v[70:73], v[122:125], a[180:183]\n" \
  "v_mfma_f32_16x16x32_f16 a[180:183], v[74:77], v[154:157], a[180:183]\n" \
  "v_mfma_f32_16x16x32_f16 a[148:151], v[18:21], v[154:157], a[148:151]\n" \
  "ds_read_b128 v[154:157], v245 offset:62464\n" \
  "v_mfma_f32_16x16x32_f16 a[148:151], v[14:17], v[122:125], a[148:151]\n" \
  "s_waitcnt lgkmcnt(13)\n" \
  "ds_read_b128 v[122:125], v245 offset:54272\n" \
  "v_mfma_f32_16x16x32_f16 a[152:155], v[14:17], v[126:129], a[152:155]\n" \
  "v_mfma_f32_16x16x32_f16 a[152:155], v[18:21], v[158:161], a[152:155]\n" \
  "v_mfma_f32_16x16x32_f16 a[184:187], v[74:77], v[158:161], a[184:187]\n" \
  "ds_read_b128 v[158:161], v245 offset:63488\n" \
  "v_mfma_f32_16x16x32_f16 a[184:187], v[70:73], v[126:129], a[184:187]\n" \
  "s_waitcnt lgkmcnt(13)\n" \
  "ds_read_b128 v[126:129], v245 offset:55296\n" \
  "v_mfma_f32_16x16x32_f16 a[188:191], v[70:73], v[130:133], a[188:191]\n" \
  "buffer_load_dwordx4 v[70:73], %[bxr], s[40:43], s73 offen offset:2048 sc0 sc1\n" \
  "v_mfma_f32_16x16x32_f16 a[188:191], v[74:77], v[162:165], a[188:191]\n" \
  "v_readlane_b32 s74, %[vtabA], 27\n" \
  "v_mfma_f32_16x16x32_f16 a[156:159], v[18:21], v[162:165], a[156:159]\n" \
  "ds_read_b128 v[162:165], v245 offset:64512\n" \
  "v_mfma_f32_16x16x32_f16 a[156:159], v[14:17], v[130:133], a[156:159]\n" \
  "s_waitcnt vmcnt(20)\n" \
  "ds_read_b128 v[130:133], v245 offset:56320\n" \
  "s_waitcnt lgkmcnt(14)\n" \
  "v_mfma_f32_16x16x32_f16 a[128:131], v[30:33], v[102:105], a[128:131]\n" \
  "buffer_load_dwordx4 v[74:77], %[bxr], s[40:43], s73 offen offset:3072 sc0 sc1\n" \
  "v_mfma_f32_16x16x32_f16 a[128:131], v[34:37], v[134:137], a[128:131]\n" \
  "s_add_u32 s72, s74, s70\n" \
  "s_nop 7\n" \
  "v_mfma_f32_16x16x32_f16 a[160:163], v[90:93], v[134:137], a[160:163]\n" \
  "ds_read_b128 v[134:137], v246 offset:24576\n" \
  "v_mfma_f32_16x16x32_f16 a[160:163], v[86:89], v[102:105], a[160:163]\n" \
  "s_waitcnt lgkmcnt(13)\n" \
  "ds_read_b128 v[102:105], v246 offset:16384\n" \
  "v_mfma_f32_16x16x32_f16 a[164:167], v[86:89], v[106:109], a[164:167]\n" \
  "buffer_load_dwordx4 v[18:21], %[bxr], s[40:43], s72 offen offset:0 sc0 sc1\n" \
  "v_mfma_f32_16x16x32_f16 a[164:167], v[90:93], v[138:141], a[164:167]\n" \
  "v_mfma_f32_16x16x32_f16 a[132:135], v[34:37], v[138:141], a[132:135]\n" \
  "ds_read_b128 v[138:141], v246 offset:25600\n" \
  "v_mfma_f32_16x16x32_f16 a[132:135], v[30:33], v[106:109], a[132:135]\n" \
  "s_waitcnt lgkmcnt(13)\n" \
  "ds_read_b128 v[106:109], v246 offset:17408\n" \
  "v_mfma_f32_16x16x32_f16 a[136:139], v[30:33], v[110:113], a[136:139]\n" \
  "buffer_load_dwordx4 v[14:17], %[bxr], s[40:43], s72 offen offset:1024 sc0 sc1\n" \
  "v_mfma_f32_16x16x32_f16 a[136:139], v[34:37], v[142:145], a[136:139]\n" \
  "buffer_store_dwordx4 a[52:55], %[voffO], s[44:47], s84 offen offset:256 sc1\n" \
  "v_mfma_f32_16x16x32_f16 a[168:171], v[90:93], v[142:145], a[168:171]\n" \
  "ds_read_b128 v[142:145], v246 offset:26624\n" \
  "v_mfma_f32_16x16x32_f16 a[168:171], v[86:89], v[110:113], a[168:171]\n" \
  "s_waitcnt lgkmcnt(13)\n" \
  "ds_read_b128 v[110:113], v246 offset:18432\n" \
  "v_mfma_f32_16x16x32_f16 a[172:175], v[86:89], v[114:117], a[172:175]\n" \
  "v_mfma_f32_16x16x32_f16 a[172:175], v[90:93], v[146:149], a[172:175]\n" \
  "s_waitcnt vmcnt(16)\n" \
  "v_mov_b32 v250, s76\n" \
  "s_nop 0\n" \
  "global_store_dword v249, v250, %[xfo] offset:0 sc1\n" \
  "v_mfma_f32_16x16x32_f16 a[140:143], v[34:37], v[146:149], a[140:143]\n" \
  "ds_read_b128 v[146:149], v246 offset:27648\n" \
  "v_mfma_f32_16x16x32_f16 a[140:143], v[30:33], v[114:117], a[140:143]\n" \
  "s_waitcnt lgkmcnt(13)\n" \
  "ds_read_b128 v[114:117], v246 offset:19456\n" \
  "v_mfma_f32_16x16x32_f16 a[144:147], v[30:33], v[118:121], a[144:147]\n" \
  "v_mfma_f32_16x16x32_f16 a[144:147], v[34:37], v[150:153], a[144:147]\n" \
  "s_nop 7\n" \
  "v_mfma_f32_16x16x32_f16 a[176:179], v[90:93], v[150:153], a[176:179]\n" \
  "ds_read_b128 v[150:153], v246 offset:28672\n" \
  "v_mfma_f32_16x16x32_f16 a[176:179], v[86:89], v[118:121], a[176:179]\n" \
  "s_waitcnt lgkmcnt(13)\n" \
  "ds_read_b128 v[118:121], v246 offset:20480\n" \
  "v_mfma_f32_16x16x32_f16 a[180:183], v[86:89], v[122:125], a[180:183]\n" \
  "v_mfma_f32_16x16x32_f16 a[180:183], v[90:93], v[154:157], a[180:183]\n" \
  "v_mfma_f32_16x16x32_f16 a[148:151], v[34:37], v[154:157], a[148:151]\n" \
  "ds_read_b128 v[154:157], v246 offset:29696\n" \
  "v_mfma_f32_16x16x32_f16 a[148:151], v[30:33], v[122:125], a[148:151]\n" \
  "s_waitcnt lgkmcnt(13)\n" \
  "ds_read_b128 v[122:125], v246 offset:21504\n" \
  "v_mfma_f32_16x16x32_f16 a[152:155], v[30:33], v[126:129], a[152:155]\n" \
  "v_mfma_f32_16x16x32_f16 a[152:155], v[34:37], v[158:161], a[152:155]\n" \
  "s_add_u32 s83, s90, 0x120000\n" \
  "v_mfma_f32_16x16x32_f16 a[184:187], v[90:93], v[158:161], a[184:187]\n" \
  "ds_read_b128 v[158:161], v246 offset:30720\n" \
  "v_mfma_f32_16x16x32_f16 a[184:187], v[86:89], v[126:129], a[184:187]\n" \
  "s_waitcnt lgkmcnt(13)\n" \
  "ds_read_b128 v[126:129], v246 offset:22528\n" \
  "v_mfma_f32_16x16x32_f16 a[188:191], v[86:89], v[130:133], a[188:191]\n" \
  "buffer_load_dwordx4 v[86:89], %[bxr], s[40:43], s72 offen offset:2048 sc0 sc1\n" \
  "v_mfma_f32_16x16x32_f16 a[188:191], v[90:93], v[162:165], a[188:191]\n" \
  "v_readlane_b32 s74, %[vtabA], 28\n" \
  "v_mfma_f32_16x16x32_f16 a[156:159], v[34:37], v[162:165], a[156:159]\n" \
  "ds_read_b128 v[162:165], v246 offset:31744\n" \
  "v_mfma_f32_16x16x32_f16 a[156:159], v[30:33], v[130:133], a[156:159]\n" \
  "ds_read_b128 v[130:133], v246 offset:23552\n" \
  "s_waitcnt lgkmcnt(14)\n" \
  "v_mfma_f32_16x16x32_f16 a[64:67], v[46:49], v[102:105], a[64:67]\n" \
  "buffer_load_dwordx4 v[90:93], %[bxr], s[40:43], s72 offen offset:3072 sc0 sc1\n" \
  "v_mfma_f32_16x16x32_f16 a[64:67], v[50:53], v[134:137], a[64:67]\n" \
  "buffer_store_dwordx4 a[40:43], %[voffO], s[44:47], s83 offen offset:0 sc1\n" \
  "s_nop 7\n" \
  "v_mfma_f32_16x16x32_f16 a[96:99], v[26:29], v[134:137], a[96:99]\n" \
  "s_add_u32 s73, s74, s70\n" \
  "ds_read_b128 v[134:137], v246 offset:40960\n" \
  "v_mfma_f32_16x16x32_f16 a[96:99], v[22:25], v[102:105], a[96:99]\n" \
  "s_waitcnt lgkmcnt(13)\n" \
  "ds_read_b128 v[102:105], v246 offset:32768\n" \
  "v_mfma_f32_16x16x32_f16 a[100:103], v[22:25], v[106:109], a[100:103]\n" \
  "buffer_load_dwordx4 v[34:37], %[bxr], s[40:43], s73 offen offset:0 sc0 sc1\n" \
  "v_mfma_f32_16x16x32_f16 a[100:103], v[26:29], v[138:141], a[100:103]\n" \
  "v_mfma_f32_16x16x32_f16 a[68:71], v[50:53], v[138:141], a[68:71]\n" \
  "ds_read_b128 v[138:141], v246 offset:41984\n" \
  "v_mfma_f32_16x16x32_f16 a[68:71], v[46:49], v[106:109], a[68:71]\n" \
  "s_waitcnt lgkmcnt(13)\n" \
  "ds_read_b128 v[106:109], v246 offset:33792\n" \
  "v_mfma_f32_16x16x32_f16 a[72:75], v[46:49], v[110:113], a[72:75]\n" \
  "buffer_load_dwordx4 v[30:33], %[bxr], s[40:43], s73 offen offset:1024 sc0 sc1\n" \
  "v_mfma_f32_16x16x32_f16 a[72:75], v[50:53], v[142:145], a[72:75]\n" \
  "v_mfma_f32_16x16x32_f16 a[104:107], v[26:29], v[142:145], a[104:107]\n" \
  "ds_read_b128 v[142:145], v246 offset:43008\n" \
  "v_mfma_f32_16x16x32_f16 a[104:107], v[22:25], v[110:113], a[104:107]\n" \
  "s_waitcnt lgkmcnt(13)\n" \
  "ds_read_b128 v[110:113], v246 offset:34816\n" \
  "v_mfma_f32_16x16x32_f16 a[108:111], v[22:25], v[114:117], a[108:111]\n" \
  "v_mfma_f32_16x16x32_f16 a[108:111], v[26:29], v[146:149], a[108:111]\n" \
  "v_mfma_f32_16x16x32_f16 a[76:79], v[50:53], v[146:149], a[76:79]\n" \
  "ds_read_b128 v[146:149], v246 offset:44032\n" \
  "v_mfma_f32_16x16x32_f16 a[76:79], v[46:49], v[114:117], a[76:79]\n" \
  "s_waitcnt lgkmcnt(13)\n" \
  "ds_read_b128 v[114:117], v246 offset:35840\n" \
  "v_mfma_f32_16x16x32_f16 a[80:83], v[46:49], v[118:121], a[80:83]\n" \
  "v_mfma_f32_16x16x32_f16 a[80:83], v[50:53], v[150:153], a[80:83]\n" \
  "s_nop 7\n" \
  "v_mfma_f32_16x16x32_f16 a[112:115], v[26:29], v[150:153], a[112:115]\n" \
  "ds_read_b128 v[150:153], v246 offset:45056\n" \
  "v_mfma_f32_16x16x32_f16 a[112:115], v[22:25], v[118:121], a[112:115]\n" \
  "s_waitcnt lgkmcnt(13)\n" \
  "ds_read_b128 v[118:121], v246 offset:36864\n" \
  "v_mfma_f32_16x16x32_f16 a[116:119], v[22:25], v[122:125], a[116:119]\n" \
  "v_mfma_f32_16x16x32_f16 a[116:119], v[26:29], v[154:157], a[116:119]\n" \
  "v_mfma_f32_16x16x32_f16 a[84:87], v[50:53], v[154:157], a[84:87]\n" \
  "ds_read_b128 v[154:157], v246 offset:46080\n" \
  "v_mfma_f32_16x16x32_f16 a[84:87], v[46:49], v[122:125], a[84:87]\n" \
  "s_waitcnt lgkmcnt(13)\n" \
  "ds_read_b128 v[122:125], v246 offset:37888\n" \
  "v_mfma_f32_16x16x32_f16 a[88:91], v[46:49], v[126:129], a[88:91]\n" \
  "v_mfma_f32_16x16x32_f16 a[88:91], v[50:53], v[158:161], a[88:91]\n" \
  "buffer_store_dwordx4 a[56:59], %[voffO], s[44:47], s83 offen offset:256 sc1\n" \
  "v_mfma_f32_16x16x32_f16 a[120:123], v[26:29], v[158:161], a[120:123]\n" \
  "ds_read_b128 v[158:161], v246 offset:47104\n" \
  "v_mfma_f32_16x16x32_f16 a[120:123], v[22:25], v[126:129], a[120:123]\n" \
  "s_waitcnt lgkmcnt(13)\n" \
  "ds_read_b128 v[126:129], v246 offset:38912\n" \
  "v_mfma_f32_16x16x32_f16 a[124:127], v[22:25], v[130:133], a[124:127]\n" \
  "buffer_load_dwordx4 v[22:25], %[bxr], s[40:43], s73 offen offset:2048 sc0 sc1\n" \
  "v_mfma_f32_16x16x32_f16 a[124:127], v[26:29], v[162:165], a[124:127]\n" \
  "v_readlane_b32 s74, %[vtabA], 29\n" \
  "v_mfma_f32_16x16x32_f16 a[92:95], v[50:53], v[162:165], a[92:95]\n" \
  "ds_read_b128 v[162:165], v246 offset:48128\n" \
  "v_mfma_f32_16x16x32_f16 a[92:95], v[46:49], v[130:133], a[92:95]\n" \
  "s_waitcnt vmcnt(21)\n" \
  "ds_read_b128 v[130:133], v246 offset:39936\n" \
  "s_waitcnt lgkmcnt(14)\n" \
  "v_mfma_f32_16x16x32_f16 a[64:67], v[62:65], v[102:105], a[64:67]\n" \
  "buffer_load_dwordx4 v[26:29], %[bxr], s[40:43], s73 offen offset:3072 sc0 sc1\n" \
  "v_mfma_f32_16x16x32_f16 a[64:67], v[66:69], v[134:137], a[64:67]\n" \
  "s_add_u32 s72, s74, s70\n" \
  "s_nop 7\n" \
  "v_mfma_f32_16x16x32_f16 a[96:99], v[42:45], v[134:137], a[96:99]\n" \
  "ds_read_b128 v[134:137], v246 offset:57344\n" \
  "v_mfma_f32_16x16x32_f16 a[96:99], v[38:41], v[102:105], a[96:99]\n" \
  "s_waitcnt lgkmcnt(13)\n" \
  "ds_read_b128 v[102:105], v246 offset:49152\n" \
  "v_mfma_f32_16x16x32_f16 a[100:103], v[38:41], v[106:109], a[100:103]\n" \
  "buffer_load_dwordx4 v[50:53], %[bxr], s[40:43], s72 offen offset:0 sc0 sc1\n" \
  "v_mfma_f32_16x16x32_f16 a[100:103], v[42:45], v[138:141], a[100:103]\n" \
  "v_mfma_f32_16x16x32_f16 a[68:71], v[66:69], v[138:141], a[68:71]\n" \
  "ds_read_b128 v[138:141], v246 offset:58368\n" \
  "v_mfma_f32_16x16x32_f16 a[68:71], v[62:65], v[106:109], a[68:71]\n" \
  "s_waitcnt lgkmcnt(13)\n" \
  "ds_read_b128 v[106:109], v246 offset:50176\n" \
  "v_mfma_f32_16x16x32_f16 a[72:75], v[62:65], v[110:113], a[72:75]\n" \
  "buffer_load_dwordx4 v[46:49], %[bxr], s[40:43], s72 offen offset:1024 sc0 sc1\n" \
  "v_mfma_f32_16x16x32_f16 a[72:75], v[66:69], v[142:145], a[72:75]\n" \
  "s_add_u32 s84, s90, 0x130000\n" \
  "v_mfma_f32_16x16x32_f16 a[104:107], v[42:45], v[142:145], a[104:107]\n" \
  "ds_read_b128 v[142:145], v246 offset:59392\n" \
  "v_mfma_f32_16x16x32_f16 a[104:107], v[38:41], v[110:113], a[104:107]\n" \
  "s_waitcnt lgkmcnt(13)\n" \
  "ds_read_b128 v[110:113], v246 offset:51200\n" \
  "v_mfma_f32_16x16x32_f16 a[108:111], v[38:41], v[114:117], a[108:111]\n" \
  "v_mfma_f32_16x16x32_f16 a[108:111], v[42:45], v[146:149], a[108:111]\n" \
  "v_mfma_f32_16x16x32_f16 a[76:79], v[66:69], v[146:149], a[76:79]\n" \
  "ds_read_b128 v[146:149], v246 offset:60416\n" \
  "v_mfma_f32_16x16x32_f16 a[76:79], v[62:65], v[114:117], a[76:79]\n" \
  "s_waitcnt lgkmcnt(13)\n" \
  "ds_read_b128 v[114:117], v246 offset:52224\n" \
  "v_mfma_f32_16x16x32_f16 a[80:83], v[62:65], v[118:121], a[80:83]\n" \
  "v_mfma_f32_16x16x32_f16 a[80:83], v[66:69], v[150:153], a[80:83]\n" \
  "buffer_store_dwordx4 a[44:47], %[voffO], s[44:47], s84 offen offset:0 sc1\n" \
  "s_nop 7\n" \
  "v_mfma_f32_16x16x32_f16 a[112:115], v[42:45], v[150:153], a[112:115]\n" \
  "ds_read_b128 v[150:153], v246 offset:61440\n" \
  "v_mfma_f32_16x16x32_f16 a[112:115], v[38:41], v[118:121], a[112:115]\n" \
  "s_waitcnt lgkmcnt(13)\n" \
  "ds_read_b128 v[118:121], v246 offset:53248\n" \
  "v_mfma_f32_16x16x32_f16 a[116:119], v[38:41], v[122:125], a[116:119]\n" \
  "v_mfma_f32_16x16x32_f16 a[116:119], v[42:45], v[154:157], a[116:119]\n" \
  "v_mfma_f32_16x16x32_f16 a[84:87], v[66:69], v[154:157], a[84:87]\n" \
  "ds_read_b128 v[154:157], v246 offset:62464\n" \
  "v_mfma_f32_16x16x32_f16 a[84:87], v[62:65], v[122:125], a[84:87]\n" \
  "s_waitcnt lgkmcnt(13)\n" \
  "ds_read_b128 v[122:125], v246 offset:54272\n" \
  "v_mfma_f32_16x16x32_f16 a[88:91], v[62:65], v[126:129], a[88:91]\n" \
  "v_mfma_f32_16x16x32_f16 a[88:91], v[66:69], v[158:161], a[88:91]\n" \
  "v_mfma_f32_16x16x32_f16 a[120:123], v[42:45], v[158:161], a[120:123]\n" \
  "ds_read_b128 v[158:161], v246 offset:63488\n" \
  "v_mfma_f32_16x16x32_f16 a[120:123], v[38:41], v[126:129], a[120:123]\n" \
  "s_waitcnt lgkmcnt(13)\n" \
  "ds_read_b128 v[126:129], v246 offset:55296\n" \
  "v_mfma_f32_16x16x32_f16 a[124:127], v[38:41], v[130:133], a[124:127]\n" \
  "buffer_load_dwordx4 v[38:41], %[bxr], s[40:43], s72 offen offset:2048 sc0 sc1\n" \
  "v_mfma_f32_16x16x32_f16 a[124:127], v[42:45], v[162:165], a[124:127]\n" \
  "v_readlane_b32 s74, %[vtabA], 30\n" \
  "v_mfma_f32_16x16x32_f16 a[92:95], v[66:69], v[162:165], a[92:95]\n" \
  "ds_read_b128 v[162:165], v246 offset:64512\n" \
  "v_mfma_f32_16x16x32_f16 a[92:95], v[62:65], v[130:133], a[92:95]\n" \
  "s_waitcnt vmcnt(20)\n" \
  "ds_read_b128 v[130:133], v246 offset:56320\n" \
  "s_waitcnt lgkmcnt(14)\n" \
  "v_mfma_f32_16x16x32_f16 a[64:67], v[78:81], v[102:105], a[64:67]\n" \
  "buffer_load_dwordx4 v[42:45], %[bxr], s[40:43], s72 offen offset:3072 sc0 sc1\n" \
  "v_mfma_f32_16x16x32_f16 a[64:67], v[82:85], v[134:137], a[64:67]\n" \
  "s_add_u32 s73, s74, s70\n" \
  "s_nop 7\n" \
  "v_mfma_f32_16x16x32_f16 a[96:99], v[58:61], v[134:137], a[96:99]\n" \
  "ds_read_b128 v[134:137], v247 offset:8192\n" \
  "v_mfma_f32_16x16x32_f16 a[96:99], v[54:57], v[102:105], a[96:99]\n" \
  "s_waitcnt lgkmcnt(13)\n" \
  "ds_read_b128 v[102:105], v247 offset:0\n" \
  "v_mfma_f32_16x16x32_f16 a[100:103], v[54:57], v[106:109], a[100:103]\n" \
  "buffer_load_dwordx4 v[66:69], %[bxr], s[40:43], s73 offen offset:0 sc0 sc1\n" \
  "v_mfma_f32_16x16x32_f16 a[100:103], v[58:61], v[138:141], a[100:103]\n" \
  "global_load_dwordx4 v[238:241], v249, %[xfi] sc1\n" \
  "v_mfma_f32_16x16x32_f16 a[68:71], v[82:85], v[138:141], a[68:71]\n" \
  "ds_read_b128 v[138:141], v247 offset:9216\n" \
  "v_mfma_f32_16x16x32_f16 a[68:71], v[78:81], v[106:109], a[68:71]\n" \
  "s_waitcnt lgkmcnt(13)\n" \
  "ds_read_b128 v[106:109], v247 offset:1024\n" \
  "v_mfma_f32_16x16x32_f16 a[72:75], v[78:81], v[110:113], a[72:75]\n" \
  "buffer_load_dwordx4 v[62:65], %[bxr], s[40:43], s73 offen offset:1024 sc0 sc1\n" \
  "v_mfma_f32_16x16x32_f16 a[72:75], v[82:85], v[142:145], a[72:75]\n" \
  "buffer_store_dwordx4 a[60:63], %[voffO], s[44:47], s84 offen offset:256 sc1\n" \
  "v_mfma_f32_16x16x32_f16 a[104:107], v[58:61], v[142:145], a[104:107]\n" \
  "ds_read_b128 v[142:145], v247 offset:10240\n" \
  "v_mfma_f32_16x16x32_f16 a[104:107], v[54:57], v[110:113], a[104:107]\n" \
  "s_waitcnt lgkmcnt(13)\n" \
  "ds_read_b128 v[110:113], v247 offset:2048\n" \
  "v_mfma_f32_16x16x32_f16 a[108:111], v[54:57], v[114:117], a[108:111]\n" \
  "v_mfma_f32_16x16x32_f16 a[108:111], v[58:61], v[146:149], a[108:111]\n" \
  "v_mfma_f32_16x16x32_f16 a[76:79], v[82:85], v[146:149], a[76:79]\n" \
  "ds_read_b128 v[146:149], v247 offset:11264\n" \
  "v_mfma_f32_16x16x32_f16 a[76:79], v[78:81], v[114:117], a[76:79]\n" \
  "s_waitcnt lgkmcnt(13)\n" \
  "ds_read_b128 v[114:117], v247 offset:3072\n" \
  "v_mfma_f32_16x16x32_f16 a[80:83], v[78:81], v[118:121], a[80:83]\n" \
  "v_mfma_f32_16x16x32_f16 a[80:83], v[82:85], v[150:153], a[80:83]\n" \
  "s_nop 7\n" \
  "v_mfma_f32_16x16x32_f16 a[112:115], v[58:61], v[150:153], a[112:115]\n" \
  "ds_read_b128 v[150:153], v247 offset:12288\n" \
  "v_mfma_f32_16x16x32_f16 a[112:115], v[54:57], v[118:121], a[112:115]\n" \
  "s_waitcnt lgkmcnt(13)\n" \
  "ds_read_b128 v[118:121], v247 offset:4096\n" \
  "v_mfma_f32_16x16x32_f16 a[116:119], v[54:57], v[122:125], a[116:119]\n" \
  "v_mfma_f32_16x16x32_f16 a[116:119], v[58:61], v[154:157], a[116:119]\n" \
  "v_mfma_f32_16x16x32_f16 a[84:87], v[82:85], v[154:157], a[84:87]\n" \
  "ds_read_b128 v[154:157], v247 offset:13312\n" \
  "v_mfma_f32_16x16x32_f16 a[84:87], v[78:81], v[122:125], a[84:87]\n" \
  "s_waitcnt lgkmcnt(13)\n" \
  "ds_read_b128 v[122:125], v247 offset:5120\n" \
  "v_mfma_f32_16x16x32_f16 a[88:91], v[78:81], v[126:129], a[88:91]\n" \
  "v_mfma_f32_16x16x32_f16 a[88:91], v[82:85], v[158:161], a[88:91]\n" \
  "s_add_u32 s83, s90, 0x400000\n" \
  "v_mfma_f32_16x16x32_f16 a[120:123], v[58:61], v[158:161], a[120:123]\n" \
  "ds_read_b128 v[158:161], v247 offset:14336\n" \
  "v_mfma_f32_16x16x32_f16 a[120:123], v[54:57], v[126:129], a[120:123]\n" \
  "s_waitcnt lgkmcnt(13)\n" \
  "ds_read_b128 v[126:129], v247 offset:6144\n" \
  "v_mfma_f32_16x16x32_f16 a[124:127], v[54:57], v[130:133], a[124:127]\n" \
  "buffer_load_dwordx4 v[54:57], %[bxr], s[40:43], s73 offen offset:2048 sc0 sc1\n" \
  "v_mfma_f32_16x16x32_f16 a[124:127], v[58:61], v[162:165], a[124:127]\n" \
  "v_readlane_b32 s74, %[vtabA], 31\n" \
  "v_mfma_f32_16x16x32_f16 a[92:95], v[82:85], v[162:165], a[92:95]\n" \
  "ds_read_b128 v[162:165], v247 offset:15360\n" \
  "v_mfma_f32_16x16x32_f16 a[92:95], v[78:81], v[130:133], a[92:95]\n" \
  "s_waitcnt vmcnt(20)\n" \
  "ds_read_b128 v[130:133], v247 offset:7168\n" \
  "s_waitcnt lgkmcnt(14)\n" \
  "v_mfma_f32_16x16x32_f16 a[64:67], v[94:97], v[102:105], a[64:67]\n" \
  "buffer_load_dwordx4 v[58:61], %[bxr], s[40:43], s73 offen offset:3072 sc0 sc1\n" \
  "v_mfma_f32_16x16x32_f16 a[64:67], v[98:101], v[134:137], a[64:67]\n" \
  "buffer_store_dwordx4 a[128:131], %[voffO], s[44:47], s83 offen offset:0 sc1\n" \
  "s_nop 7\n" \
  "v_mfma_f32_16x16x32_f16 a[96:99], v[74:77], v[134:137], a[96:99]\n" \
  "s_add_u32 s72, s74, s70\n" \
  "ds_read_b128 v[134:137], v247 offset:24576\n" \
  "v_mfma_f32_16x16x32_f16 a[96:99], v[70:73], v[102:105], a[96:99]\n" \
  "s_waitcnt lgkmcnt(13)\n" \
  "ds_read_b128 v[102:105], v247 offset:16384\n" \
  "v_mfma_f32_16x16x32_f16 a[100:103], v[70:73], v[106:109], a[100:103]\n" \
  "buffer_load_dwordx4 v[82:85], %[bxr], s[40:43], s72 offen offset:0 sc0 sc1\n" \
  "v_mfma_f32_16x16x32_f16 a[100:103], v[74:77], v[138:141], a[100:103]\n" \
  "v_mfma_f32_16x16x32_f16 a[68:71], v[98:101], v[138:141], a[68:71]\n" \
  "ds_read_b128 v[138:141], v247 offset:25600\n" \
  "v_mfma_f32_16x16x32_f16 a[68:71], v[94:97], v[106:109], a[68:71]\n" \
  "s_waitcnt lgkmcnt(13)\n" \
  "ds_read_b128 v[106:109], v247 offset:17408\n" \
  "v_mfma_f32_16x16x32_f16 a[72:75], v[94:97], v[110:113], a[72:75]\n" \
  "buffer_load_dwordx4 v[78:81], %[bxr], s[40:43], s72 offen offset:1024 sc0 sc1\n" \
  "v_mfma_f32_16x16x32_f16 a[72:75], v[98:101], v[142:145], a[72:75]\n" \
  "v_mfma_f32_16x16x32_f16 a[104:107], v[74:77], v[142:145], a[104:107]\n" \
  "ds_read_b128 v[142:145], v247 offset:26624\n" \
  "v_mfma_f32_16x16x32_f16 a[104:107], v[70:73], v[110:113], a[104:107]\n" \
  "s_waitcnt lgkmcnt(13)\n" \
  "ds_read_b128 v[110:113], v247 offset:18432\n" \
  "v_mfma_f32_16x16x32_f16 a[108:111], v[70:73], v[114:117], a[108:111]\n" \
  "v_mfma_f32_16x16x32_f16 a[108:111], v[74:77], v[146:149], a[108:111]\n" \
  "v_mfma_f32_16x16x32_f16 a[76:79], v[98:101], v[146:149], a[76:79]\n" \
  "ds_read_b128 v[146:149], v247 offset:27648\n" \
  "v_mfma_f32_16x16x32_f16 a[76:79], v[94:97], v[114:117], a[76:79]\n" \
  "s_waitcnt lgkmcnt(13)\n" \
  "ds_read_b128 v[114:117], v247 offset:19456\n" \
  "v_mfma_f32_16x16x32_f16 a[80:83], v[94:97], v[118:121], a[80:83]\n" \
  "v_mfma_f32_16x16x32_f16 a[80:83], v[98:101], v[150:153], a[80:83]\n" \
  "s_nop 7\n" \
  "v_mfma_f32_16x16x32_f16 a[112:115], v[74:77], v[150:153], a[112:115]\n" \
  "ds_read_b128 v[150:153], v247 offset:28672\n" \
  "v_mfma_f32_16x16x32_f16 a[112:115], v[70:73], v[118:121], a[112:115]\n" \
  "s_waitcnt lgkmcnt(13)\n" \
  "ds_read_b128 v[118:121], v247 offset:20480\n" \
  "v_mfma_f32_16x16x32_f16 a[116:119], v[70:73], v[122:125], a[116:119]\n" \
  "v_mfma_f32_16x16x32_f16 a[116:119], v[74:77], v[154:157], a[116:119]\n" \
  "v_mfma_f32_16x16x32_f16 a[84:87], v[98:101], v[154:157], a[84:87]\n" \
  "ds_read_b128 v[154:157], v247 offset:29696\n" \
  "v_mfma_f32_16x16x32_f16 a[84:87], v[94:97], v[122:125], a[84:87]\n" \
  "s_waitcnt lgkmcnt(13)\n" \
  "ds_read_b128 v[122:125], v247 offset:21504\n" \
  "v_mfma_f32_16x16x32_f16 a[88:91], v[94:97], v[126:129], a[88:91]\n" \
  "v_mfma_f32_16x16x32_f16 a[88:91], v[98:101], v[158:161], a[88:91]\n" \
  "buffer_store_dwordx4 a[144:147], %[voffO], s[44:47], s83 offen offset:256 sc1\n" \
  "v_mfma_f32_16x16x32_f16 a[120:123], v[74:77], v[158:161], a[120:123]\n" \
  "ds_read_b128 v[158:161], v247 offset:30720\n" \
  "v_mfma_f32_16x16x32_f16 a[120:123], v[70:73], v[126:129], a[120:123]\n" \
  "s_waitcnt lgkmcnt(13)\n" \
  "ds_read_b128 v[126:129], v247 offset:22528\n" \
  "v_mfma_f32_16x16x32_f16 a[124:127], v[70:73], v[130:133], a[124:127]\n" \
  "buffer_load_dwordx4 v[70:73], %[bxr], s[40:43], s72 offen offset:2048 sc0 sc1\n" \
  "v_mfma_f32_16x16x32_f16 a[124:127], v[74:77], v[162:165], a[124:127]\n" \
  "v_readlane_b32 s74, %[vtabA], 23\n" \
  "v_mfma_f32_16x16x32_f16 a[92:95], v[98:101], v[162:165], a[92:95]\n" \
  "ds_read_b128 v[162:165], v247 offset:31744\n" \
  "v_mfma_f32_16x16x32_f16 a[92:95], v[94:97], v[130:133], a[92:95]\n" \
  "s_waitcnt vmcnt(21)\n" \
  "ds_read_b128 v[130:133], v247 offset:23552\n" \
  "s_waitcnt lgkmcnt(14)\n" \
  "v_mfma_f32_16x16x32_f16 a[64:67], v[18:21], v[102:105], a[64:67]\n" \
  "buffer_load_dwordx4 v[74:77], %[bxr], s[40:43], s72 offen offset:3072 sc0 sc1\n" \
  "v_mfma_f32_16x16x32_f16 a[64:67], v[14:17], v[134:137], a[64:67]\n" \
  "s_add_u32 s73, s74, s71\n" \
  "s_nop 7\n" \
  "v_mfma_f32_16x16x32_f16 a[96:99], v[90:93], v[134:137], a[96:99]\n" \
  "ds_read_b128 v[134:137], v245 offset:8192\n" \
  "v_mfma_f32_16x16x32_f16 a[96:99], v[86:89], v[102:105], a[96:99]\n" \
  "s_waitcnt lgkmcnt(13)\n" \
  "ds_read_b128 v[102:105], v245 offset:0\n" \
  "v_mfma_f32_16x16x32_f16 a[100:103], v[86:89], v[106:109], a[100:103]\n" \
  "buffer_load_dwordx4 v[98:101], %[bxr], s[40:43], s73 offen offset:0 sc0 sc1\n" \
  "v_mfma_f32_16x16x32_f16 a[100:103], v[90:93], v[138:141], a[100:103]\n" \
  "v_mfma_f32_16x16x32_f16 a[68:71], v[14:17], v[138:141], a[68:71]\n" \
  "ds_read_b128 v[138:141], v245 offset:9216\n" \
  "v_mfma_f32_16x16x32_f16 a[68:71], v[18:21], v[106:109], a[68:71]\n" \
  "s_waitcnt lgkmcnt(13)\n" \
  "ds_read_b128 v[106:109], v245 offset:1024\n" \
  "v_mfma_f32_16x16x32_f16 a[72:75], v[18:21], v[110:113], a[72:75]\n" \
  "buffer_load_dwordx4 v[94:97], %[bxr], s[40:43], s73 offen offset:1024 sc0 sc1\n" \
  "v_mfma_f32_16x16x32_f16 a[72:75], v[14:17], v[142:145], a[72:75]\n" \
  "s_add_u32 s84, s90, 0x410000\n" \
  "v_mfma_f32_16x16x32_f16 a[104:107], v[90:93], v[142:145], a[104:107]\n" \
  "ds_read_b128 v[142:145], v245 offset:10240\n" \
  "v_mfma_f32_16x16x32_f16 a[104:107], v[86:89], v[110:113], a[104:107]\n" \
  "s_waitcnt lgkmcnt(13)\n" \
  "ds_read_b128 v[110:113], v245 offset:2048\n" \
  "v_mfma_f32_16x16x32_f16 a[108:111], v[86:89], v[114:117], a[108:111]\n" \
  "v_mfma_f32_16x16x32_f16 a[108:111], v[90:93], v[146:149], a[108:111]\n" \
  "v_mfma_f32_16x16x32_f16 a[76:79], v[14:17], v[146:149], a[76:79]\n" \
  "ds_read_b128 v[146:149], v245 offset:11264\n" \
  "v_mfma_f32_16x16x32_f16 a[76:79], v[18:21], v[114:117], a[76:79]\n" \
  "s_waitcnt lgkmcnt(13)\n" \
  "ds_read_b128 v[114:117], v245 offset:3072\n" \
  "v_mfma_f32_16x16x32_f16 a[80:83], v[18:21], v[118:121], a[80:83]\n" \
  "v_mfma_f32_16x16x32_f16 a[80:83], v[14:17], v[150:153], a[80:83]\n" \
  "buffer_store_dwordx4 a[132:135], %[voffO], s[44:47], s84 offen offset:0 sc1\n" \
  "s_nop 7\n" \
  "v_mfma_f32_16x16x32_f16 a[112:115], v[90:93], v[150:153], a[112:115]\n" \
  "ds_read_b128 v[150:153], v245 offset:12288\n" \
  "v_mfma_f32_16x16x32_f16 a[112:115], v[86:89], v[118:121], a[112:115]\n" \
  "s_waitcnt lgkmcnt(13)\n" \
  "ds_read_b128 v[118:121], v245 offset:4096\n" \
  "v_mfma_f32_16x16x32_f16 a[116:119], v[86:89], v[122:125], a[116:119]\n" \
  "v_mfma_f32_16x16x32_f16 a[116:119], v[90:93], v[154:157], a[116:119]\n" \
  "v_mfma_f32_16x16x32_f16 a[84:87], v[14:17], v[154:157], a[84:87]\n" \
  "ds_read_b128 v[154:157], v245 offset:13312\n" \
  "v_mfma_f32_16x16x32_f16 a[84:87], v[18:21], v[122:125], a[84:87]\n" \
  "s_waitcnt lgkmcnt(13)\n" \
  "ds_read_b128 v[122:125], v245 offset:5120\n" \
  "v_mfma_f32_16x16x32_f16 a[88:91], v[18:21], v[126:129], a[88:91]\n" \
  "v_mfma_f32_16x16x32_f16 a[88:91], v[14:17], v[158:161], a[88:91]\n" \
  "v_mfma_f32_16x16x32_f16 a[120:123], v[90:93], v[158:161], a[120:123]\n" \
  "ds_read_b128 v[158:161], v245 offset:14336\n" \
  "v_mfma_f32_16x16x32_f16 a[120:123], v[86:89], v[126:129], a[120:123]\n" \
  "s_waitcnt lgkmcnt(13)\n" \
  "ds_read_b128 v[126:129], v245 offset:6144\n" \
  "v_mfma_f32_16x16x32_f16 a[124:127], v[86:89], v[130:133], a[124:127]\n" \
  "buffer_load_dwordx4 v[86:89], %[bxr], s[40:43], s73 offen offset:2048 sc0 sc1\n" \
  "v_mfma_f32_16x16x32_f16 a[124:127], v[90:93], v[162:165], a[124:127]\n" \
  "v_readlane_b32 s74, %[vtabA], 24\n" \
  "v_mfma_f32_16x16x32_f16 a[92:95], v[14:17], v[162:165], a[92:95]\n" \
  "ds_read_b128 v[162:165], v245 offset:15360\n" \
  "v_mfma_f32_16x16x32_f16 a[92:95], v[18:21], v[130:133], a[92:95]\n" \
  "s_waitcnt vmcnt(20)\n" \
  "ds_read_b128 v[130:133], v245 offset:7168\n" \
  "s_waitcnt lgkmcnt(14)\n" \
  "v_mfma_f32_16x16x32_f16 a[64:67], v[34:37], v[102:105], a[64:67]\n" \
  "buffer_load_dwordx4 v[90:93], %[bxr], s[40:43], s73 offen offset:3072 sc0 sc1\n" \
  "v_mfma_f32_16x16x32_f16 a[64:67], v[30:33], v[134:137], a[64:67]\n" \
  "s_add_u32 s72, s74, s71\n" \
  "s_nop 7\n" \
  "v_mfma_f32_16x16x32_f16 a[96:99], v[26:29], v[134:137], a[96:99]\n" \
  "ds_read_b128 v[134:137], v245 offset:24576\n" \
  "v_mfma_f32_16x16x32_f16 a[96:99], v[22:25], v[102:105], a[96:99]\n" \
  "s_waitcnt lgkmcnt(13)\n" \
  "ds_read_b128 v[102:105], v245 offset:16384\n" \
  "v_mfma_f32_16x16x32_f16 a[100:103], v[22:25], v[106:109], a[100:103]\n" \
  "buffer_load_dwordx4 v[14:17], %[bxr], s[40:43], s72 offen offset:0 sc0 sc1\n" \
  "v_mfma_f32_16x16x32_f16 a[100:103], v[26:29], v[138:141], a[100:103]\n" \
  "v_mfma_f32_16x16x32_f16 a[68:71], v[30:33], v[138:141], a[68:71]\n" \
  "ds_read_b128 v[138:141], v245 offset:25600\n" \
  "v_mfma_f32_16x16x32_f16 a[68:71], v[34:37], v[106:109], a[68:71]\n" \
  "s_waitcnt lgkmcnt(13)\n" \
  "ds_read_b128 v[106:109], v245 offset:17408\n" \
  "v_mfma_f32_16x16x32_f16 a[72:75], v[34:37], v[110:113], a[72:75]\n" \
  "buffer_load_dwordx4 v[18:21], %[bxr], s[40:43], s72 offen offset:1024 sc0 sc1\n" \
  "v_mfma_f32_16x16x32_f16 a[72:75], v[30:33], v[142:145], a[72:75]\n" \
  "buffer_store_dwordx4 a[148:151], %[voffO], s[44:47], s84 offen offset:256 sc1\n" \
  "v_mfma_f32_16x16x32_f16 a[104:107], v[26:29], v[142:145], a[104:107]\n" \
  "ds_read_b128 v[142:145], v245 offset:26624\n" \
  "v_mfma_f32_16x16x32_f16 a[104:107], v[22:25], v[110:113], a[104:107]\n" \
  "s_waitcnt lgkmcnt(13)\n" \
  "ds_read_b128 v[110:113], v245 offset:18432\n" \
  "v_mfma_f32_16x16x32_f16 a[108:111], v[22:25], v[114:117], a[108:111]\n" \
  "v_mfma_f32_16x16x32_f16 a[108:111], v[26:29], v[146:149], a[108:111]\n" \
  "s_waitcnt vmcnt(16)\n" \
  "v_mov_b32 v250, s76\n" \
  "s_nop 0\n" \
  "global_store_dword v249, v250, %[xfo] offset:4 sc1\n" \
  "v_mfma_f32_16x16x32_f16 a[76:79], v[30:33], v[146:149], a[76:79]\n" \
  "ds_read_b128 v[146:149], v245 offset:27648\n" \
  "v_mfma_f32_16x16x32_f16 a[76:79], v[34:37], v[114:117], a[76:79]\n" \
  "s_waitcnt lgkmcnt(13)\n" \
  "ds_read_b128 v[114:117], v245 offset:19456\n" \
  "v_mfma_f32_16x16x32_f16 a[80:83], v[34:37], v[118:121], a[80:83]\n" \
  "s_add_u32 s60, s91, 0x0\n" \
  "v_readfirstlane_b32 s79, v238\n" \
  "s_nop 0\n" \
  "s_cmp_ge_u32 s79, s76\n" \
  "s_cbranch_scc0 L_s91x_%=\n" \
  "L_b91x_%=:\n" \
  "global_load_dword v251, v249, %[wsp] sc1\n" \
  "v_mfma_f32_16x16x32_f16 a[80:83], v[30:33], v[150:153], a[80:83]\n" \
  "s_nop 7\n" \
  "v_mfma_f32_16x16x32_f16 a[112:115], v[26:29], v[150:153], a[112:115]\n" \
  "ds_read_b128 v[150:153], v245 offset:28672\n" \
  "v_mfma_f32_16x16x32_f16 a[112:115], v[22:25], v[118:121], a[112:115]\n" \
  "s_waitcnt lgkmcnt(13)\n" \
  "ds_read_b128 v[118:121], v245 offset:20480\n" \
  "v_mfma_f32_16x16x32_f16 a[116:119], v[22:25], v[122:125], a[116:119]\n" \
  "buffer_load_dwordx4 a[0:3], %[voffO], s[44:47], s60 offen offset:0 sc1\n" \
  "v_mfma_f32_16x16x32_f16 a[116:119], v[26:29], v[154:157], a[116:119]\n" \
  "v_mfma_f32_16x16x32_f16 a[84:87], v[30:33], v[154:157], a[84:87]\n" \
  "ds_read_b128 v[154:157], v245 offset:29696\n" \
  "v_mfma_f32_16x16x32_f16 a[84:87], v[34:37], v[122:125], a[84:87]\n" \
  "s_waitcnt lgkmcnt(13)\n" \
  "ds_read_b128 v[122:125], v245 offset:21504\n" \
  "v_mfma_f32_16x16x32_f16 a[88:91], v[34:37], v[126:129], a[88:91]\n" \
  "buffer_load_dwordx4 a[16:19], %[voffO], s[44:47], s60 offen offset:256 sc1\n" \
  "v_mfma_f32_16x16x32_f16 a[88:91], v[30:33], v[158:161], a[88:91]\n" \
  "s_add_u32 s83, s90, 0x420000\n" \
  "s_add_u32 s61, s91, 0x10000\n" \
  "v_mfma_f32_16x16x32_f16 a[120:123], v[26:29], v[158:161], a[120:123]\n" \
  "ds_read_b128 v[158:161], v245 offset:30720\n" \
  "v_mfma_f32_16x16x32_f16 a[120:123], v[22:25], v[126:129], a[120:123]\n" \
  "s_waitcnt lgkmcnt(13)\n" \
  "ds_read_b128 v[126:129], v245 offset:22528\n" \
  "v_mfma_f32_16x16x32_f16 a[124:127], v[22:25], v[130:133], a[124:127]\n" \
  "buffer_load_dwordx4 a[4:7], %[voffO], s[44:47], s61 offen offset:0 sc1\n" \
  "v_mfma_f32_16x16x32_f16 a[124:127], v[26:29], v[162:165], a[124:127]\n" \
  "buffer_load_dwordx4 v[22:25], %[bxr], s[40:43], s72 offen offset:2048 sc0 sc1\n" \
  "v_mfma_f32_16x16x32_f16 a[92:95], v[30:33], v[162:165], a[92:95]\n" \
  "ds_read_b128 v[162:165], v245 offset:31744\n" \
  "v_mfma_f32_16x16x32_f16 a[92:95], v[34:37], v[130:133], a[92:95]\n" \
  "ds_read_b128 v[130:133], v245 offset:23552\n" \
  "s_waitcnt lgkmcnt(14)\n" \
  "v_mfma_f32_16x16x32_f16 a[64:67], v[50:53], v[102:105], a[64:67]\n" \
  "buffer_load_dwordx4 a[20:23], %[voffO], s[44:47], s61 offen offset:256 sc1\n" \
  "v_mfma_f32_16x16x32_f16 a[64:67], v[46:49], v[134:137], a[64:67]\n" \
  "buffer_store_dwordx4 a[136:139], %[voffO], s[44:47], s83 offen offset:0 sc1\n" \
  "s_nop 7\n" \
  "v_mfma_f32_16x16x32_f16 a[96:99], v[42:45], v[134:137], a[96:99]\n" \
  "s_add_u32 s60, s91, 0x20000\n" \
  "ds_read_b128 v[134:137], v245 offset:40960\n" \
  "v_readlane_b32 s74, %[vtabA], 25\n" \
  "v_mfma_f32_16x16x32_f16 a[96:99], v[38:41], v[102:105], a[96:99]\n" \
  "s_waitcnt lgkmcnt(13)\n" \
  "ds_read_b128 v[102:105], v245 offset:32768\n" \
  "v_mfma_f32_16x16x32_f16 a[100:103], v[38:41], v[106:109], a[100:103]\n" \
  "buffer_load_dwordx4 a[8:11], %[voffO], s[44:47], s60 offen offset:0 sc1\n" \
  "v_mfma_f32_16x16x32_f16 a[100:103], v[42:45], v[138:141], a[100:103]\n" \
  "buffer_load_dwordx4 v[26:29], %[bxr], s[40:43], s72 offen offset:3072 sc0 sc1\n" \
  "v_mfma_f32_16x16x32_f16 a[68:71], v[46:49], v[138:141], a[68:71]\n" \
  "s_add_u32 s73, s74, s71\n" \
  "ds_read_b128 v[138:141], v245 offset:41984\n" \
  "v_mfma_f32_16x16x32_f16 a[68:71], v[50:53], v[106:109], a[68:71]\n" \
  "s_waitcnt lgkmcnt(13)\n" \
  "ds_read_b128 v[106:109], v245 offset:33792\n" \
  "v_mfma_f32_16x16x32_f16 a[72:75], v[50:53], v[110:113], a[72:75]\n" \
  "buffer_load_dwordx4 a[24:27], %[voffO], s[44:47], s60 offen offset:256 sc1\n" \
  "v_mfma_f32_16x16x32_f16 a[72:75], v[46:49], v[142:145], a[72:75]\n" \
  "buffer_load_dwordx4 v[30:33], %[bxr], s[40:43], s73 offen offset:0 sc0 sc1\n" \
  "v_mfma_f32_16x16x32_f16 a[104:107], v[42:45], v[142:145], a[104:107]\n" \
  "s_add_u32 s61, s91, 0x30000\n" \
  "ds_read_b128 v[142:145], v245 offset:43008\n" \
  "v_mfma_f32_16x16x32_f16 a[104:107], v[38:41], v[110:113], a[104:107]\n" \
  "s_waitcnt lgkmcnt(13)\n" \
  "ds_read_b128 v[110:113], v245 offset:34816\n" \
  "v_mfma_f32_16x16x32_f16 a[108:111], v[38:41], v[114:117], a[108:111]\n" \
  "buffer_load_dwordx4 a[12:15], %[voffO], s[44:47], s61 offen offset:0 sc1\n" \
  "v_mfma_f32_16x16x32_f16 a[108:111], v[42:45], v[146:149], a[108:111]\n" \
  "buffer_load_dwordx4 v[34:37], %[bxr], s[40:43], s73 offen offset:1024 sc0 sc1\n" \
  "v_mfma_f32_16x16x32_f16 a[76:79], v[46:49], v[146:149], a[76:79]\n" \
  "ds_read_b128 v[146:149], v245 offset:44032\n" \
  "v_mfma_f32_16x16x32_f16 a[76:79], v[50:53], v[114:117], a[76:79]\n" \
  "s_waitcnt lgkmcnt(13)\n" \
  "ds_read_b128 v[114:117], v245 offset:35840\n" \
  "v_mfma_f32_16x16x32_f16 a[80:83], v[50:53], v[118:121], a[80:83]\n" \
  "buffer_load_dwordx4 a[28:31], %[voffO], s[44:47], s61 offen offset:256 sc1\n" \
  "v_mfma_f32_16x16x32_f16 a[80:83], v[46:49], v[150:153], a[80:83]\n" \
  "s_nop 7\n" \
  "v_mfma_f32_16x16x32_f16 a[112:115], v[42:45], v[150:153], a[112:115]\n" \
  "ds_read_b128 v[150:153], v245 offset:45056\n" \
  "v_mfma_f32_16x16x32_f16 a[112:115], v[38:41], v[118:121], a[112:115]\n" \
  "s_waitcnt lgkmcnt(13)\n" \
  "ds_read_b128 v[118:121], v245 offset:36864\n" \
  "v_mfma_f32_16x16x32_f16 a[116:119], v[38:41], v[122:125], a[116:119]\n" \
  "v_mfma_f32_16x16x32_f16 a[116:119], v[42:45], v[154:157], a[116:119]\n" \
  "v_mfma_f32_16x16x32_f16 a[84:87], v[46:49], v[154:157], a[84:87]\n" \
  "ds_read_b128 v[154:157], v245 offset:46080\n" \
  "v_mfma_f32_16x16x32_f16 a[84:87], v[50:53], v[122:125], a[84:87]\n" \
  "s_waitcnt lgkmcnt(13)\n" \
  "ds_read_b128 v[122:125], v245 offset:37888\n" \
  "v_mfma_f32_16x16x32_f16 a[88:91], v[50:53], v[126:129], a[88:91]\n" \
  "v_mfma_f32_16x16x32_f16 a[88:91], v[46:49], v[158:161], a[88:91]\n" \
  "buffer_store_dwordx4 a[152:155], %[voffO], s[44:47], s83 offen offset:256 sc1\n" \
  "v_mfma_f32_16x16x32_f16 a[120:123], v[42:45], v[158:161], a[120:123]\n" \
  "ds_read_b128 v[158:161], v245 offset:47104\n" \
  "v_mfma_f32_16x16x32_f16 a[120:123], v[38:41], v[126:129], a[120:123]\n" \
  "s_waitcnt lgkmcnt(13)\n" \
  "ds_read_b128 v[126:129], v245 offset:38912\n" \
  "v_mfma_f32_16x16x32_f16 a[124:127], v[38:41], v[130:133], a[124:127]\n" \
  "buffer_load_dwordx4 v[38:41], %[bxr], s[40:43], s73 offen offset:2048 sc0 sc1\n" \
  "v_mfma_f32_16x16x32_f16 a[124:127], v[42:45], v[162:165], a[124:127]\n" \
  "v_readlane_b32 s74, %[vtabA], 26\n" \
  "v_mfma_f32_16x16x32_f16 a[92:95], v[46:49], v[162:165], a[92:95]\n" \
  "ds_read_b128 v[162:165], v245 offset:48128\n" \
  "v_mfma_f32_16x16x32_f16 a[92:95], v[50:53], v[130:133], a[92:95]\n" \
  "s_waitcnt vmcnt(29)\n" \
  "ds_read_b128 v[130:133], v245 offset:39936\n" \
  "s_waitcnt lgkmcnt(14)\n" \
  "v_mfma_f32_16x16x32_f16 a[64:67], v[66:69], v[102:105], a[64:67]\n" \
  "buffer_load_dwordx4 v[42:45], %[bxr], s[40:43], s73 offen offset:3072 sc0 sc1\n" \
  "v_mfma_f32_16x16x32_f16 a[64:67], v[62:65], v[134:137], a[64:67]\n" \
  "s_add_u32 s72, s74, s71\n" \
  "s_nop 7\n" \
  "v_mfma_f32_16x16x32_f16 a[96:99], v[58:61], v[134:137], a[96:99]\n" \
  "ds_read_b128 v[134:137], v245 offset:57344\n" \
  "v_mfma_f32_16x16x32_f16 a[96:99], v[54:57], v[102:105], a[96:99]\n" \
  "s_waitcnt lgkmcnt(13)\n" \
  "ds_read_b128 v[102:105], v245 offset:49152\n" \
  "v_mfma_f32_16x16x32_f16 a[100:103], v[54:57], v[106:109], a[100:103]\n" \
  "buffer_load_dwordx4 v[46:49], %[bxr], s[40:43], s72 offen offset:0 sc0 sc1\n" \
  "v_mfma_f32_16x16x32_f16 a[100:103], v[58:61], v[138:141], a[100:103]\n" \
  "v_mfma_f32_16x16x32_f16 a[68:71], v[62:65], v[138:141], a[68:71]\n" \
  "ds_read_b128 v[138:141], v245 offset:58368\n" \
  "v_mfma_f32_16x16x32_f16 a[68:71], v[66:69], v[106:109], a[68:71]\n" \
  "s_waitcnt lgkmcnt(13)\n" \
  "ds_read_b128 v[106:109], v245 offset:50176\n" \
  "v_mfma_f32_16x16x32_f16 a[72:75], v[66:69], v[110:113], a[72:75]\n" \
  "buffer_load_dwordx4 v[50:53], %[bxr], s[40:43], s72 offen offset:1024 sc0 sc1\n" \
  "v_mfma_f32_16x16x32_f16 a[72:75], v[62:65], v[142:145], a[72:75]\n" \
  "s_add_u32 s84, s90, 0x430000\n" \
  "v_mfma_f32_16x16x32_f16 a[104:107], v[58:61], v[142:145], a[104:107]\n" \
  "ds_read_b128 v[142:145], v245 offset:59392\n" \
  "v_mfma_f32_16x16x32_f16 a[104:107], v[54:57], v[110:113], a[104:107]\n" \
  "s_waitcnt lgkmcnt(13)\n" \
  "ds_read_b128 v[110:113], v245 offset:51200\n" \
  "v_mfma_f32_16x16x32_f16 a[108:111], v[54:57], v[114:117], a[108:111]\n" \
  "global_load_dwordx4 v[238:241], v249, %[xfi] sc1\n" \
  "v_mfma_f32_16x16x32_f16 a[108:111], v[58:61], v[146:149], a[108:111]\n" \
  "v_mfma_f32_16x16x32_f16 a[76:79], v[62:65], v[146:149], a[76:79]\n" \
  "ds_read_b128 v[146:149], v245 offset:60416\n" \
  "v_mfma_f32_16x16x32_f16 a[76:79], v[66:69], v[114:117], a[76:79]\n" \
  "s_waitcnt lgkmcnt(13)\n" \
  "ds_read_b128 v[114:117], v245 offset:52224\n" \
  "v_mfma_f32_16x16x32_f16 a[80:83], v[66:69], v[118:121], a[80:83]\n" \
  "v_mfma_f32_16x16x32_f16 a[80:83], v[62:65], v[150:153], a[80:83]\n" \
  "buffer_store_dwordx4 a[140:143], %[voffO], s[44:47], s84 offen offset:0 sc1\n" \
  "s_nop 7\n" \
  "v_mfma_f32_16x16x32_f16 a[112:115], v[58:61], v[150:153], a[112:115]\n" \
  "ds_read_b128 v[150:153], v245 offset:61440\n" \
  "v_mfma_f32_16x16x32_f16 a[112:115], v[54:57], v[118:121], a[112:115]\n" \
  "s_waitcnt lgkmcnt(13)\n" \
  "ds_read_b128 v[118:121], v245 offset:53248\n" \
  "v_mfma_f32_16x16x32_f16 a[116:119], v[54:57], v[122:125], a[116:119]\n" \
  "v_mfma_f32_16x16x32_f16 a[116:119], v[58:61], v[154:157], a[116:119]\n" \
  "v_mfma_f32_16x16x32_f16 a[84:87], v[62:65], v[154:157], a[84:87]\n" \
  "ds_read_b128 v[154:157], v245 offset:62464\n" \
  "v_mfma_f32_16x16x32_f16 a[84:87], v[66:69], v[122:125], a[84:87]\n" \
  "s_waitcnt lgkmcnt(13)\n" \
  "ds_read_b128 v[122:125], v245 offset:54272\n" \
  "v_mfma_f32_16x16x32_f16 a[88:91], v[66:69], v[126:129], a[88:91]\n" \
  "v_mfma_f32_16x16x32_f16 a[88:91], v[62:65], v[158:161], a[88:91]\n" \
  "v_mfma_f32_16x16x32_f16 a[120:123], v[58:61], v[158:161], a[120:123]\n" \
  "ds_read_b128 v[158:161], v245 offset:63488\n" \
  "v_mfma_f32_16x16x32_f16 a[120:123], v[54:57], v[126:129], a[120:123]\n" \
  "s_waitcnt lgkmcnt(13)\n" \
  "ds_read_b128 v[126:129], v245 offset:55296\n" \
  "v_mfma_f32_16x16x32_f16 a[124:127], v[54:57], v[130:133], a[124:127]\n" \
  "buffer_load_dwordx4 v[54:57], %[bxr], s[40:43], s72 offen offset:2048 sc0 sc1\n" \
  "v_mfma_f32_16x16x32_f16 a[124:127], v[58:61], v[162:165], a[124:127]\n" \
  "v_readlane_b32 s74, %[vtabA], 27\n" \
  "v_mfma_f32_16x16x32_f16 a[92:95], v[62:65], v[162:165], a[92:95]\n" \
  "ds_read_b128 v[162:165], v245 offset:64512\n" \
  "v_mfma_f32_16x16x32_f16 a[92:95], v[66:69], v[130:133], a[92:95]\n" \
  "s_waitcnt vmcnt(28)\n" \
  "ds_read_b128 v[130:133], v245 offset:56320\n" \
  "s_waitcnt lgkmcnt(14)\n" \
  "v_mfma_f32_16x16x32_f16 a[64:67], v[82:85], v[102:105], a[64:67]\n" \
  "buffer_load_dwordx4 v[58:61], %[bxr], s[40:43], s72 offen offset:3072 sc0 sc1\n" \
  "v_mfma_f32_16x16x32_f16 a[64:67], v[78:81], v[134:137], a[64:67]\n" \
  "s_add_u32 s73, s74, s71\n" \
  "s_nop 7\n" \
  "v_mfma_f32_16x16x32_f16 a[96:99], v[74:77], v[134:137], a[96:99]\n" \
  "ds_read_b128 v[134:137], v246 offset:24576\n" \
  "v_mfma_f32_16x16x32_f16 a[96:99], v[70:73], v[102:105], a[96:99]\n" \
  "s_waitcnt lgkmcnt(13)\n" \
  "ds_read_b128 v[102:105], v246 offset:16384\n" \
  "v_mfma_f32_16x16x32_f16 a[100:103], v[70:73], v[106:109], a[100:103]\n" \
  "buffer_load_dwordx4 v[62:65], %[bxr], s[40:43], s73 offen offset:0 sc0 sc1\n" \
  "v_mfma_f32_16x16x32_f16 a[100:103], v[74:77], v[138:141], a[100:103]\n" \
  "v_mfma_f32_16x16x32_f16 a[68:71], v[78:81], v[138:141], a[68:71]\n" \
  "ds_read_b128 v[138:141], v246 offset:25600\n" \
  "v_mfma_f32_16x16x32_f16 a[68:71], v[82:85], v[106:109], a[68:71]\n" \
  "s_waitcnt lgkmcnt(13)\n" \
  "ds_read_b128 v[106:109], v246 offset:17408\n" \
  "v_mfma_f32_16x16x32_f16 a[72:75], v[82:85], v[110:113], a[72:75]\n" \
  "buffer_load_dwordx4 v[66:69], %[bxr], s[40:43], s73 offen offset:1024 sc0 sc1\n" \
  "v_mfma_f32_16x16x32_f16 a[72:75], v[78:81], v[142:145], a[72:75]\n" \
  "buffer_store_dwordx4 a[156:159], %[voffO], s[44:47], s84 offen offset:256 sc1\n" \
  "v_mfma_f32_16x16x32_f16 a[104:107], v[74:77], v[142:145], a[104:107]\n" \
  "ds_read_b128 v[142:145], v246 offset:26624\n" \
  "v_mfma_f32_16x16x32_f16 a[104:107], v[70:73], v[110:113], a[104:107]\n" \
  "s_waitcnt lgkmcnt(13)\n" \
  "ds_read_b128 v[110:113], v246 offset:18432\n" \
  "v_mfma_f32_16x16x32_f16 a[108:111], v[70:73], v[114:117], a[108:111]\n" \
  "v_mfma_f32_16x16x32_f16 a[108:111], v[74:77], v[146:149], a[108:111]\n" \
  "v_mfma_f32_16x16x32_f16 a[76:79], v[78:81], v[146:149], a[76:79]\n" \
  "ds_read_b128 v[146:149], v246 offset:27648\n" \
  "v_mfma_f32_16x16x32_f16 a[76:79], v[82:85], v[114:117], a[76:79]\n" \
  "s_waitcnt lgkmcnt(13)\n" \
  "ds_read_b128 v[114:117], v246 offset:19456\n" \
  "v_mfma_f32_16x16x32_f16 a[80:83], v[82:85], v[118:121], a[80:83]\n" \
  "v_mfma_f32_16x16x32_f16 a[80:83], v[78:81], v[150:153], a[80:83]\n" \
  "s_nop 7\n" \
  "v_mfma_f32_16x16x32_f16 a[112:115], v[74:77], v[150:153], a[112:115]\n" \
  "ds_read_b128 v[150:153], v246 offset:28672\n" \
  "v_mfma_f32_16x16x32_f16 a[112:115], v[70:73], v[118:121], a[112:115]\n" \
  "s_waitcnt lgkmcnt(13)\n" \
  "ds_read_b128 v[118:121], v246 offset:20480\n" \
  "v_mfma_f32_16x16x32_f16 a[116:119], v[70:73], v[122:125], a[116:119]\n" \
  "v_mfma_f32_16x16x32_f16 a[116:119], v[74:77], v[154:157], a[116:119]\n" \
  "v_mfma_f32_16x16x32_f16 a[84:87], v[78:81], v[154:157], a[84:87]\n" \
  "ds_read_b128 v[154:157], v246 offset:29696\n" \
  "v_mfma_f32_16x16x32_f16 a[84:87], v[82:85], v[122:125], a[84:87]\n" \
  "s_waitcnt lgkmcnt(13)\n" \
  "ds_read_b128 v[122:125], v246 offset:21504\n" \
  "v_mfma_f32_16x16x32_f16 a[88:91], v[82:85], v[126:129], a[88:91]\n" \
  "v_mfma_f32_16x16x32_f16 a[88:91], v[78:81], v[158:161], a[88:91]\n" \
  "s_add_u32 s83, s90, 0x500000\n" \
  "v_mfma_f32_16x16x32_f16 a[120:123], v[74:77], v[158:161], a[120:123]\n" \
  "ds_read_b128 v[158:161], v246 offset:30720\n" \
  "v_mfma_f32_16x16x32_f16 a[120:123], v[70:73], v[126:129], a[120:123]\n" \
  "s_waitcnt lgkmcnt(13)\n" \
  "ds_read_b128 v[126:129], v246 offset:22528\n" \
  "v_mfma_f32_16x16x32_f16 a[124:127], v[70:73], v[130:133], a[124:127]\n" \
  "buffer_load_dwordx4 v[70:73], %[bxr], s[40:43], s73 offen offset:2048 sc0 sc1\n" \
  "v_mfma_f32_16x16x32_f16 a[124:127], v[74:77], v[162:165], a[124:127]\n" \
  "v_readlane_b32 s74, %[vtabA], 28\n" \
  "v_mfma_f32_16x16x32_f16 a[92:95], v[78:81], v[162:165], a[92:95]\n" \
  "ds_read_b128 v[162:165], v246 offset:31744\n" \
  "v_mfma_f32_16x16x32_f16 a[92:95], v[82:85], v[130:133], a[92:95]\n" \
  "s_waitcnt vmcnt(28)\n" \
  "ds_read_b128 v[130:133], v246 offset:23552\n" \
  "s_waitcnt lgkmcnt(14)\n" \
  "v_mfma_f32_16x16x32_f16 a[192:195], v[98:101], v[102:105], a[192:195]\n" \
  "buffer_load_dwordx4 v[74:77], %[bxr], s[40:43], s73 offen offset:3072 sc0 sc1\n" \
  "v_mfma_f32_16x16x32_f16 a[192:195], v[94:97], v[134:137], a[192:195]\n" \
  "buffer_store_dwordx4 a[160:163], %[voffO], s[44:47], s83 offen offset:0 sc1\n" \
  "v_mfma_f32_16x16x32_f16 a[224:227], v[90:93], v[134:137], a[224:227]\n" \
  "s_add_u32 s72, s74, s71\n" \
  "ds_read_b128 v[134:137], v246 offset:40960\n" \
  "v_mfma_f32_16x16x32_f16 a[224:227], v[86:89], v[102:105], a[224:227]\n" \
  "s_waitcnt lgkmcnt(13)\n" \
  "ds_read_b128 v[102:105], v246 offset:32768\n" \
  "v_mfma_f32_16x16x32_f16 a[228:231], v[86:89], v[106:109], a[228:231]\n" \
  "buffer_load_dwordx4 v[78:81], %[bxr], s[40:43], s72 offen offset:0 sc0 sc1\n" \
  "v_mfma_f32_16x16x32_f16 a[228:231], v[90:93], v[138:141], a[228:231]\n" \
  "v_mfma_f32_16x16x32_f16 a[196:199], v[94:97], v[138:141], a[196:199]\n" \
  "ds_read_b128 v[138:141], v246 offset:41984\n" \
  "v_mfma_f32_16x16x32_f16 a[196:199], v[98:101], v[106:109], a[196:199]\n" \
  "s_waitcnt lgkmcnt(13)\n" \
  "ds_read_b128 v[106:109], v246 offset:33792\n" \
  "v_mfma_f32_16x16x32_f16 a[200:203], v[98:101], v[110:113], a[200:203]\n" \
  "buffer_load_dwordx4 v[82:85], %[bxr], s[40:43], s72 offen offset:1024 sc0 sc1\n" \
  "v_mfma_f32_16x16x32_f16 a[200:203], v[94:97], v[142:145], a[200:203]\n" \
  "v_mfma_f32_16x16x32_f16 a[232:235], v[90:93], v[142:145], a[232:235]\n" \
  "ds_read_b128 v[142:145], v246 offset:43008\n" \
  "v_mfma_f32_16x16x32_f16 a[232:235], v[86:89], v[110:113], a[232:235]\n" \
  "s_waitcnt lgkmcnt(13)\n" \
  "ds_read_b128 v[110:113], v246 offset:34816\n" \
  "v_mfma_f32_16x16x32_f16 a[236:239], v[86:89], v[114:117], a[236:239]\n" \
  "v_mfma_f32_16x16x32_f16 a[236:239], v[90:93], v[146:149], a[236:239]\n" \
  "v_mfma_f32_16x16x32_f16 a[204:207], v[94:97], v[146:149], a[204:207]\n" \
  "ds_read_b128 v[146:149], v246 offset:44032\n" \
  "v_mfma_f32_16x16x32_f16 a[204:207], v[98:101], v[114:117], a[204:207]\n" \
  "s_waitcnt lgkmcnt(13)\n" \
  "ds_read_b128 v[114:117], v246 offset:35840\n" \
  "v_mfma_f32_16x16x32_f16 a[208:211], v[98:101], v[118:121], a[208:211]\n" \
  "v_mfma_f32_16x16x32_f16 a[208:211], v[94:97], v[150:153], a[208:211]\n" \
  "v_mfma_f32_16x16x32_f16 a[240:243], v[90:93], v[150:153], a[240:243]\n" \
  "ds_read_b128 v[150:153], v246 offset:45056\n" \
  "s_waitcnt vmcnt(29)\n" \
  "v_readfirstlane_b32 s79, v251\n" \
  "s_nop 0\n" \
  "s_cmp_eq_u32 s79, s76\n" \
  "s_cselect_b32 s78, 1, s78\n" \
  "s_cmp_eq_u32 s78, 0\n" \
  "s_mov_b32 s48, 0xc0135761\n" \
  "s_cselect_b32 s48, s48, 0x7fc00000\n" \
  "v_mov_b32 v230, 0xbdd2d3e8\n" \
  "v_mov_b32 v231, 0xbdd2d3e8\n" \
  "v_mov_b32 v232, s48\n" \
  "v_mov_b32 v233, s48\n" \
  "v_accvgpr_read_b32 v182, a64\n" \
  "v_accvgpr_read_b32 v186, a65\n" \
  "v_accvgpr_read_b32 v190, a66\n" \
  "v_accvgpr_read_b32 v194, a67\n" \
  "v_accvgpr_read_b32 v183, a68\n" \
  "v_accvgpr_read_b32 v187, a69\n" \
  "v_accvgpr_read_b32 v191, a70\n" \
  "v_accvgpr_read_b32 v195, a71\n" \
  "v_accvgpr_read_b32 v184, a72\n" \
  "v_accvgpr_read_b32 v188, a73\n" \
  "v_accvgpr_read_b32 v192, a74\n" \
  "v_accvgpr_read_b32 v196, a75\n" \
  "v_accvgpr_read_b32 v185, a76\n" \
  "v_accvgpr_read_b32 v189, a77\n" \
  "v_accvgpr_read_b32 v193, a78\n" \
  "v_accvgpr_read_b32 v197, a79\n" \
  "s_waitcnt vmcnt(18)\n" \
  "v_accvgpr_read_b32 v166, a0\n" \
  "v_accvgpr_read_b32 v170, a1\n" \
  "v_accvgpr_read_b32 v174, a2\n" \
  "v_accvgpr_read_b32 v178, a3\n" \
  "v_accvgpr_read_b32 v167, a4\n" \
  "v_accvgpr_read_b32 v171, a5\n" \
  "v_accvgpr_read_b32 v175, a6\n" \
  "v_accvgpr_read_b32 v179, a7\n" \
  "v_accvgpr_read_b32 v168, a8\n" \
  "v_accvgpr_read_b32 v172, a9\n" \
  "v_accvgpr_read_b32 v176, a10\n" \
  "v_accvgpr_read_b32 v180, a11\n" \
  "v_accvgpr_read_b32 v169, a12\n" \
  "v_accvgpr_read_b32 v173, a13\n" \
  "v_accvgpr_read_b32 v177, a14\n" \
  "v_accvgpr_read_b32 v181, a15\n" \
  "v_pk_add_f32 v[166:167], v[166:167], v[182:183]\n" \
  "v_pk_add_f32 v[168:169], v[168:169], v[184:185]\n" \
  "v_pk_add_f32 v[170:171], v[170:171], v[186:187]\n" \
  "v_pk_add_f32 v[172:173], v[172:173], v[188:189]\n" \
  "v_pk_add_f32 v[174:175], v[174:175], v[190:191]\n" \
  "v_pk_add_f32 v[176:177], v[176:177], v[192:193]\n" \
  "v_pk_add_f32 v[178:179], v[178:179], v[194:195]\n" \
  "v_pk_add_f32 v[180:181], v[180:181], v[196:197]\n" \
  "v_pk_mul_f32 v[182:183], v[166:167], v[166:167]\n" \
  "v_pk_mul_f32 v[184:185], v[168:169], v[168:169]\n" \
  "v_pk_mul_f32 v[186:187], v[170:171], v[170:171]\n" \
  "v_pk_mul_f32 v[188:189], v[172:173], v[172:173]\n" \
  "v_pk_mul_f32 v[190:191], v[174:175], v[174:175]\n" \
  "v_pk_mul_f32 v[192:193], v[176:177], v[176:177]\n" \
  "v_pk_mul_f32 v[194:195], v[178:179], v[178:179]\n" \
  "v_pk_mul_f32 v[196:197], v[180:181], v[180:181]\n" \
  "v_pk_fma_f32 v[182:183], v[182:183], v[230:231], v[232:233]\n" \
  "v_pk_fma_f32 v[184:185], v[184:185], v[230:231], v[232:233]\n" \
  "v_pk_fma_f32 v[186:187], v[186:187], v[230:231], v[232:233]\n" \
  "v_pk_fma_f32 v[188:189], v[188:189], v[230:231], v[232:233]\n" \
  "v_pk_fma_f32 v[190:191], v[190:191], v[230:231], v[232:233]\n" \
  "v_pk_fma_f32 v[192:193], v[192:193], v[230:231], v[232:233]\n" \
  "v_pk_fma_f32 v[194:195], v[194:195], v[230:231], v[232:233]\n" \
  "v_pk_fma_f32 v[196:197], v[196:197], v[230:231], v[232:233]\n" \
  "v_pk_mul_f32 v[182:183], v[182:183], v[166:167]\n" \
  "v_pk_mul_f32 v[184:185], v[184:185], v[168:169]\n" \
  "v_pk_mul_f32 v[186:187], v[186:187], v[170:171]\n" \
  "v_pk_mul_f32 v[188:189], v[188:189], v[172:173]\n" \
  "v_pk_mul_f32 v[190:191], v[190:191], v[174:175]\n" \
  "v_pk_mul_f32 v[192:193], v[192:193], v[176:177]\n" \
  "v_pk_mul_f32 v[194:195], v[194:195], v[178:179]\n" \
  "v_pk_mul_f32 v[196:197], v[196:197], v[180:181]\n" \
  "v_exp_f32 v182, v182\n" \
  "v_exp_f32 v183, v183\n" \
  "v_exp_f32 v184, v184\n" \
  "v_exp_f32 v185, v185\n" \
  "v_exp_f32 v186, v186\n" \
  "v_exp_f32 v187, v187\n" \
  "v_exp_f32 v188, v188\n" \
  "v_exp_f32 v189, v189\n" \
  "v_exp_f32 v190, v190\n" \
  "v_exp_f32 v191, v191\n" \
  "v_exp_f32 v192, v192\n" \
  "v_exp_f32 v193, v193\n" \
  "v_exp_f32 v194, v194\n" \
  "v_exp_f32 v195, v195\n" \
  "v_exp_f32 v196, v196\n" \
  "v_exp_f32 v197, v197\n" \
  "v_pk_add_f32 v[182:183], v[182:183], 1.0 op_sel_hi:[1,0]\n" \
  "v_pk_add_f32 v[184:185], v[184:185], 1.0 op_sel_hi:[1,0]\n" \
  "v_pk_add_f32 v[186:187], v[186:187], 1.0 op_sel_hi:[1,0]\n" \
  "v_pk_add_f32 v[188:189], v[188:189], 1.0 op_sel_hi:[1,0]\n" \
  "v_pk_add_f32 v[190:191], v[190:191], 1.0 op_sel_hi:[1,0]\n" \
  "v_pk_add_f32 v[192:193], v[192:193], 1.0 op_sel_hi:[1,0]\n" \
  "v_pk_add_f32 v[194:195], v[194:195], 1.0 op_sel_hi:[1,0]\n" \
  "v_pk_add_f32 v[196:197], v[196:197], 1.0 op_sel_hi:[1,0]\n" \
  "v_rcp_f32 v182, v182\n" \
  "v_rcp_f32 v183, v183\n" \
  "v_rcp_f32 v184, v184\n" \
  "v_rcp_f32 v185, v185\n" \
  "v_rcp_f32 v186, v186\n" \
  "v_rcp_f32 v187, v187\n" \
  "v_rcp_f32 v188, v188\n" \
  "v_rcp_f32 v189, v189\n" \
  "v_rcp_f32 v190, v190\n" \
  "v_rcp_f32 v191, v191\n" \
  "v_rcp_f32 v192, v192\n" \
  "v_rcp_f32 v193, v193\n" \
  "v_rcp_f32 v194, v194\n" \
  "v_rcp_f32 v195, v195\n" \
  "v_rcp_f32 v196, v196\n" \
  "v_rcp_f32 v197, v197\n" \
  "v_pk_mul_f32 v[166:167], v[166:167], v[182:183]\n" \
  "v_pk_mul_f32 v[168:169], v[168:169], v[184:185]\n" \
  "v_pk_mul_f32 v[170:171], v[170:171], v[186:187]\n" \
  "v_pk_mul_f32 v[172:173], v[172:173], v[188:189]\n" \
  "v_pk_mul_f32 v[174:175], v[174:175], v[190:191]\n" \
  "v_pk_mul_f32 v[176:177], v[176:177], v[192:193]\n" \
  "v_pk_mul_f32 v[178:179], v[178:179], v[194:195]\n" \
  "v_pk_mul_f32 v[180:181], v[180:181], v[196:197]\n" \
  "s_add_u32 s62, s91, 0x0\n" \
  "buffer_store_dwordx4 v[166:169], %[voffO], s[44:47], s62 offen offset:0 sc1\n" \
  "s_add_u32 s62, s91, 0x10000\n" \
  "buffer_store_dwordx4 v[170:173], %[voffO], s[44:47], s62 offen offset:0 sc1\n" \
  "s_add_u32 s62, s91, 0x20000\n" \
  "buffer_store_dwordx4 v[174:177], %[voffO], s[44:47], s62 offen offset:0 sc1\n" \
  "s_add_u32 s62, s91, 0x30000\n" \
  "buffer_store_dwordx4 v[178:181], %[voffO], s[44:47], s62 offen offset:0 sc1\n" \
  "v_mfma_f32_16x16x32_f16 a[240:243], v[86:89], v[118:121], a[240:243]\n" \
  "s_waitcnt lgkmcnt(13)\n" \
  "ds_read_b128 v[118:121], v246 offset:36864\n" \
  "v_mfma_f32_16x16x32_f16 a[244:247], v[86:89], v[122:125], a[244:247]\n" \
  "v_mfma_f32_16x16x32_f16 a[244:247], v[90:93], v[154:157], a[244:247]\n" \
  "v_mfma_f32_16x16x32_f16 a[212:215], v[94:97], v[154:157], a[212:215]\n" \
  "ds_read_b128 v[154:157], v246 offset:46080\n" \
  "v_mfma_f32_16x16x32_f16 a[212:215], v[98:101], v[122:125], a[212:215]\n" \
  "s_waitcnt lgkmcnt(13)\n" \
  "ds_read_b128 v[122:125], v246 offset:37888\n" \
  "v_mfma_f32_16x16x32_f16 a[216:219], v[98:101], v[126:129], a[216:219]\n" \
  "v_mfma_f32_16x16x32_f16 a[216:219], v[94:97], v[158:161], a[216:219]\n" \
  "buffer_store_dwordx4 a[176:179], %[voffO], s[44:47], s83 offen offset:256 sc1\n" \
  "v_mfma_f32_16x16x32_f16 a[248:251], v[90:93], v[158:161], a[248:251]\n" \
  "ds_read_b128 v[158:161], v246 offset:47104\n" \
  "v_mfma_f32_16x16x32_f16 a[248:251], v[86:89], v[126:129], a[248:251]\n" \
  "s_waitcnt lgkmcnt(13)\n" \
  "ds_read_b128 v[126:129], v246 offset:38912\n" \
  "v_mfma_f32_16x16x32_f16 a[252:255], v[86:89], v[130:133], a[252:255]\n" \
  "buffer_load_dwordx4 v[86:89], %[bxr], s[40:43], s72 offen offset:2048 sc0 sc1\n" \
  "v_mfma_f32_16x16x32_f16 a[252:255], v[90:93], v[162:165], a[252:255]\n" \
  "v_readlane_b32 s74, %[vtabA], 29\n" \
  "v_mfma_f32_16x16x32_f16 a[220:223], v[94:97], v[162:165], a[220:223]\n" \
  "ds_read_b128 v[162:165], v246 offset:48128\n" \
  "v_mfma_f32_16x16x32_f16 a[220:223], v[98:101], v[130:133], a[220:223]\n" \
  "ds_read_b128 v[130:133], v246 offset:39936\n" \
  "s_waitcnt lgkmcnt(14)\n" \
  "v_mfma_f32_16x16x32_f16 a[192:195], v[14:17], v[102:105], a[192:195]\n" \
  "buffer_load_dwordx4 v[90:93], %[bxr], s[40:43], s72 offen offset:3072 sc0 sc1\n" \
  "v_mfma_f32_16x16x32_f16 a[192:195], v[18:21], v[134:137], a[192:195]\n" \
  "s_add_u32 s73, s74, s71\n" \
  "v_mfma_f32_16x16x32_f16 a[224:227], v[26:29], v[134:137], a[224:227]\n" \
  "ds_read_b128 v[134:137], v246 offset:57344\n" \
  "v_mfma_f32_16x16x32_f16 a[224:227], v[22:25], v[102:105], a[224:227]\n" \
  "s_waitcnt lgkmcnt(13)\n" \
  "ds_read_b128 v[102:105], v246 offset:49152\n" \
  "v_mfma_f32_16x16x32_f16 a[228:231], v[22:25], v[106:109], a[228:231]\n" \
  "buffer_load_dwordx4 v[94:97], %[bxr], s[40:43], s73 offen offset:0 sc0 sc1\n" \
  "v_mfma_f32_16x16x32_f16 a[228:231], v[26:29], v[138:141], a[228:231]\n" \
  "v_mfma_f32_16x16x32_f16 a[196:199], v[18:21], v[138:141], a[196:199]\n" \
  "ds_read_b128 v[138:141], v246 offset:58368\n" \
  "v_mfma_f32_16x16x32_f16 a[196:199], v[14:17], v[106:109], a[196:199]\n" \
  "s_waitcnt lgkmcnt(13)\n" \
  "ds_read_b128 v[106:109], v246 offset:50176\n" \
  "v_mfma_f32_16x16x32_f16 a[200:203], v[14:17], v[110:113], a[200:203]\n" \
  "buffer_load_dwordx4 v[98:101], %[bxr], s[40:43], s73 offen offset:1024 sc0 sc1\n" \
  "v_mfma_f32_16x16x32_f16 a[200:203], v[18:21], v[142:145], a[200:203]\n" \
  "s_add_u32 s84, s90, 0x510000\n" \
  "v_mfma_f32_16x16x32_f16 a[232:235], v[26:29], v[142:145], a[232:235]\n" \
  "ds_read_b128 v[142:145], v246 offset:59392\n" \
  "v_mfma_f32_16x16x32_f16 a[232:235], v[22:25], v[110:113], a[232:235]\n" \
  "s_waitcnt lgkmcnt(13)\n" \
  "ds_read_b128 v[110:113], v246 offset:51200\n" \
  "v_accvgpr_read_b32 v214, a80\n" \
  "v_accvgpr_read_b32 v218, a81\n" \
  "v_accvgpr_read_b32 v222, a82\n" \
  "v_accvgpr_read_b32 v226, a83\n" \
  "v_accvgpr_read_b32 v215, a84\n" \
  "v_accvgpr_read_b32 v219, a85\n" \
  "v_accvgpr_read_b32 v223, a86\n" \
  "v_accvgpr_read_b32 v227, a87\n" \
  "v_accvgpr_read_b32 v216, a88\n" \
  "v_accvgpr_read_b32 v220, a89\n" \
  "v_accvgpr_read_b32 v224, a90\n" \
  "v_accvgpr_read_b32 v228, a91\n" \
  "v_accvgpr_read_b32 v217, a92\n" \
  "v_accvgpr_read_b32 v221, a93\n" \
  "v_accvgpr_read_b32 v225, a94\n" \
  "v_accvgpr_read_b32 v229, a95\n" \
  "s_waitcnt vmcnt(25)\n" \
  "v_accvgpr_read_b32 v198, a16\n" \
  "v_accvgpr_read_b32 v202, a17\n" \
  "v_accvgpr_read_b32 v206, a18\n" \
  "v_accvgpr_read_b32 v210, a19\n" \
  "v_accvgpr_read_b32 v199, a20\n" \
  "v_accvgpr_read_b32 v203, a21\n" \
  "v_accvgpr_read_b32 v207, a22\n" \
  "v_accvgpr_read_b32 v211, a23\n" \
  "v_accvgpr_read_b32 v200, a24\n" \
  "v_accvgpr_read_b32 v204, a25\n" \
  "v_accvgpr_read_b32 v208, a26\n" \
  "v_accvgpr_read_b32 v212, a27\n" \
  "v_accvgpr_read_b32 v201, a28\n" \
  "v_accvgpr_read_b32 v205, a29\n" \
  "v_accvgpr_read_b32 v209, a30\n" \
  "v_accvgpr_read_b32 v213, a31\n" \
  "v_pk_add_f32 v[198:199], v[198:199], v[214:215]\n" \
  "v_pk_add_f32 v[200:201], v[200:201], v[216:217]\n" \
  "v_pk_add_f32 v[202:203], v[202:203], v[218:219]\n" \
  "v_pk_add_f32 v[204:205], v[204:205], v[220:221]\n" \
  "v_pk_add_f32 v[206:207], v[206:207], v[222:223]\n" \
  "v_pk_add_f32 v[208:209], v[208:209], v[224:225]\n" \
  "v_pk_add_f32 v[210:211], v[210:211], v[226:227]\n" \
  "v_pk_add_f32 v[212:213], v[212:213], v[228:229]\n" \
  "v_pk_mul_f32 v[214:215], v[198:199], v[198:199]\n" \
  "v_pk_mul_f32 v[216:217], v[200:201], v[200:201]\n" \
  "v_pk_mul_f32 v[218:219], v[202:203], v[202:203]\n" \
  "v_pk_mul_f32 v[220:221], v[204:205], v[204:205]\n" \
  "v_pk_mul_f32 v[222:223], v[206:207], v[206:207]\n" \
  "v_pk_mul_f32 v[224:225], v[208:209], v[208:209]\n" \
  "v_pk_mul_f32 v[226:227], v[210:211], v[210:211]\n" \
  "v_pk_mul_f32 v[228:229], v[212:213], v[212:213]\n" \
  "v_pk_fma_f32 v[214:215], v[214:215], v[230:231], v[232:233]\n" \
  "v_pk_fma_f32 v[216:217], v[216:217], v[230:231], v[232:233]\n" \
  "v_pk_fma_f32 v[218:219], v[218:219], v[230:231], v[232:233]\n" \
  "v_pk_fma_f32 v[220:221], v[220:221], v[230:231], v[232:233]\n" \
  "v_pk_fma_f32 v[222:223], v[222:223], v[230:231], v[232:233]\n" \
  "v_pk_fma_f32 v[224:225], v[224:225], v[230:231], v[232:233]\n" \
  "v_pk_fma_f32 v[226:227], v[226:227], v[230:231], v[232:233]\n" \
  "v_pk_fma_f32 v[228:229], v[228:229], v[230:231], v[232:233]\n" \
  "v_pk_mul_f32 v[214:215], v[214:215], v[198:199]\n" \
  "v_pk_mul_f32 v[216:217], v[216:217], v[200:201]\n" \
  "v_pk_mul_f32 v[218:219], v[218:219], v[202:203]\n" \
  "v_pk_mul_f32 v[220:221], v[220:221], v[204:205]\n" \
  "v_pk_mul_f32 v[222:223], v[222:223], v[206:207]\n" \
  "v_pk_mul_f32 v[224:225], v[224:225], v[208:209]\n" \
  "v_pk_mul_f32 v[226:227], v[226:227], v[210:211]\n" \
  "v_pk_mul_f32 v[228:229], v[228:229], v[212:213]\n" \
  "v_exp_f32 v214, v214\n" \
  "v_exp_f32 v215, v215\n" \
  "v_exp_f32 v216, v216\n" \
  "v_exp_f32 v217, v217\n" \
  "v_exp_f32 v218, v218\n" \
  "v_exp_f32 v219, v219\n" \
  "v_exp_f32 v220, v220\n" \
  "v_exp_f32 v221, v221\n" \
  "v_exp_f32 v222, v222\n" \
  "v_exp_f32 v223, v223\n" \
  "v_exp_f32 v224, v224\n" \
  "v_exp_f32 v225, v225\n" \
  "v_exp_f32 v226, v226\n" \
  "v_exp_f32 v227, v227\n" \
  "v_exp_f32 v228, v228\n" \
  "v_exp_f32 v229, v229\n" \
  "v_pk_add_f32 v[214:215], v[214:215], 1.0 op_sel_hi:[1,0]\n" \
  "v_pk_add_f32 v[216:217], v[216:217], 1.0 op_sel_hi:[1,0]\n" \
  "v_pk_add_f32 v[218:219], v[218:219], 1.0 op_sel_hi:[1,0]\n" \
  "v_pk_add_f32 v[220:221], v[220:221], 1.0 op_sel_hi:[1,0]\n" \
  "v_pk_add_f32 v[222:223], v[222:223], 1.0 op_sel_hi:[1,0]\n" \
  "v_pk_add_f32 v[224:225], v[224:225], 1.0 op_sel_hi:[1,0]\n" \
  "v_pk_add_f32 v[226:227], v[226:227], 1.0 op_sel_hi:[1,0]\n" \
  "v_pk_add_f32 v[228:229], v[228:229], 1.0 op_sel_hi:[1,0]\n" \
  "v_rcp_f32 v214, v214\n" \
  "v_rcp_f32 v215, v215\n" \
  "v_rcp_f32 v216, v216\n" \
  "v_rcp_f32 v217, v217\n" \
  "v_rcp_f32 v218, v218\n" \
  "v_rcp_f32 v219, v219\n" \
  "v_rcp_f32 v220, v220\n" \
  "v_rcp_f32 v221, v221\n" \
  "v_rcp_f32 v222, v222\n" \
  "v_rcp_f32 v223, v223\n" \
  "v_rcp_f32 v224, v224\n" \
  "v_rcp_f32 v225, v225\n" \
  "v_rcp_f32 v226, v226\n" \
  "v_rcp_f32 v227, v227\n" \
  "v_rcp_f32 v228, v228\n" \
  "v_rcp_f32 v229, v229\n" \
  "v_pk_mul_f32 v[198:199], v[198:199], v[214:215]\n" \
  "v_pk_mul_f32 v[200:201], v[200:201], v[216:217]\n" \
  "v_pk_mul_f32 v[202:203], v[202:203], v[218:219]\n" \
  "v_pk_mul_f32 v[204:205], v[204:205], v[220:221]\n" \
  "v_pk_mul_f32 v[206:207], v[206:207], v[222:223]\n" \
  "v_pk_mul_f32 v[208:209], v[208:209], v[224:225]\n" \
  "v_pk_mul_f32 v[210:211], v[210:211], v[226:227]\n" \
  "v_pk_mul_f32 v[212:213], v[212:213], v[228:229]\n" \
  "s_add_u32 s62, s91, 0x0\n" \
  "buffer_store_dwordx4 v[198:201], %[voffO], s[44:47], s62 offen offset:256 sc1\n" \
  "s_add_u32 s62, s91, 0x10000\n" \
  "buffer_store_dwordx4 v[202:205], %[voffO], s[44:47], s62 offen offset:256 sc1\n" \
  "s_add_u32 s62, s91, 0x20000\n" \
  "buffer_store_dwordx4 v[206:209], %[voffO], s[44:47], s62 offen offset:256 sc1\n" \
  "s_add_u32 s62, s91, 0x30000\n" \
  "buffer_store_dwordx4 v[210:213], %[voffO], s[44:47], s62 offen offset:256 sc1\n" \
  "v_mfma_f32_16x16x32_f16 a[236:239], v[22:25], v[114:117], a[236:239]\n" \
  "v_mfma_f32_16x16x32_f16 a[236:239], v[26:29], v[146:149], a[236:239]\n" \
  "v_mfma_f32_16x16x32_f16 a[204:207], v[18:21], v[146:149], a[204:207]\n" \
  "ds_read_b128 v[146:149], v246 offset:60416\n" \
  "v_mfma_f32_16x16x32_f16 a[204:207], v[14:17], v[114:117], a[204:207]\n" \
  "s_waitcnt lgkmcnt(13)\n" \
  "ds_read_b128 v[114:117], v246 offset:52224\n" \
  "v_mfma_f32_16x16x32_f16 a[208:211], v[14:17], v[118:121], a[208:211]\n" \
  "v_mfma_f32_16x16x32_f16 a[208:211], v[18:21], v[150:153], a[208:211]\n" \
  "buffer_store_dwordx4 a[164:167], %[voffO], s[44:47], s84 offen offset:0 sc1\n" \
  "v_mfma_f32_16x16x32_f16 a[240:243], v[26:29], v[150:153], a[240:243]\n" \
  "ds_read_b128 v[150:153], v246 offset:61440\n" \
  "v_mfma_f32_16x16x32_f16 a[240:243], v[22:25], v[118:121], a[240:243]\n" \
  "s_waitcnt lgkmcnt(13)\n" \
  "ds_read_b128 v[118:121], v246 offset:53248\n" \
  "v_mfma_f32_16x16x32_f16 a[244:247], v[22:25], v[122:125], a[244:247]\n" \
  "v_mfma_f32_16x16x32_f16 a[244:247], v[26:29], v[154:157], a[244:247]\n" \
  "v_mfma_f32_16x16x32_f16 a[212:215], v[18:21], v[154:157], a[212:215]\n" \
  "ds_read_b128 v[154:157], v246 offset:62464\n" \
  "v_mfma_f32_16x16x32_f16 a[212:215], v[14:17], v[122:125], a[212:215]\n" \
  "s_waitcnt lgkmcnt(13)\n" \
  "ds_read_b128 v[122:125], v246 offset:54272\n" \
  "v_mfma_f32_16x16x32_f16 a[216:219], v[14:17], v[126:129], a[216:219]\n" \
  "s_add_u32 s60, s91, 0x100000\n" \
  "v_mfma_f32_16x16x32_f16 a[216:219], v[18:21], v[158:161], a[216:219]\n" \
  "v_mfma_f32_16x16x32_f16 a[248:251], v[26:29], v[158:161], a[248:251]\n" \
  "ds_read_b128 v[158:161], v246 offset:63488\n" \
  "s_waitcnt vmcnt(25)\n" \
  "v_readfirstlane_b32 s79, v239\n" \
  "s_nop 0\n" \
  "s_cmp_ge_u32 s79, s76\n" \
  "s_cbranch_scc0 L_s92x_%=\n" \
  "L_b92x_%=:\n" \
  "global_load_dword v251, v249, %[wsp] sc1\n" \
  "v_mfma_f32_16x16x32_f16 a[248:251], v[22:25], v[126:129], a[248:251]\n" \
  "s_waitcnt lgkmcnt(13)\n" \
  "ds_read_b128 v[126:129], v246 offset:55296\n" \
  "v_mfma_f32_16x16x32_f16 a[252:255], v[22:25], v[130:133], a[252:255]\n" \
  "buffer_load_dwordx4 a[32:35], %[voffO], s[44:47], s60 offen offset:0 sc1\n" \
  "v_mfma_f32_16x16x32_f16 a[252:255], v[26:29], v[162:165], a[252:255]\n" \
  "buffer_load_dwordx4 v[22:25], %[bxr], s[40:43], s73 offen offset:2048 sc0 sc1\n" \
  "v_mfma_f32_16x16x32_f16 a[220:223], v[18:21], v[162:165], a[220:223]\n" \
  "ds_read_b128 v[162:165], v246 offset:64512\n" \
  "v_readlane_b32 s74, %[vtabA], 30\n" \
  "v_mfma_f32_16x16x32_f16 a[220:223], v[14:17], v[130:133], a[220:223]\n" \
  "ds_read_b128 v[130:133], v246 offset:56320\n" \
  "s_waitcnt lgkmcnt(14)\n" \
  "v_mfma_f32_16x16x32_f16 a[192:195], v[30:33], v[102:105], a[192:195]\n" \
  "buffer_load_dwordx4 a[48:51], %[voffO], s[44:47], s60 offen offset:256 sc1\n" \
  "v_mfma_f32_16x16x32_f16 a[192:195], v[34:37], v[134:137], a[192:195]\n" \
  "buffer_load_dwordx4 v[26:29], %[bxr], s[40:43], s73 offen offset:3072 sc0 sc1\n" \
  "v_mfma_f32_16x16x32_f16 a[224:227], v[42:45], v[134:137], a[224:227]\n" \
  "s_add_u32 s72, s74, s71\n" \
  "ds_read_b128 v[134:137], v247 offset:8192\n" \
  "s_add_u32 s61, s91, 0x110000\n" \
  "v_mfma_f32_16x16x32_f16 a[224:227], v[38:41], v[102:105], a[224:227]\n" \
  "s_waitcnt lgkmcnt(13)\n" \
  "ds_read_b128 v[102:105], v247 offset:0\n" \
  "v_mfma_f32_16x16x32_f16 a[228:231], v[38:41], v[106:109], a[228:231]\n" \
  "buffer_load_dwordx4 a[36:39], %[voffO], s[44:47], s61 offen offset:0 sc1\n" \
  "v_mfma_f32_16x16x32_f16 a[228:231], v[42:45], v[138:141], a[228:231]\n" \
  "buffer_load_dwordx4 v[18:21], %[bxr], s[40:43], s72 offen offset:0 sc0 sc1\n" \
  "v_mfma_f32_16x16x32_f16 a[196:199], v[34:37], v[138:141], a[196:199]\n" \
  "ds_read_b128 v[138:141], v247 offset:9216\n" \
  "v_mfma_f32_16x16x32_f16 a[196:199], v[30:33], v[106:109], a[196:199]\n" \
  "s_waitcnt lgkmcnt(13)\n" \
  "ds_read_b128 v[106:109], v247 offset:1024\n" \
  "v_mfma_f32_16x16x32_f16 a[200:203], v[30:33], v[110:113], a[200:203]\n" \
  "buffer_load_dwordx4 a[52:55], %[voffO], s[44:47], s61 offen offset:256 sc1\n" \
  "v_mfma_f32_16x16x32_f16 a[200:203], v[34:37], v[142:145], a[200:203]\n" \
  "buffer_store_dwordx4 a[180:183], %[voffO], s[44:47], s84 offen offset:256 sc1\n" \
  "v_mfma_f32_16x16x32_f16 a[232:235], v[42:45], v[142:145], a[232:235]\n" \
  "s_add_u32 s60, s91, 0x120000\n" \
  "ds_read_b128 v[142:145], v247 offset:10240\n" \
  "v_mfma_f32_16x16x32_f16 a[232:235], v[38:41], v[110:113], a[232:235]\n" \
  "s_waitcnt lgkmcnt(13)\n" \
  "ds_read_b128 v[110:113], v247 offset:2048\n" \
  "v_mfma_f32_16x16x32_f16 a[236:239], v[38:41], v[114:117], a[236:239]\n" \
  "buffer_load_dwordx4 a[40:43], %[voffO], s[44:47], s60 offen offset:0 sc1\n" \
  "v_mfma_f32_16x16x32_f16 a[236:239], v[42:45], v[146:149], a[236:239]\n" \
  "buffer_load_dwordx4 v[14:17], %[bxr], s[40:43], s72 offen offset:1024 sc0 sc1\n" \
  "s_waitcnt vmcnt(29)\n" \
  "v_mov_b32 v250, s76\n" \
  "s_nop 0\n" \
  "global_store_dword v249, v250, %[xfo] offset:8 sc1\n" \
  "v_mfma_f32_16x16x32_f16 a[204:207], v[34:37], v[146:149], a[204:207]\n" \
  "ds_read_b128 v[146:149], v247 offset:11264\n" \
  "v_mfma_f32_16x16x32_f16 a[204:207], v[30:33], v[114:117], a[204:207]\n" \
  "s_waitcnt lgkmcnt(13)\n" \
  "ds_read_b128 v[114:117], v247 offset:3072\n" \
  "v_mfma_f32_16x16x32_f16 a[208:211], v[30:33], v[118:121], a[208:211]\n" \
  "buffer_load_dwordx4 a[56:59], %[voffO], s[44:47], s60 offen offset:256 sc1\n" \
  "v_mfma_f32_16x16x32_f16 a[208:211], v[34:37], v[150:153], a[208:211]\n" \
  "s_add_u32 s61, s91, 0x130000\n" \
  "v_mfma_f32_16x16x32_f16 a[240:243], v[42:45], v[150:153], a[240:243]\n" \
  "ds_read_b128 v[150:153], v247 offset:12288\n" \
  "v_mfma_f32_16x16x32_f16 a[240:243], v[38:41], v[118:121], a[240:243]\n" \
  "s_waitcnt lgkmcnt(13)\n" \
  "ds_read_b128 v[118:121], v247 offset:4096\n" \
  "v_mfma_f32_16x16x32_f16 a[244:247], v[38:41], v[122:125], a[244:247]\n" \
  "buffer_load_dwordx4 a[44:47], %[voffO], s[44:47], s61 offen offset:0 sc1\n" \
  "v_mfma_f32_16x16x32_f16 a[244:247], v[42:45], v[154:157], a[244:247]\n" \
  "v_mfma_f32_16x16x32_f16 a[212:215], v[34:37], v[154:157], a[212:215]\n" \
  "ds_read_b128 v[154:157], v247 offset:13312\n" \
  "v_mfma_f32_16x16x32_f16 a[212:215], v[30:33], v[122:125], a[212:215]\n" \
  "s_waitcnt lgkmcnt(13)\n" \
  "ds_read_b128 v[122:125], v247 offset:5120\n" \
  "v_mfma_f32_16x16x32_f16 a[216:219], v[30:33], v[126:129], a[216:219]\n" \
  "buffer_load_dwordx4 a[60:63], %[voffO], s[44:47], s61 offen offset:256 sc1\n" \
  "v_mfma_f32_16x16x32_f16 a[216:219], v[34:37], v[158:161], a[216:219]\n" \
  "s_add_u32 s83, s90, 0x520000\n" \
  "v_mfma_f32_16x16x32_f16 a[248:251], v[42:45], v[158:161], a[248:251]\n" \
  "ds_read_b128 v[158:161], v247 offset:14336\n" \
  "v_mfma_f32_16x16x32_f16 a[248:251], v[38:41], v[126:129], a[248:251]\n" \
  "s_waitcnt lgkmcnt(13)\n" \
  "ds_read_b128 v[126:129], v247 offset:6144\n" \
  "v_mfma_f32_16x16x32_f16 a[252:255], v[38:41], v[130:133], a[252:255]\n" \
  "buffer_load_dwordx4 v[38:41], %[bxr], s[40:43], s72 offen offset:2048 sc0 sc1\n" \
  "v_mfma_f32_16x16x32_f16 a[252:255], v[42:45], v[162:165], a[252:255]\n" \
  "v_readlane_b32 s74, %[vtabA], 31\n" \
  "v_mfma_f32_16x16x32_f16 a[220:223], v[34:37], v[162:165], a[220:223]\n" \
  "ds_read_b128 v[162:165], v247 offset:15360\n" \
  "v_mfma_f32_16x16x32_f16 a[220:223], v[30:33], v[130:133], a[220:223]\n" \
  "ds_read_b128 v[130:133], v247 offset:7168\n" \
  "s_waitcnt lgkmcnt(14)\n" \
  "v_mfma_f32_16x16x32_f16 a[192:195], v[46:49], v[102:105], a[192:195]\n" \
  "buffer_load_dwordx4 v[42:45], %[bxr], s[40:43], s72 offen offset:3072 sc0 sc1\n" \
  "v_mfma_f32_16x16x32_f16 a[192:195], v[50:53], v[134:137], a[192:195]\n" \
  "buffer_store_dwordx4 a[168:171], %[voffO], s[44:47], s83 offen offset:0 sc1\n" \
  "v_mfma_f32_16x16x32_f16 a[224:227], v[58:61], v[134:137], a[224:227]\n" \
  "s_add_u32 s73, s74, s71\n" \
  "ds_read_b128 v[134:137], v247 offset:24576\n" \
  "v_mfma_f32_16x16x32_f16 a[224:227], v[54:57], v[102:105], a[224:227]\n" \
  "s_waitcnt lgkmcnt(13)\n" \
  "ds_read_b128 v[102:105], v247 offset:16384\n" \
  "v_mfma_f32_16x16x32_f16 a[228:231], v[54:57], v[106:109], a[228:231]\n" \
  "buffer_load_dwordx4 v[34:37], %[bxr], s[40:43], s73 offen offset:0 sc0 sc1\n" \
  "v_mfma_f32_16x16x32_f16 a[228:231], v[58:61], v[138:141], a[228:231]\n" \
  "v_mfma_f32_16x16x32_f16 a[196:199], v[50:53], v[138:141], a[196:199]\n" \
  "ds_read_b128 v[138:141], v247 offset:25600\n" \
  "v_mfma_f32_16x16x32_f16 a[196:199], v[46:49], v[106:109], a[196:199]\n" \
  "s_waitcnt lgkmcnt(13)\n" \
  "ds_read_b128 v[106:109], v247 offset:17408\n" \
  "v_mfma_f32_16x16x32_f16 a[200:203], v[46:49], v[110:113], a[200:203]\n" \
  "buffer_load_dwordx4 v[30:33], %[bxr], s[40:43], s73 offen offset:1024 sc0 sc1\n" \
  "v_mfma_f32_16x16x32_f16 a[200:203], v[50:53], v[142:145], a[200:203]\n" \
  "v_mfma_f32_16x16x32_f16 a[232:235], v[58:61], v[142:145], a[232:235]\n" \
  "ds_read_b128 v[142:145], v247 offset:26624\n" \
  "v_mfma_f32_16x16x32_f16 a[232:235], v[54:57], v[110:113], a[232:235]\n" \
  "s_waitcnt lgkmcnt(13)\n" \
  "ds_read_b128 v[110:113], v247 offset:18432\n" \
  "v_mfma_f32_16x16x32_f16 a[236:239], v[54:57], v[114:117], a[236:239]\n" \
  "v_mfma_f32_16x16x32_f16 a[236:239], v[58:61], v[146:149], a[236:239]\n" \
  "v_mfma_f32_16x16x32_f16 a[204:207], v[50:53], v[146:149], a[204:207]\n" \
  "ds_read_b128 v[146:149], v247 offset:27648\n" \
  "v_mfma_f32_16x16x32_f16 a[204:207], v[46:49], v[114:117], a[204:207]\n" \
  "s_waitcnt lgkmcnt(13)\n" \
  "ds_read_b128 v[114:117], v247 offset:19456\n" \
  "v_mfma_f32_16x16x32_f16 a[208:211], v[46:49], v[118:121], a[208:211]\n" \
  "v_mfma_f32_16x16x32_f16 a[208:211], v[50:53], v[150:153], a[208:211]\n" \
  "v_mfma_f32_16x16x32_f16 a[240:243], v[58:61], v[150:153], a[240:243]\n" \
  "ds_read_b128 v[150:153], v247 offset:28672\n" \
  "v_mfma_f32_16x16x32_f16 a[240:243], v[54:57], v[118:121], a[240:243]\n" \
  "s_waitcnt lgkmcnt(13)\n" \
  "ds_read_b128 v[118:121], v247 offset:20480\n" \
  "v_mfma_f32_16x16x32_f16 a[244:247], v[54:57], v[122:125], a[244:247]\n" \
  "v_mfma_f32_16x16x32_f16 a[244:247], v[58:61], v[154:157], a[244:247]\n" \
  "v_mfma_f32_16x16x32_f16 a[212:215], v[50:53], v[154:157], a[212:215]\n" \
  "ds_read_b128 v[154:157], v247 offset:29696\n" \
  "v_mfma_f32_16x16x32_f16 a[212:215], v[46:49], v[122:125], a[212:215]\n" \
  "s_waitcnt lgkmcnt(13)\n" \
  "ds_read_b128 v[122:125], v247 offset:21504\n" \
  "v_mfma_f32_16x16x32_f16 a[216:219], v[46:49], v[126:129], a[216:219]\n" \
  "v_mfma_f32_16x16x32_f16 a[216:219], v[50:53], v[158:161], a[216:219]\n" \
  "buffer_store_dwordx4 a[184:187], %[voffO], s[44:47], s83 offen offset:256 sc1\n" \
  "v_mfma_f32_16x16x32_f16 a[248:251], v[58:61], v[158:161], a[248:251]\n" \
  "ds_read_b128 v[158:161], v247 offset:30720\n" \
  "v_mfma_f32_16x16x32_f16 a[248:251], v[54:57], v[126:129], a[248:251]\n" \
  "s_waitcnt lgkmcnt(13)\n" \
  "ds_read_b128 v[126:129], v247 offset:22528\n" \
  "v_mfma_f32_16x16x32_f16 a[252:255], v[54:57], v[130:133], a[252:255]\n" \
  "buffer_load_dwordx4 v[54:57], %[bxr], s[40:43], s73 offen offset:2048 sc0 sc1\n" \
  "v_mfma_f32_16x16x32_f16 a[252:255], v[58:61], v[162:165], a[252:255]\n" \
  "v_mfma_f32_16x16x32_f16 a[220:223], v[50:53], v[162:165], a[220:223]\n" \
  "ds_read_b128 v[162:165], v247 offset:31744\n" \
  "v_mfma_f32_16x16x32_f16 a[220:223], v[46:49], v[130:133], a[220:223]\n" \
  "s_waitcnt vmcnt(37)\n" \
  "ds_read_b128 v[130:133], v247 offset:23552\n" \
  "s_waitcnt lgkmcnt(14)\n" \
  "v_mfma_f32_16x16x32_f16 a[192:195], v[62:65], v[102:105], a[192:195]\n" \
  "buffer_load_dwordx4 v[58:61], %[bxr], s[40:43], s73 offen offset:3072 sc0 sc1\n" \
  "v_mfma_f32_16x16x32_f16 a[192:195], v[66:69], v[134:137], a[192:195]\n" \
  "global_load_dwordx4 v[238:241], v249, %[xfi] sc1\n" \
  "v_mfma_f32_16x16x32_f16 a[224:227], v[74:77], v[134:137], a[224:227]\n" \
  "ds_read_b128 v[134:137], v245 offset:8192\n" \
  "v_mfma_f32_16x16x32_f16 a[224:227], v[70:73], v[102:105], a[224:227]\n" \
  "s_waitcnt lgkmcnt(13)\n" \
  "ds_read_b128 v[102:105], v245 offset:0\n" \
  "v_mfma_f32_16x16x32_f16 a[228:231], v[70:73], v[106:109], a[228:231]\n" \
  "v_mfma_f32_16x16x32_f16 a[228:231], v[74:77], v[138:141], a[228:231]\n" \
  "v_mfma_f32_16x16x32_f16 a[196:199], v[66:69], v[138:141], a[196:199]\n" \
  "ds_read_b128 v[138:141], v245 offset:9216\n" \
  "v_mfma_f32_16x16x32_f16 a[196:199], v[62:65], v[106:109], a[196:199]\n" \
  "s_waitcnt lgkmcnt(13)\n" \
  "ds_read_b128 v[106:109], v245 offset:1024\n" \
  "v_mfma_f32_16x16x32_f16 a[200:203], v[62:65], v[110:113], a[200:203]\n" \
  "v_mfma_f32_16x16x32_f16 a[200:203], v[66:69], v[142:145], a[200:203]\n" \
  "s_add_u32 s84, s90, 0x530000\n" \
  "v_mfma_f32_16x16x32_f16 a[232:235], v[74:77], v[142:145], a[232:235]\n" \
  "ds_read_b128 v[142:145], v245 offset:10240\n" \
  "v_mfma_f32_16x16x32_f16 a[232:235], v[70:73], v[110:113], a[232:235]\n" \
  "s_waitcnt lgkmcnt(13)\n" \
  "ds_read_b128 v[110:113], v245 offset:2048\n" \
  "v_mfma_f32_16x16x32_f16 a[236:239], v[70:73], v[114:117], a[236:239]\n" \
  "v_mfma_f32_16x16x32_f16 a[236:239], v[74:77], v[146:149], a[236:239]\n" \
  "v_mfma_f32_16x16x32_f16 a[204:207], v[66:69], v[146:149], a[204:207]\n" \
  "ds_read_b128 v[146:149], v245 offset:11264\n" \
  "v_mfma_f32_16x16x32_f16 a[204:207], v[62:65], v[114:117], a[204:207]\n" \
  "s_waitcnt lgkmcnt(13)\n" \
  "ds_read_b128 v[114:117], v245 offset:3072\n" \
  "v_mfma_f32_16x16x32_f16 a[208:211], v[62:65], v[118:121], a[208:211]\n" \
  "v_mfma_f32_16x16x32_f16 a[208:211], v[66:69], v[150:153], a[208:211]\n" \
  "buffer_store_dwordx4 a[172:175], %[voffO], s[44:47], s84 offen offset:0 sc1\n" \
  "v_mfma_f32_16x16x32_f16 a[240:243], v[74:77], v[150:153], a[240:243]\n" \
  "ds_read_b128 v[150:153], v245 offset:12288\n" \
  "v_mfma_f32_16x16x32_f16 a[240:243], v[70:73], v[118:121], a[240:243]\n" \
  "s_waitcnt lgkmcnt(13)\n" \
  "ds_read_b128 v[118:121], v245 offset:4096\n" \
  "v_mfma_f32_16x16x32_f16 a[244:247], v[70:73], v[122:125], a[244:247]\n" \
  "v_mfma_f32_16x16x32_f16 a[244:247], v[74:77], v[154:157], a[244:247]\n" \
  "v_mfma_f32_16x16x32_f16 a[212:215], v[66:69], v[154:157], a[212:215]\n" \
  "ds_read_b128 v[154:157], v245 offset:13312\n" \
  "v_mfma_f32_16x16x32_f16 a[212:215], v[62:65], v[122:125], a[212:215]\n" \
  "s_waitcnt lgkmcnt(13)\n" \
  "ds_read_b128 v[122:125], v245 offset:5120\n" \
  "v_mfma_f32_16x16x32_f16 a[216:219], v[62:65], v[126:129], a[216:219]\n" \
  "v_mfma_f32_16x16x32_f16 a[216:219], v[66:69], v[158:161], a[216:219]\n" \
  "v_mfma_f32_16x16x32_f16 a[248:251], v[74:77], v[158:161], a[248:251]\n" \
  "ds_read_b128 v[158:161], v245 offset:14336\n" \
  "v_mfma_f32_16x16x32_f16 a[248:251], v[70:73], v[126:129], a[248:251]\n" \
  "s_waitcnt lgkmcnt(13)\n" \
  "ds_read_b128 v[126:129], v245 offset:6144\n" \
  "v_mfma_f32_16x16x32_f16 a[252:255], v[70:73], v[130:133], a[252:255]\n" \
  "v_mfma_f32_16x16x32_f16 a[252:255], v[74:77], v[162:165], a[252:255]\n" \
  "v_mfma_f32_16x16x32_f16 a[220:223], v[66:69], v[162:165], a[220:223]\n" \
  "ds_read_b128 v[162:165], v245 offset:15360\n" \
  "v_mfma_f32_16x16x32_f16 a[220:223], v[62:65], v[130:133], a[220:223]\n" \
  "s_waitcnt vmcnt(29)\n" \
  "ds_read_b128 v[130:133], v245 offset:7168\n" \
  "s_waitcnt lgkmcnt(14)\n" \
  "v_mfma_f32_16x16x32_f16 a[192:195], v[78:81], v[102:105], a[192:195]\n" \
  "v_mfma_f32_16x16x32_f16 a[192:195], v[82:85], v[134:137], a[192:195]\n" \
  "v_mfma_f32_16x16x32_f16 a[224:227], v[90:93], v[134:137], a[224:227]\n" \
  "ds_read_b128 v[134:137], v245 offset:24576\n" \
  "v_mfma_f32_16x16x32_f16 a[224:227], v[86:89], v[102:105], a[224:227]\n" \
  "s_waitcnt lgkmcnt(13)\n" \
  "ds_read_b128 v[102:105], v245 offset:16384\n" \
  "v_mfma_f32_16x16x32_f16 a[228:231], v[86:89], v[106:109], a[228:231]\n" \
  "v_mfma_f32_16x16x32_f16 a[228:231], v[90:93], v[138:141], a[228:231]\n" \
  "v_mfma_f32_16x16x32_f16 a[196:199], v[82:85], v[138:141], a[196:199]\n" \
  "ds_read_b128 v[138:141], v245 offset:25600\n" \
  "v_mfma_f32_16x16x32_f16 a[196:199], v[78:81], v[106:109], a[196:199]\n" \
  "s_waitcnt lgkmcnt(13)\n" \
  "ds_read_b128 v[106:109], v245 offset:17408\n" \
  "v_mfma_f32_16x16x32_f16 a[200:203], v[78:81], v[110:113], a[200:203]\n" \
  "s_waitcnt vmcnt(22)\n" \
  "v_readfirstlane_b32 s79, v251\n" \
  "s_nop 0\n" \
  "s_cmp_eq_u32 s79, s76\n" \
  "s_cselect_b32 s78, 1, s78\n" \
  "s_cmp_eq_u32 s78, 0\n" \
  "s_mov_b32 s48, 0xc0135761\n" \
  "s_cselect_b32 s48, s48, 0x7fc00000\n" \
  "v_mov_b32 v230, 0xbdd2d3e8\n" \
  "v_mov_b32 v231, 0xbdd2d3e8\n" \
  "v_mov_b32 v232, s48\n" \
  "v_mov_b32 v233, s48\n" \
  "v_accvgpr_read_b32 v182, a96\n" \
  "v_accvgpr_read_b32 v186, a97\n" \
  "v_accvgpr_read_b32 v190, a98\n" \
  "v_accvgpr_read_b32 v194, a99\n" \
  "v_accvgpr_read_b32 v183, a100\n" \
  "v_accvgpr_read_b32 v187, a101\n" \
  "v_accvgpr_read_b32 v191, a102\n" \
  "v_accvgpr_read_b32 v195, a103\n" \
  "v_accvgpr_read_b32 v184, a104\n" \
  "v_accvgpr_read_b32 v188, a105\n" \
  "v_accvgpr_read_b32 v192, a106\n" \
  "v_accvgpr_read_b32 v196, a107\n" \
  "v_accvgpr_read_b32 v185, a108\n" \
  "v_accvgpr_read_b32 v189, a109\n" \
  "v_accvgpr_read_b32 v193, a110\n" \
  "v_accvgpr_read_b32 v197, a111\n" \
  "s_waitcnt vmcnt(10)\n" \
  "v_accvgpr_read_b32 v166, a32\n" \
  "v_accvgpr_read_b32 v170, a33\n" \
  "v_accvgpr_read_b32 v174, a34\n" \
  "v_accvgpr_read_b32 v178, a35\n" \
  "v_accvgpr_read_b32 v167, a36\n" \
  "v_accvgpr_read_b32 v171, a37\n" \
  "v_accvgpr_read_b32 v175, a38\n" \
  "v_accvgpr_read_b32 v179, a39\n" \
  "v_accvgpr_read_b32 v168, a40\n" \
  "v_accvgpr_read_b32 v172, a41\n" \
  "v_accvgpr_read_b32 v176, a42\n" \
  "v_accvgpr_read_b32 v180, a43\n" \
  "v_accvgpr_read_b32 v169, a44\n" \
  "v_accvgpr_read_b32 v173, a45\n" \
  "v_accvgpr_read_b32 v177, a46\n" \
  "v_accvgpr_read_b32 v181, a47\n" \
  "v_pk_add_f32 v[166:167], v[166:167], v[182:183]\n" \
  "v_pk_add_f32 v[168:169], v[168:169], v[184:185]\n" \
  "v_pk_add_f32 v[170:171], v[170:171], v[186:187]\n" \
  "v_pk_add_f32 v[172:173], v[172:173], v[188:189]\n" \
  "v_pk_add_f32 v[174:175], v[174:175], v[190:191]\n" \
  "v_pk_add_f32 v[176:177], v[176:177], v[192:193]\n" \
  "v_pk_add_f32 v[178:179], v[178:179], v[194:195]\n" \
  "v_pk_add_f32 v[180:181], v[180:181], v[196:197]\n" \
  "v_pk_mul_f32 v[182:183], v[166:167], v[166:167]\n" \
  "v_pk_mul_f32 v[184:185], v[168:169], v[168:169]\n" \
  "v_pk_mul_f32 v[186:187], v[170:171], v[170:171]\n" \
  "v_pk_mul_f32 v[188:189], v[172:173], v[172:173]\n" \
  "v_pk_mul_f32 v[190:191], v[174:175], v[174:175]\n" \
  "v_pk_mul_f32 v[192:193], v[176:177], v[176:177]\n" \
  "v_pk_mul_f32 v[194:195], v[178:179], v[178:179]\n" \
  "v_pk_mul_f32 v[196:197], v[180:181], v[180:181]\n" \
  "v_pk_fma_f32 v[182:183], v[182:183], v[230:231], v[232:233]\n" \
  "v_pk_fma_f32 v[184:185], v[184:185], v[230:231], v[232:233]\n" \
  "v_pk_fma_f32 v[186:187], v[186:187], v[230:231], v[232:233]\n" \
  "v_pk_fma_f32 v[188:189], v[188:189], v[230:231], v[232:233]\n" \
  "v_pk_fma_f32 v[190:191], v[190:191], v[230:231], v[232:233]\n" \
  "v_pk_fma_f32 v[192:193], v[192:193], v[230:231], v[232:233]\n" \
  "v_pk_fma_f32 v[194:195], v[194:195], v[230:231], v[232:233]\n" \
  "v_pk_fma_f32 v[196:197], v[196:197], v[230:231], v[232:233]\n" \
  "v_pk_mul_f32 v[182:183], v[182:183], v[166:167]\n" \
  "v_pk_mul_f32 v[184:185], v[184:185], v[168:169]\n" \
  "v_pk_mul_f32 v[186:187], v[186:187], v[170:171]\n" \
  "v_pk_mul_f32 v[188:189], v[188:189], v[172:173]\n" \
  "v_pk_mul_f32 v[190:191], v[190:191], v[174:175]\n" \
  "v_pk_mul_f32 v[192:193], v[192:193], v[176:177]\n" \
  "v_pk_mul_f32 v[194:195], v[194:195], v[178:179]\n" \
  "v_pk_mul_f32 v[196:197], v[196:197], v[180:181]\n" \
  "v_exp_f32 v182, v182\n" \
  "v_exp_f32 v183, v183\n" \
  "v_exp_f32 v184, v184\n" \
  "v_exp_f32 v185, v185\n" \
  "v_exp_f32 v186, v186\n" \
  "v_exp_f32 v187, v187\n" \
  "v_exp_f32 v188, v188\n" \
  "v_exp_f32 v189, v189\n" \
  "v_exp_f32 v190, v190\n" \
  "v_exp_f32 v191, v191\n" \
  "v_exp_f32 v192, v192\n" \
  "v_exp_f32 v193, v193\n" \
  "v_exp_f32 v194, v194\n" \
  "v_exp_f32 v195, v195\n" \
  "v_exp_f32 v196, v196\n" \
  "v_exp_f32 v197, v197\n" \
  "v_pk_add_f32 v[182:183], v[182:183], 1.0 op_sel_hi:[1,0]\n" \
  "v_pk_add_f32 v[184:185], v[184:185], 1.0 op_sel_hi:[1,0]\n" \
  "v_pk_add_f32 v[186:187], v[186:187], 1.0 op_sel_hi:[1,0]\n" \
  "v_pk_add_f32 v[188:189], v[188:189], 1.0 op_sel_hi:[1,0]\n" \
  "v_pk_add_f32 v[190:191], v[190:191], 1.0 op_sel_hi:[1,0]\n" \
  "v_pk_add_f32 v[192:193], v[192:193], 1.0 op_sel_hi:[1,0]\n" \
  "v_pk_add_f32 v[194:195], v[194:195], 1.0 op_sel_hi:[1,0]\n" \
  "v_pk_add_f32 v[196:197], v[196:197], 1.0 op_sel_hi:[1,0]\n" \
  "v_rcp_f32 v182, v182\n" \
  "v_rcp_f32 v183, v183\n" \
  "v_rcp_f32 v184, v184\n" \
  "v_rcp_f32 v185, v185\n" \
  "v_rcp_f32 v186, v186\n" \
  "v_rcp_f32 v187, v187\n" \
  "v_rcp_f32 v188, v188\n" \
  "v_rcp_f32 v189, v189\n" \
  "v_rcp_f32 v190, v190\n" \
  "v_rcp_f32 v191, v191\n" \
  "v_rcp_f32 v192, v192\n" \
  "v_rcp_f32 v193, v193\n" \
  "v_rcp_f32 v194, v194\n" \
  "v_rcp_f32 v195, v195\n" \
  "v_rcp_f32 v196, v196\n" \
  "v_rcp_f32 v197, v197\n" \
  "v_pk_mul_f32 v[166:167], v[166:167], v[182:183]\n" \
  "v_pk_mul_f32 v[168:169], v[168:169], v[184:185]\n" \
  "v_pk_mul_f32 v[170:171], v[170:171], v[186:187]\n" \
  "v_pk_mul_f32 v[172:173], v[172:173], v[188:189]\n" \
  "v_pk_mul_f32 v[174:175], v[174:175], v[190:191]\n" \
  "v_pk_mul_f32 v[176:177], v[176:177], v[192:193]\n" \
  "v_pk_mul_f32 v[178:179], v[178:179], v[194:195]\n" \
  "v_pk_mul_f32 v[180:181], v[180:181], v[196:197]\n" \
  "s_add_u32 s62, s91, 0x100000\n" \
  "buffer_store_dwordx4 v[166:169], %[voffO], s[44:47], s62 offen offset:0 sc1\n" \
  "s_add_u32 s62, s91, 0x110000\n" \
  "buffer_store_dwordx4 v[170:173], %[voffO], s[44:47], s62 offen offset:0 sc1\n" \
  "s_add_u32 s62, s91, 0x120000\n" \
  "buffer_store_dwordx4 v[174:177], %[voffO], s[44:47], s62 offen offset:0 sc1\n" \
  "s_add_u32 s62, s91, 0x130000\n" \
  "buffer_store_dwordx4 v[178:181], %[voffO], s[44:47], s62 offen offset:0 sc1\n" \
  "v_mfma_f32_16x16x32_f16 a[200:203], v[82:85], v[142:145], a[200:203]\n" \
  "buffer_store_dwordx4 a[188:191], %[voffO], s[44:47], s84 offen offset:256 sc1\n" \
  "v_mfma_f32_16x16x32_f16 a[232:235], v[90:93], v[142:145], a[232:235]\n" \
  "ds_read_b128 v[142:145], v245 offset:26624\n" \
  "v_mfma_f32_16x16x32_f16 a[232:235], v[86:89], v[110:113], a[232:235]\n" \
  "s_waitcnt lgkmcnt(13)\n" \
  "ds_read_b128 v[110:113], v245 offset:18432\n" \
  "v_mfma_f32_16x16x32_f16 a[236:239], v[86:89], v[114:117], a[236:239]\n" \
  "v_mfma_f32_16x16x32_f16 a[236:239], v[90:93], v[146:149], a[236:239]\n" \
  "v_mfma_f32_16x16x32_f16 a[204:207], v[82:85], v[146:149], a[204:207]\n" \
  "ds_read_b128 v[146:149], v245 offset:27648\n" \
  "v_mfma_f32_16x16x32_f16 a[204:207], v[78:81], v[114:117], a[204:207]\n" \
  "s_waitcnt lgkmcnt(13)\n" \
  "ds_read_b128 v[114:117], v245 offset:19456\n" \
  "v_mfma_f32_16x16x32_f16 a[208:211], v[78:81], v[118:121], a[208:211]\n" \
  "v_mfma_f32_16x16x32_f16 a[208:211], v[82:85], v[150:153], a[208:211]\n" \
  "v_mfma_f32_16x16x32_f16 a[240:243], v[90:93], v[150:153], a[240:243]\n" \
  "ds_read_b128 v[150:153], v245 offset:28672\n" \
  "v_mfma_f32_16x16x32_f16 a[240:243], v[86:89], v[118:121], a[240:243]\n" \
  "s_waitcnt lgkmcnt(13)\n" \
  "ds_read_b128 v[118:121], v245 offset:20480\n" \
  "v_mfma_f32_16x16x32_f16 a[244:247], v[86:89], v[122:125], a[244:247]\n" \
  "v_mfma_f32_16x16x32_f16 a[244:247], v[90:93], v[154:157], a[244:247]\n" \
  "v_mfma_f32_16x16x32_f16 a[212:215], v[82:85], v[154:157], a[212:215]\n" \
  "ds_read_b128 v[154:157], v245 offset:29696\n" \
  "v_mfma_f32_16x16x32_f16 a[212:215], v[78:81], v[122:125], a[212:215]\n" \
  "s_waitcnt lgkmcnt(13)\n" \
  "ds_read_b128 v[122:125], v245 offset:21504\n" \
  "v_mfma_f32_16x16x32_f16 a[216:219], v[78:81], v[126:129], a[216:219]\n" \
  "v_mfma_f32_16x16x32_f16 a[216:219], v[82:85], v[158:161], a[216:219]\n" \
  "v_mfma_f32_16x16x32_f16 a[248:251], v[90:93], v[158:161], a[248:251]\n" \
  "ds_read_b128 v[158:161], v245 offset:30720\n" \
  "v_mfma_f32_16x16x32_f16 a[248:251], v[86:89], v[126:129], a[248:251]\n" \
  "s_waitcnt lgkmcnt(13)\n" \
  "ds_read_b128 v[126:129], v245 offset:22528\n" \
  "v_mfma_f32_16x16x32_f16 a[252:255], v[86:89], v[130:133], a[252:255]\n" \
  "v_mfma_f32_16x16x32_f16 a[252:255], v[90:93], v[162:165], a[252:255]\n" \
  "v_mfma_f32_16x16x32_f16 a[220:223], v[82:85], v[162:165], a[220:223]\n" \
  "ds_read_b128 v[162:165], v245 offset:31744\n" \
  "v_mfma_f32_16x16x32_f16 a[220:223], v[78:81], v[130:133], a[220:223]\n" \
  "ds_read_b128 v[130:133], v245 offset:23552\n" \
  "s_waitcnt lgkmcnt(14)\n" \
  "v_mfma_f32_16x16x32_f16 a[192:195], v[94:97], v[102:105], a[192:195]\n" \
  "v_mfma_f32_16x16x32_f16 a[192:195], v[98:101], v[134:137], a[192:195]\n" \
  "v_accvgpr_read_b32 v214, a112\n" \
  "v_accvgpr_read_b32 v218, a113\n" \
  "v_accvgpr_read_b32 v222, a114\n" \
  "v_accvgpr_read_b32 v226, a115\n" \
  "v_accvgpr_read_b32 v215, a116\n" \
  "v_accvgpr_read_b32 v219, a117\n" \
  "v_accvgpr_read_b32 v223, a118\n" \
  "v_accvgpr_read_b32 v227, a119\n" \
  "v_accvgpr_read_b32 v216, a120\n" \
  "v_accvgpr_read_b32 v220, a121\n" \
  "v_accvgpr_read_b32 v224, a122\n" \
  "v_accvgpr_read_b32 v228, a123\n" \
  "v_accvgpr_read_b32 v217, a124\n" \
  "v_accvgpr_read_b32 v221, a125\n" \
  "v_accvgpr_read_b32 v225, a126\n" \
  "v_accvgpr_read_b32 v229, a127\n" \
  "s_waitcnt vmcnt(14)\n" \
  "v_accvgpr_read_b32 v198, a48\n" \
  "v_accvgpr_read_b32 v202, a49\n" \
  "v_accvgpr_read_b32 v206, a50\n" \
  "v_accvgpr_read_b32 v210, a51\n" \
  "v_accvgpr_read_b32 v199, a52\n" \
  "v_accvgpr_read_b32 v203, a53\n" \
  "v_accvgpr_read_b32 v207, a54\n" \
  "v_accvgpr_read_b32 v211, a55\n" \
  "v_accvgpr_read_b32 v200, a56\n" \
  "v_accvgpr_read_b32 v204, a57\n" \
  "v_accvgpr_read_b32 v208, a58\n" \
  "v_accvgpr_read_b32 v212, a59\n" \
  "v_accvgpr_read_b32 v201, a60\n" \
  "v_accvgpr_read_b32 v205, a61\n" \
  "v_accvgpr_read_b32 v209, a62\n" \
  "v_accvgpr_read_b32 v213, a63\n" \
  "v_pk_add_f32 v[198:199], v[198:199], v[214:215]\n" \
  "v_pk_add_f32 v[200:201], v[200:201], v[216:217]\n" \
  "v_pk_add_f32 v[202:203], v[202:203], v[218:219]\n" \
  "v_pk_add_f32 v[204:205], v[204:205], v[220:221]\n" \
  "v_pk_add_f32 v[206:207], v[206:207], v[222:223]\n" \
  "v_pk_add_f32 v[208:209], v[208:209], v[224:225]\n" \
  "v_pk_add_f32 v[210:211], v[210:211], v[226:227]\n" \
  "v_pk_add_f32 v[212:213], v[212:213], v[228:229]\n" \
  "v_pk_mul_f32 v[214:215], v[198:199], v[198:199]\n" \
  "v_pk_mul_f32 v[216:217], v[200:201], v[200:201]\n" \
  "v_pk_mul_f32 v[218:219], v[202:203], v[202:203]\n" \
  "v_pk_mul_f32 v[220:221], v[204:205], v[204:205]\n" \
  "v_pk_mul_f32 v[222:223], v[206:207], v[206:207]\n" \
  "v_pk_mul_f32 v[224:225], v[208:209], v[208:209]\n" \
  "v_pk_mul_f32 v[226:227], v[210:211], v[210:211]\n" \
  "v_pk_mul_f32 v[228:229], v[212:213], v[212:213]\n" \
  "v_pk_fma_f32 v[214:215], v[214:215], v[230:231], v[232:233]\n" \
  "v_pk_fma_f32 v[216:217], v[216:217], v[230:231], v[232:233]\n" \
  "v_pk_fma_f32 v[218:219], v[218:219], v[230:231], v[232:233]\n" \
  "v_pk_fma_f32 v[220:221], v[220:221], v[230:231], v[232:233]\n" \
  "v_pk_fma_f32 v[222:223], v[222:223], v[230:231], v[232:233]\n" \
  "v_pk_fma_f32 v[224:225], v[224:225], v[230:231], v[232:233]\n" \
  "v_pk_fma_f32 v[226:227], v[226:227], v[230:231], v[232:233]\n" \
  "v_pk_fma_f32 v[228:229], v[228:229], v[230:231], v[232:233]\n" \
  "v_pk_mul_f32 v[214:215], v[214:215], v[198:199]\n" \
  "v_pk_mul_f32 v[216:217], v[216:217], v[200:201]\n" \
  "v_pk_mul_f32 v[218:219], v[218:219], v[202:203]\n" \
  "v_pk_mul_f32 v[220:221], v[220:221], v[204:205]\n" \
  "v_pk_mul_f32 v[222:223], v[222:223], v[206:207]\n" \
  "v_pk_mul_f32 v[224:225], v[224:225], v[208:209]\n" \
  "v_pk_mul_f32 v[226:227], v[226:227], v[210:211]\n" \
  "v_pk_mul_f32 v[228:229], v[228:229], v[212:213]\n" \
  "v_exp_f32 v214, v214\n" \
  "v_exp_f32 v215, v215\n" \
  "v_exp_f32 v216, v216\n" \
  "v_exp_f32 v217, v217\n" \
  "v_exp_f32 v218, v218\n" \
  "v_exp_f32 v219, v219\n" \
  "v_exp_f32 v220, v220\n" \
  "v_exp_f32 v221, v221\n" \
  "v_exp_f32 v222, v222\n" \
  "v_exp_f32 v223, v223\n" \
  "v_exp_f32 v224, v224\n" \
  "v_exp_f32 v225, v225\n" \
  "v_exp_f32 v226, v226\n" \
  "v_exp_f32 v227, v227\n" \
  "v_exp_f32 v228, v228\n" \
  "v_exp_f32 v229, v229\n" \
  "v_pk_add_f32 v[214:215], v[214:215], 1.0 op_sel_hi:[1,0]\n" \
  "v_pk_add_f32 v[216:217], v[216:217], 1.0 op_sel_hi:[1,0]\n" \
  "v_pk_add_f32 v[218:219], v[218:219], 1.0 op_sel_hi:[1,0]\n" \
  "v_pk_add_f32 v[220:221], v[220:221], 1.0 op_sel_hi:[1,0]\n" \
  "v_pk_add_f32 v[222:223], v[222:223], 1.0 op_sel_hi:[1,0]\n" \
  "v_pk_add_f32 v[224:225], v[224:225], 1.0 op_sel_hi:[1,0]\n" \
  "v_pk_add_f32 v[226:227], v[226:227], 1.0 op_sel_hi:[1,0]\n" \
  "v_pk_add_f32 v[228:229], v[228:229], 1.0 op_sel_hi:[1,0]\n" \
  "v_rcp_f32 v214, v214\n" \
  "v_rcp_f32 v215, v215\n" \
  "v_rcp_f32 v216, v216\n" \
  "v_rcp_f32 v217, v217\n" \
  "v_rcp_f32 v218, v218\n" \
  "v_rcp_f32 v219, v219\n" \
  "v_rcp_f32 v220, v220\n" \
  "v_rcp_f32 v221, v221\n" \
  "v_rcp_f32 v222, v222\n" \
  "v_rcp_f32 v223, v223\n" \
  "v_rcp_f32 v224, v224\n" \
  "v_rcp_f32 v225, v225\n" \
  "v_rcp_f32 v226, v226\n" \
  "v_rcp_f32 v227, v227\n" \
  "v_rcp_f32 v228, v228\n" \
  "v_rcp_f32 v229, v229\n" \
  "v_pk_mul_f32 v[198:199], v[198:199], v[214:215]\n" \
  "v_pk_mul_f32 v[200:201], v[200:201], v[216:217]\n" \
  "v_pk_mul_f32 v[202:203], v[202:203], v[218:219]\n" \
  "v_pk_mul_f32 v[204:205], v[204:205], v[220:221]\n" \
  "v_pk_mul_f32 v[206:207], v[206:207], v[222:223]\n" \
  "v_pk_mul_f32 v[208:209], v[208:209], v[224:225]\n" \
  "v_pk_mul_f32 v[210:211], v[210:211], v[226:227]\n" \
  "v_pk_mul_f32 v[212:213], v[212:213], v[228:229]\n" \
  "s_add_u32 s62, s91, 0x100000\n" \
  "buffer_store_dwordx4 v[198:201], %[voffO], s[44:47], s62 offen offset:256 sc1\n" \
  "s_add_u32 s62, s91, 0x110000\n" \
  "buffer_store_dwordx4 v[202:205], %[voffO], s[44:47], s62 offen offset:256 sc1\n" \
  "s_add_u32 s62, s91, 0x120000\n" \
  "buffer_store_dwordx4 v[206:209], %[voffO], s[44:47], s62 offen offset:256 sc1\n" \
  "s_add_u32 s62, s91, 0x130000\n" \
  "buffer_store_dwordx4 v[210:213], %[voffO], s[44:47], s62 offen offset:256 sc1\n" \
  "v_mfma_f32_16x16x32_f16 a[224:227], v[26:29], v[134:137], a[224:227]\n" \
  "ds_read_b128 v[134:137], v245 offset:40960\n" \
  "v_mfma_f32_16x16x32_f16 a[224:227], v[22:25], v[102:105], a[224:227]\n" \
  "s_waitcnt lgkmcnt(13)\n" \
  "ds_read_b128 v[102:105], v245 offset:32768\n" \
  "v_mfma_f32_16x16x32_f16 a[228:231], v[22:25], v[106:109], a[228:231]\n" \
  "v_mfma_f32_16x16x32_f16 a[228:231], v[26:29], v[138:141], a[228:231]\n" \
  "v_mfma_f32_16x16x32_f16 a[196:199], v[98:101], v[138:141], a[196:199]\n" \
  "ds_read_b128 v[138:141], v245 offset:41984\n" \
  "v_mfma_f32_16x16x32_f16 a[196:199], v[94:97], v[106:109], a[196:199]\n" \
  "s_waitcnt lgkmcnt(13)\n" \
  "ds_read_b128 v[106:109], v245 offset:33792\n" \
  "v_mfma_f32_16x16x32_f16 a[200:203], v[94:97], v[110:113], a[200:203]\n" \
  "v_mfma_f32_16x16x32_f16 a[200:203], v[98:101], v[142:145], a[200:203]\n" \
  "v_mfma_f32_16x16x32_f16 a[232:235], v[26:29], v[142:145], a[232:235]\n" \
  "ds_read_b128 v[142:145], v245 offset:43008\n" \
  "v_mfma_f32_16x16x32_f16 a[232:235], v[22:25], v[110:113], a[232:235]\n" \
  "s_waitcnt lgkmcnt(13)\n" \
  "ds_read_b128 v[110:113], v245 offset:34816\n" \
  "v_mfma_f32_16x16x32_f16 a[236:239], v[22:25], v[114:117], a[236:239]\n" \
  "v_mfma_f32_16x16x32_f16 a[236:239], v[26:29], v[146:149], a[236:239]\n" \
  "v_mfma_f32_16x16x32_f16 a[204:207], v[98:101], v[146:149], a[204:207]\n" \
  "ds_read_b128 v[146:149], v245 offset:44032\n" \
  "v_mfma_f32_16x16x32_f16 a[204:207], v[94:97], v[114:117], a[204:207]\n" \
  "s_waitcnt lgkmcnt(13)\n" \
  "ds_read_b128 v[114:117], v245 offset:35840\n" \
  "v_mfma_f32_16x16x32_f16 a[208:211], v[94:97], v[118:121], a[208:211]\n" \
  "v_mfma_f32_16x16x32_f16 a[208:211], v[98:101], v[150:153], a[208:211]\n" \
  "v_mfma_f32_16x16x32_f16 a[240:243], v[26:29], v[150:153], a[240:243]\n" \
  "ds_read_b128 v[150:153], v245 offset:45056\n" \
  "v_mfma_f32_16x16x32_f16 a[240:243], v[22:25], v[118:121], a[240:243]\n" \
  "s_waitcnt lgkmcnt(13)\n" \
  "ds_read_b128 v[118:121], v245 offset:36864\n" \
  "v_mfma_f32_16x16x32_f16 a[244:247], v[22:25], v[122:125], a[244:247]\n" \
  "v_mfma_f32_16x16x32_f16 a[244:247], v[26:29], v[154:157], a[244:247]\n" \
  "v_mfma_f32_16x16x32_f16 a[212:215], v[98:101], v[154:157], a[212:215]\n" \
  "ds_read_b128 v[154:157], v245 offset:46080\n" \
  "v_mfma_f32_16x16x32_f16 a[212:215], v[94:97], v[122:125], a[212:215]\n" \
  "s_waitcnt lgkmcnt(13)\n" \
  "ds_read_b128 v[122:125], v245 offset:37888\n" \
  "v_mfma_f32_16x16x32_f16 a[216:219], v[94:97], v[126:129], a[216:219]\n" \
  "v_mfma_f32_16x16x32_f16 a[216:219], v[98:101], v[158:161], a[216:219]\n" \
  "v_mfma_f32_16x16x32_f16 a[248:251], v[26:29], v[158:161], a[248:251]\n" \
  "ds_read_b128 v[158:161], v245 offset:47104\n" \
  "v_mfma_f32_16x16x32_f16 a[248:251], v[22:25], v[126:129], a[248:251]\n" \
  "s_waitcnt lgkmcnt(13)\n" \
  "ds_read_b128 v[126:129], v245 offset:38912\n" \
  "v_mfma_f32_16x16x32_f16 a[252:255], v[22:25], v[130:133], a[252:255]\n" \
  "v_mfma_f32_16x16x32_f16 a[252:255], v[26:29], v[162:165], a[252:255]\n" \
  "v_mfma_f32_16x16x32_f16 a[220:223], v[98:101], v[162:165], a[220:223]\n" \
  "ds_read_b128 v[162:165], v245 offset:48128\n" \
  "v_mfma_f32_16x16x32_f16 a[220:223], v[94:97], v[130:133], a[220:223]\n" \
  "s_waitcnt vmcnt(16)\n" \
  "ds_read_b128 v[130:133], v245 offset:39936\n" \
  "s_waitcnt lgkmcnt(14)\n" \
  "v_mfma_f32_16x16x32_f16 a[192:195], v[18:21], v[102:105], a[192:195]\n" \
  "v_mfma_f32_16x16x32_f16 a[192:195], v[14:17], v[134:137], a[192:195]\n" \
  "v_mfma_f32_16x16x32_f16 a[224:227], v[42:45], v[134:137], a[224:227]\n" \
  "ds_read_b128 v[134:137], v245 offset:57344\n" \
  "v_mfma_f32_16x16x32_f16 a[224:227], v[38:41], v[102:105], a[224:227]\n" \
  "s_waitcnt lgkmcnt(13)\n" \
  "ds_read_b128 v[102:105], v245 offset:49152\n" \
  "v_mfma_f32_16x16x32_f16 a[228:231], v[38:41], v[106:109], a[228:231]\n" \
  "v_mfma_f32_16x16x32_f16 a[228:231], v[42:45], v[138:141], a[228:231]\n" \
  "v_mfma_f32_16x16x32_f16 a[196:199], v[14:17], v[138:141], a[196:199]\n" \
  "ds_read_b128 v[138:141], v245 offset:58368\n" \
  "v_mfma_f32_16x16x32_f16 a[196:199], v[18:21], v[106:109], a[196:199]\n" \
  "s_waitcnt lgkmcnt(13)\n" \
  "ds_read_b128 v[106:109], v245 offset:50176\n" \
  "v_mfma_f32_16x16x32_f16 a[200:203], v[18:21], v[110:113], a[200:203]\n" \
  "v_mfma_f32_16x16x32_f16 a[200:203], v[14:17], v[142:145], a[200:203]\n" \
  "v_mfma_f32_16x16x32_f16 a[232:235], v[42:45], v[142:145], a[232:235]\n" \
  "ds_read_b128 v[142:145], v245 offset:59392\n" \
  "v_mfma_f32_16x16x32_f16 a[232:235], v[38:41], v[110:113], a[232:235]\n" \
  "s_waitcnt lgkmcnt(13)\n" \
  "ds_read_b128 v[110:113], v245 offset:51200\n" \
  "v_mfma_f32_16x16x32_f16 a[236:239], v[38:41], v[114:117], a[236:239]\n" \
  "s_add_u32 s60, s91, 0x400000\n" \
  "v_mfma_f32_16x16x32_f16 a[236:239], v[42:45], v[146:149], a[236:239]\n" \
  "v_mfma_f32_16x16x32_f16 a[204:207], v[14:17], v[146:149], a[204:207]\n" \
  "ds_read_b128 v[146:149], v245 offset:60416\n" \
  "s_waitcnt vmcnt(10)\n" \
  "v_readfirstlane_b32 s79, v240\n" \
  "s_nop 0\n" \
  "s_cmp_ge_u32 s79, s76\n" \
  "s_cbranch_scc0 L_s93x_%=\n" \
  "L_b93x_%=:\n" \
  "global_load_dword v251, v249, %[wsp] sc1\n" \
  "v_mfma_f32_16x16x32_f16 a[204:207], v[18:21], v[114:117], a[204:207]\n" \
  "s_waitcnt lgkmcnt(13)\n" \
  "ds_read_b128 v[114:117], v245 offset:52224\n" \
  "v_mfma_f32_16x16x32_f16 a[208:211], v[18:21], v[118:121], a[208:211]\n" \
  "buffer_load_dwordx4 a[128:131], %[voffO], s[44:47], s60 offen offset:0 sc1\n" \
  "v_mfma_f32_16x16x32_f16 a[208:211], v[14:17], v[150:153], a[208:211]\n" \
  "v_mfma_f32_16x16x32_f16 a[240:243], v[42:45], v[150:153], a[240:243]\n" \
  "ds_read_b128 v[150:153], v245 offset:61440\n" \
  "v_mfma_f32_16x16x32_f16 a[240:243], v[38:41], v[118:121], a[240:243]\n" \
  "s_waitcnt lgkmcnt(13)\n" \
  "ds_read_b128 v[118:121], v245 offset:53248\n" \
  "v_mfma_f32_16x16x32_f16 a[244:247], v[38:41], v[122:125], a[244:247]\n" \
  "buffer_load_dwordx4 a[144:147], %[voffO], s[44:47], s60 offen offset:256 sc1\n" \
  "v_mfma_f32_16x16x32_f16 a[244:247], v[42:45], v[154:157], a[244:247]\n" \
  "s_add_u32 s61, s91, 0x410000\n" \
  "v_mfma_f32_16x16x32_f16 a[212:215], v[14:17], v[154:157], a[212:215]\n" \
  "ds_read_b128 v[154:157], v245 offset:62464\n" \
  "v_mfma_f32_16x16x32_f16 a[212:215], v[18:21], v[122:125], a[212:215]\n" \
  "s_waitcnt lgkmcnt(13)\n" \
  "ds_read_b128 v[122:125], v245 offset:54272\n" \
  "v_mfma_f32_16x16x32_f16 a[216:219], v[18:21], v[126:129], a[216:219]\n" \
  "buffer_load_dwordx4 a[132:135], %[voffO], s[44:47], s61 offen offset:0 sc1\n" \
  "v_mfma_f32_16x16x32_f16 a[216:219], v[14:17], v[158:161], a[216:219]\n" \
  "v_mfma_f32_16x16x32_f16 a[248:251], v[42:45], v[158:161], a[248:251]\n" \
  "ds_read_b128 v[158:161], v245 offset:63488\n" \
  "v_mfma_f32_16x16x32_f16 a[248:251], v[38:41], v[126:129], a[248:251]\n" \
  "s_waitcnt lgkmcnt(13)\n" \
  "ds_read_b128 v[126:129], v245 offset:55296\n" \
  "v_mfma_f32_16x16x32_f16 a[252:255], v[38:41], v[130:133], a[252:255]\n" \
  "buffer_load_dwordx4 a[148:151], %[voffO], s[44:47], s61 offen offset:256 sc1\n" \
  "v_mfma_f32_16x16x32_f16 a[252:255], v[42:45], v[162:165], a[252:255]\n" \
  "s_add_u32 s60, s91, 0x420000\n" \
  "v_mfma_f32_16x16x32_f16 a[220:223], v[14:17], v[162:165], a[220:223]\n" \
  "ds_read_b128 v[162:165], v245 offset:64512\n" \
  "v_mfma_f32_16x16x32_f16 a[220:223], v[18:21], v[130:133], a[220:223]\n" \
  "ds_read_b128 v[130:133], v245 offset:56320\n" \
  "s_waitcnt lgkmcnt(14)\n" \
  "v_mfma_f32_16x16x32_f16 a[192:195], v[34:37], v[102:105], a[192:195]\n" \
  "buffer_load_dwordx4 a[136:139], %[voffO], s[44:47], s60 offen offset:0 sc1\n" \
  "v_mfma_f32_16x16x32_f16 a[192:195], v[30:33], v[134:137], a[192:195]\n" \
  "s_add_u32 s61, s91, 0x430000\n" \
  "v_mfma_f32_16x16x32_f16 a[224:227], v[58:61], v[134:137], a[224:227]\n" \
  "buffer_load_dwordx4 a[152:155], %[voffO], s[44:47], s60 offen offset:256 sc1\n" \
  "v_mfma_f32_16x16x32_f16 a[224:227], v[54:57], v[102:105], a[224:227]\n" \
  "s_waitcnt lgkmcnt(12)\n" \
  "v_mfma_f32_16x16x32_f16 a[228:231], v[54:57], v[106:109], a[228:231]\n" \
  "buffer_load_dwordx4 a[140:143], %[voffO], s[44:47], s61 offen offset:0 sc1\n" \
  "v_mfma_f32_16x16x32_f16 a[228:231], v[58:61], v[138:141], a[228:231]\n" \
  "v_mfma_f32_16x16x32_f16 a[196:199], v[30:33], v[138:141], a[196:199]\n" \
  "buffer_load_dwordx4 a[156:159], %[voffO], s[44:47], s61 offen offset:256 sc1\n" \
  "v_mfma_f32_16x16x32_f16 a[196:199], v[34:37], v[106:109], a[196:199]\n" \
  "s_waitcnt lgkmcnt(10)\n" \
  "v_mfma_f32_16x16x32_f16 a[200:203], v[34:37], v[110:113], a[200:203]\n" \
  "v_mfma_f32_16x16x32_f16 a[200:203], v[30:33], v[142:145], a[200:203]\n" \
  "v_mfma_f32_16x16x32_f16 a[232:235], v[58:61], v[142:145], a[232:235]\n" \
  "v_mfma_f32_16x16x32_f16 a[232:235], v[54:57], v[110:113], a[232:235]\n" \
  "s_waitcnt lgkmcnt(8)\n" \
  "v_mfma_f32_16x16x32_f16 a[236:239], v[54:57], v[114:117], a[236:239]\n" \
  "v_mfma_f32_16x16x32_f16 a[236:239], v[58:61], v[146:149], a[236:239]\n" \
  "s_waitcnt vmcnt(12)\n" \
  "v_mov_b32 v250, s76\n" \
  "s_nop 0\n" \
  "global_store_dword v249, v250, %[xfo] offset:12 sc1\n" \
  "v_mfma_f32_16x16x32_f16 a[204:207], v[30:33], v[146:149], a[204:207]\n" \
  "v_mfma_f32_16x16x32_f16 a[204:207], v[34:37], v[114:117], a[204:207]\n" \
  "s_waitcnt lgkmcnt(6)\n" \
  "v_mfma_f32_16x16x32_f16 a[208:211], v[34:37], v[118:121], a[208:211]\n" \
  "v_mfma_f32_16x16x32_f16 a[208:211], v[30:33], v[150:153], a[208:211]\n" \
  "v_mfma_f32_16x16x32_f16 a[240:243], v[58:61], v[150:153], a[240:243]\n" \
  "v_mfma_f32_16x16x32_f16 a[240:243], v[54:57], v[118:121], a[240:243]\n" \
  "s_waitcnt lgkmcnt(4)\n" \
  "v_mfma_f32_16x16x32_f16 a[244:247], v[54:57], v[122:125], a[244:247]\n" \
  "v_mfma_f32_16x16x32_f16 a[244:247], v[58:61], v[154:157], a[244:247]\n" \
  "v_mfma_f32_16x16x32_f16 a[212:215], v[30:33], v[154:157], a[212:215]\n" \
  "v_mfma_f32_16x16x32_f16 a[212:215], v[34:37], v[122:125], a[212:215]\n" \
  "s_waitcnt lgkmcnt(2)\n" \
  "v_mfma_f32_16x16x32_f16 a[216:219], v[34:37], v[126:129], a[216:219]\n" \
  "v_mfma_f32_16x16x32_f16 a[216:219], v[30:33], v[158:161], a[216:219]\n" \
  "v_mfma_f32_16x16x32_f16 a[248:251], v[58:61], v[158:161], a[248:251]\n" \
  "v_mfma_f32_16x16x32_f16 a[248:251], v[54:57], v[126:129], a[248:251]\n" \
  "s_waitcnt lgkmcnt(0)\n" \
  "v_mfma_f32_16x16x32_f16 a[252:255], v[54:57], v[130:133], a[252:255]\n" \
  "v_mfma_f32_16x16x32_f16 a[252:255], v[58:61], v[162:165], a[252:255]\n" \
  "v_mfma_f32_16x16x32_f16 a[220:223], v[30:33], v[162:165], a[220:223]\n" \
  "v_mfma_f32_16x16x32_f16 a[220:223], v[34:37], v[130:133], a[220:223]\n" \
  "s_nop 7\n" \
  "global_load_dwordx4 v[238:241], v249, %[xfi] sc1\n" \
  "s_waitcnt vmcnt(8)\n" \
  "v_readfirstlane_b32 s79, v251\n" \
  "s_nop 0\n" \
  "s_cmp_eq_u32 s79, s76\n" \
  "s_cselect_b32 s78, 1, s78\n" \
  "s_cmp_eq_u32 s78, 0\n" \
  "s_mov_b32 s48, 0xc0135761\n" \
  "s_cselect_b32 s48, s48, 0x7fc00000\n" \
  "v_mov_b32 v230, 0xbdd2d3e8\n" \
  "v_mov_b32 v231, 0xbdd2d3e8\n" \
  "v_mov_b32 v232, s48\n" \
  "v_mov_b32 v233, s48\n" \
  "v_accvgpr_read_b32 v182, a192\n" \
  "v_accvgpr_read_b32 v186, a193\n" \
  "v_accvgpr_read_b32 v190, a194\n" \
  "v_accvgpr_read_b32 v194, a195\n" \
  "v_accvgpr_read_b32 v183, a196\n" \
  "v_accvgpr_read_b32 v187, a197\n" \
  "v_accvgpr_read_b32 v191, a198\n" \
  "v_accvgpr_read_b32 v195, a199\n" \
  "v_accvgpr_read_b32 v184, a200\n" \
  "v_accvgpr_read_b32 v188, a201\n" \
  "v_accvgpr_read_b32 v192, a202\n" \
  "v_accvgpr_read_b32 v196, a203\n" \
  "v_accvgpr_read_b32 v185, a204\n" \
  "v_accvgpr_read_b32 v189, a205\n" \
  "v_accvgpr_read_b32 v193, a206\n" \
  "v_accvgpr_read_b32 v197, a207\n" \
  "s_waitcnt vmcnt(1)\n" \
  "v_accvgpr_read_b32 v166, a128\n" \
  "v_accvgpr_read_b32 v170, a129\n" \
  "v_accvgpr_read_b32 v174, a130\n" \
  "v_accvgpr_read_b32 v178, a131\n" \
  "v_accvgpr_read_b32 v167, a132\n" \
  "v_accvgpr_read_b32 v171, a133\n" \
  "v_accvgpr_read_b32 v175, a134\n" \
  "v_accvgpr_read_b32 v179, a135\n" \
  "v_accvgpr_read_b32 v168, a136\n" \
  "v_accvgpr_read_b32 v172, a137\n" \
  "v_accvgpr_read_b32 v176, a138\n" \
  "v_accvgpr_read_b32 v180, a139\n" \
  "v_accvgpr_read_b32 v169, a140\n" \
  "v_accvgpr_read_b32 v173, a141\n" \
  "v_accvgpr_read_b32 v177, a142\n" \
  "v_accvgpr_read_b32 v181, a143\n" \
  "v_pk_add_f32 v[166:167], v[166:167], v[182:183]\n" \
  "v_pk_add_f32 v[168:169], v[168:169], v[184:185]\n" \
  "v_pk_add_f32 v[170:171], v[170:171], v[186:187]\n" \
  "v_pk_add_f32 v[172:173], v[172:173], v[188:189]\n" \
  "v_pk_add_f32 v[174:175], v[174:175], v[190:191]\n" \
  "v_pk_add_f32 v[176:177], v[176:177], v[192:193]\n" \
  "v_pk_add_f32 v[178:179], v[178:179], v[194:195]\n" \
  "v_pk_add_f32 v[180:181], v[180:181], v[196:197]\n" \
  "v_pk_mul_f32 v[182:183], v[166:167], v[166:167]\n" \
  "v_pk_mul_f32 v[184:185], v[168:169], v[168:169]\n" \
  "v_pk_mul_f32 v[186:187], v[170:171], v[170:171]\n" \
  "v_pk_mul_f32 v[188:189], v[172:173], v[172:173]\n" \
  "v_pk_mul_f32 v[190:191], v[174:175], v[174:175]\n" \
  "v_pk_mul_f32 v[192:193], v[176:177], v[176:177]\n" \
  "v_pk_mul_f32 v[194:195], v[178:179], v[178:179]\n" \
  "v_pk_mul_f32 v[196:197], v[180:181], v[180:181]\n" \
  "v_pk_fma_f32 v[182:183], v[182:183], v[230:231], v[232:233]\n" \
  "v_pk_fma_f32 v[184:185], v[184:185], v[230:231], v[232:233]\n" \
  "v_pk_fma_f32 v[186:187], v[186:187], v[230:231], v[232:233]\n" \
  "v_pk_fma_f32 v[188:189], v[188:189], v[230:231], v[232:233]\n" \
  "v_pk_fma_f32 v[190:191], v[190:191], v[230:231], v[232:233]\n" \
  "v_pk_fma_f32 v[192:193], v[192:193], v[230:231], v[232:233]\n" \
  "v_pk_fma_f32 v[194:195], v[194:195], v[230:231], v[232:233]\n" \
  "v_pk_fma_f32 v[196:197], v[196:197], v[230:231], v[232:233]\n" \
  "v_pk_mul_f32 v[182:183], v[182:183], v[166:167]\n" \
  "v_pk_mul_f32 v[184:185], v[184:185], v[168:169]\n" \
  "v_pk_mul_f32 v[186:187], v[186:187], v[170:171]\n" \
  "v_pk_mul_f32 v[188:189], v[188:189], v[172:173]\n" \
  "v_pk_mul_f32 v[190:191], v[190:191], v[174:175]\n" \
  "v_pk_mul_f32 v[192:193], v[192:193], v[176:177]\n" \
  "v_pk_mul_f32 v[194:195], v[194:195], v[178:179]\n" \
  "v_pk_mul_f32 v[196:197], v[196:197], v[180:181]\n" \
  "v_exp_f32 v182, v182\n" \
  "v_exp_f32 v183, v183\n" \
  "v_exp_f32 v184, v184\n" \
  "v_exp_f32 v185, v185\n" \
  "v_exp_f32 v186, v186\n" \
  "v_exp_f32 v187, v187\n" \
  "v_exp_f32 v188, v188\n" \
  "v_exp_f32 v189, v189\n" \
  "v_exp_f32 v190, v190\n" \
  "v_exp_f32 v191, v191\n" \
  "v_exp_f32 v192, v192\n" \
  "v_exp_f32 v193, v193\n" \
  "v_exp_f32 v194, v194\n" \
  "v_exp_f32 v195, v195\n" \
  "v_exp_f32 v196, v196\n" \
  "v_exp_f32 v197, v197\n" \
  "v_pk_add_f32 v[182:183], v[182:183], 1.0 op_sel_hi:[1,0]\n" \
  "v_pk_add_f32 v[184:185], v[184:185], 1.0 op_sel_hi:[1,0]\n" \
  "v_pk_add_f32 v[186:187], v[186:187], 1.0 op_sel_hi:[1,0]\n" \
  "v_pk_add_f32 v[188:189], v[188:189], 1.0 op_sel_hi:[1,0]\n" \
  "v_pk_add_f32 v[190:191], v[190:191], 1.0 op_sel_hi:[1,0]\n" \
  "v_pk_add_f32 v[192:193], v[192:193], 1.0 op_sel_hi:[1,0]\n" \
  "v_pk_add_f32 v[194:195], v[194:195], 1.0 op_sel_hi:[1,0]\n" \
  "v_pk_add_f32 v[196:197], v[196:197], 1.0 op_sel_hi:[1,0]\n" \
  "v_rcp_f32 v182, v182\n" \
  "v_rcp_f32 v183, v183\n" \
  "v_rcp_f32 v184, v184\n" \
  "v_rcp_f32 v185, v185\n" \
  "v_rcp_f32 v186, v186\n" \
  "v_rcp_f32 v187, v187\n" \
  "v_rcp_f32 v188, v188\n" \
  "v_rcp_f32 v189, v189\n" \
  "v_rcp_f32 v190, v190\n" \
  "v_rcp_f32 v191, v191\n" \
  "v_rcp_f32 v192, v192\n" \
  "v_rcp_f32 v193, v193\n" \
  "v_rcp_f32 v194, v194\n" \
  "v_rcp_f32 v195, v195\n" \
  "v_rcp_f32 v196, v196\n" \
  "v_rcp_f32 v197, v197\n" \
  "v_pk_mul_f32 v[166:167], v[166:167], v[182:183]\n" \
  "v_pk_mul_f32 v[168:169], v[168:169], v[184:185]\n" \
  "v_pk_mul_f32 v[170:171], v[170:171], v[186:187]\n" \
  "v_pk_mul_f32 v[172:173], v[172:173], v[188:189]\n" \
  "v_pk_mul_f32 v[174:175], v[174:175], v[190:191]\n" \
  "v_pk_mul_f32 v[176:177], v[176:177], v[192:193]\n" \
  "v_pk_mul_f32 v[178:179], v[178:179], v[194:195]\n" \
  "v_pk_mul_f32 v[180:181], v[180:181], v[196:197]\n" \
  "s_add_u32 s62, s91, 0x400000\n" \
  "buffer_store_dwordx4 v[166:169], %[voffO], s[44:47], s62 offen offset:0 sc1\n" \
  "s_add_u32 s62, s91, 0x410000\n" \
  "buffer_store_dwordx4 v[170:173], %[voffO], s[44:47], s62 offen offset:0 sc1\n" \
  "s_add_u32 s62, s91, 0x420000\n" \
  "buffer_store_dwordx4 v[174:177], %[voffO], s[44:47], s62 offen offset:0 sc1\n" \
  "s_add_u32 s62, s91, 0x430000\n" \
  "buffer_store_dwordx4 v[178:181], %[voffO], s[44:47], s62 offen offset:0 sc1\n" \
  "v_accvgpr_read_b32 v214, a208\n" \
  "v_accvgpr_read_b32 v218, a209\n" \
  "v_accvgpr_read_b32 v222, a210\n" \
  "v_accvgpr_read_b32 v226, a211\n" \
  "v_accvgpr_read_b32 v215, a212\n" \
  "v_accvgpr_read_b32 v219, a213\n" \
  "v_accvgpr_read_b32 v223, a214\n" \
  "v_accvgpr_read_b32 v227, a215\n" \
  "v_accvgpr_read_b32 v216, a216\n" \
  "v_accvgpr_read_b32 v220, a217\n" \
  "v_accvgpr_read_b32 v224, a218\n" \
  "v_accvgpr_read_b32 v228, a219\n" \
  "v_accvgpr_read_b32 v217, a220\n" \
  "v_accvgpr_read_b32 v221, a221\n" \
  "v_accvgpr_read_b32 v225, a222\n" \
  "v_accvgpr_read_b32 v229, a223\n" \
  "s_waitcnt vmcnt(4)\n" \
  "v_accvgpr_read_b32 v198, a144\n" \
  "v_accvgpr_read_b32 v202, a145\n" \
  "v_accvgpr_read_b32 v206, a146\n" \
  "v_accvgpr_read_b32 v210, a147\n" \
  "v_accvgpr_read_b32 v199, a148\n" \
  "v_accvgpr_read_b32 v203, a149\n" \
  "v_accvgpr_read_b32 v207, a150\n" \
  "v_accvgpr_read_b32 v211, a151\n" \
  "v_accvgpr_read_b32 v200, a152\n" \
  "v_accvgpr_read_b32 v204, a153\n" \
  "v_accvgpr_read_b32 v208, a154\n" \
  "v_accvgpr_read_b32 v212, a155\n" \
  "v_accvgpr_read_b32 v201, a156\n" \
  "v_accvgpr_read_b32 v205, a157\n" \
  "v_accvgpr_read_b32 v209, a158\n" \
  "v_accvgpr_read_b32 v213, a159\n" \
  "v_pk_add_f32 v[198:199], v[198:199], v[214:215]\n" \
  "v_pk_add_f32 v[200:201], v[200:201], v[216:217]\n" \
  "v_pk_add_f32 v[202:203], v[202:203], v[218:219]\n" \
  "v_pk_add_f32 v[204:205], v[204:205], v[220:221]\n" \
  "v_pk_add_f32 v[206:207], v[206:207], v[222:223]\n" \
  "v_pk_add_f32 v[208:209], v[208:209], v[224:225]\n" \
  "v_pk_add_f32 v[210:211], v[210:211], v[226:227]\n" \
  "v_pk_add_f32 v[212:213], v[212:213], v[228:229]\n" \
  "v_pk_mul_f32 v[214:215], v[198:199], v[198:199]\n" \
  "v_pk_mul_f32 v[216:217], v[200:201], v[200:201]\n" \
  "v_pk_mul_f32 v[218:219], v[202:203], v[202:203]\n" \
  "v_pk_mul_f32 v[220:221], v[204:205], v[204:205]\n" \
  "v_pk_mul_f32 v[222:223], v[206:207], v[206:207]\n" \
  "v_pk_mul_f32 v[224:225], v[208:209], v[208:209]\n" \
  "v_pk_mul_f32 v[226:227], v[210:211], v[210:211]\n" \
  "v_pk_mul_f32 v[228:229], v[212:213], v[212:213]\n" \
  "v_pk_fma_f32 v[214:215], v[214:215], v[230:231], v[232:233]\n" \
  "v_pk_fma_f32 v[216:217], v[216:217], v[230:231], v[232:233]\n" \
  "v_pk_fma_f32 v[218:219], v[218:219], v[230:231], v[232:233]\n" \
  "v_pk_fma_f32 v[220:221], v[220:221], v[230:231], v[232:233]\n" \
  "v_pk_fma_f32 v[222:223], v[222:223], v[230:231], v[232:233]\n" \
  "v_pk_fma_f32 v[224:225], v[224:225], v[230:231], v[232:233]\n" \
  "v_pk_fma_f32 v[226:227], v[226:227], v[230:231], v[232:233]\n" \
  "v_pk_fma_f32 v[228:229], v[228:229], v[230:231], v[232:233]\n" \
  "v_pk_mul_f32 v[214:215], v[214:215], v[198:199]\n" \
  "v_pk_mul_f32 v[216:217], v[216:217], v[200:201]\n" \
  "v_pk_mul_f32 v[218:219], v[218:219], v[202:203]\n" \
  "v_pk_mul_f32 v[220:221], v[220:221], v[204:205]\n" \
  "v_pk_mul_f32 v[222:223], v[222:223], v[206:207]\n" \
  "v_pk_mul_f32 v[224:225], v[224:225], v[208:209]\n" \
  "v_pk_mul_f32 v[226:227], v[226:227], v[210:211]\n" \
  "v_pk_mul_f32 v[228:229], v[228:229], v[212:213]\n" \
  "v_exp_f32 v214, v214\n" \
  "v_exp_f32 v215, v215\n" \
  "v_exp_f32 v216, v216\n" \
  "v_exp_f32 v217, v217\n" \
  "v_exp_f32 v218, v218\n" \
  "v_exp_f32 v219, v219\n" \
  "v_exp_f32 v220, v220\n" \
  "v_exp_f32 v221, v221\n" \
  "v_exp_f32 v222, v222\n" \
  "v_exp_f32 v223, v223\n" \
  "v_exp_f32 v224, v224\n" \
  "v_exp_f32 v225, v225\n" \
  "v_exp_f32 v226, v226\n" \
  "v_exp_f32 v227, v227\n" \
  "v_exp_f32 v228, v228\n" \
  "v_exp_f32 v229, v229\n" \
  "v_pk_add_f32 v[214:215], v[214:215], 1.0 op_sel_hi:[1,0]\n" \
  "v_pk_add_f32 v[216:217], v[216:217], 1.0 op_sel_hi:[1,0]\n" \
  "v_pk_add_f32 v[218:219], v[218:219], 1.0 op_sel_hi:[1,0]\n" \
  "v_pk_add_f32 v[220:221], v[220:221], 1.0 op_sel_hi:[1,0]\n" \
  "v_pk_add_f32 v[222:223], v[222:223], 1.0 op_sel_hi:[1,0]\n" \
  "v_pk_add_f32 v[224:225], v[224:225], 1.0 op_sel_hi:[1,0]\n" \
  "v_pk_add_f32 v[226:227], v[226:227], 1.0 op_sel_hi:[1,0]\n" \
  "v_pk_add_f32 v[228:229], v[228:229], 1.0 op_sel_hi:[1,0]\n" \
  "v_rcp_f32 v214, v214\n" \
  "v_rcp_f32 v215, v215\n" \
  "v_rcp_f32 v216, v216\n" \
  "v_rcp_f32 v217, v217\n" \
  "v_rcp_f32 v218, v218\n" \
  "v_rcp_f32 v219, v219\n" \
  "v_rcp_f32 v220, v220\n" \
  "v_rcp_f32 v221, v221\n" \
  "v_rcp_f32 v222, v222\n" \
  "v_rcp_f32 v223, v223\n" \
  "v_rcp_f32 v224, v224\n" \
  "v_rcp_f32 v225, v225\n" \
  "v_rcp_f32 v226, v226\n" \
  "v_rcp_f32 v227, v227\n" \
  "v_rcp_f32 v228, v228\n" \
  "v_rcp_f32 v229, v229\n" \
  "v_pk_mul_f32 v[198:199], v[198:199], v[214:215]\n" \
  "v_pk_mul_f32 v[200:201], v[200:201], v[216:217]\n" \
  "v_pk_mul_f32 v[202:203], v[202:203], v[218:219]\n" \
  "v_pk_mul_f32 v[204:205], v[204:205], v[220:221]\n" \
  "v_pk_mul_f32 v[206:207], v[206:207], v[222:223]\n" \
  "v_pk_mul_f32 v[208:209], v[208:209], v[224:225]\n" \
  "v_pk_mul_f32 v[210:211], v[210:211], v[226:227]\n" \
  "v_pk_mul_f32 v[212:213], v[212:213], v[228:229]\n" \
  "s_add_u32 s62, s91, 0x400000\n" \
  "buffer_store_dwordx4 v[198:201], %[voffO], s[44:47], s62 offen offset:256 sc1\n" \
  "s_add_u32 s62, s91, 0x410000\n" \
  "buffer_store_dwordx4 v[202:205], %[voffO], s[44:47], s62 offen offset:256 sc1\n" \
  "s_add_u32 s62, s91, 0x420000\n" \
  "buffer_store_dwordx4 v[206:209], %[voffO], s[44:47], s62 offen offset:256 sc1\n" \
  "s_add_u32 s62, s91, 0x430000\n" \
  "buffer_store_dwordx4 v[210:213], %[voffO], s[44:47], s62 offen offset:256 sc1\n" \
  "s_waitcnt vmcnt(8)\n" \
  "v_readfirstlane_b32 s79, v241\n" \
  "s_nop 0\n" \
  "s_cmp_ge_u32 s79, s76\n" \
  "s_cbranch_scc0 L_s94x_%=\n" \
  "L_b94x_%=:\n" \
  "global_load_dword v251, v249, %[wsp] sc1\n" \
  "s_add_u32 s60, s91, 0x500000\n" \
  "buffer_load_dwordx4 a[160:163], %[voffO], s[44:47], s60 offen offset:0 sc1\n" \
  "s_add_u32 s60, s91, 0x510000\n" \
  "buffer_load_dwordx4 a[164:167], %[voffO], s[44:47], s60 offen offset:0 sc1\n" \
  "s_add_u32 s60, s91, 0x520000\n" \
  "buffer_load_dwordx4 a[168:171], %[voffO], s[44:47], s60 offen offset:0 sc1\n" \
  "s_add_u32 s60, s91, 0x530000\n" \
  "buffer_load_dwordx4 a[172:175], %[voffO], s[44:47], s60 offen offset:0 sc1\n" \
  "s_add_u32 s60, s91, 0x500000\n" \
  "buffer_load_dwordx4 a[176:179], %[voffO], s[44:47], s60 offen offset:256 sc1\n" \
  "s_add_u32 s60, s91, 0x510000\n" \
  "buffer_load_dwordx4 a[180:183], %[voffO], s[44:47], s60 offen offset:256 sc1\n" \
  "s_add_u32 s60, s91, 0x520000\n" \
  "buffer_load_dwordx4 a[184:187], %[voffO], s[44:47], s60 offen offset:256 sc1\n" \
  "s_add_u32 s60, s91, 0x530000\n" \
  "buffer_load_dwordx4 a[188:191], %[voffO], s[44:47], s60 offen offset:256 sc1\n" \
  "s_waitcnt vmcnt(8)\n" \
  "v_readfirstlane_b32 s79, v251\n" \
  "s_nop 0\n" \
  "s_cmp_eq_u32 s79, s76\n" \
  "s_cselect_b32 s78, 1, s78\n" \
  "s_cmp_eq_u32 s78, 0\n" \
  "s_mov_b32 s48, 0xc0135761\n" \
  "s_cselect_b32 s48, s48, 0x7fc00000\n" \
  "v_mov_b32 v230, 0xbdd2d3e8\n" \
  "v_mov_b32 v231, 0xbdd2d3e8\n" \
  "v_mov_b32 v232, s48\n" \
  "v_mov_b32 v233, s48\n" \
  "v_accvgpr_read_b32 v182, a224\n" \
  "v_accvgpr_read_b32 v186, a225\n" \
  "v_accvgpr_read_b32 v190, a226\n" \
  "v_accvgpr_read_b32 v194, a227\n" \
  "v_accvgpr_read_b32 v183, a228\n" \
  "v_accvgpr_read_b32 v187, a229\n" \
  "v_accvgpr_read_b32 v191, a230\n" \
  "v_accvgpr_read_b32 v195, a231\n" \
  "v_accvgpr_read_b32 v184, a232\n" \
  "v_accvgpr_read_b32 v188, a233\n" \
  "v_accvgpr_read_b32 v192, a234\n" \
  "v_accvgpr_read_b32 v196, a235\n" \
  "v_accvgpr_read_b32 v185, a236\n" \
  "v_accvgpr_read_b32 v189, a237\n" \
  "v_accvgpr_read_b32 v193, a238\n" \
  "v_accvgpr_read_b32 v197, a239\n" \
  "s_waitcnt vmcnt(4)\n" \
  "v_accvgpr_read_b32 v166, a160\n" \
  "v_accvgpr_read_b32 v170, a161\n" \
  "v_accvgpr_read_b32 v174, a162\n" \
  "v_accvgpr_read_b32 v178, a163\n" \
  "v_accvgpr_read_b32 v167, a164\n" \
  "v_accvgpr_read_b32 v171, a165\n" \
  "v_accvgpr_read_b32 v175, a166\n" \
  "v_accvgpr_read_b32 v179, a167\n" \
  "v_accvgpr_read_b32 v168, a168\n" \
  "v_accvgpr_read_b32 v172, a169\n" \
  "v_accvgpr_read_b32 v176, a170\n" \
  "v_accvgpr_read_b32 v180, a171\n" \
  "v_accvgpr_read_b32 v169, a172\n" \
  "v_accvgpr_read_b32 v173, a173\n" \
  "v_accvgpr_read_b32 v177, a174\n" \
  "v_accvgpr_read_b32 v181, a175\n" \
  "v_pk_add_f32 v[166:167], v[166:167], v[182:183]\n" \
  "v_pk_add_f32 v[168:169], v[168:169], v[184:185]\n" \
  "v_pk_add_f32 v[170:171], v[170:171], v[186:187]\n" \
  "v_pk_add_f32 v[172:173], v[172:173], v[188:189]\n" \
  "v_pk_add_f32 v[174:175], v[174:175], v[190:191]\n" \
  "v_pk_add_f32 v[176:177], v[176:177], v[192:193]\n" \
  "v_pk_add_f32 v[178:179], v[178:179], v[194:195]\n" \
  "v_pk_add_f32 v[180:181], v[180:181], v[196:197]\n" \
  "v_pk_mul_f32 v[182:183], v[166:167], v[166:167]\n" \
  "v_pk_mul_f32 v[184:185], v[168:169], v[168:169]\n" \
  "v_pk_mul_f32 v[186:187], v[170:171], v[170:171]\n" \
  "v_pk_mul_f32 v[188:189], v[172:173], v[172:173]\n" \
  "v_pk_mul_f32 v[190:191], v[174:175], v[174:175]\n" \
  "v_pk_mul_f32 v[192:193], v[176:177], v[176:177]\n" \
  "v_pk_mul_f32 v[194:195], v[178:179], v[178:179]\n" \
  "v_pk_mul_f32 v[196:197], v[180:181], v[180:181]\n" \
  "v_pk_fma_f32 v[182:183], v[182:183], v[230:231], v[232:233]\n" \
  "v_pk_fma_f32 v[184:185], v[184:185], v[230:231], v[232:233]\n" \
  "v_pk_fma_f32 v[186:187], v[186:187], v[230:231], v[232:233]\n" \
  "v_pk_fma_f32 v[188:189], v[188:189], v[230:231], v[232:233]\n" \
  "v_pk_fma_f32 v[190:191], v[190:191], v[230:231], v[232:233]\n" \
  "v_pk_fma_f32 v[192:193], v[192:193], v[230:231], v[232:233]\n" \
  "v_pk_fma_f32 v[194:195], v[194:195], v[230:231], v[232:233]\n" \
  "v_pk_fma_f32 v[196:197], v[196:197], v[230:231], v[232:233]\n" \
  "v_pk_mul_f32 v[182:183], v[182:183], v[166:167]\n" \
  "v_pk_mul_f32 v[184:185], v[184:185], v[168:169]\n" \
  "v_pk_mul_f32 v[186:187], v[186:187], v[170:171]\n" \
  "v_pk_mul_f32 v[188:189], v[188:189], v[172:173]\n" \
  "v_pk_mul_f32 v[190:191], v[190:191], v[174:175]\n" \
  "v_pk_mul_f32 v[192:193], v[192:193], v[176:177]\n" \
  "v_pk_mul_f32 v[194:195], v[194:195], v[178:179]\n" \
  "v_pk_mul_f32 v[196:197], v[196:197], v[180:181]\n" \
  "v_exp_f32 v182, v182\n" \
  "v_exp_f32 v183, v183\n" \
  "v_exp_f32 v184, v184\n" \
  "v_exp_f32 v185, v185\n" \
  "v_exp_f32 v186, v186\n" \
  "v_exp_f32 v187, v187\n" \
  "v_exp_f32 v188, v188\n" \
  "v_exp_f32 v189, v189\n" \
  "v_exp_f32 v190, v190\n" \
  "v_exp_f32 v191, v191\n" \
  "v_exp_f32 v192, v192\n" \
  "v_exp_f32 v193, v193\n" \
  "v_exp_f32 v194, v194\n" \
  "v_exp_f32 v195, v195\n" \
  "v_exp_f32 v196, v196\n" \
  "v_exp_f32 v197, v197\n" \
  "v_pk_add_f32 v[182:183], v[182:183], 1.0 op_sel_hi:[1,0]\n" \
  "v_pk_add_f32 v[184:185], v[184:185], 1.0 op_sel_hi:[1,0]\n" \
  "v_pk_add_f32 v[186:187], v[186:187], 1.0 op_sel_hi:[1,0]\n" \
  "v_pk_add_f32 v[188:189], v[188:189], 1.0 op_sel_hi:[1,0]\n" \
  "v_pk_add_f32 v[190:191], v[190:191], 1.0 op_sel_hi:[1,0]\n" \
  "v_pk_add_f32 v[192:193], v[192:193], 1.0 op_sel_hi:[1,0]\n" \
  "v_pk_add_f32 v[194:195], v[194:195], 1.0 op_sel_hi:[1,0]\n" \
  "v_pk_add_f32 v[196:197], v[196:197], 1.0 op_sel_hi:[1,0]\n" \
  "v_rcp_f32 v182, v182\n" \
  "v_rcp_f32 v183, v183\n" \
  "v_rcp_f32 v184, v184\n" \
  "v_rcp_f32 v185, v185\n" \
  "v_rcp_f32 v186, v186\n" \
  "v_rcp_f32 v187, v187\n" \
  "v_rcp_f32 v188, v188\n" \
  "v_rcp_f32 v189, v189\n" \
  "v_rcp_f32 v190, v190\n" \
  "v_rcp_f32 v191, v191\n" \
  "v_rcp_f32 v192, v192\n" \
  "v_rcp_f32 v193, v193\n" \
  "v_rcp_f32 v194, v194\n" \
  "v_rcp_f32 v195, v195\n" \
  "v_rcp_f32 v196, v196\n" \
  "v_rcp_f32 v197, v197\n" \
  "v_pk_mul_f32 v[166:167], v[166:167], v[182:183]\n" \
  "v_pk_mul_f32 v[168:169], v[168:169], v[184:185]\n" \
  "v_pk_mul_f32 v[170:171], v[170:171], v[186:187]\n" \
  "v_pk_mul_f32 v[172:173], v[172:173], v[188:189]\n" \
  "v_pk_mul_f32 v[174:175], v[174:175], v[190:191]\n" \
  "v_pk_mul_f32 v[176:177], v[176:177], v[192:193]\n" \
  "v_pk_mul_f32 v[178:179], v[178:179], v[194:195]\n" \
  "v_pk_mul_f32 v[180:181], v[180:181], v[196:197]\n" \
  "s_add_u32 s62, s91, 0x500000\n" \
  "buffer_store_dwordx4 v[166:169], %[voffO], s[44:47], s62 offen offset:0 sc1\n" \
  "s_add_u32 s62, s91, 0x510000\n" \
  "buffer_store_dwordx4 v[170:173], %[voffO], s[44:47], s62 offen offset:0 sc1\n" \
  "s_add_u32 s62, s91, 0x520000\n" \
  "buffer_store_dwordx4 v[174:177], %[voffO], s[44:47], s62 offen offset:0 sc1\n" \
  "s_add_u32 s62, s91, 0x530000\n" \
  "buffer_store_dwordx4 v[178:181], %[voffO], s[44:47], s62 offen offset:0 sc1\n" \
  "v_accvgpr_read_b32 v214, a240\n" \
  "v_accvgpr_read_b32 v218, a241\n" \
  "v_accvgpr_read_b32 v222, a242\n" \
  "v_accvgpr_read_b32 v226, a243\n" \
  "v_accvgpr_read_b32 v215, a244\n" \
  "v_accvgpr_read_b32 v219, a245\n" \
  "v_accvgpr_read_b32 v223, a246\n" \
  "v_accvgpr_read_b32 v227, a247\n" \
  "v_accvgpr_read_b32 v216, a248\n" \
  "v_accvgpr_read_b32 v220, a249\n" \
  "v_accvgpr_read_b32 v224, a250\n" \
  "v_accvgpr_read_b32 v228, a251\n" \
  "v_accvgpr_read_b32 v217, a252\n" \
  "v_accvgpr_read_b32 v221, a253\n" \
  "v_accvgpr_read_b32 v225, a254\n" \
  "v_accvgpr_read_b32 v229, a255\n" \
  "s_waitcnt vmcnt(4)\n" \
  "v_accvgpr_read_b32 v198, a176\n" \
  "v_accvgpr_read_b32 v202, a177\n" \
  "v_accvgpr_read_b32 v206, a178\n" \
  "v_accvgpr_read_b32 v210, a179\n" \
  "v_accvgpr_read_b32 v199, a180\n" \
  "v_accvgpr_read_b32 v203, a181\n" \
  "v_accvgpr_read_b32 v207, a182\n" \
  "v_accvgpr_read_b32 v211, a183\n" \
  "v_accvgpr_read_b32 v200, a184\n" \
  "v_accvgpr_read_b32 v204, a185\n" \
  "v_accvgpr_read_b32 v208, a186\n" \
  "v_accvgpr_read_b32 v212, a187\n" \
  "v_accvgpr_read_b32 v201, a188\n" \
  "v_accvgpr_read_b32 v205, a189\n" \
  "v_accvgpr_read_b32 v209, a190\n" \
  "v_accvgpr_read_b32 v213, a191\n" \
  "v_pk_add_f32 v[198:199], v[198:199], v[214:215]\n" \
  "v_pk_add_f32 v[200:201], v[200:201], v[216:217]\n" \
  "v_pk_add_f32 v[202:203], v[202:203], v[218:219]\n" \
  "v_pk_add_f32 v[204:205], v[204:205], v[220:221]\n" \
  "v_pk_add_f32 v[206:207], v[206:207], v[222:223]\n" \
  "v_pk_add_f32 v[208:209], v[208:209], v[224:225]\n" \
  "v_pk_add_f32 v[210:211], v[210:211], v[226:227]\n" \
  "v_pk_add_f32 v[212:213], v[212:213], v[228:229]\n" \
  "v_pk_mul_f32 v[214:215], v[198:199], v[198:199]\n" \
  "v_pk_mul_f32 v[216:217], v[200:201], v[200:201]\n" \
  "v_pk_mul_f32 v[218:219], v[202:203], v[202:203]\n" \
  "v_pk_mul_f32 v[220:221], v[204:205], v[204:205]\n" \
  "v_pk_mul_f32 v[222:223], v[206:207], v[206:207]\n" \
  "v_pk_mul_f32 v[224:225], v[208:209], v[208:209]\n" \
  "v_pk_mul_f32 v[226:227], v[210:211], v[210:211]\n" \
  "v_pk_mul_f32 v[228:229], v[212:213], v[212:213]\n" \
  "v_pk_fma_f32 v[214:215], v[214:215], v[230:231], v[232:233]\n" \
  "v_pk_fma_f32 v[216:217], v[216:217], v[230:231], v[232:233]\n" \
  "v_pk_fma_f32 v[218:219], v[218:219], v[230:231], v[232:233]\n" \
  "v_pk_fma_f32 v[220:221], v[220:221], v[230:231], v[232:233]\n" \
  "v_pk_fma_f32 v[222:223], v[222:223], v[230:231], v[232:233]\n" \
  "v_pk_fma_f32 v[224:225], v[224:225], v[230:231], v[232:233]\n" \
  "v_pk_fma_f32 v[226:227], v[226:227], v[230:231], v[232:233]\n" \
  "v_pk_fma_f32 v[228:229], v[228:229], v[230:231], v[232:233]\n" \
  "v_pk_mul_f32 v[214:215], v[214:215], v[198:199]\n" \
  "v_pk_mul_f32 v[216:217], v[216:217], v[200:201]\n" \
  "v_pk_mul_f32 v[218:219], v[218:219], v[202:203]\n" \
  "v_pk_mul_f32 v[220:221], v[220:221], v[204:205]\n" \
  "v_pk_mul_f32 v[222:223], v[222:223], v[206:207]\n" \
  "v_pk_mul_f32 v[224:225], v[224:225], v[208:209]\n" \
  "v_pk_mul_f32 v[226:227], v[226:227], v[210:211]\n" \
  "v_pk_mul_f32 v[228:229], v[228:229], v[212:213]\n" \
  "v_exp_f32 v214, v214\n" \
  "v_exp_f32 v215, v215\n" \
  "v_exp_f32 v216, v216\n" \
  "v_exp_f32 v217, v217\n" \
  "v_exp_f32 v218, v218\n" \
  "v_exp_f32 v219, v219\n" \
  "v_exp_f32 v220, v220\n" \
  "v_exp_f32 v221, v221\n" \
  "v_exp_f32 v222, v222\n" \
  "v_exp_f32 v223, v223\n" \
  "v_exp_f32 v224, v224\n" \
  "v_exp_f32 v225, v225\n" \
  "v_exp_f32 v226, v226\n" \
  "v_exp_f32 v227, v227\n" \
  "v_exp_f32 v228, v228\n" \
  "v_exp_f32 v229, v229\n" \
  "v_pk_add_f32 v[214:215], v[214:215], 1.0 op_sel_hi:[1,0]\n" \
  "v_pk_add_f32 v[216:217], v[216:217], 1.0 op_sel_hi:[1,0]\n" \
  "v_pk_add_f32 v[218:219], v[218:219], 1.0 op_sel_hi:[1,0]\n" \
  "v_pk_add_f32 v[220:221], v[220:221], 1.0 op_sel_hi:[1,0]\n" \
  "v_pk_add_f32 v[222:223], v[222:223], 1.0 op_sel_hi:[1,0]\n" \
  "v_pk_add_f32 v[224:225], v[224:225], 1.0 op_sel_hi:[1,0]\n" \
  "v_pk_add_f32 v[226:227], v[226:227], 1.0 op_sel_hi:[1,0]\n" \
  "v_pk_add_f32 v[228:229], v[228:229], 1.0 op_sel_hi:[1,0]\n" \
  "v_rcp_f32 v214, v214\n" \
  "v_rcp_f32 v215, v215\n" \
  "v_rcp_f32 v216, v216\n" \
  "v_rcp_f32 v217, v217\n" \
  "v_rcp_f32 v218, v218\n" \
  "v_rcp_f32 v219, v219\n" \
  "v_rcp_f32 v220, v220\n" \
  "v_rcp_f32 v221, v221\n" \
  "v_rcp_f32 v222, v222\n" \
  "v_rcp_f32 v223, v223\n" \
  "v_rcp_f32 v224, v224\n" \
  "v_rcp_f32 v225, v225\n" \
  "v_rcp_f32 v226, v226\n" \
  "v_rcp_f32 v227, v227\n" \
  "v_rcp_f32 v228, v228\n" \
  "v_rcp_f32 v229, v229\n" \
  "v_pk_mul_f32 v[198:199], v[198:199], v[214:215]\n" \
  "v_pk_mul_f32 v[200:201], v[200:201], v[216:217]\n" \
  "v_pk_mul_f32 v[202:203], v[202:203], v[218:219]\n" \
  "v_pk_mul_f32 v[204:205], v[204:205], v[220:221]\n" \
  "v_pk_mul_f32 v[206:207], v[206:207], v[222:223]\n" \
  "v_pk_mul_f32 v[208:209], v[208:209], v[224:225]\n" \
  "v_pk_mul_f32 v[210:211], v[210:211], v[226:227]\n" \
  "v_pk_mul_f32 v[212:213], v[212:213], v[228:229]\n" \
  "s_add_u32 s62, s91, 0x500000\n" \
  "buffer_store_dwordx4 v[198:201], %[voffO], s[44:47], s62 offen offset:256 sc1\n" \
  "s_add_u32 s62, s91, 0x510000\n" \
  "buffer_store_dwordx4 v[202:205], %[voffO], s[44:47], s62 offen offset:256 sc1\n" \
  "s_add_u32 s62, s91, 0x520000\n" \
  "buffer_store_dwordx4 v[206:209], %[voffO], s[44:47], s62 offen offset:256 sc1\n" \
  "s_add_u32 s62, s91, 0x530000\n" \
  "buffer_store_dwordx4 v[210:213], %[voffO], s[44:47], s62 offen offset:256 sc1\n" \
  "s_branch L_done_%=\n" \
    \
  "L_ev_%=:\n" \
  "s_cmp_eq_u32 s80, 2\n" \
  "s_cbranch_scc1 L_ev2_%=\n" \
  "L_ev1_%=:\n" \
  "s_waitcnt vmcnt(16)\n" \
  "L_ev1b_%=:\n" \
  "v_and_b32 v230, 0xf0, %[bxr]\n" \
  "v_lshrrev_b32 v231, 8, %[bxr]\n" \
  "v_lshlrev_b32 v230, 3, v230\n" \
  "v_lshl_add_u32 v230, v231, 4, v230\n" \
  "v_add_u32 v250, %[xlds], v230\n" \
  "ds_read_b128 v[230:233], v250 offset:0\n" \
  "ds_read_b128 v[234:237], v250 offset:64\n" \
  "s_waitcnt lgkmcnt(0)\n" \
  "v_cvt_pk_f16_f32 v230, v230, v231\n" \
  "v_cvt_pk_f16_f32 v231, v232, v233\n" \
  "v_cvt_pk_f16_f32 v232, v234, v235\n" \
  "v_cvt_pk_f16_f32 v233, v236, v237\n" \
  "global_store_dwordx4 %[bxr], v[230:233], s[52:53] offset:0 sc0 sc1\n" \
  "s_nop 7\n" \
  "ds_read_b128 v[230:233], v250 offset:2048\n" \
  "ds_read_b128 v[234:237], v250 offset:2112\n" \
  "s_waitcnt lgkmcnt(0)\n" \
  "v_cvt_pk_f16_f32 v230, v230, v231\n" \
  "v_cvt_pk_f16_f32 v231, v232, v233\n" \
  "v_cvt_pk_f16_f32 v232, v234, v235\n" \
  "v_cvt_pk_f16_f32 v233, v236, v237\n" \
  "global_store_dwordx4 %[bxr], v[230:233], s[52:53] offset:1024 sc0 sc1\n" \
  "s_nop 7\n" \
  "ds_read_b128 v[230:233], v250 offset:4096\n" \
  "ds_read_b128 v[234:237], v250 offset:4160\n" \
  "s_waitcnt lgkmcnt(0)\n" \
  "v_cvt_pk_f16_f32 v230, v230, v231\n" \
  "v_cvt_pk_f16_f32 v231, v232, v233\n" \
  "v_cvt_pk_f16_f32 v232, v234, v235\n" \
  "v_cvt_pk_f16_f32 v233, v236, v237\n" \
  "global_store_dwordx4 %[bxr], v[230:233], s[52:53] offset:2048 sc0 sc1\n" \
  "s_nop 7\n" \
  "ds_read_b128 v[230:233], v250 offset:6144\n" \
  "ds_read_b128 v[234:237], v250 offset:6208\n" \
  "s_waitcnt lgkmcnt(0)\n" \
  "v_cvt_pk_f16_f32 v230, v230, v231\n" \
  "v_cvt_pk_f16_f32 v231, v232, v233\n" \
  "v_cvt_pk_f16_f32 v232, v234, v235\n" \
  "v_cvt_pk_f16_f32 v233, v236, v237\n" \
  "global_store_dwordx4 %[bxr], v[230:233], s[52:53] offset:3072 sc0 sc1\n" \
  "s_nop 7\n" \
  "s_mov_b32 s80, 2\n" \
  "s_add_u32 s77, s77, 3\n" \
  "s_add_u32 s54, s54, 4\n" \
  "s_addc_u32 s55, s55, 0\n" \
  "s_setpc_b64 s[54:55]\n" \
  "L_ev2_%=:\n" \
  "s_waitcnt vmcnt(39)\n" \
  "v_mov_b32 v250, s76\n" \
  "s_nop 0\n" \
  "global_store_dword v249, v250, s[56:57] sc0 sc1\n" \
  "s_add_u32 s54, s54, 4\n" \
  "s_addc_u32 s55, s55, 0\n" \
  "s_setpc_b64 s[54:55]\n" \
    \
  "L_wa_%=:\n" \
  "s_memrealtime s[48:49]\n" \
  "L_wa_%=l:\n" \
  "global_load_dwordx4 v[238:241], %[voffF], %[wsp] sc0 sc1\n" \
  "global_load_dword v250, v249, %[wsp] sc1\n" \
  "s_memrealtime s[50:51]\n" \
  "s_waitcnt vmcnt(0) lgkmcnt(0)\n" \
  "v_readfirstlane_b32 s79, v250\n" \
  "v_min3_u32 v248, v238, v239, v240\n" \
  "v_min_u32 v248, v248, v241\n" \
  "v_cmp_le_u32 s[34:35], s76, v248\n" \
  "s_nop 3\n" \
  "s_bitcmp1_b32 s34, s85\n" \
  "s_cbranch_scc1 L_wa_%=r\n" \
  "s_cmp_eq_u32 s79, s76\n" \
  "s_cbranch_scc1 L_wa_%=t\n" \
  "s_sub_u32 s50, s50, s48\n" \
  "s_cmp_gt_u32 s50, 190000000\n" \
  "s_cbranch_scc1 L_wa_%=t\n" \
  "s_sleep 1\n" \
  "s_branch L_wa_%=l\n" \
  "L_wa_%=t:\n" \
  "v_mov_b32 v250, s76\n" \
  "s_nop 0\n" \
  "global_store_dword v249, v250, %[wsp] sc1\n" \
  "s_mov_b32 s78, 1\n" \
  "L_wa_%=r:\n" \
  "s_memrealtime s[50:51]\n" \
  "s_waitcnt lgkmcnt(0)\n" \
  "s_sub_u32 s48, s50, s48\n" \
  "s_cmp_lt_u32 s48, 40\n" \
  "s_cbranch_scc1 L_wae_%=\n" \
  "s_cmp_lt_u32 s85, 4\n" \
  "s_cbranch_scc1 L_wae_%=\n" \
  "s_add_u32 s79, s50, %[phi]\n" \
  "s_add_u32 s81, s79, 150\n" \
  "L_wad_%=:\n" \
  "s_sleep 1\n" \
  "s_memrealtime s[48:49]\n" \
  "s_waitcnt lgkmcnt(0)\n" \
  "s_sub_u32 s48, s79, s48\n" \
  "s_cmp_gt_i32 s48, 0\n" \
  "s_cbranch_scc1 L_wad_%=\n" \
  "L_wae_%=:\n" \
  "s_add_u32 s54, s54, 4\n" \
  "s_addc_u32 s55, s55, 0\n" \
  "s_setpc_b64 s[54:55]\n" \
    \
  "L_wv_%=:\n" \
  "s_memrealtime s[48:49]\n" \
  "L_wv_%=l:\n" \
  "v_lshrrev_b32 v238, 1, %[bxr]\n" \
  "global_load_dwordx2 v[238:239], v238, %[wsp] offset:1024 sc1\n" \
  "global_load_dword v250, v249, %[wsp] sc1\n" \
  "s_memrealtime s[50:51]\n" \
  "s_waitcnt vmcnt(0) lgkmcnt(0)\n" \
  "v_readfirstlane_b32 s79, v250\n" \
  "v_xor_b32 v240, s85, v238\n" \
  "v_xor_b32 v241, s85, v239\n" \
  "v_or_b32 v248, v240, v241\n" \
  "v_cmp_lt_u32 vcc, 0xffff, v248\n" \
  "s_nop 4\n" \
  "s_cmp_eq_u64 vcc, 0\n" \
  "s_cbranch_scc1 L_wv_%=r\n" \
  "s_cmp_eq_u32 s79, s76\n" \
  "s_cbranch_scc1 L_wv_%=t\n" \
  "s_sub_u32 s50, s50, s48\n" \
  "s_cmp_gt_u32 s50, 190000000\n" \
  "s_cbranch_scc1 L_wv_%=t\n" \
  "s_sleep 1\n" \
  "s_branch L_wv_%=l\n" \
  "L_wv_%=t:\n" \
  "v_mov_b32 v250, s76\n" \
  "s_nop 0\n" \
  "global_store_dword v249, v250, %[wsp] sc1\n" \
  "s_mov_b32 s78, 1\n" \
  "L_wv_%=r:\n" \
  "s_add_u32 s54, s54, 4\n" \
  "s_addc_u32 s55, s55, 0\n" \
  "s_setpc_b64 s[54:55]\n" \
    \
  "L_wx0_%=:\n" \
  "s_memrealtime s[48:49]\n" \
  "L_wx0_%=l:\n" \
  "global_load_dwordx4 v[238:241], v249, %[xfi] sc1\n" \
  "global_load_dword v250, v249, %[wsp] sc1\n" \
  "s_memrealtime s[50:51]\n" \
  "s_waitcnt vmcnt(0) lgkmcnt(0)\n" \
  "v_readfirstlane_b32 s79, v250\n" \
  "v_readfirstlane_b32 s85, v238\n" \
  "s_nop 0\n" \
  "s_cmp_ge_u32 s85, s76\n" \
  "s_cbranch_scc1 L_wx0_%=r\n" \
  "s_cmp_eq_u32 s79, s76\n" \
  "s_cbranch_scc1 L_wx0_%=t\n" \
  "s_sub_u32 s50, s50, s48\n" \
  "s_cmp_gt_u32 s50, 190000000\n" \
  "s_cbranch_scc1 L_wx0_%=t\n" \
  "s_sleep 1\n" \
  "s_branch L_wx0_%=l\n" \
  "L_wx0_%=t:\n" \
  "v_mov_b32 v250, s76\n" \
  "s_nop 0\n" \
  "global_store_dword v249, v250, %[wsp] sc1\n" \
  "s_mov_b32 s78, 1\n" \
  "L_wx0_%=r:\n" \
  "s_add_u32 s54, s54, 4\n" \
  "s_addc_u32 s55, s55, 0\n" \
  "s_setpc_b64 s[54:55]\n" \
  "L_wx1_%=:\n" \
  "s_memrealtime s[48:49]\n" \
  "L_wx1_%=l:\n" \
  "global_load_dwordx4 v[238:241], v249, %[xfi] sc1\n" \
  "global_load_dword v250, v249, %[wsp] sc1\n" \
  "s_memrealtime s[50:51]\n" \
  "s_waitcnt vmcnt(0) lgkmcnt(0)\n" \
  "v_readfirstlane_b32 s79, v250\n" \
  "v_readfirstlane_b32 s85, v239\n" \
  "s_nop 0\n" \
  "s_cmp_ge_u32 s85, s76\n" \
  "s_cbranch_scc1 L_wx1_%=r\n" \
  "s_cmp_eq_u32 s79, s76\n" \
  "s_cbranch_scc1 L_wx1_%=t\n" \
  "s_sub_u32 s50, s50, s48\n" \
  "s_cmp_gt_u32 s50, 190000000\n" \
  "s_cbranch_scc1 L_wx1_%=t\n" \
  "s_sleep 1\n" \
  "s_branch L_wx1_%=l\n" \
  "L_wx1_%=t:\n" \
  "v_mov_b32 v250, s76\n" \
  "s_nop 0\n" \
  "global_store_dword v249, v250, %[wsp] sc1\n" \
  "s_mov_b32 s78, 1\n" \
  "L_wx1_%=r:\n" \
  "s_add_u32 s54, s54, 4\n" \
  "s_addc_u32 s55, s55, 0\n" \
  "s_setpc_b64 s[54:55]\n" \
  "L_wx2_%=:\n" \
  "s_memrealtime s[48:49]\n" \
  "L_wx2_%=l:\n" \
  "global_load_dwordx4 v[238:241], v249, %[xfi] sc1\n" \
  "global_load_dword v250, v249, %[wsp] sc1\n" \
  "s_memrealtime s[50:51]\n" \
  "s_waitcnt vmcnt(0) lgkmcnt(0)\n" \
  "v_readfirstlane_b32 s79, v250\n" \
  "v_readfirstlane_b32 s85, v240\n" \
  "s_nop 0\n" \
  "s_cmp_ge_u32 s85, s76\n" \
  "s_cbranch_scc1 L_wx2_%=r\n" \
  "s_cmp_eq_u32 s79, s76\n" \
  "s_cbranch_scc1 L_wx2_%=t\n" \
  "s_sub_u32 s50, s50, s48\n" \
  "s_cmp_gt_u32 s50, 190000000\n" \
  "s_cbranch_scc1 L_wx2_%=t\n" \
  "s_sleep 1\n" \
  "s_branch L_wx2_%=l\n" \
  "L_wx2_%=t:\n" \
  "v_mov_b32 v250, s76\n" \
  "s_nop 0\n" \
  "global_store_dword v249, v250, %[wsp] sc1\n" \
  "s_mov_b32 s78, 1\n" \
  "L_wx2_%=r:\n" \
  "s_add_u32 s54, s54, 4\n" \
  "s_addc_u32 s55, s55, 0\n" \
  "s_setpc_b64 s[54:55]\n" \
  "L_wx3_%=:\n" \
  "s_memrealtime s[48:49]\n" \
  "L_wx3_%=l:\n" \
  "global_load_dwordx4 v[238:241], v249, %[xfi] sc1\n" \
  "global_load_dword v250, v249, %[wsp] sc1\n" \
  "s_memrealtime s[50:51]\n" \
  "s_waitcnt vmcnt(0) lgkmcnt(0)\n" \
  "v_readfirstlane_b32 s79, v250\n" \
  "v_readfirstlane_b32 s85, v241\n" \
  "s_nop 0\n" \
  "s_cmp_ge_u32 s85, s76\n" \
  "s_cbranch_scc1 L_wx3_%=r\n" \
  "s_cmp_eq_u32 s79, s76\n" \
  "s_cbranch_scc1 L_wx3_%=t\n" \
  "s_sub_u32 s50, s50, s48\n" \
  "s_cmp_gt_u32 s50, 190000000\n" \
  "s_cbranch_scc1 L_wx3_%=t\n" \
  "s_sleep 1\n" \
  "s_branch L_wx3_%=l\n" \
  "L_wx3_%=t:\n" \
  "v_mov_b32 v250, s76\n" \
  "s_nop 0\n" \
  "global_store_dword v249, v250, %[wsp] sc1\n" \
  "s_mov_b32 s78, 1\n" \
  "L_wx3_%=r:\n" \
  "s_add_u32 s54, s54, 4\n" \
  "s_addc_u32 s55, s55, 0\n" \
  "s_setpc_b64 s[54:55]\n" \
    \
  "L_pace_%=:\n" \
  "s_sleep 1\n" \
  "s_memrealtime s[48:49]\n" \
  "s_waitcnt lgkmcnt(0)\n" \
  "s_sub_u32 s48, s50, s48\n" \
  "s_cmp_gt_i32 s48, 0\n" \
  "s_cbranch_scc1 L_pace_%=\n" \
  "s_add_u32 s54, s54, 4\n" \
  "s_addc_u32 s55, s55, 0\n" \
  "s_setpc_b64 s[54:55]\n" \
  "L_pl1_%=:\n" \
  "global_load_dwordx4 v[238:241], %[voffF], %[wsp] sc0 sc1\n" \
  "s_branch L_pb1_%=\n" \
  "L_pl2_%=:\n" \
  "global_load_dwordx4 v[238:241], %[voffF], %[wsp] sc0 sc1\n" \
  "s_branch L_pb2_%=\n" \
  "L_pl3_%=:\n" \
  "global_load_dwordx4 v[238:241], %[voffF], %[wsp] sc0 sc1\n" \
  "s_branch L_pb3_%=\n" \
  "L_pl4_%=:\n" \
  "global_load_dwordx4 v[238:241], %[voffF], %[wsp] sc0 sc1\n" \
  "s_branch L_pb4_%=\n" \
  "L_pl5_%=:\n" \
  "global_load_dwordx4 v[238:241], %[voffF], %[wsp] sc0 sc1\n" \
  "s_branch L_pb5_%=\n" \
  "L_pl6_%=:\n" \
  "global_load_dwordx4 v[238:241], %[voffF], %[wsp] sc0 sc1\n" \
  "s_branch L_pb6_%=\n" \
  "L_pl7_%=:\n" \
  "global_load_dwordx4 v[238:241], %[voffF], %[wsp] sc0 sc1\n" \
  "s_branch L_pb7_%=\n" \
  "L_pl8_%=:\n" \
  "global_load_dwordx4 v[238:241], %[voffF], %[wsp] sc0 sc1\n" \
  "s_branch L_pb8_%=\n" \
  "L_pl9_%=:\n" \
  "global_load_dwordx4 v[238:241], %[voffF], %[wsp] sc0 sc1\n" \
  "s_branch L_pb9_%=\n" \
  "L_pl10_%=:\n" \
  "global_load_dwordx4 v[238:241], %[voffF], %[wsp] sc0 sc1\n" \
  "s_branch L_pb10_%=\n" \
  "L_pl11_%=:\n" \
  "global_load_dwordx4 v[238:241], %[voffF], %[wsp] sc0 sc1\n" \
  "s_branch L_pb11_%=\n" \
  "L_pl12_%=:\n" \
  "global_load_dwordx4 v[238:241], %[voffF], %[wsp] sc0 sc1\n" \
  "s_branch L_pb12_%=\n" \
  "L_pl13_%=:\n" \
  "global_load_dwordx4 v[238:241], %[voffF], %[wsp] sc0 sc1\n" \
  "s_branch L_pb13_%=\n" \
  "L_pl14_%=:\n" \
  "global_load_dwordx4 v[238:241], %[voffF], %[wsp] sc0 sc1\n" \
  "s_branch L_pb14_%=\n" \
  "L_pl15_%=:\n" \
  "global_load_dwordx4 v[238:241], %[voffF], %[wsp] sc0 sc1\n" \
  "s_branch L_pb15_%=\n" \
  "L_pl16_%=:\n" \
  "global_load_dwordx4 v[238:241], %[voffF], %[wsp] sc0 sc1\n" \
  "s_branch L_pb16_%=\n" \
  "L_pl17_%=:\n" \
  "global_load_dwordx4 v[238:241], %[voffF], %[wsp] sc0 sc1\n" \
  "s_branch L_pb17_%=\n" \
  "L_pl18_%=:\n" \
  "global_load_dwordx4 v[238:241], %[voffF], %[wsp] sc0 sc1\n" \
  "s_branch L_pb18_%=\n" \
  "L_pl19_%=:\n" \
  "global_load_dwordx4 v[238:241], %[voffF], %[wsp] sc0 sc1\n" \
  "s_branch L_pb19_%=\n" \
  "L_pl20_%=:\n" \
  "global_load_dwordx4 v[238:241], %[voffF], %[wsp] sc0 sc1\n" \
  "s_branch L_pb20_%=\n" \
  "L_pl21_%=:\n" \
  "global_load_dwordx4 v[238:241], %[voffF], %[wsp] sc0 sc1\n" \
  "s_branch L_pb21_%=\n" \
  "L_s22_%=:\n" \
  "s_and_b32 s50, s93, 0\n" \
  "s_max_i32 s50, s50, s82\n" \
  "s_add_u32 s50, s50, s58\n" \
  "s_getpc_b64 s[54:55]\n" \
  "s_branch L_pace_%=\n" \
  "s_branch L_b22_%=\n" \
  "L_s23_%=:\n" \
  "s_and_b32 s50, s93, 0\n" \
  "s_max_i32 s50, s50, s82\n" \
  "s_add_u32 s50, s50, s58\n" \
  "s_getpc_b64 s[54:55]\n" \
  "s_branch L_pace_%=\n" \
  "s_branch L_b23_%=\n" \
  "L_s24_%=:\n" \
  "s_and_b32 s50, s93, 0\n" \
  "s_max_i32 s50, s50, s82\n" \
  "s_add_u32 s50, s50, s58\n" \
  "s_getpc_b64 s[54:55]\n" \
  "s_branch L_pace_%=\n" \
  "s_branch L_b24_%=\n" \
  "L_s25_%=:\n" \
  "s_and_b32 s50, s93, 0\n" \
  "s_max_i32 s50, s50, s82\n" \
  "s_add_u32 s50, s50, s58\n" \
  "s_getpc_b64 s[54:55]\n" \
  "s_branch L_pace_%=\n" \
  "s_branch L_b25_%=\n" \
  "L_s26_%=:\n" \
  "s_and_b32 s50, s93, 34\n" \
  "s_max_i32 s50, s50, s82\n" \
  "s_add_u32 s50, s50, s58\n" \
  "s_getpc_b64 s[54:55]\n" \
  "s_branch L_pace_%=\n" \
  "s_branch L_b26_%=\n" \
  "L_s27_%=:\n" \
  "s_and_b32 s50, s93, 34\n" \
  "s_max_i32 s50, s50, s82\n" \
  "s_add_u32 s50, s50, s58\n" \
  "s_getpc_b64 s[54:55]\n" \
  "s_branch L_pace_%=\n" \
  "s_branch L_b27_%=\n" \
  "L_s28_%=:\n" \
  "s_and_b32 s50, s93, 34\n" \
  "s_max_i32 s50, s50, s82\n" \
  "s_add_u32 s50, s50, s58\n" \
  "s_getpc_b64 s[54:55]\n" \
  "s_branch L_pace_%=\n" \
  "s_branch L_b28_%=\n" \
  "L_s29_%=:\n" \
  "s_and_b32 s50, s93, 34\n" \
  "s_max_i32 s50, s50, s82\n" \
  "s_add_u32 s50, s50, s58\n" \
  "s_getpc_b64 s[54:55]\n" \
  "s_branch L_pace_%=\n" \
  "s_branch L_b29_%=\n" \
  "L_s30_%=:\n" \
  "s_and_b32 s50, s93, 34\n" \
  "s_max_i32 s50, s50, s82\n" \
  "s_add_u32 s50, s50, s58\n" \
  "s_getpc_b64 s[54:55]\n" \
  "s_branch L_pace_%=\n" \
  "s_branch L_b30_%=\n" \
  "L_s31_%=:\n" \
  "s_and_b32 s50, s93, 34\n" \
  "s_max_i32 s50, s50, s82\n" \
  "s_add_u32 s50, s50, s58\n" \
  "s_getpc_b64 s[54:55]\n" \
  "s_branch L_pace_%=\n" \
  "s_branch L_b31_%=\n" \
  "L_s32_%=:\n" \
  "s_and_b32 s50, s93, 34\n" \
  "s_max_i32 s50, s50, s82\n" \
  "s_add_u32 s50, s50, s58\n" \
  "s_getpc_b64 s[54:55]\n" \
  "s_branch L_pace_%=\n" \
  "s_branch L_b32_%=\n" \
  "L_s33_%=:\n" \
  "s_and_b32 s50, s93, 34\n" \
  "s_max_i32 s50, s50, s82\n" \
  "s_add_u32 s50, s50, s58\n" \
  "s_getpc_b64 s[54:55]\n" \
  "s_branch L_pace_%=\n" \
  "s_branch L_b33_%=\n" \
  "L_s34_%=:\n" \
  "s_and_b32 s50, s93, 34\n" \
  "s_max_i32 s50, s50, s82\n" \
  "s_add_u32 s50, s50, s58\n" \
  "s_getpc_b64 s[54:55]\n" \
  "s_branch L_pace_%=\n" \
  "s_branch L_b34_%=\n" \
  "L_s35_%=:\n" \
  "s_and_b32 s50, s93, 34\n" \
  "s_max_i32 s50, s50, s82\n" \
  "s_add_u32 s50, s50, s58\n" \
  "s_getpc_b64 s[54:55]\n" \
  "s_branch L_pace_%=\n" \
  "s_branch L_b35_%=\n" \
  "L_s36_%=:\n" \
  "s_and_b32 s50, s93, 36\n" \
  "s_max_i32 s50, s50, s82\n" \
  "s_add_u32 s50, s50, s58\n" \
  "s_getpc_b64 s[54:55]\n" \
  "s_branch L_pace_%=\n" \
  "s_branch L_b36_%=\n" \
  "L_s37_%=:\n" \
  "s_and_b32 s50, s93, 18\n" \
  "s_max_i32 s50, s50, s82\n" \
  "s_add_u32 s50, s50, s58\n" \
  "s_getpc_b64 s[54:55]\n" \
  "s_branch L_pace_%=\n" \
  "s_branch L_b37_%=\n" \
  "L_s38_%=:\n" \
  "s_and_b32 s50, s93, 0\n" \
  "s_max_i32 s50, s50, s82\n" \
  "s_add_u32 s50, s50, s58\n" \
  "s_getpc_b64 s[54:55]\n" \
  "s_branch L_pace_%=\n" \
  "s_branch L_b38_%=\n" \
  "L_s39_%=:\n" \
  "s_and_b32 s50, s93, 0\n" \
  "s_max_i32 s50, s50, s82\n" \
  "s_add_u32 s50, s50, s58\n" \
  "s_getpc_b64 s[54:55]\n" \
  "s_branch L_pace_%=\n" \
  "s_branch L_b39_%=\n" \
  "L_s40_%=:\n" \
  "s_and_b32 s50, s93, 0\n" \
  "s_max_i32 s50, s50, s82\n" \
  "s_add_u32 s50, s50, s58\n" \
  "s_getpc_b64 s[54:55]\n" \
  "s_branch L_pace_%=\n" \
  "s_branch L_b40_%=\n" \
  "L_s41_%=:\n" \
  "s_and_b32 s50, s93, 0\n" \
  "s_max_i32 s50, s50, s82\n" \
  "s_add_u32 s50, s50, s58\n" \
  "s_getpc_b64 s[54:55]\n" \
  "s_branch L_pace_%=\n" \
  "s_branch L_b41_%=\n" \
  "L_s42_%=:\n" \
  "s_and_b32 s50, s93, 0\n" \
  "s_max_i32 s50, s50, s82\n" \
  "s_add_u32 s50, s50, s58\n" \
  "s_getpc_b64 s[54:55]\n" \
  "s_branch L_pace_%=\n" \
  "s_branch L_b42_%=\n" \
  "L_s43_%=:\n" \
  "s_and_b32 s50, s93, 0\n" \
  "s_max_i32 s50, s50, s82\n" \
  "s_add_u32 s50, s50, s58\n" \
  "s_getpc_b64 s[54:55]\n" \
  "s_branch L_pace_%=\n" \
  "s_branch L_b43_%=\n" \
  "L_s44_%=:\n" \
  "s_and_b32 s50, s93, 0\n" \
  "s_max_i32 s50, s50, s82\n" \
  "s_add_u32 s50, s50, s58\n" \
  "s_getpc_b64 s[54:55]\n" \
  "s_branch L_pace_%=\n" \
  "s_branch L_b44_%=\n" \
  "L_s45_%=:\n" \
  "s_getpc_b64 s[54:55]\n" \
  "s_branch L_wv_%=\n" \
  "s_branch L_b45_%=\n" \
  "L_s46_%=:\n" \
  "s_mov_b32 s85, 1\n" \
  "s_getpc_b64 s[54:55]\n" \
  "s_branch L_wa_%=\n" \
  "s_branch L_b46_%=\n" \
  "L_s47_%=:\n" \
  "s_mov_b32 s85, 2\n" \
  "s_getpc_b64 s[54:55]\n" \
  "s_branch L_wa_%=\n" \
  "s_branch L_b47_%=\n" \
  "L_s48_%=:\n" \
  "s_mov_b32 s85, 3\n" \
  "s_getpc_b64 s[54:55]\n" \
  "s_branch L_wa_%=\n" \
  "s_branch L_b48_%=\n" \
  "L_s49_%=:\n" \
  "s_mov_b32 s85, 4\n" \
  "s_getpc_b64 s[54:55]\n" \
  "s_branch L_wa_%=\n" \
  "s_branch L_b49_%=\n" \
  "L_s50_%=:\n" \
  "s_mov_b32 s85, 5\n" \
  "s_getpc_b64 s[54:55]\n" \
  "s_branch L_wa_%=\n" \
  "s_branch L_b50_%=\n" \
  "L_s51_%=:\n" \
  "s_mov_b32 s85, 6\n" \
  "s_getpc_b64 s[54:55]\n" \
  "s_branch L_wa_%=\n" \
  "s_branch L_b51_%=\n" \
  "L_s52_%=:\n" \
  "s_mov_b32 s85, 7\n" \
  "s_getpc_b64 s[54:55]\n" \
  "s_branch L_wa_%=\n" \
  "s_branch L_b52_%=\n" \
  "L_s53_%=:\n" \
  "s_mov_b32 s85, 8\n" \
  "s_getpc_b64 s[54:55]\n" \
  "s_branch L_wa_%=\n" \
  "s_branch L_b53_%=\n" \
  "L_s54_%=:\n" \
  "s_mov_b32 s85, 9\n" \
  "s_getpc_b64 s[54:55]\n" \
  "s_branch L_wa_%=\n" \
  "s_branch L_b54_%=\n" \
  "L_s55_%=:\n" \
  "s_mov_b32 s85, 10\n" \
  "s_getpc_b64 s[54:55]\n" \
  "s_branch L_wa_%=\n" \
  "s_branch L_b55_%=\n" \
  "L_s56_%=:\n" \
  "s_mov_b32 s85, 11\n" \
  "s_getpc_b64 s[54:55]\n" \
  "s_branch L_wa_%=\n" \
  "s_branch L_b56_%=\n" \
  "L_s57_%=:\n" \
  "s_mov_b32 s85, 12\n" \
  "s_getpc_b64 s[54:55]\n" \
  "s_branch L_wa_%=\n" \
  "s_branch L_b57_%=\n" \
  "L_s58_%=:\n" \
  "s_mov_b32 s85, 13\n" \
  "s_getpc_b64 s[54:55]\n" \
  "s_branch L_wa_%=\n" \
  "s_branch L_b58_%=\n" \
  "L_s59_%=:\n" \
  "s_mov_b32 s85, 14\n" \
  "s_getpc_b64 s[54:55]\n" \
  "s_branch L_wa_%=\n" \
  "s_branch L_b59_%=\n" \
  "L_s60_%=:\n" \
  "s_mov_b32 s85, 15\n" \
  "s_getpc_b64 s[54:55]\n" \
  "s_branch L_wa_%=\n" \
  "s_branch L_b60_%=\n" \
  "L_s61_%=:\n" \
  "s_mov_b32 s85, 16\n" \
  "s_getpc_b64 s[54:55]\n" \
  "s_branch L_wa_%=\n" \
  "s_branch L_b61_%=\n" \
  "L_s62_%=:\n" \
  "s_mov_b32 s85, 17\n" \
  "s_getpc_b64 s[54:55]\n" \
  "s_branch L_wa_%=\n" \
  "s_branch L_b62_%=\n" \
  "L_s63_%=:\n" \
  "s_mov_b32 s85, 18\n" \
  "s_getpc_b64 s[54:55]\n" \
  "s_branch L_wa_%=\n" \
  "s_branch L_b63_%=\n" \
  "L_s64_%=:\n" \
  "s_mov_b32 s85, 19\n" \
  "s_getpc_b64 s[54:55]\n" \
  "s_branch L_wa_%=\n" \
  "s_branch L_b64_%=\n" \
  "L_s65_%=:\n" \
  "s_mov_b32 s85, 20\n" \
  "s_getpc_b64 s[54:55]\n" \
  "s_branch L_wa_%=\n" \
  "s_branch L_b65_%=\n" \
  "L_s66_%=:\n" \
  "s_mov_b32 s85, 21\n" \
  "s_getpc_b64 s[54:55]\n" \
  "s_branch L_wa_%=\n" \
  "s_branch L_b66_%=\n" \
  "L_s67_%=:\n" \
  "s_mov_b32 s85, 22\n" \
  "s_getpc_b64 s[54:55]\n" \
  "s_branch L_wa_%=\n" \
  "s_branch L_b67_%=\n" \
  "L_s68_%=:\n" \
  "s_mov_b32 s85, 23\n" \
  "s_getpc_b64 s[54:55]\n" \
  "s_branch L_wa_%=\n" \
  "s_branch L_b68_%=\n" \
  "L_s69_%=:\n" \
  "s_mov_b32 s85, 24\n" \
  "s_getpc_b64 s[54:55]\n" \
  "s_branch L_wa_%=\n" \
  "s_branch L_b69_%=\n" \
  "L_s70_%=:\n" \
  "s_mov_b32 s85, 25\n" \
  "s_getpc_b64 s[54:55]\n" \
  "s_branch L_wa_%=\n" \
  "s_branch L_b70_%=\n" \
  "L_s71_%=:\n" \
  "s_mov_b32 s85, 26\n" \
  "s_getpc_b64 s[54:55]\n" \
  "s_branch L_wa_%=\n" \
  "s_branch L_b71_%=\n" \
  "L_s72_%=:\n" \
  "s_mov_b32 s85, 27\n" \
  "s_getpc_b64 s[54:55]\n" \
  "s_branch L_wa_%=\n" \
  "s_branch L_b72_%=\n" \
  "L_s73_%=:\n" \
  "s_mov_b32 s85, 28\n" \
  "s_getpc_b64 s[54:55]\n" \
  "s_branch L_wa_%=\n" \
  "s_branch L_b73_%=\n" \
  "L_s74_%=:\n" \
  "s_mov_b32 s85, 29\n" \
  "s_getpc_b64 s[54:55]\n" \
  "s_branch L_wa_%=\n" \
  "s_branch L_b74_%=\n" \
  "L_s75_%=:\n" \
  "s_mov_b32 s85, 30\n" \
  "s_getpc_b64 s[54:55]\n" \
  "s_branch L_wa_%=\n" \
  "s_branch L_b75_%=\n" \
  "L_s76_%=:\n" \
  "s_mov_b32 s85, 31\n" \
  "s_getpc_b64 s[54:55]\n" \
  "s_branch L_wa_%=\n" \
  "s_branch L_b76_%=\n" \
  "L_s77_%=:\n" \
  "s_getpc_b64 s[54:55]\n" \
  "s_branch L_ev_%=\n" \
  "s_branch L_b77_%=\n" \
  "L_s78_%=:\n" \
  "s_getpc_b64 s[54:55]\n" \
  "s_branch L_ev_%=\n" \
  "s_branch L_b78_%=\n" \
  "L_s79_%=:\n" \
  "s_getpc_b64 s[54:55]\n" \
  "s_branch L_ev_%=\n" \
  "s_branch L_b79_%=\n" \
  "L_s80_%=:\n" \
  "s_getpc_b64 s[54:55]\n" \
  "s_branch L_ev_%=\n" \
  "s_branch L_b80_%=\n" \
  "L_s81_%=:\n" \
  "s_getpc_b64 s[54:55]\n" \
  "s_branch L_ev_%=\n" \
  "s_branch L_b81_%=\n" \
  "L_s82_%=:\n" \
  "s_getpc_b64 s[54:55]\n" \
  "s_branch L_ev_%=\n" \
  "s_branch L_b82_%=\n" \
  "L_s83_%=:\n" \
  "s_getpc_b64 s[54:55]\n" \
  "s_branch L_ev_%=\n" \
  "s_branch L_b83_%=\n" \
  "L_s84_%=:\n" \
  "s_getpc_b64 s[54:55]\n" \
  "s_branch L_ev_%=\n" \
  "s_branch L_b84_%=\n" \
  "L_s85_%=:\n" \
  "s_getpc_b64 s[54:55]\n" \
  "s_branch L_ev_%=\n" \
  "s_branch L_b85_%=\n" \
  "L_s86_%=:\n" \
  "s_getpc_b64 s[54:55]\n" \
  "s_branch L_ev_%=\n" \
  "s_branch L_b86_%=\n" \
  "L_s87_%=:\n" \
  "s_getpc_b64 s[54:55]\n" \
  "s_branch L_ev_%=\n" \
  "s_branch L_b87_%=\n" \
  "L_s88_%=:\n" \
  "s_getpc_b64 s[54:55]\n" \
  "s_branch L_ev_%=\n" \
  "s_branch L_b88_%=\n" \
  "L_s89_%=:\n" \
  "s_getpc_b64 s[54:55]\n" \
  "s_branch L_ev_%=\n" \
  "s_branch L_b89_%=\n" \
  "L_s90_%=:\n" \
  "s_getpc_b64 s[54:55]\n" \
  "s_branch L_ev_%=\n" \
  "s_branch L_b90_%=\n" \
  "L_s91_%=:\n" \
  "s_getpc_b64 s[54:55]\n" \
  "s_branch L_wx0_%=\n" \
  "s_branch L_b91_%=\n" \
  "L_s92_%=:\n" \
  "s_getpc_b64 s[54:55]\n" \
  "s_branch L_wx1_%=\n" \
  "s_branch L_b92_%=\n" \
  "L_s93_%=:\n" \
  "s_getpc_b64 s[54:55]\n" \
  "s_branch L_wx2_%=\n" \
  "s_branch L_b93_%=\n" \
  "L_s94_%=:\n" \
  "s_getpc_b64 s[54:55]\n" \
  "s_branch L_wx3_%=\n" \
  "s_branch L_b94_%=\n" \
  "L_s95_%=:\n" \
  "s_mov_b32 s85, 0\n" \
  "s_getpc_b64 s[54:55]\n" \
  "s_branch L_wa_%=\n" \
  "s_branch L_b95_%=\n" \
  "L_pl21x_%=:\n" \
  "global_load_dwordx4 v[238:241], %[voffF], %[wsp] sc0 sc1\n" \
  "s_branch L_pb21x_%=\n" \
  "L_s73x_%=:\n" \
  "s_mov_b32 s85, 28\n" \
  "s_getpc_b64 s[54:55]\n" \
  "s_branch L_wa_%=\n" \
  "s_branch L_b73x_%=\n" \
  "L_s74x_%=:\n" \
  "s_mov_b32 s85, 29\n" \
  "s_getpc_b64 s[54:55]\n" \
  "s_branch L_wa_%=\n" \
  "s_branch L_b74x_%=\n" \
  "L_s75x_%=:\n" \
  "s_mov_b32 s85, 30\n" \
  "s_getpc_b64 s[54:55]\n" \
  "s_branch L_wa_%=\n" \
  "s_branch L_b75x_%=\n" \
  "L_s76x_%=:\n" \
  "s_mov_b32 s85, 31\n" \
  "s_getpc_b64 s[54:55]\n" \
  "s_branch L_wa_%=\n" \
  "s_branch L_b76x_%=\n" \
  "L_s91x_%=:\n" \
  "s_getpc_b64 s[54:55]\n" \
  "s_branch L_wx0_%=\n" \
  "s_branch L_b91x_%=\n" \
  "L_s92x_%=:\n" \
  "s_getpc_b64 s[54:55]\n" \
  "s_branch L_wx1_%=\n" \
  "s_branch L_b92x_%=\n" \
  "L_s93x_%=:\n" \
  "s_getpc_b64 s[54:55]\n" \
  "s_branch L_wx2_%=\n" \
  "s_branch L_b93x_%=\n" \
  "L_s94x_%=:\n" \
  "s_getpc_b64 s[54:55]\n" \
  "s_branch L_wx3_%=\n" \
  "s_branch L_b94x_%=\n" \
  "L_done_%=:\n" \
  "s_memrealtime s[48:49]\n" \
  "s_waitcnt lgkmcnt(0)\n" \
  "v_mov_b32 v248, s48\n" \
  "s_nop 0\n" \
  "global_store_dword v249, v248, %[callp] offset:4 sc1\n" \
  ""
#define GEMM_CLOB "v14","v15","v16","v17","v18","v19","v20","v21","v22","v23","v24","v25","v26","v27","v28","v29","v30","v31","v32","v33","v34","v35","v36","v37","v38","v39","v40","v41","v42","v43","v44","v45","v46","v47","v48","v49","v50","v51","v52","v53","v54","v55","v56","v57","v58","v59","v60","v61","v62","v63","v64","v65","v66","v67","v68","v69","v70","v71","v72","v73","v74","v75","v76","v77","v78","v79","v80","v81","v82","v83","v84","v85","v86","v87","v88","v89","v90","v91","v92","v93","v94","v95","v96","v97","v98","v99","v100","v101","v102","v103","v104","v105","v106","v107","v108","v109","v110","v111","v112","v113","v114","v115","v116","v117","v118","v119","v120","v121","v122","v123","v124","v125","v126","v127","v128","v129","v130","v131","v132","v133","v134","v135","v136","v137","v138","v139","v140","v141","v142","v143","v144","v145","v146","v147","v148","v149","v150","v151","v152","v153","v154","v155","v156","v157","v158","v159","v160","v161","v162","v163","v164","v165","v166","v167","v168","v169","v170","v171","v172","v173","v174","v175","v176","v177","v178","v179","v180","v181","v182","v183","v184","v185","v186","v187","v188","v189","v190","v191","v192","v193","v194","v195","v196","v197","v198","v199","v200","v201","v202","v203","v204","v205","v206","v207","v208","v209","v210","v211","v212","v213","v214","v215","v216","v217","v218","v219","v220","v221","v222","v223","v224","v225","v226","v227","v228","v229","v230","v231","v232","v233","v234","v235","v236","v237","v238","v239","v240","v241","v242","v243","v244","v245","v246","v247","v248","v249","v250","v251","v252","v253","v254","v255","a0","a1","a2","a3","a4","a5","a6","a7","a8","a9","a10","a11","a12","a13","a14","a15","a16","a17","a18","a19","a20","a21","a22","a23","a24","a25","a26","a27","a28","a29","a30","a31","a32","a33","a34","a35","a36","a37","a38","a39","a40","a41","a42","a43","a44","a45","a46","a47","a48","a49","a50","a51","a52","a53","a54","a55","a56","a57","a58","a59","a60","a61","a62","a63","a64","a65","a66","a67","a68","a69","a70","a71","a72","a73","a74","a75","a76","a77","a78","a79","a80","a81","a82","a83","a84","a85","a86","a87","a88","a89","a90","a91","a92","a93","a94","a95","a96","a97","a98","a99","a100","a101","a102","a103","a104","a105","a106","a107","a108","a109","a110","a111","a112","a113","a114","a115","a116","a117","a118","a119","a120","a121","a122","a123","a124","a125","a126","a127","a128","a129","a130","a131","a132","a133","a134","a135","a136","a137","a138","a139","a140","a141","a142","a143","a144","a145","a146","a147","a148","a149","a150","a151","a152","a153","a154","a155","a156","a157","a158","a159","a160","a161","a162","a163","a164","a165","a166","a167","a168","a169","a170","a171","a172","a173","a174","a175","a176","a177","a178","a179","a180","a181","a182","a183","a184","a185","a186","a187","a188","a189","a190","a191","a192","a193","a194","a195","a196","a197","a198","a199","a200","a201","a202","a203","a204","a205","a206","a207","a208","a209","a210","a211","a212","a213","a214","a215","a216","a217","a218","a219","a220","a221","a222","a223","a224","a225","a226","a227","a228","a229","a230","a231","a232","a233","a234","a235","a236","a237","a238","a239","a240","a241","a242","a243","a244","a245","a246","a247","a248","a249","a250","a251","a252","a253","a254","a255","s34","s35","s36","s37","s38","s39","s40","s41","s42","s43","s44","s45","s46","s47","s48","s49","s50","s51","s52","s53","s54","s55","s56","s57","s58","s59","s60","s61","s62","s63","s64","s65","s66","s67","s68","s69","s70","s71","s72","s73","s74","s75","s76","s77","s78","s79","s80","s81","s82","s83","s84","s85","s86","s87","s88","s89","s90","s91","s92","s93","s94","memory","scc","vcc"
#define LFL 1
#define SPC 3
#define SLOT0 0
#define NSTATIC 3
#define NEVENT 14
#define EVD 3


constexpr int NR = 8;
constexpr unsigned LDS_BYTES = 163840;
constexpr int PHI = 7, PHIC = 15;
constexpr size_t WS_TMO = 0;
constexpr size_t WS_VOTE = 1024;
constexpr size_t WS_CALL = 4096;
constexpr size_t WS_AFLAG = 16384;
constexpr size_t WS_LFLAG = 32768;
constexpr size_t WS_XFLAG = 65536;
constexpr size_t WS_A = 1u << 20;
constexpr size_t WS_END = WS_A + (8u << 20);

struct Ptrs { char* own; char* const* tab; };

__global__ __launch_bounds__(256, 1) void a2a_gemm_gelu(const float* __restrict__ X, const float* __restrict__ W, float* out, Ptrs P8, unsigned rank) {
  const unsigned b = blockIdx.x, lab = b & 7, idx = b >> 3, role = lab >> 2, cls = lab & 3;
  const unsigned tid = threadIdx.x, lane = tid & 63;
  const unsigned w = __builtin_amdgcn_readfirstlane(tid >> 6);
  const unsigned gw = b * 4 + w;
  char* const ws = P8.own;
  const unsigned Pl = b >> 6, pk = (b + idx) & 7, v = (idx & 7) * 4 + w;
  const unsigned locm = 0u - ((pk + 1) >> 3), firstm = 0u - (unsigned)(Pl == 0);
  const int slot = (int)(((SLOT0 + (Pl - 1) * SPC + (((idx & 7) * SPC) >> 3)) | locm | firstm) + locm);
  const unsigned d = (rank + 7 - pk) & 7;
  const unsigned ukh = v >> 4, ucw = (v >> 2) & 3, q = v & 3;
  const unsigned xoff0 = (d * 512 + ucw * 128 + q * 32 + (lane >> 3)) * 2048u + (Pl * 4 + ukh) * 128u + (lane & 7) * 16u;
  const unsigned xoff1 = xoff0 + 32768u;
  const unsigned xlds = 131072 + w * 8192;
  char* const callp = ws + WS_CALL + gw * 8;
  asm volatile(GEMM_ASM1 :: [callp] "s"(callp), [slot] "s"(slot), [xptr] "s"(X), [xlds] "s"(xlds), [xoff0] "v"(xoff0), [xoff1] "v"(xoff1) : GEMM_CLOB1);
  __builtin_amdgcn_sched_barrier(0);

  const unsigned nt = (idx & 15) | ((cls >> 1) << 4) | ((idx >> 4) << 5) | ((cls & 1) << 6);
  const unsigned g = lane >> 4, c = lane & 15;
  const unsigned novote = 0u - (unsigned)((b >> 7) + ((w + 1) >> 2) != 2), outv = (b >> 6) & 1;
  const int prb = (int)((((b & (63u >> (3 * outv))) << 22) + (idx & 1) * 0x2000 + ((idx >> 1) & 1) * 0x1000 + ((idx >> 2) & 1) * 0x4000) | novote);
  const char* const pbase = (const char*)((size_t)W ^ (((size_t)W ^ (size_t)out) & (0ull - outv)));
  const unsigned doff = (unsigned)WS_A + (rank * 4 + Pl) * 131072 + (ukh * 4 + ucw) * 16384 + q * 4096;
  const unsigned fl = (unsigned)WS_LFLAG + Pl * 1024, fr = (unsigned)WS_AFLAG + rank * 1024 + Pl * 16;
  const unsigned foff = ((fl & locm) | (fr & ~locm)) + (ukh * 4 + ucw) * 128 + q * 4;
  const unsigned n0 = nt * 128;
  const unsigned voffW = ((w >> 1) * 64 + 4 * g) * 65536u + (n0 + (w & 1) * 64 + 4 * c) * 4u;
  const unsigned voffC = ((idx & 7) * 4 + w) * 4096 + lane * 64;
  const unsigned voffO = (4 * g) * 65536u + (n0 + 4 * c) * 4u;
  const unsigned voffT = (((idx * 4 + w) << 21) + 0x100000u + lab * 4096u) | ((0u - lane) & 0x40000000u);
  const unsigned Pi = lane & 31, ownm = 0u - (unsigned)(Pi < 4), j = Pi - 4, j7 = ((j & 31) * 37) >> 8;
  const unsigned Pq = ((rank * 4 + Pi) & ownm) | ((((rank + 1 + j - 7 * j7) & 7) * 4 + j7) & ~ownm);
  static_assert(NSTATIC <= 4 && LFL == 1, "");
  const unsigned kh0 = ((role ^ idx ^ Pi) & 1) & (0u - (unsigned)(Pi < NSTATIC));
  unsigned vtabW = Pq * (8u << 20) + kh0 * (2u << 20);
  unsigned vtabA = Pq * 131072u + kh0 * 65536u;
  unsigned voffF = ((((unsigned)WS_LFLAG + Pi * 1024) & ownm) | (((unsigned)WS_AFLAG + (Pq >> 2) * 1024 + (Pq & 3) * 16) & ~ownm)) + (kh0 * 4 + w) * 128;
  const unsigned bxr = lane * 16;
  static_assert(WS_TMO == 0 && WS_VOTE == 1024, "offsets used by the assembly");
  static_assert(WS_A == 0x100000 && SLOT0 + 3 * SPC - 1 + EVD < NEVENT, "push schedule does not fit the event sites");
  asm volatile(GEMM_ASM
               : [vtabW] "+v"(vtabW), [vtabA] "+v"(vtabA), [voffF] "+v"(voffF)
               : [wptr] "s"(W), [xptr] "s"(X), [wsp] "s"(ws), [optr] "s"(out), [callp] "s"(callp), [slot] "s"(slot),
                 [prb] "s"(prb), [votep] "s"(ws + WS_VOTE + (b & 63) * 8 + ((b >> 6) & 1) * 4), [cls] "s"(cls), [khx] "s"((cls >> 1) ^ role), [w4k] "s"(w * 4096), [phi] "s"(((idx * 13 + cls * 8) & 31) * PHI), [phic] "s"(PHIC - (int)((idx * 13 + cls * 8) & 31)),
                 [tabp] "s"(P8.tab + d), [doff] "s"(doff), [foff] "s"(foff), [xfo] "s"(ws + WS_XFLAG + ((b ^ 4) * 4 + w) * 128), [xfi] "s"(ws + WS_XFLAG + gw * 128),
                 [pbase] "s"(pbase), [xlds] "s"(xlds),
                 [voffW] "v"(voffW), [voffC] "v"(voffC), [voffO] "v"(voffO), [voffT] "v"(voffT),
                 [xoff0] "v"(xoff0), [xoff1] "v"(xoff1), [bxr] "v"(bxr)
               : GEMM_CLOB);
}

#define HIP_CHECK(expr) do { hipError_t _e = (expr); if (_e != hipSuccess) { \
    fprintf(stderr, "HIP error %s at %s:%d\n", hipGetErrorString(_e), __FILE__, __LINE__); return; } } while (0)

extern "C" void kernel_launch(int rank, int nranks, ncclComm_t comm, hipStream_t stream, void* const* d_in, const int* in_sizes, int n_in,
                              void* d_out, int out_size, const int* in_dtypes, int out_dtype, void* const* d_out_all,
                              void* const* d_ws_all, size_t ws_bytes) {
  if (nranks != NR || (unsigned)rank >= (unsigned)NR || n_in != 2 || ws_bytes < WS_END || in_sizes[0] != 4096 * 512 ||
      in_sizes[1] != 4096 * 16384 || out_size != 512 * 16384) {
    fprintf(stderr, "unexpected problem shape\n");
    return;
  }
  HIP_CHECK(hipFuncSetAttribute((const void*)a2a_gemm_gelu, hipFuncAttributeMaxDynamicSharedMemorySize, LDS_BYTES));
  Ptrs P8;
  P8.own = (char*)d_ws_all[rank];
  P8.tab = (char* const*)d_ws_all;
  a2a_gemm_gelu<<<256, 256, LDS_BYTES, stream>>>((const float*)d_in[0], (const float*)d_in[1], (float*)d_out, P8, rank);
  HIP_CHECK(hipGetLastError());
}
```
